```python
import numpy as np

kernel_name = "gpu_a2a_gemm_m8192_k8192_n8192_bf16_none"


ABSMAX_RTOL = 0.008

COMPUTE_DTYPE = "bfloat16"
EPILOGUE = "none"
IN_QUANT = None
OUT_QUANT = None
IN_DTYPES = {"x": "bfloat16", "w_mat": "bfloat16"}
WORLD_SIZE = 8

def _to_bf16(x):
    u = np.ascontiguousarray(x, dtype=np.float32).view(np.uint32)
    return ((u + 0x7FFF + ((u >> 16) & 1)) & 0xFFFF0000).view(np.float32)

def setup_inputs():

    x = _to_bf16(np.random.randn(8192, 8192).astype(np.float32))
    w_mat = _to_bf16((np.random.randn(8192, 8192) / np.sqrt(8192)).astype(np.float32))
    return {"x": x, "w_mat": w_mat}

def _epilogue(y):
    return y

def reference(x, w_mat):
    y = (x @ w_mat).astype(np.float32)
    return _epilogue(y)
```

```cpp
#include <hip/hip_runtime.h>
#include <rccl/rccl.h>
#include <cstdio>
#include <cstddef>
#define K_KH 0
#define K_PD 2
#define K_PROF 0
#define K_WC 1
#define K_NR 8
#define K_I0 4
#define K_I 6
#define K_DBG 0
#define K_ASM1 \
  "s_add_u32 s79, s78, 256\n\t" \
  "s_add_u32 m0, s68, 65536\n\t" \
  "s_nop 0\n\t" \
  "buffer_load_dwordx4 v4, s[40:43], s78 offen offset:0 lds\n\t" \
  "buffer_load_dwordx4 v5, s[40:43], s78 offen offset:1024 lds\n\t" \
  "buffer_load_dwordx4 v6, s[40:43], s78 offen offset:2048 lds\n\t" \
  "buffer_load_dwordx4 v7, s[40:43], s78 offen offset:3072 lds\n\t" \
  "s_add_u32 m0, s68, 0\n\t" \
  "s_nop 0\n\t" \
  "buffer_load_dwordx4 v0, s[36:39], s69 offen offset:0 sc0 sc1 lds\n\t" \
  "buffer_load_dwordx4 v1, s[36:39], s69 offen offset:1024 sc0 sc1 lds\n\t" \
  "buffer_load_dwordx4 v2, s[36:39], s69 offen offset:2048 sc0 sc1 lds\n\t" \
  "buffer_load_dwordx4 v3, s[36:39], s69 offen offset:3072 sc0 sc1 lds\n\t" \
  "s_add_u32 m0, s68, 81920\n\t" \
  "s_nop 0\n\t" \
  "buffer_load_dwordx4 v4, s[40:43], s79 offen offset:0 lds\n\t" \
  "buffer_load_dwordx4 v5, s[40:43], s79 offen offset:1024 lds\n\t" \
  "buffer_load_dwordx4 v6, s[40:43], s79 offen offset:2048 lds\n\t" \
  "buffer_load_dwordx4 v7, s[40:43], s79 offen offset:3072 lds\n\t" \
  "global_load_ushort v16, v16, s[58:59]\n\t" \
  "v_mov_b32 v47, s62\n\t" \
  "v_mov_b32 v31, 1\n\t" \
  "s_mov_b64 exec, 1\n\t" \
  "global_atomic_add v46, v47, v31, s[50:51] sc0\n\t" \
  "s_mov_b64 exec, -1\n\t" \
  "global_load_dwordx4 v[32:35], v8, s[48:49] nt\n\t" \
  "s_add_u32 m0, s68, 16384\n\t" \
  "s_nop 0\n\t" \
  "buffer_load_dwordx4 v0, s[36:39], s70 offen offset:0 sc0 sc1 lds\n\t" \
  "buffer_load_dwordx4 v1, s[36:39], s70 offen offset:1024 sc0 sc1 lds\n\t" \
  "buffer_load_dwordx4 v2, s[36:39], s70 offen offset:2048 sc0 sc1 lds\n\t" \
  "buffer_load_dwordx4 v3, s[36:39], s70 offen offset:3072 sc0 sc1 lds\n\t" \
  ""

#define K_ASM2 \
  "s_load_dwordx4 s[52:55], s[58:59], s52\n\t" \
  "s_add_u32 s64, s69, 256\n\t" \
  "s_add_u32 s65, s70, 256\n\t" \
  "s_add_u32 s78, s78, 0x200000\n\t" \
  "s_add_u32 s79, s78, 256\n\t" \
  "s_add_u32 m0, s68, 98304\n\t" \
  "s_nop 0\n\t" \
  "buffer_load_dwordx4 v4, s[40:43], s78 offen offset:0 lds\n\t" \
  "buffer_load_dwordx4 v5, s[40:43], s78 offen offset:1024 lds\n\t" \
  "buffer_load_dwordx4 v6, s[40:43], s78 offen offset:2048 lds\n\t" \
  "buffer_load_dwordx4 v7, s[40:43], s78 offen offset:3072 lds\n\t" \
  "s_add_u32 m0, s68, 32768\n\t" \
  "s_nop 0\n\t" \
  "buffer_load_dwordx4 v0, s[36:39], s64 offen offset:0 sc0 sc1 lds\n\t" \
  "buffer_load_dwordx4 v1, s[36:39], s64 offen offset:1024 sc0 sc1 lds\n\t" \
  "buffer_load_dwordx4 v2, s[36:39], s64 offen offset:2048 sc0 sc1 lds\n\t" \
  "buffer_load_dwordx4 v3, s[36:39], s64 offen offset:3072 sc0 sc1 lds\n\t" \
  "v_mov_b32 v30, 0\n\t" \
  "v_mov_b32 v31, 1\n\t" \
  "s_mov_b32 s14, 0\n\t" \
  "s_mov_b32 s15, 0\n\t" \
  "s_mov_b32 s16, 16384\n\t" \
  "s_mov_b32 s17, 1\n\t" \
  "s_mov_b32 s19, 0\n\t" \
  "s_mov_b32 s81, 0\n\t" \
  "s_mov_b32 s84, 144\n\t" \
  "s_mov_b32 s77, 2\n\t" \
  "s_mov_b32 s88, 0\n\t" \
  "s_mov_b32 s30, 0\n\t" \
  "s_add_u32 s29, s71, 0\n\t" \
  "s_mov_b32 s31, s74\n\t" \
  "s_add_u32 s87, s72, 0\n\t" \
  "s_mov_b32 s72, 0\n\t" \
  "s_mov_b32 s26, 1431655731\n\t" \
  "s_mov_b32 s27, 644245094\n\t" \
  "s_cmp_eq_u32 s71, 0\n\t" \
  "s_cselect_b64 s[90:91], 0xff, -1\n\t" \
  "s_cselect_b64 s[58:59], 0, -1\n\t" \
  "s_mov_b32 s76, 26\n\t" \
  "s_cmp_eq_u32 s74, 128\n\t" \
  "s_cselect_b32 s80, 0, 4294966784\n\t" \
  "s_cselect_b32 s98, 0, 128\n\t" \
  "v_lshrrev_b32 v39, 14, v8\n\t" \
  "v_lshlrev_b32 v39, 10, v39\n\t" \
  "v_lshl_add_u32 v39, v24, 4, v39\n\t" \
  "s_lshl_b32 s99, s98, 10\n\t" \
  "v_add_u32 v39, s99, v39\n\t" \
  "v_mov_b32 v38, s98\n\t" \
  "v_mov_b32 v36, 0\n\t" \
  "v_mov_b32 v37, 0\n\t" \
  "s_sub_u32 s69, s69, s63\n\t" \
  "s_sub_u32 s70, s70, s63\n\t" \
  "s_mov_b32 s67, 1\n\t" \
  "s_mov_b32 s89, 2095104\n\t" \
  "s_sub_u32 s63, 16777216, s38\n\t" \
  "s_lshr_b32 s63, s63, 4\n\t" \
  "s_sub_u32 s63, 16777216, s63\n\t" \
  "s_getpc_b64 s[94:95]\n\t" \
  "CW0_%=:\n\t" \
  "s_lshr_b32 s98, s75, 10\n\t" \
  "s_and_b32 s98, s98, 31\n\t" \
  "s_lshl_b32 s98, s98, 2\n\t" \
  "s_add_u32 s98, s98, s71\n\t" \
  "s_lshl_b32 s98, s98, 10\n\t" \
  "v_lshl_add_u32 v51, v24, 7, s98\n\t" \
  "s_mov_b32 s99, END_%=-CW0_%=-4\n\t" \
  "v_min_u32 v51, s99, v51\n\t" \
  "v_and_b32 v51, -4, v51\n\t" \
  "s_mov_b64 exec, 0xff\n\t" \
  "global_load_dword v51, v51, s[94:95]\n\t" \
  "s_mov_b64 exec, -1\n\t" \
  "s_waitcnt vmcnt(16)\n\t" \
  "s_barrier\n\t" \
  "ds_read_b128 v[64:67], v10 offset:0\n\t" \
  "ds_read_b128 v[68:71], v11 offset:0\n\t" \
  "ds_read_b64_tr_b16 a[128:129], v12 offset:0\n\t" \
  "ds_read_b64_tr_b16 a[130:131], v12 offset:1024\n\t" \
  "ds_read_b64_tr_b16 a[132:133], v12 offset:8192\n\t" \
  "ds_read_b64_tr_b16 a[134:135], v12 offset:9216\n\t" \
  "ds_read_b64_tr_b16 a[136:137], v13 offset:0\n\t" \
  "ds_read_b64_tr_b16 a[138:139], v13 offset:1024\n\t" \
  "ds_read_b64_tr_b16 a[140:141], v13 offset:8192\n\t" \
  "ds_read_b64_tr_b16 a[142:143], v13 offset:9216\n\t" \
  "ds_read_b64_tr_b16 a[144:145], v14 offset:0\n\t" \
  "ds_read_b64_tr_b16 a[146:147], v14 offset:1024\n\t" \
  "ds_read_b64_tr_b16 a[148:149], v14 offset:8192\n\t" \
  "ds_read_b64_tr_b16 a[150:151], v14 offset:9216\n\t" \
  "ds_read_b64_tr_b16 a[152:153], v15 offset:0\n\t" \
  "ds_read_b64_tr_b16 a[154:155], v15 offset:1024\n\t" \
  "ds_read_b64_tr_b16 a[156:157], v15 offset:8192\n\t" \
  "ds_read_b64_tr_b16 a[158:159], v15 offset:9216\n\t" \
  "ds_read_b64_tr_b16 a[160:161], v12 offset:16384\n\t" \
  "ds_read_b64_tr_b16 a[162:163], v12 offset:17408\n\t" \
  "ds_read_b64_tr_b16 a[164:165], v12 offset:24576\n\t" \
  "ds_read_b64_tr_b16 a[166:167], v12 offset:25600\n\t" \
  "ds_read_b64_tr_b16 a[168:169], v13 offset:16384\n\t" \
  "ds_read_b64_tr_b16 a[170:171], v13 offset:17408\n\t" \
  "ds_read_b64_tr_b16 a[172:173], v13 offset:24576\n\t" \
  "ds_read_b64_tr_b16 a[174:175], v13 offset:25600\n\t" \
  "ds_read_b64_tr_b16 a[176:177], v14 offset:16384\n\t" \
  "ds_read_b64_tr_b16 a[178:179], v14 offset:17408\n\t" \
  "ds_read_b64_tr_b16 a[180:181], v14 offset:24576\n\t" \
  "ds_read_b64_tr_b16 a[182:183], v14 offset:25600\n\t" \
  "ds_read_b64_tr_b16 a[184:185], v15 offset:16384\n\t" \
  "ds_read_b64_tr_b16 a[186:187], v15 offset:17408\n\t" \
  "s_add_u32 m0, s68, 114688\n\t" \
  "s_nop 0\n\t" \
  "buffer_load_dwordx4 v4, s[40:43], s79 offen offset:0 lds\n\t" \
  "buffer_load_dwordx4 v5, s[40:43], s79 offen offset:1024 lds\n\t" \
  "buffer_load_dwordx4 v6, s[40:43], s79 offen offset:2048 lds\n\t" \
  "buffer_load_dwordx4 v7, s[40:43], s79 offen offset:3072 lds\n\t" \
  "s_add_u32 m0, s68, 49152\n\t" \
  "s_nop 0\n\t" \
  "buffer_load_dwordx4 v0, s[36:39], s65 offen offset:0 sc0 sc1 lds\n\t" \
  "buffer_load_dwordx4 v1, s[36:39], s65 offen offset:1024 sc0 sc1 lds\n\t" \
  "buffer_load_dwordx4 v2, s[36:39], s65 offen offset:2048 sc0 sc1 lds\n\t" \
  "buffer_load_dwordx4 v3, s[36:39], s65 offen offset:3072 sc0 sc1 lds\n\t" \
  "s_add_u32 s64, s64, 256\n\t" \
  "s_add_u32 s65, s65, 256\n\t" \
  "s_add_u32 s78, s78, 0x200000\n\t" \
  "s_add_u32 s79, s78, 256\n\t" \
  "s_waitcnt lgkmcnt(14)\n\t" \
  "v_mfma_f32_16x16x32_bf16 a[0:3], a[128:131], v[64:67], 0\n\t" \
  "ds_read_b64_tr_b16 a[188:189], v15 offset:24576\n\t" \
  "ds_read_b64_tr_b16 a[190:191], v15 offset:25600\n\t" \
  "v_mfma_f32_16x16x32_bf16 a[0:3], a[132:135], v[68:71], a[0:3]\n\t" \
  "ds_read_b128 v[72:75], v10 offset:2048\n\t" \
  "ds_read_b128 v[76:79], v11 offset:2048\n\t" \
  "v_mfma_f32_16x16x32_bf16 a[4:7], a[140:143], v[68:71], 0\n\t" \
  "ds_read_b128 v[80:83], v10 offset:4096\n\t" \
  "ds_read_b128 v[84:87], v11 offset:4096\n\t" \
  "v_mfma_f32_16x16x32_bf16 a[4:7], a[136:139], v[64:67], a[4:7]\n\t" \
  "ds_read_b128 v[88:91], v10 offset:6144\n\t" \
  "ds_read_b128 v[92:95], v11 offset:6144\n\t" \
  "v_mfma_f32_16x16x32_bf16 a[8:11], a[144:147], v[64:67], 0\n\t" \
  "v_mfma_f32_16x16x32_bf16 a[8:11], a[148:151], v[68:71], a[8:11]\n\t" \
  "v_mfma_f32_16x16x32_bf16 a[12:15], a[156:159], v[68:71], 0\n\t" \
  "v_mfma_f32_16x16x32_bf16 a[12:15], a[152:155], v[64:67], a[12:15]\n\t" \
  "s_waitcnt vmcnt(9) lgkmcnt(0)\n\t" \
  "s_barrier\n\t" \
  "v_mfma_f32_16x16x32_bf16 a[16:19], a[160:163], v[64:67], 0\n\t" \
  "ds_read_b64_tr_b16 a[192:193], v12 offset:32768\n\t" \
  "v_readfirstlane_b32 s85, v46\n\t" \
  "v_mfma_f32_16x16x32_bf16 a[16:19], a[164:167], v[68:71], a[16:19]\n\t" \
  "s_add_u32 m0, s68, 0\n\t" \
  "v_lshrrev_b32 v17, 8, v16\n\t" \
  "v_mfma_f32_16x16x32_bf16 a[20:23], a[172:175], v[68:71], 0\n\t" \
  "ds_read_b64_tr_b16 a[194:195], v12 offset:33792\n\t" \
  "buffer_load_dwordx4 v0, s[36:39], s64 offen offset:0 sc0 sc1 lds\n\t" \
  "v_mfma_f32_16x16x32_bf16 a[20:23], a[168:171], v[64:67], a[20:23]\n\t" \
  "ds_read_b128 v[96:99], v10 offset:16384\n\t" \
  "v_and_b32 v16, 0xff, v16\n\t" \
  "v_mfma_f32_16x16x32_bf16 a[24:27], a[176:179], v[64:67], 0\n\t" \
  "ds_read_b64_tr_b16 a[196:197], v12 offset:40960\n\t" \
  "v_lshlrev_b32 v17, 7, v17\n\t" \
  "v_mfma_f32_16x16x32_bf16 a[24:27], a[180:183], v[68:71], a[24:27]\n\t" \
  "buffer_load_dwordx4 v1, s[36:39], s64 offen offset:1024 sc0 sc1 lds\n\t" \
  "v_mfma_f32_16x16x32_bf16 a[28:31], a[188:191], v[68:71], 0\n\t" \
  "ds_read_b64_tr_b16 a[198:199], v12 offset:41984\n\t" \
  "v_lshlrev_b32 v16, 7, v16\n\t" \
  "v_mfma_f32_16x16x32_bf16 a[28:31], a[184:187], v[64:67], a[28:31]\n\t" \
  "ds_read_b128 v[64:67], v10 offset:32768\n\t" \
  "ds_read_b128 v[68:71], v11 offset:32768\n\t" \
  "v_mfma_f32_16x16x32_bf16 a[60:63], a[184:187], v[72:75], 0\n\t" \
  "ds_read_b64_tr_b16 a[200:201], v13 offset:32768\n\t" \
  "buffer_load_dwordx4 v2, s[36:39], s64 offen offset:2048 sc0 sc1 lds\n\t" \
  "v_mfma_f32_16x16x32_bf16 a[60:63], a[188:191], v[76:79], a[60:63]\n\t" \
  "ds_read_b128 v[100:103], v11 offset:16384\n\t" \
  "s_add_u32 s85, s85, 1\n\t" \
  "v_mfma_f32_16x16x32_bf16 a[56:59], a[180:183], v[76:79], 0\n\t" \
  "ds_read_b64_tr_b16 a[202:203], v13 offset:33792\n\t" \
  "s_lshl_b32 s86, s85, 5\n\t" \
  "v_mfma_f32_16x16x32_bf16 a[56:59], a[176:179], v[72:75], a[56:59]\n\t" \
  "buffer_load_dwordx4 v3, s[36:39], s64 offen offset:3072 sc0 sc1 lds\n\t" \
  "v_mfma_f32_16x16x32_bf16 a[52:55], a[168:171], v[72:75], 0\n\t" \
  "ds_read_b64_tr_b16 a[204:205], v13 offset:40960\n\t" \
  "s_and_b32 s98, s85, 1\n\t" \
  "v_mfma_f32_16x16x32_bf16 a[52:55], a[172:175], v[76:79], a[52:55]\n\t" \
  "s_add_u32 m0, s68, 65536\n\t" \
  "s_lshl_b32 s99, s98, 24\n\t" \
  "v_mfma_f32_16x16x32_bf16 a[48:51], a[164:167], v[76:79], 0\n\t" \
  "ds_read_b64_tr_b16 a[206:207], v13 offset:41984\n\t" \
  "buffer_load_dwordx4 v4, s[40:43], s78 offen offset:0 lds\n\t" \
  "v_mfma_f32_16x16x32_bf16 a[48:51], a[160:163], v[72:75], a[48:51]\n\t" \
  "ds_read_b128 v[104:107], v10 offset:18432\n\t" \
  "s_add_u32 s52, s52, s99\n\t" \
  "s_addc_u32 s53, s53, 0\n\t" \
  "v_mfma_f32_16x16x32_bf16 a[44:47], a[152:155], v[72:75], 0\n\t" \
  "ds_read_b64_tr_b16 a[208:209], v14 offset:32768\n\t" \
  "s_add_u32 s56, s56, s99\n\t" \
  "s_addc_u32 s57, s57, 0\n\t" \
  "v_mfma_f32_16x16x32_bf16 a[44:47], a[156:159], v[76:79], a[44:47]\n\t" \
  "buffer_load_dwordx4 v5, s[40:43], s78 offen offset:1024 lds\n\t" \
  "v_mfma_f32_16x16x32_bf16 a[40:43], a[148:151], v[76:79], 0\n\t" \
  "ds_read_b64_tr_b16 a[210:211], v14 offset:33792\n\t" \
  "s_and_b32 s57, s57, 0xffff\n\t" \
  "v_mfma_f32_16x16x32_bf16 a[40:43], a[144:147], v[72:75], a[40:43]\n\t" \
  "ds_read_b128 v[108:111], v11 offset:18432\n\t" \
  "v_mfma_f32_16x16x32_bf16 a[36:39], a[136:139], v[72:75], 0\n\t" \
  "ds_read_b64_tr_b16 a[212:213], v14 offset:40960\n\t" \
  "buffer_load_dwordx4 v6, s[40:43], s78 offen offset:2048 lds\n\t" \
  "v_mfma_f32_16x16x32_bf16 a[36:39], a[140:143], v[76:79], a[36:39]\n\t" \
  "ds_read_b128 v[112:115], v10 offset:20480\n\t" \
  "v_mfma_f32_16x16x32_bf16 a[32:35], a[132:135], v[76:79], 0\n\t" \
  "ds_read_b64_tr_b16 a[214:215], v14 offset:41984\n\t" \
  "v_mfma_f32_16x16x32_bf16 a[32:35], a[128:131], v[72:75], a[32:35]\n\t" \
  "ds_read_b128 v[72:75], v10 offset:34816\n\t" \
  "ds_read_b128 v[76:79], v11 offset:34816\n\t" \
  "buffer_load_dwordx4 v7, s[40:43], s78 offen offset:3072 lds\n\t" \
  "v_mfma_f32_16x16x32_bf16 a[64:67], a[128:131], v[80:83], 0\n\t" \
  "ds_read_b64_tr_b16 a[216:217], v15 offset:32768\n\t" \
  "v_mfma_f32_16x16x32_bf16 a[64:67], a[132:135], v[84:87], a[64:67]\n\t" \
  "s_sub_u32 s87, s87, 1\n\t" \
  "v_mfma_f32_16x16x32_bf16 a[68:71], a[140:143], v[84:87], 0\n\t" \
  "ds_read_b64_tr_b16 a[218:219], v15 offset:33792\n\t" \
  "v_mfma_f32_16x16x32_bf16 a[68:71], a[136:139], v[80:83], a[68:71]\n\t" \
  "s_cbranch_scc1 EVT1_%=\n\t" \
  "v_mfma_f32_16x16x32_bf16 a[72:75], a[144:147], v[80:83], 0\n\t" \
  "ds_read_b64_tr_b16 a[220:221], v15 offset:40960\n\t" \
  "v_mfma_f32_16x16x32_bf16 a[72:75], a[148:151], v[84:87], a[72:75]\n\t" \
  "v_mfma_f32_16x16x32_bf16 a[76:79], a[156:159], v[84:87], 0\n\t" \
  "ds_read_b64_tr_b16 a[222:223], v15 offset:41984\n\t" \
  "v_mfma_f32_16x16x32_bf16 a[76:79], a[152:155], v[80:83], a[76:79]\n\t" \
  "v_mfma_f32_16x16x32_bf16 a[80:83], a[160:163], v[80:83], 0\n\t" \
  "v_mfma_f32_16x16x32_bf16 a[80:83], a[164:167], v[84:87], a[80:83]\n\t" \
  "ds_read_b128 v[116:119], v11 offset:20480\n\t" \
  "v_mfma_f32_16x16x32_bf16 a[84:87], a[172:175], v[84:87], 0\n\t" \
  "ds_read_b128 v[120:123], v10 offset:22528\n\t" \
  "v_mfma_f32_16x16x32_bf16 a[84:87], a[168:171], v[80:83], a[84:87]\n\t" \
  "v_mfma_f32_16x16x32_bf16 a[88:91], a[176:179], v[80:83], 0\n\t" \
  "ds_read_b128 v[124:127], v11 offset:22528\n\t" \
  "v_mfma_f32_16x16x32_bf16 a[88:91], a[180:183], v[84:87], a[88:91]\n\t" \
  "v_mfma_f32_16x16x32_bf16 a[92:95], a[188:191], v[84:87], 0\n\t" \
  "RET2_%=:\n\t" \
  "v_mfma_f32_16x16x32_bf16 a[92:95], a[184:187], v[80:83], a[92:95]\n\t" \
  "ds_read_b128 v[80:83], v10 offset:36864\n\t" \
  "ds_read_b128 v[84:87], v11 offset:36864\n\t" \
  "v_mfma_f32_16x16x32_bf16 a[124:127], a[184:187], v[88:91], 0\n\t" \
  "v_mfma_f32_16x16x32_bf16 a[124:127], a[188:191], v[92:95], a[124:127]\n\t" \
  "v_mfma_f32_16x16x32_bf16 a[120:123], a[180:183], v[92:95], 0\n\t" \
  "v_mfma_f32_16x16x32_bf16 a[120:123], a[176:179], v[88:91], a[120:123]\n\t" \
  "s_add_u32 m0, s68, 81920\n\t" \
  "v_mfma_f32_16x16x32_bf16 a[116:119], a[168:171], v[88:91], 0\n\t" \
  "buffer_load_dwordx4 v4, s[40:43], s79 offen offset:0 lds\n\t" \
  "v_mfma_f32_16x16x32_bf16 a[116:119], a[172:175], v[92:95], a[116:119]\n\t" \
  "v_mfma_f32_16x16x32_bf16 a[112:115], a[164:167], v[92:95], 0\n\t" \
  "v_mfma_f32_16x16x32_bf16 a[112:115], a[160:163], v[88:91], a[112:115]\n\t" \
  "buffer_load_dwordx4 v5, s[40:43], s79 offen offset:1024 lds\n\t" \
  "v_mfma_f32_16x16x32_bf16 a[108:111], a[152:155], v[88:91], 0\n\t" \
  "v_mfma_f32_16x16x32_bf16 a[108:111], a[156:159], v[92:95], a[108:111]\n\t" \
  "v_mfma_f32_16x16x32_bf16 a[104:107], a[148:151], v[92:95], 0\n\t" \
  "buffer_load_dwordx4 v6, s[40:43], s79 offen offset:2048 lds\n\t" \
  "v_mfma_f32_16x16x32_bf16 a[104:107], a[144:147], v[88:91], a[104:107]\n\t" \
  "v_mfma_f32_16x16x32_bf16 a[100:103], a[136:139], v[88:91], 0\n\t" \
  "v_mfma_f32_16x16x32_bf16 a[100:103], a[140:143], v[92:95], a[100:103]\n\t" \
  "buffer_load_dwordx4 v7, s[40:43], s79 offen offset:3072 lds\n\t" \
  "v_mfma_f32_16x16x32_bf16 a[96:99], a[132:135], v[92:95], 0\n\t" \
  "v_mfma_f32_16x16x32_bf16 a[96:99], a[128:131], v[88:91], a[96:99]\n\t" \
  "ds_read_b128 v[88:91], v10 offset:38912\n\t" \
  "ds_read_b128 v[92:95], v11 offset:38912\n\t" \
  "s_waitcnt lgkmcnt(4)\n\t" \
  "v_mfma_f32_16x16x32_bf16 v[128:131], a[128:131], v[96:99], 0\n\t" \
  "s_mov_b32 s99, s65\n\t" \
  "s_add_u32 s77, s77, 1\n\t" \
  "v_mfma_f32_16x16x32_bf16 v[128:131], a[132:135], v[100:103], v[128:131]\n\t" \
  "v_readlane_b32 s66, v16, s77\n\t" \
  "v_and_b32 v52, 15, v24\n\t" \
  "v_mfma_f32_16x16x32_bf16 v[132:135], a[140:143], v[100:103], 0\n\t" \
  "v_lshrrev_b32 v53, 4, v24\n\t" \
  "v_lshlrev_b32 v29, 14, v53\n\t" \
  "v_mfma_f32_16x16x32_bf16 v[132:135], a[136:139], v[96:99], v[132:135]\n\t" \
  "v_lshl_add_u32 v29, v52, 4, v29\n\t" \
  "v_and_b32 v54, 8, v52\n\t" \
  "v_mfma_f32_16x16x32_bf16 v[136:139], a[144:147], v[96:99], 0\n\t" \
  "s_lshl_b32 s78, s66, 13\n\t" \
  "v_lshl_add_u32 v29, v54, 4, v29\n\t" \
  "v_mfma_f32_16x16x32_bf16 v[136:139], a[148:151], v[100:103], v[136:139]\n\t" \
  "s_lshr_b32 s65, s66, 11\n\t" \
  "v_xor_b32 v54, v52, v53\n\t" \
  "v_mfma_f32_16x16x32_bf16 v[140:143], a[156:159], v[100:103], 0\n\t" \
  "s_mul_i32 s64, s66, s67\n\t" \
  "v_lshlrev_b32 v54, 4, v54\n\t" \
  "v_mfma_f32_16x16x32_bf16 v[140:143], a[152:155], v[96:99], v[140:143]\n\t" \
  "v_lshl_add_u32 v54, v53, 8, v54\n\t" \
  "v_add_u32 v28, s68, v54\n\t" \
  "s_waitcnt vmcnt(12) lgkmcnt(0)\n\t" \
  "s_barrier\n\t" \
  "v_mfma_f32_16x16x32_bf16 v[144:147], a[160:163], v[96:99], 0\n\t" \
  "ds_read_b64_tr_b16 a[224:225], v12 offset:49152\n\t" \
  "s_add_u32 m0, s68, 16384\n\t" \
  "v_mfma_f32_16x16x32_bf16 v[144:147], a[164:167], v[100:103], v[144:147]\n\t" \
  "s_add_u32 s79, s78, 256\n\t" \
  "v_lshrrev_b32 v54, 1, v53\n\t" \
  "v_mfma_f32_16x16x32_bf16 v[148:151], a[172:175], v[100:103], 0\n\t" \
  "ds_read_b64_tr_b16 a[226:227], v12 offset:50176\n\t" \
  "buffer_load_dwordx4 v0, s[36:39], s99 offen offset:0 sc0 sc1 lds\n\t" \
  "v_mfma_f32_16x16x32_bf16 v[148:151], a[168:171], v[96:99], v[148:151]\n\t" \
  "s_mul_i32 s65, s65, s89\n\t" \
  "v_xor_b32 v54, v54, v52\n\t" \
  "v_mfma_f32_16x16x32_bf16 v[152:155], a[176:179], v[96:99], 0\n\t" \
  "ds_read_b64_tr_b16 a[228:229], v12 offset:57344\n\t" \
  "buffer_load_dwordx4 v1, s[36:39], s99 offen offset:1024 sc0 sc1 lds\n\t" \
  "v_mfma_f32_16x16x32_bf16 v[152:155], a[180:183], v[100:103], v[152:155]\n\t" \
  "s_add_u32 s65, s65, s64\n\t" \
  "v_lshlrev_b32 v54, 4, v54\n\t" \
  "v_mfma_f32_16x16x32_bf16 v[156:159], a[188:191], v[100:103], 0\n\t" \
  "ds_read_b64_tr_b16 a[230:231], v12 offset:58368\n\t" \
  "v_lshl_add_u32 v54, v52, 8, v54\n\t" \
  "v_mfma_f32_16x16x32_bf16 v[156:159], a[184:187], v[96:99], v[156:159]\n\t" \
  "ds_read_b128 v[96:99], v10 offset:49152\n\t" \
  "ds_read_b128 v[100:103], v11 offset:49152\n\t" \
  "v_mfma_f32_16x16x32_bf16 v[188:191], a[184:187], v[104:107], 0\n\t" \
  "ds_read_b64_tr_b16 a[232:233], v13 offset:49152\n\t" \
  "buffer_load_dwordx4 v2, s[36:39], s99 offen offset:2048 sc0 sc1 lds\n\t" \
  "v_mfma_f32_16x16x32_bf16 v[188:191], a[188:191], v[108:111], v[188:191]\n\t" \
  "s_add_u32 s64, s65, s69\n\t" \
  "v_and_b32 v55, 1, v53\n\t" \
  "v_mfma_f32_16x16x32_bf16 v[184:187], a[180:183], v[108:111], 0\n\t" \
  "ds_read_b64_tr_b16 a[234:235], v13 offset:50176\n\t" \
  "v_lshl_add_u32 v54, v55, 3, v54\n\t" \
  "v_mfma_f32_16x16x32_bf16 v[184:187], a[176:179], v[104:107], v[184:187]\n\t" \
  "s_add_u32 s65, s65, s70\n\t" \
  "v_add_u32 v27, s68, v54\n\t" \
  "v_mfma_f32_16x16x32_bf16 v[180:183], a[168:171], v[104:107], 0\n\t" \
  "ds_read_b64_tr_b16 a[236:237], v13 offset:57344\n\t" \
  "buffer_load_dwordx4 v3, s[36:39], s99 offen offset:3072 sc0 sc1 lds\n\t" \
  "v_mfma_f32_16x16x32_bf16 v[180:183], a[172:175], v[108:111], v[180:183]\n\t" \
  "v_lshlrev_b32 v54, 2, v53\n\t" \
  "v_sub_u32 v54, v52, v54\n\t" \
  "v_mfma_f32_16x16x32_bf16 v[176:179], a[164:167], v[108:111], 0\n\t" \
  "ds_read_b64_tr_b16 a[238:239], v13 offset:58368\n\t" \
  "v_and_b32 v55, 1, v54\n\t" \
  "v_mfma_f32_16x16x32_bf16 v[176:179], a[160:163], v[104:107], v[176:179]\n\t" \
  "s_sub_u32 s87, s87, 1\n\t" \
  "v_mfma_f32_16x16x32_bf16 v[172:175], a[152:155], v[104:107], 0\n\t" \
  "ds_read_b64_tr_b16 a[240:241], v14 offset:49152\n\t" \
  "v_mfma_f32_16x16x32_bf16 v[172:175], a[156:159], v[108:111], v[172:175]\n\t" \
  "s_cbranch_scc1 EVT7_%=\n\t" \
  "v_mfma_f32_16x16x32_bf16 v[168:171], a[148:151], v[108:111], 0\n\t" \
  "ds_read_b64_tr_b16 a[242:243], v14 offset:50176\n\t" \
  "v_mfma_f32_16x16x32_bf16 v[168:171], a[144:147], v[104:107], v[168:171]\n\t" \
  "v_mfma_f32_16x16x32_bf16 v[164:167], a[136:139], v[104:107], 0\n\t" \
  "ds_read_b64_tr_b16 a[244:245], v14 offset:57344\n\t" \
  "v_mfma_f32_16x16x32_bf16 v[164:167], a[140:143], v[108:111], v[164:167]\n\t" \
  "v_mfma_f32_16x16x32_bf16 v[160:163], a[132:135], v[108:111], 0\n\t" \
  "ds_read_b64_tr_b16 a[246:247], v14 offset:58368\n\t" \
  "v_mfma_f32_16x16x32_bf16 v[160:163], a[128:131], v[104:107], v[160:163]\n\t" \
  "ds_read_b128 v[104:107], v10 offset:51200\n\t" \
  "ds_read_b128 v[108:111], v11 offset:51200\n\t" \
  "v_mfma_f32_16x16x32_bf16 v[192:195], a[128:131], v[112:115], 0\n\t" \
  "ds_read_b64_tr_b16 a[248:249], v15 offset:49152\n\t" \
  "v_mfma_f32_16x16x32_bf16 v[192:195], a[132:135], v[116:119], v[192:195]\n\t" \
  "v_mfma_f32_16x16x32_bf16 v[196:199], a[140:143], v[116:119], 0\n\t" \
  "ds_read_b64_tr_b16 a[250:251], v15 offset:50176\n\t" \
  "v_mfma_f32_16x16x32_bf16 v[196:199], a[136:139], v[112:115], v[196:199]\n\t" \
  "v_mfma_f32_16x16x32_bf16 v[200:203], a[144:147], v[112:115], 0\n\t" \
  "ds_read_b64_tr_b16 a[252:253], v15 offset:57344\n\t" \
  "v_mfma_f32_16x16x32_bf16 v[200:203], a[148:151], v[116:119], v[200:203]\n\t" \
  "v_mfma_f32_16x16x32_bf16 v[204:207], a[156:159], v[116:119], 0\n\t" \
  "ds_read_b64_tr_b16 a[254:255], v15 offset:58368\n\t" \
  "RET8_%=:\n\t" \
  "v_mfma_f32_16x16x32_bf16 v[204:207], a[152:155], v[112:115], v[204:207]\n\t" \
  "s_add_u32 m0, s68, 32768\n\t" \
  "v_lshlrev_b32 v55, 4, v55\n\t" \
  "v_mfma_f32_16x16x32_bf16 v[208:211], a[160:163], v[112:115], 0\n\t" \
  "buffer_load_dwordx4 v0, s[36:39], s64 offen offset:0 sc0 sc1 lds\n\t" \
  "v_mov_b32 v51, 0x3f80\n\t" \
  "v_mfma_f32_16x16x32_bf16 v[208:211], a[164:167], v[116:119], v[208:211]\n\t" \
  "v_lshlrev_b32 v55, v55, v51\n\t" \
  "v_lshrrev_b32 v54, 1, v54\n\t" \
  "v_mfma_f32_16x16x32_bf16 v[212:215], a[172:175], v[116:119], 0\n\t" \
  "v_cmp_eq_u32 vcc, 0, v54\n\t" \
  "v_mov_b32 v40, 0\n\t" \
  "v_mfma_f32_16x16x32_bf16 v[212:215], a[168:171], v[112:115], v[212:215]\n\t" \
  "buffer_load_dwordx4 v1, s[36:39], s64 offen offset:1024 sc0 sc1 lds\n\t" \
  "v_mov_b32 v41, 0\n\t" \
  "v_mfma_f32_16x16x32_bf16 v[216:219], a[176:179], v[112:115], 0\n\t" \
  "v_cndmask_b32 v42, 0, v55, vcc\n\t" \
  "v_cmp_eq_u32 vcc, 1, v54\n\t" \
  "v_mfma_f32_16x16x32_bf16 v[216:219], a[180:183], v[116:119], v[216:219]\n\t" \
  "v_mov_b32 v44, 0\n\t" \
  "v_mov_b32 v45, 0\n\t" \
  "v_mfma_f32_16x16x32_bf16 v[220:223], a[188:191], v[116:119], 0\n\t" \
  "buffer_load_dwordx4 v2, s[36:39], s64 offen offset:2048 sc0 sc1 lds\n\t" \
  "v_cndmask_b32 v43, 0, v55, vcc\n\t" \
  "v_mfma_f32_16x16x32_bf16 v[220:223], a[184:187], v[112:115], v[220:223]\n\t" \
  "ds_read_b128 v[112:115], v10 offset:53248\n\t" \
  "ds_read_b128 v[116:119], v11 offset:53248\n\t" \
  "v_mfma_f32_16x16x32_bf16 v[252:255], a[184:187], v[120:123], 0\n\t" \
  "v_lshl_add_u32 v9, v24, 4, s60\n\t" \
  "v_mov_b32 v18, s61\n\t" \
  "v_mfma_f32_16x16x32_bf16 v[252:255], a[188:191], v[124:127], v[252:255]\n\t" \
  "buffer_load_dwordx4 v3, s[36:39], s64 offen offset:3072 sc0 sc1 lds\n\t" \
  "v_xor_b32 v19, 0x80, v18\n\t" \
  "v_mfma_f32_16x16x32_bf16 v[248:251], a[180:183], v[124:127], 0\n\t" \
  "v_lshlrev_b32 v54, 7, v24\n\t" \
  "v_and_b32 v54, 0x80, v54\n\t" \
  "v_mfma_f32_16x16x32_bf16 v[248:251], a[176:179], v[120:123], v[248:251]\n\t" \
  "s_add_u32 m0, s68, 98304\n\t" \
  "v_xor_b32 v54, v54, v18\n\t" \
  "v_mfma_f32_16x16x32_bf16 v[244:247], a[168:171], v[120:123], 0\n\t" \
  "buffer_load_dwordx4 v4, s[40:43], s78 offen offset:0 lds\n\t" \
  "v_add_u32 v22, 64, v54\n\t" \
  "v_mfma_f32_16x16x32_bf16 v[244:247], a[172:175], v[124:127], v[244:247]\n\t" \
  "v_mov_b32 v20, 0\n\t" \
  "v_mov_b32 v21, 0\n\t" \
  "v_mfma_f32_16x16x32_bf16 v[240:243], a[164:167], v[124:127], 0\n\t" \
  "v_bfe_u32 v54, v24, 2, 3\n\t" \
  "v_and_b32 v55, 3, v24\n\t" \
  "v_mfma_f32_16x16x32_bf16 v[240:243], a[160:163], v[120:123], v[240:243]\n\t" \
  "buffer_load_dwordx4 v5, s[40:43], s78 offen offset:1024 lds\n\t" \
  "v_lshlrev_b32 v55, 7, v55\n\t" \
  "v_mfma_f32_16x16x32_bf16 v[236:239], a[152:155], v[120:123], 0\n\t" \
  "v_lshl_add_u32 v54, v54, 14, v55\n\t" \
  "v_add_u32 v23, s62, v54\n\t" \
  "v_mfma_f32_16x16x32_bf16 v[236:239], a[156:159], v[124:127], v[236:239]\n\t" \
  "s_and_b32 s62, s62, 4294836223\n\t" \
  "v_lshrrev_b32 v54, 2, v24\n\t" \
  "v_mfma_f32_16x16x32_bf16 v[232:235], a[148:151], v[124:127], 0\n\t" \
  "buffer_load_dwordx4 v6, s[40:43], s78 offen offset:2048 lds\n\t" \
  "v_lshl_add_u32 v54, v54, 14, v55\n\t" \
  "v_mfma_f32_16x16x32_bf16 v[232:235], a[144:147], v[120:123], v[232:235]\n\t" \
  "v_add_u32 v25, s62, v54\n\t" \
  "v_mfma_f32_16x16x32_bf16 v[228:231], a[136:139], v[120:123], 0\n\t" \
  "v_mfma_f32_16x16x32_bf16 v[228:231], a[140:143], v[124:127], v[228:231]\n\t" \
  "buffer_load_dwordx4 v7, s[40:43], s78 offen offset:3072 lds\n\t" \
  "v_mfma_f32_16x16x32_bf16 v[224:227], a[132:135], v[124:127], 0\n\t" \
  "v_mfma_f32_16x16x32_bf16 v[224:227], a[128:131], v[120:123], v[224:227]\n\t" \
  "ds_read_b128 v[120:123], v10 offset:55296\n\t" \
  "ds_read_b128 v[124:127], v11 offset:55296\n\t" \
  "v_mfma_f32_16x16x32_bf16 a[0:3], a[192:195], v[64:67], a[0:3]\n\t" \
  "v_mfma_f32_16x16x32_bf16 a[0:3], a[196:199], v[68:71], a[0:3]\n\t" \
  "v_mfma_f32_16x16x32_bf16 a[4:7], a[204:207], v[68:71], a[4:7]\n\t" \
  "v_mfma_f32_16x16x32_bf16 a[4:7], a[200:203], v[64:67], a[4:7]\n\t" \
  "v_mfma_f32_16x16x32_bf16 a[8:11], a[208:211], v[64:67], a[8:11]\n\t" \
  "v_mfma_f32_16x16x32_bf16 a[8:11], a[212:215], v[68:71], a[8:11]\n\t" \
  "v_mfma_f32_16x16x32_bf16 a[12:15], a[220:223], v[68:71], a[12:15]\n\t" \
  "v_mfma_f32_16x16x32_bf16 a[12:15], a[216:219], v[64:67], a[12:15]\n\t" \
  "s_waitcnt vmcnt(16) lgkmcnt(0)\n\t" \
  "s_barrier\n\t" \
  "v_mfma_f32_16x16x32_bf16 a[16:19], a[224:227], v[64:67], a[16:19]\n\t" \
  "ds_read_b64_tr_b16 a[128:129], v12 offset:0\n\t" \
  "v_mfma_f32_16x16x32_bf16 a[16:19], a[228:231], v[68:71], a[16:19]\n\t" \
  "v_mfma_f32_16x16x32_bf16 a[20:23], a[236:239], v[68:71], a[20:23]\n\t" \
  "ds_read_b64_tr_b16 a[130:131], v12 offset:1024\n\t" \
  "v_mfma_f32_16x16x32_bf16 a[20:23], a[232:235], v[64:67], a[20:23]\n\t" \
  "v_mfma_f32_16x16x32_bf16 a[24:27], a[240:243], v[64:67], a[24:27]\n\t" \
  "ds_read_b64_tr_b16 a[132:133], v12 offset:8192\n\t" \
  "v_mfma_f32_16x16x32_bf16 a[24:27], a[244:247], v[68:71], a[24:27]\n\t" \
  "v_mfma_f32_16x16x32_bf16 a[28:31], a[252:255], v[68:71], a[28:31]\n\t" \
  "ds_read_b64_tr_b16 a[134:135], v12 offset:9216\n\t" \
  "v_mfma_f32_16x16x32_bf16 a[28:31], a[248:251], v[64:67], a[28:31]\n\t" \
  "ds_read_b128 v[64:67], v10 offset:0\n\t" \
  "ds_read_b128 v[68:71], v11 offset:0\n\t" \
  "v_mfma_f32_16x16x32_bf16 a[60:63], a[248:251], v[72:75], a[60:63]\n\t" \
  "ds_read_b64_tr_b16 a[136:137], v13 offset:0\n\t" \
  "v_mfma_f32_16x16x32_bf16 a[60:63], a[252:255], v[76:79], a[60:63]\n\t" \
  "v_mfma_f32_16x16x32_bf16 a[56:59], a[244:247], v[76:79], a[56:59]\n\t" \
  "ds_read_b64_tr_b16 a[138:139], v13 offset:1024\n\t" \
  "v_mfma_f32_16x16x32_bf16 a[56:59], a[240:243], v[72:75], a[56:59]\n\t" \
  "v_mfma_f32_16x16x32_bf16 a[52:55], a[232:235], v[72:75], a[52:55]\n\t" \
  "ds_read_b64_tr_b16 a[140:141], v13 offset:8192\n\t" \
  "v_mfma_f32_16x16x32_bf16 a[52:55], a[236:239], v[76:79], a[52:55]\n\t" \
  "v_mfma_f32_16x16x32_bf16 a[48:51], a[228:231], v[76:79], a[48:51]\n\t" \
  "ds_read_b64_tr_b16 a[142:143], v13 offset:9216\n\t" \
  "v_mfma_f32_16x16x32_bf16 a[48:51], a[224:227], v[72:75], a[48:51]\n\t" \
  "s_sub_u32 s87, s87, 1\n\t" \
  "v_mfma_f32_16x16x32_bf16 a[44:47], a[216:219], v[72:75], a[44:47]\n\t" \
  "ds_read_b64_tr_b16 a[144:145], v14 offset:0\n\t" \
  "v_mfma_f32_16x16x32_bf16 a[44:47], a[220:223], v[76:79], a[44:47]\n\t" \
  "s_cbranch_scc1 EVT11_%=\n\t" \
  "v_mfma_f32_16x16x32_bf16 a[40:43], a[212:215], v[76:79], a[40:43]\n\t" \
  "ds_read_b64_tr_b16 a[146:147], v14 offset:1024\n\t" \
  "v_mfma_f32_16x16x32_bf16 a[40:43], a[208:211], v[72:75], a[40:43]\n\t" \
  "v_mfma_f32_16x16x32_bf16 a[36:39], a[200:203], v[72:75], a[36:39]\n\t" \
  "ds_read_b64_tr_b16 a[148:149], v14 offset:8192\n\t" \
  "v_mfma_f32_16x16x32_bf16 a[36:39], a[204:207], v[76:79], a[36:39]\n\t" \
  "v_mfma_f32_16x16x32_bf16 a[32:35], a[196:199], v[76:79], a[32:35]\n\t" \
  "ds_read_b64_tr_b16 a[150:151], v14 offset:9216\n\t" \
  "v_mfma_f32_16x16x32_bf16 a[32:35], a[192:195], v[72:75], a[32:35]\n\t" \
  "ds_read_b128 v[72:75], v10 offset:2048\n\t" \
  "ds_read_b128 v[76:79], v11 offset:2048\n\t" \
  "v_mfma_f32_16x16x32_bf16 a[64:67], a[192:195], v[80:83], a[64:67]\n\t" \
  "ds_read_b64_tr_b16 a[152:153], v15 offset:0\n\t" \
  "v_mfma_f32_16x16x32_bf16 a[64:67], a[196:199], v[84:87], a[64:67]\n\t" \
  "v_mfma_f32_16x16x32_bf16 a[68:71], a[204:207], v[84:87], a[68:71]\n\t" \
  "ds_read_b64_tr_b16 a[154:155], v15 offset:1024\n\t" \
  "v_mfma_f32_16x16x32_bf16 a[68:71], a[200:203], v[80:83], a[68:71]\n\t" \
  "v_mfma_f32_16x16x32_bf16 a[72:75], a[208:211], v[80:83], a[72:75]\n\t" \
  "ds_read_b64_tr_b16 a[156:157], v15 offset:8192\n\t" \
  "v_mfma_f32_16x16x32_bf16 a[72:75], a[212:215], v[84:87], a[72:75]\n\t" \
  "v_mfma_f32_16x16x32_bf16 a[76:79], a[220:223], v[84:87], a[76:79]\n\t" \
  "ds_read_b64_tr_b16 a[158:159], v15 offset:9216\n\t" \
  "RET12_%=:\n\t" \
  "v_mfma_f32_16x16x32_bf16 a[76:79], a[216:219], v[80:83], a[76:79]\n\t" \
  "s_add_u32 m0, s68, 49152\n\t" \
  "v_mfma_f32_16x16x32_bf16 a[80:83], a[224:227], v[80:83], a[80:83]\n\t" \
  "buffer_load_dwordx4 v0, s[36:39], s65 offen offset:0 sc0 sc1 lds\n\t" \
  "v_mfma_f32_16x16x32_bf16 a[80:83], a[228:231], v[84:87], a[80:83]\n\t" \
  "v_mfma_f32_16x16x32_bf16 a[84:87], a[236:239], v[84:87], a[84:87]\n\t" \
  "v_mfma_f32_16x16x32_bf16 a[84:87], a[232:235], v[80:83], a[84:87]\n\t" \
  "buffer_load_dwordx4 v1, s[36:39], s65 offen offset:1024 sc0 sc1 lds\n\t" \
  "v_mfma_f32_16x16x32_bf16 a[88:91], a[240:243], v[80:83], a[88:91]\n\t" \
  "v_mfma_f32_16x16x32_bf16 a[88:91], a[244:247], v[84:87], a[88:91]\n\t" \
  "v_mfma_f32_16x16x32_bf16 a[92:95], a[252:255], v[84:87], a[92:95]\n\t" \
  "buffer_load_dwordx4 v2, s[36:39], s65 offen offset:2048 sc0 sc1 lds\n\t" \
  "v_mfma_f32_16x16x32_bf16 a[92:95], a[248:251], v[80:83], a[92:95]\n\t" \
  "ds_read_b128 v[80:83], v10 offset:4096\n\t" \
  "ds_read_b128 v[84:87], v11 offset:4096\n\t" \
  "v_mfma_f32_16x16x32_bf16 a[124:127], a[248:251], v[88:91], a[124:127]\n\t" \
  "v_mfma_f32_16x16x32_bf16 a[124:127], a[252:255], v[92:95], a[124:127]\n\t" \
  "buffer_load_dwordx4 v3, s[36:39], s65 offen offset:3072 sc0 sc1 lds\n\t" \
  "v_mfma_f32_16x16x32_bf16 a[120:123], a[244:247], v[92:95], a[120:123]\n\t" \
  "v_mfma_f32_16x16x32_bf16 a[120:123], a[240:243], v[88:91], a[120:123]\n\t" \
  "s_add_u32 m0, s68, 114688\n\t" \
  "v_mfma_f32_16x16x32_bf16 a[116:119], a[232:235], v[88:91], a[116:119]\n\t" \
  "buffer_load_dwordx4 v4, s[40:43], s79 offen offset:0 lds\n\t" \
  "v_mfma_f32_16x16x32_bf16 a[116:119], a[236:239], v[92:95], a[116:119]\n\t" \
  "v_mfma_f32_16x16x32_bf16 a[112:115], a[228:231], v[92:95], a[112:115]\n\t" \
  "v_mfma_f32_16x16x32_bf16 a[112:115], a[224:227], v[88:91], a[112:115]\n\t" \
  "buffer_load_dwordx4 v5, s[40:43], s79 offen offset:1024 lds\n\t" \
  "v_mfma_f32_16x16x32_bf16 a[108:111], a[216:219], v[88:91], a[108:111]\n\t" \
  "v_mfma_f32_16x16x32_bf16 a[108:111], a[220:223], v[92:95], a[108:111]\n\t" \
  "v_mfma_f32_16x16x32_bf16 a[104:107], a[212:215], v[92:95], a[104:107]\n\t" \
  "buffer_load_dwordx4 v6, s[40:43], s79 offen offset:2048 lds\n\t" \
  "v_mfma_f32_16x16x32_bf16 a[104:107], a[208:211], v[88:91], a[104:107]\n\t" \
  "v_mfma_f32_16x16x32_bf16 a[100:103], a[200:203], v[88:91], a[100:103]\n\t" \
  "v_mfma_f32_16x16x32_bf16 a[100:103], a[204:207], v[92:95], a[100:103]\n\t" \
  "buffer_load_dwordx4 v7, s[40:43], s79 offen offset:3072 lds\n\t" \
  "v_mfma_f32_16x16x32_bf16 a[96:99], a[196:199], v[92:95], a[96:99]\n\t" \
  "v_mfma_f32_16x16x32_bf16 a[96:99], a[192:195], v[88:91], a[96:99]\n\t" \
  "ds_read_b128 v[88:91], v10 offset:6144\n\t" \
  "ds_read_b128 v[92:95], v11 offset:6144\n\t" \
  "v_mfma_f32_16x16x32_bf16 v[128:131], a[192:195], v[96:99], v[128:131]\n\t" \
  "s_add_u32 s77, s77, 1\n\t" \
  "v_mfma_f32_16x16x32_bf16 v[128:131], a[196:199], v[100:103], v[128:131]\n\t" \
  "v_readlane_b32 s66, v16, s77\n\t" \
  "v_mfma_f32_16x16x32_bf16 v[132:135], a[204:207], v[100:103], v[132:135]\n\t" \
  "v_mfma_f32_16x16x32_bf16 v[132:135], a[200:203], v[96:99], v[132:135]\n\t" \
  "v_mfma_f32_16x16x32_bf16 v[136:139], a[208:211], v[96:99], v[136:139]\n\t" \
  "s_lshl_b32 s78, s66, 13\n\t" \
  "v_mfma_f32_16x16x32_bf16 v[136:139], a[212:215], v[100:103], v[136:139]\n\t" \
  "s_lshr_b32 s65, s66, 11\n\t" \
  "v_mfma_f32_16x16x32_bf16 v[140:143], a[220:223], v[100:103], v[140:143]\n\t" \
  "s_mul_i32 s64, s66, s67\n\t" \
  "v_mfma_f32_16x16x32_bf16 v[140:143], a[216:219], v[96:99], v[140:143]\n\t" \
  "s_waitcnt vmcnt(16) lgkmcnt(0)\n\t" \
  "s_barrier\n\t" \
  "v_mfma_f32_16x16x32_bf16 v[144:147], a[224:227], v[96:99], v[144:147]\n\t" \
  "ds_read_b64_tr_b16 a[160:161], v12 offset:16384\n\t" \
  "v_mfma_f32_16x16x32_bf16 v[144:147], a[228:231], v[100:103], v[144:147]\n\t" \
  "s_add_u32 s79, s78, 256\n\t" \
  "v_mfma_f32_16x16x32_bf16 v[148:151], a[236:239], v[100:103], v[148:151]\n\t" \
  "ds_read_b64_tr_b16 a[162:163], v12 offset:17408\n\t" \
  "v_mfma_f32_16x16x32_bf16 v[148:151], a[232:235], v[96:99], v[148:151]\n\t" \
  "s_mul_i32 s65, s65, s89\n\t" \
  "v_mfma_f32_16x16x32_bf16 v[152:155], a[240:243], v[96:99], v[152:155]\n\t" \
  "ds_read_b64_tr_b16 a[164:165], v12 offset:24576\n\t" \
  "v_mfma_f32_16x16x32_bf16 v[152:155], a[244:247], v[100:103], v[152:155]\n\t" \
  "s_add_u32 s65, s65, s64\n\t" \
  "v_mfma_f32_16x16x32_bf16 v[156:159], a[252:255], v[100:103], v[156:159]\n\t" \
  "ds_read_b64_tr_b16 a[166:167], v12 offset:25600\n\t" \
  "v_mfma_f32_16x16x32_bf16 v[156:159], a[248:251], v[96:99], v[156:159]\n\t" \
  "ds_read_b128 v[96:99], v10 offset:16384\n\t" \
  "ds_read_b128 v[100:103], v11 offset:16384\n\t" \
  "v_mfma_f32_16x16x32_bf16 v[188:191], a[248:251], v[104:107], v[188:191]\n\t" \
  "ds_read_b64_tr_b16 a[168:169], v13 offset:16384\n\t" \
  "v_mfma_f32_16x16x32_bf16 v[188:191], a[252:255], v[108:111], v[188:191]\n\t" \
  "s_add_u32 s64, s65, s69\n\t" \
  "v_mfma_f32_16x16x32_bf16 v[184:187], a[244:247], v[108:111], v[184:187]\n\t" \
  "ds_read_b64_tr_b16 a[170:171], v13 offset:17408\n\t" \
  "v_mfma_f32_16x16x32_bf16 v[184:187], a[240:243], v[104:107], v[184:187]\n\t" \
  "s_add_u32 s65, s65, s70\n\t" \
  "v_mfma_f32_16x16x32_bf16 v[180:183], a[232:235], v[104:107], v[180:183]\n\t" \
  "ds_read_b64_tr_b16 a[172:173], v13 offset:24576\n\t" \
  "v_mfma_f32_16x16x32_bf16 v[180:183], a[236:239], v[108:111], v[180:183]\n\t" \
  "v_mfma_f32_16x16x32_bf16 v[176:179], a[228:231], v[108:111], v[176:179]\n\t" \
  "ds_read_b64_tr_b16 a[174:175], v13 offset:25600\n\t" \
  "v_mfma_f32_16x16x32_bf16 v[176:179], a[224:227], v[104:107], v[176:179]\n\t" \
  "s_sub_u32 s87, s87, 1\n\t" \
  "v_mfma_f32_16x16x32_bf16 v[172:175], a[216:219], v[104:107], v[172:175]\n\t" \
  "ds_read_b64_tr_b16 a[176:177], v14 offset:16384\n\t" \
  "v_mfma_f32_16x16x32_bf16 v[172:175], a[220:223], v[108:111], v[172:175]\n\t" \
  "s_cbranch_scc1 EVT17_%=\n\t" \
  "v_mfma_f32_16x16x32_bf16 v[168:171], a[212:215], v[108:111], v[168:171]\n\t" \
  "ds_read_b64_tr_b16 a[178:179], v14 offset:17408\n\t" \
  "v_mfma_f32_16x16x32_bf16 v[168:171], a[208:211], v[104:107], v[168:171]\n\t" \
  "v_mfma_f32_16x16x32_bf16 v[164:167], a[200:203], v[104:107], v[164:167]\n\t" \
  "ds_read_b64_tr_b16 a[180:181], v14 offset:24576\n\t" \
  "v_mfma_f32_16x16x32_bf16 v[164:167], a[204:207], v[108:111], v[164:167]\n\t" \
  "v_mfma_f32_16x16x32_bf16 v[160:163], a[196:199], v[108:111], v[160:163]\n\t" \
  "ds_read_b64_tr_b16 a[182:183], v14 offset:25600\n\t" \
  "v_mfma_f32_16x16x32_bf16 v[160:163], a[192:195], v[104:107], v[160:163]\n\t" \
  "ds_read_b128 v[104:107], v10 offset:18432\n\t" \
  "ds_read_b128 v[108:111], v11 offset:18432\n\t" \
  "v_mfma_f32_16x16x32_bf16 v[192:195], a[192:195], v[112:115], v[192:195]\n\t" \
  "ds_read_b64_tr_b16 a[184:185], v15 offset:16384\n\t" \
  "v_mfma_f32_16x16x32_bf16 v[192:195], a[196:199], v[116:119], v[192:195]\n\t" \
  "v_mfma_f32_16x16x32_bf16 v[196:199], a[204:207], v[116:119], v[196:199]\n\t" \
  "ds_read_b64_tr_b16 a[186:187], v15 offset:17408\n\t" \
  "v_mfma_f32_16x16x32_bf16 v[196:199], a[200:203], v[112:115], v[196:199]\n\t" \
  "v_mfma_f32_16x16x32_bf16 v[200:203], a[208:211], v[112:115], v[200:203]\n\t" \
  "ds_read_b64_tr_b16 a[188:189], v15 offset:24576\n\t" \
  "v_mfma_f32_16x16x32_bf16 v[200:203], a[212:215], v[116:119], v[200:203]\n\t" \
  "v_mfma_f32_16x16x32_bf16 v[204:207], a[220:223], v[116:119], v[204:207]\n\t" \
  "ds_read_b64_tr_b16 a[190:191], v15 offset:25600\n\t" \
  "RET18_%=:\n\t" \
  "v_mfma_f32_16x16x32_bf16 v[204:207], a[216:219], v[112:115], v[204:207]\n\t" \
  "s_add_u32 m0, s68, 0\n\t" \
  "v_mfma_f32_16x16x32_bf16 v[208:211], a[224:227], v[112:115], v[208:211]\n\t" \
  "buffer_load_dwordx4 v0, s[36:39], s64 offen offset:0 sc0 sc1 lds\n\t" \
  "v_mfma_f32_16x16x32_bf16 v[208:211], a[228:231], v[116:119], v[208:211]\n\t" \
  "v_mfma_f32_16x16x32_bf16 v[212:215], a[236:239], v[116:119], v[212:215]\n\t" \
  "v_mfma_f32_16x16x32_bf16 v[212:215], a[232:235], v[112:115], v[212:215]\n\t" \
  "buffer_load_dwordx4 v1, s[36:39], s64 offen offset:1024 sc0 sc1 lds\n\t" \
  "v_mfma_f32_16x16x32_bf16 v[216:219], a[240:243], v[112:115], v[216:219]\n\t" \
  "v_mfma_f32_16x16x32_bf16 v[216:219], a[244:247], v[116:119], v[216:219]\n\t" \
  "v_mfma_f32_16x16x32_bf16 v[220:223], a[252:255], v[116:119], v[220:223]\n\t" \
  "buffer_load_dwordx4 v2, s[36:39], s64 offen offset:2048 sc0 sc1 lds\n\t" \
  "v_mfma_f32_16x16x32_bf16 v[220:223], a[248:251], v[112:115], v[220:223]\n\t" \
  "ds_read_b128 v[112:115], v10 offset:20480\n\t" \
  "ds_read_b128 v[116:119], v11 offset:20480\n\t" \
  "v_mfma_f32_16x16x32_bf16 v[252:255], a[248:251], v[120:123], v[252:255]\n\t" \
  "v_mfma_f32_16x16x32_bf16 v[252:255], a[252:255], v[124:127], v[252:255]\n\t" \
  "buffer_load_dwordx4 v3, s[36:39], s64 offen offset:3072 sc0 sc1 lds\n\t" \
  "v_mfma_f32_16x16x32_bf16 v[248:251], a[244:247], v[124:127], v[248:251]\n\t" \
  "v_mfma_f32_16x16x32_bf16 v[248:251], a[240:243], v[120:123], v[248:251]\n\t" \
  "s_add_u32 m0, s68, 65536\n\t" \
  "v_mfma_f32_16x16x32_bf16 v[244:247], a[232:235], v[120:123], v[244:247]\n\t" \
  "buffer_load_dwordx4 v4, s[40:43], s78 offen offset:0 lds\n\t" \
  "v_mfma_f32_16x16x32_bf16 v[244:247], a[236:239], v[124:127], v[244:247]\n\t" \
  "v_mfma_f32_16x16x32_bf16 v[240:243], a[228:231], v[124:127], v[240:243]\n\t" \
  "v_mfma_f32_16x16x32_bf16 v[240:243], a[224:227], v[120:123], v[240:243]\n\t" \
  "buffer_load_dwordx4 v5, s[40:43], s78 offen offset:1024 lds\n\t" \
  "v_mfma_f32_16x16x32_bf16 v[236:239], a[216:219], v[120:123], v[236:239]\n\t" \
  "v_mfma_f32_16x16x32_bf16 v[236:239], a[220:223], v[124:127], v[236:239]\n\t" \
  "v_mfma_f32_16x16x32_bf16 v[232:235], a[212:215], v[124:127], v[232:235]\n\t" \
  "buffer_load_dwordx4 v6, s[40:43], s78 offen offset:2048 lds\n\t" \
  "v_mfma_f32_16x16x32_bf16 v[232:235], a[208:211], v[120:123], v[232:235]\n\t" \
  "v_mfma_f32_16x16x32_bf16 v[228:231], a[200:203], v[120:123], v[228:231]\n\t" \
  "v_mfma_f32_16x16x32_bf16 v[228:231], a[204:207], v[124:127], v[228:231]\n\t" \
  "buffer_load_dwordx4 v7, s[40:43], s78 offen offset:3072 lds\n\t" \
  "v_mfma_f32_16x16x32_bf16 v[224:227], a[196:199], v[124:127], v[224:227]\n\t" \
  "v_mfma_f32_16x16x32_bf16 v[224:227], a[192:195], v[120:123], v[224:227]\n\t" \
  "ds_read_b128 v[120:123], v10 offset:22528\n\t" \
  "ds_read_b128 v[124:127], v11 offset:22528\n\t" \
  "v_mfma_f32_16x16x32_bf16 a[0:3], a[128:131], v[64:67], a[0:3]\n\t" \
  "v_mfma_f32_16x16x32_bf16 a[0:3], a[132:135], v[68:71], a[0:3]\n\t" \
  "v_mfma_f32_16x16x32_bf16 a[4:7], a[140:143], v[68:71], a[4:7]\n\t" \
  "LOOP_%=:\n\t" \
  "s_cmp_eq_u32 s71, 1\n\t" \
  "s_cbranch_scc1 LOOP1_%=\n\t" \
  "s_cmp_eq_u32 s71, 2\n\t" \
  "s_cbranch_scc1 LOOP2_%=\n\t" \
  "s_cmp_eq_u32 s71, 3\n\t" \
  "s_cbranch_scc1 LOOP3_%=\n\t" \
  ".p2align 6\n\t" \
  "LOOP0_%=:\n\t" \
  "v_mfma_f32_16x16x32_bf16 a[4:7], a[136:139], v[64:67], a[4:7]\n\t" \
  "s_add_u32 s99, s77, 1\n\t" \
  "v_mfma_f32_16x16x32_bf16 a[8:11], a[144:147], v[64:67], a[8:11]\n\t" \
  "s_add_u32 s66, s77, 2\n\t" \
  "v_mfma_f32_16x16x32_bf16 a[8:11], a[148:151], v[68:71], a[8:11]\n\t" \
  "v_mfma_f32_16x16x32_bf16 a[12:15], a[156:159], v[68:71], a[12:15]\n\t" \
  "v_mfma_f32_16x16x32_bf16 a[12:15], a[152:155], v[64:67], a[12:15]\n\t" \
  "s_waitcnt vmcnt(16) lgkmcnt(0)\n\t" \
  "s_barrier\n\t" \
  "v_mfma_f32_16x16x32_bf16 a[16:19], a[160:163], v[64:67], a[16:19]\n\t" \
  "ds_read_b64_tr_b16 a[192:193], v12 offset:32768\n\t" \
  "v_mfma_f32_16x16x32_bf16 a[16:19], a[164:167], v[68:71], a[16:19]\n\t" \
  "v_mfma_f32_16x16x32_bf16 a[20:23], a[172:175], v[68:71], a[20:23]\n\t" \
  "ds_read_b64_tr_b16 a[194:195], v12 offset:33792\n\t" \
  "v_mfma_f32_16x16x32_bf16 a[20:23], a[168:171], v[64:67], a[20:23]\n\t" \
  "s_lshl_b64 exec, 1, s66\n\t" \
  "global_load_dword v36, v16, s[50:51] sc0 sc1\n\t" \
  "s_mov_b64 exec, -1\n\t" \
  "v_mfma_f32_16x16x32_bf16 a[24:27], a[176:179], v[64:67], a[24:27]\n\t" \
  "ds_read_b64_tr_b16 a[196:197], v12 offset:40960\n\t" \
  "v_mfma_f32_16x16x32_bf16 a[24:27], a[180:183], v[68:71], a[24:27]\n\t" \
  "v_cmp_le_u32 s[92:93], s86, v36\n\t" \
  "v_mfma_f32_16x16x32_bf16 a[28:31], a[188:191], v[68:71], a[28:31]\n\t" \
  "ds_read_b64_tr_b16 a[198:199], v12 offset:41984\n\t" \
  "v_mfma_f32_16x16x32_bf16 a[28:31], a[184:187], v[64:67], a[28:31]\n\t" \
  "ds_read_b128 v[64:67], v10 offset:32768\n\t" \
  "ds_read_b128 v[68:71], v11 offset:32768\n\t" \
  "v_mfma_f32_16x16x32_bf16 a[60:63], a[184:187], v[72:75], a[60:63]\n\t" \
  "ds_read_b64_tr_b16 a[200:201], v13 offset:32768\n\t" \
  "v_mfma_f32_16x16x32_bf16 a[60:63], a[188:191], v[76:79], a[60:63]\n\t" \
  "s_or_b64 s[90:91], s[90:91], s[92:93]\n\t" \
  "v_mfma_f32_16x16x32_bf16 a[56:59], a[180:183], v[76:79], a[56:59]\n\t" \
  "ds_read_b64_tr_b16 a[202:203], v13 offset:33792\n\t" \
  "v_mfma_f32_16x16x32_bf16 a[56:59], a[176:179], v[72:75], a[56:59]\n\t" \
  "s_bitcmp1_b64 s[90:91], s99\n\t" \
  "v_mfma_f32_16x16x32_bf16 a[52:55], a[168:171], v[72:75], a[52:55]\n\t" \
  "ds_read_b64_tr_b16 a[204:205], v13 offset:40960\n\t" \
  "v_mfma_f32_16x16x32_bf16 a[52:55], a[172:175], v[76:79], a[52:55]\n\t" \
  "s_cbranch_scc0 POLL28_%=\n\t" \
  "RET27_%=:\n\t" \
  "v_mfma_f32_16x16x32_bf16 a[48:51], a[164:167], v[76:79], a[48:51]\n\t" \
  "ds_read_b64_tr_b16 a[206:207], v13 offset:41984\n\t" \
  "v_mfma_f32_16x16x32_bf16 a[48:51], a[160:163], v[72:75], a[48:51]\n\t" \
  "s_sub_u32 s87, s87, 1\n\t" \
  "v_mfma_f32_16x16x32_bf16 a[44:47], a[152:155], v[72:75], a[44:47]\n\t" \
  "ds_read_b64_tr_b16 a[208:209], v14 offset:32768\n\t" \
  "v_mfma_f32_16x16x32_bf16 a[44:47], a[156:159], v[76:79], a[44:47]\n\t" \
  "s_cbranch_scc1 EVT30_%=\n\t" \
  "v_mfma_f32_16x16x32_bf16 a[40:43], a[148:151], v[76:79], a[40:43]\n\t" \
  "ds_read_b64_tr_b16 a[210:211], v14 offset:33792\n\t" \
  "v_mfma_f32_16x16x32_bf16 a[40:43], a[144:147], v[72:75], a[40:43]\n\t" \
  "v_mfma_f32_16x16x32_bf16 a[36:39], a[136:139], v[72:75], a[36:39]\n\t" \
  "ds_read_b64_tr_b16 a[212:213], v14 offset:40960\n\t" \
  "v_mfma_f32_16x16x32_bf16 a[36:39], a[140:143], v[76:79], a[36:39]\n\t" \
  "v_mfma_f32_16x16x32_bf16 a[32:35], a[132:135], v[76:79], a[32:35]\n\t" \
  "ds_read_b64_tr_b16 a[214:215], v14 offset:41984\n\t" \
  "v_mfma_f32_16x16x32_bf16 a[32:35], a[128:131], v[72:75], a[32:35]\n\t" \
  "ds_read_b128 v[72:75], v10 offset:34816\n\t" \
  "ds_read_b128 v[76:79], v11 offset:34816\n\t" \
  "v_mfma_f32_16x16x32_bf16 a[64:67], a[128:131], v[80:83], a[64:67]\n\t" \
  "ds_read_b64_tr_b16 a[216:217], v15 offset:32768\n\t" \
  "v_mfma_f32_16x16x32_bf16 a[64:67], a[132:135], v[84:87], a[64:67]\n\t" \
  "v_mfma_f32_16x16x32_bf16 a[68:71], a[140:143], v[84:87], a[68:71]\n\t" \
  "ds_read_b64_tr_b16 a[218:219], v15 offset:33792\n\t" \
  "v_mfma_f32_16x16x32_bf16 a[68:71], a[136:139], v[80:83], a[68:71]\n\t" \
  "v_mfma_f32_16x16x32_bf16 a[72:75], a[144:147], v[80:83], a[72:75]\n\t" \
  "ds_read_b64_tr_b16 a[220:221], v15 offset:40960\n\t" \
  "v_mfma_f32_16x16x32_bf16 a[72:75], a[148:151], v[84:87], a[72:75]\n\t" \
  "v_mfma_f32_16x16x32_bf16 a[76:79], a[156:159], v[84:87], a[76:79]\n\t" \
  "ds_read_b64_tr_b16 a[222:223], v15 offset:41984\n\t" \
  "v_mfma_f32_16x16x32_bf16 a[76:79], a[152:155], v[80:83], a[76:79]\n\t" \
  "s_add_u32 m0, s68, 16384\n\t" \
  "v_mfma_f32_16x16x32_bf16 a[80:83], a[160:163], v[80:83], a[80:83]\n\t" \
  "buffer_load_dwordx4 v0, s[36:39], s65 offen offset:0 sc0 sc1 lds\n\t" \
  "v_mfma_f32_16x16x32_bf16 a[80:83], a[164:167], v[84:87], a[80:83]\n\t" \
  "v_mfma_f32_16x16x32_bf16 a[84:87], a[172:175], v[84:87], a[84:87]\n\t" \
  "RET31_%=:\n\t" \
  "v_mfma_f32_16x16x32_bf16 a[84:87], a[168:171], v[80:83], a[84:87]\n\t" \
  "buffer_load_dwordx4 v1, s[36:39], s65 offen offset:1024 sc0 sc1 lds\n\t" \
  "v_mfma_f32_16x16x32_bf16 a[88:91], a[176:179], v[80:83], a[88:91]\n\t" \
  "v_mfma_f32_16x16x32_bf16 a[88:91], a[180:183], v[84:87], a[88:91]\n\t" \
  "v_mfma_f32_16x16x32_bf16 a[92:95], a[188:191], v[84:87], a[92:95]\n\t" \
  "buffer_load_dwordx4 v2, s[36:39], s65 offen offset:2048 sc0 sc1 lds\n\t" \
  "v_mfma_f32_16x16x32_bf16 a[92:95], a[184:187], v[80:83], a[92:95]\n\t" \
  "ds_read_b128 v[80:83], v10 offset:36864\n\t" \
  "ds_read_b128 v[84:87], v11 offset:36864\n\t" \
  "v_mfma_f32_16x16x32_bf16 a[124:127], a[184:187], v[88:91], a[124:127]\n\t" \
  "v_mfma_f32_16x16x32_bf16 a[124:127], a[188:191], v[92:95], a[124:127]\n\t" \
  "buffer_load_dwordx4 v3, s[36:39], s65 offen offset:3072 sc0 sc1 lds\n\t" \
  "v_mfma_f32_16x16x32_bf16 a[120:123], a[180:183], v[92:95], a[120:123]\n\t" \
  "v_mfma_f32_16x16x32_bf16 a[120:123], a[176:179], v[88:91], a[120:123]\n\t" \
  "s_add_u32 m0, s68, 81920\n\t" \
  "v_mfma_f32_16x16x32_bf16 a[116:119], a[168:171], v[88:91], a[116:119]\n\t" \
  "buffer_load_dwordx4 v4, s[40:43], s79 offen offset:0 lds\n\t" \
  "v_mfma_f32_16x16x32_bf16 a[116:119], a[172:175], v[92:95], a[116:119]\n\t" \
  "s_bitcmp1_b64 s[90:91], s99\n\t" \
  "v_mfma_f32_16x16x32_bf16 a[112:115], a[164:167], v[92:95], a[112:115]\n\t" \
  "s_cbranch_scc0 STG29_%=\n\t" \
  "RET23_%=:\n\t" \
  "v_mfma_f32_16x16x32_bf16 a[112:115], a[160:163], v[88:91], a[112:115]\n\t" \
  "buffer_load_dwordx4 v5, s[40:43], s79 offen offset:1024 lds\n\t" \
  "v_mfma_f32_16x16x32_bf16 a[108:111], a[152:155], v[88:91], a[108:111]\n\t" \
  "v_mfma_f32_16x16x32_bf16 a[108:111], a[156:159], v[92:95], a[108:111]\n\t" \
  "v_mfma_f32_16x16x32_bf16 a[104:107], a[148:151], v[92:95], a[104:107]\n\t" \
  "buffer_load_dwordx4 v6, s[40:43], s79 offen offset:2048 lds\n\t" \
  "v_mfma_f32_16x16x32_bf16 a[104:107], a[144:147], v[88:91], a[104:107]\n\t" \
  "v_mfma_f32_16x16x32_bf16 a[100:103], a[136:139], v[88:91], a[100:103]\n\t" \
  "v_mfma_f32_16x16x32_bf16 a[100:103], a[140:143], v[92:95], a[100:103]\n\t" \
  "buffer_load_dwordx4 v7, s[40:43], s79 offen offset:3072 lds\n\t" \
  "v_mfma_f32_16x16x32_bf16 a[96:99], a[132:135], v[92:95], a[96:99]\n\t" \
  "v_mfma_f32_16x16x32_bf16 a[96:99], a[128:131], v[88:91], a[96:99]\n\t" \
  "ds_read_b128 v[88:91], v10 offset:38912\n\t" \
  "ds_read_b128 v[92:95], v11 offset:38912\n\t" \
  "v_mfma_f32_16x16x32_bf16 v[128:131], a[128:131], v[96:99], v[128:131]\n\t" \
  "s_add_u32 s77, s77, 1\n\t" \
  "v_mfma_f32_16x16x32_bf16 v[128:131], a[132:135], v[100:103], v[128:131]\n\t" \
  "v_readlane_b32 s66, v16, s77\n\t" \
  "v_mfma_f32_16x16x32_bf16 v[132:135], a[140:143], v[100:103], v[132:135]\n\t" \
  "s_cmp_eq_u32 s77, 8\n\t" \
  "v_mfma_f32_16x16x32_bf16 v[132:135], a[136:139], v[96:99], v[132:135]\n\t" \
  "s_cbranch_scc1 SW33_%=\n\t" \
  "RET34_%=:\n\t" \
  "v_mfma_f32_16x16x32_bf16 v[136:139], a[144:147], v[96:99], v[136:139]\n\t" \
  "s_lshl_b32 s78, s66, 13\n\t" \
  "v_mfma_f32_16x16x32_bf16 v[136:139], a[148:151], v[100:103], v[136:139]\n\t" \
  "s_lshr_b32 s65, s66, 11\n\t" \
  "v_mfma_f32_16x16x32_bf16 v[140:143], a[156:159], v[100:103], v[140:143]\n\t" \
  "s_mul_i32 s64, s66, s67\n\t" \
  "v_mfma_f32_16x16x32_bf16 v[140:143], a[152:155], v[96:99], v[140:143]\n\t" \
  "s_waitcnt vmcnt(16) lgkmcnt(0)\n\t" \
  "s_barrier\n\t" \
  "v_mfma_f32_16x16x32_bf16 v[144:147], a[160:163], v[96:99], v[144:147]\n\t" \
  "ds_read_b64_tr_b16 a[224:225], v12 offset:49152\n\t" \
  "v_mfma_f32_16x16x32_bf16 v[144:147], a[164:167], v[100:103], v[144:147]\n\t" \
  "s_add_u32 s79, s78, 256\n\t" \
  "v_mfma_f32_16x16x32_bf16 v[148:151], a[172:175], v[100:103], v[148:151]\n\t" \
  "ds_read_b64_tr_b16 a[226:227], v12 offset:50176\n\t" \
  "v_mfma_f32_16x16x32_bf16 v[148:151], a[168:171], v[96:99], v[148:151]\n\t" \
  "s_mul_i32 s65, s65, s89\n\t" \
  "v_mfma_f32_16x16x32_bf16 v[152:155], a[176:179], v[96:99], v[152:155]\n\t" \
  "ds_read_b64_tr_b16 a[228:229], v12 offset:57344\n\t" \
  "v_mfma_f32_16x16x32_bf16 v[152:155], a[180:183], v[100:103], v[152:155]\n\t" \
  "s_add_u32 s65, s65, s64\n\t" \
  "v_mfma_f32_16x16x32_bf16 v[156:159], a[188:191], v[100:103], v[156:159]\n\t" \
  "ds_read_b64_tr_b16 a[230:231], v12 offset:58368\n\t" \
  "v_mfma_f32_16x16x32_bf16 v[156:159], a[184:187], v[96:99], v[156:159]\n\t" \
  "ds_read_b128 v[96:99], v10 offset:49152\n\t" \
  "ds_read_b128 v[100:103], v11 offset:49152\n\t" \
  "v_mfma_f32_16x16x32_bf16 v[188:191], a[184:187], v[104:107], v[188:191]\n\t" \
  "ds_read_b64_tr_b16 a[232:233], v13 offset:49152\n\t" \
  "v_mfma_f32_16x16x32_bf16 v[188:191], a[188:191], v[108:111], v[188:191]\n\t" \
  "s_add_u32 s64, s65, s69\n\t" \
  "v_mfma_f32_16x16x32_bf16 v[184:187], a[180:183], v[108:111], v[184:187]\n\t" \
  "ds_read_b64_tr_b16 a[234:235], v13 offset:50176\n\t" \
  "v_mfma_f32_16x16x32_bf16 v[184:187], a[176:179], v[104:107], v[184:187]\n\t" \
  "s_add_u32 s65, s65, s70\n\t" \
  "v_mfma_f32_16x16x32_bf16 v[180:183], a[168:171], v[104:107], v[180:183]\n\t" \
  "ds_read_b64_tr_b16 a[236:237], v13 offset:57344\n\t" \
  "v_mfma_f32_16x16x32_bf16 v[180:183], a[172:175], v[108:111], v[180:183]\n\t" \
  "v_mfma_f32_16x16x32_bf16 v[176:179], a[164:167], v[108:111], v[176:179]\n\t" \
  "ds_read_b64_tr_b16 a[238:239], v13 offset:58368\n\t" \
  "v_mfma_f32_16x16x32_bf16 v[176:179], a[160:163], v[104:107], v[176:179]\n\t" \
  "s_sub_u32 s87, s87, 1\n\t" \
  "v_mfma_f32_16x16x32_bf16 v[172:175], a[152:155], v[104:107], v[172:175]\n\t" \
  "ds_read_b64_tr_b16 a[240:241], v14 offset:49152\n\t" \
  "v_mfma_f32_16x16x32_bf16 v[172:175], a[156:159], v[108:111], v[172:175]\n\t" \
  "s_cbranch_scc1 EVT35_%=\n\t" \
  "v_mfma_f32_16x16x32_bf16 v[168:171], a[148:151], v[108:111], v[168:171]\n\t" \
  "ds_read_b64_tr_b16 a[242:243], v14 offset:50176\n\t" \
  "v_mfma_f32_16x16x32_bf16 v[168:171], a[144:147], v[104:107], v[168:171]\n\t" \
  "v_mfma_f32_16x16x32_bf16 v[164:167], a[136:139], v[104:107], v[164:167]\n\t" \
  "ds_read_b64_tr_b16 a[244:245], v14 offset:57344\n\t" \
  "v_mfma_f32_16x16x32_bf16 v[164:167], a[140:143], v[108:111], v[164:167]\n\t" \
  "v_mfma_f32_16x16x32_bf16 v[160:163], a[132:135], v[108:111], v[160:163]\n\t" \
  "ds_read_b64_tr_b16 a[246:247], v14 offset:58368\n\t" \
  "v_mfma_f32_16x16x32_bf16 v[160:163], a[128:131], v[104:107], v[160:163]\n\t" \
  "ds_read_b128 v[104:107], v10 offset:51200\n\t" \
  "ds_read_b128 v[108:111], v11 offset:51200\n\t" \
  "v_mfma_f32_16x16x32_bf16 v[192:195], a[128:131], v[112:115], v[192:195]\n\t" \
  "ds_read_b64_tr_b16 a[248:249], v15 offset:49152\n\t" \
  "v_mfma_f32_16x16x32_bf16 v[192:195], a[132:135], v[116:119], v[192:195]\n\t" \
  "v_mfma_f32_16x16x32_bf16 v[196:199], a[140:143], v[116:119], v[196:199]\n\t" \
  "ds_read_b64_tr_b16 a[250:251], v15 offset:50176\n\t" \
  "v_mfma_f32_16x16x32_bf16 v[196:199], a[136:139], v[112:115], v[196:199]\n\t" \
  "v_mfma_f32_16x16x32_bf16 v[200:203], a[144:147], v[112:115], v[200:203]\n\t" \
  "ds_read_b64_tr_b16 a[252:253], v15 offset:57344\n\t" \
  "v_mfma_f32_16x16x32_bf16 v[200:203], a[148:151], v[116:119], v[200:203]\n\t" \
  "v_mfma_f32_16x16x32_bf16 v[204:207], a[156:159], v[116:119], v[204:207]\n\t" \
  "ds_read_b64_tr_b16 a[254:255], v15 offset:58368\n\t" \
  "v_mfma_f32_16x16x32_bf16 v[204:207], a[152:155], v[112:115], v[204:207]\n\t" \
  "s_add_u32 m0, s68, 32768\n\t" \
  "v_mfma_f32_16x16x32_bf16 v[208:211], a[160:163], v[112:115], v[208:211]\n\t" \
  "buffer_load_dwordx4 v0, s[36:39], s64 offen offset:0 sc0 sc1 lds\n\t" \
  "v_mfma_f32_16x16x32_bf16 v[208:211], a[164:167], v[116:119], v[208:211]\n\t" \
  "v_mfma_f32_16x16x32_bf16 v[212:215], a[172:175], v[116:119], v[212:215]\n\t" \
  "RET36_%=:\n\t" \
  "v_mfma_f32_16x16x32_bf16 v[212:215], a[168:171], v[112:115], v[212:215]\n\t" \
  "buffer_load_dwordx4 v1, s[36:39], s64 offen offset:1024 sc0 sc1 lds\n\t" \
  "v_mfma_f32_16x16x32_bf16 v[216:219], a[176:179], v[112:115], v[216:219]\n\t" \
  "s_mul_i32 s12, s16, 246\n\t" \
  "v_mfma_f32_16x16x32_bf16 v[216:219], a[180:183], v[116:119], v[216:219]\n\t" \
  "s_lshr_b32 s12, s12, 8\n\t" \
  "v_mfma_f32_16x16x32_bf16 v[220:223], a[188:191], v[116:119], v[220:223]\n\t" \
  "buffer_load_dwordx4 v2, s[36:39], s64 offen offset:2048 sc0 sc1 lds\n\t" \
  "v_mfma_f32_16x16x32_bf16 v[220:223], a[184:187], v[112:115], v[220:223]\n\t" \
  "ds_read_b128 v[112:115], v10 offset:53248\n\t" \
  "ds_read_b128 v[116:119], v11 offset:53248\n\t" \
  "v_mfma_f32_16x16x32_bf16 v[252:255], a[184:187], v[120:123], v[252:255]\n\t" \
  "s_cmp_eq_u32 s76, 21\n\t" \
  "s_cselect_b32 s16, s12, s16\n\t" \
  "v_mfma_f32_16x16x32_bf16 v[252:255], a[188:191], v[124:127], v[252:255]\n\t" \
  "buffer_load_dwordx4 v3, s[36:39], s64 offen offset:3072 sc0 sc1 lds\n\t" \
  "v_mfma_f32_16x16x32_bf16 v[248:251], a[180:183], v[124:127], v[248:251]\n\t" \
  "v_mfma_f32_16x16x32_bf16 v[248:251], a[176:179], v[120:123], v[248:251]\n\t" \
  "s_add_u32 m0, s68, 98304\n\t" \
  "v_mfma_f32_16x16x32_bf16 v[244:247], a[168:171], v[120:123], v[244:247]\n\t" \
  "buffer_load_dwordx4 v4, s[40:43], s78 offen offset:0 lds\n\t" \
  "v_mfma_f32_16x16x32_bf16 v[244:247], a[172:175], v[124:127], v[244:247]\n\t" \
  "s_bitcmp1_b32 4177920, s76\n\t" \
  "v_mfma_f32_16x16x32_bf16 v[240:243], a[164:167], v[124:127], v[240:243]\n\t" \
  "s_cbranch_scc1 IDL37_%=\n\t" \
  "IDLR38_%=:\n\t" \
  "v_mfma_f32_16x16x32_bf16 v[240:243], a[160:163], v[120:123], v[240:243]\n\t" \
  "buffer_load_dwordx4 v5, s[40:43], s78 offen offset:1024 lds\n\t" \
  "v_mfma_f32_16x16x32_bf16 v[236:239], a[152:155], v[120:123], v[236:239]\n\t" \
  "v_mfma_f32_16x16x32_bf16 v[236:239], a[156:159], v[124:127], v[236:239]\n\t" \
  "v_mfma_f32_16x16x32_bf16 v[232:235], a[148:151], v[124:127], v[232:235]\n\t" \
  "buffer_load_dwordx4 v6, s[40:43], s78 offen offset:2048 lds\n\t" \
  "v_mfma_f32_16x16x32_bf16 v[232:235], a[144:147], v[120:123], v[232:235]\n\t" \
  "v_mfma_f32_16x16x32_bf16 v[228:231], a[136:139], v[120:123], v[228:231]\n\t" \
  "v_mfma_f32_16x16x32_bf16 v[228:231], a[140:143], v[124:127], v[228:231]\n\t" \
  "buffer_load_dwordx4 v7, s[40:43], s78 offen offset:3072 lds\n\t" \
  "v_mfma_f32_16x16x32_bf16 v[224:227], a[132:135], v[124:127], v[224:227]\n\t" \
  "v_mfma_f32_16x16x32_bf16 v[224:227], a[128:131], v[120:123], v[224:227]\n\t" \
  "ds_read_b128 v[120:123], v10 offset:55296\n\t" \
  "ds_read_b128 v[124:127], v11 offset:55296\n\t" \
  "v_mfma_f32_16x16x32_bf16 a[0:3], a[192:195], v[64:67], a[0:3]\n\t" \
  "v_mfma_f32_16x16x32_bf16 a[0:3], a[196:199], v[68:71], a[0:3]\n\t" \
  "v_mfma_f32_16x16x32_bf16 a[4:7], a[204:207], v[68:71], a[4:7]\n\t" \
  "v_mfma_f32_16x16x32_bf16 a[4:7], a[200:203], v[64:67], a[4:7]\n\t" \
  "s_add_u32 s99, s77, 1\n\t" \
  "v_mfma_f32_16x16x32_bf16 a[8:11], a[208:211], v[64:67], a[8:11]\n\t" \
  "s_add_u32 s66, s77, 2\n\t" \
  "v_mfma_f32_16x16x32_bf16 a[8:11], a[212:215], v[68:71], a[8:11]\n\t" \
  "v_mfma_f32_16x16x32_bf16 a[12:15], a[220:223], v[68:71], a[12:15]\n\t" \
  "v_mfma_f32_16x16x32_bf16 a[12:15], a[216:219], v[64:67], a[12:15]\n\t" \
  "s_waitcnt vmcnt(16) lgkmcnt(0)\n\t" \
  "s_barrier\n\t" \
  "v_mfma_f32_16x16x32_bf16 a[16:19], a[224:227], v[64:67], a[16:19]\n\t" \
  "ds_read_b64_tr_b16 a[128:129], v12 offset:0\n\t" \
  "v_mfma_f32_16x16x32_bf16 a[16:19], a[228:231], v[68:71], a[16:19]\n\t" \
  "v_mfma_f32_16x16x32_bf16 a[20:23], a[236:239], v[68:71], a[20:23]\n\t" \
  "ds_read_b64_tr_b16 a[130:131], v12 offset:1024\n\t" \
  "v_mfma_f32_16x16x32_bf16 a[20:23], a[232:235], v[64:67], a[20:23]\n\t" \
  "s_lshl_b64 exec, 1, s66\n\t" \
  "global_load_dword v36, v16, s[50:51] sc0 sc1\n\t" \
  "s_mov_b64 exec, -1\n\t" \
  "v_mfma_f32_16x16x32_bf16 a[24:27], a[240:243], v[64:67], a[24:27]\n\t" \
  "ds_read_b64_tr_b16 a[132:133], v12 offset:8192\n\t" \
  "v_mfma_f32_16x16x32_bf16 a[24:27], a[244:247], v[68:71], a[24:27]\n\t" \
  "v_cmp_le_u32 s[92:93], s86, v36\n\t" \
  "v_mfma_f32_16x16x32_bf16 a[28:31], a[252:255], v[68:71], a[28:31]\n\t" \
  "ds_read_b64_tr_b16 a[134:135], v12 offset:9216\n\t" \
  "v_mfma_f32_16x16x32_bf16 a[28:31], a[248:251], v[64:67], a[28:31]\n\t" \
  "ds_read_b128 v[64:67], v10 offset:0\n\t" \
  "ds_read_b128 v[68:71], v11 offset:0\n\t" \
  "v_mfma_f32_16x16x32_bf16 a[60:63], a[248:251], v[72:75], a[60:63]\n\t" \
  "ds_read_b64_tr_b16 a[136:137], v13 offset:0\n\t" \
  "v_mfma_f32_16x16x32_bf16 a[60:63], a[252:255], v[76:79], a[60:63]\n\t" \
  "s_or_b64 s[90:91], s[90:91], s[92:93]\n\t" \
  "v_mfma_f32_16x16x32_bf16 a[56:59], a[244:247], v[76:79], a[56:59]\n\t" \
  "ds_read_b64_tr_b16 a[138:139], v13 offset:1024\n\t" \
  "v_mfma_f32_16x16x32_bf16 a[56:59], a[240:243], v[72:75], a[56:59]\n\t" \
  "s_bitcmp1_b64 s[90:91], s99\n\t" \
  "v_mfma_f32_16x16x32_bf16 a[52:55], a[232:235], v[72:75], a[52:55]\n\t" \
  "ds_read_b64_tr_b16 a[140:141], v13 offset:8192\n\t" \
  "v_mfma_f32_16x16x32_bf16 a[52:55], a[236:239], v[76:79], a[52:55]\n\t" \
  "s_cbranch_scc0 POLL47_%=\n\t" \
  "RET46_%=:\n\t" \
  "v_mfma_f32_16x16x32_bf16 a[48:51], a[228:231], v[76:79], a[48:51]\n\t" \
  "ds_read_b64_tr_b16 a[142:143], v13 offset:9216\n\t" \
  "v_mfma_f32_16x16x32_bf16 a[48:51], a[224:227], v[72:75], a[48:51]\n\t" \
  "s_sub_u32 s87, s87, 1\n\t" \
  "v_mfma_f32_16x16x32_bf16 a[44:47], a[216:219], v[72:75], a[44:47]\n\t" \
  "ds_read_b64_tr_b16 a[144:145], v14 offset:0\n\t" \
  "v_mfma_f32_16x16x32_bf16 a[44:47], a[220:223], v[76:79], a[44:47]\n\t" \
  "s_cbranch_scc1 EVT49_%=\n\t" \
  "v_mfma_f32_16x16x32_bf16 a[40:43], a[212:215], v[76:79], a[40:43]\n\t" \
  "ds_read_b64_tr_b16 a[146:147], v14 offset:1024\n\t" \
  "v_mfma_f32_16x16x32_bf16 a[40:43], a[208:211], v[72:75], a[40:43]\n\t" \
  "v_mfma_f32_16x16x32_bf16 a[36:39], a[200:203], v[72:75], a[36:39]\n\t" \
  "ds_read_b64_tr_b16 a[148:149], v14 offset:8192\n\t" \
  "v_mfma_f32_16x16x32_bf16 a[36:39], a[204:207], v[76:79], a[36:39]\n\t" \
  "v_mfma_f32_16x16x32_bf16 a[32:35], a[196:199], v[76:79], a[32:35]\n\t" \
  "ds_read_b64_tr_b16 a[150:151], v14 offset:9216\n\t" \
  "v_mfma_f32_16x16x32_bf16 a[32:35], a[192:195], v[72:75], a[32:35]\n\t" \
  "ds_read_b128 v[72:75], v10 offset:2048\n\t" \
  "ds_read_b128 v[76:79], v11 offset:2048\n\t" \
  "v_mfma_f32_16x16x32_bf16 a[64:67], a[192:195], v[80:83], a[64:67]\n\t" \
  "ds_read_b64_tr_b16 a[152:153], v15 offset:0\n\t" \
  "v_mfma_f32_16x16x32_bf16 a[64:67], a[196:199], v[84:87], a[64:67]\n\t" \
  "v_mfma_f32_16x16x32_bf16 a[68:71], a[204:207], v[84:87], a[68:71]\n\t" \
  "ds_read_b64_tr_b16 a[154:155], v15 offset:1024\n\t" \
  "v_mfma_f32_16x16x32_bf16 a[68:71], a[200:203], v[80:83], a[68:71]\n\t" \
  "v_mfma_f32_16x16x32_bf16 a[72:75], a[208:211], v[80:83], a[72:75]\n\t" \
  "ds_read_b64_tr_b16 a[156:157], v15 offset:8192\n\t" \
  "v_mfma_f32_16x16x32_bf16 a[72:75], a[212:215], v[84:87], a[72:75]\n\t" \
  "v_mfma_f32_16x16x32_bf16 a[76:79], a[220:223], v[84:87], a[76:79]\n\t" \
  "ds_read_b64_tr_b16 a[158:159], v15 offset:9216\n\t" \
  "v_mfma_f32_16x16x32_bf16 a[76:79], a[216:219], v[80:83], a[76:79]\n\t" \
  "s_add_u32 m0, s68, 49152\n\t" \
  "v_mfma_f32_16x16x32_bf16 a[80:83], a[224:227], v[80:83], a[80:83]\n\t" \
  "buffer_load_dwordx4 v0, s[36:39], s65 offen offset:0 sc0 sc1 lds\n\t" \
  "v_mfma_f32_16x16x32_bf16 a[80:83], a[228:231], v[84:87], a[80:83]\n\t" \
  "v_mfma_f32_16x16x32_bf16 a[84:87], a[236:239], v[84:87], a[84:87]\n\t" \
  "RET50_%=:\n\t" \
  "v_mfma_f32_16x16x32_bf16 a[84:87], a[232:235], v[80:83], a[84:87]\n\t" \
  "buffer_load_dwordx4 v1, s[36:39], s65 offen offset:1024 sc0 sc1 lds\n\t" \
  "v_mfma_f32_16x16x32_bf16 a[88:91], a[240:243], v[80:83], a[88:91]\n\t" \
  "s_memtime s[20:21]\n\t" \
  "v_mfma_f32_16x16x32_bf16 a[88:91], a[244:247], v[84:87], a[88:91]\n\t" \
  "s_memrealtime s[22:23]\n\t" \
  "v_mfma_f32_16x16x32_bf16 a[92:95], a[252:255], v[84:87], a[92:95]\n\t" \
  "buffer_load_dwordx4 v2, s[36:39], s65 offen offset:2048 sc0 sc1 lds\n\t" \
  "v_mfma_f32_16x16x32_bf16 a[92:95], a[248:251], v[80:83], a[92:95]\n\t" \
  "ds_read_b128 v[80:83], v10 offset:4096\n\t" \
  "ds_read_b128 v[84:87], v11 offset:4096\n\t" \
  "v_mfma_f32_16x16x32_bf16 a[124:127], a[248:251], v[88:91], a[124:127]\n\t" \
  "v_mfma_f32_16x16x32_bf16 a[124:127], a[252:255], v[92:95], a[124:127]\n\t" \
  "buffer_load_dwordx4 v3, s[36:39], s65 offen offset:3072 sc0 sc1 lds\n\t" \
  "v_mfma_f32_16x16x32_bf16 a[120:123], a[244:247], v[92:95], a[120:123]\n\t" \
  "v_mfma_f32_16x16x32_bf16 a[120:123], a[240:243], v[88:91], a[120:123]\n\t" \
  "s_add_u32 m0, s68, 114688\n\t" \
  "v_mfma_f32_16x16x32_bf16 a[116:119], a[232:235], v[88:91], a[116:119]\n\t" \
  "buffer_load_dwordx4 v4, s[40:43], s79 offen offset:0 lds\n\t" \
  "v_mfma_f32_16x16x32_bf16 a[116:119], a[236:239], v[92:95], a[116:119]\n\t" \
  "s_bitcmp1_b64 s[90:91], s99\n\t" \
  "v_mfma_f32_16x16x32_bf16 a[112:115], a[228:231], v[92:95], a[112:115]\n\t" \
  "s_cbranch_scc0 STG48_%=\n\t" \
  "RET42_%=:\n\t" \
  "v_mfma_f32_16x16x32_bf16 a[112:115], a[224:227], v[88:91], a[112:115]\n\t" \
  "buffer_load_dwordx4 v5, s[40:43], s79 offen offset:1024 lds\n\t" \
  "v_mfma_f32_16x16x32_bf16 a[108:111], a[216:219], v[88:91], a[108:111]\n\t" \
  "v_mfma_f32_16x16x32_bf16 a[108:111], a[220:223], v[92:95], a[108:111]\n\t" \
  "v_mfma_f32_16x16x32_bf16 a[104:107], a[212:215], v[92:95], a[104:107]\n\t" \
  "buffer_load_dwordx4 v6, s[40:43], s79 offen offset:2048 lds\n\t" \
  "v_mfma_f32_16x16x32_bf16 a[104:107], a[208:211], v[88:91], a[104:107]\n\t" \
  "v_mfma_f32_16x16x32_bf16 a[100:103], a[200:203], v[88:91], a[100:103]\n\t" \
  "v_mfma_f32_16x16x32_bf16 a[100:103], a[204:207], v[92:95], a[100:103]\n\t" \
  "buffer_load_dwordx4 v7, s[40:43], s79 offen offset:3072 lds\n\t" \
  "v_mfma_f32_16x16x32_bf16 a[96:99], a[196:199], v[92:95], a[96:99]\n\t" \
  "v_mfma_f32_16x16x32_bf16 a[96:99], a[192:195], v[88:91], a[96:99]\n\t" \
  "ds_read_b128 v[88:91], v10 offset:6144\n\t" \
  "ds_read_b128 v[92:95], v11 offset:6144\n\t" \
  "v_mfma_f32_16x16x32_bf16 v[128:131], a[192:195], v[96:99], v[128:131]\n\t" \
  "s_add_u32 s77, s77, 1\n\t" \
  "v_mfma_f32_16x16x32_bf16 v[128:131], a[196:199], v[100:103], v[128:131]\n\t" \
  "v_readlane_b32 s66, v16, s77\n\t" \
  "v_mfma_f32_16x16x32_bf16 v[132:135], a[204:207], v[100:103], v[132:135]\n\t" \
  "s_cmp_eq_u32 s77, 8\n\t" \
  "v_mfma_f32_16x16x32_bf16 v[132:135], a[200:203], v[96:99], v[132:135]\n\t" \
  "s_cbranch_scc1 SW52_%=\n\t" \
  "RET53_%=:\n\t" \
  "v_mfma_f32_16x16x32_bf16 v[136:139], a[208:211], v[96:99], v[136:139]\n\t" \
  "s_lshl_b32 s78, s66, 13\n\t" \
  "v_mfma_f32_16x16x32_bf16 v[136:139], a[212:215], v[100:103], v[136:139]\n\t" \
  "s_lshr_b32 s65, s66, 11\n\t" \
  "v_mfma_f32_16x16x32_bf16 v[140:143], a[220:223], v[100:103], v[140:143]\n\t" \
  "s_mul_i32 s64, s66, s67\n\t" \
  "v_mfma_f32_16x16x32_bf16 v[140:143], a[216:219], v[96:99], v[140:143]\n\t" \
  "s_waitcnt vmcnt(16) lgkmcnt(0)\n\t" \
  "s_barrier\n\t" \
  "v_mfma_f32_16x16x32_bf16 v[144:147], a[224:227], v[96:99], v[144:147]\n\t" \
  "ds_read_b64_tr_b16 a[160:161], v12 offset:16384\n\t" \
  "v_mfma_f32_16x16x32_bf16 v[144:147], a[228:231], v[100:103], v[144:147]\n\t" \
  "s_add_u32 s79, s78, 256\n\t" \
  "v_mfma_f32_16x16x32_bf16 v[148:151], a[236:239], v[100:103], v[148:151]\n\t" \
  "ds_read_b64_tr_b16 a[162:163], v12 offset:17408\n\t" \
  "v_mfma_f32_16x16x32_bf16 v[148:151], a[232:235], v[96:99], v[148:151]\n\t" \
  "s_mul_i32 s65, s65, s89\n\t" \
  "v_mfma_f32_16x16x32_bf16 v[152:155], a[240:243], v[96:99], v[152:155]\n\t" \
  "ds_read_b64_tr_b16 a[164:165], v12 offset:24576\n\t" \
  "v_mfma_f32_16x16x32_bf16 v[152:155], a[244:247], v[100:103], v[152:155]\n\t" \
  "s_add_u32 s65, s65, s64\n\t" \
  "v_mfma_f32_16x16x32_bf16 v[156:159], a[252:255], v[100:103], v[156:159]\n\t" \
  "ds_read_b64_tr_b16 a[166:167], v12 offset:25600\n\t" \
  "v_mfma_f32_16x16x32_bf16 v[156:159], a[248:251], v[96:99], v[156:159]\n\t" \
  "ds_read_b128 v[96:99], v10 offset:16384\n\t" \
  "ds_read_b128 v[100:103], v11 offset:16384\n\t" \
  "v_mfma_f32_16x16x32_bf16 v[188:191], a[248:251], v[104:107], v[188:191]\n\t" \
  "ds_read_b64_tr_b16 a[168:169], v13 offset:16384\n\t" \
  "v_mfma_f32_16x16x32_bf16 v[188:191], a[252:255], v[108:111], v[188:191]\n\t" \
  "s_add_u32 s64, s65, s69\n\t" \
  "v_mfma_f32_16x16x32_bf16 v[184:187], a[244:247], v[108:111], v[184:187]\n\t" \
  "ds_read_b64_tr_b16 a[170:171], v13 offset:17408\n\t" \
  "v_mfma_f32_16x16x32_bf16 v[184:187], a[240:243], v[104:107], v[184:187]\n\t" \
  "s_add_u32 s65, s65, s70\n\t" \
  "v_mfma_f32_16x16x32_bf16 v[180:183], a[232:235], v[104:107], v[180:183]\n\t" \
  "ds_read_b64_tr_b16 a[172:173], v13 offset:24576\n\t" \
  "v_mfma_f32_16x16x32_bf16 v[180:183], a[236:239], v[108:111], v[180:183]\n\t" \
  "v_mfma_f32_16x16x32_bf16 v[176:179], a[228:231], v[108:111], v[176:179]\n\t" \
  "ds_read_b64_tr_b16 a[174:175], v13 offset:25600\n\t" \
  "v_mfma_f32_16x16x32_bf16 v[176:179], a[224:227], v[104:107], v[176:179]\n\t" \
  "s_sub_u32 s87, s87, 1\n\t" \
  "v_mfma_f32_16x16x32_bf16 v[172:175], a[216:219], v[104:107], v[172:175]\n\t" \
  "ds_read_b64_tr_b16 a[176:177], v14 offset:16384\n\t" \
  "v_mfma_f32_16x16x32_bf16 v[172:175], a[220:223], v[108:111], v[172:175]\n\t" \
  "s_cbranch_scc1 EVT54_%=\n\t" \
  "v_mfma_f32_16x16x32_bf16 v[168:171], a[212:215], v[108:111], v[168:171]\n\t" \
  "ds_read_b64_tr_b16 a[178:179], v14 offset:17408\n\t" \
  "v_mfma_f32_16x16x32_bf16 v[168:171], a[208:211], v[104:107], v[168:171]\n\t" \
  "v_mfma_f32_16x16x32_bf16 v[164:167], a[200:203], v[104:107], v[164:167]\n\t" \
  "ds_read_b64_tr_b16 a[180:181], v14 offset:24576\n\t" \
  "v_mfma_f32_16x16x32_bf16 v[164:167], a[204:207], v[108:111], v[164:167]\n\t" \
  "v_mfma_f32_16x16x32_bf16 v[160:163], a[196:199], v[108:111], v[160:163]\n\t" \
  "ds_read_b64_tr_b16 a[182:183], v14 offset:25600\n\t" \
  "v_mfma_f32_16x16x32_bf16 v[160:163], a[192:195], v[104:107], v[160:163]\n\t" \
  "ds_read_b128 v[104:107], v10 offset:18432\n\t" \
  "ds_read_b128 v[108:111], v11 offset:18432\n\t" \
  "v_mfma_f32_16x16x32_bf16 v[192:195], a[192:195], v[112:115], v[192:195]\n\t" \
  "ds_read_b64_tr_b16 a[184:185], v15 offset:16384\n\t" \
  "v_mfma_f32_16x16x32_bf16 v[192:195], a[196:199], v[116:119], v[192:195]\n\t" \
  "v_mfma_f32_16x16x32_bf16 v[196:199], a[204:207], v[116:119], v[196:199]\n\t" \
  "ds_read_b64_tr_b16 a[186:187], v15 offset:17408\n\t" \
  "v_mfma_f32_16x16x32_bf16 v[196:199], a[200:203], v[112:115], v[196:199]\n\t" \
  "v_mfma_f32_16x16x32_bf16 v[200:203], a[208:211], v[112:115], v[200:203]\n\t" \
  "ds_read_b64_tr_b16 a[188:189], v15 offset:24576\n\t" \
  "v_mfma_f32_16x16x32_bf16 v[200:203], a[212:215], v[116:119], v[200:203]\n\t" \
  "v_mfma_f32_16x16x32_bf16 v[204:207], a[220:223], v[116:119], v[204:207]\n\t" \
  "ds_read_b64_tr_b16 a[190:191], v15 offset:25600\n\t" \
  "v_mfma_f32_16x16x32_bf16 v[204:207], a[216:219], v[112:115], v[204:207]\n\t" \
  "s_add_u32 m0, s68, 0\n\t" \
  "v_mfma_f32_16x16x32_bf16 v[208:211], a[224:227], v[112:115], v[208:211]\n\t" \
  "buffer_load_dwordx4 v0, s[36:39], s64 offen offset:0 sc0 sc1 lds\n\t" \
  "v_mfma_f32_16x16x32_bf16 v[208:211], a[228:231], v[116:119], v[208:211]\n\t" \
  "v_mfma_f32_16x16x32_bf16 v[212:215], a[236:239], v[116:119], v[212:215]\n\t" \
  "RET55_%=:\n\t" \
  "v_mfma_f32_16x16x32_bf16 v[212:215], a[232:235], v[112:115], v[212:215]\n\t" \
  "buffer_load_dwordx4 v1, s[36:39], s64 offen offset:1024 sc0 sc1 lds\n\t" \
  "v_mfma_f32_16x16x32_bf16 v[216:219], a[240:243], v[112:115], v[216:219]\n\t" \
  "s_sub_u32 s24, s20, s14\n\t" \
  "s_sub_u32 s25, s22, s15\n\t" \
  "v_mfma_f32_16x16x32_bf16 v[216:219], a[244:247], v[116:119], v[216:219]\n\t" \
  "s_mul_i32 s18, s24, s17\n\t" \
  "s_mul_i32 s13, s25, s16\n\t" \
  "v_mfma_f32_16x16x32_bf16 v[220:223], a[252:255], v[116:119], v[220:223]\n\t" \
  "buffer_load_dwordx4 v2, s[36:39], s64 offen offset:2048 sc0 sc1 lds\n\t" \
  "v_mfma_f32_16x16x32_bf16 v[220:223], a[248:251], v[112:115], v[220:223]\n\t" \
  "ds_read_b128 v[112:115], v10 offset:20480\n\t" \
  "ds_read_b128 v[116:119], v11 offset:20480\n\t" \
  "v_mfma_f32_16x16x32_bf16 v[252:255], a[248:251], v[120:123], v[252:255]\n\t" \
  "s_mov_b32 s14, s20\n\t" \
  "s_mov_b32 s15, s22\n\t" \
  "v_mfma_f32_16x16x32_bf16 v[252:255], a[252:255], v[124:127], v[252:255]\n\t" \
  "buffer_load_dwordx4 v3, s[36:39], s64 offen offset:3072 sc0 sc1 lds\n\t" \
  "v_mfma_f32_16x16x32_bf16 v[248:251], a[244:247], v[124:127], v[248:251]\n\t" \
  "s_cmp_lt_u32 s18, s13\n\t" \
  "s_cselect_b32 s12, 1, s19\n\t" \
  "v_mfma_f32_16x16x32_bf16 v[248:251], a[240:243], v[120:123], v[248:251]\n\t" \
  "s_add_u32 m0, s68, 65536\n\t" \
  "v_mfma_f32_16x16x32_bf16 v[244:247], a[232:235], v[120:123], v[244:247]\n\t" \
  "buffer_load_dwordx4 v4, s[40:43], s78 offen offset:0 lds\n\t" \
  "v_mfma_f32_16x16x32_bf16 v[244:247], a[236:239], v[124:127], v[244:247]\n\t" \
  "s_cmp_lg_u32 s12, 0\n\t" \
  "v_mfma_f32_16x16x32_bf16 v[240:243], a[228:231], v[124:127], v[240:243]\n\t" \
  "s_cbranch_scc1 AD56_%=\n\t" \
  "ADR57_%=:\n\t" \
  "v_mfma_f32_16x16x32_bf16 v[240:243], a[224:227], v[120:123], v[240:243]\n\t" \
  "buffer_load_dwordx4 v5, s[40:43], s78 offen offset:1024 lds\n\t" \
  "v_mfma_f32_16x16x32_bf16 v[236:239], a[216:219], v[120:123], v[236:239]\n\t" \
  "s_bitcmp1_b32 s84, s76\n\t" \
  "v_mfma_f32_16x16x32_bf16 v[236:239], a[220:223], v[124:127], v[236:239]\n\t" \
  "s_cbranch_scc1 TRG59_%=\n\t" \
  "TRGR60_%=:\n\t" \
  "v_mfma_f32_16x16x32_bf16 v[232:235], a[212:215], v[124:127], v[232:235]\n\t" \
  "buffer_load_dwordx4 v6, s[40:43], s78 offen offset:2048 lds\n\t" \
  "v_mfma_f32_16x16x32_bf16 v[232:235], a[208:211], v[120:123], v[232:235]\n\t" \
  "s_sub_u32 s76, s76, 1\n\t" \
  "v_mfma_f32_16x16x32_bf16 v[228:231], a[200:203], v[120:123], v[228:231]\n\t" \
  "s_cmp_lg_u32 s76, 0\n\t" \
  "v_mfma_f32_16x16x32_bf16 v[228:231], a[204:207], v[124:127], v[228:231]\n\t" \
  "buffer_load_dwordx4 v7, s[40:43], s78 offen offset:3072 lds\n\t" \
  "v_mfma_f32_16x16x32_bf16 v[224:227], a[196:199], v[124:127], v[224:227]\n\t" \
  "v_mfma_f32_16x16x32_bf16 v[224:227], a[192:195], v[120:123], v[224:227]\n\t" \
  "ds_read_b128 v[120:123], v10 offset:22528\n\t" \
  "ds_read_b128 v[124:127], v11 offset:22528\n\t" \
  "v_mfma_f32_16x16x32_bf16 a[0:3], a[128:131], v[64:67], a[0:3]\n\t" \
  "v_mfma_f32_16x16x32_bf16 a[0:3], a[132:135], v[68:71], a[0:3]\n\t" \
  "v_mfma_f32_16x16x32_bf16 a[4:7], a[140:143], v[68:71], a[4:7]\n\t" \
  "s_cbranch_scc1 LOOP0_%=\n\t" \
  "s_branch LEND_%=\n\t" \
  ".p2align 6\n\t" \
  "LOOP1_%=:\n\t" \
  "v_mfma_f32_16x16x32_bf16 a[4:7], a[136:139], v[64:67], a[4:7]\n\t" \
  "v_mfma_f32_16x16x32_bf16 a[8:11], a[144:147], v[64:67], a[8:11]\n\t" \
  "v_mfma_f32_16x16x32_bf16 a[8:11], a[148:151], v[68:71], a[8:11]\n\t" \
  "v_mfma_f32_16x16x32_bf16 a[12:15], a[156:159], v[68:71], a[12:15]\n\t" \
  "s_waitcnt vmcnt(16) lgkmcnt(0)\n\t" \
  "s_barrier\n\t" \
  "v_mfma_f32_16x16x32_bf16 a[12:15], a[152:155], v[64:67], a[12:15]\n\t" \
  "v_mfma_f32_16x16x32_bf16 a[16:19], a[160:163], v[64:67], a[16:19]\n\t" \
  "ds_read_b64_tr_b16 a[192:193], v12 offset:32768\n\t" \
  "v_mfma_f32_16x16x32_bf16 a[16:19], a[164:167], v[68:71], a[16:19]\n\t" \
  "v_mfma_f32_16x16x32_bf16 a[20:23], a[172:175], v[68:71], a[20:23]\n\t" \
  "ds_read_b64_tr_b16 a[194:195], v12 offset:33792\n\t" \
  "v_mfma_f32_16x16x32_bf16 a[20:23], a[168:171], v[64:67], a[20:23]\n\t" \
  "v_mfma_f32_16x16x32_bf16 a[24:27], a[176:179], v[64:67], a[24:27]\n\t" \
  "ds_read_b64_tr_b16 a[196:197], v12 offset:40960\n\t" \
  "v_mfma_f32_16x16x32_bf16 a[24:27], a[180:183], v[68:71], a[24:27]\n\t" \
  "v_mfma_f32_16x16x32_bf16 a[28:31], a[188:191], v[68:71], a[28:31]\n\t" \
  "ds_read_b64_tr_b16 a[198:199], v12 offset:41984\n\t" \
  "v_mfma_f32_16x16x32_bf16 a[28:31], a[184:187], v[64:67], a[28:31]\n\t" \
  "ds_read_b128 v[64:67], v10 offset:32768\n\t" \
  "ds_read_b128 v[68:71], v11 offset:32768\n\t" \
  "v_mfma_f32_16x16x32_bf16 a[60:63], a[184:187], v[72:75], a[60:63]\n\t" \
  "ds_read_b64_tr_b16 a[200:201], v13 offset:32768\n\t" \
  "v_mfma_f32_16x16x32_bf16 a[60:63], a[188:191], v[76:79], a[60:63]\n\t" \
  "v_mfma_f32_16x16x32_bf16 a[56:59], a[180:183], v[76:79], a[56:59]\n\t" \
  "ds_read_b64_tr_b16 a[202:203], v13 offset:33792\n\t" \
  "v_mfma_f32_16x16x32_bf16 a[56:59], a[176:179], v[72:75], a[56:59]\n\t" \
  "v_mfma_f32_16x16x32_bf16 a[52:55], a[168:171], v[72:75], a[52:55]\n\t" \
  "ds_read_b64_tr_b16 a[204:205], v13 offset:40960\n\t" \
  "v_mfma_f32_16x16x32_bf16 a[52:55], a[172:175], v[76:79], a[52:55]\n\t" \
  "v_mfma_f32_16x16x32_bf16 a[48:51], a[164:167], v[76:79], a[48:51]\n\t" \
  "ds_read_b64_tr_b16 a[206:207], v13 offset:41984\n\t" \
  "v_mfma_f32_16x16x32_bf16 a[48:51], a[160:163], v[72:75], a[48:51]\n\t" \
  "s_sub_u32 s87, s87, 1\n\t" \
  "v_mfma_f32_16x16x32_bf16 a[44:47], a[152:155], v[72:75], a[44:47]\n\t" \
  "ds_read_b64_tr_b16 a[208:209], v14 offset:32768\n\t" \
  "v_mfma_f32_16x16x32_bf16 a[44:47], a[156:159], v[76:79], a[44:47]\n\t" \
  "s_cbranch_scc1 EVT63_%=\n\t" \
  "v_mfma_f32_16x16x32_bf16 a[40:43], a[148:151], v[76:79], a[40:43]\n\t" \
  "ds_read_b64_tr_b16 a[210:211], v14 offset:33792\n\t" \
  "v_mfma_f32_16x16x32_bf16 a[40:43], a[144:147], v[72:75], a[40:43]\n\t" \
  "v_mfma_f32_16x16x32_bf16 a[36:39], a[136:139], v[72:75], a[36:39]\n\t" \
  "ds_read_b64_tr_b16 a[212:213], v14 offset:40960\n\t" \
  "v_mfma_f32_16x16x32_bf16 a[36:39], a[140:143], v[76:79], a[36:39]\n\t" \
  "v_mfma_f32_16x16x32_bf16 a[32:35], a[132:135], v[76:79], a[32:35]\n\t" \
  "ds_read_b64_tr_b16 a[214:215], v14 offset:41984\n\t" \
  "v_mfma_f32_16x16x32_bf16 a[32:35], a[128:131], v[72:75], a[32:35]\n\t" \
  "ds_read_b128 v[72:75], v10 offset:34816\n\t" \
  "ds_read_b128 v[76:79], v11 offset:34816\n\t" \
  "v_mfma_f32_16x16x32_bf16 a[64:67], a[128:131], v[80:83], a[64:67]\n\t" \
  "ds_read_b64_tr_b16 a[216:217], v15 offset:32768\n\t" \
  "v_mfma_f32_16x16x32_bf16 a[64:67], a[132:135], v[84:87], a[64:67]\n\t" \
  "v_mfma_f32_16x16x32_bf16 a[68:71], a[140:143], v[84:87], a[68:71]\n\t" \
  "ds_read_b64_tr_b16 a[218:219], v15 offset:33792\n\t" \
  "v_mfma_f32_16x16x32_bf16 a[68:71], a[136:139], v[80:83], a[68:71]\n\t" \
  "v_mfma_f32_16x16x32_bf16 a[72:75], a[144:147], v[80:83], a[72:75]\n\t" \
  "ds_read_b64_tr_b16 a[220:221], v15 offset:40960\n\t" \
  "v_mfma_f32_16x16x32_bf16 a[72:75], a[148:151], v[84:87], a[72:75]\n\t" \
  "v_mfma_f32_16x16x32_bf16 a[76:79], a[156:159], v[84:87], a[76:79]\n\t" \
  "ds_read_b64_tr_b16 a[222:223], v15 offset:41984\n\t" \
  "v_mfma_f32_16x16x32_bf16 a[76:79], a[152:155], v[80:83], a[76:79]\n\t" \
  "s_add_u32 m0, s68, 16384\n\t" \
  "v_mfma_f32_16x16x32_bf16 a[80:83], a[160:163], v[80:83], a[80:83]\n\t" \
  "buffer_load_dwordx4 v0, s[36:39], s65 offen offset:0 sc0 sc1 lds\n\t" \
  "v_mfma_f32_16x16x32_bf16 a[80:83], a[164:167], v[84:87], a[80:83]\n\t" \
  "v_mfma_f32_16x16x32_bf16 a[84:87], a[172:175], v[84:87], a[84:87]\n\t" \
  "RET64_%=:\n\t" \
  "v_mfma_f32_16x16x32_bf16 a[84:87], a[168:171], v[80:83], a[84:87]\n\t" \
  "buffer_load_dwordx4 v1, s[36:39], s65 offen offset:1024 sc0 sc1 lds\n\t" \
  "v_mfma_f32_16x16x32_bf16 a[88:91], a[176:179], v[80:83], a[88:91]\n\t" \
  "v_mfma_f32_16x16x32_bf16 a[88:91], a[180:183], v[84:87], a[88:91]\n\t" \
  "v_mfma_f32_16x16x32_bf16 a[92:95], a[188:191], v[84:87], a[92:95]\n\t" \
  "buffer_load_dwordx4 v2, s[36:39], s65 offen offset:2048 sc0 sc1 lds\n\t" \
  "v_mfma_f32_16x16x32_bf16 a[92:95], a[184:187], v[80:83], a[92:95]\n\t" \
  "ds_read_b128 v[80:83], v10 offset:36864\n\t" \
  "ds_read_b128 v[84:87], v11 offset:36864\n\t" \
  "v_mfma_f32_16x16x32_bf16 a[124:127], a[184:187], v[88:91], a[124:127]\n\t" \
  "v_mfma_f32_16x16x32_bf16 a[124:127], a[188:191], v[92:95], a[124:127]\n\t" \
  "buffer_load_dwordx4 v3, s[36:39], s65 offen offset:3072 sc0 sc1 lds\n\t" \
  "v_mfma_f32_16x16x32_bf16 a[120:123], a[180:183], v[92:95], a[120:123]\n\t" \
  "v_mfma_f32_16x16x32_bf16 a[120:123], a[176:179], v[88:91], a[120:123]\n\t" \
  "s_add_u32 m0, s68, 81920\n\t" \
  "v_mfma_f32_16x16x32_bf16 a[116:119], a[168:171], v[88:91], a[116:119]\n\t" \
  "buffer_load_dwordx4 v4, s[40:43], s79 offen offset:0 lds\n\t" \
  "v_mfma_f32_16x16x32_bf16 a[116:119], a[172:175], v[92:95], a[116:119]\n\t" \
  "v_mfma_f32_16x16x32_bf16 a[112:115], a[164:167], v[92:95], a[112:115]\n\t" \
  "v_mfma_f32_16x16x32_bf16 a[112:115], a[160:163], v[88:91], a[112:115]\n\t" \
  "buffer_load_dwordx4 v5, s[40:43], s79 offen offset:1024 lds\n\t" \
  "v_mfma_f32_16x16x32_bf16 a[108:111], a[152:155], v[88:91], a[108:111]\n\t" \
  "v_mfma_f32_16x16x32_bf16 a[108:111], a[156:159], v[92:95], a[108:111]\n\t" \
  "v_mfma_f32_16x16x32_bf16 a[104:107], a[148:151], v[92:95], a[104:107]\n\t" \
  "buffer_load_dwordx4 v6, s[40:43], s79 offen offset:2048 lds\n\t" \
  "v_mfma_f32_16x16x32_bf16 a[104:107], a[144:147], v[88:91], a[104:107]\n\t" \
  "v_mfma_f32_16x16x32_bf16 a[100:103], a[136:139], v[88:91], a[100:103]\n\t" \
  "v_mfma_f32_16x16x32_bf16 a[100:103], a[140:143], v[92:95], a[100:103]\n\t" \
  "buffer_load_dwordx4 v7, s[40:43], s79 offen offset:3072 lds\n\t" \
  "v_mfma_f32_16x16x32_bf16 a[96:99], a[132:135], v[92:95], a[96:99]\n\t" \
  "v_mfma_f32_16x16x32_bf16 a[96:99], a[128:131], v[88:91], a[96:99]\n\t" \
  "ds_read_b128 v[88:91], v10 offset:38912\n\t" \
  "ds_read_b128 v[92:95], v11 offset:38912\n\t" \
  "v_mfma_f32_16x16x32_bf16 v[128:131], a[128:131], v[96:99], v[128:131]\n\t" \
  "s_add_u32 s77, s77, 1\n\t" \
  "v_mfma_f32_16x16x32_bf16 v[128:131], a[132:135], v[100:103], v[128:131]\n\t" \
  "v_readlane_b32 s66, v16, s77\n\t" \
  "v_mfma_f32_16x16x32_bf16 v[132:135], a[140:143], v[100:103], v[132:135]\n\t" \
  "s_cmp_eq_u32 s77, 8\n\t" \
  "v_mfma_f32_16x16x32_bf16 v[132:135], a[136:139], v[96:99], v[132:135]\n\t" \
  "s_cbranch_scc1 SW66_%=\n\t" \
  "RET67_%=:\n\t" \
  "v_mfma_f32_16x16x32_bf16 v[136:139], a[144:147], v[96:99], v[136:139]\n\t" \
  "s_lshl_b32 s78, s66, 13\n\t" \
  "v_mfma_f32_16x16x32_bf16 v[136:139], a[148:151], v[100:103], v[136:139]\n\t" \
  "s_lshr_b32 s65, s66, 11\n\t" \
  "v_mfma_f32_16x16x32_bf16 v[140:143], a[156:159], v[100:103], v[140:143]\n\t" \
  "s_mul_i32 s64, s66, s67\n\t" \
  "s_waitcnt vmcnt(16) lgkmcnt(0)\n\t" \
  "s_barrier\n\t" \
  "v_mfma_f32_16x16x32_bf16 v[140:143], a[152:155], v[96:99], v[140:143]\n\t" \
  "v_mfma_f32_16x16x32_bf16 v[144:147], a[160:163], v[96:99], v[144:147]\n\t" \
  "ds_read_b64_tr_b16 a[224:225], v12 offset:49152\n\t" \
  "v_mfma_f32_16x16x32_bf16 v[144:147], a[164:167], v[100:103], v[144:147]\n\t" \
  "s_add_u32 s79, s78, 256\n\t" \
  "v_mfma_f32_16x16x32_bf16 v[148:151], a[172:175], v[100:103], v[148:151]\n\t" \
  "ds_read_b64_tr_b16 a[226:227], v12 offset:50176\n\t" \
  "v_mfma_f32_16x16x32_bf16 v[148:151], a[168:171], v[96:99], v[148:151]\n\t" \
  "s_mul_i32 s65, s65, s89\n\t" \
  "v_mfma_f32_16x16x32_bf16 v[152:155], a[176:179], v[96:99], v[152:155]\n\t" \
  "ds_read_b64_tr_b16 a[228:229], v12 offset:57344\n\t" \
  "v_mfma_f32_16x16x32_bf16 v[152:155], a[180:183], v[100:103], v[152:155]\n\t" \
  "s_add_u32 s65, s65, s64\n\t" \
  "v_mfma_f32_16x16x32_bf16 v[156:159], a[188:191], v[100:103], v[156:159]\n\t" \
  "ds_read_b64_tr_b16 a[230:231], v12 offset:58368\n\t" \
  "v_mfma_f32_16x16x32_bf16 v[156:159], a[184:187], v[96:99], v[156:159]\n\t" \
  "ds_read_b128 v[96:99], v10 offset:49152\n\t" \
  "ds_read_b128 v[100:103], v11 offset:49152\n\t" \
  "v_mfma_f32_16x16x32_bf16 v[188:191], a[184:187], v[104:107], v[188:191]\n\t" \
  "ds_read_b64_tr_b16 a[232:233], v13 offset:49152\n\t" \
  "v_mfma_f32_16x16x32_bf16 v[188:191], a[188:191], v[108:111], v[188:191]\n\t" \
  "s_add_u32 s64, s65, s69\n\t" \
  "v_mfma_f32_16x16x32_bf16 v[184:187], a[180:183], v[108:111], v[184:187]\n\t" \
  "ds_read_b64_tr_b16 a[234:235], v13 offset:50176\n\t" \
  "v_mfma_f32_16x16x32_bf16 v[184:187], a[176:179], v[104:107], v[184:187]\n\t" \
  "s_add_u32 s65, s65, s70\n\t" \
  "v_mfma_f32_16x16x32_bf16 v[180:183], a[168:171], v[104:107], v[180:183]\n\t" \
  "ds_read_b64_tr_b16 a[236:237], v13 offset:57344\n\t" \
  "v_mfma_f32_16x16x32_bf16 v[180:183], a[172:175], v[108:111], v[180:183]\n\t" \
  "v_mfma_f32_16x16x32_bf16 v[176:179], a[164:167], v[108:111], v[176:179]\n\t" \
  "ds_read_b64_tr_b16 a[238:239], v13 offset:58368\n\t" \
  "v_mfma_f32_16x16x32_bf16 v[176:179], a[160:163], v[104:107], v[176:179]\n\t" \
  "s_sub_u32 s87, s87, 1\n\t" \
  "v_mfma_f32_16x16x32_bf16 v[172:175], a[152:155], v[104:107], v[172:175]\n\t" \
  "ds_read_b64_tr_b16 a[240:241], v14 offset:49152\n\t" \
  "v_mfma_f32_16x16x32_bf16 v[172:175], a[156:159], v[108:111], v[172:175]\n\t" \
  "s_cbranch_scc1 EVT68_%=\n\t" \
  "v_mfma_f32_16x16x32_bf16 v[168:171], a[148:151], v[108:111], v[168:171]\n\t" \
  "ds_read_b64_tr_b16 a[242:243], v14 offset:50176\n\t" \
  "v_mfma_f32_16x16x32_bf16 v[168:171], a[144:147], v[104:107], v[168:171]\n\t" \
  "v_mfma_f32_16x16x32_bf16 v[164:167], a[136:139], v[104:107], v[164:167]\n\t" \
  "ds_read_b64_tr_b16 a[244:245], v14 offset:57344\n\t" \
  "v_mfma_f32_16x16x32_bf16 v[164:167], a[140:143], v[108:111], v[164:167]\n\t" \
  "v_mfma_f32_16x16x32_bf16 v[160:163], a[132:135], v[108:111], v[160:163]\n\t" \
  "ds_read_b64_tr_b16 a[246:247], v14 offset:58368\n\t" \
  "v_mfma_f32_16x16x32_bf16 v[160:163], a[128:131], v[104:107], v[160:163]\n\t" \
  "ds_read_b128 v[104:107], v10 offset:51200\n\t" \
  "ds_read_b128 v[108:111], v11 offset:51200\n\t" \
  "v_mfma_f32_16x16x32_bf16 v[192:195], a[128:131], v[112:115], v[192:195]\n\t" \
  "ds_read_b64_tr_b16 a[248:249], v15 offset:49152\n\t" \
  "v_mfma_f32_16x16x32_bf16 v[192:195], a[132:135], v[116:119], v[192:195]\n\t" \
  "v_mfma_f32_16x16x32_bf16 v[196:199], a[140:143], v[116:119], v[196:199]\n\t" \
  "ds_read_b64_tr_b16 a[250:251], v15 offset:50176\n\t" \
  "v_mfma_f32_16x16x32_bf16 v[196:199], a[136:139], v[112:115], v[196:199]\n\t" \
  "v_mfma_f32_16x16x32_bf16 v[200:203], a[144:147], v[112:115], v[200:203]\n\t" \
  "ds_read_b64_tr_b16 a[252:253], v15 offset:57344\n\t" \
  "v_mfma_f32_16x16x32_bf16 v[200:203], a[148:151], v[116:119], v[200:203]\n\t" \
  "v_mfma_f32_16x16x32_bf16 v[204:207], a[156:159], v[116:119], v[204:207]\n\t" \
  "ds_read_b64_tr_b16 a[254:255], v15 offset:58368\n\t" \
  "v_mfma_f32_16x16x32_bf16 v[204:207], a[152:155], v[112:115], v[204:207]\n\t" \
  "s_add_u32 m0, s68, 32768\n\t" \
  "v_mfma_f32_16x16x32_bf16 v[208:211], a[160:163], v[112:115], v[208:211]\n\t" \
  "buffer_load_dwordx4 v0, s[36:39], s64 offen offset:0 sc0 sc1 lds\n\t" \
  "v_mfma_f32_16x16x32_bf16 v[208:211], a[164:167], v[116:119], v[208:211]\n\t" \
  "v_mfma_f32_16x16x32_bf16 v[212:215], a[172:175], v[116:119], v[212:215]\n\t" \
  "RET69_%=:\n\t" \
  "v_mfma_f32_16x16x32_bf16 v[212:215], a[168:171], v[112:115], v[212:215]\n\t" \
  "buffer_load_dwordx4 v1, s[36:39], s64 offen offset:1024 sc0 sc1 lds\n\t" \
  "v_mfma_f32_16x16x32_bf16 v[216:219], a[176:179], v[112:115], v[216:219]\n\t" \
  "v_mfma_f32_16x16x32_bf16 v[216:219], a[180:183], v[116:119], v[216:219]\n\t" \
  "v_mfma_f32_16x16x32_bf16 v[220:223], a[188:191], v[116:119], v[220:223]\n\t" \
  "buffer_load_dwordx4 v2, s[36:39], s64 offen offset:2048 sc0 sc1 lds\n\t" \
  "v_mfma_f32_16x16x32_bf16 v[220:223], a[184:187], v[112:115], v[220:223]\n\t" \
  "ds_read_b128 v[112:115], v10 offset:53248\n\t" \
  "ds_read_b128 v[116:119], v11 offset:53248\n\t" \
  "v_mfma_f32_16x16x32_bf16 v[252:255], a[184:187], v[120:123], v[252:255]\n\t" \
  "v_mfma_f32_16x16x32_bf16 v[252:255], a[188:191], v[124:127], v[252:255]\n\t" \
  "buffer_load_dwordx4 v3, s[36:39], s64 offen offset:3072 sc0 sc1 lds\n\t" \
  "v_mfma_f32_16x16x32_bf16 v[248:251], a[180:183], v[124:127], v[248:251]\n\t" \
  "v_mfma_f32_16x16x32_bf16 v[248:251], a[176:179], v[120:123], v[248:251]\n\t" \
  "s_add_u32 m0, s68, 98304\n\t" \
  "v_mfma_f32_16x16x32_bf16 v[244:247], a[168:171], v[120:123], v[244:247]\n\t" \
  "buffer_load_dwordx4 v4, s[40:43], s78 offen offset:0 lds\n\t" \
  "v_mfma_f32_16x16x32_bf16 v[244:247], a[172:175], v[124:127], v[244:247]\n\t" \
  "v_mfma_f32_16x16x32_bf16 v[240:243], a[164:167], v[124:127], v[240:243]\n\t" \
  "v_mfma_f32_16x16x32_bf16 v[240:243], a[160:163], v[120:123], v[240:243]\n\t" \
  "buffer_load_dwordx4 v5, s[40:43], s78 offen offset:1024 lds\n\t" \
  "v_mfma_f32_16x16x32_bf16 v[236:239], a[152:155], v[120:123], v[236:239]\n\t" \
  "v_mfma_f32_16x16x32_bf16 v[236:239], a[156:159], v[124:127], v[236:239]\n\t" \
  "v_mfma_f32_16x16x32_bf16 v[232:235], a[148:151], v[124:127], v[232:235]\n\t" \
  "buffer_load_dwordx4 v6, s[40:43], s78 offen offset:2048 lds\n\t" \
  "v_mfma_f32_16x16x32_bf16 v[232:235], a[144:147], v[120:123], v[232:235]\n\t" \
  "v_mfma_f32_16x16x32_bf16 v[228:231], a[136:139], v[120:123], v[228:231]\n\t" \
  "v_mfma_f32_16x16x32_bf16 v[228:231], a[140:143], v[124:127], v[228:231]\n\t" \
  "buffer_load_dwordx4 v7, s[40:43], s78 offen offset:3072 lds\n\t" \
  "v_mfma_f32_16x16x32_bf16 v[224:227], a[132:135], v[124:127], v[224:227]\n\t" \
  "v_mfma_f32_16x16x32_bf16 v[224:227], a[128:131], v[120:123], v[224:227]\n\t" \
  "ds_read_b128 v[120:123], v10 offset:55296\n\t" \
  "ds_read_b128 v[124:127], v11 offset:55296\n\t" \
  "v_mfma_f32_16x16x32_bf16 a[0:3], a[192:195], v[64:67], a[0:3]\n\t" \
  "v_mfma_f32_16x16x32_bf16 a[0:3], a[196:199], v[68:71], a[0:3]\n\t" \
  "v_mfma_f32_16x16x32_bf16 a[4:7], a[204:207], v[68:71], a[4:7]\n\t" \
  "v_mfma_f32_16x16x32_bf16 a[4:7], a[200:203], v[64:67], a[4:7]\n\t" \
  "v_mfma_f32_16x16x32_bf16 a[8:11], a[208:211], v[64:67], a[8:11]\n\t" \
  "v_mfma_f32_16x16x32_bf16 a[8:11], a[212:215], v[68:71], a[8:11]\n\t" \
  "v_mfma_f32_16x16x32_bf16 a[12:15], a[220:223], v[68:71], a[12:15]\n\t" \
  "s_waitcnt vmcnt(16) lgkmcnt(0)\n\t" \
  "s_barrier\n\t" \
  "v_mfma_f32_16x16x32_bf16 a[12:15], a[216:219], v[64:67], a[12:15]\n\t" \
  "v_mfma_f32_16x16x32_bf16 a[16:19], a[224:227], v[64:67], a[16:19]\n\t" \
  "ds_read_b64_tr_b16 a[128:129], v12 offset:0\n\t" \
  "v_mfma_f32_16x16x32_bf16 a[16:19], a[228:231], v[68:71], a[16:19]\n\t" \
  "v_mfma_f32_16x16x32_bf16 a[20:23], a[236:239], v[68:71], a[20:23]\n\t" \
  "ds_read_b64_tr_b16 a[130:131], v12 offset:1024\n\t" \
  "v_mfma_f32_16x16x32_bf16 a[20:23], a[232:235], v[64:67], a[20:23]\n\t" \
  "v_mfma_f32_16x16x32_bf16 a[24:27], a[240:243], v[64:67], a[24:27]\n\t" \
  "ds_read_b64_tr_b16 a[132:133], v12 offset:8192\n\t" \
  "v_mfma_f32_16x16x32_bf16 a[24:27], a[244:247], v[68:71], a[24:27]\n\t" \
  "v_mfma_f32_16x16x32_bf16 a[28:31], a[252:255], v[68:71], a[28:31]\n\t" \
  "ds_read_b64_tr_b16 a[134:135], v12 offset:9216\n\t" \
  "v_mfma_f32_16x16x32_bf16 a[28:31], a[248:251], v[64:67], a[28:31]\n\t" \
  "ds_read_b128 v[64:67], v10 offset:0\n\t" \
  "ds_read_b128 v[68:71], v11 offset:0\n\t" \
  "v_mfma_f32_16x16x32_bf16 a[60:63], a[248:251], v[72:75], a[60:63]\n\t" \
  "ds_read_b64_tr_b16 a[136:137], v13 offset:0\n\t" \
  "v_mfma_f32_16x16x32_bf16 a[60:63], a[252:255], v[76:79], a[60:63]\n\t" \
  "v_mfma_f32_16x16x32_bf16 a[56:59], a[244:247], v[76:79], a[56:59]\n\t" \
  "ds_read_b64_tr_b16 a[138:139], v13 offset:1024\n\t" \
  "v_mfma_f32_16x16x32_bf16 a[56:59], a[240:243], v[72:75], a[56:59]\n\t" \
  "v_mfma_f32_16x16x32_bf16 a[52:55], a[232:235], v[72:75], a[52:55]\n\t" \
  "ds_read_b64_tr_b16 a[140:141], v13 offset:8192\n\t" \
  "v_mfma_f32_16x16x32_bf16 a[52:55], a[236:239], v[76:79], a[52:55]\n\t" \
  "v_mfma_f32_16x16x32_bf16 a[48:51], a[228:231], v[76:79], a[48:51]\n\t" \
  "ds_read_b64_tr_b16 a[142:143], v13 offset:9216\n\t" \
  "v_mfma_f32_16x16x32_bf16 a[48:51], a[224:227], v[72:75], a[48:51]\n\t" \
  "s_sub_u32 s87, s87, 1\n\t" \
  "v_mfma_f32_16x16x32_bf16 a[44:47], a[216:219], v[72:75], a[44:47]\n\t" \
  "ds_read_b64_tr_b16 a[144:145], v14 offset:0\n\t" \
  "v_mfma_f32_16x16x32_bf16 a[44:47], a[220:223], v[76:79], a[44:47]\n\t" \
  "s_cbranch_scc1 EVT71_%=\n\t" \
  "v_mfma_f32_16x16x32_bf16 a[40:43], a[212:215], v[76:79], a[40:43]\n\t" \
  "ds_read_b64_tr_b16 a[146:147], v14 offset:1024\n\t" \
  "v_mfma_f32_16x16x32_bf16 a[40:43], a[208:211], v[72:75], a[40:43]\n\t" \
  "v_mfma_f32_16x16x32_bf16 a[36:39], a[200:203], v[72:75], a[36:39]\n\t" \
  "ds_read_b64_tr_b16 a[148:149], v14 offset:8192\n\t" \
  "v_mfma_f32_16x16x32_bf16 a[36:39], a[204:207], v[76:79], a[36:39]\n\t" \
  "v_mfma_f32_16x16x32_bf16 a[32:35], a[196:199], v[76:79], a[32:35]\n\t" \
  "ds_read_b64_tr_b16 a[150:151], v14 offset:9216\n\t" \
  "v_mfma_f32_16x16x32_bf16 a[32:35], a[192:195], v[72:75], a[32:35]\n\t" \
  "ds_read_b128 v[72:75], v10 offset:2048\n\t" \
  "ds_read_b128 v[76:79], v11 offset:2048\n\t" \
  "v_mfma_f32_16x16x32_bf16 a[64:67], a[192:195], v[80:83], a[64:67]\n\t" \
  "ds_read_b64_tr_b16 a[152:153], v15 offset:0\n\t" \
  "v_mfma_f32_16x16x32_bf16 a[64:67], a[196:199], v[84:87], a[64:67]\n\t" \
  "v_mfma_f32_16x16x32_bf16 a[68:71], a[204:207], v[84:87], a[68:71]\n\t" \
  "ds_read_b64_tr_b16 a[154:155], v15 offset:1024\n\t" \
  "v_mfma_f32_16x16x32_bf16 a[68:71], a[200:203], v[80:83], a[68:71]\n\t" \
  "v_mfma_f32_16x16x32_bf16 a[72:75], a[208:211], v[80:83], a[72:75]\n\t" \
  "ds_read_b64_tr_b16 a[156:157], v15 offset:8192\n\t" \
  "v_mfma_f32_16x16x32_bf16 a[72:75], a[212:215], v[84:87], a[72:75]\n\t" \
  "v_mfma_f32_16x16x32_bf16 a[76:79], a[220:223], v[84:87], a[76:79]\n\t" \
  "ds_read_b64_tr_b16 a[158:159], v15 offset:9216\n\t" \
  "v_mfma_f32_16x16x32_bf16 a[76:79], a[216:219], v[80:83], a[76:79]\n\t" \
  "s_add_u32 m0, s68, 49152\n\t" \
  "v_mfma_f32_16x16x32_bf16 a[80:83], a[224:227], v[80:83], a[80:83]\n\t" \
  "buffer_load_dwordx4 v0, s[36:39], s65 offen offset:0 sc0 sc1 lds\n\t" \
  "v_mfma_f32_16x16x32_bf16 a[80:83], a[228:231], v[84:87], a[80:83]\n\t" \
  "v_mfma_f32_16x16x32_bf16 a[84:87], a[236:239], v[84:87], a[84:87]\n\t" \
  "RET72_%=:\n\t" \
  "v_mfma_f32_16x16x32_bf16 a[84:87], a[232:235], v[80:83], a[84:87]\n\t" \
  "buffer_load_dwordx4 v1, s[36:39], s65 offen offset:1024 sc0 sc1 lds\n\t" \
  "v_mfma_f32_16x16x32_bf16 a[88:91], a[240:243], v[80:83], a[88:91]\n\t" \
  "v_mfma_f32_16x16x32_bf16 a[88:91], a[244:247], v[84:87], a[88:91]\n\t" \
  "v_mfma_f32_16x16x32_bf16 a[92:95], a[252:255], v[84:87], a[92:95]\n\t" \
  "buffer_load_dwordx4 v2, s[36:39], s65 offen offset:2048 sc0 sc1 lds\n\t" \
  "v_mfma_f32_16x16x32_bf16 a[92:95], a[248:251], v[80:83], a[92:95]\n\t" \
  "ds_read_b128 v[80:83], v10 offset:4096\n\t" \
  "ds_read_b128 v[84:87], v11 offset:4096\n\t" \
  "v_mfma_f32_16x16x32_bf16 a[124:127], a[248:251], v[88:91], a[124:127]\n\t" \
  "v_mfma_f32_16x16x32_bf16 a[124:127], a[252:255], v[92:95], a[124:127]\n\t" \
  "buffer_load_dwordx4 v3, s[36:39], s65 offen offset:3072 sc0 sc1 lds\n\t" \
  "v_mfma_f32_16x16x32_bf16 a[120:123], a[244:247], v[92:95], a[120:123]\n\t" \
  "v_mfma_f32_16x16x32_bf16 a[120:123], a[240:243], v[88:91], a[120:123]\n\t" \
  "s_add_u32 m0, s68, 114688\n\t" \
  "v_mfma_f32_16x16x32_bf16 a[116:119], a[232:235], v[88:91], a[116:119]\n\t" \
  "buffer_load_dwordx4 v4, s[40:43], s79 offen offset:0 lds\n\t" \
  "v_mfma_f32_16x16x32_bf16 a[116:119], a[236:239], v[92:95], a[116:119]\n\t" \
  "v_mfma_f32_16x16x32_bf16 a[112:115], a[228:231], v[92:95], a[112:115]\n\t" \
  "v_mfma_f32_16x16x32_bf16 a[112:115], a[224:227], v[88:91], a[112:115]\n\t" \
  "buffer_load_dwordx4 v5, s[40:43], s79 offen offset:1024 lds\n\t" \
  "v_mfma_f32_16x16x32_bf16 a[108:111], a[216:219], v[88:91], a[108:111]\n\t" \
  "v_mfma_f32_16x16x32_bf16 a[108:111], a[220:223], v[92:95], a[108:111]\n\t" \
  "v_mfma_f32_16x16x32_bf16 a[104:107], a[212:215], v[92:95], a[104:107]\n\t" \
  "buffer_load_dwordx4 v6, s[40:43], s79 offen offset:2048 lds\n\t" \
  "v_mfma_f32_16x16x32_bf16 a[104:107], a[208:211], v[88:91], a[104:107]\n\t" \
  "v_mfma_f32_16x16x32_bf16 a[100:103], a[200:203], v[88:91], a[100:103]\n\t" \
  "v_mfma_f32_16x16x32_bf16 a[100:103], a[204:207], v[92:95], a[100:103]\n\t" \
  "buffer_load_dwordx4 v7, s[40:43], s79 offen offset:3072 lds\n\t" \
  "v_mfma_f32_16x16x32_bf16 a[96:99], a[196:199], v[92:95], a[96:99]\n\t" \
  "v_mfma_f32_16x16x32_bf16 a[96:99], a[192:195], v[88:91], a[96:99]\n\t" \
  "ds_read_b128 v[88:91], v10 offset:6144\n\t" \
  "ds_read_b128 v[92:95], v11 offset:6144\n\t" \
  "v_mfma_f32_16x16x32_bf16 v[128:131], a[192:195], v[96:99], v[128:131]\n\t" \
  "s_add_u32 s77, s77, 1\n\t" \
  "v_mfma_f32_16x16x32_bf16 v[128:131], a[196:199], v[100:103], v[128:131]\n\t" \
  "v_readlane_b32 s66, v16, s77\n\t" \
  "v_mfma_f32_16x16x32_bf16 v[132:135], a[204:207], v[100:103], v[132:135]\n\t" \
  "s_cmp_eq_u32 s77, 8\n\t" \
  "v_mfma_f32_16x16x32_bf16 v[132:135], a[200:203], v[96:99], v[132:135]\n\t" \
  "s_cbranch_scc1 SW74_%=\n\t" \
  "RET75_%=:\n\t" \
  "v_mfma_f32_16x16x32_bf16 v[136:139], a[208:211], v[96:99], v[136:139]\n\t" \
  "s_lshl_b32 s78, s66, 13\n\t" \
  "v_mfma_f32_16x16x32_bf16 v[136:139], a[212:215], v[100:103], v[136:139]\n\t" \
  "s_lshr_b32 s65, s66, 11\n\t" \
  "v_mfma_f32_16x16x32_bf16 v[140:143], a[220:223], v[100:103], v[140:143]\n\t" \
  "s_mul_i32 s64, s66, s67\n\t" \
  "s_waitcnt vmcnt(16) lgkmcnt(0)\n\t" \
  "s_barrier\n\t" \
  "v_mfma_f32_16x16x32_bf16 v[140:143], a[216:219], v[96:99], v[140:143]\n\t" \
  "v_mfma_f32_16x16x32_bf16 v[144:147], a[224:227], v[96:99], v[144:147]\n\t" \
  "ds_read_b64_tr_b16 a[160:161], v12 offset:16384\n\t" \
  "v_mfma_f32_16x16x32_bf16 v[144:147], a[228:231], v[100:103], v[144:147]\n\t" \
  "s_add_u32 s79, s78, 256\n\t" \
  "v_mfma_f32_16x16x32_bf16 v[148:151], a[236:239], v[100:103], v[148:151]\n\t" \
  "ds_read_b64_tr_b16 a[162:163], v12 offset:17408\n\t" \
  "v_mfma_f32_16x16x32_bf16 v[148:151], a[232:235], v[96:99], v[148:151]\n\t" \
  "s_mul_i32 s65, s65, s89\n\t" \
  "v_mfma_f32_16x16x32_bf16 v[152:155], a[240:243], v[96:99], v[152:155]\n\t" \
  "ds_read_b64_tr_b16 a[164:165], v12 offset:24576\n\t" \
  "v_mfma_f32_16x16x32_bf16 v[152:155], a[244:247], v[100:103], v[152:155]\n\t" \
  "s_add_u32 s65, s65, s64\n\t" \
  "v_mfma_f32_16x16x32_bf16 v[156:159], a[252:255], v[100:103], v[156:159]\n\t" \
  "ds_read_b64_tr_b16 a[166:167], v12 offset:25600\n\t" \
  "v_mfma_f32_16x16x32_bf16 v[156:159], a[248:251], v[96:99], v[156:159]\n\t" \
  "ds_read_b128 v[96:99], v10 offset:16384\n\t" \
  "ds_read_b128 v[100:103], v11 offset:16384\n\t" \
  "v_mfma_f32_16x16x32_bf16 v[188:191], a[248:251], v[104:107], v[188:191]\n\t" \
  "ds_read_b64_tr_b16 a[168:169], v13 offset:16384\n\t" \
  "v_mfma_f32_16x16x32_bf16 v[188:191], a[252:255], v[108:111], v[188:191]\n\t" \
  "s_add_u32 s64, s65, s69\n\t" \
  "v_mfma_f32_16x16x32_bf16 v[184:187], a[244:247], v[108:111], v[184:187]\n\t" \
  "ds_read_b64_tr_b16 a[170:171], v13 offset:17408\n\t" \
  "v_mfma_f32_16x16x32_bf16 v[184:187], a[240:243], v[104:107], v[184:187]\n\t" \
  "s_add_u32 s65, s65, s70\n\t" \
  "v_mfma_f32_16x16x32_bf16 v[180:183], a[232:235], v[104:107], v[180:183]\n\t" \
  "ds_read_b64_tr_b16 a[172:173], v13 offset:24576\n\t" \
  "v_mfma_f32_16x16x32_bf16 v[180:183], a[236:239], v[108:111], v[180:183]\n\t" \
  "v_mfma_f32_16x16x32_bf16 v[176:179], a[228:231], v[108:111], v[176:179]\n\t" \
  "ds_read_b64_tr_b16 a[174:175], v13 offset:25600\n\t" \
  "v_mfma_f32_16x16x32_bf16 v[176:179], a[224:227], v[104:107], v[176:179]\n\t" \
  "s_sub_u32 s87, s87, 1\n\t" \
  "v_mfma_f32_16x16x32_bf16 v[172:175], a[216:219], v[104:107], v[172:175]\n\t" \
  "ds_read_b64_tr_b16 a[176:177], v14 offset:16384\n\t" \
  "v_mfma_f32_16x16x32_bf16 v[172:175], a[220:223], v[108:111], v[172:175]\n\t" \
  "s_cbranch_scc1 EVT76_%=\n\t" \
  "v_mfma_f32_16x16x32_bf16 v[168:171], a[212:215], v[108:111], v[168:171]\n\t" \
  "ds_read_b64_tr_b16 a[178:179], v14 offset:17408\n\t" \
  "v_mfma_f32_16x16x32_bf16 v[168:171], a[208:211], v[104:107], v[168:171]\n\t" \
  "v_mfma_f32_16x16x32_bf16 v[164:167], a[200:203], v[104:107], v[164:167]\n\t" \
  "ds_read_b64_tr_b16 a[180:181], v14 offset:24576\n\t" \
  "v_mfma_f32_16x16x32_bf16 v[164:167], a[204:207], v[108:111], v[164:167]\n\t" \
  "v_mfma_f32_16x16x32_bf16 v[160:163], a[196:199], v[108:111], v[160:163]\n\t" \
  "ds_read_b64_tr_b16 a[182:183], v14 offset:25600\n\t" \
  "v_mfma_f32_16x16x32_bf16 v[160:163], a[192:195], v[104:107], v[160:163]\n\t" \
  "ds_read_b128 v[104:107], v10 offset:18432\n\t" \
  "ds_read_b128 v[108:111], v11 offset:18432\n\t" \
  "v_mfma_f32_16x16x32_bf16 v[192:195], a[192:195], v[112:115], v[192:195]\n\t" \
  "ds_read_b64_tr_b16 a[184:185], v15 offset:16384\n\t" \
  "v_mfma_f32_16x16x32_bf16 v[192:195], a[196:199], v[116:119], v[192:195]\n\t" \
  "v_mfma_f32_16x16x32_bf16 v[196:199], a[204:207], v[116:119], v[196:199]\n\t" \
  "ds_read_b64_tr_b16 a[186:187], v15 offset:17408\n\t" \
  "v_mfma_f32_16x16x32_bf16 v[196:199], a[200:203], v[112:115], v[196:199]\n\t" \
  "v_mfma_f32_16x16x32_bf16 v[200:203], a[208:211], v[112:115], v[200:203]\n\t" \
  "ds_read_b64_tr_b16 a[188:189], v15 offset:24576\n\t" \
  "v_mfma_f32_16x16x32_bf16 v[200:203], a[212:215], v[116:119], v[200:203]\n\t" \
  "v_mfma_f32_16x16x32_bf16 v[204:207], a[220:223], v[116:119], v[204:207]\n\t" \
  "ds_read_b64_tr_b16 a[190:191], v15 offset:25600\n\t" \
  "v_mfma_f32_16x16x32_bf16 v[204:207], a[216:219], v[112:115], v[204:207]\n\t" \
  "s_add_u32 m0, s68, 0\n\t" \
  "v_mfma_f32_16x16x32_bf16 v[208:211], a[224:227], v[112:115], v[208:211]\n\t" \
  "buffer_load_dwordx4 v0, s[36:39], s64 offen offset:0 sc0 sc1 lds\n\t" \
  "v_mfma_f32_16x16x32_bf16 v[208:211], a[228:231], v[116:119], v[208:211]\n\t" \
  "v_mfma_f32_16x16x32_bf16 v[212:215], a[236:239], v[116:119], v[212:215]\n\t" \
  "RET77_%=:\n\t" \
  "v_mfma_f32_16x16x32_bf16 v[212:215], a[232:235], v[112:115], v[212:215]\n\t" \
  "buffer_load_dwordx4 v1, s[36:39], s64 offen offset:1024 sc0 sc1 lds\n\t" \
  "v_mfma_f32_16x16x32_bf16 v[216:219], a[240:243], v[112:115], v[216:219]\n\t" \
  "v_mfma_f32_16x16x32_bf16 v[216:219], a[244:247], v[116:119], v[216:219]\n\t" \
  "v_mfma_f32_16x16x32_bf16 v[220:223], a[252:255], v[116:119], v[220:223]\n\t" \
  "buffer_load_dwordx4 v2, s[36:39], s64 offen offset:2048 sc0 sc1 lds\n\t" \
  "v_mfma_f32_16x16x32_bf16 v[220:223], a[248:251], v[112:115], v[220:223]\n\t" \
  "ds_read_b128 v[112:115], v10 offset:20480\n\t" \
  "ds_read_b128 v[116:119], v11 offset:20480\n\t" \
  "v_mfma_f32_16x16x32_bf16 v[252:255], a[248:251], v[120:123], v[252:255]\n\t" \
  "v_mfma_f32_16x16x32_bf16 v[252:255], a[252:255], v[124:127], v[252:255]\n\t" \
  "buffer_load_dwordx4 v3, s[36:39], s64 offen offset:3072 sc0 sc1 lds\n\t" \
  "v_mfma_f32_16x16x32_bf16 v[248:251], a[244:247], v[124:127], v[248:251]\n\t" \
  "v_mfma_f32_16x16x32_bf16 v[248:251], a[240:243], v[120:123], v[248:251]\n\t" \
  "s_add_u32 m0, s68, 65536\n\t" \
  "v_mfma_f32_16x16x32_bf16 v[244:247], a[232:235], v[120:123], v[244:247]\n\t" \
  "buffer_load_dwordx4 v4, s[40:43], s78 offen offset:0 lds\n\t" \
  "v_mfma_f32_16x16x32_bf16 v[244:247], a[236:239], v[124:127], v[244:247]\n\t" \
  "v_mfma_f32_16x16x32_bf16 v[240:243], a[228:231], v[124:127], v[240:243]\n\t" \
  "v_mfma_f32_16x16x32_bf16 v[240:243], a[224:227], v[120:123], v[240:243]\n\t" \
  "buffer_load_dwordx4 v5, s[40:43], s78 offen offset:1024 lds\n\t" \
  "v_mfma_f32_16x16x32_bf16 v[236:239], a[216:219], v[120:123], v[236:239]\n\t" \
  "s_bitcmp1_b32 s84, s76\n\t" \
  "v_mfma_f32_16x16x32_bf16 v[236:239], a[220:223], v[124:127], v[236:239]\n\t" \
  "s_cbranch_scc1 TRG78_%=\n\t" \
  "TRGR79_%=:\n\t" \
  "v_mfma_f32_16x16x32_bf16 v[232:235], a[212:215], v[124:127], v[232:235]\n\t" \
  "buffer_load_dwordx4 v6, s[40:43], s78 offen offset:2048 lds\n\t" \
  "v_mfma_f32_16x16x32_bf16 v[232:235], a[208:211], v[120:123], v[232:235]\n\t" \
  "s_sub_u32 s76, s76, 1\n\t" \
  "v_mfma_f32_16x16x32_bf16 v[228:231], a[200:203], v[120:123], v[228:231]\n\t" \
  "s_cmp_lg_u32 s76, 0\n\t" \
  "v_mfma_f32_16x16x32_bf16 v[228:231], a[204:207], v[124:127], v[228:231]\n\t" \
  "buffer_load_dwordx4 v7, s[40:43], s78 offen offset:3072 lds\n\t" \
  "v_mfma_f32_16x16x32_bf16 v[224:227], a[196:199], v[124:127], v[224:227]\n\t" \
  "v_mfma_f32_16x16x32_bf16 v[224:227], a[192:195], v[120:123], v[224:227]\n\t" \
  "ds_read_b128 v[120:123], v10 offset:22528\n\t" \
  "ds_read_b128 v[124:127], v11 offset:22528\n\t" \
  "v_mfma_f32_16x16x32_bf16 a[0:3], a[128:131], v[64:67], a[0:3]\n\t" \
  "v_mfma_f32_16x16x32_bf16 a[0:3], a[132:135], v[68:71], a[0:3]\n\t" \
  "v_mfma_f32_16x16x32_bf16 a[4:7], a[140:143], v[68:71], a[4:7]\n\t" \
  "s_cbranch_scc1 LOOP1_%=\n\t" \
  "s_branch LEND_%=\n\t" \
  ".p2align 6\n\t" \
  "LOOP2_%=:\n\t" \
  "v_mfma_f32_16x16x32_bf16 a[4:7], a[136:139], v[64:67], a[4:7]\n\t" \
  "v_mfma_f32_16x16x32_bf16 a[8:11], a[144:147], v[64:67], a[8:11]\n\t" \
  "v_mfma_f32_16x16x32_bf16 a[8:11], a[148:151], v[68:71], a[8:11]\n\t" \
  "s_waitcnt vmcnt(16) lgkmcnt(0)\n\t" \
  "s_barrier\n\t" \
  "v_mfma_f32_16x16x32_bf16 a[12:15], a[156:159], v[68:71], a[12:15]\n\t" \
  "v_mfma_f32_16x16x32_bf16 a[12:15], a[152:155], v[64:67], a[12:15]\n\t" \
  "v_mfma_f32_16x16x32_bf16 a[16:19], a[160:163], v[64:67], a[16:19]\n\t" \
  "ds_read_b64_tr_b16 a[192:193], v12 offset:32768\n\t" \
  "v_mfma_f32_16x16x32_bf16 a[16:19], a[164:167], v[68:71], a[16:19]\n\t" \
  "v_mfma_f32_16x16x32_bf16 a[20:23], a[172:175], v[68:71], a[20:23]\n\t" \
  "ds_read_b64_tr_b16 a[194:195], v12 offset:33792\n\t" \
  "v_mfma_f32_16x16x32_bf16 a[20:23], a[168:171], v[64:67], a[20:23]\n\t" \
  "v_mfma_f32_16x16x32_bf16 a[24:27], a[176:179], v[64:67], a[24:27]\n\t" \
  "ds_read_b64_tr_b16 a[196:197], v12 offset:40960\n\t" \
  "v_mfma_f32_16x16x32_bf16 a[24:27], a[180:183], v[68:71], a[24:27]\n\t" \
  "v_mfma_f32_16x16x32_bf16 a[28:31], a[188:191], v[68:71], a[28:31]\n\t" \
  "ds_read_b64_tr_b16 a[198:199], v12 offset:41984\n\t" \
  "v_mfma_f32_16x16x32_bf16 a[28:31], a[184:187], v[64:67], a[28:31]\n\t" \
  "ds_read_b128 v[64:67], v10 offset:32768\n\t" \
  "ds_read_b128 v[68:71], v11 offset:32768\n\t" \
  "v_mfma_f32_16x16x32_bf16 a[60:63], a[184:187], v[72:75], a[60:63]\n\t" \
  "ds_read_b64_tr_b16 a[200:201], v13 offset:32768\n\t" \
  "v_mfma_f32_16x16x32_bf16 a[60:63], a[188:191], v[76:79], a[60:63]\n\t" \
  "v_mfma_f32_16x16x32_bf16 a[56:59], a[180:183], v[76:79], a[56:59]\n\t" \
  "ds_read_b64_tr_b16 a[202:203], v13 offset:33792\n\t" \
  "v_mfma_f32_16x16x32_bf16 a[56:59], a[176:179], v[72:75], a[56:59]\n\t" \
  "v_mfma_f32_16x16x32_bf16 a[52:55], a[168:171], v[72:75], a[52:55]\n\t" \
  "ds_read_b64_tr_b16 a[204:205], v13 offset:40960\n\t" \
  "v_mfma_f32_16x16x32_bf16 a[52:55], a[172:175], v[76:79], a[52:55]\n\t" \
  "v_mfma_f32_16x16x32_bf16 a[48:51], a[164:167], v[76:79], a[48:51]\n\t" \
  "ds_read_b64_tr_b16 a[206:207], v13 offset:41984\n\t" \
  "v_mfma_f32_16x16x32_bf16 a[48:51], a[160:163], v[72:75], a[48:51]\n\t" \
  "s_sub_u32 s87, s87, 1\n\t" \
  "v_mfma_f32_16x16x32_bf16 a[44:47], a[152:155], v[72:75], a[44:47]\n\t" \
  "ds_read_b64_tr_b16 a[208:209], v14 offset:32768\n\t" \
  "v_mfma_f32_16x16x32_bf16 a[44:47], a[156:159], v[76:79], a[44:47]\n\t" \
  "s_cbranch_scc1 EVT82_%=\n\t" \
  "v_mfma_f32_16x16x32_bf16 a[40:43], a[148:151], v[76:79], a[40:43]\n\t" \
  "ds_read_b64_tr_b16 a[210:211], v14 offset:33792\n\t" \
  "v_mfma_f32_16x16x32_bf16 a[40:43], a[144:147], v[72:75], a[40:43]\n\t" \
  "v_mfma_f32_16x16x32_bf16 a[36:39], a[136:139], v[72:75], a[36:39]\n\t" \
  "ds_read_b64_tr_b16 a[212:213], v14 offset:40960\n\t" \
  "v_mfma_f32_16x16x32_bf16 a[36:39], a[140:143], v[76:79], a[36:39]\n\t" \
  "v_mfma_f32_16x16x32_bf16 a[32:35], a[132:135], v[76:79], a[32:35]\n\t" \
  "ds_read_b64_tr_b16 a[214:215], v14 offset:41984\n\t" \
  "v_mfma_f32_16x16x32_bf16 a[32:35], a[128:131], v[72:75], a[32:35]\n\t" \
  "ds_read_b128 v[72:75], v10 offset:34816\n\t" \
  "ds_read_b128 v[76:79], v11 offset:34816\n\t" \
  "v_mfma_f32_16x16x32_bf16 a[64:67], a[128:131], v[80:83], a[64:67]\n\t" \
  "ds_read_b64_tr_b16 a[216:217], v15 offset:32768\n\t" \
  "v_mfma_f32_16x16x32_bf16 a[64:67], a[132:135], v[84:87], a[64:67]\n\t" \
  "v_mfma_f32_16x16x32_bf16 a[68:71], a[140:143], v[84:87], a[68:71]\n\t" \
  "ds_read_b64_tr_b16 a[218:219], v15 offset:33792\n\t" \
  "v_mfma_f32_16x16x32_bf16 a[68:71], a[136:139], v[80:83], a[68:71]\n\t" \
  "v_mfma_f32_16x16x32_bf16 a[72:75], a[144:147], v[80:83], a[72:75]\n\t" \
  "ds_read_b64_tr_b16 a[220:221], v15 offset:40960\n\t" \
  "v_mfma_f32_16x16x32_bf16 a[72:75], a[148:151], v[84:87], a[72:75]\n\t" \
  "v_mfma_f32_16x16x32_bf16 a[76:79], a[156:159], v[84:87], a[76:79]\n\t" \
  "ds_read_b64_tr_b16 a[222:223], v15 offset:41984\n\t" \
  "v_mfma_f32_16x16x32_bf16 a[76:79], a[152:155], v[80:83], a[76:79]\n\t" \
  "s_add_u32 m0, s68, 16384\n\t" \
  "v_mfma_f32_16x16x32_bf16 a[80:83], a[160:163], v[80:83], a[80:83]\n\t" \
  "buffer_load_dwordx4 v0, s[36:39], s65 offen offset:0 sc0 sc1 lds\n\t" \
  "v_mfma_f32_16x16x32_bf16 a[80:83], a[164:167], v[84:87], a[80:83]\n\t" \
  "v_mfma_f32_16x16x32_bf16 a[84:87], a[172:175], v[84:87], a[84:87]\n\t" \
  "RET83_%=:\n\t" \
  "v_mfma_f32_16x16x32_bf16 a[84:87], a[168:171], v[80:83], a[84:87]\n\t" \
  "buffer_load_dwordx4 v1, s[36:39], s65 offen offset:1024 sc0 sc1 lds\n\t" \
  "v_mfma_f32_16x16x32_bf16 a[88:91], a[176:179], v[80:83], a[88:91]\n\t" \
  "v_mfma_f32_16x16x32_bf16 a[88:91], a[180:183], v[84:87], a[88:91]\n\t" \
  "v_mfma_f32_16x16x32_bf16 a[92:95], a[188:191], v[84:87], a[92:95]\n\t" \
  "buffer_load_dwordx4 v2, s[36:39], s65 offen offset:2048 sc0 sc1 lds\n\t" \
  "v_mfma_f32_16x16x32_bf16 a[92:95], a[184:187], v[80:83], a[92:95]\n\t" \
  "ds_read_b128 v[80:83], v10 offset:36864\n\t" \
  "ds_read_b128 v[84:87], v11 offset:36864\n\t" \
  "v_mfma_f32_16x16x32_bf16 a[124:127], a[184:187], v[88:91], a[124:127]\n\t" \
  "v_mfma_f32_16x16x32_bf16 a[124:127], a[188:191], v[92:95], a[124:127]\n\t" \
  "buffer_load_dwordx4 v3, s[36:39], s65 offen offset:3072 sc0 sc1 lds\n\t" \
  "v_mfma_f32_16x16x32_bf16 a[120:123], a[180:183], v[92:95], a[120:123]\n\t" \
  "v_mfma_f32_16x16x32_bf16 a[120:123], a[176:179], v[88:91], a[120:123]\n\t" \
  "s_add_u32 m0, s68, 81920\n\t" \
  "v_mfma_f32_16x16x32_bf16 a[116:119], a[168:171], v[88:91], a[116:119]\n\t" \
  "buffer_load_dwordx4 v4, s[40:43], s79 offen offset:0 lds\n\t" \
  "v_mfma_f32_16x16x32_bf16 a[116:119], a[172:175], v[92:95], a[116:119]\n\t" \
  "v_mfma_f32_16x16x32_bf16 a[112:115], a[164:167], v[92:95], a[112:115]\n\t" \
  "v_mfma_f32_16x16x32_bf16 a[112:115], a[160:163], v[88:91], a[112:115]\n\t" \
  "buffer_load_dwordx4 v5, s[40:43], s79 offen offset:1024 lds\n\t" \
  "v_mfma_f32_16x16x32_bf16 a[108:111], a[152:155], v[88:91], a[108:111]\n\t" \
  "v_mfma_f32_16x16x32_bf16 a[108:111], a[156:159], v[92:95], a[108:111]\n\t" \
  "v_mfma_f32_16x16x32_bf16 a[104:107], a[148:151], v[92:95], a[104:107]\n\t" \
  "buffer_load_dwordx4 v6, s[40:43], s79 offen offset:2048 lds\n\t" \
  "v_mfma_f32_16x16x32_bf16 a[104:107], a[144:147], v[88:91], a[104:107]\n\t" \
  "v_mfma_f32_16x16x32_bf16 a[100:103], a[136:139], v[88:91], a[100:103]\n\t" \
  "v_mfma_f32_16x16x32_bf16 a[100:103], a[140:143], v[92:95], a[100:103]\n\t" \
  "buffer_load_dwordx4 v7, s[40:43], s79 offen offset:3072 lds\n\t" \
  "v_mfma_f32_16x16x32_bf16 a[96:99], a[132:135], v[92:95], a[96:99]\n\t" \
  "v_mfma_f32_16x16x32_bf16 a[96:99], a[128:131], v[88:91], a[96:99]\n\t" \
  "ds_read_b128 v[88:91], v10 offset:38912\n\t" \
  "ds_read_b128 v[92:95], v11 offset:38912\n\t" \
  "v_mfma_f32_16x16x32_bf16 v[128:131], a[128:131], v[96:99], v[128:131]\n\t" \
  "s_add_u32 s77, s77, 1\n\t" \
  "v_mfma_f32_16x16x32_bf16 v[128:131], a[132:135], v[100:103], v[128:131]\n\t" \
  "v_readlane_b32 s66, v16, s77\n\t" \
  "v_mfma_f32_16x16x32_bf16 v[132:135], a[140:143], v[100:103], v[132:135]\n\t" \
  "s_cmp_eq_u32 s77, 8\n\t" \
  "v_mfma_f32_16x16x32_bf16 v[132:135], a[136:139], v[96:99], v[132:135]\n\t" \
  "s_cbranch_scc1 SW85_%=\n\t" \
  "RET86_%=:\n\t" \
  "v_mfma_f32_16x16x32_bf16 v[136:139], a[144:147], v[96:99], v[136:139]\n\t" \
  "s_lshl_b32 s78, s66, 13\n\t" \
  "v_mfma_f32_16x16x32_bf16 v[136:139], a[148:151], v[100:103], v[136:139]\n\t" \
  "s_lshr_b32 s65, s66, 11\n\t" \
  "s_waitcnt vmcnt(16) lgkmcnt(0)\n\t" \
  "s_barrier\n\t" \
  "v_mfma_f32_16x16x32_bf16 v[140:143], a[156:159], v[100:103], v[140:143]\n\t" \
  "s_mul_i32 s64, s66, s67\n\t" \
  "v_mfma_f32_16x16x32_bf16 v[140:143], a[152:155], v[96:99], v[140:143]\n\t" \
  "v_mfma_f32_16x16x32_bf16 v[144:147], a[160:163], v[96:99], v[144:147]\n\t" \
  "ds_read_b64_tr_b16 a[224:225], v12 offset:49152\n\t" \
  "v_mfma_f32_16x16x32_bf16 v[144:147], a[164:167], v[100:103], v[144:147]\n\t" \
  "s_add_u32 s79, s78, 256\n\t" \
  "v_mfma_f32_16x16x32_bf16 v[148:151], a[172:175], v[100:103], v[148:151]\n\t" \
  "ds_read_b64_tr_b16 a[226:227], v12 offset:50176\n\t" \
  "v_mfma_f32_16x16x32_bf16 v[148:151], a[168:171], v[96:99], v[148:151]\n\t" \
  "s_mul_i32 s65, s65, s89\n\t" \
  "v_mfma_f32_16x16x32_bf16 v[152:155], a[176:179], v[96:99], v[152:155]\n\t" \
  "ds_read_b64_tr_b16 a[228:229], v12 offset:57344\n\t" \
  "v_mfma_f32_16x16x32_bf16 v[152:155], a[180:183], v[100:103], v[152:155]\n\t" \
  "s_add_u32 s65, s65, s64\n\t" \
  "v_mfma_f32_16x16x32_bf16 v[156:159], a[188:191], v[100:103], v[156:159]\n\t" \
  "ds_read_b64_tr_b16 a[230:231], v12 offset:58368\n\t" \
  "v_mfma_f32_16x16x32_bf16 v[156:159], a[184:187], v[96:99], v[156:159]\n\t" \
  "ds_read_b128 v[96:99], v10 offset:49152\n\t" \
  "ds_read_b128 v[100:103], v11 offset:49152\n\t" \
  "v_mfma_f32_16x16x32_bf16 v[188:191], a[184:187], v[104:107], v[188:191]\n\t" \
  "ds_read_b64_tr_b16 a[232:233], v13 offset:49152\n\t" \
  "v_mfma_f32_16x16x32_bf16 v[188:191], a[188:191], v[108:111], v[188:191]\n\t" \
  "s_add_u32 s64, s65, s69\n\t" \
  "v_mfma_f32_16x16x32_bf16 v[184:187], a[180:183], v[108:111], v[184:187]\n\t" \
  "ds_read_b64_tr_b16 a[234:235], v13 offset:50176\n\t" \
  "v_mfma_f32_16x16x32_bf16 v[184:187], a[176:179], v[104:107], v[184:187]\n\t" \
  "s_add_u32 s65, s65, s70\n\t" \
  "v_mfma_f32_16x16x32_bf16 v[180:183], a[168:171], v[104:107], v[180:183]\n\t" \
  "ds_read_b64_tr_b16 a[236:237], v13 offset:57344\n\t" \
  "v_mfma_f32_16x16x32_bf16 v[180:183], a[172:175], v[108:111], v[180:183]\n\t" \
  "v_mfma_f32_16x16x32_bf16 v[176:179], a[164:167], v[108:111], v[176:179]\n\t" \
  "ds_read_b64_tr_b16 a[238:239], v13 offset:58368\n\t" \
  "v_mfma_f32_16x16x32_bf16 v[176:179], a[160:163], v[104:107], v[176:179]\n\t" \
  "s_sub_u32 s87, s87, 1\n\t" \
  "v_mfma_f32_16x16x32_bf16 v[172:175], a[152:155], v[104:107], v[172:175]\n\t" \
  "ds_read_b64_tr_b16 a[240:241], v14 offset:49152\n\t" \
  "v_mfma_f32_16x16x32_bf16 v[172:175], a[156:159], v[108:111], v[172:175]\n\t" \
  "s_cbranch_scc1 EVT87_%=\n\t" \
  "v_mfma_f32_16x16x32_bf16 v[168:171], a[148:151], v[108:111], v[168:171]\n\t" \
  "ds_read_b64_tr_b16 a[242:243], v14 offset:50176\n\t" \
  "v_mfma_f32_16x16x32_bf16 v[168:171], a[144:147], v[104:107], v[168:171]\n\t" \
  "v_mfma_f32_16x16x32_bf16 v[164:167], a[136:139], v[104:107], v[164:167]\n\t" \
  "ds_read_b64_tr_b16 a[244:245], v14 offset:57344\n\t" \
  "v_mfma_f32_16x16x32_bf16 v[164:167], a[140:143], v[108:111], v[164:167]\n\t" \
  "v_mfma_f32_16x16x32_bf16 v[160:163], a[132:135], v[108:111], v[160:163]\n\t" \
  "ds_read_b64_tr_b16 a[246:247], v14 offset:58368\n\t" \
  "v_mfma_f32_16x16x32_bf16 v[160:163], a[128:131], v[104:107], v[160:163]\n\t" \
  "ds_read_b128 v[104:107], v10 offset:51200\n\t" \
  "ds_read_b128 v[108:111], v11 offset:51200\n\t" \
  "v_mfma_f32_16x16x32_bf16 v[192:195], a[128:131], v[112:115], v[192:195]\n\t" \
  "ds_read_b64_tr_b16 a[248:249], v15 offset:49152\n\t" \
  "v_mfma_f32_16x16x32_bf16 v[192:195], a[132:135], v[116:119], v[192:195]\n\t" \
  "v_mfma_f32_16x16x32_bf16 v[196:199], a[140:143], v[116:119], v[196:199]\n\t" \
  "ds_read_b64_tr_b16 a[250:251], v15 offset:50176\n\t" \
  "v_mfma_f32_16x16x32_bf16 v[196:199], a[136:139], v[112:115], v[196:199]\n\t" \
  "v_mfma_f32_16x16x32_bf16 v[200:203], a[144:147], v[112:115], v[200:203]\n\t" \
  "ds_read_b64_tr_b16 a[252:253], v15 offset:57344\n\t" \
  "v_mfma_f32_16x16x32_bf16 v[200:203], a[148:151], v[116:119], v[200:203]\n\t" \
  "v_mfma_f32_16x16x32_bf16 v[204:207], a[156:159], v[116:119], v[204:207]\n\t" \
  "ds_read_b64_tr_b16 a[254:255], v15 offset:58368\n\t" \
  "v_mfma_f32_16x16x32_bf16 v[204:207], a[152:155], v[112:115], v[204:207]\n\t" \
  "s_add_u32 m0, s68, 32768\n\t" \
  "v_mfma_f32_16x16x32_bf16 v[208:211], a[160:163], v[112:115], v[208:211]\n\t" \
  "buffer_load_dwordx4 v0, s[36:39], s64 offen offset:0 sc0 sc1 lds\n\t" \
  "v_mfma_f32_16x16x32_bf16 v[208:211], a[164:167], v[116:119], v[208:211]\n\t" \
  "v_mfma_f32_16x16x32_bf16 v[212:215], a[172:175], v[116:119], v[212:215]\n\t" \
  "RET88_%=:\n\t" \
  "v_mfma_f32_16x16x32_bf16 v[212:215], a[168:171], v[112:115], v[212:215]\n\t" \
  "buffer_load_dwordx4 v1, s[36:39], s64 offen offset:1024 sc0 sc1 lds\n\t" \
  "v_mfma_f32_16x16x32_bf16 v[216:219], a[176:179], v[112:115], v[216:219]\n\t" \
  "v_mfma_f32_16x16x32_bf16 v[216:219], a[180:183], v[116:119], v[216:219]\n\t" \
  "v_mfma_f32_16x16x32_bf16 v[220:223], a[188:191], v[116:119], v[220:223]\n\t" \
  "buffer_load_dwordx4 v2, s[36:39], s64 offen offset:2048 sc0 sc1 lds\n\t" \
  "v_mfma_f32_16x16x32_bf16 v[220:223], a[184:187], v[112:115], v[220:223]\n\t" \
  "ds_read_b128 v[112:115], v10 offset:53248\n\t" \
  "ds_read_b128 v[116:119], v11 offset:53248\n\t" \
  "v_mfma_f32_16x16x32_bf16 v[252:255], a[184:187], v[120:123], v[252:255]\n\t" \
  "v_mfma_f32_16x16x32_bf16 v[252:255], a[188:191], v[124:127], v[252:255]\n\t" \
  "buffer_load_dwordx4 v3, s[36:39], s64 offen offset:3072 sc0 sc1 lds\n\t" \
  "v_mfma_f32_16x16x32_bf16 v[248:251], a[180:183], v[124:127], v[248:251]\n\t" \
  "v_mfma_f32_16x16x32_bf16 v[248:251], a[176:179], v[120:123], v[248:251]\n\t" \
  "s_add_u32 m0, s68, 98304\n\t" \
  "v_mfma_f32_16x16x32_bf16 v[244:247], a[168:171], v[120:123], v[244:247]\n\t" \
  "buffer_load_dwordx4 v4, s[40:43], s78 offen offset:0 lds\n\t" \
  "v_mfma_f32_16x16x32_bf16 v[244:247], a[172:175], v[124:127], v[244:247]\n\t" \
  "v_mfma_f32_16x16x32_bf16 v[240:243], a[164:167], v[124:127], v[240:243]\n\t" \
  "v_mfma_f32_16x16x32_bf16 v[240:243], a[160:163], v[120:123], v[240:243]\n\t" \
  "buffer_load_dwordx4 v5, s[40:43], s78 offen offset:1024 lds\n\t" \
  "v_mfma_f32_16x16x32_bf16 v[236:239], a[152:155], v[120:123], v[236:239]\n\t" \
  "v_mfma_f32_16x16x32_bf16 v[236:239], a[156:159], v[124:127], v[236:239]\n\t" \
  "v_mfma_f32_16x16x32_bf16 v[232:235], a[148:151], v[124:127], v[232:235]\n\t" \
  "buffer_load_dwordx4 v6, s[40:43], s78 offen offset:2048 lds\n\t" \
  "v_mfma_f32_16x16x32_bf16 v[232:235], a[144:147], v[120:123], v[232:235]\n\t" \
  "v_mfma_f32_16x16x32_bf16 v[228:231], a[136:139], v[120:123], v[228:231]\n\t" \
  "v_mfma_f32_16x16x32_bf16 v[228:231], a[140:143], v[124:127], v[228:231]\n\t" \
  "buffer_load_dwordx4 v7, s[40:43], s78 offen offset:3072 lds\n\t" \
  "v_mfma_f32_16x16x32_bf16 v[224:227], a[132:135], v[124:127], v[224:227]\n\t" \
  "v_mfma_f32_16x16x32_bf16 v[224:227], a[128:131], v[120:123], v[224:227]\n\t" \
  "ds_read_b128 v[120:123], v10 offset:55296\n\t" \
  "ds_read_b128 v[124:127], v11 offset:55296\n\t" \
  "v_mfma_f32_16x16x32_bf16 a[0:3], a[192:195], v[64:67], a[0:3]\n\t" \
  "v_mfma_f32_16x16x32_bf16 a[0:3], a[196:199], v[68:71], a[0:3]\n\t" \
  "v_mfma_f32_16x16x32_bf16 a[4:7], a[204:207], v[68:71], a[4:7]\n\t" \
  "v_mfma_f32_16x16x32_bf16 a[4:7], a[200:203], v[64:67], a[4:7]\n\t" \
  "v_mfma_f32_16x16x32_bf16 a[8:11], a[208:211], v[64:67], a[8:11]\n\t" \
  "v_mfma_f32_16x16x32_bf16 a[8:11], a[212:215], v[68:71], a[8:11]\n\t" \
  "s_waitcnt vmcnt(16) lgkmcnt(0)\n\t" \
  "s_barrier\n\t" \
  "v_mfma_f32_16x16x32_bf16 a[12:15], a[220:223], v[68:71], a[12:15]\n\t" \
  "v_mfma_f32_16x16x32_bf16 a[12:15], a[216:219], v[64:67], a[12:15]\n\t" \
  "v_mfma_f32_16x16x32_bf16 a[16:19], a[224:227], v[64:67], a[16:19]\n\t" \
  "ds_read_b64_tr_b16 a[128:129], v12 offset:0\n\t" \
  "v_mfma_f32_16x16x32_bf16 a[16:19], a[228:231], v[68:71], a[16:19]\n\t" \
  "v_mfma_f32_16x16x32_bf16 a[20:23], a[236:239], v[68:71], a[20:23]\n\t" \
  "ds_read_b64_tr_b16 a[130:131], v12 offset:1024\n\t" \
  "v_mfma_f32_16x16x32_bf16 a[20:23], a[232:235], v[64:67], a[20:23]\n\t" \
  "v_mfma_f32_16x16x32_bf16 a[24:27], a[240:243], v[64:67], a[24:27]\n\t" \
  "ds_read_b64_tr_b16 a[132:133], v12 offset:8192\n\t" \
  "v_mfma_f32_16x16x32_bf16 a[24:27], a[244:247], v[68:71], a[24:27]\n\t" \
  "v_mfma_f32_16x16x32_bf16 a[28:31], a[252:255], v[68:71], a[28:31]\n\t" \
  "ds_read_b64_tr_b16 a[134:135], v12 offset:9216\n\t" \
  "v_mfma_f32_16x16x32_bf16 a[28:31], a[248:251], v[64:67], a[28:31]\n\t" \
  "ds_read_b128 v[64:67], v10 offset:0\n\t" \
  "ds_read_b128 v[68:71], v11 offset:0\n\t" \
  "v_mfma_f32_16x16x32_bf16 a[60:63], a[248:251], v[72:75], a[60:63]\n\t" \
  "ds_read_b64_tr_b16 a[136:137], v13 offset:0\n\t" \
  "v_mfma_f32_16x16x32_bf16 a[60:63], a[252:255], v[76:79], a[60:63]\n\t" \
  "v_mfma_f32_16x16x32_bf16 a[56:59], a[244:247], v[76:79], a[56:59]\n\t" \
  "ds_read_b64_tr_b16 a[138:139], v13 offset:1024\n\t" \
  "v_mfma_f32_16x16x32_bf16 a[56:59], a[240:243], v[72:75], a[56:59]\n\t" \
  "v_mfma_f32_16x16x32_bf16 a[52:55], a[232:235], v[72:75], a[52:55]\n\t" \
  "ds_read_b64_tr_b16 a[140:141], v13 offset:8192\n\t" \
  "v_mfma_f32_16x16x32_bf16 a[52:55], a[236:239], v[76:79], a[52:55]\n\t" \
  "v_mfma_f32_16x16x32_bf16 a[48:51], a[228:231], v[76:79], a[48:51]\n\t" \
  "ds_read_b64_tr_b16 a[142:143], v13 offset:9216\n\t" \
  "v_mfma_f32_16x16x32_bf16 a[48:51], a[224:227], v[72:75], a[48:51]\n\t" \
  "s_sub_u32 s87, s87, 1\n\t" \
  "v_mfma_f32_16x16x32_bf16 a[44:47], a[216:219], v[72:75], a[44:47]\n\t" \
  "ds_read_b64_tr_b16 a[144:145], v14 offset:0\n\t" \
  "v_mfma_f32_16x16x32_bf16 a[44:47], a[220:223], v[76:79], a[44:47]\n\t" \
  "s_cbranch_scc1 EVT90_%=\n\t" \
  "v_mfma_f32_16x16x32_bf16 a[40:43], a[212:215], v[76:79], a[40:43]\n\t" \
  "ds_read_b64_tr_b16 a[146:147], v14 offset:1024\n\t" \
  "v_mfma_f32_16x16x32_bf16 a[40:43], a[208:211], v[72:75], a[40:43]\n\t" \
  "v_mfma_f32_16x16x32_bf16 a[36:39], a[200:203], v[72:75], a[36:39]\n\t" \
  "ds_read_b64_tr_b16 a[148:149], v14 offset:8192\n\t" \
  "v_mfma_f32_16x16x32_bf16 a[36:39], a[204:207], v[76:79], a[36:39]\n\t" \
  "v_mfma_f32_16x16x32_bf16 a[32:35], a[196:199], v[76:79], a[32:35]\n\t" \
  "ds_read_b64_tr_b16 a[150:151], v14 offset:9216\n\t" \
  "v_mfma_f32_16x16x32_bf16 a[32:35], a[192:195], v[72:75], a[32:35]\n\t" \
  "ds_read_b128 v[72:75], v10 offset:2048\n\t" \
  "ds_read_b128 v[76:79], v11 offset:2048\n\t" \
  "v_mfma_f32_16x16x32_bf16 a[64:67], a[192:195], v[80:83], a[64:67]\n\t" \
  "ds_read_b64_tr_b16 a[152:153], v15 offset:0\n\t" \
  "v_mfma_f32_16x16x32_bf16 a[64:67], a[196:199], v[84:87], a[64:67]\n\t" \
  "v_mfma_f32_16x16x32_bf16 a[68:71], a[204:207], v[84:87], a[68:71]\n\t" \
  "ds_read_b64_tr_b16 a[154:155], v15 offset:1024\n\t" \
  "v_mfma_f32_16x16x32_bf16 a[68:71], a[200:203], v[80:83], a[68:71]\n\t" \
  "v_mfma_f32_16x16x32_bf16 a[72:75], a[208:211], v[80:83], a[72:75]\n\t" \
  "ds_read_b64_tr_b16 a[156:157], v15 offset:8192\n\t" \
  "v_mfma_f32_16x16x32_bf16 a[72:75], a[212:215], v[84:87], a[72:75]\n\t" \
  "v_mfma_f32_16x16x32_bf16 a[76:79], a[220:223], v[84:87], a[76:79]\n\t" \
  "ds_read_b64_tr_b16 a[158:159], v15 offset:9216\n\t" \
  "v_mfma_f32_16x16x32_bf16 a[76:79], a[216:219], v[80:83], a[76:79]\n\t" \
  "s_add_u32 m0, s68, 49152\n\t" \
  "v_mfma_f32_16x16x32_bf16 a[80:83], a[224:227], v[80:83], a[80:83]\n\t" \
  "buffer_load_dwordx4 v0, s[36:39], s65 offen offset:0 sc0 sc1 lds\n\t" \
  "v_mfma_f32_16x16x32_bf16 a[80:83], a[228:231], v[84:87], a[80:83]\n\t" \
  "v_mfma_f32_16x16x32_bf16 a[84:87], a[236:239], v[84:87], a[84:87]\n\t" \
  "RET91_%=:\n\t" \
  "v_mfma_f32_16x16x32_bf16 a[84:87], a[232:235], v[80:83], a[84:87]\n\t" \
  "buffer_load_dwordx4 v1, s[36:39], s65 offen offset:1024 sc0 sc1 lds\n\t" \
  "v_mfma_f32_16x16x32_bf16 a[88:91], a[240:243], v[80:83], a[88:91]\n\t" \
  "v_mfma_f32_16x16x32_bf16 a[88:91], a[244:247], v[84:87], a[88:91]\n\t" \
  "v_mfma_f32_16x16x32_bf16 a[92:95], a[252:255], v[84:87], a[92:95]\n\t" \
  "buffer_load_dwordx4 v2, s[36:39], s65 offen offset:2048 sc0 sc1 lds\n\t" \
  "v_mfma_f32_16x16x32_bf16 a[92:95], a[248:251], v[80:83], a[92:95]\n\t" \
  "ds_read_b128 v[80:83], v10 offset:4096\n\t" \
  "ds_read_b128 v[84:87], v11 offset:4096\n\t" \
  "v_mfma_f32_16x16x32_bf16 a[124:127], a[248:251], v[88:91], a[124:127]\n\t" \
  "v_mfma_f32_16x16x32_bf16 a[124:127], a[252:255], v[92:95], a[124:127]\n\t" \
  "buffer_load_dwordx4 v3, s[36:39], s65 offen offset:3072 sc0 sc1 lds\n\t" \
  "v_mfma_f32_16x16x32_bf16 a[120:123], a[244:247], v[92:95], a[120:123]\n\t" \
  "v_mfma_f32_16x16x32_bf16 a[120:123], a[240:243], v[88:91], a[120:123]\n\t" \
  "s_add_u32 m0, s68, 114688\n\t" \
  "v_mfma_f32_16x16x32_bf16 a[116:119], a[232:235], v[88:91], a[116:119]\n\t" \
  "buffer_load_dwordx4 v4, s[40:43], s79 offen offset:0 lds\n\t" \
  "v_mfma_f32_16x16x32_bf16 a[116:119], a[236:239], v[92:95], a[116:119]\n\t" \
  "v_mfma_f32_16x16x32_bf16 a[112:115], a[228:231], v[92:95], a[112:115]\n\t" \
  "v_mfma_f32_16x16x32_bf16 a[112:115], a[224:227], v[88:91], a[112:115]\n\t" \
  "buffer_load_dwordx4 v5, s[40:43], s79 offen offset:1024 lds\n\t" \
  "v_mfma_f32_16x16x32_bf16 a[108:111], a[216:219], v[88:91], a[108:111]\n\t" \
  "v_mfma_f32_16x16x32_bf16 a[108:111], a[220:223], v[92:95], a[108:111]\n\t" \
  "v_mfma_f32_16x16x32_bf16 a[104:107], a[212:215], v[92:95], a[104:107]\n\t" \
  "buffer_load_dwordx4 v6, s[40:43], s79 offen offset:2048 lds\n\t" \
  "v_mfma_f32_16x16x32_bf16 a[104:107], a[208:211], v[88:91], a[104:107]\n\t" \
  "v_mfma_f32_16x16x32_bf16 a[100:103], a[200:203], v[88:91], a[100:103]\n\t" \
  "v_mfma_f32_16x16x32_bf16 a[100:103], a[204:207], v[92:95], a[100:103]\n\t" \
  "buffer_load_dwordx4 v7, s[40:43], s79 offen offset:3072 lds\n\t" \
  "v_mfma_f32_16x16x32_bf16 a[96:99], a[196:199], v[92:95], a[96:99]\n\t" \
  "v_mfma_f32_16x16x32_bf16 a[96:99], a[192:195], v[88:91], a[96:99]\n\t" \
  "ds_read_b128 v[88:91], v10 offset:6144\n\t" \
  "ds_read_b128 v[92:95], v11 offset:6144\n\t" \
  "v_mfma_f32_16x16x32_bf16 v[128:131], a[192:195], v[96:99], v[128:131]\n\t" \
  "s_add_u32 s77, s77, 1\n\t" \
  "v_mfma_f32_16x16x32_bf16 v[128:131], a[196:199], v[100:103], v[128:131]\n\t" \
  "v_readlane_b32 s66, v16, s77\n\t" \
  "v_mfma_f32_16x16x32_bf16 v[132:135], a[204:207], v[100:103], v[132:135]\n\t" \
  "s_cmp_eq_u32 s77, 8\n\t" \
  "v_mfma_f32_16x16x32_bf16 v[132:135], a[200:203], v[96:99], v[132:135]\n\t" \
  "s_cbranch_scc1 SW93_%=\n\t" \
  "RET94_%=:\n\t" \
  "v_mfma_f32_16x16x32_bf16 v[136:139], a[208:211], v[96:99], v[136:139]\n\t" \
  "s_lshl_b32 s78, s66, 13\n\t" \
  "v_mfma_f32_16x16x32_bf16 v[136:139], a[212:215], v[100:103], v[136:139]\n\t" \
  "s_lshr_b32 s65, s66, 11\n\t" \
  "s_waitcnt vmcnt(16) lgkmcnt(0)\n\t" \
  "s_barrier\n\t" \
  "v_mfma_f32_16x16x32_bf16 v[140:143], a[220:223], v[100:103], v[140:143]\n\t" \
  "s_mul_i32 s64, s66, s67\n\t" \
  "v_mfma_f32_16x16x32_bf16 v[140:143], a[216:219], v[96:99], v[140:143]\n\t" \
  "v_mfma_f32_16x16x32_bf16 v[144:147], a[224:227], v[96:99], v[144:147]\n\t" \
  "ds_read_b64_tr_b16 a[160:161], v12 offset:16384\n\t" \
  "v_mfma_f32_16x16x32_bf16 v[144:147], a[228:231], v[100:103], v[144:147]\n\t" \
  "s_add_u32 s79, s78, 256\n\t" \
  "v_mfma_f32_16x16x32_bf16 v[148:151], a[236:239], v[100:103], v[148:151]\n\t" \
  "ds_read_b64_tr_b16 a[162:163], v12 offset:17408\n\t" \
  "v_mfma_f32_16x16x32_bf16 v[148:151], a[232:235], v[96:99], v[148:151]\n\t" \
  "s_mul_i32 s65, s65, s89\n\t" \
  "v_mfma_f32_16x16x32_bf16 v[152:155], a[240:243], v[96:99], v[152:155]\n\t" \
  "ds_read_b64_tr_b16 a[164:165], v12 offset:24576\n\t" \
  "v_mfma_f32_16x16x32_bf16 v[152:155], a[244:247], v[100:103], v[152:155]\n\t" \
  "s_add_u32 s65, s65, s64\n\t" \
  "v_mfma_f32_16x16x32_bf16 v[156:159], a[252:255], v[100:103], v[156:159]\n\t" \
  "ds_read_b64_tr_b16 a[166:167], v12 offset:25600\n\t" \
  "v_mfma_f32_16x16x32_bf16 v[156:159], a[248:251], v[96:99], v[156:159]\n\t" \
  "ds_read_b128 v[96:99], v10 offset:16384\n\t" \
  "ds_read_b128 v[100:103], v11 offset:16384\n\t" \
  "v_mfma_f32_16x16x32_bf16 v[188:191], a[248:251], v[104:107], v[188:191]\n\t" \
  "ds_read_b64_tr_b16 a[168:169], v13 offset:16384\n\t" \
  "v_mfma_f32_16x16x32_bf16 v[188:191], a[252:255], v[108:111], v[188:191]\n\t" \
  "s_add_u32 s64, s65, s69\n\t" \
  "v_mfma_f32_16x16x32_bf16 v[184:187], a[244:247], v[108:111], v[184:187]\n\t" \
  "ds_read_b64_tr_b16 a[170:171], v13 offset:17408\n\t" \
  "v_mfma_f32_16x16x32_bf16 v[184:187], a[240:243], v[104:107], v[184:187]\n\t" \
  "s_add_u32 s65, s65, s70\n\t" \
  "v_mfma_f32_16x16x32_bf16 v[180:183], a[232:235], v[104:107], v[180:183]\n\t" \
  "ds_read_b64_tr_b16 a[172:173], v13 offset:24576\n\t" \
  "v_mfma_f32_16x16x32_bf16 v[180:183], a[236:239], v[108:111], v[180:183]\n\t" \
  "v_mfma_f32_16x16x32_bf16 v[176:179], a[228:231], v[108:111], v[176:179]\n\t" \
  "ds_read_b64_tr_b16 a[174:175], v13 offset:25600\n\t" \
  "v_mfma_f32_16x16x32_bf16 v[176:179], a[224:227], v[104:107], v[176:179]\n\t" \
  "s_sub_u32 s87, s87, 1\n\t" \
  "v_mfma_f32_16x16x32_bf16 v[172:175], a[216:219], v[104:107], v[172:175]\n\t" \
  "ds_read_b64_tr_b16 a[176:177], v14 offset:16384\n\t" \
  "v_mfma_f32_16x16x32_bf16 v[172:175], a[220:223], v[108:111], v[172:175]\n\t" \
  "s_cbranch_scc1 EVT95_%=\n\t" \
  "v_mfma_f32_16x16x32_bf16 v[168:171], a[212:215], v[108:111], v[168:171]\n\t" \
  "ds_read_b64_tr_b16 a[178:179], v14 offset:17408\n\t" \
  "v_mfma_f32_16x16x32_bf16 v[168:171], a[208:211], v[104:107], v[168:171]\n\t" \
  "v_mfma_f32_16x16x32_bf16 v[164:167], a[200:203], v[104:107], v[164:167]\n\t" \
  "ds_read_b64_tr_b16 a[180:181], v14 offset:24576\n\t" \
  "v_mfma_f32_16x16x32_bf16 v[164:167], a[204:207], v[108:111], v[164:167]\n\t" \
  "v_mfma_f32_16x16x32_bf16 v[160:163], a[196:199], v[108:111], v[160:163]\n\t" \
  "ds_read_b64_tr_b16 a[182:183], v14 offset:25600\n\t" \
  "v_mfma_f32_16x16x32_bf16 v[160:163], a[192:195], v[104:107], v[160:163]\n\t" \
  "ds_read_b128 v[104:107], v10 offset:18432\n\t" \
  "ds_read_b128 v[108:111], v11 offset:18432\n\t" \
  "v_mfma_f32_16x16x32_bf16 v[192:195], a[192:195], v[112:115], v[192:195]\n\t" \
  "ds_read_b64_tr_b16 a[184:185], v15 offset:16384\n\t" \
  "v_mfma_f32_16x16x32_bf16 v[192:195], a[196:199], v[116:119], v[192:195]\n\t" \
  "v_mfma_f32_16x16x32_bf16 v[196:199], a[204:207], v[116:119], v[196:199]\n\t" \
  "ds_read_b64_tr_b16 a[186:187], v15 offset:17408\n\t" \
  "v_mfma_f32_16x16x32_bf16 v[196:199], a[200:203], v[112:115], v[196:199]\n\t" \
  "v_mfma_f32_16x16x32_bf16 v[200:203], a[208:211], v[112:115], v[200:203]\n\t" \
  "ds_read_b64_tr_b16 a[188:189], v15 offset:24576\n\t" \
  "v_mfma_f32_16x16x32_bf16 v[200:203], a[212:215], v[116:119], v[200:203]\n\t" \
  "v_mfma_f32_16x16x32_bf16 v[204:207], a[220:223], v[116:119], v[204:207]\n\t" \
  "ds_read_b64_tr_b16 a[190:191], v15 offset:25600\n\t" \
  "v_mfma_f32_16x16x32_bf16 v[204:207], a[216:219], v[112:115], v[204:207]\n\t" \
  "s_add_u32 m0, s68, 0\n\t" \
  "v_mfma_f32_16x16x32_bf16 v[208:211], a[224:227], v[112:115], v[208:211]\n\t" \
  "buffer_load_dwordx4 v0, s[36:39], s64 offen offset:0 sc0 sc1 lds\n\t" \
  "v_mfma_f32_16x16x32_bf16 v[208:211], a[228:231], v[116:119], v[208:211]\n\t" \
  "v_mfma_f32_16x16x32_bf16 v[212:215], a[236:239], v[116:119], v[212:215]\n\t" \
  "RET96_%=:\n\t" \
  "v_mfma_f32_16x16x32_bf16 v[212:215], a[232:235], v[112:115], v[212:215]\n\t" \
  "buffer_load_dwordx4 v1, s[36:39], s64 offen offset:1024 sc0 sc1 lds\n\t" \
  "v_mfma_f32_16x16x32_bf16 v[216:219], a[240:243], v[112:115], v[216:219]\n\t" \
  "v_mfma_f32_16x16x32_bf16 v[216:219], a[244:247], v[116:119], v[216:219]\n\t" \
  "v_mfma_f32_16x16x32_bf16 v[220:223], a[252:255], v[116:119], v[220:223]\n\t" \
  "buffer_load_dwordx4 v2, s[36:39], s64 offen offset:2048 sc0 sc1 lds\n\t" \
  "v_mfma_f32_16x16x32_bf16 v[220:223], a[248:251], v[112:115], v[220:223]\n\t" \
  "ds_read_b128 v[112:115], v10 offset:20480\n\t" \
  "ds_read_b128 v[116:119], v11 offset:20480\n\t" \
  "v_mfma_f32_16x16x32_bf16 v[252:255], a[248:251], v[120:123], v[252:255]\n\t" \
  "v_mfma_f32_16x16x32_bf16 v[252:255], a[252:255], v[124:127], v[252:255]\n\t" \
  "buffer_load_dwordx4 v3, s[36:39], s64 offen offset:3072 sc0 sc1 lds\n\t" \
  "v_mfma_f32_16x16x32_bf16 v[248:251], a[244:247], v[124:127], v[248:251]\n\t" \
  "v_mfma_f32_16x16x32_bf16 v[248:251], a[240:243], v[120:123], v[248:251]\n\t" \
  "s_add_u32 m0, s68, 65536\n\t" \
  "v_mfma_f32_16x16x32_bf16 v[244:247], a[232:235], v[120:123], v[244:247]\n\t" \
  "buffer_load_dwordx4 v4, s[40:43], s78 offen offset:0 lds\n\t" \
  "v_mfma_f32_16x16x32_bf16 v[244:247], a[236:239], v[124:127], v[244:247]\n\t" \
  "v_mfma_f32_16x16x32_bf16 v[240:243], a[228:231], v[124:127], v[240:243]\n\t" \
  "v_mfma_f32_16x16x32_bf16 v[240:243], a[224:227], v[120:123], v[240:243]\n\t" \
  "buffer_load_dwordx4 v5, s[40:43], s78 offen offset:1024 lds\n\t" \
  "v_mfma_f32_16x16x32_bf16 v[236:239], a[216:219], v[120:123], v[236:239]\n\t" \
  "s_bitcmp1_b32 s84, s76\n\t" \
  "v_mfma_f32_16x16x32_bf16 v[236:239], a[220:223], v[124:127], v[236:239]\n\t" \
  "s_cbranch_scc1 TRG97_%=\n\t" \
  "TRGR98_%=:\n\t" \
  "v_mfma_f32_16x16x32_bf16 v[232:235], a[212:215], v[124:127], v[232:235]\n\t" \
  "buffer_load_dwordx4 v6, s[40:43], s78 offen offset:2048 lds\n\t" \
  "v_mfma_f32_16x16x32_bf16 v[232:235], a[208:211], v[120:123], v[232:235]\n\t" \
  "s_sub_u32 s76, s76, 1\n\t" \
  "v_mfma_f32_16x16x32_bf16 v[228:231], a[200:203], v[120:123], v[228:231]\n\t" \
  "s_cmp_lg_u32 s76, 0\n\t" \
  "v_mfma_f32_16x16x32_bf16 v[228:231], a[204:207], v[124:127], v[228:231]\n\t" \
  "buffer_load_dwordx4 v7, s[40:43], s78 offen offset:3072 lds\n\t" \
  "v_mfma_f32_16x16x32_bf16 v[224:227], a[196:199], v[124:127], v[224:227]\n\t" \
  "v_mfma_f32_16x16x32_bf16 v[224:227], a[192:195], v[120:123], v[224:227]\n\t" \
  "ds_read_b128 v[120:123], v10 offset:22528\n\t" \
  "ds_read_b128 v[124:127], v11 offset:22528\n\t" \
  "v_mfma_f32_16x16x32_bf16 a[0:3], a[128:131], v[64:67], a[0:3]\n\t" \
  "v_mfma_f32_16x16x32_bf16 a[0:3], a[132:135], v[68:71], a[0:3]\n\t" \
  "v_mfma_f32_16x16x32_bf16 a[4:7], a[140:143], v[68:71], a[4:7]\n\t" \
  "s_cbranch_scc1 LOOP2_%=\n\t" \
  "s_branch LEND_%=\n\t" \
  ".p2align 6\n\t" \
  "LOOP3_%=:\n\t" \
  "v_mfma_f32_16x16x32_bf16 a[4:7], a[136:139], v[64:67], a[4:7]\n\t" \
  "v_mfma_f32_16x16x32_bf16 a[8:11], a[144:147], v[64:67], a[8:11]\n\t" \
  "s_waitcnt vmcnt(16) lgkmcnt(0)\n\t" \
  "s_barrier\n\t" \
  "v_mfma_f32_16x16x32_bf16 a[8:11], a[148:151], v[68:71], a[8:11]\n\t" \
  "v_mfma_f32_16x16x32_bf16 a[12:15], a[156:159], v[68:71], a[12:15]\n\t" \
  "v_mfma_f32_16x16x32_bf16 a[12:15], a[152:155], v[64:67], a[12:15]\n\t" \
  "v_mfma_f32_16x16x32_bf16 a[16:19], a[160:163], v[64:67], a[16:19]\n\t" \
  "ds_read_b64_tr_b16 a[192:193], v12 offset:32768\n\t" \
  "v_mfma_f32_16x16x32_bf16 a[16:19], a[164:167], v[68:71], a[16:19]\n\t" \
  "v_mfma_f32_16x16x32_bf16 a[20:23], a[172:175], v[68:71], a[20:23]\n\t" \
  "ds_read_b64_tr_b16 a[194:195], v12 offset:33792\n\t" \
  "v_mfma_f32_16x16x32_bf16 a[20:23], a[168:171], v[64:67], a[20:23]\n\t" \
  "v_mfma_f32_16x16x32_bf16 a[24:27], a[176:179], v[64:67], a[24:27]\n\t" \
  "ds_read_b64_tr_b16 a[196:197], v12 offset:40960\n\t" \
  "v_mfma_f32_16x16x32_bf16 a[24:27], a[180:183], v[68:71], a[24:27]\n\t" \
  "v_mfma_f32_16x16x32_bf16 a[28:31], a[188:191], v[68:71], a[28:31]\n\t" \
  "ds_read_b64_tr_b16 a[198:199], v12 offset:41984\n\t" \
  "v_mfma_f32_16x16x32_bf16 a[28:31], a[184:187], v[64:67], a[28:31]\n\t" \
  "ds_read_b128 v[64:67], v10 offset:32768\n\t" \
  "ds_read_b128 v[68:71], v11 offset:32768\n\t" \
  "v_mfma_f32_16x16x32_bf16 a[60:63], a[184:187], v[72:75], a[60:63]\n\t" \
  "ds_read_b64_tr_b16 a[200:201], v13 offset:32768\n\t" \
  "v_mfma_f32_16x16x32_bf16 a[60:63], a[188:191], v[76:79], a[60:63]\n\t" \
  "v_mfma_f32_16x16x32_bf16 a[56:59], a[180:183], v[76:79], a[56:59]\n\t" \
  "ds_read_b64_tr_b16 a[202:203], v13 offset:33792\n\t" \
  "v_mfma_f32_16x16x32_bf16 a[56:59], a[176:179], v[72:75], a[56:59]\n\t" \
  "v_mfma_f32_16x16x32_bf16 a[52:55], a[168:171], v[72:75], a[52:55]\n\t" \
  "ds_read_b64_tr_b16 a[204:205], v13 offset:40960\n\t" \
  "v_mfma_f32_16x16x32_bf16 a[52:55], a[172:175], v[76:79], a[52:55]\n\t" \
  "v_mfma_f32_16x16x32_bf16 a[48:51], a[164:167], v[76:79], a[48:51]\n\t" \
  "ds_read_b64_tr_b16 a[206:207], v13 offset:41984\n\t" \
  "v_mfma_f32_16x16x32_bf16 a[48:51], a[160:163], v[72:75], a[48:51]\n\t" \
  "s_sub_u32 s87, s87, 1\n\t" \
  "v_mfma_f32_16x16x32_bf16 a[44:47], a[152:155], v[72:75], a[44:47]\n\t" \
  "ds_read_b64_tr_b16 a[208:209], v14 offset:32768\n\t" \
  "v_mfma_f32_16x16x32_bf16 a[44:47], a[156:159], v[76:79], a[44:47]\n\t" \
  "s_cbranch_scc1 EVT101_%=\n\t" \
  "v_mfma_f32_16x16x32_bf16 a[40:43], a[148:151], v[76:79], a[40:43]\n\t" \
  "ds_read_b64_tr_b16 a[210:211], v14 offset:33792\n\t" \
  "v_mfma_f32_16x16x32_bf16 a[40:43], a[144:147], v[72:75], a[40:43]\n\t" \
  "v_mfma_f32_16x16x32_bf16 a[36:39], a[136:139], v[72:75], a[36:39]\n\t" \
  "ds_read_b64_tr_b16 a[212:213], v14 offset:40960\n\t" \
  "v_mfma_f32_16x16x32_bf16 a[36:39], a[140:143], v[76:79], a[36:39]\n\t" \
  "v_mfma_f32_16x16x32_bf16 a[32:35], a[132:135], v[76:79], a[32:35]\n\t" \
  "ds_read_b64_tr_b16 a[214:215], v14 offset:41984\n\t" \
  "v_mfma_f32_16x16x32_bf16 a[32:35], a[128:131], v[72:75], a[32:35]\n\t" \
  "ds_read_b128 v[72:75], v10 offset:34816\n\t" \
  "ds_read_b128 v[76:79], v11 offset:34816\n\t" \
  "v_mfma_f32_16x16x32_bf16 a[64:67], a[128:131], v[80:83], a[64:67]\n\t" \
  "ds_read_b64_tr_b16 a[216:217], v15 offset:32768\n\t" \
  "v_mfma_f32_16x16x32_bf16 a[64:67], a[132:135], v[84:87], a[64:67]\n\t" \
  "v_mfma_f32_16x16x32_bf16 a[68:71], a[140:143], v[84:87], a[68:71]\n\t" \
  "ds_read_b64_tr_b16 a[218:219], v15 offset:33792\n\t" \
  "v_mfma_f32_16x16x32_bf16 a[68:71], a[136:139], v[80:83], a[68:71]\n\t" \
  "v_mfma_f32_16x16x32_bf16 a[72:75], a[144:147], v[80:83], a[72:75]\n\t" \
  "ds_read_b64_tr_b16 a[220:221], v15 offset:40960\n\t" \
  "v_mfma_f32_16x16x32_bf16 a[72:75], a[148:151], v[84:87], a[72:75]\n\t" \
  "v_mfma_f32_16x16x32_bf16 a[76:79], a[156:159], v[84:87], a[76:79]\n\t" \
  "ds_read_b64_tr_b16 a[222:223], v15 offset:41984\n\t" \
  "v_mfma_f32_16x16x32_bf16 a[76:79], a[152:155], v[80:83], a[76:79]\n\t" \
  "s_add_u32 m0, s68, 16384\n\t" \
  "v_mfma_f32_16x16x32_bf16 a[80:83], a[160:163], v[80:83], a[80:83]\n\t" \
  "buffer_load_dwordx4 v0, s[36:39], s65 offen offset:0 sc0 sc1 lds\n\t" \
  "v_mfma_f32_16x16x32_bf16 a[80:83], a[164:167], v[84:87], a[80:83]\n\t" \
  "v_mfma_f32_16x16x32_bf16 a[84:87], a[172:175], v[84:87], a[84:87]\n\t" \
  "RET102_%=:\n\t" \
  "v_mfma_f32_16x16x32_bf16 a[84:87], a[168:171], v[80:83], a[84:87]\n\t" \
  "buffer_load_dwordx4 v1, s[36:39], s65 offen offset:1024 sc0 sc1 lds\n\t" \
  "v_mfma_f32_16x16x32_bf16 a[88:91], a[176:179], v[80:83], a[88:91]\n\t" \
  "v_mfma_f32_16x16x32_bf16 a[88:91], a[180:183], v[84:87], a[88:91]\n\t" \
  "v_mfma_f32_16x16x32_bf16 a[92:95], a[188:191], v[84:87], a[92:95]\n\t" \
  "buffer_load_dwordx4 v2, s[36:39], s65 offen offset:2048 sc0 sc1 lds\n\t" \
  "v_mfma_f32_16x16x32_bf16 a[92:95], a[184:187], v[80:83], a[92:95]\n\t" \
  "ds_read_b128 v[80:83], v10 offset:36864\n\t" \
  "ds_read_b128 v[84:87], v11 offset:36864\n\t" \
  "v_mfma_f32_16x16x32_bf16 a[124:127], a[184:187], v[88:91], a[124:127]\n\t" \
  "v_mfma_f32_16x16x32_bf16 a[124:127], a[188:191], v[92:95], a[124:127]\n\t" \
  "buffer_load_dwordx4 v3, s[36:39], s65 offen offset:3072 sc0 sc1 lds\n\t" \
  "v_mfma_f32_16x16x32_bf16 a[120:123], a[180:183], v[92:95], a[120:123]\n\t" \
  "v_mfma_f32_16x16x32_bf16 a[120:123], a[176:179], v[88:91], a[120:123]\n\t" \
  "s_add_u32 m0, s68, 81920\n\t" \
  "v_mfma_f32_16x16x32_bf16 a[116:119], a[168:171], v[88:91], a[116:119]\n\t" \
  "buffer_load_dwordx4 v4, s[40:43], s79 offen offset:0 lds\n\t" \
  "v_mfma_f32_16x16x32_bf16 a[116:119], a[172:175], v[92:95], a[116:119]\n\t" \
  "v_mfma_f32_16x16x32_bf16 a[112:115], a[164:167], v[92:95], a[112:115]\n\t" \
  "v_mfma_f32_16x16x32_bf16 a[112:115], a[160:163], v[88:91], a[112:115]\n\t" \
  "buffer_load_dwordx4 v5, s[40:43], s79 offen offset:1024 lds\n\t" \
  "v_mfma_f32_16x16x32_bf16 a[108:111], a[152:155], v[88:91], a[108:111]\n\t" \
  "v_mfma_f32_16x16x32_bf16 a[108:111], a[156:159], v[92:95], a[108:111]\n\t" \
  "v_mfma_f32_16x16x32_bf16 a[104:107], a[148:151], v[92:95], a[104:107]\n\t" \
  "buffer_load_dwordx4 v6, s[40:43], s79 offen offset:2048 lds\n\t" \
  "v_mfma_f32_16x16x32_bf16 a[104:107], a[144:147], v[88:91], a[104:107]\n\t" \
  "v_mfma_f32_16x16x32_bf16 a[100:103], a[136:139], v[88:91], a[100:103]\n\t" \
  "v_mfma_f32_16x16x32_bf16 a[100:103], a[140:143], v[92:95], a[100:103]\n\t" \
  "buffer_load_dwordx4 v7, s[40:43], s79 offen offset:3072 lds\n\t" \
  "v_mfma_f32_16x16x32_bf16 a[96:99], a[132:135], v[92:95], a[96:99]\n\t" \
  "v_mfma_f32_16x16x32_bf16 a[96:99], a[128:131], v[88:91], a[96:99]\n\t" \
  "ds_read_b128 v[88:91], v10 offset:38912\n\t" \
  "ds_read_b128 v[92:95], v11 offset:38912\n\t" \
  "v_mfma_f32_16x16x32_bf16 v[128:131], a[128:131], v[96:99], v[128:131]\n\t" \
  "s_add_u32 s77, s77, 1\n\t" \
  "v_mfma_f32_16x16x32_bf16 v[128:131], a[132:135], v[100:103], v[128:131]\n\t" \
  "v_readlane_b32 s66, v16, s77\n\t" \
  "v_mfma_f32_16x16x32_bf16 v[132:135], a[140:143], v[100:103], v[132:135]\n\t" \
  "s_cmp_eq_u32 s77, 8\n\t" \
  "v_mfma_f32_16x16x32_bf16 v[132:135], a[136:139], v[96:99], v[132:135]\n\t" \
  "s_cbranch_scc1 SW104_%=\n\t" \
  "RET105_%=:\n\t" \
  "v_mfma_f32_16x16x32_bf16 v[136:139], a[144:147], v[96:99], v[136:139]\n\t" \
  "s_lshl_b32 s78, s66, 13\n\t" \
  "s_waitcnt vmcnt(16) lgkmcnt(0)\n\t" \
  "s_barrier\n\t" \
  "v_mfma_f32_16x16x32_bf16 v[136:139], a[148:151], v[100:103], v[136:139]\n\t" \
  "s_lshr_b32 s65, s66, 11\n\t" \
  "v_mfma_f32_16x16x32_bf16 v[140:143], a[156:159], v[100:103], v[140:143]\n\t" \
  "s_mul_i32 s64, s66, s67\n\t" \
  "v_mfma_f32_16x16x32_bf16 v[140:143], a[152:155], v[96:99], v[140:143]\n\t" \
  "v_mfma_f32_16x16x32_bf16 v[144:147], a[160:163], v[96:99], v[144:147]\n\t" \
  "ds_read_b64_tr_b16 a[224:225], v12 offset:49152\n\t" \
  "v_mfma_f32_16x16x32_bf16 v[144:147], a[164:167], v[100:103], v[144:147]\n\t" \
  "s_add_u32 s79, s78, 256\n\t" \
  "v_mfma_f32_16x16x32_bf16 v[148:151], a[172:175], v[100:103], v[148:151]\n\t" \
  "ds_read_b64_tr_b16 a[226:227], v12 offset:50176\n\t" \
  "v_mfma_f32_16x16x32_bf16 v[148:151], a[168:171], v[96:99], v[148:151]\n\t" \
  "s_mul_i32 s65, s65, s89\n\t" \
  "v_mfma_f32_16x16x32_bf16 v[152:155], a[176:179], v[96:99], v[152:155]\n\t" \
  "ds_read_b64_tr_b16 a[228:229], v12 offset:57344\n\t" \
  "v_mfma_f32_16x16x32_bf16 v[152:155], a[180:183], v[100:103], v[152:155]\n\t" \
  "s_add_u32 s65, s65, s64\n\t" \
  "v_mfma_f32_16x16x32_bf16 v[156:159], a[188:191], v[100:103], v[156:159]\n\t" \
  "ds_read_b64_tr_b16 a[230:231], v12 offset:58368\n\t" \
  "v_mfma_f32_16x16x32_bf16 v[156:159], a[184:187], v[96:99], v[156:159]\n\t" \
  "ds_read_b128 v[96:99], v10 offset:49152\n\t" \
  "ds_read_b128 v[100:103], v11 offset:49152\n\t" \
  "v_mfma_f32_16x16x32_bf16 v[188:191], a[184:187], v[104:107], v[188:191]\n\t" \
  "ds_read_b64_tr_b16 a[232:233], v13 offset:49152\n\t" \
  "v_mfma_f32_16x16x32_bf16 v[188:191], a[188:191], v[108:111], v[188:191]\n\t" \
  "s_add_u32 s64, s65, s69\n\t" \
  "v_mfma_f32_16x16x32_bf16 v[184:187], a[180:183], v[108:111], v[184:187]\n\t" \
  "ds_read_b64_tr_b16 a[234:235], v13 offset:50176\n\t" \
  "v_mfma_f32_16x16x32_bf16 v[184:187], a[176:179], v[104:107], v[184:187]\n\t" \
  "s_add_u32 s65, s65, s70\n\t" \
  "v_mfma_f32_16x16x32_bf16 v[180:183], a[168:171], v[104:107], v[180:183]\n\t" \
  "ds_read_b64_tr_b16 a[236:237], v13 offset:57344\n\t" \
  "v_mfma_f32_16x16x32_bf16 v[180:183], a[172:175], v[108:111], v[180:183]\n\t" \
  "v_mfma_f32_16x16x32_bf16 v[176:179], a[164:167], v[108:111], v[176:179]\n\t" \
  "ds_read_b64_tr_b16 a[238:239], v13 offset:58368\n\t" \
  "v_mfma_f32_16x16x32_bf16 v[176:179], a[160:163], v[104:107], v[176:179]\n\t" \
  "s_sub_u32 s87, s87, 1\n\t" \
  "v_mfma_f32_16x16x32_bf16 v[172:175], a[152:155], v[104:107], v[172:175]\n\t" \
  "ds_read_b64_tr_b16 a[240:241], v14 offset:49152\n\t" \
  "v_mfma_f32_16x16x32_bf16 v[172:175], a[156:159], v[108:111], v[172:175]\n\t" \
  "s_cbranch_scc1 EVT106_%=\n\t" \
  "v_mfma_f32_16x16x32_bf16 v[168:171], a[148:151], v[108:111], v[168:171]\n\t" \
  "ds_read_b64_tr_b16 a[242:243], v14 offset:50176\n\t" \
  "v_mfma_f32_16x16x32_bf16 v[168:171], a[144:147], v[104:107], v[168:171]\n\t" \
  "v_mfma_f32_16x16x32_bf16 v[164:167], a[136:139], v[104:107], v[164:167]\n\t" \
  "ds_read_b64_tr_b16 a[244:245], v14 offset:57344\n\t" \
  "v_mfma_f32_16x16x32_bf16 v[164:167], a[140:143], v[108:111], v[164:167]\n\t" \
  "v_mfma_f32_16x16x32_bf16 v[160:163], a[132:135], v[108:111], v[160:163]\n\t" \
  "ds_read_b64_tr_b16 a[246:247], v14 offset:58368\n\t" \
  "v_mfma_f32_16x16x32_bf16 v[160:163], a[128:131], v[104:107], v[160:163]\n\t" \
  "ds_read_b128 v[104:107], v10 offset:51200\n\t" \
  "ds_read_b128 v[108:111], v11 offset:51200\n\t" \
  "v_mfma_f32_16x16x32_bf16 v[192:195], a[128:131], v[112:115], v[192:195]\n\t" \
  "ds_read_b64_tr_b16 a[248:249], v15 offset:49152\n\t" \
  "v_mfma_f32_16x16x32_bf16 v[192:195], a[132:135], v[116:119], v[192:195]\n\t" \
  "v_mfma_f32_16x16x32_bf16 v[196:199], a[140:143], v[116:119], v[196:199]\n\t" \
  "ds_read_b64_tr_b16 a[250:251], v15 offset:50176\n\t" \
  "v_mfma_f32_16x16x32_bf16 v[196:199], a[136:139], v[112:115], v[196:199]\n\t" \
  "v_mfma_f32_16x16x32_bf16 v[200:203], a[144:147], v[112:115], v[200:203]\n\t" \
  "ds_read_b64_tr_b16 a[252:253], v15 offset:57344\n\t" \
  "v_mfma_f32_16x16x32_bf16 v[200:203], a[148:151], v[116:119], v[200:203]\n\t" \
  "v_mfma_f32_16x16x32_bf16 v[204:207], a[156:159], v[116:119], v[204:207]\n\t" \
  "ds_read_b64_tr_b16 a[254:255], v15 offset:58368\n\t" \
  "v_mfma_f32_16x16x32_bf16 v[204:207], a[152:155], v[112:115], v[204:207]\n\t" \
  "s_add_u32 m0, s68, 32768\n\t" \
  "v_mfma_f32_16x16x32_bf16 v[208:211], a[160:163], v[112:115], v[208:211]\n\t" \
  "buffer_load_dwordx4 v0, s[36:39], s64 offen offset:0 sc0 sc1 lds\n\t" \
  "v_mfma_f32_16x16x32_bf16 v[208:211], a[164:167], v[116:119], v[208:211]\n\t" \
  "v_mfma_f32_16x16x32_bf16 v[212:215], a[172:175], v[116:119], v[212:215]\n\t" \
  "RET107_%=:\n\t" \
  "v_mfma_f32_16x16x32_bf16 v[212:215], a[168:171], v[112:115], v[212:215]\n\t" \
  "buffer_load_dwordx4 v1, s[36:39], s64 offen offset:1024 sc0 sc1 lds\n\t" \
  "v_mfma_f32_16x16x32_bf16 v[216:219], a[176:179], v[112:115], v[216:219]\n\t" \
  "v_mfma_f32_16x16x32_bf16 v[216:219], a[180:183], v[116:119], v[216:219]\n\t" \
  "v_mfma_f32_16x16x32_bf16 v[220:223], a[188:191], v[116:119], v[220:223]\n\t" \
  "buffer_load_dwordx4 v2, s[36:39], s64 offen offset:2048 sc0 sc1 lds\n\t" \
  "v_mfma_f32_16x16x32_bf16 v[220:223], a[184:187], v[112:115], v[220:223]\n\t" \
  "ds_read_b128 v[112:115], v10 offset:53248\n\t" \
  "ds_read_b128 v[116:119], v11 offset:53248\n\t" \
  "v_mfma_f32_16x16x32_bf16 v[252:255], a[184:187], v[120:123], v[252:255]\n\t" \
  "v_mfma_f32_16x16x32_bf16 v[252:255], a[188:191], v[124:127], v[252:255]\n\t" \
  "buffer_load_dwordx4 v3, s[36:39], s64 offen offset:3072 sc0 sc1 lds\n\t" \
  "v_mfma_f32_16x16x32_bf16 v[248:251], a[180:183], v[124:127], v[248:251]\n\t" \
  "v_mfma_f32_16x16x32_bf16 v[248:251], a[176:179], v[120:123], v[248:251]\n\t" \
  "s_add_u32 m0, s68, 98304\n\t" \
  "v_mfma_f32_16x16x32_bf16 v[244:247], a[168:171], v[120:123], v[244:247]\n\t" \
  "buffer_load_dwordx4 v4, s[40:43], s78 offen offset:0 lds\n\t" \
  "v_mfma_f32_16x16x32_bf16 v[244:247], a[172:175], v[124:127], v[244:247]\n\t" \
  "v_mfma_f32_16x16x32_bf16 v[240:243], a[164:167], v[124:127], v[240:243]\n\t" \
  "v_mfma_f32_16x16x32_bf16 v[240:243], a[160:163], v[120:123], v[240:243]\n\t" \
  "buffer_load_dwordx4 v5, s[40:43], s78 offen offset:1024 lds\n\t" \
  "v_mfma_f32_16x16x32_bf16 v[236:239], a[152:155], v[120:123], v[236:239]\n\t" \
  "v_mfma_f32_16x16x32_bf16 v[236:239], a[156:159], v[124:127], v[236:239]\n\t" \
  "v_mfma_f32_16x16x32_bf16 v[232:235], a[148:151], v[124:127], v[232:235]\n\t" \
  "buffer_load_dwordx4 v6, s[40:43], s78 offen offset:2048 lds\n\t" \
  "v_mfma_f32_16x16x32_bf16 v[232:235], a[144:147], v[120:123], v[232:235]\n\t" \
  "v_mfma_f32_16x16x32_bf16 v[228:231], a[136:139], v[120:123], v[228:231]\n\t" \
  "v_mfma_f32_16x16x32_bf16 v[228:231], a[140:143], v[124:127], v[228:231]\n\t" \
  "buffer_load_dwordx4 v7, s[40:43], s78 offen offset:3072 lds\n\t" \
  "v_mfma_f32_16x16x32_bf16 v[224:227], a[132:135], v[124:127], v[224:227]\n\t" \
  "v_mfma_f32_16x16x32_bf16 v[224:227], a[128:131], v[120:123], v[224:227]\n\t" \
  "ds_read_b128 v[120:123], v10 offset:55296\n\t" \
  "ds_read_b128 v[124:127], v11 offset:55296\n\t" \
  "v_mfma_f32_16x16x32_bf16 a[0:3], a[192:195], v[64:67], a[0:3]\n\t" \
  "v_mfma_f32_16x16x32_bf16 a[0:3], a[196:199], v[68:71], a[0:3]\n\t" \
  "v_mfma_f32_16x16x32_bf16 a[4:7], a[204:207], v[68:71], a[4:7]\n\t" \
  "v_mfma_f32_16x16x32_bf16 a[4:7], a[200:203], v[64:67], a[4:7]\n\t" \
  "v_mfma_f32_16x16x32_bf16 a[8:11], a[208:211], v[64:67], a[8:11]\n\t" \
  "s_waitcnt vmcnt(16) lgkmcnt(0)\n\t" \
  "s_barrier\n\t" \
  "v_mfma_f32_16x16x32_bf16 a[8:11], a[212:215], v[68:71], a[8:11]\n\t" \
  "v_mfma_f32_16x16x32_bf16 a[12:15], a[220:223], v[68:71], a[12:15]\n\t" \
  "v_mfma_f32_16x16x32_bf16 a[12:15], a[216:219], v[64:67], a[12:15]\n\t" \
  "v_mfma_f32_16x16x32_bf16 a[16:19], a[224:227], v[64:67], a[16:19]\n\t" \
  "ds_read_b64_tr_b16 a[128:129], v12 offset:0\n\t" \
  "v_mfma_f32_16x16x32_bf16 a[16:19], a[228:231], v[68:71], a[16:19]\n\t" \
  "v_mfma_f32_16x16x32_bf16 a[20:23], a[236:239], v[68:71], a[20:23]\n\t" \
  "ds_read_b64_tr_b16 a[130:131], v12 offset:1024\n\t" \
  "v_mfma_f32_16x16x32_bf16 a[20:23], a[232:235], v[64:67], a[20:23]\n\t" \
  "v_mfma_f32_16x16x32_bf16 a[24:27], a[240:243], v[64:67], a[24:27]\n\t" \
  "ds_read_b64_tr_b16 a[132:133], v12 offset:8192\n\t" \
  "v_mfma_f32_16x16x32_bf16 a[24:27], a[244:247], v[68:71], a[24:27]\n\t" \
  "v_mfma_f32_16x16x32_bf16 a[28:31], a[252:255], v[68:71], a[28:31]\n\t" \
  "ds_read_b64_tr_b16 a[134:135], v12 offset:9216\n\t" \
  "v_mfma_f32_16x16x32_bf16 a[28:31], a[248:251], v[64:67], a[28:31]\n\t" \
  "ds_read_b128 v[64:67], v10 offset:0\n\t" \
  "ds_read_b128 v[68:71], v11 offset:0\n\t" \
  "v_mfma_f32_16x16x32_bf16 a[60:63], a[248:251], v[72:75], a[60:63]\n\t" \
  "ds_read_b64_tr_b16 a[136:137], v13 offset:0\n\t" \
  "v_mfma_f32_16x16x32_bf16 a[60:63], a[252:255], v[76:79], a[60:63]\n\t" \
  "v_mfma_f32_16x16x32_bf16 a[56:59], a[244:247], v[76:79], a[56:59]\n\t" \
  "ds_read_b64_tr_b16 a[138:139], v13 offset:1024\n\t" \
  "v_mfma_f32_16x16x32_bf16 a[56:59], a[240:243], v[72:75], a[56:59]\n\t" \
  "v_mfma_f32_16x16x32_bf16 a[52:55], a[232:235], v[72:75], a[52:55]\n\t" \
  "ds_read_b64_tr_b16 a[140:141], v13 offset:8192\n\t" \
  "v_mfma_f32_16x16x32_bf16 a[52:55], a[236:239], v[76:79], a[52:55]\n\t" \
  "v_mfma_f32_16x16x32_bf16 a[48:51], a[228:231], v[76:79], a[48:51]\n\t" \
  "ds_read_b64_tr_b16 a[142:143], v13 offset:9216\n\t" \
  "v_mfma_f32_16x16x32_bf16 a[48:51], a[224:227], v[72:75], a[48:51]\n\t" \
  "s_sub_u32 s87, s87, 1\n\t" \
  "v_mfma_f32_16x16x32_bf16 a[44:47], a[216:219], v[72:75], a[44:47]\n\t" \
  "ds_read_b64_tr_b16 a[144:145], v14 offset:0\n\t" \
  "v_mfma_f32_16x16x32_bf16 a[44:47], a[220:223], v[76:79], a[44:47]\n\t" \
  "s_cbranch_scc1 EVT109_%=\n\t" \
  "v_mfma_f32_16x16x32_bf16 a[40:43], a[212:215], v[76:79], a[40:43]\n\t" \
  "ds_read_b64_tr_b16 a[146:147], v14 offset:1024\n\t" \
  "v_mfma_f32_16x16x32_bf16 a[40:43], a[208:211], v[72:75], a[40:43]\n\t" \
  "v_mfma_f32_16x16x32_bf16 a[36:39], a[200:203], v[72:75], a[36:39]\n\t" \
  "ds_read_b64_tr_b16 a[148:149], v14 offset:8192\n\t" \
  "v_mfma_f32_16x16x32_bf16 a[36:39], a[204:207], v[76:79], a[36:39]\n\t" \
  "v_mfma_f32_16x16x32_bf16 a[32:35], a[196:199], v[76:79], a[32:35]\n\t" \
  "ds_read_b64_tr_b16 a[150:151], v14 offset:9216\n\t" \
  "v_mfma_f32_16x16x32_bf16 a[32:35], a[192:195], v[72:75], a[32:35]\n\t" \
  "ds_read_b128 v[72:75], v10 offset:2048\n\t" \
  "ds_read_b128 v[76:79], v11 offset:2048\n\t" \
  "v_mfma_f32_16x16x32_bf16 a[64:67], a[192:195], v[80:83], a[64:67]\n\t" \
  "ds_read_b64_tr_b16 a[152:153], v15 offset:0\n\t" \
  "v_mfma_f32_16x16x32_bf16 a[64:67], a[196:199], v[84:87], a[64:67]\n\t" \
  "v_mfma_f32_16x16x32_bf16 a[68:71], a[204:207], v[84:87], a[68:71]\n\t" \
  "ds_read_b64_tr_b16 a[154:155], v15 offset:1024\n\t" \
  "v_mfma_f32_16x16x32_bf16 a[68:71], a[200:203], v[80:83], a[68:71]\n\t" \
  "v_mfma_f32_16x16x32_bf16 a[72:75], a[208:211], v[80:83], a[72:75]\n\t" \
  "ds_read_b64_tr_b16 a[156:157], v15 offset:8192\n\t" \
  "v_mfma_f32_16x16x32_bf16 a[72:75], a[212:215], v[84:87], a[72:75]\n\t" \
  "v_mfma_f32_16x16x32_bf16 a[76:79], a[220:223], v[84:87], a[76:79]\n\t" \
  "ds_read_b64_tr_b16 a[158:159], v15 offset:9216\n\t" \
  "v_mfma_f32_16x16x32_bf16 a[76:79], a[216:219], v[80:83], a[76:79]\n\t" \
  "s_add_u32 m0, s68, 49152\n\t" \
  "v_mfma_f32_16x16x32_bf16 a[80:83], a[224:227], v[80:83], a[80:83]\n\t" \
  "buffer_load_dwordx4 v0, s[36:39], s65 offen offset:0 sc0 sc1 lds\n\t" \
  "v_mfma_f32_16x16x32_bf16 a[80:83], a[228:231], v[84:87], a[80:83]\n\t" \
  "v_mfma_f32_16x16x32_bf16 a[84:87], a[236:239], v[84:87], a[84:87]\n\t" \
  "RET110_%=:\n\t" \
  "v_mfma_f32_16x16x32_bf16 a[84:87], a[232:235], v[80:83], a[84:87]\n\t" \
  "buffer_load_dwordx4 v1, s[36:39], s65 offen offset:1024 sc0 sc1 lds\n\t" \
  "v_mfma_f32_16x16x32_bf16 a[88:91], a[240:243], v[80:83], a[88:91]\n\t" \
  "v_mfma_f32_16x16x32_bf16 a[88:91], a[244:247], v[84:87], a[88:91]\n\t" \
  "v_mfma_f32_16x16x32_bf16 a[92:95], a[252:255], v[84:87], a[92:95]\n\t" \
  "buffer_load_dwordx4 v2, s[36:39], s65 offen offset:2048 sc0 sc1 lds\n\t" \
  "v_mfma_f32_16x16x32_bf16 a[92:95], a[248:251], v[80:83], a[92:95]\n\t" \
  "ds_read_b128 v[80:83], v10 offset:4096\n\t" \
  "ds_read_b128 v[84:87], v11 offset:4096\n\t" \
  "v_mfma_f32_16x16x32_bf16 a[124:127], a[248:251], v[88:91], a[124:127]\n\t" \
  "v_mfma_f32_16x16x32_bf16 a[124:127], a[252:255], v[92:95], a[124:127]\n\t" \
  "buffer_load_dwordx4 v3, s[36:39], s65 offen offset:3072 sc0 sc1 lds\n\t" \
  "v_mfma_f32_16x16x32_bf16 a[120:123], a[244:247], v[92:95], a[120:123]\n\t" \
  "v_mfma_f32_16x16x32_bf16 a[120:123], a[240:243], v[88:91], a[120:123]\n\t" \
  "s_add_u32 m0, s68, 114688\n\t" \
  "v_mfma_f32_16x16x32_bf16 a[116:119], a[232:235], v[88:91], a[116:119]\n\t" \
  "buffer_load_dwordx4 v4, s[40:43], s79 offen offset:0 lds\n\t" \
  "v_mfma_f32_16x16x32_bf16 a[116:119], a[236:239], v[92:95], a[116:119]\n\t" \
  "v_mfma_f32_16x16x32_bf16 a[112:115], a[228:231], v[92:95], a[112:115]\n\t" \
  "v_mfma_f32_16x16x32_bf16 a[112:115], a[224:227], v[88:91], a[112:115]\n\t" \
  "buffer_load_dwordx4 v5, s[40:43], s79 offen offset:1024 lds\n\t" \
  "v_mfma_f32_16x16x32_bf16 a[108:111], a[216:219], v[88:91], a[108:111]\n\t" \
  "v_mfma_f32_16x16x32_bf16 a[108:111], a[220:223], v[92:95], a[108:111]\n\t" \
  "v_mfma_f32_16x16x32_bf16 a[104:107], a[212:215], v[92:95], a[104:107]\n\t" \
  "buffer_load_dwordx4 v6, s[40:43], s79 offen offset:2048 lds\n\t" \
  "v_mfma_f32_16x16x32_bf16 a[104:107], a[208:211], v[88:91], a[104:107]\n\t" \
  "v_mfma_f32_16x16x32_bf16 a[100:103], a[200:203], v[88:91], a[100:103]\n\t" \
  "v_mfma_f32_16x16x32_bf16 a[100:103], a[204:207], v[92:95], a[100:103]\n\t" \
  "buffer_load_dwordx4 v7, s[40:43], s79 offen offset:3072 lds\n\t" \
  "v_mfma_f32_16x16x32_bf16 a[96:99], a[196:199], v[92:95], a[96:99]\n\t" \
  "v_mfma_f32_16x16x32_bf16 a[96:99], a[192:195], v[88:91], a[96:99]\n\t" \
  "ds_read_b128 v[88:91], v10 offset:6144\n\t" \
  "ds_read_b128 v[92:95], v11 offset:6144\n\t" \
  "v_mfma_f32_16x16x32_bf16 v[128:131], a[192:195], v[96:99], v[128:131]\n\t" \
  "s_add_u32 s77, s77, 1\n\t" \
  "v_mfma_f32_16x16x32_bf16 v[128:131], a[196:199], v[100:103], v[128:131]\n\t" \
  "v_readlane_b32 s66, v16, s77\n\t" \
  "v_mfma_f32_16x16x32_bf16 v[132:135], a[204:207], v[100:103], v[132:135]\n\t" \
  "s_cmp_eq_u32 s77, 8\n\t" \
  "v_mfma_f32_16x16x32_bf16 v[132:135], a[200:203], v[96:99], v[132:135]\n\t" \
  "s_cbranch_scc1 SW112_%=\n\t" \
  "RET113_%=:\n\t" \
  "v_mfma_f32_16x16x32_bf16 v[136:139], a[208:211], v[96:99], v[136:139]\n\t" \
  "s_lshl_b32 s78, s66, 13\n\t" \
  "s_waitcnt vmcnt(16) lgkmcnt(0)\n\t" \
  "s_barrier\n\t" \
  "v_mfma_f32_16x16x32_bf16 v[136:139], a[212:215], v[100:103], v[136:139]\n\t" \
  "s_lshr_b32 s65, s66, 11\n\t" \
  "v_mfma_f32_16x16x32_bf16 v[140:143], a[220:223], v[100:103], v[140:143]\n\t" \
  "s_mul_i32 s64, s66, s67\n\t" \
  "v_mfma_f32_16x16x32_bf16 v[140:143], a[216:219], v[96:99], v[140:143]\n\t" \
  "v_mfma_f32_16x16x32_bf16 v[144:147], a[224:227], v[96:99], v[144:147]\n\t" \
  "ds_read_b64_tr_b16 a[160:161], v12 offset:16384\n\t" \
  "v_mfma_f32_16x16x32_bf16 v[144:147], a[228:231], v[100:103], v[144:147]\n\t" \
  "s_add_u32 s79, s78, 256\n\t" \
  "v_mfma_f32_16x16x32_bf16 v[148:151], a[236:239], v[100:103], v[148:151]\n\t" \
  "ds_read_b64_tr_b16 a[162:163], v12 offset:17408\n\t" \
  "v_mfma_f32_16x16x32_bf16 v[148:151], a[232:235], v[96:99], v[148:151]\n\t" \
  "s_mul_i32 s65, s65, s89\n\t" \
  "v_mfma_f32_16x16x32_bf16 v[152:155], a[240:243], v[96:99], v[152:155]\n\t" \
  "ds_read_b64_tr_b16 a[164:165], v12 offset:24576\n\t" \
  "v_mfma_f32_16x16x32_bf16 v[152:155], a[244:247], v[100:103], v[152:155]\n\t" \
  "s_add_u32 s65, s65, s64\n\t" \
  "v_mfma_f32_16x16x32_bf16 v[156:159], a[252:255], v[100:103], v[156:159]\n\t" \
  "ds_read_b64_tr_b16 a[166:167], v12 offset:25600\n\t" \
  "v_mfma_f32_16x16x32_bf16 v[156:159], a[248:251], v[96:99], v[156:159]\n\t" \
  "ds_read_b128 v[96:99], v10 offset:16384\n\t" \
  "ds_read_b128 v[100:103], v11 offset:16384\n\t" \
  "v_mfma_f32_16x16x32_bf16 v[188:191], a[248:251], v[104:107], v[188:191]\n\t" \
  "ds_read_b64_tr_b16 a[168:169], v13 offset:16384\n\t" \
  "v_mfma_f32_16x16x32_bf16 v[188:191], a[252:255], v[108:111], v[188:191]\n\t" \
  "s_add_u32 s64, s65, s69\n\t" \
  "v_mfma_f32_16x16x32_bf16 v[184:187], a[244:247], v[108:111], v[184:187]\n\t" \
  "ds_read_b64_tr_b16 a[170:171], v13 offset:17408\n\t" \
  "v_mfma_f32_16x16x32_bf16 v[184:187], a[240:243], v[104:107], v[184:187]\n\t" \
  "s_add_u32 s65, s65, s70\n\t" \
  "v_mfma_f32_16x16x32_bf16 v[180:183], a[232:235], v[104:107], v[180:183]\n\t" \
  "ds_read_b64_tr_b16 a[172:173], v13 offset:24576\n\t" \
  "v_mfma_f32_16x16x32_bf16 v[180:183], a[236:239], v[108:111], v[180:183]\n\t" \
  "v_mfma_f32_16x16x32_bf16 v[176:179], a[228:231], v[108:111], v[176:179]\n\t" \
  "ds_read_b64_tr_b16 a[174:175], v13 offset:25600\n\t" \
  "v_mfma_f32_16x16x32_bf16 v[176:179], a[224:227], v[104:107], v[176:179]\n\t" \
  "s_sub_u32 s87, s87, 1\n\t" \
  "v_mfma_f32_16x16x32_bf16 v[172:175], a[216:219], v[104:107], v[172:175]\n\t" \
  "ds_read_b64_tr_b16 a[176:177], v14 offset:16384\n\t" \
  "v_mfma_f32_16x16x32_bf16 v[172:175], a[220:223], v[108:111], v[172:175]\n\t" \
  "s_cbranch_scc1 EVT114_%=\n\t" \
  "v_mfma_f32_16x16x32_bf16 v[168:171], a[212:215], v[108:111], v[168:171]\n\t" \
  "ds_read_b64_tr_b16 a[178:179], v14 offset:17408\n\t" \
  "v_mfma_f32_16x16x32_bf16 v[168:171], a[208:211], v[104:107], v[168:171]\n\t" \
  "v_mfma_f32_16x16x32_bf16 v[164:167], a[200:203], v[104:107], v[164:167]\n\t" \
  "ds_read_b64_tr_b16 a[180:181], v14 offset:24576\n\t" \
  "v_mfma_f32_16x16x32_bf16 v[164:167], a[204:207], v[108:111], v[164:167]\n\t" \
  "v_mfma_f32_16x16x32_bf16 v[160:163], a[196:199], v[108:111], v[160:163]\n\t" \
  "ds_read_b64_tr_b16 a[182:183], v14 offset:25600\n\t" \
  "v_mfma_f32_16x16x32_bf16 v[160:163], a[192:195], v[104:107], v[160:163]\n\t" \
  "ds_read_b128 v[104:107], v10 offset:18432\n\t" \
  "ds_read_b128 v[108:111], v11 offset:18432\n\t" \
  "v_mfma_f32_16x16x32_bf16 v[192:195], a[192:195], v[112:115], v[192:195]\n\t" \
  "ds_read_b64_tr_b16 a[184:185], v15 offset:16384\n\t" \
  "v_mfma_f32_16x16x32_bf16 v[192:195], a[196:199], v[116:119], v[192:195]\n\t" \
  "v_mfma_f32_16x16x32_bf16 v[196:199], a[204:207], v[116:119], v[196:199]\n\t" \
  "ds_read_b64_tr_b16 a[186:187], v15 offset:17408\n\t" \
  "v_mfma_f32_16x16x32_bf16 v[196:199], a[200:203], v[112:115], v[196:199]\n\t" \
  "v_mfma_f32_16x16x32_bf16 v[200:203], a[208:211], v[112:115], v[200:203]\n\t" \
  "ds_read_b64_tr_b16 a[188:189], v15 offset:24576\n\t" \
  "v_mfma_f32_16x16x32_bf16 v[200:203], a[212:215], v[116:119], v[200:203]\n\t" \
  "v_mfma_f32_16x16x32_bf16 v[204:207], a[220:223], v[116:119], v[204:207]\n\t" \
  "ds_read_b64_tr_b16 a[190:191], v15 offset:25600\n\t" \
  "v_mfma_f32_16x16x32_bf16 v[204:207], a[216:219], v[112:115], v[204:207]\n\t" \
  "s_add_u32 m0, s68, 0\n\t" \
  "v_mfma_f32_16x16x32_bf16 v[208:211], a[224:227], v[112:115], v[208:211]\n\t" \
  "buffer_load_dwordx4 v0, s[36:39], s64 offen offset:0 sc0 sc1 lds\n\t" \
  "v_mfma_f32_16x16x32_bf16 v[208:211], a[228:231], v[116:119], v[208:211]\n\t" \
  "v_mfma_f32_16x16x32_bf16 v[212:215], a[236:239], v[116:119], v[212:215]\n\t" \
  "RET115_%=:\n\t" \
  "v_mfma_f32_16x16x32_bf16 v[212:215], a[232:235], v[112:115], v[212:215]\n\t" \
  "buffer_load_dwordx4 v1, s[36:39], s64 offen offset:1024 sc0 sc1 lds\n\t" \
  "v_mfma_f32_16x16x32_bf16 v[216:219], a[240:243], v[112:115], v[216:219]\n\t" \
  "v_mfma_f32_16x16x32_bf16 v[216:219], a[244:247], v[116:119], v[216:219]\n\t" \
  "v_mfma_f32_16x16x32_bf16 v[220:223], a[252:255], v[116:119], v[220:223]\n\t" \
  "buffer_load_dwordx4 v2, s[36:39], s64 offen offset:2048 sc0 sc1 lds\n\t" \
  "v_mfma_f32_16x16x32_bf16 v[220:223], a[248:251], v[112:115], v[220:223]\n\t" \
  "ds_read_b128 v[112:115], v10 offset:20480\n\t" \
  "ds_read_b128 v[116:119], v11 offset:20480\n\t" \
  "v_mfma_f32_16x16x32_bf16 v[252:255], a[248:251], v[120:123], v[252:255]\n\t" \
  "v_mfma_f32_16x16x32_bf16 v[252:255], a[252:255], v[124:127], v[252:255]\n\t" \
  "buffer_load_dwordx4 v3, s[36:39], s64 offen offset:3072 sc0 sc1 lds\n\t" \
  "v_mfma_f32_16x16x32_bf16 v[248:251], a[244:247], v[124:127], v[248:251]\n\t" \
  "v_mfma_f32_16x16x32_bf16 v[248:251], a[240:243], v[120:123], v[248:251]\n\t" \
  "s_add_u32 m0, s68, 65536\n\t" \
  "v_mfma_f32_16x16x32_bf16 v[244:247], a[232:235], v[120:123], v[244:247]\n\t" \
  "buffer_load_dwordx4 v4, s[40:43], s78 offen offset:0 lds\n\t" \
  "v_mfma_f32_16x16x32_bf16 v[244:247], a[236:239], v[124:127], v[244:247]\n\t" \
  "v_mfma_f32_16x16x32_bf16 v[240:243], a[228:231], v[124:127], v[240:243]\n\t" \
  "v_mfma_f32_16x16x32_bf16 v[240:243], a[224:227], v[120:123], v[240:243]\n\t" \
  "buffer_load_dwordx4 v5, s[40:43], s78 offen offset:1024 lds\n\t" \
  "v_mfma_f32_16x16x32_bf16 v[236:239], a[216:219], v[120:123], v[236:239]\n\t" \
  "s_bitcmp1_b32 s84, s76\n\t" \
  "v_mfma_f32_16x16x32_bf16 v[236:239], a[220:223], v[124:127], v[236:239]\n\t" \
  "s_cbranch_scc1 TRG116_%=\n\t" \
  "TRGR117_%=:\n\t" \
  "v_mfma_f32_16x16x32_bf16 v[232:235], a[212:215], v[124:127], v[232:235]\n\t" \
  "buffer_load_dwordx4 v6, s[40:43], s78 offen offset:2048 lds\n\t" \
  "v_mfma_f32_16x16x32_bf16 v[232:235], a[208:211], v[120:123], v[232:235]\n\t" \
  "s_sub_u32 s76, s76, 1\n\t" \
  "v_mfma_f32_16x16x32_bf16 v[228:231], a[200:203], v[120:123], v[228:231]\n\t" \
  "s_cmp_lg_u32 s76, 0\n\t" \
  "v_mfma_f32_16x16x32_bf16 v[228:231], a[204:207], v[124:127], v[228:231]\n\t" \
  "buffer_load_dwordx4 v7, s[40:43], s78 offen offset:3072 lds\n\t" \
  "v_mfma_f32_16x16x32_bf16 v[224:227], a[196:199], v[124:127], v[224:227]\n\t" \
  "v_mfma_f32_16x16x32_bf16 v[224:227], a[192:195], v[120:123], v[224:227]\n\t" \
  "ds_read_b128 v[120:123], v10 offset:22528\n\t" \
  "ds_read_b128 v[124:127], v11 offset:22528\n\t" \
  "v_mfma_f32_16x16x32_bf16 a[0:3], a[128:131], v[64:67], a[0:3]\n\t" \
  "v_mfma_f32_16x16x32_bf16 a[0:3], a[132:135], v[68:71], a[0:3]\n\t" \
  "v_mfma_f32_16x16x32_bf16 a[4:7], a[140:143], v[68:71], a[4:7]\n\t" \
  "s_cbranch_scc1 LOOP3_%=\n\t" \
  "LEND_%=:\n\t" \
  "s_cmp_lg_u32 s81, 0\n\t" \
  "s_cbranch_scc1 TSEL_%=\n\t" \
  "DEC_%=:\n\t" \
  "v_readlane_b32 s98, v21, 0\n\t" \
  "v_readlane_b32 s99, v21, 1\n\t" \
  "v_readlane_b32 s94, v20, 0\n\t" \
  "v_readlane_b32 s95, v20, 1\n\t" \
  "s_nop 1\n\t" \
  "s_min_u32 s98, s98, s99\n\t" \
  "s_cmp_ge_u32 s98, s85\n\t" \
  "s_cbranch_scc0 DSLOW_%=\n\t" \
  "s_sub_i32 s98, s95, s94\n\t" \
  "DECB_%=:\n\t" \
  "s_mov_b32 s81, 2\n\t" \
  "s_cmp_ge_i32 s98, 90\n\t" \
  "s_addc_u32 s81, s81, 0\n\t" \
  "s_cmp_le_i32 s98, -90\n\t" \
  "s_subb_u32 s81, s81, 0\n\t" \
  "s_cmp_ge_i32 s98, 270\n\t" \
  "s_addc_u32 s81, s81, 0\n\t" \
  "s_cmp_le_i32 s98, -270\n\t" \
  "s_subb_u32 s81, s81, 0\n\t" \
  "s_lshr_b32 s76, s81, 1\n\t" \
  "s_add_u32 s81, s81, 8\n\t" \
  "s_cmp_lg_u32 s76, 0\n\t" \
  "s_cbranch_scc1 LOOP_%=\n\t" \
  "TSEL_%=:\n\t" \
  "s_bitcmp1_b32 s81, 0\n\t" \
  "s_cbranch_scc1 TAILO_%=\n\t" \
  "v_mfma_f32_16x16x32_bf16 a[4:7], a[136:139], v[64:67], a[4:7]\n\t" \
  "s_add_u32 s99, s77, 1\n\t" \
  "v_mfma_f32_16x16x32_bf16 a[8:11], a[144:147], v[64:67], a[8:11]\n\t" \
  "v_mfma_f32_16x16x32_bf16 a[8:11], a[148:151], v[68:71], a[8:11]\n\t" \
  "v_mfma_f32_16x16x32_bf16 a[12:15], a[156:159], v[68:71], a[12:15]\n\t" \
  "v_mfma_f32_16x16x32_bf16 a[12:15], a[152:155], v[64:67], a[12:15]\n\t" \
  "s_waitcnt vmcnt(16) lgkmcnt(0)\n\t" \
  "s_barrier\n\t" \
  "v_mfma_f32_16x16x32_bf16 a[16:19], a[160:163], v[64:67], a[16:19]\n\t" \
  "ds_read_b64_tr_b16 a[192:193], v12 offset:32768\n\t" \
  "v_mfma_f32_16x16x32_bf16 a[16:19], a[164:167], v[68:71], a[16:19]\n\t" \
  "s_andn2_b64 s[92:93], 255, s[58:59]\n\t" \
  "s_cbranch_scc1 POLL128_%=\n\t" \
  "RET127_%=:\n\t" \
  "v_mfma_f32_16x16x32_bf16 a[20:23], a[172:175], v[68:71], a[20:23]\n\t" \
  "ds_read_b64_tr_b16 a[194:195], v12 offset:33792\n\t" \
  "v_mfma_f32_16x16x32_bf16 a[20:23], a[168:171], v[64:67], a[20:23]\n\t" \
  "v_mfma_f32_16x16x32_bf16 a[24:27], a[176:179], v[64:67], a[24:27]\n\t" \
  "ds_read_b64_tr_b16 a[196:197], v12 offset:40960\n\t" \
  "v_mfma_f32_16x16x32_bf16 a[24:27], a[180:183], v[68:71], a[24:27]\n\t" \
  "v_cmp_le_u32 s[92:93], s86, v36\n\t" \
  "v_mfma_f32_16x16x32_bf16 a[28:31], a[188:191], v[68:71], a[28:31]\n\t" \
  "ds_read_b64_tr_b16 a[198:199], v12 offset:41984\n\t" \
  "v_mfma_f32_16x16x32_bf16 a[28:31], a[184:187], v[64:67], a[28:31]\n\t" \
  "ds_read_b128 v[64:67], v10 offset:32768\n\t" \
  "ds_read_b128 v[68:71], v11 offset:32768\n\t" \
  "v_mfma_f32_16x16x32_bf16 a[60:63], a[184:187], v[72:75], a[60:63]\n\t" \
  "ds_read_b64_tr_b16 a[200:201], v13 offset:32768\n\t" \
  "v_mfma_f32_16x16x32_bf16 a[60:63], a[188:191], v[76:79], a[60:63]\n\t" \
  "s_or_b64 s[90:91], s[90:91], s[92:93]\n\t" \
  "v_mfma_f32_16x16x32_bf16 a[56:59], a[180:183], v[76:79], a[56:59]\n\t" \
  "ds_read_b64_tr_b16 a[202:203], v13 offset:33792\n\t" \
  "v_mfma_f32_16x16x32_bf16 a[56:59], a[176:179], v[72:75], a[56:59]\n\t" \
  "s_bitcmp1_b64 s[90:91], s99\n\t" \
  "v_mfma_f32_16x16x32_bf16 a[52:55], a[168:171], v[72:75], a[52:55]\n\t" \
  "ds_read_b64_tr_b16 a[204:205], v13 offset:40960\n\t" \
  "v_mfma_f32_16x16x32_bf16 a[52:55], a[172:175], v[76:79], a[52:55]\n\t" \
  "s_cbranch_scc0 POLL125_%=\n\t" \
  "RET124_%=:\n\t" \
  "v_mfma_f32_16x16x32_bf16 a[48:51], a[164:167], v[76:79], a[48:51]\n\t" \
  "ds_read_b64_tr_b16 a[206:207], v13 offset:41984\n\t" \
  "v_mfma_f32_16x16x32_bf16 a[48:51], a[160:163], v[72:75], a[48:51]\n\t" \
  "v_mfma_f32_16x16x32_bf16 a[44:47], a[152:155], v[72:75], a[44:47]\n\t" \
  "ds_read_b64_tr_b16 a[208:209], v14 offset:32768\n\t" \
  "v_mfma_f32_16x16x32_bf16 a[44:47], a[156:159], v[76:79], a[44:47]\n\t" \
  "v_mfma_f32_16x16x32_bf16 a[40:43], a[148:151], v[76:79], a[40:43]\n\t" \
  "ds_read_b64_tr_b16 a[210:211], v14 offset:33792\n\t" \
  "v_mfma_f32_16x16x32_bf16 a[40:43], a[144:147], v[72:75], a[40:43]\n\t" \
  "v_mfma_f32_16x16x32_bf16 a[36:39], a[136:139], v[72:75], a[36:39]\n\t" \
  "ds_read_b64_tr_b16 a[212:213], v14 offset:40960\n\t" \
  "v_mfma_f32_16x16x32_bf16 a[36:39], a[140:143], v[76:79], a[36:39]\n\t" \
  "v_mfma_f32_16x16x32_bf16 a[32:35], a[132:135], v[76:79], a[32:35]\n\t" \
  "ds_read_b64_tr_b16 a[214:215], v14 offset:41984\n\t" \
  "v_mfma_f32_16x16x32_bf16 a[32:35], a[128:131], v[72:75], a[32:35]\n\t" \
  "ds_read_b128 v[72:75], v10 offset:34816\n\t" \
  "ds_read_b128 v[76:79], v11 offset:34816\n\t" \
  "v_mfma_f32_16x16x32_bf16 a[64:67], a[128:131], v[80:83], a[64:67]\n\t" \
  "ds_read_b64_tr_b16 a[216:217], v15 offset:32768\n\t" \
  "v_mfma_f32_16x16x32_bf16 a[64:67], a[132:135], v[84:87], a[64:67]\n\t" \
  "v_mfma_f32_16x16x32_bf16 a[68:71], a[140:143], v[84:87], a[68:71]\n\t" \
  "ds_read_b64_tr_b16 a[218:219], v15 offset:33792\n\t" \
  "v_mfma_f32_16x16x32_bf16 a[68:71], a[136:139], v[80:83], a[68:71]\n\t" \
  "v_mfma_f32_16x16x32_bf16 a[72:75], a[144:147], v[80:83], a[72:75]\n\t" \
  "ds_read_b64_tr_b16 a[220:221], v15 offset:40960\n\t" \
  "v_mfma_f32_16x16x32_bf16 a[72:75], a[148:151], v[84:87], a[72:75]\n\t" \
  "v_mfma_f32_16x16x32_bf16 a[76:79], a[156:159], v[84:87], a[76:79]\n\t" \
  "ds_read_b64_tr_b16 a[222:223], v15 offset:41984\n\t" \
  "v_mfma_f32_16x16x32_bf16 a[76:79], a[152:155], v[80:83], a[76:79]\n\t" \
  "s_add_u32 m0, s68, 16384\n\t" \
  "v_mfma_f32_16x16x32_bf16 a[80:83], a[160:163], v[80:83], a[80:83]\n\t" \
  "buffer_load_dwordx4 v0, s[36:39], s65 offen offset:0 sc0 sc1 lds\n\t" \
  "v_mfma_f32_16x16x32_bf16 a[80:83], a[164:167], v[84:87], a[80:83]\n\t" \
  "v_mfma_f32_16x16x32_bf16 a[84:87], a[172:175], v[84:87], a[84:87]\n\t" \
  "v_mfma_f32_16x16x32_bf16 a[84:87], a[168:171], v[80:83], a[84:87]\n\t" \
  "buffer_load_dwordx4 v1, s[36:39], s65 offen offset:1024 sc0 sc1 lds\n\t" \
  "v_mfma_f32_16x16x32_bf16 a[88:91], a[176:179], v[80:83], a[88:91]\n\t" \
  "v_mfma_f32_16x16x32_bf16 a[88:91], a[180:183], v[84:87], a[88:91]\n\t" \
  "v_mfma_f32_16x16x32_bf16 a[92:95], a[188:191], v[84:87], a[92:95]\n\t" \
  "buffer_load_dwordx4 v2, s[36:39], s65 offen offset:2048 sc0 sc1 lds\n\t" \
  "v_mfma_f32_16x16x32_bf16 a[92:95], a[184:187], v[80:83], a[92:95]\n\t" \
  "ds_read_b128 v[80:83], v10 offset:36864\n\t" \
  "ds_read_b128 v[84:87], v11 offset:36864\n\t" \
  "v_mfma_f32_16x16x32_bf16 a[124:127], a[184:187], v[88:91], a[124:127]\n\t" \
  "v_mfma_f32_16x16x32_bf16 a[124:127], a[188:191], v[92:95], a[124:127]\n\t" \
  "buffer_load_dwordx4 v3, s[36:39], s65 offen offset:3072 sc0 sc1 lds\n\t" \
  "v_mfma_f32_16x16x32_bf16 a[120:123], a[180:183], v[92:95], a[120:123]\n\t" \
  "v_mfma_f32_16x16x32_bf16 a[120:123], a[176:179], v[88:91], a[120:123]\n\t" \
  "s_add_u32 m0, s68, 81920\n\t" \
  "v_mfma_f32_16x16x32_bf16 a[116:119], a[168:171], v[88:91], a[116:119]\n\t" \
  "buffer_load_dwordx4 v4, s[40:43], s79 offen offset:0 lds\n\t" \
  "v_mfma_f32_16x16x32_bf16 a[116:119], a[172:175], v[92:95], a[116:119]\n\t" \
  "s_bitcmp1_b64 s[90:91], s99\n\t" \
  "v_mfma_f32_16x16x32_bf16 a[112:115], a[164:167], v[92:95], a[112:115]\n\t" \
  "s_cbranch_scc0 STG126_%=\n\t" \
  "RET120_%=:\n\t" \
  "v_mfma_f32_16x16x32_bf16 a[112:115], a[160:163], v[88:91], a[112:115]\n\t" \
  "buffer_load_dwordx4 v5, s[40:43], s79 offen offset:1024 lds\n\t" \
  "v_mfma_f32_16x16x32_bf16 a[108:111], a[152:155], v[88:91], a[108:111]\n\t" \
  "v_mfma_f32_16x16x32_bf16 a[108:111], a[156:159], v[92:95], a[108:111]\n\t" \
  "v_mfma_f32_16x16x32_bf16 a[104:107], a[148:151], v[92:95], a[104:107]\n\t" \
  "buffer_load_dwordx4 v6, s[40:43], s79 offen offset:2048 lds\n\t" \
  "v_mfma_f32_16x16x32_bf16 a[104:107], a[144:147], v[88:91], a[104:107]\n\t" \
  "v_mfma_f32_16x16x32_bf16 a[100:103], a[136:139], v[88:91], a[100:103]\n\t" \
  "v_mfma_f32_16x16x32_bf16 a[100:103], a[140:143], v[92:95], a[100:103]\n\t" \
  "buffer_load_dwordx4 v7, s[40:43], s79 offen offset:3072 lds\n\t" \
  "v_mfma_f32_16x16x32_bf16 a[96:99], a[132:135], v[92:95], a[96:99]\n\t" \
  "s_add_u32 s98, s71, 0\n\t" \
  "v_readlane_b32 s98, v17, s98\n\t" \
  "s_nop 1\n\t" \
  "s_lshl_b32 s98, s98, 13\n\t" \
  "s_mov_b32 exec_hi, 0\n\t" \
  "buffer_load_dword v50, v23, s[40:43], s98 offen\n\t" \
  "s_mov_b32 exec_hi, -1\n\t" \
  "v_mfma_f32_16x16x32_bf16 a[96:99], a[128:131], v[88:91], a[96:99]\n\t" \
  "ds_read_b128 v[88:91], v10 offset:38912\n\t" \
  "ds_read_b128 v[92:95], v11 offset:38912\n\t" \
  "v_mfma_f32_16x16x32_bf16 v[128:131], a[128:131], v[96:99], v[128:131]\n\t" \
  "s_add_u32 s77, s77, 1\n\t" \
  "v_mfma_f32_16x16x32_bf16 v[128:131], a[132:135], v[100:103], v[128:131]\n\t" \
  "v_readlane_b32 s66, v16, s77\n\t" \
  "v_mfma_f32_16x16x32_bf16 v[132:135], a[140:143], v[100:103], v[132:135]\n\t" \
  "v_mfma_f32_16x16x32_bf16 v[132:135], a[136:139], v[96:99], v[132:135]\n\t" \
  "v_mfma_f32_16x16x32_bf16 v[136:139], a[144:147], v[96:99], v[136:139]\n\t" \
  "s_lshl_b32 s78, s66, 13\n\t" \
  "v_mfma_f32_16x16x32_bf16 v[136:139], a[148:151], v[100:103], v[136:139]\n\t" \
  "s_lshr_b32 s65, s66, 11\n\t" \
  "v_mfma_f32_16x16x32_bf16 v[140:143], a[156:159], v[100:103], v[140:143]\n\t" \
  "s_mul_i32 s64, s66, s67\n\t" \
  "v_mfma_f32_16x16x32_bf16 v[140:143], a[152:155], v[96:99], v[140:143]\n\t" \
  "s_waitcnt vmcnt(17) lgkmcnt(0)\n\t" \
  "s_barrier\n\t" \
  "v_mfma_f32_16x16x32_bf16 v[144:147], a[160:163], v[96:99], v[144:147]\n\t" \
  "ds_read_b64_tr_b16 a[224:225], v12 offset:49152\n\t" \
  "v_mfma_f32_16x16x32_bf16 v[144:147], a[164:167], v[100:103], v[144:147]\n\t" \
  "s_add_u32 s79, s78, 256\n\t" \
  "v_mfma_f32_16x16x32_bf16 v[148:151], a[172:175], v[100:103], v[148:151]\n\t" \
  "ds_read_b64_tr_b16 a[226:227], v12 offset:50176\n\t" \
  "v_mfma_f32_16x16x32_bf16 v[148:151], a[168:171], v[96:99], v[148:151]\n\t" \
  "s_mul_i32 s65, s65, s89\n\t" \
  "v_mfma_f32_16x16x32_bf16 v[152:155], a[176:179], v[96:99], v[152:155]\n\t" \
  "ds_read_b64_tr_b16 a[228:229], v12 offset:57344\n\t" \
  "v_mfma_f32_16x16x32_bf16 v[152:155], a[180:183], v[100:103], v[152:155]\n\t" \
  "s_add_u32 s65, s65, s64\n\t" \
  "v_mfma_f32_16x16x32_bf16 v[156:159], a[188:191], v[100:103], v[156:159]\n\t" \
  "ds_read_b64_tr_b16 a[230:231], v12 offset:58368\n\t" \
  "v_mfma_f32_16x16x32_bf16 v[156:159], a[184:187], v[96:99], v[156:159]\n\t" \
  "ds_read_b128 v[96:99], v10 offset:49152\n\t" \
  "ds_read_b128 v[100:103], v11 offset:49152\n\t" \
  "v_mfma_f32_16x16x32_bf16 v[188:191], a[184:187], v[104:107], v[188:191]\n\t" \
  "ds_read_b64_tr_b16 a[232:233], v13 offset:49152\n\t" \
  "v_mfma_f32_16x16x32_bf16 v[188:191], a[188:191], v[108:111], v[188:191]\n\t" \
  "s_add_u32 s64, s65, s69\n\t" \
  "v_mfma_f32_16x16x32_bf16 v[184:187], a[180:183], v[108:111], v[184:187]\n\t" \
  "ds_read_b64_tr_b16 a[234:235], v13 offset:50176\n\t" \
  "v_mfma_f32_16x16x32_bf16 v[184:187], a[176:179], v[104:107], v[184:187]\n\t" \
  "s_add_u32 s65, s65, s70\n\t" \
  "v_mfma_f32_16x16x32_bf16 v[180:183], a[168:171], v[104:107], v[180:183]\n\t" \
  "ds_read_b64_tr_b16 a[236:237], v13 offset:57344\n\t" \
  "v_mfma_f32_16x16x32_bf16 v[180:183], a[172:175], v[108:111], v[180:183]\n\t" \
  "v_mfma_f32_16x16x32_bf16 v[176:179], a[164:167], v[108:111], v[176:179]\n\t" \
  "ds_read_b64_tr_b16 a[238:239], v13 offset:58368\n\t" \
  "v_mfma_f32_16x16x32_bf16 v[176:179], a[160:163], v[104:107], v[176:179]\n\t" \
  "v_mfma_f32_16x16x32_bf16 v[172:175], a[152:155], v[104:107], v[172:175]\n\t" \
  "ds_read_b64_tr_b16 a[240:241], v14 offset:49152\n\t" \
  "v_mfma_f32_16x16x32_bf16 v[172:175], a[156:159], v[108:111], v[172:175]\n\t" \
  "v_mfma_f32_16x16x32_bf16 v[168:171], a[148:151], v[108:111], v[168:171]\n\t" \
  "ds_read_b64_tr_b16 a[242:243], v14 offset:50176\n\t" \
  "v_mfma_f32_16x16x32_bf16 v[168:171], a[144:147], v[104:107], v[168:171]\n\t" \
  "v_mfma_f32_16x16x32_bf16 v[164:167], a[136:139], v[104:107], v[164:167]\n\t" \
  "ds_read_b64_tr_b16 a[244:245], v14 offset:57344\n\t" \
  "v_mfma_f32_16x16x32_bf16 v[164:167], a[140:143], v[108:111], v[164:167]\n\t" \
  "v_mfma_f32_16x16x32_bf16 v[160:163], a[132:135], v[108:111], v[160:163]\n\t" \
  "ds_read_b64_tr_b16 a[246:247], v14 offset:58368\n\t" \
  "v_mfma_f32_16x16x32_bf16 v[160:163], a[128:131], v[104:107], v[160:163]\n\t" \
  "ds_read_b128 v[104:107], v10 offset:51200\n\t" \
  "ds_read_b128 v[108:111], v11 offset:51200\n\t" \
  "v_mfma_f32_16x16x32_bf16 v[192:195], a[128:131], v[112:115], v[192:195]\n\t" \
  "ds_read_b64_tr_b16 a[248:249], v15 offset:49152\n\t" \
  "v_mfma_f32_16x16x32_bf16 v[192:195], a[132:135], v[116:119], v[192:195]\n\t" \
  "v_mfma_f32_16x16x32_bf16 v[196:199], a[140:143], v[116:119], v[196:199]\n\t" \
  "ds_read_b64_tr_b16 a[250:251], v15 offset:50176\n\t" \
  "v_mfma_f32_16x16x32_bf16 v[196:199], a[136:139], v[112:115], v[196:199]\n\t" \
  "v_mfma_f32_16x16x32_bf16 v[200:203], a[144:147], v[112:115], v[200:203]\n\t" \
  "ds_read_b64_tr_b16 a[252:253], v15 offset:57344\n\t" \
  "v_mfma_f32_16x16x32_bf16 v[200:203], a[148:151], v[116:119], v[200:203]\n\t" \
  "v_mfma_f32_16x16x32_bf16 v[204:207], a[156:159], v[116:119], v[204:207]\n\t" \
  "ds_read_b64_tr_b16 a[254:255], v15 offset:58368\n\t" \
  "v_mfma_f32_16x16x32_bf16 v[204:207], a[152:155], v[112:115], v[204:207]\n\t" \
  "s_add_u32 m0, s68, 32768\n\t" \
  "v_mfma_f32_16x16x32_bf16 v[208:211], a[160:163], v[112:115], v[208:211]\n\t" \
  "buffer_load_dwordx4 v0, s[36:39], s64 offen offset:0 sc0 sc1 lds\n\t" \
  "v_mfma_f32_16x16x32_bf16 v[208:211], a[164:167], v[116:119], v[208:211]\n\t" \
  "v_mfma_f32_16x16x32_bf16 v[212:215], a[172:175], v[116:119], v[212:215]\n\t" \
  "v_mfma_f32_16x16x32_bf16 v[212:215], a[168:171], v[112:115], v[212:215]\n\t" \
  "buffer_load_dwordx4 v1, s[36:39], s64 offen offset:1024 sc0 sc1 lds\n\t" \
  "v_mfma_f32_16x16x32_bf16 v[216:219], a[176:179], v[112:115], v[216:219]\n\t" \
  "v_mfma_f32_16x16x32_bf16 v[216:219], a[180:183], v[116:119], v[216:219]\n\t" \
  "v_mfma_f32_16x16x32_bf16 v[220:223], a[188:191], v[116:119], v[220:223]\n\t" \
  "buffer_load_dwordx4 v2, s[36:39], s64 offen offset:2048 sc0 sc1 lds\n\t" \
  "v_mfma_f32_16x16x32_bf16 v[220:223], a[184:187], v[112:115], v[220:223]\n\t" \
  "ds_read_b128 v[112:115], v10 offset:53248\n\t" \
  "ds_read_b128 v[116:119], v11 offset:53248\n\t" \
  "v_mfma_f32_16x16x32_bf16 v[252:255], a[184:187], v[120:123], v[252:255]\n\t" \
  "v_mfma_f32_16x16x32_bf16 v[252:255], a[188:191], v[124:127], v[252:255]\n\t" \
  "buffer_load_dwordx4 v3, s[36:39], s64 offen offset:3072 sc0 sc1 lds\n\t" \
  "v_mfma_f32_16x16x32_bf16 v[248:251], a[180:183], v[124:127], v[248:251]\n\t" \
  "v_mfma_f32_16x16x32_bf16 v[248:251], a[176:179], v[120:123], v[248:251]\n\t" \
  "s_add_u32 m0, s68, 98304\n\t" \
  "v_mfma_f32_16x16x32_bf16 v[244:247], a[168:171], v[120:123], v[244:247]\n\t" \
  "buffer_load_dwordx4 v4, s[40:43], s78 offen offset:0 lds\n\t" \
  "v_mfma_f32_16x16x32_bf16 v[244:247], a[172:175], v[124:127], v[244:247]\n\t" \
  "v_mfma_f32_16x16x32_bf16 v[240:243], a[164:167], v[124:127], v[240:243]\n\t" \
  "v_mfma_f32_16x16x32_bf16 v[240:243], a[160:163], v[120:123], v[240:243]\n\t" \
  "buffer_load_dwordx4 v5, s[40:43], s78 offen offset:1024 lds\n\t" \
  "v_mfma_f32_16x16x32_bf16 v[236:239], a[152:155], v[120:123], v[236:239]\n\t" \
  "v_mfma_f32_16x16x32_bf16 v[236:239], a[156:159], v[124:127], v[236:239]\n\t" \
  "v_mfma_f32_16x16x32_bf16 v[232:235], a[148:151], v[124:127], v[232:235]\n\t" \
  "buffer_load_dwordx4 v6, s[40:43], s78 offen offset:2048 lds\n\t" \
  "v_mfma_f32_16x16x32_bf16 v[232:235], a[144:147], v[120:123], v[232:235]\n\t" \
  "v_mfma_f32_16x16x32_bf16 v[228:231], a[136:139], v[120:123], v[228:231]\n\t" \
  "v_mfma_f32_16x16x32_bf16 v[228:231], a[140:143], v[124:127], v[228:231]\n\t" \
  "buffer_load_dwordx4 v7, s[40:43], s78 offen offset:3072 lds\n\t" \
  "v_mfma_f32_16x16x32_bf16 v[224:227], a[132:135], v[124:127], v[224:227]\n\t" \
  "s_add_u32 s98, s71, 4\n\t" \
  "v_readlane_b32 s98, v17, s98\n\t" \
  "s_nop 1\n\t" \
  "s_lshl_b32 s98, s98, 13\n\t" \
  "s_mov_b32 exec_hi, 0\n\t" \
  "buffer_load_dword v50, v23, s[40:43], s98 offen\n\t" \
  "s_mov_b32 exec_hi, -1\n\t" \
  "v_mfma_f32_16x16x32_bf16 v[224:227], a[128:131], v[120:123], v[224:227]\n\t" \
  "ds_read_b128 v[120:123], v10 offset:55296\n\t" \
  "ds_read_b128 v[124:127], v11 offset:55296\n\t" \
  "v_mfma_f32_16x16x32_bf16 a[0:3], a[192:195], v[64:67], a[0:3]\n\t" \
  "v_mfma_f32_16x16x32_bf16 a[0:3], a[196:199], v[68:71], a[0:3]\n\t" \
  "v_mfma_f32_16x16x32_bf16 a[4:7], a[204:207], v[68:71], a[4:7]\n\t" \
  "v_mfma_f32_16x16x32_bf16 a[4:7], a[200:203], v[64:67], a[4:7]\n\t" \
  "v_mfma_f32_16x16x32_bf16 a[8:11], a[208:211], v[64:67], a[8:11]\n\t" \
  "v_mfma_f32_16x16x32_bf16 a[8:11], a[212:215], v[68:71], a[8:11]\n\t" \
  "v_mfma_f32_16x16x32_bf16 a[12:15], a[220:223], v[68:71], a[12:15]\n\t" \
  "v_mfma_f32_16x16x32_bf16 a[12:15], a[216:219], v[64:67], a[12:15]\n\t" \
  "s_waitcnt vmcnt(18) lgkmcnt(0)\n\t" \
  "s_barrier\n\t" \
  "v_mfma_f32_16x16x32_bf16 a[16:19], a[224:227], v[64:67], a[16:19]\n\t" \
  "ds_read_b64_tr_b16 a[128:129], v12 offset:0\n\t" \
  "v_mfma_f32_16x16x32_bf16 a[16:19], a[228:231], v[68:71], a[16:19]\n\t" \
  "s_andn2_b64 s[92:93], 255, s[58:59]\n\t" \
  "s_cbranch_scc1 POLL132_%=\n\t" \
  "RET131_%=:\n\t" \
  "v_mfma_f32_16x16x32_bf16 a[20:23], a[236:239], v[68:71], a[20:23]\n\t" \
  "ds_read_b64_tr_b16 a[130:131], v12 offset:1024\n\t" \
  "v_mfma_f32_16x16x32_bf16 a[20:23], a[232:235], v[64:67], a[20:23]\n\t" \
  "v_mfma_f32_16x16x32_bf16 a[24:27], a[240:243], v[64:67], a[24:27]\n\t" \
  "ds_read_b64_tr_b16 a[132:133], v12 offset:8192\n\t" \
  "v_mfma_f32_16x16x32_bf16 a[24:27], a[244:247], v[68:71], a[24:27]\n\t" \
  "v_mfma_f32_16x16x32_bf16 a[28:31], a[252:255], v[68:71], a[28:31]\n\t" \
  "ds_read_b64_tr_b16 a[134:135], v12 offset:9216\n\t" \
  "v_mfma_f32_16x16x32_bf16 a[28:31], a[248:251], v[64:67], a[28:31]\n\t" \
  "ds_read_b128 v[64:67], v10 offset:0\n\t" \
  "ds_read_b128 v[68:71], v11 offset:0\n\t" \
  "v_mfma_f32_16x16x32_bf16 a[60:63], a[248:251], v[72:75], a[60:63]\n\t" \
  "ds_read_b64_tr_b16 a[136:137], v13 offset:0\n\t" \
  "v_mfma_f32_16x16x32_bf16 a[60:63], a[252:255], v[76:79], a[60:63]\n\t" \
  "v_mfma_f32_16x16x32_bf16 a[56:59], a[244:247], v[76:79], a[56:59]\n\t" \
  "ds_read_b64_tr_b16 a[138:139], v13 offset:1024\n\t" \
  "v_mfma_f32_16x16x32_bf16 a[56:59], a[240:243], v[72:75], a[56:59]\n\t" \
  "v_mfma_f32_16x16x32_bf16 a[52:55], a[232:235], v[72:75], a[52:55]\n\t" \
  "ds_read_b64_tr_b16 a[140:141], v13 offset:8192\n\t" \
  "v_mfma_f32_16x16x32_bf16 a[52:55], a[236:239], v[76:79], a[52:55]\n\t" \
  "v_mfma_f32_16x16x32_bf16 a[48:51], a[228:231], v[76:79], a[48:51]\n\t" \
  "ds_read_b64_tr_b16 a[142:143], v13 offset:9216\n\t" \
  "v_mfma_f32_16x16x32_bf16 a[48:51], a[224:227], v[72:75], a[48:51]\n\t" \
  "v_mfma_f32_16x16x32_bf16 a[44:47], a[216:219], v[72:75], a[44:47]\n\t" \
  "ds_read_b64_tr_b16 a[144:145], v14 offset:0\n\t" \
  "v_mfma_f32_16x16x32_bf16 a[44:47], a[220:223], v[76:79], a[44:47]\n\t" \
  "v_mfma_f32_16x16x32_bf16 a[40:43], a[212:215], v[76:79], a[40:43]\n\t" \
  "ds_read_b64_tr_b16 a[146:147], v14 offset:1024\n\t" \
  "v_mfma_f32_16x16x32_bf16 a[40:43], a[208:211], v[72:75], a[40:43]\n\t" \
  "v_mfma_f32_16x16x32_bf16 a[36:39], a[200:203], v[72:75], a[36:39]\n\t" \
  "ds_read_b64_tr_b16 a[148:149], v14 offset:8192\n\t" \
  "v_mfma_f32_16x16x32_bf16 a[36:39], a[204:207], v[76:79], a[36:39]\n\t" \
  "v_mfma_f32_16x16x32_bf16 a[32:35], a[196:199], v[76:79], a[32:35]\n\t" \
  "ds_read_b64_tr_b16 a[150:151], v14 offset:9216\n\t" \
  "v_mfma_f32_16x16x32_bf16 a[32:35], a[192:195], v[72:75], a[32:35]\n\t" \
  "ds_read_b128 v[72:75], v10 offset:2048\n\t" \
  "ds_read_b128 v[76:79], v11 offset:2048\n\t" \
  "v_mfma_f32_16x16x32_bf16 a[64:67], a[192:195], v[80:83], a[64:67]\n\t" \
  "ds_read_b64_tr_b16 a[152:153], v15 offset:0\n\t" \
  "v_mfma_f32_16x16x32_bf16 a[64:67], a[196:199], v[84:87], a[64:67]\n\t" \
  "v_mfma_f32_16x16x32_bf16 a[68:71], a[204:207], v[84:87], a[68:71]\n\t" \
  "ds_read_b64_tr_b16 a[154:155], v15 offset:1024\n\t" \
  "v_mfma_f32_16x16x32_bf16 a[68:71], a[200:203], v[80:83], a[68:71]\n\t" \
  "v_mfma_f32_16x16x32_bf16 a[72:75], a[208:211], v[80:83], a[72:75]\n\t" \
  "ds_read_b64_tr_b16 a[156:157], v15 offset:8192\n\t" \
  "v_mfma_f32_16x16x32_bf16 a[72:75], a[212:215], v[84:87], a[72:75]\n\t" \
  "v_mfma_f32_16x16x32_bf16 a[76:79], a[220:223], v[84:87], a[76:79]\n\t" \
  "ds_read_b64_tr_b16 a[158:159], v15 offset:9216\n\t" \
  "v_mfma_f32_16x16x32_bf16 a[76:79], a[216:219], v[80:83], a[76:79]\n\t" \
  "s_add_u32 m0, s68, 49152\n\t" \
  "v_mfma_f32_16x16x32_bf16 a[80:83], a[224:227], v[80:83], a[80:83]\n\t" \
  "buffer_load_dwordx4 v0, s[36:39], s65 offen offset:0 sc0 sc1 lds\n\t" \
  "v_mfma_f32_16x16x32_bf16 a[80:83], a[228:231], v[84:87], a[80:83]\n\t" \
  "v_mfma_f32_16x16x32_bf16 a[84:87], a[236:239], v[84:87], a[84:87]\n\t" \
  "v_mfma_f32_16x16x32_bf16 a[84:87], a[232:235], v[80:83], a[84:87]\n\t" \
  "buffer_load_dwordx4 v1, s[36:39], s65 offen offset:1024 sc0 sc1 lds\n\t" \
  "v_mfma_f32_16x16x32_bf16 a[88:91], a[240:243], v[80:83], a[88:91]\n\t" \
  "v_mfma_f32_16x16x32_bf16 a[88:91], a[244:247], v[84:87], a[88:91]\n\t" \
  "v_mfma_f32_16x16x32_bf16 a[92:95], a[252:255], v[84:87], a[92:95]\n\t" \
  "buffer_load_dwordx4 v2, s[36:39], s65 offen offset:2048 sc0 sc1 lds\n\t" \
  "v_mfma_f32_16x16x32_bf16 a[92:95], a[248:251], v[80:83], a[92:95]\n\t" \
  "ds_read_b128 v[80:83], v10 offset:4096\n\t" \
  "ds_read_b128 v[84:87], v11 offset:4096\n\t" \
  "v_mfma_f32_16x16x32_bf16 a[124:127], a[248:251], v[88:91], a[124:127]\n\t" \
  "v_mfma_f32_16x16x32_bf16 a[124:127], a[252:255], v[92:95], a[124:127]\n\t" \
  "buffer_load_dwordx4 v3, s[36:39], s65 offen offset:3072 sc0 sc1 lds\n\t" \
  "v_mfma_f32_16x16x32_bf16 a[120:123], a[244:247], v[92:95], a[120:123]\n\t" \
  "v_mfma_f32_16x16x32_bf16 a[120:123], a[240:243], v[88:91], a[120:123]\n\t" \
  "s_add_u32 m0, s68, 114688\n\t" \
  "v_mfma_f32_16x16x32_bf16 a[116:119], a[232:235], v[88:91], a[116:119]\n\t" \
  "buffer_load_dwordx4 v4, s[40:43], s79 offen offset:0 lds\n\t" \
  "v_mfma_f32_16x16x32_bf16 a[116:119], a[236:239], v[92:95], a[116:119]\n\t" \
  "v_mfma_f32_16x16x32_bf16 a[112:115], a[228:231], v[92:95], a[112:115]\n\t" \
  "v_mfma_f32_16x16x32_bf16 a[112:115], a[224:227], v[88:91], a[112:115]\n\t" \
  "buffer_load_dwordx4 v5, s[40:43], s79 offen offset:1024 lds\n\t" \
  "v_mfma_f32_16x16x32_bf16 a[108:111], a[216:219], v[88:91], a[108:111]\n\t" \
  "v_mfma_f32_16x16x32_bf16 a[108:111], a[220:223], v[92:95], a[108:111]\n\t" \
  "v_mfma_f32_16x16x32_bf16 a[104:107], a[212:215], v[92:95], a[104:107]\n\t" \
  "buffer_load_dwordx4 v6, s[40:43], s79 offen offset:2048 lds\n\t" \
  "v_mfma_f32_16x16x32_bf16 a[104:107], a[208:211], v[88:91], a[104:107]\n\t" \
  "v_mfma_f32_16x16x32_bf16 a[100:103], a[200:203], v[88:91], a[100:103]\n\t" \
  "v_mfma_f32_16x16x32_bf16 a[100:103], a[204:207], v[92:95], a[100:103]\n\t" \
  "buffer_load_dwordx4 v7, s[40:43], s79 offen offset:3072 lds\n\t" \
  "v_mfma_f32_16x16x32_bf16 a[96:99], a[196:199], v[92:95], a[96:99]\n\t" \
  "v_mfma_f32_16x16x32_bf16 a[96:99], a[192:195], v[88:91], a[96:99]\n\t" \
  "ds_read_b128 v[88:91], v10 offset:6144\n\t" \
  "ds_read_b128 v[92:95], v11 offset:6144\n\t" \
  "v_mfma_f32_16x16x32_bf16 v[128:131], a[192:195], v[96:99], v[128:131]\n\t" \
  "v_mfma_f32_16x16x32_bf16 v[128:131], a[196:199], v[100:103], v[128:131]\n\t" \
  "v_mfma_f32_16x16x32_bf16 v[132:135], a[204:207], v[100:103], v[132:135]\n\t" \
  "v_mfma_f32_16x16x32_bf16 v[132:135], a[200:203], v[96:99], v[132:135]\n\t" \
  "v_mfma_f32_16x16x32_bf16 v[136:139], a[208:211], v[96:99], v[136:139]\n\t" \
  "v_mfma_f32_16x16x32_bf16 v[136:139], a[212:215], v[100:103], v[136:139]\n\t" \
  "v_mfma_f32_16x16x32_bf16 v[140:143], a[220:223], v[100:103], v[140:143]\n\t" \
  "v_mfma_f32_16x16x32_bf16 v[140:143], a[216:219], v[96:99], v[140:143]\n\t" \
  "s_waitcnt vmcnt(18) lgkmcnt(0)\n\t" \
  "s_barrier\n\t" \
  "v_mfma_f32_16x16x32_bf16 v[144:147], a[224:227], v[96:99], v[144:147]\n\t" \
  "ds_read_b64_tr_b16 a[160:161], v12 offset:16384\n\t" \
  "v_readlane_b32 s66, v17, 0\n\t" \
  "s_nop 0\n\t" \
  "s_lshl_b32 s78, s66, 13\n\t" \
  "s_lshl_b32 s65, s66, 10\n\t" \
  "s_add_u32 s79, s78, 256\n\t" \
  "s_add_u32 s65, s65, s70\n\t" \
  "s_nop 1\n\t" \
  "v_mfma_f32_16x16x32_bf16 v[144:147], a[228:231], v[100:103], v[144:147]\n\t" \
  "v_mfma_f32_16x16x32_bf16 v[148:151], a[236:239], v[100:103], v[148:151]\n\t" \
  "ds_read_b64_tr_b16 a[162:163], v12 offset:17408\n\t" \
  "v_mfma_f32_16x16x32_bf16 v[148:151], a[232:235], v[96:99], v[148:151]\n\t" \
  "v_mfma_f32_16x16x32_bf16 v[152:155], a[240:243], v[96:99], v[152:155]\n\t" \
  "ds_read_b64_tr_b16 a[164:165], v12 offset:24576\n\t" \
  "v_mfma_f32_16x16x32_bf16 v[152:155], a[244:247], v[100:103], v[152:155]\n\t" \
  "v_mfma_f32_16x16x32_bf16 v[156:159], a[252:255], v[100:103], v[156:159]\n\t" \
  "ds_read_b64_tr_b16 a[166:167], v12 offset:25600\n\t" \
  "v_mfma_f32_16x16x32_bf16 v[156:159], a[248:251], v[96:99], v[156:159]\n\t" \
  "ds_read_b128 v[96:99], v10 offset:16384\n\t" \
  "ds_read_b128 v[100:103], v11 offset:16384\n\t" \
  "v_mfma_f32_16x16x32_bf16 v[188:191], a[248:251], v[104:107], v[188:191]\n\t" \
  "ds_read_b64_tr_b16 a[168:169], v13 offset:16384\n\t" \
  "v_mfma_f32_16x16x32_bf16 v[188:191], a[252:255], v[108:111], v[188:191]\n\t" \
  "v_mfma_f32_16x16x32_bf16 v[184:187], a[244:247], v[108:111], v[184:187]\n\t" \
  "ds_read_b64_tr_b16 a[170:171], v13 offset:17408\n\t" \
  "v_mfma_f32_16x16x32_bf16 v[184:187], a[240:243], v[104:107], v[184:187]\n\t" \
  "v_mfma_f32_16x16x32_bf16 v[180:183], a[232:235], v[104:107], v[180:183]\n\t" \
  "ds_read_b64_tr_b16 a[172:173], v13 offset:24576\n\t" \
  "v_mfma_f32_16x16x32_bf16 v[180:183], a[236:239], v[108:111], v[180:183]\n\t" \
  "v_mfma_f32_16x16x32_bf16 v[176:179], a[228:231], v[108:111], v[176:179]\n\t" \
  "ds_read_b64_tr_b16 a[174:175], v13 offset:25600\n\t" \
  "v_mfma_f32_16x16x32_bf16 v[176:179], a[224:227], v[104:107], v[176:179]\n\t" \
  "v_mfma_f32_16x16x32_bf16 v[172:175], a[216:219], v[104:107], v[172:175]\n\t" \
  "ds_read_b64_tr_b16 a[176:177], v14 offset:16384\n\t" \
  "v_mfma_f32_16x16x32_bf16 v[172:175], a[220:223], v[108:111], v[172:175]\n\t" \
  "v_mfma_f32_16x16x32_bf16 v[168:171], a[212:215], v[108:111], v[168:171]\n\t" \
  "ds_read_b64_tr_b16 a[178:179], v14 offset:17408\n\t" \
  "v_mfma_f32_16x16x32_bf16 v[168:171], a[208:211], v[104:107], v[168:171]\n\t" \
  "v_mfma_f32_16x16x32_bf16 v[164:167], a[200:203], v[104:107], v[164:167]\n\t" \
  "ds_read_b64_tr_b16 a[180:181], v14 offset:24576\n\t" \
  "v_mfma_f32_16x16x32_bf16 v[164:167], a[204:207], v[108:111], v[164:167]\n\t" \
  "v_mfma_f32_16x16x32_bf16 v[160:163], a[196:199], v[108:111], v[160:163]\n\t" \
  "ds_read_b64_tr_b16 a[182:183], v14 offset:25600\n\t" \
  "v_mfma_f32_16x16x32_bf16 v[160:163], a[192:195], v[104:107], v[160:163]\n\t" \
  "ds_read_b128 v[104:107], v10 offset:18432\n\t" \
  "ds_read_b128 v[108:111], v11 offset:18432\n\t" \
  "v_mfma_f32_16x16x32_bf16 v[192:195], a[192:195], v[112:115], v[192:195]\n\t" \
  "ds_read_b64_tr_b16 a[184:185], v15 offset:16384\n\t" \
  "v_mfma_f32_16x16x32_bf16 v[192:195], a[196:199], v[116:119], v[192:195]\n\t" \
  "v_mfma_f32_16x16x32_bf16 v[196:199], a[204:207], v[116:119], v[196:199]\n\t" \
  "ds_read_b64_tr_b16 a[186:187], v15 offset:17408\n\t" \
  "v_mfma_f32_16x16x32_bf16 v[196:199], a[200:203], v[112:115], v[196:199]\n\t" \
  "v_mfma_f32_16x16x32_bf16 v[200:203], a[208:211], v[112:115], v[200:203]\n\t" \
  "ds_read_b64_tr_b16 a[188:189], v15 offset:24576\n\t" \
  "v_mfma_f32_16x16x32_bf16 v[200:203], a[212:215], v[116:119], v[200:203]\n\t" \
  "v_mfma_f32_16x16x32_bf16 v[204:207], a[220:223], v[116:119], v[204:207]\n\t" \
  "ds_read_b64_tr_b16 a[190:191], v15 offset:25600\n\t" \
  "v_mfma_f32_16x16x32_bf16 v[204:207], a[216:219], v[112:115], v[204:207]\n\t" \
  "s_add_u32 m0, s68, 65536\n\t" \
  "v_mfma_f32_16x16x32_bf16 v[208:211], a[224:227], v[112:115], v[208:211]\n\t" \
  "buffer_load_dwordx4 v4, s[40:43], s78 offen offset:0 lds\n\t" \
  "v_mfma_f32_16x16x32_bf16 v[208:211], a[228:231], v[116:119], v[208:211]\n\t" \
  "v_mfma_f32_16x16x32_bf16 v[212:215], a[236:239], v[116:119], v[212:215]\n\t" \
  "v_mfma_f32_16x16x32_bf16 v[212:215], a[232:235], v[112:115], v[212:215]\n\t" \
  "v_mfma_f32_16x16x32_bf16 v[216:219], a[240:243], v[112:115], v[216:219]\n\t" \
  "v_mfma_f32_16x16x32_bf16 v[216:219], a[244:247], v[116:119], v[216:219]\n\t" \
  "v_mfma_f32_16x16x32_bf16 v[220:223], a[252:255], v[116:119], v[220:223]\n\t" \
  "v_mfma_f32_16x16x32_bf16 v[220:223], a[248:251], v[112:115], v[220:223]\n\t" \
  "ds_read_b128 v[112:115], v10 offset:20480\n\t" \
  "ds_read_b128 v[116:119], v11 offset:20480\n\t" \
  "buffer_load_dwordx4 v5, s[40:43], s78 offen offset:1024 lds\n\t" \
  "v_mfma_f32_16x16x32_bf16 v[252:255], a[248:251], v[120:123], v[252:255]\n\t" \
  "v_mfma_f32_16x16x32_bf16 v[252:255], a[252:255], v[124:127], v[252:255]\n\t" \
  "v_mfma_f32_16x16x32_bf16 v[248:251], a[244:247], v[124:127], v[248:251]\n\t" \
  "v_mfma_f32_16x16x32_bf16 v[248:251], a[240:243], v[120:123], v[248:251]\n\t" \
  "v_mfma_f32_16x16x32_bf16 v[244:247], a[232:235], v[120:123], v[244:247]\n\t" \
  "v_cmp_le_u32 s[92:93], s86, v37\n\t" \
  "v_mfma_f32_16x16x32_bf16 v[244:247], a[236:239], v[124:127], v[244:247]\n\t" \
  "v_mfma_f32_16x16x32_bf16 v[240:243], a[228:231], v[124:127], v[240:243]\n\t" \
  "buffer_load_dwordx4 v6, s[40:43], s78 offen offset:2048 lds\n\t" \
  "v_mfma_f32_16x16x32_bf16 v[240:243], a[224:227], v[120:123], v[240:243]\n\t" \
  "s_or_b64 s[58:59], s[58:59], s[92:93]\n\t" \
  "v_mfma_f32_16x16x32_bf16 v[236:239], a[216:219], v[120:123], v[236:239]\n\t" \
  "v_mfma_f32_16x16x32_bf16 v[236:239], a[220:223], v[124:127], v[236:239]\n\t" \
  "s_bitcmp1_b64 s[58:59], 0\n\t" \
  "s_cbranch_scc0 SLOW134_%=\n\t" \
  "RET133_%=:\n\t" \
  "v_mfma_f32_16x16x32_bf16 v[232:235], a[212:215], v[124:127], v[232:235]\n\t" \
  "v_mfma_f32_16x16x32_bf16 v[232:235], a[208:211], v[120:123], v[232:235]\n\t" \
  "v_mfma_f32_16x16x32_bf16 v[228:231], a[200:203], v[120:123], v[228:231]\n\t" \
  "v_mfma_f32_16x16x32_bf16 v[228:231], a[204:207], v[124:127], v[228:231]\n\t" \
  "v_mfma_f32_16x16x32_bf16 v[224:227], a[196:199], v[124:127], v[224:227]\n\t" \
  "buffer_load_dwordx4 v7, s[40:43], s78 offen offset:3072 lds\n\t" \
  "v_mfma_f32_16x16x32_bf16 v[224:227], a[192:195], v[120:123], v[224:227]\n\t" \
  "ds_read_b128 v[120:123], v10 offset:22528\n\t" \
  "ds_read_b128 v[124:127], v11 offset:22528\n\t" \
  "v_mfma_f32_16x16x32_bf16 a[0:3], a[128:131], v[64:67], a[0:3]\n\t" \
  "v_mfma_f32_16x16x32_bf16 a[0:3], a[132:135], v[68:71], a[0:3]\n\t" \
  "v_mfma_f32_16x16x32_bf16 a[4:7], a[140:143], v[68:71], a[4:7]\n\t" \
  "v_mfma_f32_16x16x32_bf16 a[4:7], a[136:139], v[64:67], a[4:7]\n\t" \
  "v_mfma_f32_16x16x32_bf16 a[8:11], a[144:147], v[64:67], a[8:11]\n\t" \
  "v_mfma_f32_16x16x32_bf16 a[8:11], a[148:151], v[68:71], a[8:11]\n\t" \
  "v_mfma_f32_16x16x32_bf16 a[12:15], a[156:159], v[68:71], a[12:15]\n\t" \
  "v_mfma_f32_16x16x32_bf16 a[12:15], a[152:155], v[64:67], a[12:15]\n\t" \
  "s_waitcnt vmcnt(13) lgkmcnt(0)\n\t" \
  "s_barrier\n\t" \
  "v_mfma_f32_16x16x32_bf16 a[16:19], a[160:163], v[64:67], a[16:19]\n\t" \
  "ds_read_b64_tr_b16 a[192:193], v12 offset:32768\n\t" \
  "v_mfma_f32_16x16x32_bf16 a[16:19], a[164:167], v[68:71], a[16:19]\n\t" \
  "s_andn2_b64 s[92:93], 255, s[58:59]\n\t" \
  "s_cbranch_scc1 POLL138_%=\n\t" \
  "RET137_%=:\n\t" \
  "v_mfma_f32_16x16x32_bf16 a[20:23], a[172:175], v[68:71], a[20:23]\n\t" \
  "ds_read_b64_tr_b16 a[194:195], v12 offset:33792\n\t" \
  "v_mfma_f32_16x16x32_bf16 a[20:23], a[168:171], v[64:67], a[20:23]\n\t" \
  "v_mfma_f32_16x16x32_bf16 a[24:27], a[176:179], v[64:67], a[24:27]\n\t" \
  "ds_read_b64_tr_b16 a[196:197], v12 offset:40960\n\t" \
  "v_mfma_f32_16x16x32_bf16 a[24:27], a[180:183], v[68:71], a[24:27]\n\t" \
  "v_mfma_f32_16x16x32_bf16 a[28:31], a[188:191], v[68:71], a[28:31]\n\t" \
  "ds_read_b64_tr_b16 a[198:199], v12 offset:41984\n\t" \
  "v_mfma_f32_16x16x32_bf16 a[28:31], a[184:187], v[64:67], a[28:31]\n\t" \
  "ds_read_b128 v[64:67], v10 offset:32768\n\t" \
  "ds_read_b128 v[68:71], v11 offset:32768\n\t" \
  "v_mfma_f32_16x16x32_bf16 a[60:63], a[184:187], v[72:75], a[60:63]\n\t" \
  "ds_read_b64_tr_b16 a[200:201], v13 offset:32768\n\t" \
  "v_mfma_f32_16x16x32_bf16 a[60:63], a[188:191], v[76:79], a[60:63]\n\t" \
  "v_mfma_f32_16x16x32_bf16 a[56:59], a[180:183], v[76:79], a[56:59]\n\t" \
  "ds_read_b64_tr_b16 a[202:203], v13 offset:33792\n\t" \
  "v_mfma_f32_16x16x32_bf16 a[56:59], a[176:179], v[72:75], a[56:59]\n\t" \
  "v_mfma_f32_16x16x32_bf16 a[52:55], a[168:171], v[72:75], a[52:55]\n\t" \
  "ds_read_b64_tr_b16 a[204:205], v13 offset:40960\n\t" \
  "v_mfma_f32_16x16x32_bf16 a[52:55], a[172:175], v[76:79], a[52:55]\n\t" \
  "v_mfma_f32_16x16x32_bf16 a[48:51], a[164:167], v[76:79], a[48:51]\n\t" \
  "ds_read_b64_tr_b16 a[206:207], v13 offset:41984\n\t" \
  "v_mfma_f32_16x16x32_bf16 a[48:51], a[160:163], v[72:75], a[48:51]\n\t" \
  "v_mfma_f32_16x16x32_bf16 a[44:47], a[152:155], v[72:75], a[44:47]\n\t" \
  "ds_read_b64_tr_b16 a[208:209], v14 offset:32768\n\t" \
  "v_mfma_f32_16x16x32_bf16 a[44:47], a[156:159], v[76:79], a[44:47]\n\t" \
  "v_mfma_f32_16x16x32_bf16 a[40:43], a[148:151], v[76:79], a[40:43]\n\t" \
  "ds_read_b64_tr_b16 a[210:211], v14 offset:33792\n\t" \
  "v_mfma_f32_16x16x32_bf16 a[40:43], a[144:147], v[72:75], a[40:43]\n\t" \
  "v_mfma_f32_16x16x32_bf16 a[36:39], a[136:139], v[72:75], a[36:39]\n\t" \
  "ds_read_b64_tr_b16 a[212:213], v14 offset:40960\n\t" \
  "v_mfma_f32_16x16x32_bf16 a[36:39], a[140:143], v[76:79], a[36:39]\n\t" \
  "v_mfma_f32_16x16x32_bf16 a[32:35], a[132:135], v[76:79], a[32:35]\n\t" \
  "ds_read_b64_tr_b16 a[214:215], v14 offset:41984\n\t" \
  "v_mfma_f32_16x16x32_bf16 a[32:35], a[128:131], v[72:75], a[32:35]\n\t" \
  "ds_read_b128 v[72:75], v10 offset:34816\n\t" \
  "ds_read_b128 v[76:79], v11 offset:34816\n\t" \
  "v_mfma_f32_16x16x32_bf16 a[64:67], a[128:131], v[80:83], a[64:67]\n\t" \
  "ds_read_b64_tr_b16 a[216:217], v15 offset:32768\n\t" \
  "v_mfma_f32_16x16x32_bf16 a[64:67], a[132:135], v[84:87], a[64:67]\n\t" \
  "v_mfma_f32_16x16x32_bf16 a[68:71], a[140:143], v[84:87], a[68:71]\n\t" \
  "ds_read_b64_tr_b16 a[218:219], v15 offset:33792\n\t" \
  "v_mfma_f32_16x16x32_bf16 a[68:71], a[136:139], v[80:83], a[68:71]\n\t" \
  "v_mfma_f32_16x16x32_bf16 a[72:75], a[144:147], v[80:83], a[72:75]\n\t" \
  "ds_read_b64_tr_b16 a[220:221], v15 offset:40960\n\t" \
  "v_mfma_f32_16x16x32_bf16 a[72:75], a[148:151], v[84:87], a[72:75]\n\t" \
  "v_mfma_f32_16x16x32_bf16 a[76:79], a[156:159], v[84:87], a[76:79]\n\t" \
  "ds_read_b64_tr_b16 a[222:223], v15 offset:41984\n\t" \
  "v_mfma_f32_16x16x32_bf16 a[76:79], a[152:155], v[80:83], a[76:79]\n\t" \
  "s_add_u32 m0, s68, 16384\n\t" \
  "v_mfma_f32_16x16x32_bf16 a[80:83], a[160:163], v[80:83], a[80:83]\n\t" \
  "buffer_load_dwordx4 v0, s[36:39], s65 offen offset:0 sc0 sc1 lds\n\t" \
  "v_mfma_f32_16x16x32_bf16 a[80:83], a[164:167], v[84:87], a[80:83]\n\t" \
  "v_mfma_f32_16x16x32_bf16 a[84:87], a[172:175], v[84:87], a[84:87]\n\t" \
  "v_mfma_f32_16x16x32_bf16 a[84:87], a[168:171], v[80:83], a[84:87]\n\t" \
  "buffer_load_dwordx4 v1, s[36:39], s65 offen offset:1024 sc0 sc1 lds\n\t" \
  "v_mfma_f32_16x16x32_bf16 a[88:91], a[176:179], v[80:83], a[88:91]\n\t" \
  "v_mfma_f32_16x16x32_bf16 a[88:91], a[180:183], v[84:87], a[88:91]\n\t" \
  "v_mfma_f32_16x16x32_bf16 a[92:95], a[188:191], v[84:87], a[92:95]\n\t" \
  "buffer_load_dwordx4 v2, s[36:39], s65 offen offset:2048 sc0 sc1 lds\n\t" \
  "v_mfma_f32_16x16x32_bf16 a[92:95], a[184:187], v[80:83], a[92:95]\n\t" \
  "ds_read_b128 v[80:83], v10 offset:36864\n\t" \
  "ds_read_b128 v[84:87], v11 offset:36864\n\t" \
  "v_mfma_f32_16x16x32_bf16 a[124:127], a[184:187], v[88:91], a[124:127]\n\t" \
  "v_mfma_f32_16x16x32_bf16 a[124:127], a[188:191], v[92:95], a[124:127]\n\t" \
  "buffer_load_dwordx4 v3, s[36:39], s65 offen offset:3072 sc0 sc1 lds\n\t" \
  "v_mfma_f32_16x16x32_bf16 a[120:123], a[180:183], v[92:95], a[120:123]\n\t" \
  "v_mfma_f32_16x16x32_bf16 a[120:123], a[176:179], v[88:91], a[120:123]\n\t" \
  "s_add_u32 m0, s68, 81920\n\t" \
  "v_mfma_f32_16x16x32_bf16 a[116:119], a[168:171], v[88:91], a[116:119]\n\t" \
  "buffer_load_dwordx4 v4, s[40:43], s79 offen offset:0 lds\n\t" \
  "v_cmp_le_u32 s[92:93], s86, v37\n\t" \
  "v_mfma_f32_16x16x32_bf16 a[116:119], a[172:175], v[92:95], a[116:119]\n\t" \
  "v_mfma_f32_16x16x32_bf16 a[112:115], a[164:167], v[92:95], a[112:115]\n\t" \
  "v_mfma_f32_16x16x32_bf16 a[112:115], a[160:163], v[88:91], a[112:115]\n\t" \
  "buffer_load_dwordx4 v5, s[40:43], s79 offen offset:1024 lds\n\t" \
  "s_or_b64 s[58:59], s[58:59], s[92:93]\n\t" \
  "v_mfma_f32_16x16x32_bf16 a[108:111], a[152:155], v[88:91], a[108:111]\n\t" \
  "v_mfma_f32_16x16x32_bf16 a[108:111], a[156:159], v[92:95], a[108:111]\n\t" \
  "s_bitcmp1_b64 s[58:59], 1\n\t" \
  "s_cbranch_scc0 SLOW140_%=\n\t" \
  "RET139_%=:\n\t" \
  "v_mfma_f32_16x16x32_bf16 a[104:107], a[148:151], v[92:95], a[104:107]\n\t" \
  "buffer_load_dwordx4 v6, s[40:43], s79 offen offset:2048 lds\n\t" \
  "v_mfma_f32_16x16x32_bf16 a[104:107], a[144:147], v[88:91], a[104:107]\n\t" \
  "v_mfma_f32_16x16x32_bf16 a[100:103], a[136:139], v[88:91], a[100:103]\n\t" \
  "v_mfma_f32_16x16x32_bf16 a[100:103], a[140:143], v[92:95], a[100:103]\n\t" \
  "v_mfma_f32_16x16x32_bf16 a[96:99], a[132:135], v[92:95], a[96:99]\n\t" \
  "buffer_load_dwordx4 v7, s[40:43], s79 offen offset:3072 lds\n\t" \
  "v_mfma_f32_16x16x32_bf16 a[96:99], a[128:131], v[88:91], a[96:99]\n\t" \
  "ds_read_b128 v[88:91], v10 offset:38912\n\t" \
  "ds_read_b128 v[92:95], v11 offset:38912\n\t" \
  "v_mfma_f32_16x16x32_bf16 v[128:131], a[128:131], v[96:99], v[128:131]\n\t" \
  "v_mfma_f32_16x16x32_bf16 v[128:131], a[132:135], v[100:103], v[128:131]\n\t" \
  "v_mfma_f32_16x16x32_bf16 v[132:135], a[140:143], v[100:103], v[132:135]\n\t" \
  "v_mfma_f32_16x16x32_bf16 v[132:135], a[136:139], v[96:99], v[132:135]\n\t" \
  "v_mfma_f32_16x16x32_bf16 v[136:139], a[144:147], v[96:99], v[136:139]\n\t" \
  "v_mfma_f32_16x16x32_bf16 v[136:139], a[148:151], v[100:103], v[136:139]\n\t" \
  "v_mfma_f32_16x16x32_bf16 v[140:143], a[156:159], v[100:103], v[140:143]\n\t" \
  "v_mfma_f32_16x16x32_bf16 v[140:143], a[152:155], v[96:99], v[140:143]\n\t" \
  "s_waitcnt vmcnt(12) lgkmcnt(0)\n\t" \
  "s_barrier\n\t" \
  "v_mfma_f32_16x16x32_bf16 v[144:147], a[160:163], v[96:99], v[144:147]\n\t" \
  "ds_read_b64_tr_b16 a[224:225], v12 offset:49152\n\t" \
  "v_readlane_b32 s66, v17, 1\n\t" \
  "s_nop 0\n\t" \
  "s_lshl_b32 s78, s66, 13\n\t" \
  "s_lshl_b32 s65, s66, 10\n\t" \
  "s_add_u32 s79, s78, 256\n\t" \
  "s_add_u32 s65, s65, s70\n\t" \
  "s_nop 1\n\t" \
  "v_mfma_f32_16x16x32_bf16 v[144:147], a[164:167], v[100:103], v[144:147]\n\t" \
  "v_mfma_f32_16x16x32_bf16 v[148:151], a[172:175], v[100:103], v[148:151]\n\t" \
  "ds_read_b64_tr_b16 a[226:227], v12 offset:50176\n\t" \
  "v_mfma_f32_16x16x32_bf16 v[148:151], a[168:171], v[96:99], v[148:151]\n\t" \
  "s_add_u32 m0, s68, 131072\n\t" \
  "v_mfma_f32_16x16x32_bf16 v[152:155], a[176:179], v[96:99], v[152:155]\n\t" \
  "ds_read_b64_tr_b16 a[228:229], v12 offset:57344\n\t" \
  "buffer_load_dwordx4 v4, s[40:43], s79 offen offset:0 lds\n\t" \
  "v_mfma_f32_16x16x32_bf16 v[152:155], a[180:183], v[100:103], v[152:155]\n\t" \
  "v_mfma_f32_16x16x32_bf16 v[156:159], a[188:191], v[100:103], v[156:159]\n\t" \
  "ds_read_b64_tr_b16 a[230:231], v12 offset:58368\n\t" \
  "v_mfma_f32_16x16x32_bf16 v[156:159], a[184:187], v[96:99], v[156:159]\n\t" \
  "ds_read_b128 v[96:99], v10 offset:49152\n\t" \
  "ds_read_b128 v[100:103], v11 offset:49152\n\t" \
  "v_mfma_f32_16x16x32_bf16 v[188:191], a[184:187], v[104:107], v[188:191]\n\t" \
  "ds_read_b64_tr_b16 a[232:233], v13 offset:49152\n\t" \
  "buffer_load_dwordx4 v5, s[40:43], s79 offen offset:1024 lds\n\t" \
  "v_mfma_f32_16x16x32_bf16 v[188:191], a[188:191], v[108:111], v[188:191]\n\t" \
  "v_mfma_f32_16x16x32_bf16 v[184:187], a[180:183], v[108:111], v[184:187]\n\t" \
  "ds_read_b64_tr_b16 a[234:235], v13 offset:50176\n\t" \
  "v_mfma_f32_16x16x32_bf16 v[184:187], a[176:179], v[104:107], v[184:187]\n\t" \
  "v_mfma_f32_16x16x32_bf16 v[180:183], a[168:171], v[104:107], v[180:183]\n\t" \
  "ds_read_b64_tr_b16 a[236:237], v13 offset:57344\n\t" \
  "v_mfma_f32_16x16x32_bf16 v[180:183], a[172:175], v[108:111], v[180:183]\n\t" \
  "buffer_load_dwordx4 v6, s[40:43], s79 offen offset:2048 lds\n\t" \
  "v_mfma_f32_16x16x32_bf16 v[176:179], a[164:167], v[108:111], v[176:179]\n\t" \
  "ds_read_b64_tr_b16 a[238:239], v13 offset:58368\n\t" \
  "v_mfma_f32_16x16x32_bf16 v[176:179], a[160:163], v[104:107], v[176:179]\n\t" \
  "v_mfma_f32_16x16x32_bf16 v[172:175], a[152:155], v[104:107], v[172:175]\n\t" \
  "ds_read_b64_tr_b16 a[240:241], v14 offset:49152\n\t" \
  "v_mfma_f32_16x16x32_bf16 v[172:175], a[156:159], v[108:111], v[172:175]\n\t" \
  "buffer_load_dwordx4 v7, s[40:43], s79 offen offset:3072 lds\n\t" \
  "v_mfma_f32_16x16x32_bf16 v[168:171], a[148:151], v[108:111], v[168:171]\n\t" \
  "ds_read_b64_tr_b16 a[242:243], v14 offset:50176\n\t" \
  "v_mfma_f32_16x16x32_bf16 v[168:171], a[144:147], v[104:107], v[168:171]\n\t" \
  "v_mfma_f32_16x16x32_bf16 v[164:167], a[136:139], v[104:107], v[164:167]\n\t" \
  "ds_read_b64_tr_b16 a[244:245], v14 offset:57344\n\t" \
  "v_mfma_f32_16x16x32_bf16 v[164:167], a[140:143], v[108:111], v[164:167]\n\t" \
  "s_add_u32 m0, s68, 0\n\t" \
  "v_mfma_f32_16x16x32_bf16 v[160:163], a[132:135], v[108:111], v[160:163]\n\t" \
  "ds_read_b64_tr_b16 a[246:247], v14 offset:58368\n\t" \
  "buffer_load_dwordx4 v0, s[36:39], s65 offen offset:0 sc0 sc1 lds\n\t" \
  "v_mfma_f32_16x16x32_bf16 v[160:163], a[128:131], v[104:107], v[160:163]\n\t" \
  "ds_read_b128 v[104:107], v10 offset:51200\n\t" \
  "ds_read_b128 v[108:111], v11 offset:51200\n\t" \
  "v_mfma_f32_16x16x32_bf16 v[192:195], a[128:131], v[112:115], v[192:195]\n\t" \
  "ds_read_b64_tr_b16 a[248:249], v15 offset:49152\n\t" \
  "v_mfma_f32_16x16x32_bf16 v[192:195], a[132:135], v[116:119], v[192:195]\n\t" \
  "v_mfma_f32_16x16x32_bf16 v[196:199], a[140:143], v[116:119], v[196:199]\n\t" \
  "ds_read_b64_tr_b16 a[250:251], v15 offset:50176\n\t" \
  "buffer_load_dwordx4 v1, s[36:39], s65 offen offset:1024 sc0 sc1 lds\n\t" \
  "v_mfma_f32_16x16x32_bf16 v[196:199], a[136:139], v[112:115], v[196:199]\n\t" \
  "v_mfma_f32_16x16x32_bf16 v[200:203], a[144:147], v[112:115], v[200:203]\n\t" \
  "ds_read_b64_tr_b16 a[252:253], v15 offset:57344\n\t" \
  "v_mfma_f32_16x16x32_bf16 v[200:203], a[148:151], v[116:119], v[200:203]\n\t" \
  "v_mfma_f32_16x16x32_bf16 v[204:207], a[156:159], v[116:119], v[204:207]\n\t" \
  "ds_read_b64_tr_b16 a[254:255], v15 offset:58368\n\t" \
  "v_mfma_f32_16x16x32_bf16 v[204:207], a[152:155], v[112:115], v[204:207]\n\t" \
  "buffer_load_dwordx4 v2, s[36:39], s65 offen offset:2048 sc0 sc1 lds\n\t" \
  "v_mfma_f32_16x16x32_bf16 v[208:211], a[160:163], v[112:115], v[208:211]\n\t" \
  "v_mfma_f32_16x16x32_bf16 v[208:211], a[164:167], v[116:119], v[208:211]\n\t" \
  "v_mfma_f32_16x16x32_bf16 v[212:215], a[172:175], v[116:119], v[212:215]\n\t" \
  "v_mfma_f32_16x16x32_bf16 v[212:215], a[168:171], v[112:115], v[212:215]\n\t" \
  "buffer_load_dwordx4 v3, s[36:39], s65 offen offset:3072 sc0 sc1 lds\n\t" \
  "v_mfma_f32_16x16x32_bf16 v[216:219], a[176:179], v[112:115], v[216:219]\n\t" \
  "v_mfma_f32_16x16x32_bf16 v[216:219], a[180:183], v[116:119], v[216:219]\n\t" \
  "v_mfma_f32_16x16x32_bf16 v[220:223], a[188:191], v[116:119], v[220:223]\n\t" \
  "v_mfma_f32_16x16x32_bf16 v[220:223], a[184:187], v[112:115], v[220:223]\n\t" \
  "ds_read_b128 v[112:115], v10 offset:53248\n\t" \
  "ds_read_b128 v[116:119], v11 offset:53248\n\t" \
  "s_add_u32 m0, s68, 98304\n\t" \
  "v_mfma_f32_16x16x32_bf16 v[252:255], a[184:187], v[120:123], v[252:255]\n\t" \
  "buffer_load_dwordx4 v4, s[40:43], s78 offen offset:0 lds\n\t" \
  "v_mfma_f32_16x16x32_bf16 v[252:255], a[188:191], v[124:127], v[252:255]\n\t" \
  "v_mfma_f32_16x16x32_bf16 v[248:251], a[180:183], v[124:127], v[248:251]\n\t" \
  "v_mfma_f32_16x16x32_bf16 v[248:251], a[176:179], v[120:123], v[248:251]\n\t" \
  "v_mfma_f32_16x16x32_bf16 v[244:247], a[168:171], v[120:123], v[244:247]\n\t" \
  "buffer_load_dwordx4 v5, s[40:43], s78 offen offset:1024 lds\n\t" \
  "v_cmp_le_u32 s[92:93], s86, v37\n\t" \
  "v_mfma_f32_16x16x32_bf16 v[244:247], a[172:175], v[124:127], v[244:247]\n\t" \
  "v_mfma_f32_16x16x32_bf16 v[240:243], a[164:167], v[124:127], v[240:243]\n\t" \
  "v_mfma_f32_16x16x32_bf16 v[240:243], a[160:163], v[120:123], v[240:243]\n\t" \
  "s_or_b64 s[58:59], s[58:59], s[92:93]\n\t" \
  "v_mfma_f32_16x16x32_bf16 v[236:239], a[152:155], v[120:123], v[236:239]\n\t" \
  "v_mfma_f32_16x16x32_bf16 v[236:239], a[156:159], v[124:127], v[236:239]\n\t" \
  "buffer_load_dwordx4 v6, s[40:43], s78 offen offset:2048 lds\n\t" \
  "s_bitcmp1_b64 s[58:59], 2\n\t" \
  "s_cbranch_scc0 SLOW144_%=\n\t" \
  "RET143_%=:\n\t" \
  "v_mfma_f32_16x16x32_bf16 v[232:235], a[148:151], v[124:127], v[232:235]\n\t" \
  "v_mfma_f32_16x16x32_bf16 v[232:235], a[144:147], v[120:123], v[232:235]\n\t" \
  "v_mfma_f32_16x16x32_bf16 v[228:231], a[136:139], v[120:123], v[228:231]\n\t" \
  "v_mfma_f32_16x16x32_bf16 v[228:231], a[140:143], v[124:127], v[228:231]\n\t" \
  "v_mfma_f32_16x16x32_bf16 v[224:227], a[132:135], v[124:127], v[224:227]\n\t" \
  "buffer_load_dwordx4 v7, s[40:43], s78 offen offset:3072 lds\n\t" \
  "v_mfma_f32_16x16x32_bf16 v[224:227], a[128:131], v[120:123], v[224:227]\n\t" \
  "ds_read_b128 v[120:123], v10 offset:55296\n\t" \
  "ds_read_b128 v[124:127], v11 offset:55296\n\t" \
  "v_mfma_f32_16x16x32_bf16 a[0:3], a[192:195], v[64:67], a[0:3]\n\t" \
  "v_mfma_f32_16x16x32_bf16 a[0:3], a[196:199], v[68:71], a[0:3]\n\t" \
  "v_mfma_f32_16x16x32_bf16 a[4:7], a[204:207], v[68:71], a[4:7]\n\t" \
  "v_mfma_f32_16x16x32_bf16 a[4:7], a[200:203], v[64:67], a[4:7]\n\t" \
  "v_mfma_f32_16x16x32_bf16 a[8:11], a[208:211], v[64:67], a[8:11]\n\t" \
  "v_mfma_f32_16x16x32_bf16 a[8:11], a[212:215], v[68:71], a[8:11]\n\t" \
  "v_mfma_f32_16x16x32_bf16 a[12:15], a[220:223], v[68:71], a[12:15]\n\t" \
  "v_mfma_f32_16x16x32_bf16 a[12:15], a[216:219], v[64:67], a[12:15]\n\t" \
  "s_waitcnt vmcnt(20) lgkmcnt(0)\n\t" \
  "s_barrier\n\t" \
  "v_mfma_f32_16x16x32_bf16 a[16:19], a[224:227], v[64:67], a[16:19]\n\t" \
  "ds_read_b64_tr_b16 a[128:129], v12 offset:0\n\t" \
  "v_readlane_b32 s66, v17, 2\n\t" \
  "s_nop 0\n\t" \
  "s_lshl_b32 s78, s66, 13\n\t" \
  "s_lshl_b32 s65, s66, 10\n\t" \
  "s_add_u32 s79, s78, 256\n\t" \
  "s_add_u32 s65, s65, s70\n\t" \
  "s_nop 1\n\t" \
  "v_mfma_f32_16x16x32_bf16 a[16:19], a[228:231], v[68:71], a[16:19]\n\t" \
  "s_andn2_b64 s[92:93], 255, s[58:59]\n\t" \
  "s_cbranch_scc1 POLL148_%=\n\t" \
  "RET147_%=:\n\t" \
  "v_mfma_f32_16x16x32_bf16 a[20:23], a[236:239], v[68:71], a[20:23]\n\t" \
  "ds_read_b64_tr_b16 a[130:131], v12 offset:1024\n\t" \
  "v_mfma_f32_16x16x32_bf16 a[20:23], a[232:235], v[64:67], a[20:23]\n\t" \
  "s_add_u32 m0, s68, 147456\n\t" \
  "v_mfma_f32_16x16x32_bf16 a[24:27], a[240:243], v[64:67], a[24:27]\n\t" \
  "ds_read_b64_tr_b16 a[132:133], v12 offset:8192\n\t" \
  "buffer_load_dwordx4 v4, s[40:43], s79 offen offset:0 lds\n\t" \
  "v_mfma_f32_16x16x32_bf16 a[24:27], a[244:247], v[68:71], a[24:27]\n\t" \
  "v_mfma_f32_16x16x32_bf16 a[28:31], a[252:255], v[68:71], a[28:31]\n\t" \
  "ds_read_b64_tr_b16 a[134:135], v12 offset:9216\n\t" \
  "v_mfma_f32_16x16x32_bf16 a[28:31], a[248:251], v[64:67], a[28:31]\n\t" \
  "v_mfma_f32_16x16x32_bf16 a[60:63], a[248:251], v[72:75], a[60:63]\n\t" \
  "ds_read_b64_tr_b16 a[136:137], v13 offset:0\n\t" \
  "buffer_load_dwordx4 v5, s[40:43], s79 offen offset:1024 lds\n\t" \
  "v_mfma_f32_16x16x32_bf16 a[60:63], a[252:255], v[76:79], a[60:63]\n\t" \
  "v_mfma_f32_16x16x32_bf16 a[56:59], a[244:247], v[76:79], a[56:59]\n\t" \
  "ds_read_b64_tr_b16 a[138:139], v13 offset:1024\n\t" \
  "v_mfma_f32_16x16x32_bf16 a[56:59], a[240:243], v[72:75], a[56:59]\n\t" \
  "v_mfma_f32_16x16x32_bf16 a[52:55], a[232:235], v[72:75], a[52:55]\n\t" \
  "ds_read_b64_tr_b16 a[140:141], v13 offset:8192\n\t" \
  "v_mfma_f32_16x16x32_bf16 a[52:55], a[236:239], v[76:79], a[52:55]\n\t" \
  "buffer_load_dwordx4 v6, s[40:43], s79 offen offset:2048 lds\n\t" \
  "v_mfma_f32_16x16x32_bf16 a[48:51], a[228:231], v[76:79], a[48:51]\n\t" \
  "ds_read_b64_tr_b16 a[142:143], v13 offset:9216\n\t" \
  "v_mfma_f32_16x16x32_bf16 a[48:51], a[224:227], v[72:75], a[48:51]\n\t" \
  "v_mfma_f32_16x16x32_bf16 a[44:47], a[216:219], v[72:75], a[44:47]\n\t" \
  "ds_read_b64_tr_b16 a[144:145], v14 offset:0\n\t" \
  "v_mfma_f32_16x16x32_bf16 a[44:47], a[220:223], v[76:79], a[44:47]\n\t" \
  "buffer_load_dwordx4 v7, s[40:43], s79 offen offset:3072 lds\n\t" \
  "v_mfma_f32_16x16x32_bf16 a[40:43], a[212:215], v[76:79], a[40:43]\n\t" \
  "ds_read_b64_tr_b16 a[146:147], v14 offset:1024\n\t" \
  "v_mfma_f32_16x16x32_bf16 a[40:43], a[208:211], v[72:75], a[40:43]\n\t" \
  "v_mfma_f32_16x16x32_bf16 a[36:39], a[200:203], v[72:75], a[36:39]\n\t" \
  "ds_read_b64_tr_b16 a[148:149], v14 offset:8192\n\t" \
  "v_mfma_f32_16x16x32_bf16 a[36:39], a[204:207], v[76:79], a[36:39]\n\t" \
  "s_add_u32 m0, s68, 32768\n\t" \
  "v_mfma_f32_16x16x32_bf16 a[32:35], a[196:199], v[76:79], a[32:35]\n\t" \
  "ds_read_b64_tr_b16 a[150:151], v14 offset:9216\n\t" \
  "buffer_load_dwordx4 v0, s[36:39], s65 offen offset:0 sc0 sc1 lds\n\t" \
  "v_mfma_f32_16x16x32_bf16 a[32:35], a[192:195], v[72:75], a[32:35]\n\t" \
  "v_mfma_f32_16x16x32_bf16 a[64:67], a[192:195], v[80:83], a[64:67]\n\t" \
  "ds_read_b64_tr_b16 a[152:153], v15 offset:0\n\t" \
  "v_mfma_f32_16x16x32_bf16 a[64:67], a[196:199], v[84:87], a[64:67]\n\t" \
  "v_mfma_f32_16x16x32_bf16 a[68:71], a[204:207], v[84:87], a[68:71]\n\t" \
  "ds_read_b64_tr_b16 a[154:155], v15 offset:1024\n\t" \
  "buffer_load_dwordx4 v1, s[36:39], s65 offen offset:1024 sc0 sc1 lds\n\t" \
  "v_mfma_f32_16x16x32_bf16 a[68:71], a[200:203], v[80:83], a[68:71]\n\t" \
  "v_mfma_f32_16x16x32_bf16 a[72:75], a[208:211], v[80:83], a[72:75]\n\t" \
  "ds_read_b64_tr_b16 a[156:157], v15 offset:8192\n\t" \
  "v_mfma_f32_16x16x32_bf16 a[72:75], a[212:215], v[84:87], a[72:75]\n\t" \
  "v_mfma_f32_16x16x32_bf16 a[76:79], a[220:223], v[84:87], a[76:79]\n\t" \
  "ds_read_b64_tr_b16 a[158:159], v15 offset:9216\n\t" \
  "v_mfma_f32_16x16x32_bf16 a[76:79], a[216:219], v[80:83], a[76:79]\n\t" \
  "buffer_load_dwordx4 v2, s[36:39], s65 offen offset:2048 sc0 sc1 lds\n\t" \
  "v_mfma_f32_16x16x32_bf16 a[80:83], a[224:227], v[80:83], a[80:83]\n\t" \
  "v_mfma_f32_16x16x32_bf16 a[80:83], a[228:231], v[84:87], a[80:83]\n\t" \
  "v_mfma_f32_16x16x32_bf16 a[84:87], a[236:239], v[84:87], a[84:87]\n\t" \
  "v_mfma_f32_16x16x32_bf16 a[84:87], a[232:235], v[80:83], a[84:87]\n\t" \
  "buffer_load_dwordx4 v3, s[36:39], s65 offen offset:3072 sc0 sc1 lds\n\t" \
  "v_mfma_f32_16x16x32_bf16 a[88:91], a[240:243], v[80:83], a[88:91]\n\t" \
  "v_mfma_f32_16x16x32_bf16 a[88:91], a[244:247], v[84:87], a[88:91]\n\t" \
  "v_mfma_f32_16x16x32_bf16 a[92:95], a[252:255], v[84:87], a[92:95]\n\t" \
  "v_mfma_f32_16x16x32_bf16 a[92:95], a[248:251], v[80:83], a[92:95]\n\t" \
  "s_add_u32 m0, s68, 114688\n\t" \
  "v_mfma_f32_16x16x32_bf16 a[124:127], a[248:251], v[88:91], a[124:127]\n\t" \
  "buffer_load_dwordx4 v4, s[40:43], s78 offen offset:0 lds\n\t" \
  "v_mfma_f32_16x16x32_bf16 a[124:127], a[252:255], v[92:95], a[124:127]\n\t" \
  "v_mfma_f32_16x16x32_bf16 a[120:123], a[244:247], v[92:95], a[120:123]\n\t" \
  "v_mfma_f32_16x16x32_bf16 a[120:123], a[240:243], v[88:91], a[120:123]\n\t" \
  "v_mfma_f32_16x16x32_bf16 a[116:119], a[232:235], v[88:91], a[116:119]\n\t" \
  "buffer_load_dwordx4 v5, s[40:43], s78 offen offset:1024 lds\n\t" \
  "v_mfma_f32_16x16x32_bf16 a[116:119], a[236:239], v[92:95], a[116:119]\n\t" \
  "v_mfma_f32_16x16x32_bf16 a[112:115], a[228:231], v[92:95], a[112:115]\n\t" \
  "v_mfma_f32_16x16x32_bf16 a[112:115], a[224:227], v[88:91], a[112:115]\n\t" \
  "v_mfma_f32_16x16x32_bf16 a[108:111], a[216:219], v[88:91], a[108:111]\n\t" \
  "v_mfma_f32_16x16x32_bf16 a[108:111], a[220:223], v[92:95], a[108:111]\n\t" \
  "buffer_load_dwordx4 v6, s[40:43], s78 offen offset:2048 lds\n\t" \
  "v_mfma_f32_16x16x32_bf16 a[104:107], a[212:215], v[92:95], a[104:107]\n\t" \
  "v_mfma_f32_16x16x32_bf16 a[104:107], a[208:211], v[88:91], a[104:107]\n\t" \
  "v_mfma_f32_16x16x32_bf16 a[100:103], a[200:203], v[88:91], a[100:103]\n\t" \
  "v_mfma_f32_16x16x32_bf16 a[100:103], a[204:207], v[92:95], a[100:103]\n\t" \
  "v_mfma_f32_16x16x32_bf16 a[96:99], a[196:199], v[92:95], a[96:99]\n\t" \
  "buffer_load_dwordx4 v7, s[40:43], s78 offen offset:3072 lds\n\t" \
  "v_mfma_f32_16x16x32_bf16 a[96:99], a[192:195], v[88:91], a[96:99]\n\t" \
  "v_mfma_f32_16x16x32_bf16 v[128:131], a[192:195], v[96:99], v[128:131]\n\t" \
  "v_mfma_f32_16x16x32_bf16 v[128:131], a[196:199], v[100:103], v[128:131]\n\t" \
  "v_accvgpr_read_b32 v68, a0\n\t" \
  "v_accvgpr_read_b32 v69, a1\n\t" \
  "v_accvgpr_read_b32 v70, a2\n\t" \
  "v_mfma_f32_16x16x32_bf16 v[132:135], a[204:207], v[100:103], v[132:135]\n\t" \
  "v_accvgpr_read_b32 v71, a3\n\t" \
  "v_accvgpr_read_b32 v72, a4\n\t" \
  "v_accvgpr_read_b32 v73, a5\n\t" \
  "v_mfma_f32_16x16x32_bf16 v[132:135], a[200:203], v[96:99], v[132:135]\n\t" \
  "v_accvgpr_read_b32 v74, a6\n\t" \
  "v_accvgpr_read_b32 v75, a7\n\t" \
  "v_cvt_pk_bf16_f32 v64, v68, v69\n\t" \
  "v_mfma_f32_16x16x32_bf16 v[136:139], a[208:211], v[96:99], v[136:139]\n\t" \
  "v_cvt_pk_bf16_f32 v65, v70, v71\n\t" \
  "v_cvt_pk_bf16_f32 v66, v72, v73\n\t" \
  "v_cvt_pk_bf16_f32 v67, v74, v75\n\t" \
  "v_mfma_f32_16x16x32_bf16 v[136:139], a[212:215], v[100:103], v[136:139]\n\t" \
  "global_store_dwordx4 v9, v[64:67], s[50:51] offset:-4096 sc0 sc1\n\t" \
  "v_accvgpr_read_b32 v84, a8\n\t" \
  "v_accvgpr_read_b32 v85, a9\n\t" \
  "v_mfma_f32_16x16x32_bf16 v[140:143], a[220:223], v[100:103], v[140:143]\n\t" \
  "v_accvgpr_read_b32 v86, a10\n\t" \
  "v_accvgpr_read_b32 v87, a11\n\t" \
  "v_accvgpr_read_b32 v88, a12\n\t" \
  "v_mfma_f32_16x16x32_bf16 v[140:143], a[216:219], v[96:99], v[140:143]\n\t" \
  "v_accvgpr_read_b32 v89, a13\n\t" \
  "v_accvgpr_read_b32 v90, a14\n\t" \
  "v_accvgpr_read_b32 v91, a15\n\t" \
  "s_waitcnt vmcnt(25) lgkmcnt(0)\n\t" \
  "s_barrier\n\t" \
  "v_mfma_f32_16x16x32_bf16 v[144:147], a[224:227], v[96:99], v[144:147]\n\t" \
  "ds_read_b64_tr_b16 a[160:161], v12 offset:16384\n\t" \
  "v_readlane_b32 s66, v17, 3\n\t" \
  "s_nop 0\n\t" \
  "s_lshl_b32 s78, s66, 13\n\t" \
  "s_lshl_b32 s65, s66, 10\n\t" \
  "s_add_u32 s79, s78, 256\n\t" \
  "s_add_u32 s65, s65, s70\n\t" \
  "s_nop 1\n\t" \
  "v_mfma_f32_16x16x32_bf16 v[144:147], a[228:231], v[100:103], v[144:147]\n\t" \
  "v_cvt_pk_bf16_f32 v80, v84, v85\n\t" \
  "v_cvt_pk_bf16_f32 v81, v86, v87\n\t" \
  "v_cvt_pk_bf16_f32 v82, v88, v89\n\t" \
  "v_mfma_f32_16x16x32_bf16 v[148:151], a[236:239], v[100:103], v[148:151]\n\t" \
  "ds_read_b64_tr_b16 a[162:163], v12 offset:17408\n\t" \
  "v_cvt_pk_bf16_f32 v83, v90, v91\n\t" \
  "global_store_dwordx4 v9, v[80:83], s[50:51] offset:-3072 sc0 sc1\n\t" \
  "v_mfma_f32_16x16x32_bf16 v[148:151], a[232:235], v[96:99], v[148:151]\n\t" \
  "v_accvgpr_read_b32 v68, a16\n\t" \
  "v_accvgpr_read_b32 v69, a17\n\t" \
  "v_accvgpr_read_b32 v70, a18\n\t" \
  "v_mfma_f32_16x16x32_bf16 v[152:155], a[240:243], v[96:99], v[152:155]\n\t" \
  "ds_read_b64_tr_b16 a[164:165], v12 offset:24576\n\t" \
  "v_accvgpr_read_b32 v71, a19\n\t" \
  "v_accvgpr_read_b32 v72, a20\n\t" \
  "v_mfma_f32_16x16x32_bf16 v[152:155], a[244:247], v[100:103], v[152:155]\n\t" \
  "v_accvgpr_read_b32 v73, a21\n\t" \
  "v_accvgpr_read_b32 v74, a22\n\t" \
  "v_accvgpr_read_b32 v75, a23\n\t" \
  "v_mfma_f32_16x16x32_bf16 v[156:159], a[252:255], v[100:103], v[156:159]\n\t" \
  "ds_read_b64_tr_b16 a[166:167], v12 offset:25600\n\t" \
  "v_cvt_pk_bf16_f32 v64, v68, v69\n\t" \
  "v_cvt_pk_bf16_f32 v65, v70, v71\n\t" \
  "v_mfma_f32_16x16x32_bf16 v[156:159], a[248:251], v[96:99], v[156:159]\n\t" \
  "ds_read_b128 v[96:99], v10 offset:16384\n\t" \
  "ds_read_b128 v[100:103], v11 offset:16384\n\t" \
  "v_cvt_pk_bf16_f32 v66, v72, v73\n\t" \
  "v_mfma_f32_16x16x32_bf16 v[188:191], a[248:251], v[104:107], v[188:191]\n\t" \
  "ds_read_b64_tr_b16 a[168:169], v13 offset:16384\n\t" \
  "v_cvt_pk_bf16_f32 v67, v74, v75\n\t" \
  "global_store_dwordx4 v9, v[64:67], s[50:51] offset:-2048 sc0 sc1\n\t" \
  "v_mfma_f32_16x16x32_bf16 v[188:191], a[252:255], v[108:111], v[188:191]\n\t" \
  "v_accvgpr_read_b32 v84, a24\n\t" \
  "v_accvgpr_read_b32 v85, a25\n\t" \
  "v_accvgpr_read_b32 v86, a26\n\t" \
  "v_mfma_f32_16x16x32_bf16 v[184:187], a[244:247], v[108:111], v[184:187]\n\t" \
  "ds_read_b64_tr_b16 a[170:171], v13 offset:17408\n\t" \
  "v_accvgpr_read_b32 v87, a27\n\t" \
  "v_accvgpr_read_b32 v88, a28\n\t" \
  "v_mfma_f32_16x16x32_bf16 v[184:187], a[240:243], v[104:107], v[184:187]\n\t" \
  "v_accvgpr_read_b32 v89, a29\n\t" \
  "v_accvgpr_read_b32 v90, a30\n\t" \
  "v_accvgpr_read_b32 v91, a31\n\t" \
  "v_mfma_f32_16x16x32_bf16 v[180:183], a[232:235], v[104:107], v[180:183]\n\t" \
  "ds_read_b64_tr_b16 a[172:173], v13 offset:24576\n\t" \
  "v_cvt_pk_bf16_f32 v80, v84, v85\n\t" \
  "v_cvt_pk_bf16_f32 v81, v86, v87\n\t" \
  "v_mfma_f32_16x16x32_bf16 v[180:183], a[236:239], v[108:111], v[180:183]\n\t" \
  "v_cvt_pk_bf16_f32 v82, v88, v89\n\t" \
  "v_cvt_pk_bf16_f32 v83, v90, v91\n\t" \
  "global_store_dwordx4 v9, v[80:83], s[50:51] offset:-1024 sc0 sc1\n\t" \
  "v_mfma_f32_16x16x32_bf16 v[176:179], a[228:231], v[108:111], v[176:179]\n\t" \
  "ds_read_b64_tr_b16 a[174:175], v13 offset:25600\n\t" \
  "v_accvgpr_read_b32 v68, a32\n\t" \
  "v_accvgpr_read_b32 v69, a33\n\t" \
  "v_mfma_f32_16x16x32_bf16 v[176:179], a[224:227], v[104:107], v[176:179]\n\t" \
  "v_accvgpr_read_b32 v70, a34\n\t" \
  "v_accvgpr_read_b32 v71, a35\n\t" \
  "v_accvgpr_read_b32 v72, a36\n\t" \
  "v_mfma_f32_16x16x32_bf16 v[172:175], a[216:219], v[104:107], v[172:175]\n\t" \
  "ds_read_b64_tr_b16 a[176:177], v14 offset:16384\n\t" \
  "v_accvgpr_read_b32 v73, a37\n\t" \
  "v_accvgpr_read_b32 v74, a38\n\t" \
  "v_mfma_f32_16x16x32_bf16 v[172:175], a[220:223], v[108:111], v[172:175]\n\t" \
  "v_accvgpr_read_b32 v75, a39\n\t" \
  "v_cvt_pk_bf16_f32 v64, v68, v69\n\t" \
  "v_cvt_pk_bf16_f32 v65, v70, v71\n\t" \
  "v_mfma_f32_16x16x32_bf16 v[168:171], a[212:215], v[108:111], v[168:171]\n\t" \
  "ds_read_b64_tr_b16 a[178:179], v14 offset:17408\n\t" \
  "v_cvt_pk_bf16_f32 v66, v72, v73\n\t" \
  "v_cvt_pk_bf16_f32 v67, v74, v75\n\t" \
  "v_mfma_f32_16x16x32_bf16 v[168:171], a[208:211], v[104:107], v[168:171]\n\t" \
  "global_store_dwordx4 v9, v[64:67], s[50:51] offset:0 sc0 sc1\n\t" \
  "v_accvgpr_read_b32 v84, a40\n\t" \
  "v_accvgpr_read_b32 v85, a41\n\t" \
  "v_mfma_f32_16x16x32_bf16 v[164:167], a[200:203], v[104:107], v[164:167]\n\t" \
  "ds_read_b64_tr_b16 a[180:181], v14 offset:24576\n\t" \
  "v_accvgpr_read_b32 v86, a42\n\t" \
  "v_accvgpr_read_b32 v87, a43\n\t" \
  "v_mfma_f32_16x16x32_bf16 v[164:167], a[204:207], v[108:111], v[164:167]\n\t" \
  "v_accvgpr_read_b32 v88, a44\n\t" \
  "v_accvgpr_read_b32 v89, a45\n\t" \
  "v_accvgpr_read_b32 v90, a46\n\t" \
  "v_mfma_f32_16x16x32_bf16 v[160:163], a[196:199], v[108:111], v[160:163]\n\t" \
  "ds_read_b64_tr_b16 a[182:183], v14 offset:25600\n\t" \
  "v_accvgpr_read_b32 v91, a47\n\t" \
  "v_cvt_pk_bf16_f32 v80, v84, v85\n\t" \
  "v_mfma_f32_16x16x32_bf16 v[160:163], a[192:195], v[104:107], v[160:163]\n\t" \
  "ds_read_b128 v[104:107], v10 offset:18432\n\t" \
  "ds_read_b128 v[108:111], v11 offset:18432\n\t" \
  "v_cvt_pk_bf16_f32 v81, v86, v87\n\t" \
  "v_mfma_f32_16x16x32_bf16 v[192:195], a[192:195], v[112:115], v[192:195]\n\t" \
  "ds_read_b64_tr_b16 a[184:185], v15 offset:16384\n\t" \
  "v_cvt_pk_bf16_f32 v82, v88, v89\n\t" \
  "v_cvt_pk_bf16_f32 v83, v90, v91\n\t" \
  "v_mfma_f32_16x16x32_bf16 v[192:195], a[196:199], v[116:119], v[192:195]\n\t" \
  "global_store_dwordx4 v9, v[80:83], s[50:51] offset:1024 sc0 sc1\n\t" \
  "v_accvgpr_read_b32 v68, a48\n\t" \
  "v_accvgpr_read_b32 v69, a49\n\t" \
  "v_mfma_f32_16x16x32_bf16 v[196:199], a[204:207], v[116:119], v[196:199]\n\t" \
  "ds_read_b64_tr_b16 a[186:187], v15 offset:17408\n\t" \
  "v_accvgpr_read_b32 v70, a50\n\t" \
  "v_accvgpr_read_b32 v71, a51\n\t" \
  "v_mfma_f32_16x16x32_bf16 v[196:199], a[200:203], v[112:115], v[196:199]\n\t" \
  "v_accvgpr_read_b32 v72, a52\n\t" \
  "v_accvgpr_read_b32 v73, a53\n\t" \
  "v_accvgpr_read_b32 v74, a54\n\t" \
  "v_mfma_f32_16x16x32_bf16 v[200:203], a[208:211], v[112:115], v[200:203]\n\t" \
  "ds_read_b64_tr_b16 a[188:189], v15 offset:24576\n\t" \
  "v_accvgpr_read_b32 v75, a55\n\t" \
  "v_cvt_pk_bf16_f32 v64, v68, v69\n\t" \
  "v_mfma_f32_16x16x32_bf16 v[200:203], a[212:215], v[116:119], v[200:203]\n\t" \
  "v_cvt_pk_bf16_f32 v65, v70, v71\n\t" \
  "v_cvt_pk_bf16_f32 v66, v72, v73\n\t" \
  "v_cvt_pk_bf16_f32 v67, v74, v75\n\t" \
  "v_mfma_f32_16x16x32_bf16 v[204:207], a[220:223], v[116:119], v[204:207]\n\t" \
  "ds_read_b64_tr_b16 a[190:191], v15 offset:25600\n\t" \
  "global_store_dwordx4 v9, v[64:67], s[50:51] offset:2048 sc0 sc1\n\t" \
  "v_accvgpr_read_b32 v84, a56\n\t" \
  "v_mfma_f32_16x16x32_bf16 v[204:207], a[216:219], v[112:115], v[204:207]\n\t" \
  "s_add_u32 m0, s68, 49152\n\t" \
  "v_accvgpr_read_b32 v85, a57\n\t" \
  "v_accvgpr_read_b32 v86, a58\n\t" \
  "v_mfma_f32_16x16x32_bf16 v[208:211], a[224:227], v[112:115], v[208:211]\n\t" \
  "buffer_load_dwordx4 v4, s[40:43], s78 offen offset:0 lds\n\t" \
  "v_accvgpr_read_b32 v87, a59\n\t" \
  "v_accvgpr_read_b32 v88, a60\n\t" \
  "v_mfma_f32_16x16x32_bf16 v[208:211], a[228:231], v[116:119], v[208:211]\n\t" \
  "v_accvgpr_read_b32 v89, a61\n\t" \
  "v_accvgpr_read_b32 v90, a62\n\t" \
  "v_accvgpr_read_b32 v91, a63\n\t" \
  "v_mfma_f32_16x16x32_bf16 v[212:215], a[236:239], v[116:119], v[212:215]\n\t" \
  "v_cvt_pk_bf16_f32 v80, v84, v85\n\t" \
  "v_cvt_pk_bf16_f32 v81, v86, v87\n\t" \
  "v_cvt_pk_bf16_f32 v82, v88, v89\n\t" \
  "v_mfma_f32_16x16x32_bf16 v[212:215], a[232:235], v[112:115], v[212:215]\n\t" \
  "buffer_load_dwordx4 v5, s[40:43], s78 offen offset:1024 lds\n\t" \
  "v_cvt_pk_bf16_f32 v83, v90, v91\n\t" \
  "global_store_dwordx4 v9, v[80:83], s[50:51] offset:3072 sc0 sc1\n\t" \
  "v_mfma_f32_16x16x32_bf16 v[216:219], a[240:243], v[112:115], v[216:219]\n\t" \
  "v_mfma_f32_16x16x32_bf16 v[216:219], a[244:247], v[116:119], v[216:219]\n\t" \
  "v_mfma_f32_16x16x32_bf16 v[220:223], a[252:255], v[116:119], v[220:223]\n\t" \
  "buffer_load_dwordx4 v6, s[40:43], s78 offen offset:2048 lds\n\t" \
  "v_mfma_f32_16x16x32_bf16 v[220:223], a[248:251], v[112:115], v[220:223]\n\t" \
  "ds_read_b128 v[112:115], v10 offset:20480\n\t" \
  "ds_read_b128 v[116:119], v11 offset:20480\n\t" \
  "v_mfma_f32_16x16x32_bf16 v[252:255], a[248:251], v[120:123], v[252:255]\n\t" \
  "v_mfma_f32_16x16x32_bf16 v[252:255], a[252:255], v[124:127], v[252:255]\n\t" \
  "buffer_load_dwordx4 v7, s[40:43], s78 offen offset:3072 lds\n\t" \
  "v_mfma_f32_16x16x32_bf16 v[248:251], a[244:247], v[124:127], v[248:251]\n\t" \
  "v_mfma_f32_16x16x32_bf16 v[248:251], a[240:243], v[120:123], v[248:251]\n\t" \
  "s_add_u32 m0, s68, 65536\n\t" \
  "v_mfma_f32_16x16x32_bf16 v[244:247], a[232:235], v[120:123], v[244:247]\n\t" \
  "buffer_load_dwordx4 v4, s[40:43], s79 offen offset:0 lds\n\t" \
  "v_cmp_le_u32 s[92:93], s86, v37\n\t" \
  "v_mfma_f32_16x16x32_bf16 v[244:247], a[236:239], v[124:127], v[244:247]\n\t" \
  "v_mfma_f32_16x16x32_bf16 v[240:243], a[228:231], v[124:127], v[240:243]\n\t" \
  "v_mfma_f32_16x16x32_bf16 v[240:243], a[224:227], v[120:123], v[240:243]\n\t" \
  "buffer_load_dwordx4 v5, s[40:43], s79 offen offset:1024 lds\n\t" \
  "s_or_b64 s[58:59], s[58:59], s[92:93]\n\t" \
  "v_mfma_f32_16x16x32_bf16 v[236:239], a[216:219], v[120:123], v[236:239]\n\t" \
  "v_mfma_f32_16x16x32_bf16 v[236:239], a[220:223], v[124:127], v[236:239]\n\t" \
  "s_bitcmp1_b64 s[58:59], 3\n\t" \
  "s_cbranch_scc0 SLOW150_%=\n\t" \
  "RET149_%=:\n\t" \
  "v_mfma_f32_16x16x32_bf16 v[232:235], a[212:215], v[124:127], v[232:235]\n\t" \
  "buffer_load_dwordx4 v6, s[40:43], s79 offen offset:2048 lds\n\t" \
  "v_mfma_f32_16x16x32_bf16 v[232:235], a[208:211], v[120:123], v[232:235]\n\t" \
  "v_mfma_f32_16x16x32_bf16 v[228:231], a[200:203], v[120:123], v[228:231]\n\t" \
  "v_mfma_f32_16x16x32_bf16 v[228:231], a[204:207], v[124:127], v[228:231]\n\t" \
  "v_mfma_f32_16x16x32_bf16 v[224:227], a[196:199], v[124:127], v[224:227]\n\t" \
  "buffer_load_dwordx4 v7, s[40:43], s79 offen offset:3072 lds\n\t" \
  "v_mfma_f32_16x16x32_bf16 v[224:227], a[192:195], v[120:123], v[224:227]\n\t" \
  "ds_read_b128 v[120:123], v10 offset:22528\n\t" \
  "ds_read_b128 v[124:127], v11 offset:22528\n\t" \
  "s_waitcnt lgkmcnt(4)\n\t" \
  "v_mfma_f32_16x16x32_bf16 v[128:131], a[128:131], v[96:99], v[128:131]\n\t" \
  "v_mfma_f32_16x16x32_bf16 v[128:131], a[132:135], v[100:103], v[128:131]\n\t" \
  "v_add_u32 v12, 32768, v12\n\t" \
  "v_accvgpr_read_b32 v68, a64\n\t" \
  "v_accvgpr_read_b32 v69, a65\n\t" \
  "v_mfma_f32_16x16x32_bf16 v[132:135], a[140:143], v[100:103], v[132:135]\n\t" \
  "v_add_u32 v13, 32768, v13\n\t" \
  "v_accvgpr_read_b32 v70, a66\n\t" \
  "v_accvgpr_read_b32 v71, a67\n\t" \
  "v_mfma_f32_16x16x32_bf16 v[132:135], a[136:139], v[96:99], v[132:135]\n\t" \
  "v_add_u32 v14, 32768, v14\n\t" \
  "v_accvgpr_read_b32 v72, a68\n\t" \
  "v_accvgpr_read_b32 v73, a69\n\t" \
  "v_mfma_f32_16x16x32_bf16 v[136:139], a[144:147], v[96:99], v[136:139]\n\t" \
  "v_add_u32 v15, 32768, v15\n\t" \
  "v_accvgpr_read_b32 v74, a70\n\t" \
  "v_accvgpr_read_b32 v75, a71\n\t" \
  "v_mfma_f32_16x16x32_bf16 v[136:139], a[148:151], v[100:103], v[136:139]\n\t" \
  "v_cvt_pk_bf16_f32 v64, v68, v69\n\t" \
  "v_cvt_pk_bf16_f32 v65, v70, v71\n\t" \
  "v_cvt_pk_bf16_f32 v66, v72, v73\n\t" \
  "v_mfma_f32_16x16x32_bf16 v[140:143], a[156:159], v[100:103], v[140:143]\n\t" \
  "v_cvt_pk_bf16_f32 v67, v74, v75\n\t" \
  "v_add_u32 v9, 8192, v9\n\t" \
  "global_store_dwordx4 v9, v[64:67], s[50:51] offset:-4096 sc0 sc1\n\t" \
  "v_mfma_f32_16x16x32_bf16 v[140:143], a[152:155], v[96:99], v[140:143]\n\t" \
  "v_accvgpr_read_b32 v84, a72\n\t" \
  "v_accvgpr_read_b32 v85, a73\n\t" \
  "v_accvgpr_read_b32 v86, a74\n\t" \
  "s_waitcnt vmcnt(29) lgkmcnt(0)\n\t" \
  "s_barrier\n\t" \
  "v_mfma_f32_16x16x32_bf16 v[144:147], a[160:163], v[96:99], v[144:147]\n\t" \
  "ds_read_b64_tr_b16 a[192:193], v12 offset:0\n\t" \
  "v_accvgpr_read_b32 v87, a75\n\t" \
  "v_accvgpr_read_b32 v88, a76\n\t" \
  "v_mfma_f32_16x16x32_bf16 v[144:147], a[164:167], v[100:103], v[144:147]\n\t" \
  "ds_read_b64_tr_b16 a[194:195], v12 offset:1024\n\t" \
  "v_accvgpr_read_b32 v89, a77\n\t" \
  "v_mfma_f32_16x16x32_bf16 v[148:151], a[172:175], v[100:103], v[148:151]\n\t" \
  "ds_read_b64_tr_b16 a[196:197], v12 offset:8192\n\t" \
  "s_andn2_b64 s[92:93], 255, s[58:59]\n\t" \
  "s_cbranch_scc1 POLL154_%=\n\t" \
  "RET153_%=:\n\t" \
  "v_accvgpr_read_b32 v90, a78\n\t" \
  "v_mfma_f32_16x16x32_bf16 v[148:151], a[168:171], v[96:99], v[148:151]\n\t" \
  "ds_read_b64_tr_b16 a[198:199], v12 offset:9216\n\t" \
  "v_accvgpr_read_b32 v91, a79\n\t" \
  "v_cvt_pk_bf16_f32 v80, v84, v85\n\t" \
  "v_mfma_f32_16x16x32_bf16 v[152:155], a[176:179], v[96:99], v[152:155]\n\t" \
  "ds_read_b64_tr_b16 a[200:201], v13 offset:0\n\t" \
  "v_cvt_pk_bf16_f32 v81, v86, v87\n\t" \
  "v_cvt_pk_bf16_f32 v82, v88, v89\n\t" \
  "v_mfma_f32_16x16x32_bf16 v[152:155], a[180:183], v[100:103], v[152:155]\n\t" \
  "ds_read_b64_tr_b16 a[202:203], v13 offset:1024\n\t" \
  "v_cvt_pk_bf16_f32 v83, v90, v91\n\t" \
  "global_store_dwordx4 v9, v[80:83], s[50:51] offset:-3072 sc0 sc1\n\t" \
  "v_mfma_f32_16x16x32_bf16 v[156:159], a[188:191], v[100:103], v[156:159]\n\t" \
  "ds_read_b64_tr_b16 a[204:205], v13 offset:8192\n\t" \
  "v_accvgpr_read_b32 v68, a80\n\t" \
  "v_mfma_f32_16x16x32_bf16 v[156:159], a[184:187], v[96:99], v[156:159]\n\t" \
  "ds_read_b128 v[96:99], v10 offset:0\n\t" \
  "ds_read_b128 v[100:103], v11 offset:0\n\t" \
  "v_accvgpr_read_b32 v69, a81\n\t" \
  "v_mfma_f32_16x16x32_bf16 v[188:191], a[184:187], v[104:107], v[188:191]\n\t" \
  "ds_read_b64_tr_b16 a[206:207], v13 offset:9216\n\t" \
  "v_accvgpr_read_b32 v70, a82\n\t" \
  "v_accvgpr_read_b32 v71, a83\n\t" \
  "v_mfma_f32_16x16x32_bf16 v[188:191], a[188:191], v[108:111], v[188:191]\n\t" \
  "ds_read_b64_tr_b16 a[208:209], v14 offset:0\n\t" \
  "v_accvgpr_read_b32 v72, a84\n\t" \
  "v_mfma_f32_16x16x32_bf16 v[184:187], a[180:183], v[108:111], v[184:187]\n\t" \
  "ds_read_b64_tr_b16 a[210:211], v14 offset:1024\n\t" \
  "v_accvgpr_read_b32 v73, a85\n\t" \
  "v_accvgpr_read_b32 v74, a86\n\t" \
  "v_mfma_f32_16x16x32_bf16 v[184:187], a[176:179], v[104:107], v[184:187]\n\t" \
  "ds_read_b64_tr_b16 a[212:213], v14 offset:8192\n\t" \
  "v_accvgpr_read_b32 v75, a87\n\t" \
  "v_cvt_pk_bf16_f32 v64, v68, v69\n\t" \
  "v_mfma_f32_16x16x32_bf16 v[180:183], a[168:171], v[104:107], v[180:183]\n\t" \
  "ds_read_b64_tr_b16 a[214:215], v14 offset:9216\n\t" \
  "v_cvt_pk_bf16_f32 v65, v70, v71\n\t" \
  "v_mfma_f32_16x16x32_bf16 v[180:183], a[172:175], v[108:111], v[180:183]\n\t" \
  "ds_read_b64_tr_b16 a[216:217], v15 offset:0\n\t" \
  "v_cvt_pk_bf16_f32 v66, v72, v73\n\t" \
  "v_cvt_pk_bf16_f32 v67, v74, v75\n\t" \
  "v_mfma_f32_16x16x32_bf16 v[176:179], a[164:167], v[108:111], v[176:179]\n\t" \
  "ds_read_b64_tr_b16 a[218:219], v15 offset:1024\n\t" \
  "global_store_dwordx4 v9, v[64:67], s[50:51] offset:-2048 sc0 sc1\n\t" \
  "v_accvgpr_read_b32 v84, a88\n\t" \
  "v_mfma_f32_16x16x32_bf16 v[176:179], a[160:163], v[104:107], v[176:179]\n\t" \
  "ds_read_b64_tr_b16 a[220:221], v15 offset:8192\n\t" \
  "v_accvgpr_read_b32 v85, a89\n\t" \
  "v_mfma_f32_16x16x32_bf16 v[172:175], a[152:155], v[104:107], v[172:175]\n\t" \
  "ds_read_b64_tr_b16 a[222:223], v15 offset:9216\n\t" \
  "v_accvgpr_read_b32 v86, a90\n\t" \
  "v_accvgpr_read_b32 v87, a91\n\t" \
  "v_mfma_f32_16x16x32_bf16 v[172:175], a[156:159], v[108:111], v[172:175]\n\t" \
  "ds_read_b64_tr_b16 a[224:225], v12 offset:32768\n\t" \
  "v_accvgpr_read_b32 v88, a92\n\t" \
  "v_accvgpr_read_b32 v89, a93\n\t" \
  "v_mfma_f32_16x16x32_bf16 v[168:171], a[148:151], v[108:111], v[168:171]\n\t" \
  "ds_read_b64_tr_b16 a[226:227], v12 offset:33792\n\t" \
  "v_accvgpr_read_b32 v90, a94\n\t" \
  "v_mfma_f32_16x16x32_bf16 v[168:171], a[144:147], v[104:107], v[168:171]\n\t" \
  "ds_read_b64_tr_b16 a[228:229], v12 offset:40960\n\t" \
  "v_accvgpr_read_b32 v91, a95\n\t" \
  "v_cvt_pk_bf16_f32 v80, v84, v85\n\t" \
  "v_mfma_f32_16x16x32_bf16 v[164:167], a[136:139], v[104:107], v[164:167]\n\t" \
  "ds_read_b64_tr_b16 a[230:231], v12 offset:41984\n\t" \
  "v_cvt_pk_bf16_f32 v81, v86, v87\n\t" \
  "v_cvt_pk_bf16_f32 v82, v88, v89\n\t" \
  "v_mfma_f32_16x16x32_bf16 v[164:167], a[140:143], v[108:111], v[164:167]\n\t" \
  "ds_read_b64_tr_b16 a[232:233], v13 offset:32768\n\t" \
  "v_cvt_pk_bf16_f32 v83, v90, v91\n\t" \
  "v_mfma_f32_16x16x32_bf16 v[160:163], a[132:135], v[108:111], v[160:163]\n\t" \
  "ds_read_b64_tr_b16 a[234:235], v13 offset:33792\n\t" \
  "global_store_dwordx4 v9, v[80:83], s[50:51] offset:-1024 sc0 sc1\n\t" \
  "v_accvgpr_read_b32 v68, a96\n\t" \
  "v_mfma_f32_16x16x32_bf16 v[160:163], a[128:131], v[104:107], v[160:163]\n\t" \
  "ds_read_b128 v[104:107], v10 offset:2048\n\t" \
  "ds_read_b128 v[108:111], v11 offset:2048\n\t" \
  "v_accvgpr_read_b32 v69, a97\n\t" \
  "v_mfma_f32_16x16x32_bf16 v[192:195], a[128:131], v[112:115], v[192:195]\n\t" \
  "ds_read_b64_tr_b16 a[236:237], v13 offset:40960\n\t" \
  "v_accvgpr_read_b32 v70, a98\n\t" \
  "v_mfma_f32_16x16x32_bf16 v[192:195], a[132:135], v[116:119], v[192:195]\n\t" \
  "ds_read_b64_tr_b16 a[238:239], v13 offset:41984\n\t" \
  "v_accvgpr_read_b32 v71, a99\n\t" \
  "v_accvgpr_read_b32 v72, a100\n\t" \
  "v_mfma_f32_16x16x32_bf16 v[196:199], a[140:143], v[116:119], v[196:199]\n\t" \
  "ds_read_b64_tr_b16 a[240:241], v14 offset:32768\n\t" \
  "v_accvgpr_read_b32 v73, a101\n\t" \
  "v_accvgpr_read_b32 v74, a102\n\t" \
  "v_mfma_f32_16x16x32_bf16 v[196:199], a[136:139], v[112:115], v[196:199]\n\t" \
  "ds_read_b64_tr_b16 a[242:243], v14 offset:33792\n\t" \
  "v_accvgpr_read_b32 v75, a103\n\t" \
  "v_mfma_f32_16x16x32_bf16 v[200:203], a[144:147], v[112:115], v[200:203]\n\t" \
  "ds_read_b64_tr_b16 a[244:245], v14 offset:40960\n\t" \
  "v_cvt_pk_bf16_f32 v64, v68, v69\n\t" \
  "v_cvt_pk_bf16_f32 v65, v70, v71\n\t" \
  "v_mfma_f32_16x16x32_bf16 v[200:203], a[148:151], v[116:119], v[200:203]\n\t" \
  "ds_read_b64_tr_b16 a[246:247], v14 offset:41984\n\t" \
  "v_cvt_pk_bf16_f32 v66, v72, v73\n\t" \
  "v_cvt_pk_bf16_f32 v67, v74, v75\n\t" \
  "v_mfma_f32_16x16x32_bf16 v[204:207], a[156:159], v[116:119], v[204:207]\n\t" \
  "ds_read_b64_tr_b16 a[248:249], v15 offset:32768\n\t" \
  "global_store_dwordx4 v9, v[64:67], s[50:51] offset:0 sc0 sc1\n\t" \
  "v_mfma_f32_16x16x32_bf16 v[204:207], a[152:155], v[112:115], v[204:207]\n\t" \
  "ds_read_b64_tr_b16 a[250:251], v15 offset:33792\n\t" \
  "s_add_u32 m0, s68, 16384\n\t" \
  "v_accvgpr_read_b32 v84, a104\n\t" \
  "v_mfma_f32_16x16x32_bf16 v[208:211], a[160:163], v[112:115], v[208:211]\n\t" \
  "ds_read_b64_tr_b16 a[252:253], v15 offset:40960\n\t" \
  "buffer_load_dwordx4 v0, s[36:39], s65 offen offset:0 sc0 sc1 lds\n\t" \
  "v_accvgpr_read_b32 v85, a105\n\t" \
  "v_mfma_f32_16x16x32_bf16 v[208:211], a[164:167], v[116:119], v[208:211]\n\t" \
  "ds_read_b64_tr_b16 a[254:255], v15 offset:41984\n\t" \
  "v_accvgpr_read_b32 v86, a106\n\t" \
  "v_mfma_f32_16x16x32_bf16 v[212:215], a[172:175], v[116:119], v[212:215]\n\t" \
  "v_accvgpr_read_b32 v87, a107\n\t" \
  "v_accvgpr_read_b32 v88, a108\n\t" \
  "v_accvgpr_read_b32 v89, a109\n\t" \
  "v_mfma_f32_16x16x32_bf16 v[212:215], a[168:171], v[112:115], v[212:215]\n\t" \
  "buffer_load_dwordx4 v1, s[36:39], s65 offen offset:1024 sc0 sc1 lds\n\t" \
  "v_accvgpr_read_b32 v90, a110\n\t" \
  "v_accvgpr_read_b32 v91, a111\n\t" \
  "v_mfma_f32_16x16x32_bf16 v[216:219], a[176:179], v[112:115], v[216:219]\n\t" \
  "v_cvt_pk_bf16_f32 v80, v84, v85\n\t" \
  "v_cvt_pk_bf16_f32 v81, v86, v87\n\t" \
  "v_mfma_f32_16x16x32_bf16 v[216:219], a[180:183], v[116:119], v[216:219]\n\t" \
  "v_cvt_pk_bf16_f32 v82, v88, v89\n\t" \
  "v_cvt_pk_bf16_f32 v83, v90, v91\n\t" \
  "global_store_dwordx4 v9, v[80:83], s[50:51] offset:1024 sc0 sc1\n\t" \
  "v_mfma_f32_16x16x32_bf16 v[220:223], a[188:191], v[116:119], v[220:223]\n\t" \
  "buffer_load_dwordx4 v2, s[36:39], s65 offen offset:2048 sc0 sc1 lds\n\t" \
  "v_accvgpr_read_b32 v68, a112\n\t" \
  "v_accvgpr_read_b32 v69, a113\n\t" \
  "v_mfma_f32_16x16x32_bf16 v[220:223], a[184:187], v[112:115], v[220:223]\n\t" \
  "ds_read_b128 v[112:115], v10 offset:4096\n\t" \
  "ds_read_b128 v[116:119], v11 offset:4096\n\t" \
  "v_mfma_f32_16x16x32_bf16 v[252:255], a[184:187], v[120:123], v[252:255]\n\t" \
  "v_accvgpr_read_b32 v70, a114\n\t" \
  "v_accvgpr_read_b32 v71, a115\n\t" \
  "v_accvgpr_read_b32 v72, a116\n\t" \
  "v_mfma_f32_16x16x32_bf16 v[252:255], a[188:191], v[124:127], v[252:255]\n\t" \
  "buffer_load_dwordx4 v3, s[36:39], s65 offen offset:3072 sc0 sc1 lds\n\t" \
  "v_readlane_b32 s66, v17, 4\n\t" \
  "s_nop 0\n\t" \
  "s_lshl_b32 s78, s66, 13\n\t" \
  "s_lshl_b32 s65, s66, 10\n\t" \
  "s_add_u32 s79, s78, 256\n\t" \
  "s_add_u32 s65, s65, s70\n\t" \
  "s_nop 1\n\t" \
  "v_mfma_f32_16x16x32_bf16 v[248:251], a[180:183], v[124:127], v[248:251]\n\t" \
  "v_accvgpr_read_b32 v73, a117\n\t" \
  "v_accvgpr_read_b32 v74, a118\n\t" \
  "v_mfma_f32_16x16x32_bf16 v[248:251], a[176:179], v[120:123], v[248:251]\n\t" \
  "s_add_u32 m0, s68, 81920\n\t" \
  "v_accvgpr_read_b32 v75, a119\n\t" \
  "v_cvt_pk_bf16_f32 v64, v68, v69\n\t" \
  "v_mfma_f32_16x16x32_bf16 v[244:247], a[168:171], v[120:123], v[244:247]\n\t" \
  "buffer_load_dwordx4 v4, s[40:43], s78 offen offset:0 lds\n\t" \
  "v_cmp_le_u32 s[92:93], s86, v37\n\t" \
  "v_cvt_pk_bf16_f32 v65, v70, v71\n\t" \
  "v_mfma_f32_16x16x32_bf16 v[244:247], a[172:175], v[124:127], v[244:247]\n\t" \
  "v_cvt_pk_bf16_f32 v66, v72, v73\n\t" \
  "v_cvt_pk_bf16_f32 v67, v74, v75\n\t" \
  "v_mfma_f32_16x16x32_bf16 v[240:243], a[164:167], v[124:127], v[240:243]\n\t" \
  "global_store_dwordx4 v9, v[64:67], s[50:51] offset:2048 sc0 sc1\n\t" \
  "v_accvgpr_read_b32 v84, a120\n\t" \
  "v_accvgpr_read_b32 v85, a121\n\t" \
  "v_mfma_f32_16x16x32_bf16 v[240:243], a[160:163], v[120:123], v[240:243]\n\t" \
  "buffer_load_dwordx4 v5, s[40:43], s78 offen offset:1024 lds\n\t" \
  "s_or_b64 s[58:59], s[58:59], s[92:93]\n\t" \
  "v_accvgpr_read_b32 v86, a122\n\t" \
  "v_mfma_f32_16x16x32_bf16 v[236:239], a[152:155], v[120:123], v[236:239]\n\t" \
  "v_accvgpr_read_b32 v87, a123\n\t" \
  "v_accvgpr_read_b32 v88, a124\n\t" \
  "v_mfma_f32_16x16x32_bf16 v[236:239], a[156:159], v[124:127], v[236:239]\n\t" \
  "s_bitcmp1_b64 s[58:59], 4\n\t" \
  "s_cbranch_scc0 SLOW156_%=\n\t" \
  "RET155_%=:\n\t" \
  "v_accvgpr_read_b32 v89, a125\n\t" \
  "v_accvgpr_read_b32 v90, a126\n\t" \
  "v_mfma_f32_16x16x32_bf16 v[232:235], a[148:151], v[124:127], v[232:235]\n\t" \
  "buffer_load_dwordx4 v6, s[40:43], s78 offen offset:2048 lds\n\t" \
  "v_accvgpr_read_b32 v91, a127\n\t" \
  "v_cvt_pk_bf16_f32 v80, v84, v85\n\t" \
  "v_mfma_f32_16x16x32_bf16 v[232:235], a[144:147], v[120:123], v[232:235]\n\t" \
  "v_cvt_pk_bf16_f32 v81, v86, v87\n\t" \
  "v_cvt_pk_bf16_f32 v82, v88, v89\n\t" \
  "v_mfma_f32_16x16x32_bf16 v[228:231], a[136:139], v[120:123], v[228:231]\n\t" \
  "v_cvt_pk_bf16_f32 v83, v90, v91\n\t" \
  "global_store_dwordx4 v9, v[80:83], s[50:51] offset:3072 sc0 sc1\n\t" \
  "v_mfma_f32_16x16x32_bf16 v[228:231], a[140:143], v[124:127], v[228:231]\n\t" \
  "v_mfma_f32_16x16x32_bf16 v[224:227], a[132:135], v[124:127], v[224:227]\n\t" \
  "buffer_load_dwordx4 v7, s[40:43], s78 offen offset:3072 lds\n\t" \
  "v_mfma_f32_16x16x32_bf16 v[224:227], a[128:131], v[120:123], v[224:227]\n\t" \
  "ds_read_b128 v[120:123], v10 offset:6144\n\t" \
  "ds_read_b128 v[124:127], v11 offset:6144\n\t" \
  "s_waitcnt lgkmcnt(4)\n\t" \
  "v_mfma_f32_16x16x32_bf16 v[128:131], a[192:195], v[96:99], v[128:131]\n\t" \
  "v_mfma_f32_16x16x32_bf16 v[128:131], a[196:199], v[100:103], v[128:131]\n\t" \
  "v_add_u32 v10, 32768, v10\n\t" \
  "v_mfma_f32_16x16x32_bf16 v[132:135], a[204:207], v[100:103], v[132:135]\n\t" \
  "v_add_u32 v11, 32768, v11\n\t" \
  "v_mfma_f32_16x16x32_bf16 v[132:135], a[200:203], v[96:99], v[132:135]\n\t" \
  "v_add_u32 v12, 16384, v12\n\t" \
  "v_mfma_f32_16x16x32_bf16 v[136:139], a[208:211], v[96:99], v[136:139]\n\t" \
  "v_add_u32 v13, 16384, v13\n\t" \
  "v_mfma_f32_16x16x32_bf16 v[136:139], a[212:215], v[100:103], v[136:139]\n\t" \
  "v_add_u32 v14, 16384, v14\n\t" \
  "v_mfma_f32_16x16x32_bf16 v[140:143], a[220:223], v[100:103], v[140:143]\n\t" \
  "v_add_u32 v15, 16384, v15\n\t" \
  "v_mfma_f32_16x16x32_bf16 v[140:143], a[216:219], v[96:99], v[140:143]\n\t" \
  "s_waitcnt vmcnt(32) lgkmcnt(0)\n\t" \
  "s_barrier\n\t" \
  "v_mfma_f32_16x16x32_bf16 v[144:147], a[224:227], v[96:99], v[144:147]\n\t" \
  "ds_read_b64_tr_b16 a[128:129], v12 offset:0\n\t" \
  "v_mfma_f32_16x16x32_bf16 v[144:147], a[228:231], v[100:103], v[144:147]\n\t" \
  "ds_read_b64_tr_b16 a[130:131], v12 offset:1024\n\t" \
  "v_mfma_f32_16x16x32_bf16 v[148:151], a[236:239], v[100:103], v[148:151]\n\t" \
  "ds_read_b64_tr_b16 a[132:133], v12 offset:8192\n\t" \
  "v_mfma_f32_16x16x32_bf16 v[148:151], a[232:235], v[96:99], v[148:151]\n\t" \
  "ds_read_b64_tr_b16 a[134:135], v12 offset:9216\n\t" \
  "s_add_u32 m0, s68, 0\n\t" \
  "v_mfma_f32_16x16x32_bf16 v[152:155], a[240:243], v[96:99], v[152:155]\n\t" \
  "ds_read_b64_tr_b16 a[136:137], v13 offset:0\n\t" \
  "buffer_load_dwordx4 v0, s[36:39], s65 offen offset:0 sc0 sc1 lds\n\t" \
  "v_mfma_f32_16x16x32_bf16 v[152:155], a[244:247], v[100:103], v[152:155]\n\t" \
  "ds_read_b64_tr_b16 a[138:139], v13 offset:1024\n\t" \
  "v_mfma_f32_16x16x32_bf16 v[156:159], a[252:255], v[100:103], v[156:159]\n\t" \
  "ds_read_b64_tr_b16 a[140:141], v13 offset:8192\n\t" \
  "v_mfma_f32_16x16x32_bf16 v[156:159], a[248:251], v[96:99], v[156:159]\n\t" \
  "ds_read_b128 v[96:99], v10 offset:0\n\t" \
  "ds_read_b128 v[100:103], v11 offset:0\n\t" \
  "v_mfma_f32_16x16x32_bf16 v[188:191], a[248:251], v[104:107], v[188:191]\n\t" \
  "ds_read_b64_tr_b16 a[142:143], v13 offset:9216\n\t" \
  "buffer_load_dwordx4 v1, s[36:39], s65 offen offset:1024 sc0 sc1 lds\n\t" \
  "v_mfma_f32_16x16x32_bf16 v[188:191], a[252:255], v[108:111], v[188:191]\n\t" \
  "ds_read_b64_tr_b16 a[144:145], v14 offset:0\n\t" \
  "v_mfma_f32_16x16x32_bf16 v[184:187], a[244:247], v[108:111], v[184:187]\n\t" \
  "ds_read_b64_tr_b16 a[146:147], v14 offset:1024\n\t" \
  "v_mfma_f32_16x16x32_bf16 v[184:187], a[240:243], v[104:107], v[184:187]\n\t" \
  "ds_read_b64_tr_b16 a[148:149], v14 offset:8192\n\t" \
  "v_mfma_f32_16x16x32_bf16 v[180:183], a[232:235], v[104:107], v[180:183]\n\t" \
  "ds_read_b64_tr_b16 a[150:151], v14 offset:9216\n\t" \
  "v_mfma_f32_16x16x32_bf16 v[180:183], a[236:239], v[108:111], v[180:183]\n\t" \
  "ds_read_b64_tr_b16 a[152:153], v15 offset:0\n\t" \
  "buffer_load_dwordx4 v2, s[36:39], s65 offen offset:2048 sc0 sc1 lds\n\t" \
  "v_mfma_f32_16x16x32_bf16 v[176:179], a[228:231], v[108:111], v[176:179]\n\t" \
  "ds_read_b64_tr_b16 a[154:155], v15 offset:1024\n\t" \
  "v_mfma_f32_16x16x32_bf16 v[176:179], a[224:227], v[104:107], v[176:179]\n\t" \
  "ds_read_b64_tr_b16 a[156:157], v15 offset:8192\n\t" \
  "v_mfma_f32_16x16x32_bf16 v[172:175], a[216:219], v[104:107], v[172:175]\n\t" \
  "ds_read_b64_tr_b16 a[158:159], v15 offset:9216\n\t" \
  "v_mfma_f32_16x16x32_bf16 v[172:175], a[220:223], v[108:111], v[172:175]\n\t" \
  "ds_read_b64_tr_b16 a[160:161], v12 offset:32768\n\t" \
  "buffer_load_dwordx4 v3, s[36:39], s65 offen offset:3072 sc0 sc1 lds\n\t" \
  "v_mfma_f32_16x16x32_bf16 v[168:171], a[212:215], v[108:111], v[168:171]\n\t" \
  "ds_read_b64_tr_b16 a[162:163], v12 offset:33792\n\t" \
  "v_mfma_f32_16x16x32_bf16 v[168:171], a[208:211], v[104:107], v[168:171]\n\t" \
  "ds_read_b64_tr_b16 a[164:165], v12 offset:40960\n\t" \
  "v_mfma_f32_16x16x32_bf16 v[164:167], a[200:203], v[104:107], v[164:167]\n\t" \
  "ds_read_b64_tr_b16 a[166:167], v12 offset:41984\n\t" \
  "v_mfma_f32_16x16x32_bf16 v[164:167], a[204:207], v[108:111], v[164:167]\n\t" \
  "ds_read_b64_tr_b16 a[168:169], v13 offset:32768\n\t" \
  "s_add_u32 m0, s68, 98304\n\t" \
  "v_mfma_f32_16x16x32_bf16 v[160:163], a[196:199], v[108:111], v[160:163]\n\t" \
  "ds_read_b64_tr_b16 a[170:171], v13 offset:33792\n\t" \
  "buffer_load_dwordx4 v4, s[40:43], s79 offen offset:0 lds\n\t" \
  "v_mfma_f32_16x16x32_bf16 v[160:163], a[192:195], v[104:107], v[160:163]\n\t" \
  "ds_read_b128 v[104:107], v10 offset:2048\n\t" \
  "ds_read_b128 v[108:111], v11 offset:2048\n\t" \
  "v_mfma_f32_16x16x32_bf16 v[192:195], a[192:195], v[112:115], v[192:195]\n\t" \
  "ds_read_b64_tr_b16 a[172:173], v13 offset:40960\n\t" \
  "v_mfma_f32_16x16x32_bf16 v[192:195], a[196:199], v[116:119], v[192:195]\n\t" \
  "ds_read_b64_tr_b16 a[174:175], v13 offset:41984\n\t" \
  "v_mfma_f32_16x16x32_bf16 v[196:199], a[204:207], v[116:119], v[196:199]\n\t" \
  "ds_read_b64_tr_b16 a[176:177], v14 offset:32768\n\t" \
  "buffer_load_dwordx4 v5, s[40:43], s79 offen offset:1024 lds\n\t" \
  "v_mfma_f32_16x16x32_bf16 v[196:199], a[200:203], v[112:115], v[196:199]\n\t" \
  "ds_read_b64_tr_b16 a[178:179], v14 offset:33792\n\t" \
  "v_mfma_f32_16x16x32_bf16 v[200:203], a[208:211], v[112:115], v[200:203]\n\t" \
  "ds_read_b64_tr_b16 a[180:181], v14 offset:40960\n\t" \
  "v_mfma_f32_16x16x32_bf16 v[200:203], a[212:215], v[116:119], v[200:203]\n\t" \
  "ds_read_b64_tr_b16 a[182:183], v14 offset:41984\n\t" \
  "v_mfma_f32_16x16x32_bf16 v[204:207], a[220:223], v[116:119], v[204:207]\n\t" \
  "ds_read_b64_tr_b16 a[184:185], v15 offset:32768\n\t" \
  "v_mfma_f32_16x16x32_bf16 v[204:207], a[216:219], v[112:115], v[204:207]\n\t" \
  "ds_read_b64_tr_b16 a[186:187], v15 offset:33792\n\t" \
  "buffer_load_dwordx4 v6, s[40:43], s79 offen offset:2048 lds\n\t" \
  "v_mfma_f32_16x16x32_bf16 v[208:211], a[224:227], v[112:115], v[208:211]\n\t" \
  "ds_read_b64_tr_b16 a[188:189], v15 offset:40960\n\t" \
  "v_mfma_f32_16x16x32_bf16 v[208:211], a[228:231], v[116:119], v[208:211]\n\t" \
  "ds_read_b64_tr_b16 a[190:191], v15 offset:41984\n\t" \
  "v_mfma_f32_16x16x32_bf16 v[212:215], a[236:239], v[116:119], v[212:215]\n\t" \
  "v_mfma_f32_16x16x32_bf16 v[212:215], a[232:235], v[112:115], v[212:215]\n\t" \
  "buffer_load_dwordx4 v7, s[40:43], s79 offen offset:3072 lds\n\t" \
  "v_readlane_b32 s66, v17, 5\n\t" \
  "s_nop 0\n\t" \
  "s_lshl_b32 s78, s66, 13\n\t" \
  "s_lshl_b32 s65, s66, 10\n\t" \
  "s_add_u32 s79, s78, 256\n\t" \
  "s_add_u32 s65, s65, s70\n\t" \
  "s_nop 1\n\t" \
  "v_mfma_f32_16x16x32_bf16 v[216:219], a[240:243], v[112:115], v[216:219]\n\t" \
  "v_mfma_f32_16x16x32_bf16 v[216:219], a[244:247], v[116:119], v[216:219]\n\t" \
  "v_mfma_f32_16x16x32_bf16 v[220:223], a[252:255], v[116:119], v[220:223]\n\t" \
  "v_mfma_f32_16x16x32_bf16 v[220:223], a[248:251], v[112:115], v[220:223]\n\t" \
  "ds_read_b128 v[112:115], v10 offset:4096\n\t" \
  "ds_read_b128 v[116:119], v11 offset:4096\n\t" \
  "s_add_u32 m0, s68, 131072\n\t" \
  "v_mfma_f32_16x16x32_bf16 v[252:255], a[248:251], v[120:123], v[252:255]\n\t" \
  "buffer_load_dwordx4 v4, s[40:43], s79 offen offset:0 lds\n\t" \
  "v_mfma_f32_16x16x32_bf16 v[252:255], a[252:255], v[124:127], v[252:255]\n\t" \
  "v_mfma_f32_16x16x32_bf16 v[248:251], a[244:247], v[124:127], v[248:251]\n\t" \
  "v_mfma_f32_16x16x32_bf16 v[248:251], a[240:243], v[120:123], v[248:251]\n\t" \
  "v_mfma_f32_16x16x32_bf16 v[244:247], a[232:235], v[120:123], v[244:247]\n\t" \
  "buffer_load_dwordx4 v5, s[40:43], s79 offen offset:1024 lds\n\t" \
  "v_cmp_le_u32 s[92:93], s86, v37\n\t" \
  "v_mfma_f32_16x16x32_bf16 v[244:247], a[236:239], v[124:127], v[244:247]\n\t" \
  "v_mfma_f32_16x16x32_bf16 v[240:243], a[228:231], v[124:127], v[240:243]\n\t" \
  "v_mfma_f32_16x16x32_bf16 v[240:243], a[224:227], v[120:123], v[240:243]\n\t" \
  "s_or_b64 s[58:59], s[58:59], s[92:93]\n\t" \
  "v_mfma_f32_16x16x32_bf16 v[236:239], a[216:219], v[120:123], v[236:239]\n\t" \
  "v_mfma_f32_16x16x32_bf16 v[236:239], a[220:223], v[124:127], v[236:239]\n\t" \
  "buffer_load_dwordx4 v6, s[40:43], s79 offen offset:2048 lds\n\t" \
  "s_bitcmp1_b64 s[58:59], 5\n\t" \
  "s_cbranch_scc0 SLOW160_%=\n\t" \
  "RET159_%=:\n\t" \
  "v_mfma_f32_16x16x32_bf16 v[232:235], a[212:215], v[124:127], v[232:235]\n\t" \
  "v_mfma_f32_16x16x32_bf16 v[232:235], a[208:211], v[120:123], v[232:235]\n\t" \
  "s_bitcmp1_b64 s[58:59], 6\n\t" \
  "s_cbranch_scc0 SLOW164_%=\n\t" \
  "RET163_%=:\n\t" \
  "v_mfma_f32_16x16x32_bf16 v[228:231], a[200:203], v[120:123], v[228:231]\n\t" \
  "v_mfma_f32_16x16x32_bf16 v[228:231], a[204:207], v[124:127], v[228:231]\n\t" \
  "v_mfma_f32_16x16x32_bf16 v[224:227], a[196:199], v[124:127], v[224:227]\n\t" \
  "buffer_load_dwordx4 v7, s[40:43], s79 offen offset:3072 lds\n\t" \
  "v_mfma_f32_16x16x32_bf16 v[224:227], a[192:195], v[120:123], v[224:227]\n\t" \
  "ds_read_b128 v[120:123], v10 offset:6144\n\t" \
  "ds_read_b128 v[124:127], v11 offset:6144\n\t" \
  "s_waitcnt lgkmcnt(4)\n\t" \
  "v_mfma_f32_16x16x32_bf16 v[128:131], a[128:131], v[96:99], v[128:131]\n\t" \
  "v_mfma_f32_16x16x32_bf16 v[128:131], a[132:135], v[100:103], v[128:131]\n\t" \
  "v_add_u32 v10, 4294950912, v10\n\t" \
  "v_mfma_f32_16x16x32_bf16 v[132:135], a[140:143], v[100:103], v[132:135]\n\t" \
  "v_add_u32 v11, 4294950912, v11\n\t" \
  "v_mfma_f32_16x16x32_bf16 v[132:135], a[136:139], v[96:99], v[132:135]\n\t" \
  "v_add_u32 v12, 4294901760, v12\n\t" \
  "v_mfma_f32_16x16x32_bf16 v[136:139], a[144:147], v[96:99], v[136:139]\n\t" \
  "v_add_u32 v13, 4294901760, v13\n\t" \
  "v_mfma_f32_16x16x32_bf16 v[136:139], a[148:151], v[100:103], v[136:139]\n\t" \
  "v_add_u32 v14, 4294901760, v14\n\t" \
  "v_mfma_f32_16x16x32_bf16 v[140:143], a[156:159], v[100:103], v[140:143]\n\t" \
  "v_add_u32 v15, 4294901760, v15\n\t" \
  "v_mfma_f32_16x16x32_bf16 v[140:143], a[152:155], v[96:99], v[140:143]\n\t" \
  "s_waitcnt vmcnt(18) lgkmcnt(0)\n\t" \
  "s_barrier\n\t" \
  "v_mfma_f32_16x16x32_bf16 v[144:147], a[160:163], v[96:99], v[144:147]\n\t" \
  "ds_read_b64_tr_b16 a[192:193], v12 offset:0\n\t" \
  "v_mfma_f32_16x16x32_bf16 v[144:147], a[164:167], v[100:103], v[144:147]\n\t" \
  "ds_read_b64_tr_b16 a[194:195], v12 offset:1024\n\t" \
  "v_mfma_f32_16x16x32_bf16 v[148:151], a[172:175], v[100:103], v[148:151]\n\t" \
  "ds_read_b64_tr_b16 a[196:197], v12 offset:8192\n\t" \
  "v_mfma_f32_16x16x32_bf16 v[148:151], a[168:171], v[96:99], v[148:151]\n\t" \
  "ds_read_b64_tr_b16 a[198:199], v12 offset:9216\n\t" \
  "s_add_u32 m0, s68, 32768\n\t" \
  "v_mfma_f32_16x16x32_bf16 v[152:155], a[176:179], v[96:99], v[152:155]\n\t" \
  "ds_read_b64_tr_b16 a[200:201], v13 offset:0\n\t" \
  "buffer_load_dwordx4 v0, s[36:39], s65 offen offset:0 sc0 sc1 lds\n\t" \
  "v_mfma_f32_16x16x32_bf16 v[152:155], a[180:183], v[100:103], v[152:155]\n\t" \
  "ds_read_b64_tr_b16 a[202:203], v13 offset:1024\n\t" \
  "v_mfma_f32_16x16x32_bf16 v[156:159], a[188:191], v[100:103], v[156:159]\n\t" \
  "ds_read_b64_tr_b16 a[204:205], v13 offset:8192\n\t" \
  "v_mfma_f32_16x16x32_bf16 v[156:159], a[184:187], v[96:99], v[156:159]\n\t" \
  "ds_read_b128 v[96:99], v10 offset:0\n\t" \
  "ds_read_b128 v[100:103], v11 offset:0\n\t" \
  "v_mfma_f32_16x16x32_bf16 v[188:191], a[184:187], v[104:107], v[188:191]\n\t" \
  "ds_read_b64_tr_b16 a[206:207], v13 offset:9216\n\t" \
  "buffer_load_dwordx4 v1, s[36:39], s65 offen offset:1024 sc0 sc1 lds\n\t" \
  "v_mfma_f32_16x16x32_bf16 v[188:191], a[188:191], v[108:111], v[188:191]\n\t" \
  "ds_read_b64_tr_b16 a[208:209], v14 offset:0\n\t" \
  "v_mfma_f32_16x16x32_bf16 v[184:187], a[180:183], v[108:111], v[184:187]\n\t" \
  "ds_read_b64_tr_b16 a[210:211], v14 offset:1024\n\t" \
  "v_mfma_f32_16x16x32_bf16 v[184:187], a[176:179], v[104:107], v[184:187]\n\t" \
  "ds_read_b64_tr_b16 a[212:213], v14 offset:8192\n\t" \
  "v_mfma_f32_16x16x32_bf16 v[180:183], a[168:171], v[104:107], v[180:183]\n\t" \
  "ds_read_b64_tr_b16 a[214:215], v14 offset:9216\n\t" \
  "v_mfma_f32_16x16x32_bf16 v[180:183], a[172:175], v[108:111], v[180:183]\n\t" \
  "ds_read_b64_tr_b16 a[216:217], v15 offset:0\n\t" \
  "buffer_load_dwordx4 v2, s[36:39], s65 offen offset:2048 sc0 sc1 lds\n\t" \
  "v_mfma_f32_16x16x32_bf16 v[176:179], a[164:167], v[108:111], v[176:179]\n\t" \
  "ds_read_b64_tr_b16 a[218:219], v15 offset:1024\n\t" \
  "v_mfma_f32_16x16x32_bf16 v[176:179], a[160:163], v[104:107], v[176:179]\n\t" \
  "ds_read_b64_tr_b16 a[220:221], v15 offset:8192\n\t" \
  "v_mfma_f32_16x16x32_bf16 v[172:175], a[152:155], v[104:107], v[172:175]\n\t" \
  "ds_read_b64_tr_b16 a[222:223], v15 offset:9216\n\t" \
  "v_mfma_f32_16x16x32_bf16 v[172:175], a[156:159], v[108:111], v[172:175]\n\t" \
  "ds_read_b64_tr_b16 a[224:225], v12 offset:16384\n\t" \
  "buffer_load_dwordx4 v3, s[36:39], s65 offen offset:3072 sc0 sc1 lds\n\t" \
  "v_mfma_f32_16x16x32_bf16 v[168:171], a[148:151], v[108:111], v[168:171]\n\t" \
  "ds_read_b64_tr_b16 a[226:227], v12 offset:17408\n\t" \
  "v_mfma_f32_16x16x32_bf16 v[168:171], a[144:147], v[104:107], v[168:171]\n\t" \
  "ds_read_b64_tr_b16 a[228:229], v12 offset:24576\n\t" \
  "v_mfma_f32_16x16x32_bf16 v[164:167], a[136:139], v[104:107], v[164:167]\n\t" \
  "ds_read_b64_tr_b16 a[230:231], v12 offset:25600\n\t" \
  "v_mfma_f32_16x16x32_bf16 v[164:167], a[140:143], v[108:111], v[164:167]\n\t" \
  "ds_read_b64_tr_b16 a[232:233], v13 offset:16384\n\t" \
  "s_add_u32 m0, s68, 114688\n\t" \
  "v_mfma_f32_16x16x32_bf16 v[160:163], a[132:135], v[108:111], v[160:163]\n\t" \
  "ds_read_b64_tr_b16 a[234:235], v13 offset:17408\n\t" \
  "buffer_load_dwordx4 v4, s[40:43], s78 offen offset:0 lds\n\t" \
  "v_mfma_f32_16x16x32_bf16 v[160:163], a[128:131], v[104:107], v[160:163]\n\t" \
  "ds_read_b128 v[104:107], v10 offset:2048\n\t" \
  "ds_read_b128 v[108:111], v11 offset:2048\n\t" \
  "v_mfma_f32_16x16x32_bf16 v[192:195], a[128:131], v[112:115], v[192:195]\n\t" \
  "ds_read_b64_tr_b16 a[236:237], v13 offset:24576\n\t" \
  "v_mfma_f32_16x16x32_bf16 v[192:195], a[132:135], v[116:119], v[192:195]\n\t" \
  "ds_read_b64_tr_b16 a[238:239], v13 offset:25600\n\t" \
  "v_mfma_f32_16x16x32_bf16 v[196:199], a[140:143], v[116:119], v[196:199]\n\t" \
  "ds_read_b64_tr_b16 a[240:241], v14 offset:16384\n\t" \
  "buffer_load_dwordx4 v5, s[40:43], s78 offen offset:1024 lds\n\t" \
  "v_mfma_f32_16x16x32_bf16 v[196:199], a[136:139], v[112:115], v[196:199]\n\t" \
  "ds_read_b64_tr_b16 a[242:243], v14 offset:17408\n\t" \
  "v_mfma_f32_16x16x32_bf16 v[200:203], a[144:147], v[112:115], v[200:203]\n\t" \
  "ds_read_b64_tr_b16 a[244:245], v14 offset:24576\n\t" \
  "v_mfma_f32_16x16x32_bf16 v[200:203], a[148:151], v[116:119], v[200:203]\n\t" \
  "ds_read_b64_tr_b16 a[246:247], v14 offset:25600\n\t" \
  "v_mfma_f32_16x16x32_bf16 v[204:207], a[156:159], v[116:119], v[204:207]\n\t" \
  "ds_read_b64_tr_b16 a[248:249], v15 offset:16384\n\t" \
  "v_mfma_f32_16x16x32_bf16 v[204:207], a[152:155], v[112:115], v[204:207]\n\t" \
  "ds_read_b64_tr_b16 a[250:251], v15 offset:17408\n\t" \
  "buffer_load_dwordx4 v6, s[40:43], s78 offen offset:2048 lds\n\t" \
  "v_mfma_f32_16x16x32_bf16 v[208:211], a[160:163], v[112:115], v[208:211]\n\t" \
  "ds_read_b64_tr_b16 a[252:253], v15 offset:24576\n\t" \
  "v_mfma_f32_16x16x32_bf16 v[208:211], a[164:167], v[116:119], v[208:211]\n\t" \
  "ds_read_b64_tr_b16 a[254:255], v15 offset:25600\n\t" \
  "v_mfma_f32_16x16x32_bf16 v[212:215], a[172:175], v[116:119], v[212:215]\n\t" \
  "v_mfma_f32_16x16x32_bf16 v[212:215], a[168:171], v[112:115], v[212:215]\n\t" \
  "buffer_load_dwordx4 v7, s[40:43], s78 offen offset:3072 lds\n\t" \
  "v_readlane_b32 s66, v17, 6\n\t" \
  "s_nop 0\n\t" \
  "s_lshl_b32 s78, s66, 13\n\t" \
  "s_lshl_b32 s65, s66, 10\n\t" \
  "s_add_u32 s79, s78, 256\n\t" \
  "s_add_u32 s65, s65, s70\n\t" \
  "s_nop 1\n\t" \
  "v_mfma_f32_16x16x32_bf16 v[216:219], a[176:179], v[112:115], v[216:219]\n\t" \
  "v_mfma_f32_16x16x32_bf16 v[216:219], a[180:183], v[116:119], v[216:219]\n\t" \
  "v_mfma_f32_16x16x32_bf16 v[220:223], a[188:191], v[116:119], v[220:223]\n\t" \
  "v_mfma_f32_16x16x32_bf16 v[220:223], a[184:187], v[112:115], v[220:223]\n\t" \
  "ds_read_b128 v[112:115], v10 offset:4096\n\t" \
  "ds_read_b128 v[116:119], v11 offset:4096\n\t" \
  "s_add_u32 m0, s68, 147456\n\t" \
  "v_mfma_f32_16x16x32_bf16 v[252:255], a[184:187], v[120:123], v[252:255]\n\t" \
  "buffer_load_dwordx4 v0, s[36:39], s65 offen offset:0 sc0 sc1 lds\n\t" \
  "v_mfma_f32_16x16x32_bf16 v[252:255], a[188:191], v[124:127], v[252:255]\n\t" \
  "v_mfma_f32_16x16x32_bf16 v[248:251], a[180:183], v[124:127], v[248:251]\n\t" \
  "v_mfma_f32_16x16x32_bf16 v[248:251], a[176:179], v[120:123], v[248:251]\n\t" \
  "v_mfma_f32_16x16x32_bf16 v[244:247], a[168:171], v[120:123], v[244:247]\n\t" \
  "buffer_load_dwordx4 v1, s[36:39], s65 offen offset:1024 sc0 sc1 lds\n\t" \
  "v_mfma_f32_16x16x32_bf16 v[244:247], a[172:175], v[124:127], v[244:247]\n\t" \
  "v_mfma_f32_16x16x32_bf16 v[240:243], a[164:167], v[124:127], v[240:243]\n\t" \
  "v_mfma_f32_16x16x32_bf16 v[240:243], a[160:163], v[120:123], v[240:243]\n\t" \
  "v_mfma_f32_16x16x32_bf16 v[236:239], a[152:155], v[120:123], v[236:239]\n\t" \
  "v_mfma_f32_16x16x32_bf16 v[236:239], a[156:159], v[124:127], v[236:239]\n\t" \
  "buffer_load_dwordx4 v2, s[36:39], s65 offen offset:2048 sc0 sc1 lds\n\t" \
  "v_mfma_f32_16x16x32_bf16 v[232:235], a[148:151], v[124:127], v[232:235]\n\t" \
  "v_mfma_f32_16x16x32_bf16 v[232:235], a[144:147], v[120:123], v[232:235]\n\t" \
  "v_mfma_f32_16x16x32_bf16 v[228:231], a[136:139], v[120:123], v[228:231]\n\t" \
  "v_mfma_f32_16x16x32_bf16 v[228:231], a[140:143], v[124:127], v[228:231]\n\t" \
  "v_mfma_f32_16x16x32_bf16 v[224:227], a[132:135], v[124:127], v[224:227]\n\t" \
  "buffer_load_dwordx4 v3, s[36:39], s65 offen offset:3072 sc0 sc1 lds\n\t" \
  "v_mfma_f32_16x16x32_bf16 v[224:227], a[128:131], v[120:123], v[224:227]\n\t" \
  "ds_read_b128 v[120:123], v10 offset:6144\n\t" \
  "ds_read_b128 v[124:127], v11 offset:6144\n\t" \
  "s_waitcnt lgkmcnt(4)\n\t" \
  "v_mfma_f32_16x16x32_bf16 v[128:131], a[192:195], v[96:99], v[128:131]\n\t" \
  "v_mfma_f32_16x16x32_bf16 v[128:131], a[196:199], v[100:103], v[128:131]\n\t" \
  "v_add_u32 v10, 4294950912, v10\n\t" \
  "v_mfma_f32_16x16x32_bf16 v[132:135], a[204:207], v[100:103], v[132:135]\n\t" \
  "v_add_u32 v11, 4294950912, v11\n\t" \
  "v_mfma_f32_16x16x32_bf16 v[132:135], a[200:203], v[96:99], v[132:135]\n\t" \
  "v_add_u32 v12, 32768, v12\n\t" \
  "v_mfma_f32_16x16x32_bf16 v[136:139], a[208:211], v[96:99], v[136:139]\n\t" \
  "v_add_u32 v13, 32768, v13\n\t" \
  "v_mfma_f32_16x16x32_bf16 v[136:139], a[212:215], v[100:103], v[136:139]\n\t" \
  "v_add_u32 v14, 32768, v14\n\t" \
  "v_mfma_f32_16x16x32_bf16 v[140:143], a[220:223], v[100:103], v[140:143]\n\t" \
  "v_add_u32 v15, 32768, v15\n\t" \
  "v_mfma_f32_16x16x32_bf16 v[140:143], a[216:219], v[96:99], v[140:143]\n\t" \
  "s_waitcnt vmcnt(16) lgkmcnt(0)\n\t" \
  "s_barrier\n\t" \
  "s_cmp_lg_u32 s71, 0\n\t" \
  "s_cbranch_scc1 XS167_%=\n\t" \
  "v_mov_b32 v50, s85\n\t" \
  "s_mov_b64 exec, 1\n\t" \
  "global_store_dword v18, v50, s[50:51] sc0 sc1\n\t" \
  "s_mov_b64 exec, -1\n\t" \
  "XS167_%=:\n\t" \
  "v_mfma_f32_16x16x32_bf16 v[144:147], a[224:227], v[96:99], v[144:147]\n\t" \
  "ds_read_b64_tr_b16 a[128:129], v12 offset:0\n\t" \
  "v_mfma_f32_16x16x32_bf16 v[144:147], a[228:231], v[100:103], v[144:147]\n\t" \
  "ds_read_b64_tr_b16 a[130:131], v12 offset:1024\n\t" \
  "global_load_dword v49, v19, s[50:51] sc0 sc1\n\t" \
  "v_mfma_f32_16x16x32_bf16 v[148:151], a[236:239], v[100:103], v[148:151]\n\t" \
  "ds_read_b64_tr_b16 a[132:133], v12 offset:8192\n\t" \
  "v_mfma_f32_16x16x32_bf16 v[148:151], a[232:235], v[96:99], v[148:151]\n\t" \
  "ds_read_b64_tr_b16 a[134:135], v12 offset:9216\n\t" \
  "s_add_u32 m0, s68, 16384\n\t" \
  "v_mfma_f32_16x16x32_bf16 v[152:155], a[240:243], v[96:99], v[152:155]\n\t" \
  "ds_read_b64_tr_b16 a[136:137], v13 offset:0\n\t" \
  "buffer_load_dwordx4 v4, s[40:43], s78 offen offset:0 lds\n\t" \
  "v_mfma_f32_16x16x32_bf16 v[152:155], a[244:247], v[100:103], v[152:155]\n\t" \
  "ds_read_b64_tr_b16 a[138:139], v13 offset:1024\n\t" \
  "v_mfma_f32_16x16x32_bf16 v[156:159], a[252:255], v[100:103], v[156:159]\n\t" \
  "ds_read_b64_tr_b16 a[140:141], v13 offset:8192\n\t" \
  "v_mfma_f32_16x16x32_bf16 v[156:159], a[248:251], v[96:99], v[156:159]\n\t" \
  "ds_read_b128 v[96:99], v10 offset:0\n\t" \
  "ds_read_b128 v[100:103], v11 offset:0\n\t" \
  "v_mfma_f32_16x16x32_bf16 v[188:191], a[248:251], v[104:107], v[188:191]\n\t" \
  "ds_read_b64_tr_b16 a[142:143], v13 offset:9216\n\t" \
  "buffer_load_dwordx4 v5, s[40:43], s78 offen offset:1024 lds\n\t" \
  "v_mfma_f32_16x16x32_bf16 v[188:191], a[252:255], v[108:111], v[188:191]\n\t" \
  "ds_read_b64_tr_b16 a[144:145], v14 offset:0\n\t" \
  "v_mfma_f32_16x16x32_bf16 v[184:187], a[244:247], v[108:111], v[184:187]\n\t" \
  "ds_read_b64_tr_b16 a[146:147], v14 offset:1024\n\t" \
  "v_mfma_f32_16x16x32_bf16 v[184:187], a[240:243], v[104:107], v[184:187]\n\t" \
  "ds_read_b64_tr_b16 a[148:149], v14 offset:8192\n\t" \
  "v_mfma_f32_16x16x32_bf16 v[180:183], a[232:235], v[104:107], v[180:183]\n\t" \
  "ds_read_b64_tr_b16 a[150:151], v14 offset:9216\n\t" \
  "v_mfma_f32_16x16x32_bf16 v[180:183], a[236:239], v[108:111], v[180:183]\n\t" \
  "ds_read_b64_tr_b16 a[152:153], v15 offset:0\n\t" \
  "buffer_load_dwordx4 v6, s[40:43], s78 offen offset:2048 lds\n\t" \
  "v_mfma_f32_16x16x32_bf16 v[176:179], a[228:231], v[108:111], v[176:179]\n\t" \
  "ds_read_b64_tr_b16 a[154:155], v15 offset:1024\n\t" \
  "v_mfma_f32_16x16x32_bf16 v[176:179], a[224:227], v[104:107], v[176:179]\n\t" \
  "ds_read_b64_tr_b16 a[156:157], v15 offset:8192\n\t" \
  "v_mfma_f32_16x16x32_bf16 v[172:175], a[216:219], v[104:107], v[172:175]\n\t" \
  "ds_read_b64_tr_b16 a[158:159], v15 offset:9216\n\t" \
  "v_mfma_f32_16x16x32_bf16 v[172:175], a[220:223], v[108:111], v[172:175]\n\t" \
  "ds_read_b64_tr_b16 a[160:161], v12 offset:16384\n\t" \
  "buffer_load_dwordx4 v7, s[40:43], s78 offen offset:3072 lds\n\t" \
  "v_mfma_f32_16x16x32_bf16 v[168:171], a[212:215], v[108:111], v[168:171]\n\t" \
  "ds_read_b64_tr_b16 a[162:163], v12 offset:17408\n\t" \
  "v_mfma_f32_16x16x32_bf16 v[168:171], a[208:211], v[104:107], v[168:171]\n\t" \
  "ds_read_b64_tr_b16 a[164:165], v12 offset:24576\n\t" \
  "v_mfma_f32_16x16x32_bf16 v[164:167], a[200:203], v[104:107], v[164:167]\n\t" \
  "ds_read_b64_tr_b16 a[166:167], v12 offset:25600\n\t" \
  "v_mfma_f32_16x16x32_bf16 v[164:167], a[204:207], v[108:111], v[164:167]\n\t" \
  "ds_read_b64_tr_b16 a[168:169], v13 offset:16384\n\t" \
  "s_add_u32 m0, s68, 49152\n\t" \
  "v_mfma_f32_16x16x32_bf16 v[160:163], a[196:199], v[108:111], v[160:163]\n\t" \
  "ds_read_b64_tr_b16 a[170:171], v13 offset:17408\n\t" \
  "buffer_load_dwordx4 v4, s[40:43], s79 offen offset:0 lds\n\t" \
  "v_mfma_f32_16x16x32_bf16 v[160:163], a[192:195], v[104:107], v[160:163]\n\t" \
  "ds_read_b128 v[104:107], v10 offset:2048\n\t" \
  "ds_read_b128 v[108:111], v11 offset:2048\n\t" \
  "v_mfma_f32_16x16x32_bf16 v[192:195], a[192:195], v[112:115], v[192:195]\n\t" \
  "ds_read_b64_tr_b16 a[172:173], v13 offset:24576\n\t" \
  "v_mfma_f32_16x16x32_bf16 v[192:195], a[196:199], v[116:119], v[192:195]\n\t" \
  "ds_read_b64_tr_b16 a[174:175], v13 offset:25600\n\t" \
  "v_mfma_f32_16x16x32_bf16 v[196:199], a[204:207], v[116:119], v[196:199]\n\t" \
  "ds_read_b64_tr_b16 a[176:177], v14 offset:16384\n\t" \
  "buffer_load_dwordx4 v5, s[40:43], s79 offen offset:1024 lds\n\t" \
  "v_mfma_f32_16x16x32_bf16 v[196:199], a[200:203], v[112:115], v[196:199]\n\t" \
  "ds_read_b64_tr_b16 a[178:179], v14 offset:17408\n\t" \
  "v_mfma_f32_16x16x32_bf16 v[200:203], a[208:211], v[112:115], v[200:203]\n\t" \
  "ds_read_b64_tr_b16 a[180:181], v14 offset:24576\n\t" \
  "v_mfma_f32_16x16x32_bf16 v[200:203], a[212:215], v[116:119], v[200:203]\n\t" \
  "ds_read_b64_tr_b16 a[182:183], v14 offset:25600\n\t" \
  "v_mfma_f32_16x16x32_bf16 v[204:207], a[220:223], v[116:119], v[204:207]\n\t" \
  "ds_read_b64_tr_b16 a[184:185], v15 offset:16384\n\t" \
  "v_mfma_f32_16x16x32_bf16 v[204:207], a[216:219], v[112:115], v[204:207]\n\t" \
  "ds_read_b64_tr_b16 a[186:187], v15 offset:17408\n\t" \
  "buffer_load_dwordx4 v6, s[40:43], s79 offen offset:2048 lds\n\t" \
  "v_mfma_f32_16x16x32_bf16 v[208:211], a[224:227], v[112:115], v[208:211]\n\t" \
  "ds_read_b64_tr_b16 a[188:189], v15 offset:24576\n\t" \
  "v_mfma_f32_16x16x32_bf16 v[208:211], a[228:231], v[116:119], v[208:211]\n\t" \
  "ds_read_b64_tr_b16 a[190:191], v15 offset:25600\n\t" \
  "v_mfma_f32_16x16x32_bf16 v[212:215], a[236:239], v[116:119], v[212:215]\n\t" \
  "v_mfma_f32_16x16x32_bf16 v[212:215], a[232:235], v[112:115], v[212:215]\n\t" \
  "buffer_load_dwordx4 v7, s[40:43], s79 offen offset:3072 lds\n\t" \
  "v_readlane_b32 s66, v17, 7\n\t" \
  "s_nop 0\n\t" \
  "s_lshl_b32 s78, s66, 13\n\t" \
  "s_lshl_b32 s65, s66, 10\n\t" \
  "s_add_u32 s79, s78, 256\n\t" \
  "s_add_u32 s65, s65, s70\n\t" \
  "s_nop 1\n\t" \
  "v_mfma_f32_16x16x32_bf16 v[216:219], a[240:243], v[112:115], v[216:219]\n\t" \
  "v_mfma_f32_16x16x32_bf16 v[216:219], a[244:247], v[116:119], v[216:219]\n\t" \
  "v_mfma_f32_16x16x32_bf16 v[220:223], a[252:255], v[116:119], v[220:223]\n\t" \
  "v_mfma_f32_16x16x32_bf16 v[220:223], a[248:251], v[112:115], v[220:223]\n\t" \
  "ds_read_b128 v[112:115], v10 offset:4096\n\t" \
  "ds_read_b128 v[116:119], v11 offset:4096\n\t" \
  "s_add_u32 m0, s68, 65536\n\t" \
  "v_mfma_f32_16x16x32_bf16 v[252:255], a[248:251], v[120:123], v[252:255]\n\t" \
  "buffer_load_dwordx4 v4, s[40:43], s78 offen offset:0 lds\n\t" \
  "v_mfma_f32_16x16x32_bf16 v[252:255], a[252:255], v[124:127], v[252:255]\n\t" \
  "v_mfma_f32_16x16x32_bf16 v[248:251], a[244:247], v[124:127], v[248:251]\n\t" \
  "v_mfma_f32_16x16x32_bf16 v[248:251], a[240:243], v[120:123], v[248:251]\n\t" \
  "v_mfma_f32_16x16x32_bf16 v[244:247], a[232:235], v[120:123], v[244:247]\n\t" \
  "buffer_load_dwordx4 v5, s[40:43], s78 offen offset:1024 lds\n\t" \
  "v_cmp_le_u32 s[92:93], s86, v37\n\t" \
  "v_mfma_f32_16x16x32_bf16 v[244:247], a[236:239], v[124:127], v[244:247]\n\t" \
  "v_mfma_f32_16x16x32_bf16 v[240:243], a[228:231], v[124:127], v[240:243]\n\t" \
  "v_mfma_f32_16x16x32_bf16 v[240:243], a[224:227], v[120:123], v[240:243]\n\t" \
  "s_or_b64 s[58:59], s[58:59], s[92:93]\n\t" \
  "v_mfma_f32_16x16x32_bf16 v[236:239], a[216:219], v[120:123], v[236:239]\n\t" \
  "v_mfma_f32_16x16x32_bf16 v[236:239], a[220:223], v[124:127], v[236:239]\n\t" \
  "buffer_load_dwordx4 v6, s[40:43], s78 offen offset:2048 lds\n\t" \
  "s_bitcmp1_b64 s[58:59], 7\n\t" \
  "s_cbranch_scc0 SLOW169_%=\n\t" \
  "RET168_%=:\n\t" \
  "v_mfma_f32_16x16x32_bf16 v[232:235], a[212:215], v[124:127], v[232:235]\n\t" \
  "v_mfma_f32_16x16x32_bf16 v[232:235], a[208:211], v[120:123], v[232:235]\n\t" \
  "v_mfma_f32_16x16x32_bf16 v[228:231], a[200:203], v[120:123], v[228:231]\n\t" \
  "v_mfma_f32_16x16x32_bf16 v[228:231], a[204:207], v[124:127], v[228:231]\n\t" \
  "v_mfma_f32_16x16x32_bf16 v[224:227], a[196:199], v[124:127], v[224:227]\n\t" \
  "buffer_load_dwordx4 v7, s[40:43], s78 offen offset:3072 lds\n\t" \
  "v_mfma_f32_16x16x32_bf16 v[224:227], a[192:195], v[120:123], v[224:227]\n\t" \
  "ds_read_b128 v[120:123], v10 offset:6144\n\t" \
  "ds_read_b128 v[124:127], v11 offset:6144\n\t" \
  "s_waitcnt lgkmcnt(4)\n\t" \
  "v_mfma_f32_16x16x32_bf16 v[128:131], a[128:131], v[96:99], v[128:131]\n\t" \
  "v_mfma_f32_16x16x32_bf16 v[128:131], a[132:135], v[100:103], v[128:131]\n\t" \
  "v_add_u32 v10, 32768, v10\n\t" \
  "v_mfma_f32_16x16x32_bf16 v[132:135], a[140:143], v[100:103], v[132:135]\n\t" \
  "v_add_u32 v11, 32768, v11\n\t" \
  "v_mfma_f32_16x16x32_bf16 v[132:135], a[136:139], v[96:99], v[132:135]\n\t" \
  "v_add_u32 v12, 32768, v12\n\t" \
  "v_mfma_f32_16x16x32_bf16 v[136:139], a[144:147], v[96:99], v[136:139]\n\t" \
  "v_add_u32 v13, 32768, v13\n\t" \
  "v_mfma_f32_16x16x32_bf16 v[136:139], a[148:151], v[100:103], v[136:139]\n\t" \
  "v_add_u32 v14, 32768, v14\n\t" \
  "v_mfma_f32_16x16x32_bf16 v[140:143], a[156:159], v[100:103], v[140:143]\n\t" \
  "v_add_u32 v15, 32768, v15\n\t" \
  "v_mfma_f32_16x16x32_bf16 v[140:143], a[152:155], v[96:99], v[140:143]\n\t" \
  "s_waitcnt vmcnt(17) lgkmcnt(0)\n\t" \
  "s_barrier\n\t" \
  "v_mfma_f32_16x16x32_bf16 v[144:147], a[160:163], v[96:99], v[144:147]\n\t" \
  "ds_read_b64_tr_b16 a[192:193], v12 offset:0\n\t" \
  "v_mfma_f32_16x16x32_bf16 v[144:147], a[164:167], v[100:103], v[144:147]\n\t" \
  "ds_read_b64_tr_b16 a[194:195], v12 offset:1024\n\t" \
  "v_mfma_f32_16x16x32_bf16 v[148:151], a[172:175], v[100:103], v[148:151]\n\t" \
  "ds_read_b64_tr_b16 a[196:197], v12 offset:8192\n\t" \
  "v_mfma_f32_16x16x32_bf16 v[148:151], a[168:171], v[96:99], v[148:151]\n\t" \
  "ds_read_b64_tr_b16 a[198:199], v12 offset:9216\n\t" \
  "v_mfma_f32_16x16x32_bf16 v[152:155], a[176:179], v[96:99], v[152:155]\n\t" \
  "ds_read_b64_tr_b16 a[200:201], v13 offset:0\n\t" \
  "v_mfma_f32_16x16x32_bf16 v[152:155], a[180:183], v[100:103], v[152:155]\n\t" \
  "ds_read_b64_tr_b16 a[202:203], v13 offset:1024\n\t" \
  "v_mfma_f32_16x16x32_bf16 v[156:159], a[188:191], v[100:103], v[156:159]\n\t" \
  "ds_read_b64_tr_b16 a[204:205], v13 offset:8192\n\t" \
  "v_mfma_f32_16x16x32_bf16 v[156:159], a[184:187], v[96:99], v[156:159]\n\t" \
  "ds_read_b128 v[96:99], v10 offset:0\n\t" \
  "ds_read_b128 v[100:103], v11 offset:0\n\t" \
  "v_mfma_f32_16x16x32_bf16 v[188:191], a[184:187], v[104:107], v[188:191]\n\t" \
  "ds_read_b64_tr_b16 a[206:207], v13 offset:9216\n\t" \
  "v_mfma_f32_16x16x32_bf16 v[188:191], a[188:191], v[108:111], v[188:191]\n\t" \
  "ds_read_b64_tr_b16 a[208:209], v14 offset:0\n\t" \
  "v_mfma_f32_16x16x32_bf16 v[184:187], a[180:183], v[108:111], v[184:187]\n\t" \
  "ds_read_b64_tr_b16 a[210:211], v14 offset:1024\n\t" \
  "v_mfma_f32_16x16x32_bf16 v[184:187], a[176:179], v[104:107], v[184:187]\n\t" \
  "ds_read_b64_tr_b16 a[212:213], v14 offset:8192\n\t" \
  "v_mfma_f32_16x16x32_bf16 v[180:183], a[168:171], v[104:107], v[180:183]\n\t" \
  "ds_read_b64_tr_b16 a[214:215], v14 offset:9216\n\t" \
  "v_mfma_f32_16x16x32_bf16 v[180:183], a[172:175], v[108:111], v[180:183]\n\t" \
  "ds_read_b64_tr_b16 a[216:217], v15 offset:0\n\t" \
  "v_mfma_f32_16x16x32_bf16 v[176:179], a[164:167], v[108:111], v[176:179]\n\t" \
  "ds_read_b64_tr_b16 a[218:219], v15 offset:1024\n\t" \
  "v_mfma_f32_16x16x32_bf16 v[176:179], a[160:163], v[104:107], v[176:179]\n\t" \
  "ds_read_b64_tr_b16 a[220:221], v15 offset:8192\n\t" \
  "v_mfma_f32_16x16x32_bf16 v[172:175], a[152:155], v[104:107], v[172:175]\n\t" \
  "ds_read_b64_tr_b16 a[222:223], v15 offset:9216\n\t" \
  "v_mfma_f32_16x16x32_bf16 v[172:175], a[156:159], v[108:111], v[172:175]\n\t" \
  "ds_read_b64_tr_b16 a[224:225], v12 offset:16384\n\t" \
  "v_mfma_f32_16x16x32_bf16 v[168:171], a[148:151], v[108:111], v[168:171]\n\t" \
  "ds_read_b64_tr_b16 a[226:227], v12 offset:17408\n\t" \
  "v_mfma_f32_16x16x32_bf16 v[168:171], a[144:147], v[104:107], v[168:171]\n\t" \
  "ds_read_b64_tr_b16 a[228:229], v12 offset:24576\n\t" \
  "v_mfma_f32_16x16x32_bf16 v[164:167], a[136:139], v[104:107], v[164:167]\n\t" \
  "ds_read_b64_tr_b16 a[230:231], v12 offset:25600\n\t" \
  "v_mfma_f32_16x16x32_bf16 v[164:167], a[140:143], v[108:111], v[164:167]\n\t" \
  "ds_read_b64_tr_b16 a[232:233], v13 offset:16384\n\t" \
  "v_mfma_f32_16x16x32_bf16 v[160:163], a[132:135], v[108:111], v[160:163]\n\t" \
  "ds_read_b64_tr_b16 a[234:235], v13 offset:17408\n\t" \
  "v_mfma_f32_16x16x32_bf16 v[160:163], a[128:131], v[104:107], v[160:163]\n\t" \
  "ds_read_b128 v[104:107], v10 offset:2048\n\t" \
  "ds_read_b128 v[108:111], v11 offset:2048\n\t" \
  "v_mfma_f32_16x16x32_bf16 v[192:195], a[128:131], v[112:115], v[192:195]\n\t" \
  "ds_read_b64_tr_b16 a[236:237], v13 offset:24576\n\t" \
  "v_mfma_f32_16x16x32_bf16 v[192:195], a[132:135], v[116:119], v[192:195]\n\t" \
  "ds_read_b64_tr_b16 a[238:239], v13 offset:25600\n\t" \
  "v_mfma_f32_16x16x32_bf16 v[196:199], a[140:143], v[116:119], v[196:199]\n\t" \
  "ds_read_b64_tr_b16 a[240:241], v14 offset:16384\n\t" \
  "v_mfma_f32_16x16x32_bf16 v[196:199], a[136:139], v[112:115], v[196:199]\n\t" \
  "ds_read_b64_tr_b16 a[242:243], v14 offset:17408\n\t" \
  "v_mfma_f32_16x16x32_bf16 v[200:203], a[144:147], v[112:115], v[200:203]\n\t" \
  "ds_read_b64_tr_b16 a[244:245], v14 offset:24576\n\t" \
  "v_mfma_f32_16x16x32_bf16 v[200:203], a[148:151], v[116:119], v[200:203]\n\t" \
  "ds_read_b64_tr_b16 a[246:247], v14 offset:25600\n\t" \
  "v_mfma_f32_16x16x32_bf16 v[204:207], a[156:159], v[116:119], v[204:207]\n\t" \
  "ds_read_b64_tr_b16 a[248:249], v15 offset:16384\n\t" \
  "v_mfma_f32_16x16x32_bf16 v[204:207], a[152:155], v[112:115], v[204:207]\n\t" \
  "ds_read_b64_tr_b16 a[250:251], v15 offset:17408\n\t" \
  "s_add_u32 m0, s68, 0\n\t" \
  "v_mfma_f32_16x16x32_bf16 v[208:211], a[160:163], v[112:115], v[208:211]\n\t" \
  "ds_read_b64_tr_b16 a[252:253], v15 offset:24576\n\t" \
  "buffer_load_dwordx4 v0, s[36:39], s65 offen offset:0 sc0 sc1 lds\n\t" \
  "v_mfma_f32_16x16x32_bf16 v[208:211], a[164:167], v[116:119], v[208:211]\n\t" \
  "ds_read_b64_tr_b16 a[254:255], v15 offset:25600\n\t" \
  "v_mfma_f32_16x16x32_bf16 v[212:215], a[172:175], v[116:119], v[212:215]\n\t" \
  "v_mfma_f32_16x16x32_bf16 v[212:215], a[168:171], v[112:115], v[212:215]\n\t" \
  "buffer_load_dwordx4 v1, s[36:39], s65 offen offset:1024 sc0 sc1 lds\n\t" \
  "v_mfma_f32_16x16x32_bf16 v[216:219], a[176:179], v[112:115], v[216:219]\n\t" \
  "v_mfma_f32_16x16x32_bf16 v[216:219], a[180:183], v[116:119], v[216:219]\n\t" \
  "v_mfma_f32_16x16x32_bf16 v[220:223], a[188:191], v[116:119], v[220:223]\n\t" \
  "buffer_load_dwordx4 v2, s[36:39], s65 offen offset:2048 sc0 sc1 lds\n\t" \
  "v_mfma_f32_16x16x32_bf16 v[220:223], a[184:187], v[112:115], v[220:223]\n\t" \
  "ds_read_b128 v[112:115], v10 offset:4096\n\t" \
  "ds_read_b128 v[116:119], v11 offset:4096\n\t" \
  "v_mfma_f32_16x16x32_bf16 v[252:255], a[184:187], v[120:123], v[252:255]\n\t" \
  "v_mfma_f32_16x16x32_bf16 v[252:255], a[188:191], v[124:127], v[252:255]\n\t" \
  "buffer_load_dwordx4 v3, s[36:39], s65 offen offset:3072 sc0 sc1 lds\n\t" \
  "v_mfma_f32_16x16x32_bf16 v[248:251], a[180:183], v[124:127], v[248:251]\n\t" \
  "v_mfma_f32_16x16x32_bf16 v[248:251], a[176:179], v[120:123], v[248:251]\n\t" \
  "s_add_u32 m0, s68, 81920\n\t" \
  "v_mfma_f32_16x16x32_bf16 v[244:247], a[168:171], v[120:123], v[244:247]\n\t" \
  "buffer_load_dwordx4 v4, s[40:43], s79 offen offset:0 lds\n\t" \
  "v_mfma_f32_16x16x32_bf16 v[244:247], a[172:175], v[124:127], v[244:247]\n\t" \
  "v_mfma_f32_16x16x32_bf16 v[240:243], a[164:167], v[124:127], v[240:243]\n\t" \
  "v_mfma_f32_16x16x32_bf16 v[240:243], a[160:163], v[120:123], v[240:243]\n\t" \
  "buffer_load_dwordx4 v5, s[40:43], s79 offen offset:1024 lds\n\t" \
  "v_mfma_f32_16x16x32_bf16 v[236:239], a[152:155], v[120:123], v[236:239]\n\t" \
  "v_mfma_f32_16x16x32_bf16 v[236:239], a[156:159], v[124:127], v[236:239]\n\t" \
  "v_mfma_f32_16x16x32_bf16 v[232:235], a[148:151], v[124:127], v[232:235]\n\t" \
  "buffer_load_dwordx4 v6, s[40:43], s79 offen offset:2048 lds\n\t" \
  "v_mfma_f32_16x16x32_bf16 v[232:235], a[144:147], v[120:123], v[232:235]\n\t" \
  "v_mfma_f32_16x16x32_bf16 v[228:231], a[136:139], v[120:123], v[228:231]\n\t" \
  "v_mfma_f32_16x16x32_bf16 v[228:231], a[140:143], v[124:127], v[228:231]\n\t" \
  "v_mfma_f32_16x16x32_bf16 v[224:227], a[132:135], v[124:127], v[224:227]\n\t" \
  "buffer_load_dwordx4 v7, s[40:43], s79 offen offset:3072 lds\n\t" \
  "v_mfma_f32_16x16x32_bf16 v[224:227], a[128:131], v[120:123], v[224:227]\n\t" \
  "ds_read_b128 v[120:123], v10 offset:6144\n\t" \
  "ds_read_b128 v[124:127], v11 offset:6144\n\t" \
  "s_waitcnt lgkmcnt(4)\n\t" \
  "v_mfma_f32_16x16x32_bf16 v[128:131], a[192:195], v[96:99], v[128:131]\n\t" \
  "v_mfma_f32_16x16x32_bf16 v[128:131], a[196:199], v[100:103], v[128:131]\n\t" \
  "v_add_u32 v10, 114688, v10\n\t" \
  "v_mfma_f32_16x16x32_bf16 v[132:135], a[204:207], v[100:103], v[132:135]\n\t" \
  "v_add_u32 v11, 114688, v11\n\t" \
  "v_mfma_f32_16x16x32_bf16 v[132:135], a[200:203], v[96:99], v[132:135]\n\t" \
  "v_add_u32 v12, 4294868992, v12\n\t" \
  "v_mfma_f32_16x16x32_bf16 v[136:139], a[208:211], v[96:99], v[136:139]\n\t" \
  "v_add_u32 v13, 4294868992, v13\n\t" \
  "v_mfma_f32_16x16x32_bf16 v[136:139], a[212:215], v[100:103], v[136:139]\n\t" \
  "v_add_u32 v14, 4294868992, v14\n\t" \
  "v_mfma_f32_16x16x32_bf16 v[140:143], a[220:223], v[100:103], v[140:143]\n\t" \
  "v_add_u32 v15, 4294868992, v15\n\t" \
  "v_mfma_f32_16x16x32_bf16 v[140:143], a[216:219], v[96:99], v[140:143]\n\t" \
  "s_waitcnt vmcnt(12) lgkmcnt(0)\n\t" \
  "s_barrier\n\t" \
  "v_readfirstlane_b32 s99, v49\n\t" \
  "s_nop 1\n\t" \
  "s_cmp_ge_u32 s99, s85\n\t" \
  "s_cbranch_scc0 SLOW173_%=\n\t" \
  "RET172_%=:\n\t" \
  "v_mfma_f32_16x16x32_bf16 v[144:147], a[224:227], v[96:99], v[144:147]\n\t" \
  "ds_read_b64_tr_b16 a[128:129], v12 offset:0\n\t" \
  "v_mfma_f32_16x16x32_bf16 v[144:147], a[228:231], v[100:103], v[144:147]\n\t" \
  "ds_read_b64_tr_b16 a[130:131], v12 offset:1024\n\t" \
  "global_load_dword v48, v30, s[50:51] offset:-128 sc1\n\t" \
  "v_mfma_f32_16x16x32_bf16 v[148:151], a[236:239], v[100:103], v[148:151]\n\t" \
  "ds_read_b64_tr_b16 a[132:133], v12 offset:8192\n\t" \
  "v_mfma_f32_16x16x32_bf16 v[148:151], a[232:235], v[96:99], v[148:151]\n\t" \
  "ds_read_b64_tr_b16 a[134:135], v12 offset:9216\n\t" \
  "v_mfma_f32_16x16x32_bf16 v[152:155], a[240:243], v[96:99], v[152:155]\n\t" \
  "ds_read_b64_tr_b16 a[136:137], v13 offset:0\n\t" \
  "v_mfma_f32_16x16x32_bf16 v[152:155], a[244:247], v[100:103], v[152:155]\n\t" \
  "ds_read_b64_tr_b16 a[138:139], v13 offset:1024\n\t" \
  "v_mfma_f32_16x16x32_bf16 v[156:159], a[252:255], v[100:103], v[156:159]\n\t" \
  "ds_read_b64_tr_b16 a[140:141], v13 offset:8192\n\t" \
  "v_add_u32 v9, 4294959104, v9\n\t" \
  "v_xor_b32 v9, 0x10000, v9\n\t" \
  "global_load_dwordx4 a[0:3], v9, s[50:51] offset:-4096 sc0 sc1\n\t" \
  "v_mfma_f32_16x16x32_bf16 v[156:159], a[248:251], v[96:99], v[156:159]\n\t" \
  "ds_read_b128 v[96:99], v10 offset:0\n\t" \
  "ds_read_b128 v[100:103], v11 offset:0\n\t" \
  "v_mfma_f32_16x16x32_bf16 v[188:191], a[248:251], v[104:107], v[188:191]\n\t" \
  "ds_read_b64_tr_b16 a[142:143], v13 offset:9216\n\t" \
  "v_mfma_f32_16x16x32_bf16 v[188:191], a[252:255], v[108:111], v[188:191]\n\t" \
  "ds_read_b64_tr_b16 a[144:145], v14 offset:0\n\t" \
  "global_load_dwordx4 a[4:7], v9, s[50:51] offset:-3072 sc0 sc1\n\t" \
  "v_mfma_f32_16x16x32_bf16 v[184:187], a[244:247], v[108:111], v[184:187]\n\t" \
  "ds_read_b64_tr_b16 a[146:147], v14 offset:1024\n\t" \
  "v_mfma_f32_16x16x32_bf16 v[184:187], a[240:243], v[104:107], v[184:187]\n\t" \
  "ds_read_b64_tr_b16 a[148:149], v14 offset:8192\n\t" \
  "v_mfma_f32_16x16x32_bf16 v[180:183], a[232:235], v[104:107], v[180:183]\n\t" \
  "ds_read_b64_tr_b16 a[150:151], v14 offset:9216\n\t" \
  "global_load_dwordx4 a[8:11], v9, s[50:51] offset:-2048 sc0 sc1\n\t" \
  "v_mfma_f32_16x16x32_bf16 v[180:183], a[236:239], v[108:111], v[180:183]\n\t" \
  "ds_read_b64_tr_b16 a[152:153], v15 offset:0\n\t" \
  "v_mfma_f32_16x16x32_bf16 v[176:179], a[228:231], v[108:111], v[176:179]\n\t" \
  "ds_read_b64_tr_b16 a[154:155], v15 offset:1024\n\t" \
  "v_mfma_f32_16x16x32_bf16 v[176:179], a[224:227], v[104:107], v[176:179]\n\t" \
  "ds_read_b64_tr_b16 a[156:157], v15 offset:8192\n\t" \
  "global_load_dwordx4 a[12:15], v9, s[50:51] offset:-1024 sc0 sc1\n\t" \
  "v_mfma_f32_16x16x32_bf16 v[172:175], a[216:219], v[104:107], v[172:175]\n\t" \
  "ds_read_b64_tr_b16 a[158:159], v15 offset:9216\n\t" \
  "v_mfma_f32_16x16x32_bf16 v[172:175], a[220:223], v[108:111], v[172:175]\n\t" \
  "ds_read_b64_tr_b16 a[160:161], v12 offset:32768\n\t" \
  "v_mfma_f32_16x16x32_bf16 v[168:171], a[212:215], v[108:111], v[168:171]\n\t" \
  "ds_read_b64_tr_b16 a[162:163], v12 offset:33792\n\t" \
  "global_load_dwordx4 a[16:19], v9, s[50:51] offset:0 sc0 sc1\n\t" \
  "v_mfma_f32_16x16x32_bf16 v[168:171], a[208:211], v[104:107], v[168:171]\n\t" \
  "ds_read_b64_tr_b16 a[164:165], v12 offset:40960\n\t" \
  "v_mfma_f32_16x16x32_bf16 v[164:167], a[200:203], v[104:107], v[164:167]\n\t" \
  "ds_read_b64_tr_b16 a[166:167], v12 offset:41984\n\t" \
  "v_mfma_f32_16x16x32_bf16 v[164:167], a[204:207], v[108:111], v[164:167]\n\t" \
  "ds_read_b64_tr_b16 a[168:169], v13 offset:32768\n\t" \
  "global_load_dwordx4 a[20:23], v9, s[50:51] offset:1024 sc0 sc1\n\t" \
  "v_mfma_f32_16x16x32_bf16 v[160:163], a[196:199], v[108:111], v[160:163]\n\t" \
  "ds_read_b64_tr_b16 a[170:171], v13 offset:33792\n\t" \
  "v_mfma_f32_16x16x32_bf16 v[160:163], a[192:195], v[104:107], v[160:163]\n\t" \
  "ds_read_b128 v[104:107], v10 offset:2048\n\t" \
  "ds_read_b128 v[108:111], v11 offset:2048\n\t" \
  "v_mfma_f32_16x16x32_bf16 v[192:195], a[192:195], v[112:115], v[192:195]\n\t" \
  "ds_read_b64_tr_b16 a[172:173], v13 offset:40960\n\t" \
  "global_load_dwordx4 a[24:27], v9, s[50:51] offset:2048 sc0 sc1\n\t" \
  "v_mfma_f32_16x16x32_bf16 v[192:195], a[196:199], v[116:119], v[192:195]\n\t" \
  "ds_read_b64_tr_b16 a[174:175], v13 offset:41984\n\t" \
  "v_mfma_f32_16x16x32_bf16 v[196:199], a[204:207], v[116:119], v[196:199]\n\t" \
  "ds_read_b64_tr_b16 a[176:177], v14 offset:32768\n\t" \
  "v_mfma_f32_16x16x32_bf16 v[196:199], a[200:203], v[112:115], v[196:199]\n\t" \
  "ds_read_b64_tr_b16 a[178:179], v14 offset:33792\n\t" \
  "global_load_dwordx4 a[28:31], v9, s[50:51] offset:3072 sc0 sc1\n\t" \
  "v_mfma_f32_16x16x32_bf16 v[200:203], a[208:211], v[112:115], v[200:203]\n\t" \
  "ds_read_b64_tr_b16 a[180:181], v14 offset:40960\n\t" \
  "v_mfma_f32_16x16x32_bf16 v[200:203], a[212:215], v[116:119], v[200:203]\n\t" \
  "ds_read_b64_tr_b16 a[182:183], v14 offset:41984\n\t" \
  "v_mfma_f32_16x16x32_bf16 v[204:207], a[220:223], v[116:119], v[204:207]\n\t" \
  "ds_read_b64_tr_b16 a[184:185], v15 offset:32768\n\t" \
  "v_add_u32 v9, 8192, v9\n\t" \
  "global_load_dwordx4 a[32:35], v9, s[50:51] offset:-4096 sc0 sc1\n\t" \
  "v_mfma_f32_16x16x32_bf16 v[204:207], a[216:219], v[112:115], v[204:207]\n\t" \
  "ds_read_b64_tr_b16 a[186:187], v15 offset:33792\n\t" \
  "v_mfma_f32_16x16x32_bf16 v[208:211], a[224:227], v[112:115], v[208:211]\n\t" \
  "ds_read_b64_tr_b16 a[188:189], v15 offset:40960\n\t" \
  "v_mfma_f32_16x16x32_bf16 v[208:211], a[228:231], v[116:119], v[208:211]\n\t" \
  "ds_read_b64_tr_b16 a[190:191], v15 offset:41984\n\t" \
  "global_load_dwordx4 a[36:39], v9, s[50:51] offset:-3072 sc0 sc1\n\t" \
  "v_mfma_f32_16x16x32_bf16 v[212:215], a[236:239], v[116:119], v[212:215]\n\t" \
  "v_mfma_f32_16x16x32_bf16 v[212:215], a[232:235], v[112:115], v[212:215]\n\t" \
  "v_mfma_f32_16x16x32_bf16 v[216:219], a[240:243], v[112:115], v[216:219]\n\t" \
  "global_load_dwordx4 a[40:43], v9, s[50:51] offset:-2048 sc0 sc1\n\t" \
  "v_mfma_f32_16x16x32_bf16 v[216:219], a[244:247], v[116:119], v[216:219]\n\t" \
  "v_mfma_f32_16x16x32_bf16 v[220:223], a[252:255], v[116:119], v[220:223]\n\t" \
  "v_mfma_f32_16x16x32_bf16 v[220:223], a[248:251], v[112:115], v[220:223]\n\t" \
  "ds_read_b128 v[112:115], v10 offset:4096\n\t" \
  "ds_read_b128 v[116:119], v11 offset:4096\n\t" \
  "global_load_dwordx4 a[44:47], v9, s[50:51] offset:-1024 sc0 sc1\n\t" \
  "v_mfma_f32_16x16x32_bf16 v[252:255], a[248:251], v[120:123], v[252:255]\n\t" \
  "v_mfma_f32_16x16x32_bf16 v[252:255], a[252:255], v[124:127], v[252:255]\n\t" \
  "v_mfma_f32_16x16x32_bf16 v[248:251], a[244:247], v[124:127], v[248:251]\n\t" \
  "global_load_dwordx4 a[48:51], v9, s[50:51] offset:0 sc0 sc1\n\t" \
  "v_mfma_f32_16x16x32_bf16 v[248:251], a[240:243], v[120:123], v[248:251]\n\t" \
  "v_mfma_f32_16x16x32_bf16 v[244:247], a[232:235], v[120:123], v[244:247]\n\t" \
  "v_mfma_f32_16x16x32_bf16 v[244:247], a[236:239], v[124:127], v[244:247]\n\t" \
  "global_load_dwordx4 a[52:55], v9, s[50:51] offset:1024 sc0 sc1\n\t" \
  "v_mfma_f32_16x16x32_bf16 v[240:243], a[228:231], v[124:127], v[240:243]\n\t" \
  "v_mfma_f32_16x16x32_bf16 v[240:243], a[224:227], v[120:123], v[240:243]\n\t" \
  "v_mfma_f32_16x16x32_bf16 v[236:239], a[216:219], v[120:123], v[236:239]\n\t" \
  "global_load_dwordx4 a[56:59], v9, s[50:51] offset:2048 sc0 sc1\n\t" \
  "v_mfma_f32_16x16x32_bf16 v[236:239], a[220:223], v[124:127], v[236:239]\n\t" \
  "v_mfma_f32_16x16x32_bf16 v[232:235], a[212:215], v[124:127], v[232:235]\n\t" \
  "v_mfma_f32_16x16x32_bf16 v[232:235], a[208:211], v[120:123], v[232:235]\n\t" \
  "global_load_dwordx4 a[60:63], v9, s[50:51] offset:3072 sc0 sc1\n\t" \
  "v_mfma_f32_16x16x32_bf16 v[228:231], a[200:203], v[120:123], v[228:231]\n\t" \
  "v_mfma_f32_16x16x32_bf16 v[228:231], a[204:207], v[124:127], v[228:231]\n\t" \
  "v_mfma_f32_16x16x32_bf16 v[224:227], a[196:199], v[124:127], v[224:227]\n\t" \
  "v_mfma_f32_16x16x32_bf16 v[224:227], a[192:195], v[120:123], v[224:227]\n\t" \
  "ds_read_b128 v[120:123], v10 offset:6144\n\t" \
  "ds_read_b128 v[124:127], v11 offset:6144\n\t" \
  "s_waitcnt lgkmcnt(4)\n\t" \
  "v_mfma_f32_16x16x32_bf16 v[128:131], a[128:131], v[96:99], v[128:131]\n\t" \
  "v_mfma_f32_16x16x32_bf16 v[128:131], a[132:135], v[100:103], v[128:131]\n\t" \
  "v_add_u32 v10, 4294819840, v10\n\t" \
  "v_mfma_f32_16x16x32_bf16 v[132:135], a[140:143], v[100:103], v[132:135]\n\t" \
  "v_add_u32 v11, 4294819840, v11\n\t" \
  "v_mfma_f32_16x16x32_bf16 v[132:135], a[136:139], v[96:99], v[132:135]\n\t" \
  "v_add_u32 v12, 49152, v12\n\t" \
  "v_mfma_f32_16x16x32_bf16 v[136:139], a[144:147], v[96:99], v[136:139]\n\t" \
  "v_add_u32 v13, 49152, v13\n\t" \
  "v_mfma_f32_16x16x32_bf16 v[136:139], a[148:151], v[100:103], v[136:139]\n\t" \
  "v_add_u32 v14, 49152, v14\n\t" \
  "v_mfma_f32_16x16x32_bf16 v[140:143], a[156:159], v[100:103], v[140:143]\n\t" \
  "v_add_u32 v15, 49152, v15\n\t" \
  "v_mfma_f32_16x16x32_bf16 v[140:143], a[152:155], v[96:99], v[140:143]\n\t" \
  "s_waitcnt vmcnt(17) lgkmcnt(0)\n\t" \
  "s_barrier\n\t" \
  "v_mfma_f32_16x16x32_bf16 v[144:147], a[160:163], v[96:99], v[144:147]\n\t" \
  "ds_read_b128 v[64:67], v10 offset:0\n\t" \
  "ds_read_b128 v[68:71], v11 offset:0\n\t" \
  "v_mfma_f32_16x16x32_bf16 v[144:147], a[164:167], v[100:103], v[144:147]\n\t" \
  "ds_read_b64_tr_b16 a[192:193], v12 offset:0\n\t" \
  "ds_read_b64_tr_b16 a[194:195], v12 offset:1024\n\t" \
  "v_mfma_f32_16x16x32_bf16 v[148:151], a[172:175], v[100:103], v[148:151]\n\t" \
  "ds_read_b64_tr_b16 a[196:197], v12 offset:8192\n\t" \
  "ds_read_b64_tr_b16 a[198:199], v12 offset:9216\n\t" \
  "v_mfma_f32_16x16x32_bf16 v[148:151], a[168:171], v[96:99], v[148:151]\n\t" \
  "ds_read_b64_tr_b16 a[200:201], v13 offset:0\n\t" \
  "ds_read_b64_tr_b16 a[202:203], v13 offset:1024\n\t" \
  "v_mfma_f32_16x16x32_bf16 v[152:155], a[176:179], v[96:99], v[152:155]\n\t" \
  "ds_read_b64_tr_b16 a[204:205], v13 offset:8192\n\t" \
  "ds_read_b64_tr_b16 a[206:207], v13 offset:9216\n\t" \
  "v_mfma_f32_16x16x32_bf16 v[152:155], a[180:183], v[100:103], v[152:155]\n\t" \
  "ds_read_b64_tr_b16 a[208:209], v14 offset:0\n\t" \
  "ds_read_b64_tr_b16 a[210:211], v14 offset:1024\n\t" \
  "v_mfma_f32_16x16x32_bf16 v[156:159], a[188:191], v[100:103], v[156:159]\n\t" \
  "ds_read_b64_tr_b16 a[212:213], v14 offset:8192\n\t" \
  "ds_read_b64_tr_b16 a[214:215], v14 offset:9216\n\t" \
  "v_mfma_f32_16x16x32_bf16 v[156:159], a[184:187], v[96:99], v[156:159]\n\t" \
  "ds_read_b64_tr_b16 a[216:217], v15 offset:0\n\t" \
  "ds_read_b64_tr_b16 a[218:219], v15 offset:1024\n\t" \
  "v_mfma_f32_16x16x32_bf16 v[188:191], a[184:187], v[104:107], v[188:191]\n\t" \
  "ds_read_b64_tr_b16 a[220:221], v15 offset:8192\n\t" \
  "ds_read_b64_tr_b16 a[222:223], v15 offset:9216\n\t" \
  "v_mfma_f32_16x16x32_bf16 v[188:191], a[188:191], v[108:111], v[188:191]\n\t" \
  "ds_read_b64_tr_b16 a[224:225], v12 offset:16384\n\t" \
  "ds_read_b64_tr_b16 a[226:227], v12 offset:17408\n\t" \
  "v_mfma_f32_16x16x32_bf16 v[184:187], a[180:183], v[108:111], v[184:187]\n\t" \
  "ds_read_b64_tr_b16 a[228:229], v12 offset:24576\n\t" \
  "ds_read_b64_tr_b16 a[230:231], v12 offset:25600\n\t" \
  "v_mfma_f32_16x16x32_bf16 v[184:187], a[176:179], v[104:107], v[184:187]\n\t" \
  "ds_read_b64_tr_b16 a[232:233], v13 offset:16384\n\t" \
  "ds_read_b64_tr_b16 a[234:235], v13 offset:17408\n\t" \
  "v_mfma_f32_16x16x32_bf16 v[180:183], a[168:171], v[104:107], v[180:183]\n\t" \
  "ds_read_b64_tr_b16 a[236:237], v13 offset:24576\n\t" \
  "ds_read_b64_tr_b16 a[238:239], v13 offset:25600\n\t" \
  "v_mfma_f32_16x16x32_bf16 v[180:183], a[172:175], v[108:111], v[180:183]\n\t" \
  "ds_read_b64_tr_b16 a[240:241], v14 offset:16384\n\t" \
  "ds_read_b64_tr_b16 a[242:243], v14 offset:17408\n\t" \
  "v_mfma_f32_16x16x32_bf16 v[176:179], a[164:167], v[108:111], v[176:179]\n\t" \
  "ds_read_b64_tr_b16 a[244:245], v14 offset:24576\n\t" \
  "ds_read_b64_tr_b16 a[246:247], v14 offset:25600\n\t" \
  "v_mfma_f32_16x16x32_bf16 v[176:179], a[160:163], v[104:107], v[176:179]\n\t" \
  "ds_read_b64_tr_b16 a[248:249], v15 offset:16384\n\t" \
  "ds_read_b64_tr_b16 a[250:251], v15 offset:17408\n\t" \
  "v_mfma_f32_16x16x32_bf16 v[172:175], a[152:155], v[104:107], v[172:175]\n\t" \
  "ds_read_b64_tr_b16 a[252:253], v15 offset:24576\n\t" \
  "ds_read_b64_tr_b16 a[254:255], v15 offset:25600\n\t" \
  "v_mfma_f32_16x16x32_bf16 v[172:175], a[156:159], v[108:111], v[172:175]\n\t" \
  "ds_read_b128 v[72:75], v10 offset:2048\n\t" \
  "ds_read_b128 v[76:79], v11 offset:2048\n\t" \
  "v_mfma_f32_16x16x32_bf16 v[168:171], a[148:151], v[108:111], v[168:171]\n\t" \
  "ds_read_b128 v[80:83], v10 offset:4096\n\t" \
  "ds_read_b128 v[84:87], v11 offset:4096\n\t" \
  "v_mfma_f32_16x16x32_bf16 v[168:171], a[144:147], v[104:107], v[168:171]\n\t" \
  "ds_read_b128 v[88:91], v10 offset:6144\n\t" \
  "ds_read_b128 v[92:95], v11 offset:6144\n\t" \
  "v_mfma_f32_16x16x32_bf16 v[164:167], a[136:139], v[104:107], v[164:167]\n\t" \
  "v_mfma_f32_16x16x32_bf16 v[164:167], a[140:143], v[108:111], v[164:167]\n\t" \
  "v_mfma_f32_16x16x32_bf16 v[160:163], a[132:135], v[108:111], v[160:163]\n\t" \
  "v_mfma_f32_16x16x32_bf16 v[160:163], a[128:131], v[104:107], v[160:163]\n\t" \
  "s_waitcnt lgkmcnt(6)\n\t" \
  "v_mfma_f32_16x16x32_bf16 v[128:131], a[192:195], v[64:67], v[128:131]\n\t" \
  "s_waitcnt vmcnt(0)\n\t" \
  "v_mfma_f32_16x16x32_bf16 v[128:131], a[196:199], v[68:71], v[128:131]\n\t" \
  "v_add_u32 v27, 16384, v27\n\t" \
  "v_mfma_f32_16x16x32_bf16 v[128:131], v[42:45], a[0:3], v[128:131]\n\t" \
  "v_add_u32 v28, 16384, v28\n\t" \
  "v_mfma_f32_16x16x32_bf16 v[132:135], a[204:207], v[68:71], v[132:135]\n\t" \
  "v_xor_b32 v0, 0, v27\n\t" \
  "v_mfma_f32_16x16x32_bf16 v[132:135], a[200:203], v[64:67], v[132:135]\n\t" \
  "v_xor_b32 v1, 32, v27\n\t" \
  "v_mfma_f32_16x16x32_bf16 v[132:135], v[40:43], a[0:3], v[132:135]\n\t" \
  "v_xor_b32 v2, 64, v27\n\t" \
  "v_mfma_f32_16x16x32_bf16 v[136:139], a[208:211], v[64:67], v[136:139]\n\t" \
  "v_xor_b32 v3, 96, v27\n\t" \
  "v_mfma_f32_16x16x32_bf16 v[136:139], a[212:215], v[68:71], v[136:139]\n\t" \
  "v_xor_b32 v4, 128, v27\n\t" \
  "v_mfma_f32_16x16x32_bf16 v[136:139], v[42:45], a[4:7], v[136:139]\n\t" \
  "v_xor_b32 v5, 160, v27\n\t" \
  "v_mfma_f32_16x16x32_bf16 v[140:143], a[220:223], v[68:71], v[140:143]\n\t" \
  "v_xor_b32 v6, 192, v27\n\t" \
  "v_mfma_f32_16x16x32_bf16 v[140:143], a[216:219], v[64:67], v[140:143]\n\t" \
  "v_xor_b32 v7, 224, v27\n\t" \
  "v_mfma_f32_16x16x32_bf16 v[140:143], v[40:43], a[4:7], v[140:143]\n\t" \
  "v_xor_b32 v20, 0, v28\n\t" \
  "v_mfma_f32_16x16x32_bf16 v[144:147], a[224:227], v[64:67], v[144:147]\n\t" \
  "v_xor_b32 v21, 64, v28\n\t" \
  "v_mfma_f32_16x16x32_bf16 v[144:147], a[228:231], v[68:71], v[144:147]\n\t" \
  "v_xor_b32 v22, 128, v28\n\t" \
  "v_mfma_f32_16x16x32_bf16 v[144:147], v[42:45], a[8:11], v[144:147]\n\t" \
  "v_xor_b32 v23, 192, v28\n\t" \
  "v_mfma_f32_16x16x32_bf16 v[148:151], a[236:239], v[68:71], v[148:151]\n\t" \
  "v_mfma_f32_16x16x32_bf16 v[148:151], a[232:235], v[64:67], v[148:151]\n\t" \
  "v_readfirstlane_b32 s99, v48\n\t" \
  "v_mfma_f32_16x16x32_bf16 v[148:151], v[40:43], a[8:11], v[148:151]\n\t" \
  "v_mfma_f32_16x16x32_bf16 v[152:155], a[240:243], v[64:67], v[152:155]\n\t" \
  "s_waitcnt lgkmcnt(0)\n\t" \
  "v_mfma_f32_16x16x32_bf16 v[152:155], a[244:247], v[68:71], v[152:155]\n\t" \
  "v_mfma_f32_16x16x32_bf16 v[152:155], v[42:45], a[12:15], v[152:155]\n\t" \
  "s_cmp_eq_u32 s99, s85\n\t" \
  "v_mfma_f32_16x16x32_bf16 v[156:159], a[252:255], v[68:71], v[156:159]\n\t" \
  "v_mfma_f32_16x16x32_bf16 v[156:159], a[248:251], v[64:67], v[156:159]\n\t" \
  "s_cbranch_scc1 NANP_%=\n\t" \
  "v_mfma_f32_16x16x32_bf16 v[156:159], v[40:43], a[12:15], v[156:159]\n\t" \
  "v_mfma_f32_16x16x32_bf16 v[188:191], a[248:251], v[72:75], v[188:191]\n\t" \
  "v_mfma_f32_16x16x32_bf16 v[188:191], a[252:255], v[76:79], v[188:191]\n\t" \
  "v_mfma_f32_16x16x32_bf16 v[188:191], v[40:43], a[28:31], v[188:191]\n\t" \
  "v_mfma_f32_16x16x32_bf16 v[184:187], a[244:247], v[76:79], v[184:187]\n\t" \
  "v_cvt_pk_bf16_f32 v60, v128, v129\n\t" \
  "v_cvt_pk_bf16_f32 v61, v130, v131\n\t" \
  "v_mfma_f32_16x16x32_bf16 v[184:187], a[240:243], v[72:75], v[184:187]\n\t" \
  "ds_write_b64 v0, v[60:61]\n\t" \
  "v_cvt_pk_bf16_f32 v62, v132, v133\n\t" \
  "v_mfma_f32_16x16x32_bf16 v[184:187], v[42:45], a[28:31], v[184:187]\n\t" \
  "v_cvt_pk_bf16_f32 v63, v134, v135\n\t" \
  "ds_write_b64 v1, v[62:63]\n\t" \
  "v_mfma_f32_16x16x32_bf16 v[180:183], a[232:235], v[72:75], v[180:183]\n\t" \
  "v_cvt_pk_bf16_f32 v60, v136, v137\n\t" \
  "v_cvt_pk_bf16_f32 v61, v138, v139\n\t" \
  "v_mfma_f32_16x16x32_bf16 v[180:183], a[236:239], v[76:79], v[180:183]\n\t" \
  "ds_write_b64 v2, v[60:61]\n\t" \
  "v_cvt_pk_bf16_f32 v62, v140, v141\n\t" \
  "v_mfma_f32_16x16x32_bf16 v[180:183], v[40:43], a[24:27], v[180:183]\n\t" \
  "v_cvt_pk_bf16_f32 v63, v142, v143\n\t" \
  "ds_write_b64 v3, v[62:63]\n\t" \
  "v_mfma_f32_16x16x32_bf16 v[176:179], a[228:231], v[76:79], v[176:179]\n\t" \
  "v_cvt_pk_bf16_f32 v60, v144, v145\n\t" \
  "v_cvt_pk_bf16_f32 v61, v146, v147\n\t" \
  "v_mfma_f32_16x16x32_bf16 v[176:179], a[224:227], v[72:75], v[176:179]\n\t" \
  "ds_write_b64 v4, v[60:61]\n\t" \
  "v_cvt_pk_bf16_f32 v62, v148, v149\n\t" \
  "v_mfma_f32_16x16x32_bf16 v[176:179], v[42:45], a[24:27], v[176:179]\n\t" \
  "v_cvt_pk_bf16_f32 v63, v150, v151\n\t" \
  "ds_write_b64 v5, v[62:63]\n\t" \
  "v_mfma_f32_16x16x32_bf16 v[172:175], a[216:219], v[72:75], v[172:175]\n\t" \
  "v_cvt_pk_bf16_f32 v60, v152, v153\n\t" \
  "v_cvt_pk_bf16_f32 v61, v154, v155\n\t" \
  "v_mfma_f32_16x16x32_bf16 v[172:175], a[220:223], v[76:79], v[172:175]\n\t" \
  "ds_write_b64 v6, v[60:61]\n\t" \
  "v_cvt_pk_bf16_f32 v62, v156, v157\n\t" \
  "v_mfma_f32_16x16x32_bf16 v[172:175], v[40:43], a[20:23], v[172:175]\n\t" \
  "v_cvt_pk_bf16_f32 v63, v158, v159\n\t" \
  "ds_write_b64 v7, v[62:63]\n\t" \
  "v_mfma_f32_16x16x32_bf16 v[168:171], a[212:215], v[76:79], v[168:171]\n\t" \
  "ds_read_b128 v[128:131], v20 offset:0\n\t" \
  "ds_read_b128 v[132:135], v21 offset:1024\n\t" \
  "v_mfma_f32_16x16x32_bf16 v[168:171], a[208:211], v[72:75], v[168:171]\n\t" \
  "ds_read_b128 v[136:139], v22 offset:2048\n\t" \
  "ds_read_b128 v[140:143], v23 offset:3072\n\t" \
  "v_mfma_f32_16x16x32_bf16 v[168:171], v[42:45], a[20:23], v[168:171]\n\t" \
  "v_mfma_f32_16x16x32_bf16 v[164:167], a[200:203], v[72:75], v[164:167]\n\t" \
  "v_mfma_f32_16x16x32_bf16 v[164:167], a[204:207], v[76:79], v[164:167]\n\t" \
  "v_mfma_f32_16x16x32_bf16 v[164:167], v[40:43], a[16:19], v[164:167]\n\t" \
  "s_waitcnt lgkmcnt(0)\n\t" \
  "s_mov_b32 s98, 0\n\t" \
  "buffer_store_dwordx4 v[128:131], v29, s[44:47], s98 offen sc0 sc1\n\t" \
  "v_mfma_f32_16x16x32_bf16 v[160:163], a[196:199], v[76:79], v[160:163]\n\t" \
  "v_mfma_f32_16x16x32_bf16 v[160:163], a[192:195], v[72:75], v[160:163]\n\t" \
  "v_mfma_f32_16x16x32_bf16 v[160:163], v[42:45], a[16:19], v[160:163]\n\t" \
  "v_mfma_f32_16x16x32_bf16 v[192:195], a[128:131], v[112:115], v[192:195]\n\t" \
  "v_mfma_f32_16x16x32_bf16 v[192:195], a[132:135], v[116:119], v[192:195]\n\t" \
  "v_mfma_f32_16x16x32_bf16 v[192:195], a[196:199], v[84:87], v[192:195]\n\t" \
  "s_mov_b32 s98, 65536\n\t" \
  "buffer_store_dwordx4 v[132:135], v29, s[44:47], s98 offen sc0 sc1\n\t" \
  "v_mfma_f32_16x16x32_bf16 v[192:195], a[192:195], v[80:83], v[192:195]\n\t" \
  "v_cvt_pk_bf16_f32 v62, v188, v189\n\t" \
  "v_cvt_pk_bf16_f32 v63, v190, v191\n\t" \
  "v_mfma_f32_16x16x32_bf16 v[192:195], v[42:45], a[32:35], v[192:195]\n\t" \
  "ds_write_b64 v7, v[62:63]\n\t" \
  "v_cvt_pk_bf16_f32 v60, v184, v185\n\t" \
  "v_mfma_f32_16x16x32_bf16 v[196:199], a[200:203], v[80:83], v[196:199]\n\t" \
  "v_cvt_pk_bf16_f32 v61, v186, v187\n\t" \
  "ds_write_b64 v6, v[60:61]\n\t" \
  "v_mfma_f32_16x16x32_bf16 v[196:199], a[204:207], v[84:87], v[196:199]\n\t" \
  "v_cvt_pk_bf16_f32 v62, v180, v181\n\t" \
  "v_cvt_pk_bf16_f32 v63, v182, v183\n\t" \
  "v_mfma_f32_16x16x32_bf16 v[196:199], a[140:143], v[116:119], v[196:199]\n\t" \
  "ds_write_b64 v5, v[62:63]\n\t" \
  "v_cvt_pk_bf16_f32 v60, v176, v177\n\t" \
  "v_mfma_f32_16x16x32_bf16 v[196:199], a[136:139], v[112:115], v[196:199]\n\t" \
  "v_cvt_pk_bf16_f32 v61, v178, v179\n\t" \
  "ds_write_b64 v4, v[60:61]\n\t" \
  "s_mov_b32 s98, 131072\n\t" \
  "buffer_store_dwordx4 v[136:139], v29, s[44:47], s98 offen sc0 sc1\n\t" \
  "v_mfma_f32_16x16x32_bf16 v[196:199], v[40:43], a[32:35], v[196:199]\n\t" \
  "v_cvt_pk_bf16_f32 v62, v172, v173\n\t" \
  "v_cvt_pk_bf16_f32 v63, v174, v175\n\t" \
  "v_mfma_f32_16x16x32_bf16 v[200:203], a[144:147], v[112:115], v[200:203]\n\t" \
  "ds_write_b64 v3, v[62:63]\n\t" \
  "v_cvt_pk_bf16_f32 v60, v168, v169\n\t" \
  "v_mfma_f32_16x16x32_bf16 v[200:203], a[148:151], v[116:119], v[200:203]\n\t" \
  "v_cvt_pk_bf16_f32 v61, v170, v171\n\t" \
  "ds_write_b64 v2, v[60:61]\n\t" \
  "v_mfma_f32_16x16x32_bf16 v[200:203], a[212:215], v[84:87], v[200:203]\n\t" \
  "v_cvt_pk_bf16_f32 v62, v164, v165\n\t" \
  "v_cvt_pk_bf16_f32 v63, v166, v167\n\t" \
  "v_mfma_f32_16x16x32_bf16 v[200:203], a[208:211], v[80:83], v[200:203]\n\t" \
  "ds_write_b64 v1, v[62:63]\n\t" \
  "v_cvt_pk_bf16_f32 v60, v160, v161\n\t" \
  "v_mfma_f32_16x16x32_bf16 v[200:203], v[42:45], a[36:39], v[200:203]\n\t" \
  "v_cvt_pk_bf16_f32 v61, v162, v163\n\t" \
  "ds_write_b64 v0, v[60:61]\n\t" \
  "s_mov_b32 s98, 196608\n\t" \
  "buffer_store_dwordx4 v[140:143], v29, s[44:47], s98 offen sc0 sc1\n\t" \
  "v_mfma_f32_16x16x32_bf16 v[204:207], a[216:219], v[80:83], v[204:207]\n\t" \
  "ds_read_b128 v[160:163], v20 offset:0\n\t" \
  "ds_read_b128 v[164:167], v21 offset:1024\n\t" \
  "v_mfma_f32_16x16x32_bf16 v[204:207], a[220:223], v[84:87], v[204:207]\n\t" \
  "ds_read_b128 v[168:171], v22 offset:2048\n\t" \
  "ds_read_b128 v[172:175], v23 offset:3072\n\t" \
  "v_mfma_f32_16x16x32_bf16 v[204:207], a[156:159], v[116:119], v[204:207]\n\t" \
  "v_mfma_f32_16x16x32_bf16 v[204:207], a[152:155], v[112:115], v[204:207]\n\t" \
  "v_mfma_f32_16x16x32_bf16 v[204:207], v[40:43], a[36:39], v[204:207]\n\t" \
  "v_mfma_f32_16x16x32_bf16 v[208:211], a[160:163], v[112:115], v[208:211]\n\t" \
  "s_waitcnt lgkmcnt(0)\n\t" \
  "s_mov_b32 s98, 262144\n\t" \
  "buffer_store_dwordx4 v[160:163], v29, s[44:47], s98 offen sc0 sc1\n\t" \
  "v_mfma_f32_16x16x32_bf16 v[208:211], a[164:167], v[116:119], v[208:211]\n\t" \
  "v_mfma_f32_16x16x32_bf16 v[208:211], a[228:231], v[84:87], v[208:211]\n\t" \
  "v_mfma_f32_16x16x32_bf16 v[208:211], a[224:227], v[80:83], v[208:211]\n\t" \
  "v_mfma_f32_16x16x32_bf16 v[208:211], v[42:45], a[40:43], v[208:211]\n\t" \
  "v_mfma_f32_16x16x32_bf16 v[212:215], a[232:235], v[80:83], v[212:215]\n\t" \
  "v_mfma_f32_16x16x32_bf16 v[212:215], a[236:239], v[84:87], v[212:215]\n\t" \
  "s_mov_b32 s98, 327680\n\t" \
  "buffer_store_dwordx4 v[164:167], v29, s[44:47], s98 offen sc0 sc1\n\t" \
  "v_mfma_f32_16x16x32_bf16 v[212:215], a[172:175], v[116:119], v[212:215]\n\t" \
  "v_mfma_f32_16x16x32_bf16 v[212:215], a[168:171], v[112:115], v[212:215]\n\t" \
  "v_mfma_f32_16x16x32_bf16 v[212:215], v[40:43], a[40:43], v[212:215]\n\t" \
  "v_mfma_f32_16x16x32_bf16 v[216:219], a[176:179], v[112:115], v[216:219]\n\t" \
  "v_mfma_f32_16x16x32_bf16 v[216:219], a[180:183], v[116:119], v[216:219]\n\t" \
  "v_mfma_f32_16x16x32_bf16 v[216:219], a[244:247], v[84:87], v[216:219]\n\t" \
  "s_mov_b32 s98, 393216\n\t" \
  "buffer_store_dwordx4 v[168:171], v29, s[44:47], s98 offen sc0 sc1\n\t" \
  "v_mfma_f32_16x16x32_bf16 v[216:219], a[240:243], v[80:83], v[216:219]\n\t" \
  "v_mfma_f32_16x16x32_bf16 v[216:219], v[42:45], a[44:47], v[216:219]\n\t" \
  "v_mfma_f32_16x16x32_bf16 v[220:223], a[248:251], v[80:83], v[220:223]\n\t" \
  "v_mfma_f32_16x16x32_bf16 v[220:223], a[252:255], v[84:87], v[220:223]\n\t" \
  "v_mfma_f32_16x16x32_bf16 v[220:223], a[188:191], v[116:119], v[220:223]\n\t" \
  "v_mfma_f32_16x16x32_bf16 v[220:223], a[184:187], v[112:115], v[220:223]\n\t" \
  "s_mov_b32 s98, 458752\n\t" \
  "buffer_store_dwordx4 v[172:175], v29, s[44:47], s98 offen sc0 sc1\n\t" \
  "v_mfma_f32_16x16x32_bf16 v[220:223], v[40:43], a[44:47], v[220:223]\n\t" \
  "v_mfma_f32_16x16x32_bf16 v[252:255], a[184:187], v[120:123], v[252:255]\n\t" \
  "v_mfma_f32_16x16x32_bf16 v[252:255], a[188:191], v[124:127], v[252:255]\n\t" \
  "v_mfma_f32_16x16x32_bf16 v[252:255], a[252:255], v[92:95], v[252:255]\n\t" \
  "v_mfma_f32_16x16x32_bf16 v[252:255], a[248:251], v[88:91], v[252:255]\n\t" \
  "v_cvt_pk_bf16_f32 v60, v192, v193\n\t" \
  "v_cvt_pk_bf16_f32 v61, v194, v195\n\t" \
  "v_mfma_f32_16x16x32_bf16 v[252:255], v[40:43], a[60:63], v[252:255]\n\t" \
  "ds_write_b64 v0, v[60:61]\n\t" \
  "v_cvt_pk_bf16_f32 v62, v196, v197\n\t" \
  "v_mfma_f32_16x16x32_bf16 v[248:251], a[240:243], v[88:91], v[248:251]\n\t" \
  "v_cvt_pk_bf16_f32 v63, v198, v199\n\t" \
  "ds_write_b64 v1, v[62:63]\n\t" \
  "v_mfma_f32_16x16x32_bf16 v[248:251], a[244:247], v[92:95], v[248:251]\n\t" \
  "v_cvt_pk_bf16_f32 v60, v200, v201\n\t" \
  "v_cvt_pk_bf16_f32 v61, v202, v203\n\t" \
  "v_mfma_f32_16x16x32_bf16 v[248:251], a[180:183], v[124:127], v[248:251]\n\t" \
  "ds_write_b64 v2, v[60:61]\n\t" \
  "v_cvt_pk_bf16_f32 v62, v204, v205\n\t" \
  "v_mfma_f32_16x16x32_bf16 v[248:251], a[176:179], v[120:123], v[248:251]\n\t" \
  "v_cvt_pk_bf16_f32 v63, v206, v207\n\t" \
  "ds_write_b64 v3, v[62:63]\n\t" \
  "v_mfma_f32_16x16x32_bf16 v[248:251], v[42:45], a[60:63], v[248:251]\n\t" \
  "v_cvt_pk_bf16_f32 v60, v208, v209\n\t" \
  "v_cvt_pk_bf16_f32 v61, v210, v211\n\t" \
  "v_mfma_f32_16x16x32_bf16 v[244:247], a[168:171], v[120:123], v[244:247]\n\t" \
  "ds_write_b64 v4, v[60:61]\n\t" \
  "v_cvt_pk_bf16_f32 v62, v212, v213\n\t" \
  "v_mfma_f32_16x16x32_bf16 v[244:247], a[172:175], v[124:127], v[244:247]\n\t" \
  "v_cvt_pk_bf16_f32 v63, v214, v215\n\t" \
  "ds_write_b64 v5, v[62:63]\n\t" \
  "v_mfma_f32_16x16x32_bf16 v[244:247], a[236:239], v[92:95], v[244:247]\n\t" \
  "v_cvt_pk_bf16_f32 v60, v216, v217\n\t" \
  "v_cvt_pk_bf16_f32 v61, v218, v219\n\t" \
  "v_mfma_f32_16x16x32_bf16 v[244:247], a[232:235], v[88:91], v[244:247]\n\t" \
  "ds_write_b64 v6, v[60:61]\n\t" \
  "v_cvt_pk_bf16_f32 v62, v220, v221\n\t" \
  "v_mfma_f32_16x16x32_bf16 v[244:247], v[40:43], a[56:59], v[244:247]\n\t" \
  "v_cvt_pk_bf16_f32 v63, v222, v223\n\t" \
  "ds_write_b64 v7, v[62:63]\n\t" \
  "v_mfma_f32_16x16x32_bf16 v[240:243], a[224:227], v[88:91], v[240:243]\n\t" \
  "ds_read_b128 v[192:195], v20 offset:0\n\t" \
  "ds_read_b128 v[196:199], v21 offset:1024\n\t" \
  "v_mfma_f32_16x16x32_bf16 v[240:243], a[228:231], v[92:95], v[240:243]\n\t" \
  "ds_read_b128 v[200:203], v22 offset:2048\n\t" \
  "ds_read_b128 v[204:207], v23 offset:3072\n\t" \
  "v_mfma_f32_16x16x32_bf16 v[240:243], a[164:167], v[124:127], v[240:243]\n\t" \
  "v_mfma_f32_16x16x32_bf16 v[240:243], a[160:163], v[120:123], v[240:243]\n\t" \
  "v_mfma_f32_16x16x32_bf16 v[240:243], v[42:45], a[56:59], v[240:243]\n\t" \
  "v_mfma_f32_16x16x32_bf16 v[236:239], a[152:155], v[120:123], v[236:239]\n\t" \
  "s_waitcnt lgkmcnt(0)\n\t" \
  "s_mov_b32 s98, 524288\n\t" \
  "buffer_store_dwordx4 v[192:195], v29, s[44:47], s98 offen sc0 sc1\n\t" \
  "v_mfma_f32_16x16x32_bf16 v[236:239], a[156:159], v[124:127], v[236:239]\n\t" \
  "v_mfma_f32_16x16x32_bf16 v[236:239], a[220:223], v[92:95], v[236:239]\n\t" \
  "v_mfma_f32_16x16x32_bf16 v[236:239], a[216:219], v[88:91], v[236:239]\n\t" \
  "v_mfma_f32_16x16x32_bf16 v[236:239], v[40:43], a[52:55], v[236:239]\n\t" \
  "v_mfma_f32_16x16x32_bf16 v[232:235], a[208:211], v[88:91], v[232:235]\n\t" \
  "v_mfma_f32_16x16x32_bf16 v[232:235], a[212:215], v[92:95], v[232:235]\n\t" \
  "s_mov_b32 s98, 589824\n\t" \
  "buffer_store_dwordx4 v[196:199], v29, s[44:47], s98 offen sc0 sc1\n\t" \
  "v_mfma_f32_16x16x32_bf16 v[232:235], a[148:151], v[124:127], v[232:235]\n\t" \
  "v_mfma_f32_16x16x32_bf16 v[232:235], a[144:147], v[120:123], v[232:235]\n\t" \
  "v_mfma_f32_16x16x32_bf16 v[232:235], v[42:45], a[52:55], v[232:235]\n\t" \
  "v_mfma_f32_16x16x32_bf16 v[228:231], a[136:139], v[120:123], v[228:231]\n\t" \
  "v_mfma_f32_16x16x32_bf16 v[228:231], a[140:143], v[124:127], v[228:231]\n\t" \
  "v_mfma_f32_16x16x32_bf16 v[228:231], a[204:207], v[92:95], v[228:231]\n\t" \
  "s_mov_b32 s98, 655360\n\t" \
  "buffer_store_dwordx4 v[200:203], v29, s[44:47], s98 offen sc0 sc1\n\t" \
  "v_mfma_f32_16x16x32_bf16 v[228:231], a[200:203], v[88:91], v[228:231]\n\t" \
  "v_mfma_f32_16x16x32_bf16 v[228:231], v[40:43], a[48:51], v[228:231]\n\t" \
  "v_mfma_f32_16x16x32_bf16 v[224:227], a[192:195], v[88:91], v[224:227]\n\t" \
  "v_mfma_f32_16x16x32_bf16 v[224:227], a[196:199], v[92:95], v[224:227]\n\t" \
  "v_mfma_f32_16x16x32_bf16 v[224:227], a[132:135], v[124:127], v[224:227]\n\t" \
  "v_mfma_f32_16x16x32_bf16 v[224:227], a[128:131], v[120:123], v[224:227]\n\t" \
  "s_mov_b32 s98, 720896\n\t" \
  "buffer_store_dwordx4 v[204:207], v29, s[44:47], s98 offen sc0 sc1\n\t" \
  "v_mfma_f32_16x16x32_bf16 v[224:227], v[42:45], a[48:51], v[224:227]\n\t" \
  "s_nop 7\n\t" \
  "v_cvt_pk_bf16_f32 v62, v252, v253\n\t" \
  "v_cvt_pk_bf16_f32 v63, v254, v255\n\t" \
  "ds_write_b64 v7, v[62:63]\n\t" \
  "v_cvt_pk_bf16_f32 v60, v248, v249\n\t" \
  "v_cvt_pk_bf16_f32 v61, v250, v251\n\t" \
  "ds_write_b64 v6, v[60:61]\n\t" \
  "v_cvt_pk_bf16_f32 v62, v244, v245\n\t" \
  "v_cvt_pk_bf16_f32 v63, v246, v247\n\t" \
  "ds_write_b64 v5, v[62:63]\n\t" \
  "v_cvt_pk_bf16_f32 v60, v240, v241\n\t" \
  "v_cvt_pk_bf16_f32 v61, v242, v243\n\t" \
  "ds_write_b64 v4, v[60:61]\n\t" \
  "v_cvt_pk_bf16_f32 v62, v236, v237\n\t" \
  "v_cvt_pk_bf16_f32 v63, v238, v239\n\t" \
  "ds_write_b64 v3, v[62:63]\n\t" \
  "v_cvt_pk_bf16_f32 v60, v232, v233\n\t" \
  "v_cvt_pk_bf16_f32 v61, v234, v235\n\t" \
  "ds_write_b64 v2, v[60:61]\n\t" \
  "v_cvt_pk_bf16_f32 v62, v228, v229\n\t" \
  "v_cvt_pk_bf16_f32 v63, v230, v231\n\t" \
  "ds_write_b64 v1, v[62:63]\n\t" \
  "v_cvt_pk_bf16_f32 v60, v224, v225\n\t" \
  "v_cvt_pk_bf16_f32 v61, v226, v227\n\t" \
  "ds_write_b64 v0, v[60:61]\n\t" \
  "ds_read_b128 v[224:227], v20 offset:0\n\t" \
  "ds_read_b128 v[228:231], v21 offset:1024\n\t" \
  "ds_read_b128 v[232:235], v22 offset:2048\n\t" \
  "ds_read_b128 v[236:239], v23 offset:3072\n\t" \
  "s_mov_b32 s98, 786432\n\t" \
  "s_waitcnt lgkmcnt(3)\n\t" \
  "buffer_store_dwordx4 v[224:227], v29, s[44:47], s98 offen sc0 sc1\n\t" \
  "s_mov_b32 s98, 851968\n\t" \
  "s_waitcnt lgkmcnt(2)\n\t" \
  "buffer_store_dwordx4 v[228:231], v29, s[44:47], s98 offen sc0 sc1\n\t" \
  "s_mov_b32 s98, 917504\n\t" \
  "s_waitcnt lgkmcnt(1)\n\t" \
  "buffer_store_dwordx4 v[232:235], v29, s[44:47], s98 offen sc0 sc1\n\t" \
  "s_mov_b32 s98, 983040\n\t" \
  "s_waitcnt lgkmcnt(0)\n\t" \
  "buffer_store_dwordx4 v[236:239], v29, s[44:47], s98 offen sc0 sc1\n\t" \
  "s_branch END_%=\n\t" \
  "TAILO_%=:\n\t" \
  "v_mfma_f32_16x16x32_bf16 a[4:7], a[136:139], v[64:67], a[4:7]\n\t" \
  "s_add_u32 s99, s77, 1\n\t" \
  "v_mfma_f32_16x16x32_bf16 a[8:11], a[144:147], v[64:67], a[8:11]\n\t" \
  "s_add_u32 s66, s77, 2\n\t" \
  "v_mfma_f32_16x16x32_bf16 a[8:11], a[148:151], v[68:71], a[8:11]\n\t" \
  "v_mfma_f32_16x16x32_bf16 a[12:15], a[156:159], v[68:71], a[12:15]\n\t" \
  "v_mfma_f32_16x16x32_bf16 a[12:15], a[152:155], v[64:67], a[12:15]\n\t" \
  "s_waitcnt vmcnt(16) lgkmcnt(0)\n\t" \
  "s_barrier\n\t" \
  "v_mfma_f32_16x16x32_bf16 a[16:19], a[160:163], v[64:67], a[16:19]\n\t" \
  "ds_read_b64_tr_b16 a[192:193], v12 offset:32768\n\t" \
  "v_mfma_f32_16x16x32_bf16 a[16:19], a[164:167], v[68:71], a[16:19]\n\t" \
  "s_bitcmp1_b64 s[90:91], s66\n\t" \
  "s_andn2_b64 s[92:93], 255, s[58:59]\n\t" \
  "s_cbranch_scc1 POLL186_%=\n\t" \
  "RET185_%=:\n\t" \
  "v_mfma_f32_16x16x32_bf16 a[20:23], a[172:175], v[68:71], a[20:23]\n\t" \
  "ds_read_b64_tr_b16 a[194:195], v12 offset:33792\n\t" \
  "v_mfma_f32_16x16x32_bf16 a[20:23], a[168:171], v[64:67], a[20:23]\n\t" \
  "s_cbranch_scc0 POLL177_%=\n\t" \
  "RET176_%=:\n\t" \
  "v_mfma_f32_16x16x32_bf16 a[24:27], a[176:179], v[64:67], a[24:27]\n\t" \
  "ds_read_b64_tr_b16 a[196:197], v12 offset:40960\n\t" \
  "v_mfma_f32_16x16x32_bf16 a[24:27], a[180:183], v[68:71], a[24:27]\n\t" \
  "v_cmp_le_u32 s[92:93], s86, v36\n\t" \
  "v_mfma_f32_16x16x32_bf16 a[28:31], a[188:191], v[68:71], a[28:31]\n\t" \
  "ds_read_b64_tr_b16 a[198:199], v12 offset:41984\n\t" \
  "v_mfma_f32_16x16x32_bf16 a[28:31], a[184:187], v[64:67], a[28:31]\n\t" \
  "ds_read_b128 v[64:67], v10 offset:32768\n\t" \
  "ds_read_b128 v[68:71], v11 offset:32768\n\t" \
  "v_mfma_f32_16x16x32_bf16 a[60:63], a[184:187], v[72:75], a[60:63]\n\t" \
  "ds_read_b64_tr_b16 a[200:201], v13 offset:32768\n\t" \
  "v_mfma_f32_16x16x32_bf16 a[60:63], a[188:191], v[76:79], a[60:63]\n\t" \
  "s_or_b64 s[90:91], s[90:91], s[92:93]\n\t" \
  "v_mfma_f32_16x16x32_bf16 a[56:59], a[180:183], v[76:79], a[56:59]\n\t" \
  "ds_read_b64_tr_b16 a[202:203], v13 offset:33792\n\t" \
  "v_mfma_f32_16x16x32_bf16 a[56:59], a[176:179], v[72:75], a[56:59]\n\t" \
  "s_bitcmp1_b64 s[90:91], s99\n\t" \
  "v_mfma_f32_16x16x32_bf16 a[52:55], a[168:171], v[72:75], a[52:55]\n\t" \
  "ds_read_b64_tr_b16 a[204:205], v13 offset:40960\n\t" \
  "v_mfma_f32_16x16x32_bf16 a[52:55], a[172:175], v[76:79], a[52:55]\n\t" \
  "s_cbranch_scc0 POLL183_%=\n\t" \
  "RET182_%=:\n\t" \
  "v_mfma_f32_16x16x32_bf16 a[48:51], a[164:167], v[76:79], a[48:51]\n\t" \
  "ds_read_b64_tr_b16 a[206:207], v13 offset:41984\n\t" \
  "v_mfma_f32_16x16x32_bf16 a[48:51], a[160:163], v[72:75], a[48:51]\n\t" \
  "v_mfma_f32_16x16x32_bf16 a[44:47], a[152:155], v[72:75], a[44:47]\n\t" \
  "ds_read_b64_tr_b16 a[208:209], v14 offset:32768\n\t" \
  "v_mfma_f32_16x16x32_bf16 a[44:47], a[156:159], v[76:79], a[44:47]\n\t" \
  "v_mfma_f32_16x16x32_bf16 a[40:43], a[148:151], v[76:79], a[40:43]\n\t" \
  "ds_read_b64_tr_b16 a[210:211], v14 offset:33792\n\t" \
  "v_mfma_f32_16x16x32_bf16 a[40:43], a[144:147], v[72:75], a[40:43]\n\t" \
  "v_mfma_f32_16x16x32_bf16 a[36:39], a[136:139], v[72:75], a[36:39]\n\t" \
  "ds_read_b64_tr_b16 a[212:213], v14 offset:40960\n\t" \
  "v_mfma_f32_16x16x32_bf16 a[36:39], a[140:143], v[76:79], a[36:39]\n\t" \
  "v_mfma_f32_16x16x32_bf16 a[32:35], a[132:135], v[76:79], a[32:35]\n\t" \
  "ds_read_b64_tr_b16 a[214:215], v14 offset:41984\n\t" \
  "v_mfma_f32_16x16x32_bf16 a[32:35], a[128:131], v[72:75], a[32:35]\n\t" \
  "ds_read_b128 v[72:75], v10 offset:34816\n\t" \
  "ds_read_b128 v[76:79], v11 offset:34816\n\t" \
  "v_mfma_f32_16x16x32_bf16 a[64:67], a[128:131], v[80:83], a[64:67]\n\t" \
  "ds_read_b64_tr_b16 a[216:217], v15 offset:32768\n\t" \
  "v_mfma_f32_16x16x32_bf16 a[64:67], a[132:135], v[84:87], a[64:67]\n\t" \
  "v_mfma_f32_16x16x32_bf16 a[68:71], a[140:143], v[84:87], a[68:71]\n\t" \
  "ds_read_b64_tr_b16 a[218:219], v15 offset:33792\n\t" \
  "v_mfma_f32_16x16x32_bf16 a[68:71], a[136:139], v[80:83], a[68:71]\n\t" \
  "v_mfma_f32_16x16x32_bf16 a[72:75], a[144:147], v[80:83], a[72:75]\n\t" \
  "ds_read_b64_tr_b16 a[220:221], v15 offset:40960\n\t" \
  "v_mfma_f32_16x16x32_bf16 a[72:75], a[148:151], v[84:87], a[72:75]\n\t" \
  "v_mfma_f32_16x16x32_bf16 a[76:79], a[156:159], v[84:87], a[76:79]\n\t" \
  "ds_read_b64_tr_b16 a[222:223], v15 offset:41984\n\t" \
  "v_mfma_f32_16x16x32_bf16 a[76:79], a[152:155], v[80:83], a[76:79]\n\t" \
  "s_add_u32 m0, s68, 16384\n\t" \
  "v_mfma_f32_16x16x32_bf16 a[80:83], a[160:163], v[80:83], a[80:83]\n\t" \
  "buffer_load_dwordx4 v0, s[36:39], s65 offen offset:0 sc0 sc1 lds\n\t" \
  "v_mfma_f32_16x16x32_bf16 a[80:83], a[164:167], v[84:87], a[80:83]\n\t" \
  "v_mfma_f32_16x16x32_bf16 a[84:87], a[172:175], v[84:87], a[84:87]\n\t" \
  "v_mfma_f32_16x16x32_bf16 a[84:87], a[168:171], v[80:83], a[84:87]\n\t" \
  "buffer_load_dwordx4 v1, s[36:39], s65 offen offset:1024 sc0 sc1 lds\n\t" \
  "v_mfma_f32_16x16x32_bf16 a[88:91], a[176:179], v[80:83], a[88:91]\n\t" \
  "v_mfma_f32_16x16x32_bf16 a[88:91], a[180:183], v[84:87], a[88:91]\n\t" \
  "v_mfma_f32_16x16x32_bf16 a[92:95], a[188:191], v[84:87], a[92:95]\n\t" \
  "buffer_load_dwordx4 v2, s[36:39], s65 offen offset:2048 sc0 sc1 lds\n\t" \
  "v_mfma_f32_16x16x32_bf16 a[92:95], a[184:187], v[80:83], a[92:95]\n\t" \
  "ds_read_b128 v[80:83], v10 offset:36864\n\t" \
  "ds_read_b128 v[84:87], v11 offset:36864\n\t" \
  "v_mfma_f32_16x16x32_bf16 a[124:127], a[184:187], v[88:91], a[124:127]\n\t" \
  "v_mfma_f32_16x16x32_bf16 a[124:127], a[188:191], v[92:95], a[124:127]\n\t" \
  "buffer_load_dwordx4 v3, s[36:39], s65 offen offset:3072 sc0 sc1 lds\n\t" \
  "v_mfma_f32_16x16x32_bf16 a[120:123], a[180:183], v[92:95], a[120:123]\n\t" \
  "v_mfma_f32_16x16x32_bf16 a[120:123], a[176:179], v[88:91], a[120:123]\n\t" \
  "s_add_u32 m0, s68, 81920\n\t" \
  "v_mfma_f32_16x16x32_bf16 a[116:119], a[168:171], v[88:91], a[116:119]\n\t" \
  "buffer_load_dwordx4 v4, s[40:43], s79 offen offset:0 lds\n\t" \
  "v_mfma_f32_16x16x32_bf16 a[116:119], a[172:175], v[92:95], a[116:119]\n\t" \
  "s_bitcmp1_b64 s[90:91], s99\n\t" \
  "v_mfma_f32_16x16x32_bf16 a[112:115], a[164:167], v[92:95], a[112:115]\n\t" \
  "s_cbranch_scc0 STG184_%=\n\t" \
  "RET178_%=:\n\t" \
  "v_mfma_f32_16x16x32_bf16 a[112:115], a[160:163], v[88:91], a[112:115]\n\t" \
  "buffer_load_dwordx4 v5, s[40:43], s79 offen offset:1024 lds\n\t" \
  "v_mfma_f32_16x16x32_bf16 a[108:111], a[152:155], v[88:91], a[108:111]\n\t" \
  "v_mfma_f32_16x16x32_bf16 a[108:111], a[156:159], v[92:95], a[108:111]\n\t" \
  "v_mfma_f32_16x16x32_bf16 a[104:107], a[148:151], v[92:95], a[104:107]\n\t" \
  "buffer_load_dwordx4 v6, s[40:43], s79 offen offset:2048 lds\n\t" \
  "v_mfma_f32_16x16x32_bf16 a[104:107], a[144:147], v[88:91], a[104:107]\n\t" \
  "v_mfma_f32_16x16x32_bf16 a[100:103], a[136:139], v[88:91], a[100:103]\n\t" \
  "v_mfma_f32_16x16x32_bf16 a[100:103], a[140:143], v[92:95], a[100:103]\n\t" \
  "buffer_load_dwordx4 v7, s[40:43], s79 offen offset:3072 lds\n\t" \
  "v_mfma_f32_16x16x32_bf16 a[96:99], a[132:135], v[92:95], a[96:99]\n\t" \
  "s_add_u32 s98, s71, 0\n\t" \
  "v_readlane_b32 s98, v17, s98\n\t" \
  "s_nop 1\n\t" \
  "s_lshl_b32 s98, s98, 13\n\t" \
  "s_mov_b32 exec_hi, 0\n\t" \
  "buffer_load_dword v50, v23, s[40:43], s98 offen\n\t" \
  "s_mov_b32 exec_hi, -1\n\t" \
  "v_mfma_f32_16x16x32_bf16 a[96:99], a[128:131], v[88:91], a[96:99]\n\t" \
  "ds_read_b128 v[88:91], v10 offset:38912\n\t" \
  "ds_read_b128 v[92:95], v11 offset:38912\n\t" \
  "v_mfma_f32_16x16x32_bf16 v[128:131], a[128:131], v[96:99], v[128:131]\n\t" \
  "s_add_u32 s77, s77, 1\n\t" \
  "v_mfma_f32_16x16x32_bf16 v[128:131], a[132:135], v[100:103], v[128:131]\n\t" \
  "v_readlane_b32 s66, v16, s77\n\t" \
  "v_mfma_f32_16x16x32_bf16 v[132:135], a[140:143], v[100:103], v[132:135]\n\t" \
  "v_mfma_f32_16x16x32_bf16 v[132:135], a[136:139], v[96:99], v[132:135]\n\t" \
  "v_mfma_f32_16x16x32_bf16 v[136:139], a[144:147], v[96:99], v[136:139]\n\t" \
  "s_lshl_b32 s78, s66, 13\n\t" \
  "v_mfma_f32_16x16x32_bf16 v[136:139], a[148:151], v[100:103], v[136:139]\n\t" \
  "s_lshr_b32 s65, s66, 11\n\t" \
  "v_mfma_f32_16x16x32_bf16 v[140:143], a[156:159], v[100:103], v[140:143]\n\t" \
  "s_mul_i32 s64, s66, s67\n\t" \
  "v_mfma_f32_16x16x32_bf16 v[140:143], a[152:155], v[96:99], v[140:143]\n\t" \
  "s_waitcnt vmcnt(17) lgkmcnt(0)\n\t" \
  "s_barrier\n\t" \
  "v_mfma_f32_16x16x32_bf16 v[144:147], a[160:163], v[96:99], v[144:147]\n\t" \
  "ds_read_b64_tr_b16 a[224:225], v12 offset:49152\n\t" \
  "v_mfma_f32_16x16x32_bf16 v[144:147], a[164:167], v[100:103], v[144:147]\n\t" \
  "s_add_u32 s79, s78, 256\n\t" \
  "v_mfma_f32_16x16x32_bf16 v[148:151], a[172:175], v[100:103], v[148:151]\n\t" \
  "ds_read_b64_tr_b16 a[226:227], v12 offset:50176\n\t" \
  "v_mfma_f32_16x16x32_bf16 v[148:151], a[168:171], v[96:99], v[148:151]\n\t" \
  "s_mul_i32 s65, s65, s89\n\t" \
  "v_mfma_f32_16x16x32_bf16 v[152:155], a[176:179], v[96:99], v[152:155]\n\t" \
  "ds_read_b64_tr_b16 a[228:229], v12 offset:57344\n\t" \
  "v_mfma_f32_16x16x32_bf16 v[152:155], a[180:183], v[100:103], v[152:155]\n\t" \
  "s_add_u32 s65, s65, s64\n\t" \
  "v_mfma_f32_16x16x32_bf16 v[156:159], a[188:191], v[100:103], v[156:159]\n\t" \
  "ds_read_b64_tr_b16 a[230:231], v12 offset:58368\n\t" \
  "v_mfma_f32_16x16x32_bf16 v[156:159], a[184:187], v[96:99], v[156:159]\n\t" \
  "ds_read_b128 v[96:99], v10 offset:49152\n\t" \
  "ds_read_b128 v[100:103], v11 offset:49152\n\t" \
  "v_mfma_f32_16x16x32_bf16 v[188:191], a[184:187], v[104:107], v[188:191]\n\t" \
  "ds_read_b64_tr_b16 a[232:233], v13 offset:49152\n\t" \
  "v_mfma_f32_16x16x32_bf16 v[188:191], a[188:191], v[108:111], v[188:191]\n\t" \
  "s_add_u32 s64, s65, s69\n\t" \
  "v_mfma_f32_16x16x32_bf16 v[184:187], a[180:183], v[108:111], v[184:187]\n\t" \
  "ds_read_b64_tr_b16 a[234:235], v13 offset:50176\n\t" \
  "v_mfma_f32_16x16x32_bf16 v[184:187], a[176:179], v[104:107], v[184:187]\n\t" \
  "s_add_u32 s65, s65, s70\n\t" \
  "v_mfma_f32_16x16x32_bf16 v[180:183], a[168:171], v[104:107], v[180:183]\n\t" \
  "ds_read_b64_tr_b16 a[236:237], v13 offset:57344\n\t" \
  "v_mfma_f32_16x16x32_bf16 v[180:183], a[172:175], v[108:111], v[180:183]\n\t" \
  "v_mfma_f32_16x16x32_bf16 v[176:179], a[164:167], v[108:111], v[176:179]\n\t" \
  "ds_read_b64_tr_b16 a[238:239], v13 offset:58368\n\t" \
  "v_mfma_f32_16x16x32_bf16 v[176:179], a[160:163], v[104:107], v[176:179]\n\t" \
  "v_mfma_f32_16x16x32_bf16 v[172:175], a[152:155], v[104:107], v[172:175]\n\t" \
  "ds_read_b64_tr_b16 a[240:241], v14 offset:49152\n\t" \
  "v_mfma_f32_16x16x32_bf16 v[172:175], a[156:159], v[108:111], v[172:175]\n\t" \
  "v_mfma_f32_16x16x32_bf16 v[168:171], a[148:151], v[108:111], v[168:171]\n\t" \
  "ds_read_b64_tr_b16 a[242:243], v14 offset:50176\n\t" \
  "v_mfma_f32_16x16x32_bf16 v[168:171], a[144:147], v[104:107], v[168:171]\n\t" \
  "v_mfma_f32_16x16x32_bf16 v[164:167], a[136:139], v[104:107], v[164:167]\n\t" \
  "ds_read_b64_tr_b16 a[244:245], v14 offset:57344\n\t" \
  "v_mfma_f32_16x16x32_bf16 v[164:167], a[140:143], v[108:111], v[164:167]\n\t" \
  "v_mfma_f32_16x16x32_bf16 v[160:163], a[132:135], v[108:111], v[160:163]\n\t" \
  "ds_read_b64_tr_b16 a[246:247], v14 offset:58368\n\t" \
  "v_mfma_f32_16x16x32_bf16 v[160:163], a[128:131], v[104:107], v[160:163]\n\t" \
  "ds_read_b128 v[104:107], v10 offset:51200\n\t" \
  "ds_read_b128 v[108:111], v11 offset:51200\n\t" \
  "v_mfma_f32_16x16x32_bf16 v[192:195], a[128:131], v[112:115], v[192:195]\n\t" \
  "ds_read_b64_tr_b16 a[248:249], v15 offset:49152\n\t" \
  "v_mfma_f32_16x16x32_bf16 v[192:195], a[132:135], v[116:119], v[192:195]\n\t" \
  "v_mfma_f32_16x16x32_bf16 v[196:199], a[140:143], v[116:119], v[196:199]\n\t" \
  "ds_read_b64_tr_b16 a[250:251], v15 offset:50176\n\t" \
  "v_mfma_f32_16x16x32_bf16 v[196:199], a[136:139], v[112:115], v[196:199]\n\t" \
  "v_mfma_f32_16x16x32_bf16 v[200:203], a[144:147], v[112:115], v[200:203]\n\t" \
  "ds_read_b64_tr_b16 a[252:253], v15 offset:57344\n\t" \
  "v_mfma_f32_16x16x32_bf16 v[200:203], a[148:151], v[116:119], v[200:203]\n\t" \
  "v_mfma_f32_16x16x32_bf16 v[204:207], a[156:159], v[116:119], v[204:207]\n\t" \
  "ds_read_b64_tr_b16 a[254:255], v15 offset:58368\n\t" \
  "v_mfma_f32_16x16x32_bf16 v[204:207], a[152:155], v[112:115], v[204:207]\n\t" \
  "s_add_u32 m0, s68, 32768\n\t" \
  "v_mfma_f32_16x16x32_bf16 v[208:211], a[160:163], v[112:115], v[208:211]\n\t" \
  "buffer_load_dwordx4 v0, s[36:39], s64 offen offset:0 sc0 sc1 lds\n\t" \
  "v_mfma_f32_16x16x32_bf16 v[208:211], a[164:167], v[116:119], v[208:211]\n\t" \
  "v_mfma_f32_16x16x32_bf16 v[212:215], a[172:175], v[116:119], v[212:215]\n\t" \
  "v_mfma_f32_16x16x32_bf16 v[212:215], a[168:171], v[112:115], v[212:215]\n\t" \
  "buffer_load_dwordx4 v1, s[36:39], s64 offen offset:1024 sc0 sc1 lds\n\t" \
  "v_mfma_f32_16x16x32_bf16 v[216:219], a[176:179], v[112:115], v[216:219]\n\t" \
  "v_mfma_f32_16x16x32_bf16 v[216:219], a[180:183], v[116:119], v[216:219]\n\t" \
  "v_mfma_f32_16x16x32_bf16 v[220:223], a[188:191], v[116:119], v[220:223]\n\t" \
  "buffer_load_dwordx4 v2, s[36:39], s64 offen offset:2048 sc0 sc1 lds\n\t" \
  "v_mfma_f32_16x16x32_bf16 v[220:223], a[184:187], v[112:115], v[220:223]\n\t" \
  "ds_read_b128 v[112:115], v10 offset:53248\n\t" \
  "ds_read_b128 v[116:119], v11 offset:53248\n\t" \
  "v_mfma_f32_16x16x32_bf16 v[252:255], a[184:187], v[120:123], v[252:255]\n\t" \
  "v_mfma_f32_16x16x32_bf16 v[252:255], a[188:191], v[124:127], v[252:255]\n\t" \
  "buffer_load_dwordx4 v3, s[36:39], s64 offen offset:3072 sc0 sc1 lds\n\t" \
  "v_mfma_f32_16x16x32_bf16 v[248:251], a[180:183], v[124:127], v[248:251]\n\t" \
  "v_mfma_f32_16x16x32_bf16 v[248:251], a[176:179], v[120:123], v[248:251]\n\t" \
  "s_add_u32 m0, s68, 98304\n\t" \
  "v_mfma_f32_16x16x32_bf16 v[244:247], a[168:171], v[120:123], v[244:247]\n\t" \
  "buffer_load_dwordx4 v4, s[40:43], s78 offen offset:0 lds\n\t" \
  "v_mfma_f32_16x16x32_bf16 v[244:247], a[172:175], v[124:127], v[244:247]\n\t" \
  "v_mfma_f32_16x16x32_bf16 v[240:243], a[164:167], v[124:127], v[240:243]\n\t" \
  "v_mfma_f32_16x16x32_bf16 v[240:243], a[160:163], v[120:123], v[240:243]\n\t" \
  "buffer_load_dwordx4 v5, s[40:43], s78 offen offset:1024 lds\n\t" \
  "v_mfma_f32_16x16x32_bf16 v[236:239], a[152:155], v[120:123], v[236:239]\n\t" \
  "v_mfma_f32_16x16x32_bf16 v[236:239], a[156:159], v[124:127], v[236:239]\n\t" \
  "v_mfma_f32_16x16x32_bf16 v[232:235], a[148:151], v[124:127], v[232:235]\n\t" \
  "buffer_load_dwordx4 v6, s[40:43], s78 offen offset:2048 lds\n\t" \
  "v_mfma_f32_16x16x32_bf16 v[232:235], a[144:147], v[120:123], v[232:235]\n\t" \
  "v_mfma_f32_16x16x32_bf16 v[228:231], a[136:139], v[120:123], v[228:231]\n\t" \
  "v_mfma_f32_16x16x32_bf16 v[228:231], a[140:143], v[124:127], v[228:231]\n\t" \
  "buffer_load_dwordx4 v7, s[40:43], s78 offen offset:3072 lds\n\t" \
  "v_mfma_f32_16x16x32_bf16 v[224:227], a[132:135], v[124:127], v[224:227]\n\t" \
  "s_add_u32 s98, s71, 4\n\t" \
  "v_readlane_b32 s98, v17, s98\n\t" \
  "s_nop 1\n\t" \
  "s_lshl_b32 s98, s98, 13\n\t" \
  "s_mov_b32 exec_hi, 0\n\t" \
  "buffer_load_dword v50, v23, s[40:43], s98 offen\n\t" \
  "s_mov_b32 exec_hi, -1\n\t" \
  "v_mfma_f32_16x16x32_bf16 v[224:227], a[128:131], v[120:123], v[224:227]\n\t" \
  "ds_read_b128 v[120:123], v10 offset:55296\n\t" \
  "ds_read_b128 v[124:127], v11 offset:55296\n\t" \
  "v_mfma_f32_16x16x32_bf16 a[0:3], a[192:195], v[64:67], a[0:3]\n\t" \
  "v_mfma_f32_16x16x32_bf16 a[0:3], a[196:199], v[68:71], a[0:3]\n\t" \
  "v_mfma_f32_16x16x32_bf16 a[4:7], a[204:207], v[68:71], a[4:7]\n\t" \
  "v_mfma_f32_16x16x32_bf16 a[4:7], a[200:203], v[64:67], a[4:7]\n\t" \
  "s_add_u32 s99, s77, 1\n\t" \
  "v_mfma_f32_16x16x32_bf16 a[8:11], a[208:211], v[64:67], a[8:11]\n\t" \
  "v_mfma_f32_16x16x32_bf16 a[8:11], a[212:215], v[68:71], a[8:11]\n\t" \
  "v_mfma_f32_16x16x32_bf16 a[12:15], a[220:223], v[68:71], a[12:15]\n\t" \
  "v_mfma_f32_16x16x32_bf16 a[12:15], a[216:219], v[64:67], a[12:15]\n\t" \
  "s_waitcnt vmcnt(18) lgkmcnt(0)\n\t" \
  "s_barrier\n\t" \
  "v_mfma_f32_16x16x32_bf16 a[16:19], a[224:227], v[64:67], a[16:19]\n\t" \
  "ds_read_b64_tr_b16 a[128:129], v12 offset:0\n\t" \
  "v_mfma_f32_16x16x32_bf16 a[16:19], a[228:231], v[68:71], a[16:19]\n\t" \
  "s_andn2_b64 s[92:93], 255, s[58:59]\n\t" \
  "s_cbranch_scc1 POLL197_%=\n\t" \
  "RET196_%=:\n\t" \
  "v_mfma_f32_16x16x32_bf16 a[20:23], a[236:239], v[68:71], a[20:23]\n\t" \
  "ds_read_b64_tr_b16 a[130:131], v12 offset:1024\n\t" \
  "v_mfma_f32_16x16x32_bf16 a[20:23], a[232:235], v[64:67], a[20:23]\n\t" \
  "v_mfma_f32_16x16x32_bf16 a[24:27], a[240:243], v[64:67], a[24:27]\n\t" \
  "ds_read_b64_tr_b16 a[132:133], v12 offset:8192\n\t" \
  "v_mfma_f32_16x16x32_bf16 a[24:27], a[244:247], v[68:71], a[24:27]\n\t" \
  "v_cmp_le_u32 s[92:93], s86, v36\n\t" \
  "v_mfma_f32_16x16x32_bf16 a[28:31], a[252:255], v[68:71], a[28:31]\n\t" \
  "ds_read_b64_tr_b16 a[134:135], v12 offset:9216\n\t" \
  "v_mfma_f32_16x16x32_bf16 a[28:31], a[248:251], v[64:67], a[28:31]\n\t" \
  "ds_read_b128 v[64:67], v10 offset:0\n\t" \
  "ds_read_b128 v[68:71], v11 offset:0\n\t" \
  "v_mfma_f32_16x16x32_bf16 a[60:63], a[248:251], v[72:75], a[60:63]\n\t" \
  "ds_read_b64_tr_b16 a[136:137], v13 offset:0\n\t" \
  "v_mfma_f32_16x16x32_bf16 a[60:63], a[252:255], v[76:79], a[60:63]\n\t" \
  "s_or_b64 s[90:91], s[90:91], s[92:93]\n\t" \
  "v_mfma_f32_16x16x32_bf16 a[56:59], a[244:247], v[76:79], a[56:59]\n\t" \
  "ds_read_b64_tr_b16 a[138:139], v13 offset:1024\n\t" \
  "v_mfma_f32_16x16x32_bf16 a[56:59], a[240:243], v[72:75], a[56:59]\n\t" \
  "s_bitcmp1_b64 s[90:91], s99\n\t" \
  "v_mfma_f32_16x16x32_bf16 a[52:55], a[232:235], v[72:75], a[52:55]\n\t" \
  "ds_read_b64_tr_b16 a[140:141], v13 offset:8192\n\t" \
  "v_mfma_f32_16x16x32_bf16 a[52:55], a[236:239], v[76:79], a[52:55]\n\t" \
  "s_cbranch_scc0 POLL194_%=\n\t" \
  "RET193_%=:\n\t" \
  "v_mfma_f32_16x16x32_bf16 a[48:51], a[228:231], v[76:79], a[48:51]\n\t" \
  "ds_read_b64_tr_b16 a[142:143], v13 offset:9216\n\t" \
  "v_mfma_f32_16x16x32_bf16 a[48:51], a[224:227], v[72:75], a[48:51]\n\t" \
  "v_mfma_f32_16x16x32_bf16 a[44:47], a[216:219], v[72:75], a[44:47]\n\t" \
  "ds_read_b64_tr_b16 a[144:145], v14 offset:0\n\t" \
  "v_mfma_f32_16x16x32_bf16 a[44:47], a[220:223], v[76:79], a[44:47]\n\t" \
  "v_mfma_f32_16x16x32_bf16 a[40:43], a[212:215], v[76:79], a[40:43]\n\t" \
  "ds_read_b64_tr_b16 a[146:147], v14 offset:1024\n\t" \
  "v_mfma_f32_16x16x32_bf16 a[40:43], a[208:211], v[72:75], a[40:43]\n\t" \
  "v_mfma_f32_16x16x32_bf16 a[36:39], a[200:203], v[72:75], a[36:39]\n\t" \
  "ds_read_b64_tr_b16 a[148:149], v14 offset:8192\n\t" \
  "v_mfma_f32_16x16x32_bf16 a[36:39], a[204:207], v[76:79], a[36:39]\n\t" \
  "v_mfma_f32_16x16x32_bf16 a[32:35], a[196:199], v[76:79], a[32:35]\n\t" \
  "ds_read_b64_tr_b16 a[150:151], v14 offset:9216\n\t" \
  "v_mfma_f32_16x16x32_bf16 a[32:35], a[192:195], v[72:75], a[32:35]\n\t" \
  "ds_read_b128 v[72:75], v10 offset:2048\n\t" \
  "ds_read_b128 v[76:79], v11 offset:2048\n\t" \
  "v_mfma_f32_16x16x32_bf16 a[64:67], a[192:195], v[80:83], a[64:67]\n\t" \
  "ds_read_b64_tr_b16 a[152:153], v15 offset:0\n\t" \
  "v_mfma_f32_16x16x32_bf16 a[64:67], a[196:199], v[84:87], a[64:67]\n\t" \
  "v_mfma_f32_16x16x32_bf16 a[68:71], a[204:207], v[84:87], a[68:71]\n\t" \
  "ds_read_b64_tr_b16 a[154:155], v15 offset:1024\n\t" \
  "v_mfma_f32_16x16x32_bf16 a[68:71], a[200:203], v[80:83], a[68:71]\n\t" \
  "v_mfma_f32_16x16x32_bf16 a[72:75], a[208:211], v[80:83], a[72:75]\n\t" \
  "ds_read_b64_tr_b16 a[156:157], v15 offset:8192\n\t" \
  "v_mfma_f32_16x16x32_bf16 a[72:75], a[212:215], v[84:87], a[72:75]\n\t" \
  "v_mfma_f32_16x16x32_bf16 a[76:79], a[220:223], v[84:87], a[76:79]\n\t" \
  "ds_read_b64_tr_b16 a[158:159], v15 offset:9216\n\t" \
  "v_mfma_f32_16x16x32_bf16 a[76:79], a[216:219], v[80:83], a[76:79]\n\t" \
  "s_add_u32 m0, s68, 49152\n\t" \
  "v_mfma_f32_16x16x32_bf16 a[80:83], a[224:227], v[80:83], a[80:83]\n\t" \
  "buffer_load_dwordx4 v0, s[36:39], s65 offen offset:0 sc0 sc1 lds\n\t" \
  "v_mfma_f32_16x16x32_bf16 a[80:83], a[228:231], v[84:87], a[80:83]\n\t" \
  "v_mfma_f32_16x16x32_bf16 a[84:87], a[236:239], v[84:87], a[84:87]\n\t" \
  "v_mfma_f32_16x16x32_bf16 a[84:87], a[232:235], v[80:83], a[84:87]\n\t" \
  "buffer_load_dwordx4 v1, s[36:39], s65 offen offset:1024 sc0 sc1 lds\n\t" \
  "v_mfma_f32_16x16x32_bf16 a[88:91], a[240:243], v[80:83], a[88:91]\n\t" \
  "v_mfma_f32_16x16x32_bf16 a[88:91], a[244:247], v[84:87], a[88:91]\n\t" \
  "v_mfma_f32_16x16x32_bf16 a[92:95], a[252:255], v[84:87], a[92:95]\n\t" \
  "buffer_load_dwordx4 v2, s[36:39], s65 offen offset:2048 sc0 sc1 lds\n\t" \
  "v_mfma_f32_16x16x32_bf16 a[92:95], a[248:251], v[80:83], a[92:95]\n\t" \
  "ds_read_b128 v[80:83], v10 offset:4096\n\t" \
  "ds_read_b128 v[84:87], v11 offset:4096\n\t" \
  "v_mfma_f32_16x16x32_bf16 a[124:127], a[248:251], v[88:91], a[124:127]\n\t" \
  "v_mfma_f32_16x16x32_bf16 a[124:127], a[252:255], v[92:95], a[124:127]\n\t" \
  "buffer_load_dwordx4 v3, s[36:39], s65 offen offset:3072 sc0 sc1 lds\n\t" \
  "v_mfma_f32_16x16x32_bf16 a[120:123], a[244:247], v[92:95], a[120:123]\n\t" \
  "v_mfma_f32_16x16x32_bf16 a[120:123], a[240:243], v[88:91], a[120:123]\n\t" \
  "s_add_u32 m0, s68, 114688\n\t" \
  "v_mfma_f32_16x16x32_bf16 a[116:119], a[232:235], v[88:91], a[116:119]\n\t" \
  "buffer_load_dwordx4 v4, s[40:43], s79 offen offset:0 lds\n\t" \
  "v_mfma_f32_16x16x32_bf16 a[116:119], a[236:239], v[92:95], a[116:119]\n\t" \
  "s_bitcmp1_b64 s[90:91], s99\n\t" \
  "v_mfma_f32_16x16x32_bf16 a[112:115], a[228:231], v[92:95], a[112:115]\n\t" \
  "s_cbranch_scc0 STG195_%=\n\t" \
  "RET189_%=:\n\t" \
  "v_mfma_f32_16x16x32_bf16 a[112:115], a[224:227], v[88:91], a[112:115]\n\t" \
  "buffer_load_dwordx4 v5, s[40:43], s79 offen offset:1024 lds\n\t" \
  "v_mfma_f32_16x16x32_bf16 a[108:111], a[216:219], v[88:91], a[108:111]\n\t" \
  "v_mfma_f32_16x16x32_bf16 a[108:111], a[220:223], v[92:95], a[108:111]\n\t" \
  "v_mfma_f32_16x16x32_bf16 a[104:107], a[212:215], v[92:95], a[104:107]\n\t" \
  "buffer_load_dwordx4 v6, s[40:43], s79 offen offset:2048 lds\n\t" \
  "v_mfma_f32_16x16x32_bf16 a[104:107], a[208:211], v[88:91], a[104:107]\n\t" \
  "v_mfma_f32_16x16x32_bf16 a[100:103], a[200:203], v[88:91], a[100:103]\n\t" \
  "v_mfma_f32_16x16x32_bf16 a[100:103], a[204:207], v[92:95], a[100:103]\n\t" \
  "buffer_load_dwordx4 v7, s[40:43], s79 offen offset:3072 lds\n\t" \
  "v_mfma_f32_16x16x32_bf16 a[96:99], a[196:199], v[92:95], a[96:99]\n\t" \
  "v_mfma_f32_16x16x32_bf16 a[96:99], a[192:195], v[88:91], a[96:99]\n\t" \
  "ds_read_b128 v[88:91], v10 offset:6144\n\t" \
  "ds_read_b128 v[92:95], v11 offset:6144\n\t" \
  "v_mfma_f32_16x16x32_bf16 v[128:131], a[192:195], v[96:99], v[128:131]\n\t" \
  "s_add_u32 s77, s77, 1\n\t" \
  "v_mfma_f32_16x16x32_bf16 v[128:131], a[196:199], v[100:103], v[128:131]\n\t" \
  "v_readlane_b32 s66, v16, s77\n\t" \
  "v_mfma_f32_16x16x32_bf16 v[132:135], a[204:207], v[100:103], v[132:135]\n\t" \
  "v_mfma_f32_16x16x32_bf16 v[132:135], a[200:203], v[96:99], v[132:135]\n\t" \
  "v_mfma_f32_16x16x32_bf16 v[136:139], a[208:211], v[96:99], v[136:139]\n\t" \
  "s_lshl_b32 s78, s66, 13\n\t" \
  "v_mfma_f32_16x16x32_bf16 v[136:139], a[212:215], v[100:103], v[136:139]\n\t" \
  "s_lshr_b32 s65, s66, 11\n\t" \
  "v_mfma_f32_16x16x32_bf16 v[140:143], a[220:223], v[100:103], v[140:143]\n\t" \
  "s_mul_i32 s64, s66, s67\n\t" \
  "v_mfma_f32_16x16x32_bf16 v[140:143], a[216:219], v[96:99], v[140:143]\n\t" \
  "s_waitcnt vmcnt(18) lgkmcnt(0)\n\t" \
  "s_barrier\n\t" \
  "v_mfma_f32_16x16x32_bf16 v[144:147], a[224:227], v[96:99], v[144:147]\n\t" \
  "ds_read_b64_tr_b16 a[160:161], v12 offset:16384\n\t" \
  "v_mfma_f32_16x16x32_bf16 v[144:147], a[228:231], v[100:103], v[144:147]\n\t" \
  "s_add_u32 s79, s78, 256\n\t" \
  "v_mfma_f32_16x16x32_bf16 v[148:151], a[236:239], v[100:103], v[148:151]\n\t" \
  "ds_read_b64_tr_b16 a[162:163], v12 offset:17408\n\t" \
  "v_mfma_f32_16x16x32_bf16 v[148:151], a[232:235], v[96:99], v[148:151]\n\t" \
  "s_mul_i32 s65, s65, s89\n\t" \
  "v_mfma_f32_16x16x32_bf16 v[152:155], a[240:243], v[96:99], v[152:155]\n\t" \
  "ds_read_b64_tr_b16 a[164:165], v12 offset:24576\n\t" \
  "v_mfma_f32_16x16x32_bf16 v[152:155], a[244:247], v[100:103], v[152:155]\n\t" \
  "s_add_u32 s65, s65, s64\n\t" \
  "v_mfma_f32_16x16x32_bf16 v[156:159], a[252:255], v[100:103], v[156:159]\n\t" \
  "ds_read_b64_tr_b16 a[166:167], v12 offset:25600\n\t" \
  "v_mfma_f32_16x16x32_bf16 v[156:159], a[248:251], v[96:99], v[156:159]\n\t" \
  "ds_read_b128 v[96:99], v10 offset:16384\n\t" \
  "ds_read_b128 v[100:103], v11 offset:16384\n\t" \
  "v_mfma_f32_16x16x32_bf16 v[188:191], a[248:251], v[104:107], v[188:191]\n\t" \
  "ds_read_b64_tr_b16 a[168:169], v13 offset:16384\n\t" \
  "v_mfma_f32_16x16x32_bf16 v[188:191], a[252:255], v[108:111], v[188:191]\n\t" \
  "s_add_u32 s64, s65, s69\n\t" \
  "v_mfma_f32_16x16x32_bf16 v[184:187], a[244:247], v[108:111], v[184:187]\n\t" \
  "ds_read_b64_tr_b16 a[170:171], v13 offset:17408\n\t" \
  "v_mfma_f32_16x16x32_bf16 v[184:187], a[240:243], v[104:107], v[184:187]\n\t" \
  "s_add_u32 s65, s65, s70\n\t" \
  "v_mfma_f32_16x16x32_bf16 v[180:183], a[232:235], v[104:107], v[180:183]\n\t" \
  "ds_read_b64_tr_b16 a[172:173], v13 offset:24576\n\t" \
  "v_mfma_f32_16x16x32_bf16 v[180:183], a[236:239], v[108:111], v[180:183]\n\t" \
  "v_mfma_f32_16x16x32_bf16 v[176:179], a[228:231], v[108:111], v[176:179]\n\t" \
  "ds_read_b64_tr_b16 a[174:175], v13 offset:25600\n\t" \
  "v_mfma_f32_16x16x32_bf16 v[176:179], a[224:227], v[104:107], v[176:179]\n\t" \
  "v_mfma_f32_16x16x32_bf16 v[172:175], a[216:219], v[104:107], v[172:175]\n\t" \
  "ds_read_b64_tr_b16 a[176:177], v14 offset:16384\n\t" \
  "v_mfma_f32_16x16x32_bf16 v[172:175], a[220:223], v[108:111], v[172:175]\n\t" \
  "v_mfma_f32_16x16x32_bf16 v[168:171], a[212:215], v[108:111], v[168:171]\n\t" \
  "ds_read_b64_tr_b16 a[178:179], v14 offset:17408\n\t" \
  "v_mfma_f32_16x16x32_bf16 v[168:171], a[208:211], v[104:107], v[168:171]\n\t" \
  "v_mfma_f32_16x16x32_bf16 v[164:167], a[200:203], v[104:107], v[164:167]\n\t" \
  "ds_read_b64_tr_b16 a[180:181], v14 offset:24576\n\t" \
  "v_mfma_f32_16x16x32_bf16 v[164:167], a[204:207], v[108:111], v[164:167]\n\t" \
  "v_mfma_f32_16x16x32_bf16 v[160:163], a[196:199], v[108:111], v[160:163]\n\t" \
  "ds_read_b64_tr_b16 a[182:183], v14 offset:25600\n\t" \
  "v_mfma_f32_16x16x32_bf16 v[160:163], a[192:195], v[104:107], v[160:163]\n\t" \
  "ds_read_b128 v[104:107], v10 offset:18432\n\t" \
  "ds_read_b128 v[108:111], v11 offset:18432\n\t" \
  "v_mfma_f32_16x16x32_bf16 v[192:195], a[192:195], v[112:115], v[192:195]\n\t" \
  "ds_read_b64_tr_b16 a[184:185], v15 offset:16384\n\t" \
  "v_mfma_f32_16x16x32_bf16 v[192:195], a[196:199], v[116:119], v[192:195]\n\t" \
  "v_mfma_f32_16x16x32_bf16 v[196:199], a[204:207], v[116:119], v[196:199]\n\t" \
  "ds_read_b64_tr_b16 a[186:187], v15 offset:17408\n\t" \
  "v_mfma_f32_16x16x32_bf16 v[196:199], a[200:203], v[112:115], v[196:199]\n\t" \
  "v_mfma_f32_16x16x32_bf16 v[200:203], a[208:211], v[112:115], v[200:203]\n\t" \
  "ds_read_b64_tr_b16 a[188:189], v15 offset:24576\n\t" \
  "v_mfma_f32_16x16x32_bf16 v[200:203], a[212:215], v[116:119], v[200:203]\n\t" \
  "v_mfma_f32_16x16x32_bf16 v[204:207], a[220:223], v[116:119], v[204:207]\n\t" \
  "ds_read_b64_tr_b16 a[190:191], v15 offset:25600\n\t" \
  "v_mfma_f32_16x16x32_bf16 v[204:207], a[216:219], v[112:115], v[204:207]\n\t" \
  "s_add_u32 m0, s68, 0\n\t" \
  "v_mfma_f32_16x16x32_bf16 v[208:211], a[224:227], v[112:115], v[208:211]\n\t" \
  "buffer_load_dwordx4 v0, s[36:39], s64 offen offset:0 sc0 sc1 lds\n\t" \
  "v_mfma_f32_16x16x32_bf16 v[208:211], a[228:231], v[116:119], v[208:211]\n\t" \
  "v_mfma_f32_16x16x32_bf16 v[212:215], a[236:239], v[116:119], v[212:215]\n\t" \
  "v_mfma_f32_16x16x32_bf16 v[212:215], a[232:235], v[112:115], v[212:215]\n\t" \
  "buffer_load_dwordx4 v1, s[36:39], s64 offen offset:1024 sc0 sc1 lds\n\t" \
  "v_mfma_f32_16x16x32_bf16 v[216:219], a[240:243], v[112:115], v[216:219]\n\t" \
  "v_mfma_f32_16x16x32_bf16 v[216:219], a[244:247], v[116:119], v[216:219]\n\t" \
  "v_mfma_f32_16x16x32_bf16 v[220:223], a[252:255], v[116:119], v[220:223]\n\t" \
  "buffer_load_dwordx4 v2, s[36:39], s64 offen offset:2048 sc0 sc1 lds\n\t" \
  "v_mfma_f32_16x16x32_bf16 v[220:223], a[248:251], v[112:115], v[220:223]\n\t" \
  "ds_read_b128 v[112:115], v10 offset:20480\n\t" \
  "ds_read_b128 v[116:119], v11 offset:20480\n\t" \
  "v_mfma_f32_16x16x32_bf16 v[252:255], a[248:251], v[120:123], v[252:255]\n\t" \
  "v_mfma_f32_16x16x32_bf16 v[252:255], a[252:255], v[124:127], v[252:255]\n\t" \
  "buffer_load_dwordx4 v3, s[36:39], s64 offen offset:3072 sc0 sc1 lds\n\t" \
  "v_mfma_f32_16x16x32_bf16 v[248:251], a[244:247], v[124:127], v[248:251]\n\t" \
  "v_mfma_f32_16x16x32_bf16 v[248:251], a[240:243], v[120:123], v[248:251]\n\t" \
  "s_add_u32 m0, s68, 65536\n\t" \
  "v_mfma_f32_16x16x32_bf16 v[244:247], a[232:235], v[120:123], v[244:247]\n\t" \
  "buffer_load_dwordx4 v4, s[40:43], s78 offen offset:0 lds\n\t" \
  "v_mfma_f32_16x16x32_bf16 v[244:247], a[236:239], v[124:127], v[244:247]\n\t" \
  "v_mfma_f32_16x16x32_bf16 v[240:243], a[228:231], v[124:127], v[240:243]\n\t" \
  "v_mfma_f32_16x16x32_bf16 v[240:243], a[224:227], v[120:123], v[240:243]\n\t" \
  "buffer_load_dwordx4 v5, s[40:43], s78 offen offset:1024 lds\n\t" \
  "v_mfma_f32_16x16x32_bf16 v[236:239], a[216:219], v[120:123], v[236:239]\n\t" \
  "v_mfma_f32_16x16x32_bf16 v[236:239], a[220:223], v[124:127], v[236:239]\n\t" \
  "v_mfma_f32_16x16x32_bf16 v[232:235], a[212:215], v[124:127], v[232:235]\n\t" \
  "buffer_load_dwordx4 v6, s[40:43], s78 offen offset:2048 lds\n\t" \
  "v_mfma_f32_16x16x32_bf16 v[232:235], a[208:211], v[120:123], v[232:235]\n\t" \
  "v_mfma_f32_16x16x32_bf16 v[228:231], a[200:203], v[120:123], v[228:231]\n\t" \
  "v_mfma_f32_16x16x32_bf16 v[228:231], a[204:207], v[124:127], v[228:231]\n\t" \
  "buffer_load_dwordx4 v7, s[40:43], s78 offen offset:3072 lds\n\t" \
  "v_mfma_f32_16x16x32_bf16 v[224:227], a[196:199], v[124:127], v[224:227]\n\t" \
  "v_mfma_f32_16x16x32_bf16 v[224:227], a[192:195], v[120:123], v[224:227]\n\t" \
  "ds_read_b128 v[120:123], v10 offset:22528\n\t" \
  "ds_read_b128 v[124:127], v11 offset:22528\n\t" \
  "v_mfma_f32_16x16x32_bf16 a[0:3], a[128:131], v[64:67], a[0:3]\n\t" \
  "v_mfma_f32_16x16x32_bf16 a[0:3], a[132:135], v[68:71], a[0:3]\n\t" \
  "v_mfma_f32_16x16x32_bf16 a[4:7], a[140:143], v[68:71], a[4:7]\n\t" \
  "v_mfma_f32_16x16x32_bf16 a[4:7], a[136:139], v[64:67], a[4:7]\n\t" \
  "v_mfma_f32_16x16x32_bf16 a[8:11], a[144:147], v[64:67], a[8:11]\n\t" \
  "v_mfma_f32_16x16x32_bf16 a[8:11], a[148:151], v[68:71], a[8:11]\n\t" \
  "v_mfma_f32_16x16x32_bf16 a[12:15], a[156:159], v[68:71], a[12:15]\n\t" \
  "v_mfma_f32_16x16x32_bf16 a[12:15], a[152:155], v[64:67], a[12:15]\n\t" \
  "s_waitcnt vmcnt(17) lgkmcnt(0)\n\t" \
  "s_barrier\n\t" \
  "v_mfma_f32_16x16x32_bf16 a[16:19], a[160:163], v[64:67], a[16:19]\n\t" \
  "ds_read_b64_tr_b16 a[192:193], v12 offset:32768\n\t" \
  "v_mfma_f32_16x16x32_bf16 a[16:19], a[164:167], v[68:71], a[16:19]\n\t" \
  "s_andn2_b64 s[92:93], 255, s[58:59]\n\t" \
  "s_cbranch_scc1 POLL201_%=\n\t" \
  "RET200_%=:\n\t" \
  "v_mfma_f32_16x16x32_bf16 a[20:23], a[172:175], v[68:71], a[20:23]\n\t" \
  "ds_read_b64_tr_b16 a[194:195], v12 offset:33792\n\t" \
  "v_mfma_f32_16x16x32_bf16 a[20:23], a[168:171], v[64:67], a[20:23]\n\t" \
  "v_mfma_f32_16x16x32_bf16 a[24:27], a[176:179], v[64:67], a[24:27]\n\t" \
  "ds_read_b64_tr_b16 a[196:197], v12 offset:40960\n\t" \
  "v_mfma_f32_16x16x32_bf16 a[24:27], a[180:183], v[68:71], a[24:27]\n\t" \
  "v_mfma_f32_16x16x32_bf16 a[28:31], a[188:191], v[68:71], a[28:31]\n\t" \
  "ds_read_b64_tr_b16 a[198:199], v12 offset:41984\n\t" \
  "v_mfma_f32_16x16x32_bf16 a[28:31], a[184:187], v[64:67], a[28:31]\n\t" \
  "ds_read_b128 v[64:67], v10 offset:32768\n\t" \
  "ds_read_b128 v[68:71], v11 offset:32768\n\t" \
  "v_mfma_f32_16x16x32_bf16 a[60:63], a[184:187], v[72:75], a[60:63]\n\t" \
  "ds_read_b64_tr_b16 a[200:201], v13 offset:32768\n\t" \
  "v_mfma_f32_16x16x32_bf16 a[60:63], a[188:191], v[76:79], a[60:63]\n\t" \
  "v_mfma_f32_16x16x32_bf16 a[56:59], a[180:183], v[76:79], a[56:59]\n\t" \
  "ds_read_b64_tr_b16 a[202:203], v13 offset:33792\n\t" \
  "v_mfma_f32_16x16x32_bf16 a[56:59], a[176:179], v[72:75], a[56:59]\n\t" \
  "v_mfma_f32_16x16x32_bf16 a[52:55], a[168:171], v[72:75], a[52:55]\n\t" \
  "ds_read_b64_tr_b16 a[204:205], v13 offset:40960\n\t" \
  "v_mfma_f32_16x16x32_bf16 a[52:55], a[172:175], v[76:79], a[52:55]\n\t" \
  "v_mfma_f32_16x16x32_bf16 a[48:51], a[164:167], v[76:79], a[48:51]\n\t" \
  "ds_read_b64_tr_b16 a[206:207], v13 offset:41984\n\t" \
  "v_mfma_f32_16x16x32_bf16 a[48:51], a[160:163], v[72:75], a[48:51]\n\t" \
  "v_mfma_f32_16x16x32_bf16 a[44:47], a[152:155], v[72:75], a[44:47]\n\t" \
  "ds_read_b64_tr_b16 a[208:209], v14 offset:32768\n\t" \
  "v_mfma_f32_16x16x32_bf16 a[44:47], a[156:159], v[76:79], a[44:47]\n\t" \
  "v_mfma_f32_16x16x32_bf16 a[40:43], a[148:151], v[76:79], a[40:43]\n\t" \
  "ds_read_b64_tr_b16 a[210:211], v14 offset:33792\n\t" \
  "v_mfma_f32_16x16x32_bf16 a[40:43], a[144:147], v[72:75], a[40:43]\n\t" \
  "v_mfma_f32_16x16x32_bf16 a[36:39], a[136:139], v[72:75], a[36:39]\n\t" \
  "ds_read_b64_tr_b16 a[212:213], v14 offset:40960\n\t" \
  "v_mfma_f32_16x16x32_bf16 a[36:39], a[140:143], v[76:79], a[36:39]\n\t" \
  "v_mfma_f32_16x16x32_bf16 a[32:35], a[132:135], v[76:79], a[32:35]\n\t" \
  "ds_read_b64_tr_b16 a[214:215], v14 offset:41984\n\t" \
  "v_mfma_f32_16x16x32_bf16 a[32:35], a[128:131], v[72:75], a[32:35]\n\t" \
  "ds_read_b128 v[72:75], v10 offset:34816\n\t" \
  "ds_read_b128 v[76:79], v11 offset:34816\n\t" \
  "v_mfma_f32_16x16x32_bf16 a[64:67], a[128:131], v[80:83], a[64:67]\n\t" \
  "ds_read_b64_tr_b16 a[216:217], v15 offset:32768\n\t" \
  "v_mfma_f32_16x16x32_bf16 a[64:67], a[132:135], v[84:87], a[64:67]\n\t" \
  "v_mfma_f32_16x16x32_bf16 a[68:71], a[140:143], v[84:87], a[68:71]\n\t" \
  "ds_read_b64_tr_b16 a[218:219], v15 offset:33792\n\t" \
  "v_mfma_f32_16x16x32_bf16 a[68:71], a[136:139], v[80:83], a[68:71]\n\t" \
  "v_mfma_f32_16x16x32_bf16 a[72:75], a[144:147], v[80:83], a[72:75]\n\t" \
  "ds_read_b64_tr_b16 a[220:221], v15 offset:40960\n\t" \
  "v_mfma_f32_16x16x32_bf16 a[72:75], a[148:151], v[84:87], a[72:75]\n\t" \
  "v_mfma_f32_16x16x32_bf16 a[76:79], a[156:159], v[84:87], a[76:79]\n\t" \
  "ds_read_b64_tr_b16 a[222:223], v15 offset:41984\n\t" \
  "v_mfma_f32_16x16x32_bf16 a[76:79], a[152:155], v[80:83], a[76:79]\n\t" \
  "s_add_u32 m0, s68, 16384\n\t" \
  "v_mfma_f32_16x16x32_bf16 a[80:83], a[160:163], v[80:83], a[80:83]\n\t" \
  "buffer_load_dwordx4 v0, s[36:39], s65 offen offset:0 sc0 sc1 lds\n\t" \
  "v_mfma_f32_16x16x32_bf16 a[80:83], a[164:167], v[84:87], a[80:83]\n\t" \
  "v_mfma_f32_16x16x32_bf16 a[84:87], a[172:175], v[84:87], a[84:87]\n\t" \
  "v_mfma_f32_16x16x32_bf16 a[84:87], a[168:171], v[80:83], a[84:87]\n\t" \
  "buffer_load_dwordx4 v1, s[36:39], s65 offen offset:1024 sc0 sc1 lds\n\t" \
  "v_mfma_f32_16x16x32_bf16 a[88:91], a[176:179], v[80:83], a[88:91]\n\t" \
  "v_mfma_f32_16x16x32_bf16 a[88:91], a[180:183], v[84:87], a[88:91]\n\t" \
  "v_mfma_f32_16x16x32_bf16 a[92:95], a[188:191], v[84:87], a[92:95]\n\t" \
  "buffer_load_dwordx4 v2, s[36:39], s65 offen offset:2048 sc0 sc1 lds\n\t" \
  "v_mfma_f32_16x16x32_bf16 a[92:95], a[184:187], v[80:83], a[92:95]\n\t" \
  "ds_read_b128 v[80:83], v10 offset:36864\n\t" \
  "ds_read_b128 v[84:87], v11 offset:36864\n\t" \
  "v_mfma_f32_16x16x32_bf16 a[124:127], a[184:187], v[88:91], a[124:127]\n\t" \
  "v_mfma_f32_16x16x32_bf16 a[124:127], a[188:191], v[92:95], a[124:127]\n\t" \
  "buffer_load_dwordx4 v3, s[36:39], s65 offen offset:3072 sc0 sc1 lds\n\t" \
  "v_mfma_f32_16x16x32_bf16 a[120:123], a[180:183], v[92:95], a[120:123]\n\t" \
  "v_mfma_f32_16x16x32_bf16 a[120:123], a[176:179], v[88:91], a[120:123]\n\t" \
  "s_add_u32 m0, s68, 81920\n\t" \
  "v_mfma_f32_16x16x32_bf16 a[116:119], a[168:171], v[88:91], a[116:119]\n\t" \
  "buffer_load_dwordx4 v4, s[40:43], s79 offen offset:0 lds\n\t" \
  "v_mfma_f32_16x16x32_bf16 a[116:119], a[172:175], v[92:95], a[116:119]\n\t" \
  "v_mfma_f32_16x16x32_bf16 a[112:115], a[164:167], v[92:95], a[112:115]\n\t" \
  "v_mfma_f32_16x16x32_bf16 a[112:115], a[160:163], v[88:91], a[112:115]\n\t" \
  "buffer_load_dwordx4 v5, s[40:43], s79 offen offset:1024 lds\n\t" \
  "v_mfma_f32_16x16x32_bf16 a[108:111], a[152:155], v[88:91], a[108:111]\n\t" \
  "v_mfma_f32_16x16x32_bf16 a[108:111], a[156:159], v[92:95], a[108:111]\n\t" \
  "v_mfma_f32_16x16x32_bf16 a[104:107], a[148:151], v[92:95], a[104:107]\n\t" \
  "buffer_load_dwordx4 v6, s[40:43], s79 offen offset:2048 lds\n\t" \
  "v_mfma_f32_16x16x32_bf16 a[104:107], a[144:147], v[88:91], a[104:107]\n\t" \
  "v_mfma_f32_16x16x32_bf16 a[100:103], a[136:139], v[88:91], a[100:103]\n\t" \
  "v_mfma_f32_16x16x32_bf16 a[100:103], a[140:143], v[92:95], a[100:103]\n\t" \
  "buffer_load_dwordx4 v7, s[40:43], s79 offen offset:3072 lds\n\t" \
  "v_mfma_f32_16x16x32_bf16 a[96:99], a[132:135], v[92:95], a[96:99]\n\t" \
  "v_mfma_f32_16x16x32_bf16 a[96:99], a[128:131], v[88:91], a[96:99]\n\t" \
  "ds_read_b128 v[88:91], v10 offset:38912\n\t" \
  "ds_read_b128 v[92:95], v11 offset:38912\n\t" \
  "v_mfma_f32_16x16x32_bf16 v[128:131], a[128:131], v[96:99], v[128:131]\n\t" \
  "v_mfma_f32_16x16x32_bf16 v[128:131], a[132:135], v[100:103], v[128:131]\n\t" \
  "v_mfma_f32_16x16x32_bf16 v[132:135], a[140:143], v[100:103], v[132:135]\n\t" \
  "v_mfma_f32_16x16x32_bf16 v[132:135], a[136:139], v[96:99], v[132:135]\n\t" \
  "v_mfma_f32_16x16x32_bf16 v[136:139], a[144:147], v[96:99], v[136:139]\n\t" \
  "v_mfma_f32_16x16x32_bf16 v[136:139], a[148:151], v[100:103], v[136:139]\n\t" \
  "v_mfma_f32_16x16x32_bf16 v[140:143], a[156:159], v[100:103], v[140:143]\n\t" \
  "v_mfma_f32_16x16x32_bf16 v[140:143], a[152:155], v[96:99], v[140:143]\n\t" \
  "s_waitcnt vmcnt(16) lgkmcnt(0)\n\t" \
  "s_barrier\n\t" \
  "v_mfma_f32_16x16x32_bf16 v[144:147], a[160:163], v[96:99], v[144:147]\n\t" \
  "ds_read_b64_tr_b16 a[224:225], v12 offset:49152\n\t" \
  "v_readlane_b32 s66, v17, 0\n\t" \
  "s_nop 0\n\t" \
  "s_lshl_b32 s78, s66, 13\n\t" \
  "s_lshl_b32 s65, s66, 10\n\t" \
  "s_add_u32 s79, s78, 256\n\t" \
  "s_add_u32 s65, s65, s70\n\t" \
  "s_nop 1\n\t" \
  "v_mfma_f32_16x16x32_bf16 v[144:147], a[164:167], v[100:103], v[144:147]\n\t" \
  "v_mfma_f32_16x16x32_bf16 v[148:151], a[172:175], v[100:103], v[148:151]\n\t" \
  "ds_read_b64_tr_b16 a[226:227], v12 offset:50176\n\t" \
  "v_mfma_f32_16x16x32_bf16 v[148:151], a[168:171], v[96:99], v[148:151]\n\t" \
  "v_mfma_f32_16x16x32_bf16 v[152:155], a[176:179], v[96:99], v[152:155]\n\t" \
  "ds_read_b64_tr_b16 a[228:229], v12 offset:57344\n\t" \
  "v_mfma_f32_16x16x32_bf16 v[152:155], a[180:183], v[100:103], v[152:155]\n\t" \
  "v_mfma_f32_16x16x32_bf16 v[156:159], a[188:191], v[100:103], v[156:159]\n\t" \
  "ds_read_b64_tr_b16 a[230:231], v12 offset:58368\n\t" \
  "v_mfma_f32_16x16x32_bf16 v[156:159], a[184:187], v[96:99], v[156:159]\n\t" \
  "ds_read_b128 v[96:99], v10 offset:49152\n\t" \
  "ds_read_b128 v[100:103], v11 offset:49152\n\t" \
  "v_mfma_f32_16x16x32_bf16 v[188:191], a[184:187], v[104:107], v[188:191]\n\t" \
  "ds_read_b64_tr_b16 a[232:233], v13 offset:49152\n\t" \
  "v_mfma_f32_16x16x32_bf16 v[188:191], a[188:191], v[108:111], v[188:191]\n\t" \
  "v_mfma_f32_16x16x32_bf16 v[184:187], a[180:183], v[108:111], v[184:187]\n\t" \
  "ds_read_b64_tr_b16 a[234:235], v13 offset:50176\n\t" \
  "v_mfma_f32_16x16x32_bf16 v[184:187], a[176:179], v[104:107], v[184:187]\n\t" \
  "v_mfma_f32_16x16x32_bf16 v[180:183], a[168:171], v[104:107], v[180:183]\n\t" \
  "ds_read_b64_tr_b16 a[236:237], v13 offset:57344\n\t" \
  "v_mfma_f32_16x16x32_bf16 v[180:183], a[172:175], v[108:111], v[180:183]\n\t" \
  "v_mfma_f32_16x16x32_bf16 v[176:179], a[164:167], v[108:111], v[176:179]\n\t" \
  "ds_read_b64_tr_b16 a[238:239], v13 offset:58368\n\t" \
  "v_mfma_f32_16x16x32_bf16 v[176:179], a[160:163], v[104:107], v[176:179]\n\t" \
  "v_mfma_f32_16x16x32_bf16 v[172:175], a[152:155], v[104:107], v[172:175]\n\t" \
  "ds_read_b64_tr_b16 a[240:241], v14 offset:49152\n\t" \
  "v_mfma_f32_16x16x32_bf16 v[172:175], a[156:159], v[108:111], v[172:175]\n\t" \
  "v_mfma_f32_16x16x32_bf16 v[168:171], a[148:151], v[108:111], v[168:171]\n\t" \
  "ds_read_b64_tr_b16 a[242:243], v14 offset:50176\n\t" \
  "v_mfma_f32_16x16x32_bf16 v[168:171], a[144:147], v[104:107], v[168:171]\n\t" \
  "v_mfma_f32_16x16x32_bf16 v[164:167], a[136:139], v[104:107], v[164:167]\n\t" \
  "ds_read_b64_tr_b16 a[244:245], v14 offset:57344\n\t" \
  "v_mfma_f32_16x16x32_bf16 v[164:167], a[140:143], v[108:111], v[164:167]\n\t" \
  "v_mfma_f32_16x16x32_bf16 v[160:163], a[132:135], v[108:111], v[160:163]\n\t" \
  "ds_read_b64_tr_b16 a[246:247], v14 offset:58368\n\t" \
  "v_mfma_f32_16x16x32_bf16 v[160:163], a[128:131], v[104:107], v[160:163]\n\t" \
  "ds_read_b128 v[104:107], v10 offset:51200\n\t" \
  "ds_read_b128 v[108:111], v11 offset:51200\n\t" \
  "v_mfma_f32_16x16x32_bf16 v[192:195], a[128:131], v[112:115], v[192:195]\n\t" \
  "ds_read_b64_tr_b16 a[248:249], v15 offset:49152\n\t" \
  "v_mfma_f32_16x16x32_bf16 v[192:195], a[132:135], v[116:119], v[192:195]\n\t" \
  "v_mfma_f32_16x16x32_bf16 v[196:199], a[140:143], v[116:119], v[196:199]\n\t" \
  "ds_read_b64_tr_b16 a[250:251], v15 offset:50176\n\t" \
  "v_mfma_f32_16x16x32_bf16 v[196:199], a[136:139], v[112:115], v[196:199]\n\t" \
  "v_mfma_f32_16x16x32_bf16 v[200:203], a[144:147], v[112:115], v[200:203]\n\t" \
  "ds_read_b64_tr_b16 a[252:253], v15 offset:57344\n\t" \
  "v_mfma_f32_16x16x32_bf16 v[200:203], a[148:151], v[116:119], v[200:203]\n\t" \
  "v_mfma_f32_16x16x32_bf16 v[204:207], a[156:159], v[116:119], v[204:207]\n\t" \
  "ds_read_b64_tr_b16 a[254:255], v15 offset:58368\n\t" \
  "v_mfma_f32_16x16x32_bf16 v[204:207], a[152:155], v[112:115], v[204:207]\n\t" \
  "s_add_u32 m0, s68, 98304\n\t" \
  "v_mfma_f32_16x16x32_bf16 v[208:211], a[160:163], v[112:115], v[208:211]\n\t" \
  "buffer_load_dwordx4 v4, s[40:43], s78 offen offset:0 lds\n\t" \
  "v_mfma_f32_16x16x32_bf16 v[208:211], a[164:167], v[116:119], v[208:211]\n\t" \
  "v_mfma_f32_16x16x32_bf16 v[212:215], a[172:175], v[116:119], v[212:215]\n\t" \
  "v_mfma_f32_16x16x32_bf16 v[212:215], a[168:171], v[112:115], v[212:215]\n\t" \
  "v_mfma_f32_16x16x32_bf16 v[216:219], a[176:179], v[112:115], v[216:219]\n\t" \
  "v_mfma_f32_16x16x32_bf16 v[216:219], a[180:183], v[116:119], v[216:219]\n\t" \
  "v_mfma_f32_16x16x32_bf16 v[220:223], a[188:191], v[116:119], v[220:223]\n\t" \
  "v_mfma_f32_16x16x32_bf16 v[220:223], a[184:187], v[112:115], v[220:223]\n\t" \
  "ds_read_b128 v[112:115], v10 offset:53248\n\t" \
  "ds_read_b128 v[116:119], v11 offset:53248\n\t" \
  "buffer_load_dwordx4 v5, s[40:43], s78 offen offset:1024 lds\n\t" \
  "v_mfma_f32_16x16x32_bf16 v[252:255], a[184:187], v[120:123], v[252:255]\n\t" \
  "v_mfma_f32_16x16x32_bf16 v[252:255], a[188:191], v[124:127], v[252:255]\n\t" \
  "v_mfma_f32_16x16x32_bf16 v[248:251], a[180:183], v[124:127], v[248:251]\n\t" \
  "v_mfma_f32_16x16x32_bf16 v[248:251], a[176:179], v[120:123], v[248:251]\n\t" \
  "v_mfma_f32_16x16x32_bf16 v[244:247], a[168:171], v[120:123], v[244:247]\n\t" \
  "v_cmp_le_u32 s[92:93], s86, v37\n\t" \
  "v_mfma_f32_16x16x32_bf16 v[244:247], a[172:175], v[124:127], v[244:247]\n\t" \
  "v_mfma_f32_16x16x32_bf16 v[240:243], a[164:167], v[124:127], v[240:243]\n\t" \
  "buffer_load_dwordx4 v6, s[40:43], s78 offen offset:2048 lds\n\t" \
  "v_mfma_f32_16x16x32_bf16 v[240:243], a[160:163], v[120:123], v[240:243]\n\t" \
  "s_or_b64 s[58:59], s[58:59], s[92:93]\n\t" \
  "v_mfma_f32_16x16x32_bf16 v[236:239], a[152:155], v[120:123], v[236:239]\n\t" \
  "v_mfma_f32_16x16x32_bf16 v[236:239], a[156:159], v[124:127], v[236:239]\n\t" \
  "s_bitcmp1_b64 s[58:59], 0\n\t" \
  "s_cbranch_scc0 SLOW203_%=\n\t" \
  "RET202_%=:\n\t" \
  "v_mfma_f32_16x16x32_bf16 v[232:235], a[148:151], v[124:127], v[232:235]\n\t" \
  "v_mfma_f32_16x16x32_bf16 v[232:235], a[144:147], v[120:123], v[232:235]\n\t" \
  "v_mfma_f32_16x16x32_bf16 v[228:231], a[136:139], v[120:123], v[228:231]\n\t" \
  "v_mfma_f32_16x16x32_bf16 v[228:231], a[140:143], v[124:127], v[228:231]\n\t" \
  "v_mfma_f32_16x16x32_bf16 v[224:227], a[132:135], v[124:127], v[224:227]\n\t" \
  "buffer_load_dwordx4 v7, s[40:43], s78 offen offset:3072 lds\n\t" \
  "v_mfma_f32_16x16x32_bf16 v[224:227], a[128:131], v[120:123], v[224:227]\n\t" \
  "ds_read_b128 v[120:123], v10 offset:55296\n\t" \
  "ds_read_b128 v[124:127], v11 offset:55296\n\t" \
  "v_mfma_f32_16x16x32_bf16 a[0:3], a[192:195], v[64:67], a[0:3]\n\t" \
  "v_mfma_f32_16x16x32_bf16 a[0:3], a[196:199], v[68:71], a[0:3]\n\t" \
  "v_mfma_f32_16x16x32_bf16 a[4:7], a[204:207], v[68:71], a[4:7]\n\t" \
  "v_mfma_f32_16x16x32_bf16 a[4:7], a[200:203], v[64:67], a[4:7]\n\t" \
  "v_mfma_f32_16x16x32_bf16 a[8:11], a[208:211], v[64:67], a[8:11]\n\t" \
  "v_mfma_f32_16x16x32_bf16 a[8:11], a[212:215], v[68:71], a[8:11]\n\t" \
  "v_mfma_f32_16x16x32_bf16 a[12:15], a[220:223], v[68:71], a[12:15]\n\t" \
  "v_mfma_f32_16x16x32_bf16 a[12:15], a[216:219], v[64:67], a[12:15]\n\t" \
  "s_waitcnt vmcnt(12) lgkmcnt(0)\n\t" \
  "s_barrier\n\t" \
  "v_mfma_f32_16x16x32_bf16 a[16:19], a[224:227], v[64:67], a[16:19]\n\t" \
  "ds_read_b64_tr_b16 a[128:129], v12 offset:0\n\t" \
  "v_mfma_f32_16x16x32_bf16 a[16:19], a[228:231], v[68:71], a[16:19]\n\t" \
  "s_andn2_b64 s[92:93], 255, s[58:59]\n\t" \
  "s_cbranch_scc1 POLL207_%=\n\t" \
  "RET206_%=:\n\t" \
  "v_mfma_f32_16x16x32_bf16 a[20:23], a[236:239], v[68:71], a[20:23]\n\t" \
  "ds_read_b64_tr_b16 a[130:131], v12 offset:1024\n\t" \
  "v_mfma_f32_16x16x32_bf16 a[20:23], a[232:235], v[64:67], a[20:23]\n\t" \
  "v_mfma_f32_16x16x32_bf16 a[24:27], a[240:243], v[64:67], a[24:27]\n\t" \
  "ds_read_b64_tr_b16 a[132:133], v12 offset:8192\n\t" \
  "v_mfma_f32_16x16x32_bf16 a[24:27], a[244:247], v[68:71], a[24:27]\n\t" \
  "v_mfma_f32_16x16x32_bf16 a[28:31], a[252:255], v[68:71], a[28:31]\n\t" \
  "ds_read_b64_tr_b16 a[134:135], v12 offset:9216\n\t" \
  "v_mfma_f32_16x16x32_bf16 a[28:31], a[248:251], v[64:67], a[28:31]\n\t" \
  "ds_read_b128 v[64:67], v10 offset:0\n\t" \
  "ds_read_b128 v[68:71], v11 offset:0\n\t" \
  "v_mfma_f32_16x16x32_bf16 a[60:63], a[248:251], v[72:75], a[60:63]\n\t" \
  "ds_read_b64_tr_b16 a[136:137], v13 offset:0\n\t" \
  "v_mfma_f32_16x16x32_bf16 a[60:63], a[252:255], v[76:79], a[60:63]\n\t" \
  "v_mfma_f32_16x16x32_bf16 a[56:59], a[244:247], v[76:79], a[56:59]\n\t" \
  "ds_read_b64_tr_b16 a[138:139], v13 offset:1024\n\t" \
  "v_mfma_f32_16x16x32_bf16 a[56:59], a[240:243], v[72:75], a[56:59]\n\t" \
  "v_mfma_f32_16x16x32_bf16 a[52:55], a[232:235], v[72:75], a[52:55]\n\t" \
  "ds_read_b64_tr_b16 a[140:141], v13 offset:8192\n\t" \
  "v_mfma_f32_16x16x32_bf16 a[52:55], a[236:239], v[76:79], a[52:55]\n\t" \
  "v_mfma_f32_16x16x32_bf16 a[48:51], a[228:231], v[76:79], a[48:51]\n\t" \
  "ds_read_b64_tr_b16 a[142:143], v13 offset:9216\n\t" \
  "v_mfma_f32_16x16x32_bf16 a[48:51], a[224:227], v[72:75], a[48:51]\n\t" \
  "v_mfma_f32_16x16x32_bf16 a[44:47], a[216:219], v[72:75], a[44:47]\n\t" \
  "ds_read_b64_tr_b16 a[144:145], v14 offset:0\n\t" \
  "v_mfma_f32_16x16x32_bf16 a[44:47], a[220:223], v[76:79], a[44:47]\n\t" \
  "v_mfma_f32_16x16x32_bf16 a[40:43], a[212:215], v[76:79], a[40:43]\n\t" \
  "ds_read_b64_tr_b16 a[146:147], v14 offset:1024\n\t" \
  "v_mfma_f32_16x16x32_bf16 a[40:43], a[208:211], v[72:75], a[40:43]\n\t" \
  "v_mfma_f32_16x16x32_bf16 a[36:39], a[200:203], v[72:75], a[36:39]\n\t" \
  "ds_read_b64_tr_b16 a[148:149], v14 offset:8192\n\t" \
  "v_mfma_f32_16x16x32_bf16 a[36:39], a[204:207], v[76:79], a[36:39]\n\t" \
  "v_mfma_f32_16x16x32_bf16 a[32:35], a[196:199], v[76:79], a[32:35]\n\t" \
  "ds_read_b64_tr_b16 a[150:151], v14 offset:9216\n\t" \
  "v_mfma_f32_16x16x32_bf16 a[32:35], a[192:195], v[72:75], a[32:35]\n\t" \
  "ds_read_b128 v[72:75], v10 offset:2048\n\t" \
  "ds_read_b128 v[76:79], v11 offset:2048\n\t" \
  "v_mfma_f32_16x16x32_bf16 a[64:67], a[192:195], v[80:83], a[64:67]\n\t" \
  "ds_read_b64_tr_b16 a[152:153], v15 offset:0\n\t" \
  "v_mfma_f32_16x16x32_bf16 a[64:67], a[196:199], v[84:87], a[64:67]\n\t" \
  "v_mfma_f32_16x16x32_bf16 a[68:71], a[204:207], v[84:87], a[68:71]\n\t" \
  "ds_read_b64_tr_b16 a[154:155], v15 offset:1024\n\t" \
  "v_mfma_f32_16x16x32_bf16 a[68:71], a[200:203], v[80:83], a[68:71]\n\t" \
  "v_mfma_f32_16x16x32_bf16 a[72:75], a[208:211], v[80:83], a[72:75]\n\t" \
  "ds_read_b64_tr_b16 a[156:157], v15 offset:8192\n\t" \
  "v_mfma_f32_16x16x32_bf16 a[72:75], a[212:215], v[84:87], a[72:75]\n\t" \
  "v_mfma_f32_16x16x32_bf16 a[76:79], a[220:223], v[84:87], a[76:79]\n\t" \
  "ds_read_b64_tr_b16 a[158:159], v15 offset:9216\n\t" \
  "v_mfma_f32_16x16x32_bf16 a[76:79], a[216:219], v[80:83], a[76:79]\n\t" \
  "s_add_u32 m0, s68, 49152\n\t" \
  "v_mfma_f32_16x16x32_bf16 a[80:83], a[224:227], v[80:83], a[80:83]\n\t" \
  "buffer_load_dwordx4 v0, s[36:39], s65 offen offset:0 sc0 sc1 lds\n\t" \
  "v_mfma_f32_16x16x32_bf16 a[80:83], a[228:231], v[84:87], a[80:83]\n\t" \
  "v_mfma_f32_16x16x32_bf16 a[84:87], a[236:239], v[84:87], a[84:87]\n\t" \
  "v_mfma_f32_16x16x32_bf16 a[84:87], a[232:235], v[80:83], a[84:87]\n\t" \
  "buffer_load_dwordx4 v1, s[36:39], s65 offen offset:1024 sc0 sc1 lds\n\t" \
  "v_mfma_f32_16x16x32_bf16 a[88:91], a[240:243], v[80:83], a[88:91]\n\t" \
  "v_mfma_f32_16x16x32_bf16 a[88:91], a[244:247], v[84:87], a[88:91]\n\t" \
  "v_mfma_f32_16x16x32_bf16 a[92:95], a[252:255], v[84:87], a[92:95]\n\t" \
  "buffer_load_dwordx4 v2, s[36:39], s65 offen offset:2048 sc0 sc1 lds\n\t" \
  "v_mfma_f32_16x16x32_bf16 a[92:95], a[248:251], v[80:83], a[92:95]\n\t" \
  "ds_read_b128 v[80:83], v10 offset:4096\n\t" \
  "ds_read_b128 v[84:87], v11 offset:4096\n\t" \
  "v_mfma_f32_16x16x32_bf16 a[124:127], a[248:251], v[88:91], a[124:127]\n\t" \
  "v_mfma_f32_16x16x32_bf16 a[124:127], a[252:255], v[92:95], a[124:127]\n\t" \
  "buffer_load_dwordx4 v3, s[36:39], s65 offen offset:3072 sc0 sc1 lds\n\t" \
  "v_mfma_f32_16x16x32_bf16 a[120:123], a[244:247], v[92:95], a[120:123]\n\t" \
  "v_mfma_f32_16x16x32_bf16 a[120:123], a[240:243], v[88:91], a[120:123]\n\t" \
  "s_add_u32 m0, s68, 114688\n\t" \
  "v_mfma_f32_16x16x32_bf16 a[116:119], a[232:235], v[88:91], a[116:119]\n\t" \
  "buffer_load_dwordx4 v4, s[40:43], s79 offen offset:0 lds\n\t" \
  "v_cmp_le_u32 s[92:93], s86, v37\n\t" \
  "v_mfma_f32_16x16x32_bf16 a[116:119], a[236:239], v[92:95], a[116:119]\n\t" \
  "v_mfma_f32_16x16x32_bf16 a[112:115], a[228:231], v[92:95], a[112:115]\n\t" \
  "v_mfma_f32_16x16x32_bf16 a[112:115], a[224:227], v[88:91], a[112:115]\n\t" \
  "buffer_load_dwordx4 v5, s[40:43], s79 offen offset:1024 lds\n\t" \
  "s_or_b64 s[58:59], s[58:59], s[92:93]\n\t" \
  "v_mfma_f32_16x16x32_bf16 a[108:111], a[216:219], v[88:91], a[108:111]\n\t" \
  "v_mfma_f32_16x16x32_bf16 a[108:111], a[220:223], v[92:95], a[108:111]\n\t" \
  "s_bitcmp1_b64 s[58:59], 1\n\t" \
  "s_cbranch_scc0 SLOW209_%=\n\t" \
  "RET208_%=:\n\t" \
  "v_mfma_f32_16x16x32_bf16 a[104:107], a[212:215], v[92:95], a[104:107]\n\t" \
  "buffer_load_dwordx4 v6, s[40:43], s79 offen offset:2048 lds\n\t" \
  "v_mfma_f32_16x16x32_bf16 a[104:107], a[208:211], v[88:91], a[104:107]\n\t" \
  "v_mfma_f32_16x16x32_bf16 a[100:103], a[200:203], v[88:91], a[100:103]\n\t" \
  "v_mfma_f32_16x16x32_bf16 a[100:103], a[204:207], v[92:95], a[100:103]\n\t" \
  "v_mfma_f32_16x16x32_bf16 a[96:99], a[196:199], v[92:95], a[96:99]\n\t" \
  "buffer_load_dwordx4 v7, s[40:43], s79 offen offset:3072 lds\n\t" \
  "v_mfma_f32_16x16x32_bf16 a[96:99], a[192:195], v[88:91], a[96:99]\n\t" \
  "ds_read_b128 v[88:91], v10 offset:6144\n\t" \
  "ds_read_b128 v[92:95], v11 offset:6144\n\t" \
  "v_mfma_f32_16x16x32_bf16 v[128:131], a[192:195], v[96:99], v[128:131]\n\t" \
  "v_mfma_f32_16x16x32_bf16 v[128:131], a[196:199], v[100:103], v[128:131]\n\t" \
  "v_mfma_f32_16x16x32_bf16 v[132:135], a[204:207], v[100:103], v[132:135]\n\t" \
  "v_mfma_f32_16x16x32_bf16 v[132:135], a[200:203], v[96:99], v[132:135]\n\t" \
  "v_mfma_f32_16x16x32_bf16 v[136:139], a[208:211], v[96:99], v[136:139]\n\t" \
  "v_mfma_f32_16x16x32_bf16 v[136:139], a[212:215], v[100:103], v[136:139]\n\t" \
  "v_mfma_f32_16x16x32_bf16 v[140:143], a[220:223], v[100:103], v[140:143]\n\t" \
  "v_mfma_f32_16x16x32_bf16 v[140:143], a[216:219], v[96:99], v[140:143]\n\t" \
  "s_waitcnt vmcnt(12) lgkmcnt(0)\n\t" \
  "s_barrier\n\t" \
  "v_mfma_f32_16x16x32_bf16 v[144:147], a[224:227], v[96:99], v[144:147]\n\t" \
  "ds_read_b64_tr_b16 a[160:161], v12 offset:16384\n\t" \
  "v_readlane_b32 s66, v17, 1\n\t" \
  "s_nop 0\n\t" \
  "s_lshl_b32 s78, s66, 13\n\t" \
  "s_lshl_b32 s65, s66, 10\n\t" \
  "s_add_u32 s79, s78, 256\n\t" \
  "s_add_u32 s65, s65, s70\n\t" \
  "s_nop 1\n\t" \
  "v_mfma_f32_16x16x32_bf16 v[144:147], a[228:231], v[100:103], v[144:147]\n\t" \
  "v_mfma_f32_16x16x32_bf16 v[148:151], a[236:239], v[100:103], v[148:151]\n\t" \
  "ds_read_b64_tr_b16 a[162:163], v12 offset:17408\n\t" \
  "v_mfma_f32_16x16x32_bf16 v[148:151], a[232:235], v[96:99], v[148:151]\n\t" \
  "s_add_u32 m0, s68, 32768\n\t" \
  "v_mfma_f32_16x16x32_bf16 v[152:155], a[240:243], v[96:99], v[152:155]\n\t" \
  "ds_read_b64_tr_b16 a[164:165], v12 offset:24576\n\t" \
  "buffer_load_dwordx4 v4, s[40:43], s78 offen offset:0 lds\n\t" \
  "v_mfma_f32_16x16x32_bf16 v[152:155], a[244:247], v[100:103], v[152:155]\n\t" \
  "v_mfma_f32_16x16x32_bf16 v[156:159], a[252:255], v[100:103], v[156:159]\n\t" \
  "ds_read_b64_tr_b16 a[166:167], v12 offset:25600\n\t" \
  "v_mfma_f32_16x16x32_bf16 v[156:159], a[248:251], v[96:99], v[156:159]\n\t" \
  "ds_read_b128 v[96:99], v10 offset:16384\n\t" \
  "ds_read_b128 v[100:103], v11 offset:16384\n\t" \
  "v_mfma_f32_16x16x32_bf16 v[188:191], a[248:251], v[104:107], v[188:191]\n\t" \
  "ds_read_b64_tr_b16 a[168:169], v13 offset:16384\n\t" \
  "buffer_load_dwordx4 v5, s[40:43], s78 offen offset:1024 lds\n\t" \
  "v_mfma_f32_16x16x32_bf16 v[188:191], a[252:255], v[108:111], v[188:191]\n\t" \
  "v_mfma_f32_16x16x32_bf16 v[184:187], a[244:247], v[108:111], v[184:187]\n\t" \
  "ds_read_b64_tr_b16 a[170:171], v13 offset:17408\n\t" \
  "v_mfma_f32_16x16x32_bf16 v[184:187], a[240:243], v[104:107], v[184:187]\n\t" \
  "v_mfma_f32_16x16x32_bf16 v[180:183], a[232:235], v[104:107], v[180:183]\n\t" \
  "ds_read_b64_tr_b16 a[172:173], v13 offset:24576\n\t" \
  "v_mfma_f32_16x16x32_bf16 v[180:183], a[236:239], v[108:111], v[180:183]\n\t" \
  "buffer_load_dwordx4 v6, s[40:43], s78 offen offset:2048 lds\n\t" \
  "v_mfma_f32_16x16x32_bf16 v[176:179], a[228:231], v[108:111], v[176:179]\n\t" \
  "ds_read_b64_tr_b16 a[174:175], v13 offset:25600\n\t" \
  "v_mfma_f32_16x16x32_bf16 v[176:179], a[224:227], v[104:107], v[176:179]\n\t" \
  "v_mfma_f32_16x16x32_bf16 v[172:175], a[216:219], v[104:107], v[172:175]\n\t" \
  "ds_read_b64_tr_b16 a[176:177], v14 offset:16384\n\t" \
  "v_mfma_f32_16x16x32_bf16 v[172:175], a[220:223], v[108:111], v[172:175]\n\t" \
  "buffer_load_dwordx4 v7, s[40:43], s78 offen offset:3072 lds\n\t" \
  "v_mfma_f32_16x16x32_bf16 v[168:171], a[212:215], v[108:111], v[168:171]\n\t" \
  "ds_read_b64_tr_b16 a[178:179], v14 offset:17408\n\t" \
  "v_mfma_f32_16x16x32_bf16 v[168:171], a[208:211], v[104:107], v[168:171]\n\t" \
  "v_mfma_f32_16x16x32_bf16 v[164:167], a[200:203], v[104:107], v[164:167]\n\t" \
  "ds_read_b64_tr_b16 a[180:181], v14 offset:24576\n\t" \
  "v_mfma_f32_16x16x32_bf16 v[164:167], a[204:207], v[108:111], v[164:167]\n\t" \
  "s_add_u32 m0, s68, 0\n\t" \
  "v_mfma_f32_16x16x32_bf16 v[160:163], a[196:199], v[108:111], v[160:163]\n\t" \
  "ds_read_b64_tr_b16 a[182:183], v14 offset:25600\n\t" \
  "buffer_load_dwordx4 v0, s[36:39], s65 offen offset:0 sc0 sc1 lds\n\t" \
  "v_mfma_f32_16x16x32_bf16 v[160:163], a[192:195], v[104:107], v[160:163]\n\t" \
  "ds_read_b128 v[104:107], v10 offset:18432\n\t" \
  "ds_read_b128 v[108:111], v11 offset:18432\n\t" \
  "v_mfma_f32_16x16x32_bf16 v[192:195], a[192:195], v[112:115], v[192:195]\n\t" \
  "ds_read_b64_tr_b16 a[184:185], v15 offset:16384\n\t" \
  "v_mfma_f32_16x16x32_bf16 v[192:195], a[196:199], v[116:119], v[192:195]\n\t" \
  "v_mfma_f32_16x16x32_bf16 v[196:199], a[204:207], v[116:119], v[196:199]\n\t" \
  "ds_read_b64_tr_b16 a[186:187], v15 offset:17408\n\t" \
  "buffer_load_dwordx4 v1, s[36:39], s65 offen offset:1024 sc0 sc1 lds\n\t" \
  "v_mfma_f32_16x16x32_bf16 v[196:199], a[200:203], v[112:115], v[196:199]\n\t" \
  "v_mfma_f32_16x16x32_bf16 v[200:203], a[208:211], v[112:115], v[200:203]\n\t" \
  "ds_read_b64_tr_b16 a[188:189], v15 offset:24576\n\t" \
  "v_mfma_f32_16x16x32_bf16 v[200:203], a[212:215], v[116:119], v[200:203]\n\t" \
  "v_mfma_f32_16x16x32_bf16 v[204:207], a[220:223], v[116:119], v[204:207]\n\t" \
  "ds_read_b64_tr_b16 a[190:191], v15 offset:25600\n\t" \
  "v_mfma_f32_16x16x32_bf16 v[204:207], a[216:219], v[112:115], v[204:207]\n\t" \
  "buffer_load_dwordx4 v2, s[36:39], s65 offen offset:2048 sc0 sc1 lds\n\t" \
  "v_mfma_f32_16x16x32_bf16 v[208:211], a[224:227], v[112:115], v[208:211]\n\t" \
  "v_mfma_f32_16x16x32_bf16 v[208:211], a[228:231], v[116:119], v[208:211]\n\t" \
  "v_mfma_f32_16x16x32_bf16 v[212:215], a[236:239], v[116:119], v[212:215]\n\t" \
  "v_mfma_f32_16x16x32_bf16 v[212:215], a[232:235], v[112:115], v[212:215]\n\t" \
  "buffer_load_dwordx4 v3, s[36:39], s65 offen offset:3072 sc0 sc1 lds\n\t" \
  "v_mfma_f32_16x16x32_bf16 v[216:219], a[240:243], v[112:115], v[216:219]\n\t" \
  "v_mfma_f32_16x16x32_bf16 v[216:219], a[244:247], v[116:119], v[216:219]\n\t" \
  "v_mfma_f32_16x16x32_bf16 v[220:223], a[252:255], v[116:119], v[220:223]\n\t" \
  "v_mfma_f32_16x16x32_bf16 v[220:223], a[248:251], v[112:115], v[220:223]\n\t" \
  "ds_read_b128 v[112:115], v10 offset:20480\n\t" \
  "ds_read_b128 v[116:119], v11 offset:20480\n\t" \
  "s_add_u32 m0, s68, 65536\n\t" \
  "v_mfma_f32_16x16x32_bf16 v[252:255], a[248:251], v[120:123], v[252:255]\n\t" \
  "buffer_load_dwordx4 v4, s[40:43], s79 offen offset:0 lds\n\t" \
  "v_mfma_f32_16x16x32_bf16 v[252:255], a[252:255], v[124:127], v[252:255]\n\t" \
  "v_mfma_f32_16x16x32_bf16 v[248:251], a[244:247], v[124:127], v[248:251]\n\t" \
  "v_mfma_f32_16x16x32_bf16 v[248:251], a[240:243], v[120:123], v[248:251]\n\t" \
  "v_mfma_f32_16x16x32_bf16 v[244:247], a[232:235], v[120:123], v[244:247]\n\t" \
  "buffer_load_dwordx4 v5, s[40:43], s79 offen offset:1024 lds\n\t" \
  "v_cmp_le_u32 s[92:93], s86, v37\n\t" \
  "v_mfma_f32_16x16x32_bf16 v[244:247], a[236:239], v[124:127], v[244:247]\n\t" \
  "v_mfma_f32_16x16x32_bf16 v[240:243], a[228:231], v[124:127], v[240:243]\n\t" \
  "v_mfma_f32_16x16x32_bf16 v[240:243], a[224:227], v[120:123], v[240:243]\n\t" \
  "s_or_b64 s[58:59], s[58:59], s[92:93]\n\t" \
  "v_mfma_f32_16x16x32_bf16 v[236:239], a[216:219], v[120:123], v[236:239]\n\t" \
  "v_mfma_f32_16x16x32_bf16 v[236:239], a[220:223], v[124:127], v[236:239]\n\t" \
  "buffer_load_dwordx4 v6, s[40:43], s79 offen offset:2048 lds\n\t" \
  "s_bitcmp1_b64 s[58:59], 2\n\t" \
  "s_cbranch_scc0 SLOW213_%=\n\t" \
  "RET212_%=:\n\t" \
  "v_mfma_f32_16x16x32_bf16 v[232:235], a[212:215], v[124:127], v[232:235]\n\t" \
  "v_mfma_f32_16x16x32_bf16 v[232:235], a[208:211], v[120:123], v[232:235]\n\t" \
  "v_mfma_f32_16x16x32_bf16 v[228:231], a[200:203], v[120:123], v[228:231]\n\t" \
  "v_mfma_f32_16x16x32_bf16 v[228:231], a[204:207], v[124:127], v[228:231]\n\t" \
  "v_mfma_f32_16x16x32_bf16 v[224:227], a[196:199], v[124:127], v[224:227]\n\t" \
  "buffer_load_dwordx4 v7, s[40:43], s79 offen offset:3072 lds\n\t" \
  "v_mfma_f32_16x16x32_bf16 v[224:227], a[192:195], v[120:123], v[224:227]\n\t" \
  "ds_read_b128 v[120:123], v10 offset:22528\n\t" \
  "ds_read_b128 v[124:127], v11 offset:22528\n\t" \
  "v_mfma_f32_16x16x32_bf16 a[0:3], a[128:131], v[64:67], a[0:3]\n\t" \
  "v_mfma_f32_16x16x32_bf16 a[0:3], a[132:135], v[68:71], a[0:3]\n\t" \
  "v_mfma_f32_16x16x32_bf16 a[4:7], a[140:143], v[68:71], a[4:7]\n\t" \
  "v_mfma_f32_16x16x32_bf16 a[4:7], a[136:139], v[64:67], a[4:7]\n\t" \
  "v_mfma_f32_16x16x32_bf16 a[8:11], a[144:147], v[64:67], a[8:11]\n\t" \
  "v_mfma_f32_16x16x32_bf16 a[8:11], a[148:151], v[68:71], a[8:11]\n\t" \
  "v_mfma_f32_16x16x32_bf16 a[12:15], a[156:159], v[68:71], a[12:15]\n\t" \
  "v_mfma_f32_16x16x32_bf16 a[12:15], a[152:155], v[64:67], a[12:15]\n\t" \
  "s_waitcnt vmcnt(20) lgkmcnt(0)\n\t" \
  "s_barrier\n\t" \
  "v_mfma_f32_16x16x32_bf16 a[16:19], a[160:163], v[64:67], a[16:19]\n\t" \
  "ds_read_b64_tr_b16 a[192:193], v12 offset:32768\n\t" \
  "v_readlane_b32 s66, v17, 2\n\t" \
  "s_nop 0\n\t" \
  "s_lshl_b32 s78, s66, 13\n\t" \
  "s_lshl_b32 s65, s66, 10\n\t" \
  "s_add_u32 s79, s78, 256\n\t" \
  "s_add_u32 s65, s65, s70\n\t" \
  "s_nop 1\n\t" \
  "v_mfma_f32_16x16x32_bf16 a[16:19], a[164:167], v[68:71], a[16:19]\n\t" \
  "s_andn2_b64 s[92:93], 255, s[58:59]\n\t" \
  "s_cbranch_scc1 POLL217_%=\n\t" \
  "RET216_%=:\n\t" \
  "v_mfma_f32_16x16x32_bf16 a[20:23], a[172:175], v[68:71], a[20:23]\n\t" \
  "ds_read_b64_tr_b16 a[194:195], v12 offset:33792\n\t" \
  "v_mfma_f32_16x16x32_bf16 a[20:23], a[168:171], v[64:67], a[20:23]\n\t" \
  "s_add_u32 m0, s68, 131072\n\t" \
  "v_mfma_f32_16x16x32_bf16 a[24:27], a[176:179], v[64:67], a[24:27]\n\t" \
  "ds_read_b64_tr_b16 a[196:197], v12 offset:40960\n\t" \
  "buffer_load_dwordx4 v4, s[40:43], s78 offen offset:0 lds\n\t" \
  "v_mfma_f32_16x16x32_bf16 a[24:27], a[180:183], v[68:71], a[24:27]\n\t" \
  "v_mfma_f32_16x16x32_bf16 a[28:31], a[188:191], v[68:71], a[28:31]\n\t" \
  "ds_read_b64_tr_b16 a[198:199], v12 offset:41984\n\t" \
  "v_mfma_f32_16x16x32_bf16 a[28:31], a[184:187], v[64:67], a[28:31]\n\t" \
  "v_mfma_f32_16x16x32_bf16 a[60:63], a[184:187], v[72:75], a[60:63]\n\t" \
  "ds_read_b64_tr_b16 a[200:201], v13 offset:32768\n\t" \
  "buffer_load_dwordx4 v5, s[40:43], s78 offen offset:1024 lds\n\t" \
  "v_mfma_f32_16x16x32_bf16 a[60:63], a[188:191], v[76:79], a[60:63]\n\t" \
  "v_mfma_f32_16x16x32_bf16 a[56:59], a[180:183], v[76:79], a[56:59]\n\t" \
  "ds_read_b64_tr_b16 a[202:203], v13 offset:33792\n\t" \
  "v_mfma_f32_16x16x32_bf16 a[56:59], a[176:179], v[72:75], a[56:59]\n\t" \
  "v_mfma_f32_16x16x32_bf16 a[52:55], a[168:171], v[72:75], a[52:55]\n\t" \
  "ds_read_b64_tr_b16 a[204:205], v13 offset:40960\n\t" \
  "v_mfma_f32_16x16x32_bf16 a[52:55], a[172:175], v[76:79], a[52:55]\n\t" \
  "buffer_load_dwordx4 v6, s[40:43], s78 offen offset:2048 lds\n\t" \
  "v_mfma_f32_16x16x32_bf16 a[48:51], a[164:167], v[76:79], a[48:51]\n\t" \
  "ds_read_b64_tr_b16 a[206:207], v13 offset:41984\n\t" \
  "v_mfma_f32_16x16x32_bf16 a[48:51], a[160:163], v[72:75], a[48:51]\n\t" \
  "v_mfma_f32_16x16x32_bf16 a[44:47], a[152:155], v[72:75], a[44:47]\n\t" \
  "ds_read_b64_tr_b16 a[208:209], v14 offset:32768\n\t" \
  "v_mfma_f32_16x16x32_bf16 a[44:47], a[156:159], v[76:79], a[44:47]\n\t" \
  "buffer_load_dwordx4 v7, s[40:43], s78 offen offset:3072 lds\n\t" \
  "v_mfma_f32_16x16x32_bf16 a[40:43], a[148:151], v[76:79], a[40:43]\n\t" \
  "ds_read_b64_tr_b16 a[210:211], v14 offset:33792\n\t" \
  "v_mfma_f32_16x16x32_bf16 a[40:43], a[144:147], v[72:75], a[40:43]\n\t" \
  "v_mfma_f32_16x16x32_bf16 a[36:39], a[136:139], v[72:75], a[36:39]\n\t" \
  "ds_read_b64_tr_b16 a[212:213], v14 offset:40960\n\t" \
  "v_mfma_f32_16x16x32_bf16 a[36:39], a[140:143], v[76:79], a[36:39]\n\t" \
  "s_add_u32 m0, s68, 147456\n\t" \
  "v_mfma_f32_16x16x32_bf16 a[32:35], a[132:135], v[76:79], a[32:35]\n\t" \
  "ds_read_b64_tr_b16 a[214:215], v14 offset:41984\n\t" \
  "buffer_load_dwordx4 v4, s[40:43], s79 offen offset:0 lds\n\t" \
  "v_mfma_f32_16x16x32_bf16 a[32:35], a[128:131], v[72:75], a[32:35]\n\t" \
  "v_mfma_f32_16x16x32_bf16 a[64:67], a[128:131], v[80:83], a[64:67]\n\t" \
  "ds_read_b64_tr_b16 a[216:217], v15 offset:32768\n\t" \
  "v_mfma_f32_16x16x32_bf16 a[64:67], a[132:135], v[84:87], a[64:67]\n\t" \
  "v_mfma_f32_16x16x32_bf16 a[68:71], a[140:143], v[84:87], a[68:71]\n\t" \
  "ds_read_b64_tr_b16 a[218:219], v15 offset:33792\n\t" \
  "buffer_load_dwordx4 v5, s[40:43], s79 offen offset:1024 lds\n\t" \
  "v_mfma_f32_16x16x32_bf16 a[68:71], a[136:139], v[80:83], a[68:71]\n\t" \
  "v_mfma_f32_16x16x32_bf16 a[72:75], a[144:147], v[80:83], a[72:75]\n\t" \
  "ds_read_b64_tr_b16 a[220:221], v15 offset:40960\n\t" \
  "v_mfma_f32_16x16x32_bf16 a[72:75], a[148:151], v[84:87], a[72:75]\n\t" \
  "v_mfma_f32_16x16x32_bf16 a[76:79], a[156:159], v[84:87], a[76:79]\n\t" \
  "ds_read_b64_tr_b16 a[222:223], v15 offset:41984\n\t" \
  "v_mfma_f32_16x16x32_bf16 a[76:79], a[152:155], v[80:83], a[76:79]\n\t" \
  "buffer_load_dwordx4 v6, s[40:43], s79 offen offset:2048 lds\n\t" \
  "v_mfma_f32_16x16x32_bf16 a[80:83], a[160:163], v[80:83], a[80:83]\n\t" \
  "v_mfma_f32_16x16x32_bf16 a[80:83], a[164:167], v[84:87], a[80:83]\n\t" \
  "v_mfma_f32_16x16x32_bf16 a[84:87], a[172:175], v[84:87], a[84:87]\n\t" \
  "v_mfma_f32_16x16x32_bf16 a[84:87], a[168:171], v[80:83], a[84:87]\n\t" \
  "buffer_load_dwordx4 v7, s[40:43], s79 offen offset:3072 lds\n\t" \
  "v_mfma_f32_16x16x32_bf16 a[88:91], a[176:179], v[80:83], a[88:91]\n\t" \
  "v_mfma_f32_16x16x32_bf16 a[88:91], a[180:183], v[84:87], a[88:91]\n\t" \
  "v_mfma_f32_16x16x32_bf16 a[92:95], a[188:191], v[84:87], a[92:95]\n\t" \
  "v_mfma_f32_16x16x32_bf16 a[92:95], a[184:187], v[80:83], a[92:95]\n\t" \
  "s_add_u32 m0, s68, 16384\n\t" \
  "v_mfma_f32_16x16x32_bf16 a[124:127], a[184:187], v[88:91], a[124:127]\n\t" \
  "buffer_load_dwordx4 v0, s[36:39], s65 offen offset:0 sc0 sc1 lds\n\t" \
  "v_mfma_f32_16x16x32_bf16 a[124:127], a[188:191], v[92:95], a[124:127]\n\t" \
  "v_mfma_f32_16x16x32_bf16 a[120:123], a[180:183], v[92:95], a[120:123]\n\t" \
  "v_mfma_f32_16x16x32_bf16 a[120:123], a[176:179], v[88:91], a[120:123]\n\t" \
  "v_mfma_f32_16x16x32_bf16 a[116:119], a[168:171], v[88:91], a[116:119]\n\t" \
  "buffer_load_dwordx4 v1, s[36:39], s65 offen offset:1024 sc0 sc1 lds\n\t" \
  "v_mfma_f32_16x16x32_bf16 a[116:119], a[172:175], v[92:95], a[116:119]\n\t" \
  "v_mfma_f32_16x16x32_bf16 a[112:115], a[164:167], v[92:95], a[112:115]\n\t" \
  "v_mfma_f32_16x16x32_bf16 a[112:115], a[160:163], v[88:91], a[112:115]\n\t" \
  "v_mfma_f32_16x16x32_bf16 a[108:111], a[152:155], v[88:91], a[108:111]\n\t" \
  "v_mfma_f32_16x16x32_bf16 a[108:111], a[156:159], v[92:95], a[108:111]\n\t" \
  "buffer_load_dwordx4 v2, s[36:39], s65 offen offset:2048 sc0 sc1 lds\n\t" \
  "v_mfma_f32_16x16x32_bf16 a[104:107], a[148:151], v[92:95], a[104:107]\n\t" \
  "v_mfma_f32_16x16x32_bf16 a[104:107], a[144:147], v[88:91], a[104:107]\n\t" \
  "v_mfma_f32_16x16x32_bf16 a[100:103], a[136:139], v[88:91], a[100:103]\n\t" \
  "v_mfma_f32_16x16x32_bf16 a[100:103], a[140:143], v[92:95], a[100:103]\n\t" \
  "v_mfma_f32_16x16x32_bf16 a[96:99], a[132:135], v[92:95], a[96:99]\n\t" \
  "buffer_load_dwordx4 v3, s[36:39], s65 offen offset:3072 sc0 sc1 lds\n\t" \
  "v_mfma_f32_16x16x32_bf16 a[96:99], a[128:131], v[88:91], a[96:99]\n\t" \
  "v_mfma_f32_16x16x32_bf16 v[128:131], a[128:131], v[96:99], v[128:131]\n\t" \
  "v_mfma_f32_16x16x32_bf16 v[128:131], a[132:135], v[100:103], v[128:131]\n\t" \
  "v_accvgpr_read_b32 v68, a0\n\t" \
  "v_accvgpr_read_b32 v69, a1\n\t" \
  "v_accvgpr_read_b32 v70, a2\n\t" \
  "v_mfma_f32_16x16x32_bf16 v[132:135], a[140:143], v[100:103], v[132:135]\n\t" \
  "v_accvgpr_read_b32 v71, a3\n\t" \
  "v_accvgpr_read_b32 v72, a4\n\t" \
  "v_accvgpr_read_b32 v73, a5\n\t" \
  "v_mfma_f32_16x16x32_bf16 v[132:135], a[136:139], v[96:99], v[132:135]\n\t" \
  "v_accvgpr_read_b32 v74, a6\n\t" \
  "v_accvgpr_read_b32 v75, a7\n\t" \
  "v_cvt_pk_bf16_f32 v64, v68, v69\n\t" \
  "v_mfma_f32_16x16x32_bf16 v[136:139], a[144:147], v[96:99], v[136:139]\n\t" \
  "v_cvt_pk_bf16_f32 v65, v70, v71\n\t" \
  "v_cvt_pk_bf16_f32 v66, v72, v73\n\t" \
  "v_cvt_pk_bf16_f32 v67, v74, v75\n\t" \
  "v_mfma_f32_16x16x32_bf16 v[136:139], a[148:151], v[100:103], v[136:139]\n\t" \
  "global_store_dwordx4 v9, v[64:67], s[50:51] offset:-4096 sc0 sc1\n\t" \
  "v_accvgpr_read_b32 v84, a8\n\t" \
  "v_accvgpr_read_b32 v85, a9\n\t" \
  "v_mfma_f32_16x16x32_bf16 v[140:143], a[156:159], v[100:103], v[140:143]\n\t" \
  "v_accvgpr_read_b32 v86, a10\n\t" \
  "v_accvgpr_read_b32 v87, a11\n\t" \
  "v_accvgpr_read_b32 v88, a12\n\t" \
  "v_mfma_f32_16x16x32_bf16 v[140:143], a[152:155], v[96:99], v[140:143]\n\t" \
  "v_accvgpr_read_b32 v89, a13\n\t" \
  "v_accvgpr_read_b32 v90, a14\n\t" \
  "v_accvgpr_read_b32 v91, a15\n\t" \
  "s_waitcnt vmcnt(25) lgkmcnt(0)\n\t" \
  "s_barrier\n\t" \
  "v_mfma_f32_16x16x32_bf16 v[144:147], a[160:163], v[96:99], v[144:147]\n\t" \
  "ds_read_b64_tr_b16 a[224:225], v12 offset:49152\n\t" \
  "v_readlane_b32 s66, v17, 3\n\t" \
  "s_nop 0\n\t" \
  "s_lshl_b32 s78, s66, 13\n\t" \
  "s_lshl_b32 s65, s66, 10\n\t" \
  "s_add_u32 s79, s78, 256\n\t" \
  "s_add_u32 s65, s65, s70\n\t" \
  "s_nop 1\n\t" \
  "v_mfma_f32_16x16x32_bf16 v[144:147], a[164:167], v[100:103], v[144:147]\n\t" \
  "v_cvt_pk_bf16_f32 v80, v84, v85\n\t" \
  "v_cvt_pk_bf16_f32 v81, v86, v87\n\t" \
  "v_cvt_pk_bf16_f32 v82, v88, v89\n\t" \
  "v_mfma_f32_16x16x32_bf16 v[148:151], a[172:175], v[100:103], v[148:151]\n\t" \
  "ds_read_b64_tr_b16 a[226:227], v12 offset:50176\n\t" \
  "v_cvt_pk_bf16_f32 v83, v90, v91\n\t" \
  "global_store_dwordx4 v9, v[80:83], s[50:51] offset:-3072 sc0 sc1\n\t" \
  "v_mfma_f32_16x16x32_bf16 v[148:151], a[168:171], v[96:99], v[148:151]\n\t" \
  "v_accvgpr_read_b32 v68, a16\n\t" \
  "v_accvgpr_read_b32 v69, a17\n\t" \
  "v_accvgpr_read_b32 v70, a18\n\t" \
  "v_mfma_f32_16x16x32_bf16 v[152:155], a[176:179], v[96:99], v[152:155]\n\t" \
  "ds_read_b64_tr_b16 a[228:229], v12 offset:57344\n\t" \
  "v_accvgpr_read_b32 v71, a19\n\t" \
  "v_accvgpr_read_b32 v72, a20\n\t" \
  "v_mfma_f32_16x16x32_bf16 v[152:155], a[180:183], v[100:103], v[152:155]\n\t" \
  "v_accvgpr_read_b32 v73, a21\n\t" \
  "v_accvgpr_read_b32 v74, a22\n\t" \
  "v_accvgpr_read_b32 v75, a23\n\t" \
  "v_mfma_f32_16x16x32_bf16 v[156:159], a[188:191], v[100:103], v[156:159]\n\t" \
  "ds_read_b64_tr_b16 a[230:231], v12 offset:58368\n\t" \
  "v_cvt_pk_bf16_f32 v64, v68, v69\n\t" \
  "v_cvt_pk_bf16_f32 v65, v70, v71\n\t" \
  "v_mfma_f32_16x16x32_bf16 v[156:159], a[184:187], v[96:99], v[156:159]\n\t" \
  "ds_read_b128 v[96:99], v10 offset:49152\n\t" \
  "ds_read_b128 v[100:103], v11 offset:49152\n\t" \
  "v_cvt_pk_bf16_f32 v66, v72, v73\n\t" \
  "v_mfma_f32_16x16x32_bf16 v[188:191], a[184:187], v[104:107], v[188:191]\n\t" \
  "ds_read_b64_tr_b16 a[232:233], v13 offset:49152\n\t" \
  "v_cvt_pk_bf16_f32 v67, v74, v75\n\t" \
  "global_store_dwordx4 v9, v[64:67], s[50:51] offset:-2048 sc0 sc1\n\t" \
  "v_mfma_f32_16x16x32_bf16 v[188:191], a[188:191], v[108:111], v[188:191]\n\t" \
  "v_accvgpr_read_b32 v84, a24\n\t" \
  "v_accvgpr_read_b32 v85, a25\n\t" \
  "v_accvgpr_read_b32 v86, a26\n\t" \
  "v_mfma_f32_16x16x32_bf16 v[184:187], a[180:183], v[108:111], v[184:187]\n\t" \
  "ds_read_b64_tr_b16 a[234:235], v13 offset:50176\n\t" \
  "v_accvgpr_read_b32 v87, a27\n\t" \
  "v_accvgpr_read_b32 v88, a28\n\t" \
  "v_mfma_f32_16x16x32_bf16 v[184:187], a[176:179], v[104:107], v[184:187]\n\t" \
  "v_accvgpr_read_b32 v89, a29\n\t" \
  "v_accvgpr_read_b32 v90, a30\n\t" \
  "v_accvgpr_read_b32 v91, a31\n\t" \
  "v_mfma_f32_16x16x32_bf16 v[180:183], a[168:171], v[104:107], v[180:183]\n\t" \
  "ds_read_b64_tr_b16 a[236:237], v13 offset:57344\n\t" \
  "v_cvt_pk_bf16_f32 v80, v84, v85\n\t" \
  "v_cvt_pk_bf16_f32 v81, v86, v87\n\t" \
  "v_mfma_f32_16x16x32_bf16 v[180:183], a[172:175], v[108:111], v[180:183]\n\t" \
  "v_cvt_pk_bf16_f32 v82, v88, v89\n\t" \
  "v_cvt_pk_bf16_f32 v83, v90, v91\n\t" \
  "global_store_dwordx4 v9, v[80:83], s[50:51] offset:-1024 sc0 sc1\n\t" \
  "v_mfma_f32_16x16x32_bf16 v[176:179], a[164:167], v[108:111], v[176:179]\n\t" \
  "ds_read_b64_tr_b16 a[238:239], v13 offset:58368\n\t" \
  "v_accvgpr_read_b32 v68, a32\n\t" \
  "v_accvgpr_read_b32 v69, a33\n\t" \
  "v_mfma_f32_16x16x32_bf16 v[176:179], a[160:163], v[104:107], v[176:179]\n\t" \
  "v_accvgpr_read_b32 v70, a34\n\t" \
  "v_accvgpr_read_b32 v71, a35\n\t" \
  "v_accvgpr_read_b32 v72, a36\n\t" \
  "v_mfma_f32_16x16x32_bf16 v[172:175], a[152:155], v[104:107], v[172:175]\n\t" \
  "ds_read_b64_tr_b16 a[240:241], v14 offset:49152\n\t" \
  "v_accvgpr_read_b32 v73, a37\n\t" \
  "v_accvgpr_read_b32 v74, a38\n\t" \
  "v_mfma_f32_16x16x32_bf16 v[172:175], a[156:159], v[108:111], v[172:175]\n\t" \
  "v_accvgpr_read_b32 v75, a39\n\t" \
  "v_cvt_pk_bf16_f32 v64, v68, v69\n\t" \
  "v_cvt_pk_bf16_f32 v65, v70, v71\n\t" \
  "v_mfma_f32_16x16x32_bf16 v[168:171], a[148:151], v[108:111], v[168:171]\n\t" \
  "ds_read_b64_tr_b16 a[242:243], v14 offset:50176\n\t" \
  "v_cvt_pk_bf16_f32 v66, v72, v73\n\t" \
  "v_cvt_pk_bf16_f32 v67, v74, v75\n\t" \
  "v_mfma_f32_16x16x32_bf16 v[168:171], a[144:147], v[104:107], v[168:171]\n\t" \
  "global_store_dwordx4 v9, v[64:67], s[50:51] offset:0 sc0 sc1\n\t" \
  "v_accvgpr_read_b32 v84, a40\n\t" \
  "v_accvgpr_read_b32 v85, a41\n\t" \
  "v_mfma_f32_16x16x32_bf16 v[164:167], a[136:139], v[104:107], v[164:167]\n\t" \
  "ds_read_b64_tr_b16 a[244:245], v14 offset:57344\n\t" \
  "v_accvgpr_read_b32 v86, a42\n\t" \
  "v_accvgpr_read_b32 v87, a43\n\t" \
  "v_mfma_f32_16x16x32_bf16 v[164:167], a[140:143], v[108:111], v[164:167]\n\t" \
  "v_accvgpr_read_b32 v88, a44\n\t" \
  "v_accvgpr_read_b32 v89, a45\n\t" \
  "v_accvgpr_read_b32 v90, a46\n\t" \
  "v_mfma_f32_16x16x32_bf16 v[160:163], a[132:135], v[108:111], v[160:163]\n\t" \
  "ds_read_b64_tr_b16 a[246:247], v14 offset:58368\n\t" \
  "v_accvgpr_read_b32 v91, a47\n\t" \
  "v_cvt_pk_bf16_f32 v80, v84, v85\n\t" \
  "v_mfma_f32_16x16x32_bf16 v[160:163], a[128:131], v[104:107], v[160:163]\n\t" \
  "ds_read_b128 v[104:107], v10 offset:51200\n\t" \
  "ds_read_b128 v[108:111], v11 offset:51200\n\t" \
  "v_cvt_pk_bf16_f32 v81, v86, v87\n\t" \
  "v_mfma_f32_16x16x32_bf16 v[192:195], a[128:131], v[112:115], v[192:195]\n\t" \
  "ds_read_b64_tr_b16 a[248:249], v15 offset:49152\n\t" \
  "v_cvt_pk_bf16_f32 v82, v88, v89\n\t" \
  "v_cvt_pk_bf16_f32 v83, v90, v91\n\t" \
  "v_mfma_f32_16x16x32_bf16 v[192:195], a[132:135], v[116:119], v[192:195]\n\t" \
  "global_store_dwordx4 v9, v[80:83], s[50:51] offset:1024 sc0 sc1\n\t" \
  "v_accvgpr_read_b32 v68, a48\n\t" \
  "v_accvgpr_read_b32 v69, a49\n\t" \
  "v_mfma_f32_16x16x32_bf16 v[196:199], a[140:143], v[116:119], v[196:199]\n\t" \
  "ds_read_b64_tr_b16 a[250:251], v15 offset:50176\n\t" \
  "v_accvgpr_read_b32 v70, a50\n\t" \
  "v_accvgpr_read_b32 v71, a51\n\t" \
  "v_mfma_f32_16x16x32_bf16 v[196:199], a[136:139], v[112:115], v[196:199]\n\t" \
  "v_accvgpr_read_b32 v72, a52\n\t" \
  "v_accvgpr_read_b32 v73, a53\n\t" \
  "v_accvgpr_read_b32 v74, a54\n\t" \
  "v_mfma_f32_16x16x32_bf16 v[200:203], a[144:147], v[112:115], v[200:203]\n\t" \
  "ds_read_b64_tr_b16 a[252:253], v15 offset:57344\n\t" \
  "v_accvgpr_read_b32 v75, a55\n\t" \
  "v_cvt_pk_bf16_f32 v64, v68, v69\n\t" \
  "v_mfma_f32_16x16x32_bf16 v[200:203], a[148:151], v[116:119], v[200:203]\n\t" \
  "v_cvt_pk_bf16_f32 v65, v70, v71\n\t" \
  "v_cvt_pk_bf16_f32 v66, v72, v73\n\t" \
  "v_cvt_pk_bf16_f32 v67, v74, v75\n\t" \
  "v_mfma_f32_16x16x32_bf16 v[204:207], a[156:159], v[116:119], v[204:207]\n\t" \
  "ds_read_b64_tr_b16 a[254:255], v15 offset:58368\n\t" \
  "global_store_dwordx4 v9, v[64:67], s[50:51] offset:2048 sc0 sc1\n\t" \
  "v_accvgpr_read_b32 v84, a56\n\t" \
  "v_mfma_f32_16x16x32_bf16 v[204:207], a[152:155], v[112:115], v[204:207]\n\t" \
  "s_add_u32 m0, s68, 81920\n\t" \
  "v_accvgpr_read_b32 v85, a57\n\t" \
  "v_accvgpr_read_b32 v86, a58\n\t" \
  "v_mfma_f32_16x16x32_bf16 v[208:211], a[160:163], v[112:115], v[208:211]\n\t" \
  "buffer_load_dwordx4 v4, s[40:43], s78 offen offset:0 lds\n\t" \
  "v_accvgpr_read_b32 v87, a59\n\t" \
  "v_accvgpr_read_b32 v88, a60\n\t" \
  "v_mfma_f32_16x16x32_bf16 v[208:211], a[164:167], v[116:119], v[208:211]\n\t" \
  "v_accvgpr_read_b32 v89, a61\n\t" \
  "v_accvgpr_read_b32 v90, a62\n\t" \
  "v_accvgpr_read_b32 v91, a63\n\t" \
  "v_mfma_f32_16x16x32_bf16 v[212:215], a[172:175], v[116:119], v[212:215]\n\t" \
  "v_cvt_pk_bf16_f32 v80, v84, v85\n\t" \
  "v_cvt_pk_bf16_f32 v81, v86, v87\n\t" \
  "v_cvt_pk_bf16_f32 v82, v88, v89\n\t" \
  "v_mfma_f32_16x16x32_bf16 v[212:215], a[168:171], v[112:115], v[212:215]\n\t" \
  "buffer_load_dwordx4 v5, s[40:43], s78 offen offset:1024 lds\n\t" \
  "v_cvt_pk_bf16_f32 v83, v90, v91\n\t" \
  "global_store_dwordx4 v9, v[80:83], s[50:51] offset:3072 sc0 sc1\n\t" \
  "v_mfma_f32_16x16x32_bf16 v[216:219], a[176:179], v[112:115], v[216:219]\n\t" \
  "v_mfma_f32_16x16x32_bf16 v[216:219], a[180:183], v[116:119], v[216:219]\n\t" \
  "v_mfma_f32_16x16x32_bf16 v[220:223], a[188:191], v[116:119], v[220:223]\n\t" \
  "buffer_load_dwordx4 v6, s[40:43], s78 offen offset:2048 lds\n\t" \
  "v_mfma_f32_16x16x32_bf16 v[220:223], a[184:187], v[112:115], v[220:223]\n\t" \
  "ds_read_b128 v[112:115], v10 offset:53248\n\t" \
  "ds_read_b128 v[116:119], v11 offset:53248\n\t" \
  "v_mfma_f32_16x16x32_bf16 v[252:255], a[184:187], v[120:123], v[252:255]\n\t" \
  "v_mfma_f32_16x16x32_bf16 v[252:255], a[188:191], v[124:127], v[252:255]\n\t" \
  "buffer_load_dwordx4 v7, s[40:43], s78 offen offset:3072 lds\n\t" \
  "v_mfma_f32_16x16x32_bf16 v[248:251], a[180:183], v[124:127], v[248:251]\n\t" \
  "v_mfma_f32_16x16x32_bf16 v[248:251], a[176:179], v[120:123], v[248:251]\n\t" \
  "s_add_u32 m0, s68, 98304\n\t" \
  "v_mfma_f32_16x16x32_bf16 v[244:247], a[168:171], v[120:123], v[244:247]\n\t" \
  "buffer_load_dwordx4 v4, s[40:43], s79 offen offset:0 lds\n\t" \
  "v_cmp_le_u32 s[92:93], s86, v37\n\t" \
  "v_mfma_f32_16x16x32_bf16 v[244:247], a[172:175], v[124:127], v[244:247]\n\t" \
  "v_mfma_f32_16x16x32_bf16 v[240:243], a[164:167], v[124:127], v[240:243]\n\t" \
  "v_mfma_f32_16x16x32_bf16 v[240:243], a[160:163], v[120:123], v[240:243]\n\t" \
  "buffer_load_dwordx4 v5, s[40:43], s79 offen offset:1024 lds\n\t" \
  "s_or_b64 s[58:59], s[58:59], s[92:93]\n\t" \
  "v_mfma_f32_16x16x32_bf16 v[236:239], a[152:155], v[120:123], v[236:239]\n\t" \
  "v_mfma_f32_16x16x32_bf16 v[236:239], a[156:159], v[124:127], v[236:239]\n\t" \
  "s_bitcmp1_b64 s[58:59], 3\n\t" \
  "s_cbranch_scc0 SLOW219_%=\n\t" \
  "RET218_%=:\n\t" \
  "v_mfma_f32_16x16x32_bf16 v[232:235], a[148:151], v[124:127], v[232:235]\n\t" \
  "buffer_load_dwordx4 v6, s[40:43], s79 offen offset:2048 lds\n\t" \
  "v_mfma_f32_16x16x32_bf16 v[232:235], a[144:147], v[120:123], v[232:235]\n\t" \
  "v_mfma_f32_16x16x32_bf16 v[228:231], a[136:139], v[120:123], v[228:231]\n\t" \
  "v_mfma_f32_16x16x32_bf16 v[228:231], a[140:143], v[124:127], v[228:231]\n\t" \
  "v_mfma_f32_16x16x32_bf16 v[224:227], a[132:135], v[124:127], v[224:227]\n\t" \
  "buffer_load_dwordx4 v7, s[40:43], s79 offen offset:3072 lds\n\t" \
  "v_mfma_f32_16x16x32_bf16 v[224:227], a[128:131], v[120:123], v[224:227]\n\t" \
  "ds_read_b128 v[120:123], v10 offset:55296\n\t" \
  "ds_read_b128 v[124:127], v11 offset:55296\n\t" \
  "s_waitcnt lgkmcnt(4)\n\t" \
  "v_mfma_f32_16x16x32_bf16 v[128:131], a[192:195], v[96:99], v[128:131]\n\t" \
  "v_mfma_f32_16x16x32_bf16 v[128:131], a[196:199], v[100:103], v[128:131]\n\t" \
  "v_add_u32 v12, 4294934528, v12\n\t" \
  "v_accvgpr_read_b32 v68, a64\n\t" \
  "v_accvgpr_read_b32 v69, a65\n\t" \
  "v_mfma_f32_16x16x32_bf16 v[132:135], a[204:207], v[100:103], v[132:135]\n\t" \
  "v_add_u32 v13, 4294934528, v13\n\t" \
  "v_accvgpr_read_b32 v70, a66\n\t" \
  "v_accvgpr_read_b32 v71, a67\n\t" \
  "v_mfma_f32_16x16x32_bf16 v[132:135], a[200:203], v[96:99], v[132:135]\n\t" \
  "v_add_u32 v14, 4294934528, v14\n\t" \
  "v_accvgpr_read_b32 v72, a68\n\t" \
  "v_accvgpr_read_b32 v73, a69\n\t" \
  "v_mfma_f32_16x16x32_bf16 v[136:139], a[208:211], v[96:99], v[136:139]\n\t" \
  "v_add_u32 v15, 4294934528, v15\n\t" \
  "v_accvgpr_read_b32 v74, a70\n\t" \
  "v_accvgpr_read_b32 v75, a71\n\t" \
  "v_mfma_f32_16x16x32_bf16 v[136:139], a[212:215], v[100:103], v[136:139]\n\t" \
  "v_cvt_pk_bf16_f32 v64, v68, v69\n\t" \
  "v_cvt_pk_bf16_f32 v65, v70, v71\n\t" \
  "v_cvt_pk_bf16_f32 v66, v72, v73\n\t" \
  "v_mfma_f32_16x16x32_bf16 v[140:143], a[220:223], v[100:103], v[140:143]\n\t" \
  "v_cvt_pk_bf16_f32 v67, v74, v75\n\t" \
  "v_add_u32 v9, 8192, v9\n\t" \
  "global_store_dwordx4 v9, v[64:67], s[50:51] offset:-4096 sc0 sc1\n\t" \
  "v_mfma_f32_16x16x32_bf16 v[140:143], a[216:219], v[96:99], v[140:143]\n\t" \
  "v_accvgpr_read_b32 v84, a72\n\t" \
  "v_accvgpr_read_b32 v85, a73\n\t" \
  "v_accvgpr_read_b32 v86, a74\n\t" \
  "s_waitcnt vmcnt(29) lgkmcnt(0)\n\t" \
  "s_barrier\n\t" \
  "v_mfma_f32_16x16x32_bf16 v[144:147], a[224:227], v[96:99], v[144:147]\n\t" \
  "ds_read_b64_tr_b16 a[128:129], v12 offset:0\n\t" \
  "v_accvgpr_read_b32 v87, a75\n\t" \
  "v_accvgpr_read_b32 v88, a76\n\t" \
  "v_mfma_f32_16x16x32_bf16 v[144:147], a[228:231], v[100:103], v[144:147]\n\t" \
  "ds_read_b64_tr_b16 a[130:131], v12 offset:1024\n\t" \
  "v_accvgpr_read_b32 v89, a77\n\t" \
  "v_mfma_f32_16x16x32_bf16 v[148:151], a[236:239], v[100:103], v[148:151]\n\t" \
  "ds_read_b64_tr_b16 a[132:133], v12 offset:8192\n\t" \
  "s_andn2_b64 s[92:93], 255, s[58:59]\n\t" \
  "s_cbranch_scc1 POLL223_%=\n\t" \
  "RET222_%=:\n\t" \
  "v_accvgpr_read_b32 v90, a78\n\t" \
  "v_mfma_f32_16x16x32_bf16 v[148:151], a[232:235], v[96:99], v[148:151]\n\t" \
  "ds_read_b64_tr_b16 a[134:135], v12 offset:9216\n\t" \
  "v_accvgpr_read_b32 v91, a79\n\t" \
  "v_cvt_pk_bf16_f32 v80, v84, v85\n\t" \
  "v_mfma_f32_16x16x32_bf16 v[152:155], a[240:243], v[96:99], v[152:155]\n\t" \
  "ds_read_b64_tr_b16 a[136:137], v13 offset:0\n\t" \
  "v_cvt_pk_bf16_f32 v81, v86, v87\n\t" \
  "v_cvt_pk_bf16_f32 v82, v88, v89\n\t" \
  "v_mfma_f32_16x16x32_bf16 v[152:155], a[244:247], v[100:103], v[152:155]\n\t" \
  "ds_read_b64_tr_b16 a[138:139], v13 offset:1024\n\t" \
  "v_cvt_pk_bf16_f32 v83, v90, v91\n\t" \
  "global_store_dwordx4 v9, v[80:83], s[50:51] offset:-3072 sc0 sc1\n\t" \
  "v_mfma_f32_16x16x32_bf16 v[156:159], a[252:255], v[100:103], v[156:159]\n\t" \
  "ds_read_b64_tr_b16 a[140:141], v13 offset:8192\n\t" \
  "v_accvgpr_read_b32 v68, a80\n\t" \
  "v_mfma_f32_16x16x32_bf16 v[156:159], a[248:251], v[96:99], v[156:159]\n\t" \
  "ds_read_b128 v[96:99], v10 offset:0\n\t" \
  "ds_read_b128 v[100:103], v11 offset:0\n\t" \
  "v_accvgpr_read_b32 v69, a81\n\t" \
  "v_mfma_f32_16x16x32_bf16 v[188:191], a[248:251], v[104:107], v[188:191]\n\t" \
  "ds_read_b64_tr_b16 a[142:143], v13 offset:9216\n\t" \
  "v_accvgpr_read_b32 v70, a82\n\t" \
  "v_accvgpr_read_b32 v71, a83\n\t" \
  "v_mfma_f32_16x16x32_bf16 v[188:191], a[252:255], v[108:111], v[188:191]\n\t" \
  "ds_read_b64_tr_b16 a[144:145], v14 offset:0\n\t" \
  "v_accvgpr_read_b32 v72, a84\n\t" \
  "v_mfma_f32_16x16x32_bf16 v[184:187], a[244:247], v[108:111], v[184:187]\n\t" \
  "ds_read_b64_tr_b16 a[146:147], v14 offset:1024\n\t" \
  "v_accvgpr_read_b32 v73, a85\n\t" \
  "v_accvgpr_read_b32 v74, a86\n\t" \
  "v_mfma_f32_16x16x32_bf16 v[184:187], a[240:243], v[104:107], v[184:187]\n\t" \
  "ds_read_b64_tr_b16 a[148:149], v14 offset:8192\n\t" \
  "v_accvgpr_read_b32 v75, a87\n\t" \
  "v_cvt_pk_bf16_f32 v64, v68, v69\n\t" \
  "v_mfma_f32_16x16x32_bf16 v[180:183], a[232:235], v[104:107], v[180:183]\n\t" \
  "ds_read_b64_tr_b16 a[150:151], v14 offset:9216\n\t" \
  "v_cvt_pk_bf16_f32 v65, v70, v71\n\t" \
  "v_mfma_f32_16x16x32_bf16 v[180:183], a[236:239], v[108:111], v[180:183]\n\t" \
  "ds_read_b64_tr_b16 a[152:153], v15 offset:0\n\t" \
  "v_cvt_pk_bf16_f32 v66, v72, v73\n\t" \
  "v_cvt_pk_bf16_f32 v67, v74, v75\n\t" \
  "v_mfma_f32_16x16x32_bf16 v[176:179], a[228:231], v[108:111], v[176:179]\n\t" \
  "ds_read_b64_tr_b16 a[154:155], v15 offset:1024\n\t" \
  "global_store_dwordx4 v9, v[64:67], s[50:51] offset:-2048 sc0 sc1\n\t" \
  "v_accvgpr_read_b32 v84, a88\n\t" \
  "v_mfma_f32_16x16x32_bf16 v[176:179], a[224:227], v[104:107], v[176:179]\n\t" \
  "ds_read_b64_tr_b16 a[156:157], v15 offset:8192\n\t" \
  "v_accvgpr_read_b32 v85, a89\n\t" \
  "v_mfma_f32_16x16x32_bf16 v[172:175], a[216:219], v[104:107], v[172:175]\n\t" \
  "ds_read_b64_tr_b16 a[158:159], v15 offset:9216\n\t" \
  "v_accvgpr_read_b32 v86, a90\n\t" \
  "v_accvgpr_read_b32 v87, a91\n\t" \
  "v_mfma_f32_16x16x32_bf16 v[172:175], a[220:223], v[108:111], v[172:175]\n\t" \
  "ds_read_b64_tr_b16 a[160:161], v12 offset:32768\n\t" \
  "v_accvgpr_read_b32 v88, a92\n\t" \
  "v_accvgpr_read_b32 v89, a93\n\t" \
  "v_mfma_f32_16x16x32_bf16 v[168:171], a[212:215], v[108:111], v[168:171]\n\t" \
  "ds_read_b64_tr_b16 a[162:163], v12 offset:33792\n\t" \
  "v_accvgpr_read_b32 v90, a94\n\t" \
  "v_mfma_f32_16x16x32_bf16 v[168:171], a[208:211], v[104:107], v[168:171]\n\t" \
  "ds_read_b64_tr_b16 a[164:165], v12 offset:40960\n\t" \
  "v_accvgpr_read_b32 v91, a95\n\t" \
  "v_cvt_pk_bf16_f32 v80, v84, v85\n\t" \
  "v_mfma_f32_16x16x32_bf16 v[164:167], a[200:203], v[104:107], v[164:167]\n\t" \
  "ds_read_b64_tr_b16 a[166:167], v12 offset:41984\n\t" \
  "v_cvt_pk_bf16_f32 v81, v86, v87\n\t" \
  "v_cvt_pk_bf16_f32 v82, v88, v89\n\t" \
  "v_mfma_f32_16x16x32_bf16 v[164:167], a[204:207], v[108:111], v[164:167]\n\t" \
  "ds_read_b64_tr_b16 a[168:169], v13 offset:32768\n\t" \
  "v_cvt_pk_bf16_f32 v83, v90, v91\n\t" \
  "v_mfma_f32_16x16x32_bf16 v[160:163], a[196:199], v[108:111], v[160:163]\n\t" \
  "ds_read_b64_tr_b16 a[170:171], v13 offset:33792\n\t" \
  "global_store_dwordx4 v9, v[80:83], s[50:51] offset:-1024 sc0 sc1\n\t" \
  "v_accvgpr_read_b32 v68, a96\n\t" \
  "v_mfma_f32_16x16x32_bf16 v[160:163], a[192:195], v[104:107], v[160:163]\n\t" \
  "ds_read_b128 v[104:107], v10 offset:2048\n\t" \
  "ds_read_b128 v[108:111], v11 offset:2048\n\t" \
  "v_accvgpr_read_b32 v69, a97\n\t" \
  "v_mfma_f32_16x16x32_bf16 v[192:195], a[192:195], v[112:115], v[192:195]\n\t" \
  "ds_read_b64_tr_b16 a[172:173], v13 offset:40960\n\t" \
  "v_accvgpr_read_b32 v70, a98\n\t" \
  "v_mfma_f32_16x16x32_bf16 v[192:195], a[196:199], v[116:119], v[192:195]\n\t" \
  "ds_read_b64_tr_b16 a[174:175], v13 offset:41984\n\t" \
  "v_accvgpr_read_b32 v71, a99\n\t" \
  "v_accvgpr_read_b32 v72, a100\n\t" \
  "v_mfma_f32_16x16x32_bf16 v[196:199], a[204:207], v[116:119], v[196:199]\n\t" \
  "ds_read_b64_tr_b16 a[176:177], v14 offset:32768\n\t" \
  "v_accvgpr_read_b32 v73, a101\n\t" \
  "v_accvgpr_read_b32 v74, a102\n\t" \
  "v_mfma_f32_16x16x32_bf16 v[196:199], a[200:203], v[112:115], v[196:199]\n\t" \
  "ds_read_b64_tr_b16 a[178:179], v14 offset:33792\n\t" \
  "v_accvgpr_read_b32 v75, a103\n\t" \
  "v_mfma_f32_16x16x32_bf16 v[200:203], a[208:211], v[112:115], v[200:203]\n\t" \
  "ds_read_b64_tr_b16 a[180:181], v14 offset:40960\n\t" \
  "v_cvt_pk_bf16_f32 v64, v68, v69\n\t" \
  "v_cvt_pk_bf16_f32 v65, v70, v71\n\t" \
  "v_mfma_f32_16x16x32_bf16 v[200:203], a[212:215], v[116:119], v[200:203]\n\t" \
  "ds_read_b64_tr_b16 a[182:183], v14 offset:41984\n\t" \
  "v_cvt_pk_bf16_f32 v66, v72, v73\n\t" \
  "v_cvt_pk_bf16_f32 v67, v74, v75\n\t" \
  "v_mfma_f32_16x16x32_bf16 v[204:207], a[220:223], v[116:119], v[204:207]\n\t" \
  "ds_read_b64_tr_b16 a[184:185], v15 offset:32768\n\t" \
  "global_store_dwordx4 v9, v[64:67], s[50:51] offset:0 sc0 sc1\n\t" \
  "v_mfma_f32_16x16x32_bf16 v[204:207], a[216:219], v[112:115], v[204:207]\n\t" \
  "ds_read_b64_tr_b16 a[186:187], v15 offset:33792\n\t" \
  "s_add_u32 m0, s68, 49152\n\t" \
  "v_accvgpr_read_b32 v84, a104\n\t" \
  "v_mfma_f32_16x16x32_bf16 v[208:211], a[224:227], v[112:115], v[208:211]\n\t" \
  "ds_read_b64_tr_b16 a[188:189], v15 offset:40960\n\t" \
  "buffer_load_dwordx4 v0, s[36:39], s65 offen offset:0 sc0 sc1 lds\n\t" \
  "v_accvgpr_read_b32 v85, a105\n\t" \
  "v_mfma_f32_16x16x32_bf16 v[208:211], a[228:231], v[116:119], v[208:211]\n\t" \
  "ds_read_b64_tr_b16 a[190:191], v15 offset:41984\n\t" \
  "v_accvgpr_read_b32 v86, a106\n\t" \
  "v_mfma_f32_16x16x32_bf16 v[212:215], a[236:239], v[116:119], v[212:215]\n\t" \
  "v_accvgpr_read_b32 v87, a107\n\t" \
  "v_accvgpr_read_b32 v88, a108\n\t" \
  "v_accvgpr_read_b32 v89, a109\n\t" \
  "v_mfma_f32_16x16x32_bf16 v[212:215], a[232:235], v[112:115], v[212:215]\n\t" \
  "buffer_load_dwordx4 v1, s[36:39], s65 offen offset:1024 sc0 sc1 lds\n\t" \
  "v_accvgpr_read_b32 v90, a110\n\t" \
  "v_accvgpr_read_b32 v91, a111\n\t" \
  "v_mfma_f32_16x16x32_bf16 v[216:219], a[240:243], v[112:115], v[216:219]\n\t" \
  "v_cvt_pk_bf16_f32 v80, v84, v85\n\t" \
  "v_cvt_pk_bf16_f32 v81, v86, v87\n\t" \
  "v_mfma_f32_16x16x32_bf16 v[216:219], a[244:247], v[116:119], v[216:219]\n\t" \
  "v_cvt_pk_bf16_f32 v82, v88, v89\n\t" \
  "v_cvt_pk_bf16_f32 v83, v90, v91\n\t" \
  "global_store_dwordx4 v9, v[80:83], s[50:51] offset:1024 sc0 sc1\n\t" \
  "v_mfma_f32_16x16x32_bf16 v[220:223], a[252:255], v[116:119], v[220:223]\n\t" \
  "buffer_load_dwordx4 v2, s[36:39], s65 offen offset:2048 sc0 sc1 lds\n\t" \
  "v_accvgpr_read_b32 v68, a112\n\t" \
  "v_accvgpr_read_b32 v69, a113\n\t" \
  "v_mfma_f32_16x16x32_bf16 v[220:223], a[248:251], v[112:115], v[220:223]\n\t" \
  "ds_read_b128 v[112:115], v10 offset:4096\n\t" \
  "ds_read_b128 v[116:119], v11 offset:4096\n\t" \
  "v_mfma_f32_16x16x32_bf16 v[252:255], a[248:251], v[120:123], v[252:255]\n\t" \
  "v_accvgpr_read_b32 v70, a114\n\t" \
  "v_accvgpr_read_b32 v71, a115\n\t" \
  "v_accvgpr_read_b32 v72, a116\n\t" \
  "v_mfma_f32_16x16x32_bf16 v[252:255], a[252:255], v[124:127], v[252:255]\n\t" \
  "buffer_load_dwordx4 v3, s[36:39], s65 offen offset:3072 sc0 sc1 lds\n\t" \
  "v_readlane_b32 s66, v17, 4\n\t" \
  "s_nop 0\n\t" \
  "s_lshl_b32 s78, s66, 13\n\t" \
  "s_lshl_b32 s65, s66, 10\n\t" \
  "s_add_u32 s79, s78, 256\n\t" \
  "s_add_u32 s65, s65, s70\n\t" \
  "s_nop 1\n\t" \
  "v_mfma_f32_16x16x32_bf16 v[248:251], a[244:247], v[124:127], v[248:251]\n\t" \
  "v_accvgpr_read_b32 v73, a117\n\t" \
  "v_accvgpr_read_b32 v74, a118\n\t" \
  "v_mfma_f32_16x16x32_bf16 v[248:251], a[240:243], v[120:123], v[248:251]\n\t" \
  "s_add_u32 m0, s68, 114688\n\t" \
  "v_accvgpr_read_b32 v75, a119\n\t" \
  "v_cvt_pk_bf16_f32 v64, v68, v69\n\t" \
  "v_mfma_f32_16x16x32_bf16 v[244:247], a[232:235], v[120:123], v[244:247]\n\t" \
  "buffer_load_dwordx4 v4, s[40:43], s79 offen offset:0 lds\n\t" \
  "v_cmp_le_u32 s[92:93], s86, v37\n\t" \
  "v_cvt_pk_bf16_f32 v65, v70, v71\n\t" \
  "v_mfma_f32_16x16x32_bf16 v[244:247], a[236:239], v[124:127], v[244:247]\n\t" \
  "v_cvt_pk_bf16_f32 v66, v72, v73\n\t" \
  "v_cvt_pk_bf16_f32 v67, v74, v75\n\t" \
  "v_mfma_f32_16x16x32_bf16 v[240:243], a[228:231], v[124:127], v[240:243]\n\t" \
  "global_store_dwordx4 v9, v[64:67], s[50:51] offset:2048 sc0 sc1\n\t" \
  "v_accvgpr_read_b32 v84, a120\n\t" \
  "v_accvgpr_read_b32 v85, a121\n\t" \
  "v_mfma_f32_16x16x32_bf16 v[240:243], a[224:227], v[120:123], v[240:243]\n\t" \
  "buffer_load_dwordx4 v5, s[40:43], s79 offen offset:1024 lds\n\t" \
  "s_or_b64 s[58:59], s[58:59], s[92:93]\n\t" \
  "v_accvgpr_read_b32 v86, a122\n\t" \
  "v_mfma_f32_16x16x32_bf16 v[236:239], a[216:219], v[120:123], v[236:239]\n\t" \
  "v_accvgpr_read_b32 v87, a123\n\t" \
  "v_accvgpr_read_b32 v88, a124\n\t" \
  "v_mfma_f32_16x16x32_bf16 v[236:239], a[220:223], v[124:127], v[236:239]\n\t" \
  "s_bitcmp1_b64 s[58:59], 4\n\t" \
  "s_cbranch_scc0 SLOW225_%=\n\t" \
  "RET224_%=:\n\t" \
  "v_accvgpr_read_b32 v89, a125\n\t" \
  "v_accvgpr_read_b32 v90, a126\n\t" \
  "v_mfma_f32_16x16x32_bf16 v[232:235], a[212:215], v[124:127], v[232:235]\n\t" \
  "buffer_load_dwordx4 v6, s[40:43], s79 offen offset:2048 lds\n\t" \
  "v_accvgpr_read_b32 v91, a127\n\t" \
  "v_cvt_pk_bf16_f32 v80, v84, v85\n\t" \
  "v_mfma_f32_16x16x32_bf16 v[232:235], a[208:211], v[120:123], v[232:235]\n\t" \
  "s_bitcmp1_b64 s[58:59], 5\n\t" \
  "s_cbranch_scc0 SLOW229_%=\n\t" \
  "RET228_%=:\n\t" \
  "v_cvt_pk_bf16_f32 v81, v86, v87\n\t" \
  "v_mfma_f32_16x16x32_bf16 v[228:231], a[200:203], v[120:123], v[228:231]\n\t" \
  "v_cvt_pk_bf16_f32 v82, v88, v89\n\t" \
  "v_cvt_pk_bf16_f32 v83, v90, v91\n\t" \
  "global_store_dwordx4 v9, v[80:83], s[50:51] offset:3072 sc0 sc1\n\t" \
  "v_mfma_f32_16x16x32_bf16 v[228:231], a[204:207], v[124:127], v[228:231]\n\t" \
  "v_mfma_f32_16x16x32_bf16 v[224:227], a[196:199], v[124:127], v[224:227]\n\t" \
  "buffer_load_dwordx4 v7, s[40:43], s79 offen offset:3072 lds\n\t" \
  "v_mfma_f32_16x16x32_bf16 v[224:227], a[192:195], v[120:123], v[224:227]\n\t" \
  "ds_read_b128 v[120:123], v10 offset:6144\n\t" \
  "ds_read_b128 v[124:127], v11 offset:6144\n\t" \
  "s_waitcnt lgkmcnt(4)\n\t" \
  "v_mfma_f32_16x16x32_bf16 v[128:131], a[128:131], v[96:99], v[128:131]\n\t" \
  "v_mfma_f32_16x16x32_bf16 v[128:131], a[132:135], v[100:103], v[128:131]\n\t" \
  "v_add_u32 v10, 16384, v10\n\t" \
  "v_mfma_f32_16x16x32_bf16 v[132:135], a[140:143], v[100:103], v[132:135]\n\t" \
  "v_add_u32 v11, 16384, v11\n\t" \
  "v_mfma_f32_16x16x32_bf16 v[132:135], a[136:139], v[96:99], v[132:135]\n\t" \
  "v_add_u32 v12, 98304, v12\n\t" \
  "v_mfma_f32_16x16x32_bf16 v[136:139], a[144:147], v[96:99], v[136:139]\n\t" \
  "v_add_u32 v13, 98304, v13\n\t" \
  "v_mfma_f32_16x16x32_bf16 v[136:139], a[148:151], v[100:103], v[136:139]\n\t" \
  "v_add_u32 v14, 98304, v14\n\t" \
  "v_mfma_f32_16x16x32_bf16 v[140:143], a[156:159], v[100:103], v[140:143]\n\t" \
  "v_add_u32 v15, 98304, v15\n\t" \
  "v_mfma_f32_16x16x32_bf16 v[140:143], a[152:155], v[96:99], v[140:143]\n\t" \
  "s_waitcnt vmcnt(32) lgkmcnt(0)\n\t" \
  "s_barrier\n\t" \
  "v_mfma_f32_16x16x32_bf16 v[144:147], a[160:163], v[96:99], v[144:147]\n\t" \
  "ds_read_b64_tr_b16 a[192:193], v12 offset:0\n\t" \
  "v_mfma_f32_16x16x32_bf16 v[144:147], a[164:167], v[100:103], v[144:147]\n\t" \
  "ds_read_b64_tr_b16 a[194:195], v12 offset:1024\n\t" \
  "v_mfma_f32_16x16x32_bf16 v[148:151], a[172:175], v[100:103], v[148:151]\n\t" \
  "ds_read_b64_tr_b16 a[196:197], v12 offset:8192\n\t" \
  "v_mfma_f32_16x16x32_bf16 v[148:151], a[168:171], v[96:99], v[148:151]\n\t" \
  "ds_read_b64_tr_b16 a[198:199], v12 offset:9216\n\t" \
  "s_add_u32 m0, s68, 0\n\t" \
  "v_mfma_f32_16x16x32_bf16 v[152:155], a[176:179], v[96:99], v[152:155]\n\t" \
  "ds_read_b64_tr_b16 a[200:201], v13 offset:0\n\t" \
  "buffer_load_dwordx4 v0, s[36:39], s65 offen offset:0 sc0 sc1 lds\n\t" \
  "v_mfma_f32_16x16x32_bf16 v[152:155], a[180:183], v[100:103], v[152:155]\n\t" \
  "ds_read_b64_tr_b16 a[202:203], v13 offset:1024\n\t" \
  "v_mfma_f32_16x16x32_bf16 v[156:159], a[188:191], v[100:103], v[156:159]\n\t" \
  "ds_read_b64_tr_b16 a[204:205], v13 offset:8192\n\t" \
  "v_mfma_f32_16x16x32_bf16 v[156:159], a[184:187], v[96:99], v[156:159]\n\t" \
  "ds_read_b128 v[96:99], v10 offset:0\n\t" \
  "ds_read_b128 v[100:103], v11 offset:0\n\t" \
  "v_mfma_f32_16x16x32_bf16 v[188:191], a[184:187], v[104:107], v[188:191]\n\t" \
  "ds_read_b64_tr_b16 a[206:207], v13 offset:9216\n\t" \
  "buffer_load_dwordx4 v1, s[36:39], s65 offen offset:1024 sc0 sc1 lds\n\t" \
  "v_mfma_f32_16x16x32_bf16 v[188:191], a[188:191], v[108:111], v[188:191]\n\t" \
  "ds_read_b64_tr_b16 a[208:209], v14 offset:0\n\t" \
  "v_mfma_f32_16x16x32_bf16 v[184:187], a[180:183], v[108:111], v[184:187]\n\t" \
  "ds_read_b64_tr_b16 a[210:211], v14 offset:1024\n\t" \
  "v_mfma_f32_16x16x32_bf16 v[184:187], a[176:179], v[104:107], v[184:187]\n\t" \
  "ds_read_b64_tr_b16 a[212:213], v14 offset:8192\n\t" \
  "v_mfma_f32_16x16x32_bf16 v[180:183], a[168:171], v[104:107], v[180:183]\n\t" \
  "ds_read_b64_tr_b16 a[214:215], v14 offset:9216\n\t" \
  "v_mfma_f32_16x16x32_bf16 v[180:183], a[172:175], v[108:111], v[180:183]\n\t" \
  "ds_read_b64_tr_b16 a[216:217], v15 offset:0\n\t" \
  "buffer_load_dwordx4 v2, s[36:39], s65 offen offset:2048 sc0 sc1 lds\n\t" \
  "v_mfma_f32_16x16x32_bf16 v[176:179], a[164:167], v[108:111], v[176:179]\n\t" \
  "ds_read_b64_tr_b16 a[218:219], v15 offset:1024\n\t" \
  "v_mfma_f32_16x16x32_bf16 v[176:179], a[160:163], v[104:107], v[176:179]\n\t" \
  "ds_read_b64_tr_b16 a[220:221], v15 offset:8192\n\t" \
  "v_mfma_f32_16x16x32_bf16 v[172:175], a[152:155], v[104:107], v[172:175]\n\t" \
  "ds_read_b64_tr_b16 a[222:223], v15 offset:9216\n\t" \
  "v_mfma_f32_16x16x32_bf16 v[172:175], a[156:159], v[108:111], v[172:175]\n\t" \
  "ds_read_b64_tr_b16 a[224:225], v12 offset:16384\n\t" \
  "buffer_load_dwordx4 v3, s[36:39], s65 offen offset:3072 sc0 sc1 lds\n\t" \
  "v_mfma_f32_16x16x32_bf16 v[168:171], a[148:151], v[108:111], v[168:171]\n\t" \
  "ds_read_b64_tr_b16 a[226:227], v12 offset:17408\n\t" \
  "v_mfma_f32_16x16x32_bf16 v[168:171], a[144:147], v[104:107], v[168:171]\n\t" \
  "ds_read_b64_tr_b16 a[228:229], v12 offset:24576\n\t" \
  "v_mfma_f32_16x16x32_bf16 v[164:167], a[136:139], v[104:107], v[164:167]\n\t" \
  "ds_read_b64_tr_b16 a[230:231], v12 offset:25600\n\t" \
  "v_mfma_f32_16x16x32_bf16 v[164:167], a[140:143], v[108:111], v[164:167]\n\t" \
  "ds_read_b64_tr_b16 a[232:233], v13 offset:16384\n\t" \
  "s_add_u32 m0, s68, 65536\n\t" \
  "v_mfma_f32_16x16x32_bf16 v[160:163], a[132:135], v[108:111], v[160:163]\n\t" \
  "ds_read_b64_tr_b16 a[234:235], v13 offset:17408\n\t" \
  "buffer_load_dwordx4 v4, s[40:43], s78 offen offset:0 lds\n\t" \
  "v_mfma_f32_16x16x32_bf16 v[160:163], a[128:131], v[104:107], v[160:163]\n\t" \
  "ds_read_b128 v[104:107], v10 offset:2048\n\t" \
  "ds_read_b128 v[108:111], v11 offset:2048\n\t" \
  "v_mfma_f32_16x16x32_bf16 v[192:195], a[128:131], v[112:115], v[192:195]\n\t" \
  "ds_read_b64_tr_b16 a[236:237], v13 offset:24576\n\t" \
  "v_mfma_f32_16x16x32_bf16 v[192:195], a[132:135], v[116:119], v[192:195]\n\t" \
  "ds_read_b64_tr_b16 a[238:239], v13 offset:25600\n\t" \
  "v_mfma_f32_16x16x32_bf16 v[196:199], a[140:143], v[116:119], v[196:199]\n\t" \
  "ds_read_b64_tr_b16 a[240:241], v14 offset:16384\n\t" \
  "buffer_load_dwordx4 v5, s[40:43], s78 offen offset:1024 lds\n\t" \
  "v_mfma_f32_16x16x32_bf16 v[196:199], a[136:139], v[112:115], v[196:199]\n\t" \
  "ds_read_b64_tr_b16 a[242:243], v14 offset:17408\n\t" \
  "v_mfma_f32_16x16x32_bf16 v[200:203], a[144:147], v[112:115], v[200:203]\n\t" \
  "ds_read_b64_tr_b16 a[244:245], v14 offset:24576\n\t" \
  "v_mfma_f32_16x16x32_bf16 v[200:203], a[148:151], v[116:119], v[200:203]\n\t" \
  "ds_read_b64_tr_b16 a[246:247], v14 offset:25600\n\t" \
  "v_mfma_f32_16x16x32_bf16 v[204:207], a[156:159], v[116:119], v[204:207]\n\t" \
  "ds_read_b64_tr_b16 a[248:249], v15 offset:16384\n\t" \
  "v_mfma_f32_16x16x32_bf16 v[204:207], a[152:155], v[112:115], v[204:207]\n\t" \
  "ds_read_b64_tr_b16 a[250:251], v15 offset:17408\n\t" \
  "buffer_load_dwordx4 v6, s[40:43], s78 offen offset:2048 lds\n\t" \
  "v_mfma_f32_16x16x32_bf16 v[208:211], a[160:163], v[112:115], v[208:211]\n\t" \
  "ds_read_b64_tr_b16 a[252:253], v15 offset:24576\n\t" \
  "v_mfma_f32_16x16x32_bf16 v[208:211], a[164:167], v[116:119], v[208:211]\n\t" \
  "ds_read_b64_tr_b16 a[254:255], v15 offset:25600\n\t" \
  "v_mfma_f32_16x16x32_bf16 v[212:215], a[172:175], v[116:119], v[212:215]\n\t" \
  "v_mfma_f32_16x16x32_bf16 v[212:215], a[168:171], v[112:115], v[212:215]\n\t" \
  "buffer_load_dwordx4 v7, s[40:43], s78 offen offset:3072 lds\n\t" \
  "v_readlane_b32 s66, v17, 5\n\t" \
  "s_nop 0\n\t" \
  "s_lshl_b32 s78, s66, 13\n\t" \
  "s_lshl_b32 s65, s66, 10\n\t" \
  "s_add_u32 s79, s78, 256\n\t" \
  "s_add_u32 s65, s65, s70\n\t" \
  "s_nop 1\n\t" \
  "v_mfma_f32_16x16x32_bf16 v[216:219], a[176:179], v[112:115], v[216:219]\n\t" \
  "v_mfma_f32_16x16x32_bf16 v[216:219], a[180:183], v[116:119], v[216:219]\n\t" \
  "v_mfma_f32_16x16x32_bf16 v[220:223], a[188:191], v[116:119], v[220:223]\n\t" \
  "v_mfma_f32_16x16x32_bf16 v[220:223], a[184:187], v[112:115], v[220:223]\n\t" \
  "ds_read_b128 v[112:115], v10 offset:4096\n\t" \
  "ds_read_b128 v[116:119], v11 offset:4096\n\t" \
  "s_add_u32 m0, s68, 32768\n\t" \
  "v_mfma_f32_16x16x32_bf16 v[252:255], a[184:187], v[120:123], v[252:255]\n\t" \
  "buffer_load_dwordx4 v0, s[36:39], s65 offen offset:0 sc0 sc1 lds\n\t" \
  "v_mfma_f32_16x16x32_bf16 v[252:255], a[188:191], v[124:127], v[252:255]\n\t" \
  "v_mfma_f32_16x16x32_bf16 v[248:251], a[180:183], v[124:127], v[248:251]\n\t" \
  "v_mfma_f32_16x16x32_bf16 v[248:251], a[176:179], v[120:123], v[248:251]\n\t" \
  "v_mfma_f32_16x16x32_bf16 v[244:247], a[168:171], v[120:123], v[244:247]\n\t" \
  "buffer_load_dwordx4 v1, s[36:39], s65 offen offset:1024 sc0 sc1 lds\n\t" \
  "v_cmp_le_u32 s[92:93], s86, v37\n\t" \
  "v_mfma_f32_16x16x32_bf16 v[244:247], a[172:175], v[124:127], v[244:247]\n\t" \
  "v_mfma_f32_16x16x32_bf16 v[240:243], a[164:167], v[124:127], v[240:243]\n\t" \
  "v_mfma_f32_16x16x32_bf16 v[240:243], a[160:163], v[120:123], v[240:243]\n\t" \
  "s_or_b64 s[58:59], s[58:59], s[92:93]\n\t" \
  "v_mfma_f32_16x16x32_bf16 v[236:239], a[152:155], v[120:123], v[236:239]\n\t" \
  "v_mfma_f32_16x16x32_bf16 v[236:239], a[156:159], v[124:127], v[236:239]\n\t" \
  "buffer_load_dwordx4 v2, s[36:39], s65 offen offset:2048 sc0 sc1 lds\n\t" \
  "s_bitcmp1_b64 s[58:59], 6\n\t" \
  "s_cbranch_scc0 SLOW233_%=\n\t" \
  "RET232_%=:\n\t" \
  "v_mfma_f32_16x16x32_bf16 v[232:235], a[148:151], v[124:127], v[232:235]\n\t" \
  "v_mfma_f32_16x16x32_bf16 v[232:235], a[144:147], v[120:123], v[232:235]\n\t" \
  "v_mfma_f32_16x16x32_bf16 v[228:231], a[136:139], v[120:123], v[228:231]\n\t" \
  "v_mfma_f32_16x16x32_bf16 v[228:231], a[140:143], v[124:127], v[228:231]\n\t" \
  "v_mfma_f32_16x16x32_bf16 v[224:227], a[132:135], v[124:127], v[224:227]\n\t" \
  "buffer_load_dwordx4 v3, s[36:39], s65 offen offset:3072 sc0 sc1 lds\n\t" \
  "v_mfma_f32_16x16x32_bf16 v[224:227], a[128:131], v[120:123], v[224:227]\n\t" \
  "ds_read_b128 v[120:123], v10 offset:6144\n\t" \
  "ds_read_b128 v[124:127], v11 offset:6144\n\t" \
  "s_waitcnt lgkmcnt(4)\n\t" \
  "v_mfma_f32_16x16x32_bf16 v[128:131], a[192:195], v[96:99], v[128:131]\n\t" \
  "v_mfma_f32_16x16x32_bf16 v[128:131], a[196:199], v[100:103], v[128:131]\n\t" \
  "v_add_u32 v10, 32768, v10\n\t" \
  "v_mfma_f32_16x16x32_bf16 v[132:135], a[204:207], v[100:103], v[132:135]\n\t" \
  "v_add_u32 v11, 32768, v11\n\t" \
  "v_mfma_f32_16x16x32_bf16 v[132:135], a[200:203], v[96:99], v[132:135]\n\t" \
  "v_add_u32 v12, 4294918144, v12\n\t" \
  "v_mfma_f32_16x16x32_bf16 v[136:139], a[208:211], v[96:99], v[136:139]\n\t" \
  "v_add_u32 v13, 4294918144, v13\n\t" \
  "v_mfma_f32_16x16x32_bf16 v[136:139], a[212:215], v[100:103], v[136:139]\n\t" \
  "v_add_u32 v14, 4294918144, v14\n\t" \
  "v_mfma_f32_16x16x32_bf16 v[140:143], a[220:223], v[100:103], v[140:143]\n\t" \
  "v_add_u32 v15, 4294918144, v15\n\t" \
  "v_mfma_f32_16x16x32_bf16 v[140:143], a[216:219], v[96:99], v[140:143]\n\t" \
  "s_waitcnt vmcnt(18) lgkmcnt(0)\n\t" \
  "s_barrier\n\t" \
  "v_mfma_f32_16x16x32_bf16 v[144:147], a[224:227], v[96:99], v[144:147]\n\t" \
  "ds_read_b64_tr_b16 a[128:129], v12 offset:0\n\t" \
  "v_mfma_f32_16x16x32_bf16 v[144:147], a[228:231], v[100:103], v[144:147]\n\t" \
  "ds_read_b64_tr_b16 a[130:131], v12 offset:1024\n\t" \
  "v_mfma_f32_16x16x32_bf16 v[148:151], a[236:239], v[100:103], v[148:151]\n\t" \
  "ds_read_b64_tr_b16 a[132:133], v12 offset:8192\n\t" \
  "v_mfma_f32_16x16x32_bf16 v[148:151], a[232:235], v[96:99], v[148:151]\n\t" \
  "ds_read_b64_tr_b16 a[134:135], v12 offset:9216\n\t" \
  "s_add_u32 m0, s68, 131072\n\t" \
  "v_mfma_f32_16x16x32_bf16 v[152:155], a[240:243], v[96:99], v[152:155]\n\t" \
  "ds_read_b64_tr_b16 a[136:137], v13 offset:0\n\t" \
  "buffer_load_dwordx4 v4, s[40:43], s78 offen offset:0 lds\n\t" \
  "v_mfma_f32_16x16x32_bf16 v[152:155], a[244:247], v[100:103], v[152:155]\n\t" \
  "ds_read_b64_tr_b16 a[138:139], v13 offset:1024\n\t" \
  "v_mfma_f32_16x16x32_bf16 v[156:159], a[252:255], v[100:103], v[156:159]\n\t" \
  "ds_read_b64_tr_b16 a[140:141], v13 offset:8192\n\t" \
  "v_mfma_f32_16x16x32_bf16 v[156:159], a[248:251], v[96:99], v[156:159]\n\t" \
  "ds_read_b128 v[96:99], v10 offset:0\n\t" \
  "ds_read_b128 v[100:103], v11 offset:0\n\t" \
  "v_mfma_f32_16x16x32_bf16 v[188:191], a[248:251], v[104:107], v[188:191]\n\t" \
  "ds_read_b64_tr_b16 a[142:143], v13 offset:9216\n\t" \
  "buffer_load_dwordx4 v5, s[40:43], s78 offen offset:1024 lds\n\t" \
  "v_mfma_f32_16x16x32_bf16 v[188:191], a[252:255], v[108:111], v[188:191]\n\t" \
  "ds_read_b64_tr_b16 a[144:145], v14 offset:0\n\t" \
  "v_mfma_f32_16x16x32_bf16 v[184:187], a[244:247], v[108:111], v[184:187]\n\t" \
  "ds_read_b64_tr_b16 a[146:147], v14 offset:1024\n\t" \
  "v_mfma_f32_16x16x32_bf16 v[184:187], a[240:243], v[104:107], v[184:187]\n\t" \
  "ds_read_b64_tr_b16 a[148:149], v14 offset:8192\n\t" \
  "v_mfma_f32_16x16x32_bf16 v[180:183], a[232:235], v[104:107], v[180:183]\n\t" \
  "ds_read_b64_tr_b16 a[150:151], v14 offset:9216\n\t" \
  "v_mfma_f32_16x16x32_bf16 v[180:183], a[236:239], v[108:111], v[180:183]\n\t" \
  "ds_read_b64_tr_b16 a[152:153], v15 offset:0\n\t" \
  "buffer_load_dwordx4 v6, s[40:43], s78 offen offset:2048 lds\n\t" \
  "v_mfma_f32_16x16x32_bf16 v[176:179], a[228:231], v[108:111], v[176:179]\n\t" \
  "ds_read_b64_tr_b16 a[154:155], v15 offset:1024\n\t" \
  "v_mfma_f32_16x16x32_bf16 v[176:179], a[224:227], v[104:107], v[176:179]\n\t" \
  "ds_read_b64_tr_b16 a[156:157], v15 offset:8192\n\t" \
  "v_mfma_f32_16x16x32_bf16 v[172:175], a[216:219], v[104:107], v[172:175]\n\t" \
  "ds_read_b64_tr_b16 a[158:159], v15 offset:9216\n\t" \
  "v_mfma_f32_16x16x32_bf16 v[172:175], a[220:223], v[108:111], v[172:175]\n\t" \
  "ds_read_b64_tr_b16 a[160:161], v12 offset:16384\n\t" \
  "buffer_load_dwordx4 v7, s[40:43], s78 offen offset:3072 lds\n\t" \
  "v_mfma_f32_16x16x32_bf16 v[168:171], a[212:215], v[108:111], v[168:171]\n\t" \
  "ds_read_b64_tr_b16 a[162:163], v12 offset:17408\n\t" \
  "v_mfma_f32_16x16x32_bf16 v[168:171], a[208:211], v[104:107], v[168:171]\n\t" \
  "ds_read_b64_tr_b16 a[164:165], v12 offset:24576\n\t" \
  "v_mfma_f32_16x16x32_bf16 v[164:167], a[200:203], v[104:107], v[164:167]\n\t" \
  "ds_read_b64_tr_b16 a[166:167], v12 offset:25600\n\t" \
  "v_mfma_f32_16x16x32_bf16 v[164:167], a[204:207], v[108:111], v[164:167]\n\t" \
  "ds_read_b64_tr_b16 a[168:169], v13 offset:16384\n\t" \
  "s_add_u32 m0, s68, 147456\n\t" \
  "v_mfma_f32_16x16x32_bf16 v[160:163], a[196:199], v[108:111], v[160:163]\n\t" \
  "ds_read_b64_tr_b16 a[170:171], v13 offset:17408\n\t" \
  "buffer_load_dwordx4 v4, s[40:43], s79 offen offset:0 lds\n\t" \
  "v_mfma_f32_16x16x32_bf16 v[160:163], a[192:195], v[104:107], v[160:163]\n\t" \
  "ds_read_b128 v[104:107], v10 offset:2048\n\t" \
  "ds_read_b128 v[108:111], v11 offset:2048\n\t" \
  "v_mfma_f32_16x16x32_bf16 v[192:195], a[192:195], v[112:115], v[192:195]\n\t" \
  "ds_read_b64_tr_b16 a[172:173], v13 offset:24576\n\t" \
  "v_mfma_f32_16x16x32_bf16 v[192:195], a[196:199], v[116:119], v[192:195]\n\t" \
  "ds_read_b64_tr_b16 a[174:175], v13 offset:25600\n\t" \
  "v_mfma_f32_16x16x32_bf16 v[196:199], a[204:207], v[116:119], v[196:199]\n\t" \
  "ds_read_b64_tr_b16 a[176:177], v14 offset:16384\n\t" \
  "buffer_load_dwordx4 v5, s[40:43], s79 offen offset:1024 lds\n\t" \
  "v_mfma_f32_16x16x32_bf16 v[196:199], a[200:203], v[112:115], v[196:199]\n\t" \
  "ds_read_b64_tr_b16 a[178:179], v14 offset:17408\n\t" \
  "v_mfma_f32_16x16x32_bf16 v[200:203], a[208:211], v[112:115], v[200:203]\n\t" \
  "ds_read_b64_tr_b16 a[180:181], v14 offset:24576\n\t" \
  "v_mfma_f32_16x16x32_bf16 v[200:203], a[212:215], v[116:119], v[200:203]\n\t" \
  "ds_read_b64_tr_b16 a[182:183], v14 offset:25600\n\t" \
  "v_mfma_f32_16x16x32_bf16 v[204:207], a[220:223], v[116:119], v[204:207]\n\t" \
  "ds_read_b64_tr_b16 a[184:185], v15 offset:16384\n\t" \
  "v_mfma_f32_16x16x32_bf16 v[204:207], a[216:219], v[112:115], v[204:207]\n\t" \
  "ds_read_b64_tr_b16 a[186:187], v15 offset:17408\n\t" \
  "buffer_load_dwordx4 v6, s[40:43], s79 offen offset:2048 lds\n\t" \
  "v_mfma_f32_16x16x32_bf16 v[208:211], a[224:227], v[112:115], v[208:211]\n\t" \
  "ds_read_b64_tr_b16 a[188:189], v15 offset:24576\n\t" \
  "v_mfma_f32_16x16x32_bf16 v[208:211], a[228:231], v[116:119], v[208:211]\n\t" \
  "ds_read_b64_tr_b16 a[190:191], v15 offset:25600\n\t" \
  "v_mfma_f32_16x16x32_bf16 v[212:215], a[236:239], v[116:119], v[212:215]\n\t" \
  "v_mfma_f32_16x16x32_bf16 v[212:215], a[232:235], v[112:115], v[212:215]\n\t" \
  "buffer_load_dwordx4 v7, s[40:43], s79 offen offset:3072 lds\n\t" \
  "v_readlane_b32 s66, v17, 6\n\t" \
  "s_nop 0\n\t" \
  "s_lshl_b32 s78, s66, 13\n\t" \
  "s_lshl_b32 s65, s66, 10\n\t" \
  "s_add_u32 s79, s78, 256\n\t" \
  "s_add_u32 s65, s65, s70\n\t" \
  "s_nop 1\n\t" \
  "v_mfma_f32_16x16x32_bf16 v[216:219], a[240:243], v[112:115], v[216:219]\n\t" \
  "v_mfma_f32_16x16x32_bf16 v[216:219], a[244:247], v[116:119], v[216:219]\n\t" \
  "v_mfma_f32_16x16x32_bf16 v[220:223], a[252:255], v[116:119], v[220:223]\n\t" \
  "v_mfma_f32_16x16x32_bf16 v[220:223], a[248:251], v[112:115], v[220:223]\n\t" \
  "ds_read_b128 v[112:115], v10 offset:4096\n\t" \
  "ds_read_b128 v[116:119], v11 offset:4096\n\t" \
  "s_add_u32 m0, s68, 16384\n\t" \
  "v_mfma_f32_16x16x32_bf16 v[252:255], a[248:251], v[120:123], v[252:255]\n\t" \
  "buffer_load_dwordx4 v0, s[36:39], s65 offen offset:0 sc0 sc1 lds\n\t" \
  "v_mfma_f32_16x16x32_bf16 v[252:255], a[252:255], v[124:127], v[252:255]\n\t" \
  "v_mfma_f32_16x16x32_bf16 v[248:251], a[244:247], v[124:127], v[248:251]\n\t" \
  "v_mfma_f32_16x16x32_bf16 v[248:251], a[240:243], v[120:123], v[248:251]\n\t" \
  "v_mfma_f32_16x16x32_bf16 v[244:247], a[232:235], v[120:123], v[244:247]\n\t" \
  "buffer_load_dwordx4 v1, s[36:39], s65 offen offset:1024 sc0 sc1 lds\n\t" \
  "v_mfma_f32_16x16x32_bf16 v[244:247], a[236:239], v[124:127], v[244:247]\n\t" \
  "v_mfma_f32_16x16x32_bf16 v[240:243], a[228:231], v[124:127], v[240:243]\n\t" \
  "v_mfma_f32_16x16x32_bf16 v[240:243], a[224:227], v[120:123], v[240:243]\n\t" \
  "v_mfma_f32_16x16x32_bf16 v[236:239], a[216:219], v[120:123], v[236:239]\n\t" \
  "v_mfma_f32_16x16x32_bf16 v[236:239], a[220:223], v[124:127], v[236:239]\n\t" \
  "buffer_load_dwordx4 v2, s[36:39], s65 offen offset:2048 sc0 sc1 lds\n\t" \
  "v_mfma_f32_16x16x32_bf16 v[232:235], a[212:215], v[124:127], v[232:235]\n\t" \
  "v_mfma_f32_16x16x32_bf16 v[232:235], a[208:211], v[120:123], v[232:235]\n\t" \
  "v_mfma_f32_16x16x32_bf16 v[228:231], a[200:203], v[120:123], v[228:231]\n\t" \
  "v_mfma_f32_16x16x32_bf16 v[228:231], a[204:207], v[124:127], v[228:231]\n\t" \
  "v_mfma_f32_16x16x32_bf16 v[224:227], a[196:199], v[124:127], v[224:227]\n\t" \
  "buffer_load_dwordx4 v3, s[36:39], s65 offen offset:3072 sc0 sc1 lds\n\t" \
  "v_mfma_f32_16x16x32_bf16 v[224:227], a[192:195], v[120:123], v[224:227]\n\t" \
  "ds_read_b128 v[120:123], v10 offset:6144\n\t" \
  "ds_read_b128 v[124:127], v11 offset:6144\n\t" \
  "s_waitcnt lgkmcnt(4)\n\t" \
  "v_mfma_f32_16x16x32_bf16 v[128:131], a[128:131], v[96:99], v[128:131]\n\t" \
  "v_mfma_f32_16x16x32_bf16 v[128:131], a[132:135], v[100:103], v[128:131]\n\t" \
  "v_add_u32 v10, 4294918144, v10\n\t" \
  "v_mfma_f32_16x16x32_bf16 v[132:135], a[140:143], v[100:103], v[132:135]\n\t" \
  "v_add_u32 v11, 4294918144, v11\n\t" \
  "v_mfma_f32_16x16x32_bf16 v[132:135], a[136:139], v[96:99], v[132:135]\n\t" \
  "v_add_u32 v12, 4294950912, v12\n\t" \
  "v_mfma_f32_16x16x32_bf16 v[136:139], a[144:147], v[96:99], v[136:139]\n\t" \
  "v_add_u32 v13, 4294950912, v13\n\t" \
  "v_mfma_f32_16x16x32_bf16 v[136:139], a[148:151], v[100:103], v[136:139]\n\t" \
  "v_add_u32 v14, 4294950912, v14\n\t" \
  "v_mfma_f32_16x16x32_bf16 v[140:143], a[156:159], v[100:103], v[140:143]\n\t" \
  "v_add_u32 v15, 4294950912, v15\n\t" \
  "v_mfma_f32_16x16x32_bf16 v[140:143], a[152:155], v[96:99], v[140:143]\n\t" \
  "s_waitcnt vmcnt(16) lgkmcnt(0)\n\t" \
  "s_barrier\n\t" \
  "s_cmp_lg_u32 s71, 0\n\t" \
  "s_cbranch_scc1 XS236_%=\n\t" \
  "v_mov_b32 v50, s85\n\t" \
  "s_mov_b64 exec, 1\n\t" \
  "global_store_dword v18, v50, s[50:51] sc0 sc1\n\t" \
  "s_mov_b64 exec, -1\n\t" \
  "XS236_%=:\n\t" \
  "v_mfma_f32_16x16x32_bf16 v[144:147], a[160:163], v[96:99], v[144:147]\n\t" \
  "ds_read_b64_tr_b16 a[192:193], v12 offset:0\n\t" \
  "v_mfma_f32_16x16x32_bf16 v[144:147], a[164:167], v[100:103], v[144:147]\n\t" \
  "ds_read_b64_tr_b16 a[194:195], v12 offset:1024\n\t" \
  "global_load_dword v49, v19, s[50:51] sc0 sc1\n\t" \
  "v_mfma_f32_16x16x32_bf16 v[148:151], a[172:175], v[100:103], v[148:151]\n\t" \
  "ds_read_b64_tr_b16 a[196:197], v12 offset:8192\n\t" \
  "v_mfma_f32_16x16x32_bf16 v[148:151], a[168:171], v[96:99], v[148:151]\n\t" \
  "ds_read_b64_tr_b16 a[198:199], v12 offset:9216\n\t" \
  "s_add_u32 m0, s68, 49152\n\t" \
  "v_mfma_f32_16x16x32_bf16 v[152:155], a[176:179], v[96:99], v[152:155]\n\t" \
  "ds_read_b64_tr_b16 a[200:201], v13 offset:0\n\t" \
  "buffer_load_dwordx4 v4, s[40:43], s78 offen offset:0 lds\n\t" \
  "v_mfma_f32_16x16x32_bf16 v[152:155], a[180:183], v[100:103], v[152:155]\n\t" \
  "ds_read_b64_tr_b16 a[202:203], v13 offset:1024\n\t" \
  "v_mfma_f32_16x16x32_bf16 v[156:159], a[188:191], v[100:103], v[156:159]\n\t" \
  "ds_read_b64_tr_b16 a[204:205], v13 offset:8192\n\t" \
  "v_mfma_f32_16x16x32_bf16 v[156:159], a[184:187], v[96:99], v[156:159]\n\t" \
  "ds_read_b128 v[96:99], v10 offset:0\n\t" \
  "ds_read_b128 v[100:103], v11 offset:0\n\t" \
  "v_mfma_f32_16x16x32_bf16 v[188:191], a[184:187], v[104:107], v[188:191]\n\t" \
  "ds_read_b64_tr_b16 a[206:207], v13 offset:9216\n\t" \
  "buffer_load_dwordx4 v5, s[40:43], s78 offen offset:1024 lds\n\t" \
  "v_mfma_f32_16x16x32_bf16 v[188:191], a[188:191], v[108:111], v[188:191]\n\t" \
  "ds_read_b64_tr_b16 a[208:209], v14 offset:0\n\t" \
  "v_mfma_f32_16x16x32_bf16 v[184:187], a[180:183], v[108:111], v[184:187]\n\t" \
  "ds_read_b64_tr_b16 a[210:211], v14 offset:1024\n\t" \
  "v_mfma_f32_16x16x32_bf16 v[184:187], a[176:179], v[104:107], v[184:187]\n\t" \
  "ds_read_b64_tr_b16 a[212:213], v14 offset:8192\n\t" \
  "v_mfma_f32_16x16x32_bf16 v[180:183], a[168:171], v[104:107], v[180:183]\n\t" \
  "ds_read_b64_tr_b16 a[214:215], v14 offset:9216\n\t" \
  "v_mfma_f32_16x16x32_bf16 v[180:183], a[172:175], v[108:111], v[180:183]\n\t" \
  "ds_read_b64_tr_b16 a[216:217], v15 offset:0\n\t" \
  "buffer_load_dwordx4 v6, s[40:43], s78 offen offset:2048 lds\n\t" \
  "v_mfma_f32_16x16x32_bf16 v[176:179], a[164:167], v[108:111], v[176:179]\n\t" \
  "ds_read_b64_tr_b16 a[218:219], v15 offset:1024\n\t" \
  "v_mfma_f32_16x16x32_bf16 v[176:179], a[160:163], v[104:107], v[176:179]\n\t" \
  "ds_read_b64_tr_b16 a[220:221], v15 offset:8192\n\t" \
  "v_mfma_f32_16x16x32_bf16 v[172:175], a[152:155], v[104:107], v[172:175]\n\t" \
  "ds_read_b64_tr_b16 a[222:223], v15 offset:9216\n\t" \
  "v_mfma_f32_16x16x32_bf16 v[172:175], a[156:159], v[108:111], v[172:175]\n\t" \
  "ds_read_b64_tr_b16 a[224:225], v12 offset:49152\n\t" \
  "buffer_load_dwordx4 v7, s[40:43], s78 offen offset:3072 lds\n\t" \
  "v_mfma_f32_16x16x32_bf16 v[168:171], a[148:151], v[108:111], v[168:171]\n\t" \
  "ds_read_b64_tr_b16 a[226:227], v12 offset:50176\n\t" \
  "v_mfma_f32_16x16x32_bf16 v[168:171], a[144:147], v[104:107], v[168:171]\n\t" \
  "ds_read_b64_tr_b16 a[228:229], v12 offset:57344\n\t" \
  "v_mfma_f32_16x16x32_bf16 v[164:167], a[136:139], v[104:107], v[164:167]\n\t" \
  "ds_read_b64_tr_b16 a[230:231], v12 offset:58368\n\t" \
  "v_mfma_f32_16x16x32_bf16 v[164:167], a[140:143], v[108:111], v[164:167]\n\t" \
  "ds_read_b64_tr_b16 a[232:233], v13 offset:49152\n\t" \
  "s_add_u32 m0, s68, 81920\n\t" \
  "v_mfma_f32_16x16x32_bf16 v[160:163], a[132:135], v[108:111], v[160:163]\n\t" \
  "ds_read_b64_tr_b16 a[234:235], v13 offset:50176\n\t" \
  "buffer_load_dwordx4 v4, s[40:43], s79 offen offset:0 lds\n\t" \
  "v_mfma_f32_16x16x32_bf16 v[160:163], a[128:131], v[104:107], v[160:163]\n\t" \
  "ds_read_b128 v[104:107], v10 offset:2048\n\t" \
  "ds_read_b128 v[108:111], v11 offset:2048\n\t" \
  "v_mfma_f32_16x16x32_bf16 v[192:195], a[128:131], v[112:115], v[192:195]\n\t" \
  "ds_read_b64_tr_b16 a[236:237], v13 offset:57344\n\t" \
  "v_mfma_f32_16x16x32_bf16 v[192:195], a[132:135], v[116:119], v[192:195]\n\t" \
  "ds_read_b64_tr_b16 a[238:239], v13 offset:58368\n\t" \
  "v_mfma_f32_16x16x32_bf16 v[196:199], a[140:143], v[116:119], v[196:199]\n\t" \
  "ds_read_b64_tr_b16 a[240:241], v14 offset:49152\n\t" \
  "buffer_load_dwordx4 v5, s[40:43], s79 offen offset:1024 lds\n\t" \
  "v_mfma_f32_16x16x32_bf16 v[196:199], a[136:139], v[112:115], v[196:199]\n\t" \
  "ds_read_b64_tr_b16 a[242:243], v14 offset:50176\n\t" \
  "v_mfma_f32_16x16x32_bf16 v[200:203], a[144:147], v[112:115], v[200:203]\n\t" \
  "ds_read_b64_tr_b16 a[244:245], v14 offset:57344\n\t" \
  "v_mfma_f32_16x16x32_bf16 v[200:203], a[148:151], v[116:119], v[200:203]\n\t" \
  "ds_read_b64_tr_b16 a[246:247], v14 offset:58368\n\t" \
  "v_mfma_f32_16x16x32_bf16 v[204:207], a[156:159], v[116:119], v[204:207]\n\t" \
  "ds_read_b64_tr_b16 a[248:249], v15 offset:49152\n\t" \
  "v_mfma_f32_16x16x32_bf16 v[204:207], a[152:155], v[112:115], v[204:207]\n\t" \
  "ds_read_b64_tr_b16 a[250:251], v15 offset:50176\n\t" \
  "buffer_load_dwordx4 v6, s[40:43], s79 offen offset:2048 lds\n\t" \
  "v_mfma_f32_16x16x32_bf16 v[208:211], a[160:163], v[112:115], v[208:211]\n\t" \
  "ds_read_b64_tr_b16 a[252:253], v15 offset:57344\n\t" \
  "v_mfma_f32_16x16x32_bf16 v[208:211], a[164:167], v[116:119], v[208:211]\n\t" \
  "ds_read_b64_tr_b16 a[254:255], v15 offset:58368\n\t" \
  "v_mfma_f32_16x16x32_bf16 v[212:215], a[172:175], v[116:119], v[212:215]\n\t" \
  "v_mfma_f32_16x16x32_bf16 v[212:215], a[168:171], v[112:115], v[212:215]\n\t" \
  "buffer_load_dwordx4 v7, s[40:43], s79 offen offset:3072 lds\n\t" \
  "v_readlane_b32 s66, v17, 7\n\t" \
  "s_nop 0\n\t" \
  "s_lshl_b32 s78, s66, 13\n\t" \
  "s_lshl_b32 s65, s66, 10\n\t" \
  "s_add_u32 s79, s78, 256\n\t" \
  "s_add_u32 s65, s65, s70\n\t" \
  "s_nop 1\n\t" \
  "v_mfma_f32_16x16x32_bf16 v[216:219], a[176:179], v[112:115], v[216:219]\n\t" \
  "v_mfma_f32_16x16x32_bf16 v[216:219], a[180:183], v[116:119], v[216:219]\n\t" \
  "v_mfma_f32_16x16x32_bf16 v[220:223], a[188:191], v[116:119], v[220:223]\n\t" \
  "v_mfma_f32_16x16x32_bf16 v[220:223], a[184:187], v[112:115], v[220:223]\n\t" \
  "ds_read_b128 v[112:115], v10 offset:4096\n\t" \
  "ds_read_b128 v[116:119], v11 offset:4096\n\t" \
  "s_add_u32 m0, s68, 98304\n\t" \
  "v_mfma_f32_16x16x32_bf16 v[252:255], a[184:187], v[120:123], v[252:255]\n\t" \
  "buffer_load_dwordx4 v4, s[40:43], s79 offen offset:0 lds\n\t" \
  "v_mfma_f32_16x16x32_bf16 v[252:255], a[188:191], v[124:127], v[252:255]\n\t" \
  "v_mfma_f32_16x16x32_bf16 v[248:251], a[180:183], v[124:127], v[248:251]\n\t" \
  "v_mfma_f32_16x16x32_bf16 v[248:251], a[176:179], v[120:123], v[248:251]\n\t" \
  "v_mfma_f32_16x16x32_bf16 v[244:247], a[168:171], v[120:123], v[244:247]\n\t" \
  "buffer_load_dwordx4 v5, s[40:43], s79 offen offset:1024 lds\n\t" \
  "v_cmp_le_u32 s[92:93], s86, v37\n\t" \
  "v_mfma_f32_16x16x32_bf16 v[244:247], a[172:175], v[124:127], v[244:247]\n\t" \
  "v_mfma_f32_16x16x32_bf16 v[240:243], a[164:167], v[124:127], v[240:243]\n\t" \
  "v_mfma_f32_16x16x32_bf16 v[240:243], a[160:163], v[120:123], v[240:243]\n\t" \
  "s_or_b64 s[58:59], s[58:59], s[92:93]\n\t" \
  "v_mfma_f32_16x16x32_bf16 v[236:239], a[152:155], v[120:123], v[236:239]\n\t" \
  "v_mfma_f32_16x16x32_bf16 v[236:239], a[156:159], v[124:127], v[236:239]\n\t" \
  "buffer_load_dwordx4 v6, s[40:43], s79 offen offset:2048 lds\n\t" \
  "s_bitcmp1_b64 s[58:59], 7\n\t" \
  "s_cbranch_scc0 SLOW238_%=\n\t" \
  "RET237_%=:\n\t" \
  "v_mfma_f32_16x16x32_bf16 v[232:235], a[148:151], v[124:127], v[232:235]\n\t" \
  "v_mfma_f32_16x16x32_bf16 v[232:235], a[144:147], v[120:123], v[232:235]\n\t" \
  "v_mfma_f32_16x16x32_bf16 v[228:231], a[136:139], v[120:123], v[228:231]\n\t" \
  "v_mfma_f32_16x16x32_bf16 v[228:231], a[140:143], v[124:127], v[228:231]\n\t" \
  "v_mfma_f32_16x16x32_bf16 v[224:227], a[132:135], v[124:127], v[224:227]\n\t" \
  "buffer_load_dwordx4 v7, s[40:43], s79 offen offset:3072 lds\n\t" \
  "v_mfma_f32_16x16x32_bf16 v[224:227], a[128:131], v[120:123], v[224:227]\n\t" \
  "ds_read_b128 v[120:123], v10 offset:6144\n\t" \
  "ds_read_b128 v[124:127], v11 offset:6144\n\t" \
  "s_waitcnt lgkmcnt(4)\n\t" \
  "v_mfma_f32_16x16x32_bf16 v[128:131], a[192:195], v[96:99], v[128:131]\n\t" \
  "v_mfma_f32_16x16x32_bf16 v[128:131], a[196:199], v[100:103], v[128:131]\n\t" \
  "v_add_u32 v10, 32768, v10\n\t" \
  "v_mfma_f32_16x16x32_bf16 v[132:135], a[204:207], v[100:103], v[132:135]\n\t" \
  "v_add_u32 v11, 32768, v11\n\t" \
  "v_mfma_f32_16x16x32_bf16 v[132:135], a[200:203], v[96:99], v[132:135]\n\t" \
  "v_add_u32 v12, 65536, v12\n\t" \
  "v_mfma_f32_16x16x32_bf16 v[136:139], a[208:211], v[96:99], v[136:139]\n\t" \
  "v_add_u32 v13, 65536, v13\n\t" \
  "v_mfma_f32_16x16x32_bf16 v[136:139], a[212:215], v[100:103], v[136:139]\n\t" \
  "v_add_u32 v14, 65536, v14\n\t" \
  "v_mfma_f32_16x16x32_bf16 v[140:143], a[220:223], v[100:103], v[140:143]\n\t" \
  "v_add_u32 v15, 65536, v15\n\t" \
  "v_mfma_f32_16x16x32_bf16 v[140:143], a[216:219], v[96:99], v[140:143]\n\t" \
  "s_waitcnt vmcnt(17) lgkmcnt(0)\n\t" \
  "s_barrier\n\t" \
  "v_mfma_f32_16x16x32_bf16 v[144:147], a[224:227], v[96:99], v[144:147]\n\t" \
  "ds_read_b64_tr_b16 a[128:129], v12 offset:0\n\t" \
  "v_mfma_f32_16x16x32_bf16 v[144:147], a[228:231], v[100:103], v[144:147]\n\t" \
  "ds_read_b64_tr_b16 a[130:131], v12 offset:1024\n\t" \
  "v_mfma_f32_16x16x32_bf16 v[148:151], a[236:239], v[100:103], v[148:151]\n\t" \
  "ds_read_b64_tr_b16 a[132:133], v12 offset:8192\n\t" \
  "v_mfma_f32_16x16x32_bf16 v[148:151], a[232:235], v[96:99], v[148:151]\n\t" \
  "ds_read_b64_tr_b16 a[134:135], v12 offset:9216\n\t" \
  "v_mfma_f32_16x16x32_bf16 v[152:155], a[240:243], v[96:99], v[152:155]\n\t" \
  "ds_read_b64_tr_b16 a[136:137], v13 offset:0\n\t" \
  "v_mfma_f32_16x16x32_bf16 v[152:155], a[244:247], v[100:103], v[152:155]\n\t" \
  "ds_read_b64_tr_b16 a[138:139], v13 offset:1024\n\t" \
  "v_mfma_f32_16x16x32_bf16 v[156:159], a[252:255], v[100:103], v[156:159]\n\t" \
  "ds_read_b64_tr_b16 a[140:141], v13 offset:8192\n\t" \
  "v_mfma_f32_16x16x32_bf16 v[156:159], a[248:251], v[96:99], v[156:159]\n\t" \
  "ds_read_b128 v[96:99], v10 offset:0\n\t" \
  "ds_read_b128 v[100:103], v11 offset:0\n\t" \
  "v_mfma_f32_16x16x32_bf16 v[188:191], a[248:251], v[104:107], v[188:191]\n\t" \
  "ds_read_b64_tr_b16 a[142:143], v13 offset:9216\n\t" \
  "v_mfma_f32_16x16x32_bf16 v[188:191], a[252:255], v[108:111], v[188:191]\n\t" \
  "ds_read_b64_tr_b16 a[144:145], v14 offset:0\n\t" \
  "v_mfma_f32_16x16x32_bf16 v[184:187], a[244:247], v[108:111], v[184:187]\n\t" \
  "ds_read_b64_tr_b16 a[146:147], v14 offset:1024\n\t" \
  "v_mfma_f32_16x16x32_bf16 v[184:187], a[240:243], v[104:107], v[184:187]\n\t" \
  "ds_read_b64_tr_b16 a[148:149], v14 offset:8192\n\t" \
  "v_mfma_f32_16x16x32_bf16 v[180:183], a[232:235], v[104:107], v[180:183]\n\t" \
  "ds_read_b64_tr_b16 a[150:151], v14 offset:9216\n\t" \
  "v_mfma_f32_16x16x32_bf16 v[180:183], a[236:239], v[108:111], v[180:183]\n\t" \
  "ds_read_b64_tr_b16 a[152:153], v15 offset:0\n\t" \
  "v_mfma_f32_16x16x32_bf16 v[176:179], a[228:231], v[108:111], v[176:179]\n\t" \
  "ds_read_b64_tr_b16 a[154:155], v15 offset:1024\n\t" \
  "v_mfma_f32_16x16x32_bf16 v[176:179], a[224:227], v[104:107], v[176:179]\n\t" \
  "ds_read_b64_tr_b16 a[156:157], v15 offset:8192\n\t" \
  "v_mfma_f32_16x16x32_bf16 v[172:175], a[216:219], v[104:107], v[172:175]\n\t" \
  "ds_read_b64_tr_b16 a[158:159], v15 offset:9216\n\t" \
  "v_mfma_f32_16x16x32_bf16 v[172:175], a[220:223], v[108:111], v[172:175]\n\t" \
  "ds_read_b64_tr_b16 a[160:161], v12 offset:16384\n\t" \
  "v_mfma_f32_16x16x32_bf16 v[168:171], a[212:215], v[108:111], v[168:171]\n\t" \
  "ds_read_b64_tr_b16 a[162:163], v12 offset:17408\n\t" \
  "v_mfma_f32_16x16x32_bf16 v[168:171], a[208:211], v[104:107], v[168:171]\n\t" \
  "ds_read_b64_tr_b16 a[164:165], v12 offset:24576\n\t" \
  "v_mfma_f32_16x16x32_bf16 v[164:167], a[200:203], v[104:107], v[164:167]\n\t" \
  "ds_read_b64_tr_b16 a[166:167], v12 offset:25600\n\t" \
  "v_mfma_f32_16x16x32_bf16 v[164:167], a[204:207], v[108:111], v[164:167]\n\t" \
  "ds_read_b64_tr_b16 a[168:169], v13 offset:16384\n\t" \
  "v_mfma_f32_16x16x32_bf16 v[160:163], a[196:199], v[108:111], v[160:163]\n\t" \
  "ds_read_b64_tr_b16 a[170:171], v13 offset:17408\n\t" \
  "v_mfma_f32_16x16x32_bf16 v[160:163], a[192:195], v[104:107], v[160:163]\n\t" \
  "ds_read_b128 v[104:107], v10 offset:2048\n\t" \
  "ds_read_b128 v[108:111], v11 offset:2048\n\t" \
  "v_mfma_f32_16x16x32_bf16 v[192:195], a[192:195], v[112:115], v[192:195]\n\t" \
  "ds_read_b64_tr_b16 a[172:173], v13 offset:24576\n\t" \
  "v_mfma_f32_16x16x32_bf16 v[192:195], a[196:199], v[116:119], v[192:195]\n\t" \
  "ds_read_b64_tr_b16 a[174:175], v13 offset:25600\n\t" \
  "v_mfma_f32_16x16x32_bf16 v[196:199], a[204:207], v[116:119], v[196:199]\n\t" \
  "ds_read_b64_tr_b16 a[176:177], v14 offset:16384\n\t" \
  "v_mfma_f32_16x16x32_bf16 v[196:199], a[200:203], v[112:115], v[196:199]\n\t" \
  "ds_read_b64_tr_b16 a[178:179], v14 offset:17408\n\t" \
  "v_mfma_f32_16x16x32_bf16 v[200:203], a[208:211], v[112:115], v[200:203]\n\t" \
  "ds_read_b64_tr_b16 a[180:181], v14 offset:24576\n\t" \
  "v_mfma_f32_16x16x32_bf16 v[200:203], a[212:215], v[116:119], v[200:203]\n\t" \
  "ds_read_b64_tr_b16 a[182:183], v14 offset:25600\n\t" \
  "v_mfma_f32_16x16x32_bf16 v[204:207], a[220:223], v[116:119], v[204:207]\n\t" \
  "ds_read_b64_tr_b16 a[184:185], v15 offset:16384\n\t" \
  "v_mfma_f32_16x16x32_bf16 v[204:207], a[216:219], v[112:115], v[204:207]\n\t" \
  "ds_read_b64_tr_b16 a[186:187], v15 offset:17408\n\t" \
  "s_add_u32 m0, s68, 0\n\t" \
  "v_mfma_f32_16x16x32_bf16 v[208:211], a[224:227], v[112:115], v[208:211]\n\t" \
  "ds_read_b64_tr_b16 a[188:189], v15 offset:24576\n\t" \
  "buffer_load_dwordx4 v0, s[36:39], s65 offen offset:0 sc0 sc1 lds\n\t" \
  "v_mfma_f32_16x16x32_bf16 v[208:211], a[228:231], v[116:119], v[208:211]\n\t" \
  "ds_read_b64_tr_b16 a[190:191], v15 offset:25600\n\t" \
  "v_mfma_f32_16x16x32_bf16 v[212:215], a[236:239], v[116:119], v[212:215]\n\t" \
  "v_mfma_f32_16x16x32_bf16 v[212:215], a[232:235], v[112:115], v[212:215]\n\t" \
  "buffer_load_dwordx4 v1, s[36:39], s65 offen offset:1024 sc0 sc1 lds\n\t" \
  "v_mfma_f32_16x16x32_bf16 v[216:219], a[240:243], v[112:115], v[216:219]\n\t" \
  "v_mfma_f32_16x16x32_bf16 v[216:219], a[244:247], v[116:119], v[216:219]\n\t" \
  "v_mfma_f32_16x16x32_bf16 v[220:223], a[252:255], v[116:119], v[220:223]\n\t" \
  "buffer_load_dwordx4 v2, s[36:39], s65 offen offset:2048 sc0 sc1 lds\n\t" \
  "v_mfma_f32_16x16x32_bf16 v[220:223], a[248:251], v[112:115], v[220:223]\n\t" \
  "ds_read_b128 v[112:115], v10 offset:4096\n\t" \
  "ds_read_b128 v[116:119], v11 offset:4096\n\t" \
  "v_mfma_f32_16x16x32_bf16 v[252:255], a[248:251], v[120:123], v[252:255]\n\t" \
  "v_mfma_f32_16x16x32_bf16 v[252:255], a[252:255], v[124:127], v[252:255]\n\t" \
  "buffer_load_dwordx4 v3, s[36:39], s65 offen offset:3072 sc0 sc1 lds\n\t" \
  "v_mfma_f32_16x16x32_bf16 v[248:251], a[244:247], v[124:127], v[248:251]\n\t" \
  "v_mfma_f32_16x16x32_bf16 v[248:251], a[240:243], v[120:123], v[248:251]\n\t" \
  "s_add_u32 m0, s68, 65536\n\t" \
  "v_mfma_f32_16x16x32_bf16 v[244:247], a[232:235], v[120:123], v[244:247]\n\t" \
  "buffer_load_dwordx4 v4, s[40:43], s78 offen offset:0 lds\n\t" \
  "v_mfma_f32_16x16x32_bf16 v[244:247], a[236:239], v[124:127], v[244:247]\n\t" \
  "v_mfma_f32_16x16x32_bf16 v[240:243], a[228:231], v[124:127], v[240:243]\n\t" \
  "v_mfma_f32_16x16x32_bf16 v[240:243], a[224:227], v[120:123], v[240:243]\n\t" \
  "buffer_load_dwordx4 v5, s[40:43], s78 offen offset:1024 lds\n\t" \
  "v_mfma_f32_16x16x32_bf16 v[236:239], a[216:219], v[120:123], v[236:239]\n\t" \
  "v_mfma_f32_16x16x32_bf16 v[236:239], a[220:223], v[124:127], v[236:239]\n\t" \
  "v_mfma_f32_16x16x32_bf16 v[232:235], a[212:215], v[124:127], v[232:235]\n\t" \
  "buffer_load_dwordx4 v6, s[40:43], s78 offen offset:2048 lds\n\t" \
  "v_mfma_f32_16x16x32_bf16 v[232:235], a[208:211], v[120:123], v[232:235]\n\t" \
  "v_mfma_f32_16x16x32_bf16 v[228:231], a[200:203], v[120:123], v[228:231]\n\t" \
  "v_mfma_f32_16x16x32_bf16 v[228:231], a[204:207], v[124:127], v[228:231]\n\t" \
  "v_mfma_f32_16x16x32_bf16 v[224:227], a[196:199], v[124:127], v[224:227]\n\t" \
  "buffer_load_dwordx4 v7, s[40:43], s78 offen offset:3072 lds\n\t" \
  "v_mfma_f32_16x16x32_bf16 v[224:227], a[192:195], v[120:123], v[224:227]\n\t" \
  "ds_read_b128 v[120:123], v10 offset:6144\n\t" \
  "ds_read_b128 v[124:127], v11 offset:6144\n\t" \
  "s_waitcnt lgkmcnt(4)\n\t" \
  "v_mfma_f32_16x16x32_bf16 v[128:131], a[128:131], v[96:99], v[128:131]\n\t" \
  "v_mfma_f32_16x16x32_bf16 v[128:131], a[132:135], v[100:103], v[128:131]\n\t" \
  "v_add_u32 v10, 4294950912, v10\n\t" \
  "v_mfma_f32_16x16x32_bf16 v[132:135], a[140:143], v[100:103], v[132:135]\n\t" \
  "v_add_u32 v11, 4294950912, v11\n\t" \
  "v_mfma_f32_16x16x32_bf16 v[132:135], a[136:139], v[96:99], v[132:135]\n\t" \
  "v_add_u32 v12, 4294885376, v12\n\t" \
  "v_mfma_f32_16x16x32_bf16 v[136:139], a[144:147], v[96:99], v[136:139]\n\t" \
  "v_add_u32 v13, 4294885376, v13\n\t" \
  "v_mfma_f32_16x16x32_bf16 v[136:139], a[148:151], v[100:103], v[136:139]\n\t" \
  "v_add_u32 v14, 4294885376, v14\n\t" \
  "v_mfma_f32_16x16x32_bf16 v[140:143], a[156:159], v[100:103], v[140:143]\n\t" \
  "v_add_u32 v15, 4294885376, v15\n\t" \
  "v_mfma_f32_16x16x32_bf16 v[140:143], a[152:155], v[96:99], v[140:143]\n\t" \
  "s_waitcnt vmcnt(12) lgkmcnt(0)\n\t" \
  "s_barrier\n\t" \
  "v_readfirstlane_b32 s99, v49\n\t" \
  "s_nop 1\n\t" \
  "s_cmp_ge_u32 s99, s85\n\t" \
  "s_cbranch_scc0 SLOW242_%=\n\t" \
  "RET241_%=:\n\t" \
  "v_mfma_f32_16x16x32_bf16 v[144:147], a[160:163], v[96:99], v[144:147]\n\t" \
  "ds_read_b64_tr_b16 a[192:193], v12 offset:0\n\t" \
  "v_mfma_f32_16x16x32_bf16 v[144:147], a[164:167], v[100:103], v[144:147]\n\t" \
  "ds_read_b64_tr_b16 a[194:195], v12 offset:1024\n\t" \
  "global_load_dword v48, v30, s[50:51] offset:-128 sc1\n\t" \
  "v_mfma_f32_16x16x32_bf16 v[148:151], a[172:175], v[100:103], v[148:151]\n\t" \
  "ds_read_b64_tr_b16 a[196:197], v12 offset:8192\n\t" \
  "v_mfma_f32_16x16x32_bf16 v[148:151], a[168:171], v[96:99], v[148:151]\n\t" \
  "ds_read_b64_tr_b16 a[198:199], v12 offset:9216\n\t" \
  "v_mfma_f32_16x16x32_bf16 v[152:155], a[176:179], v[96:99], v[152:155]\n\t" \
  "ds_read_b64_tr_b16 a[200:201], v13 offset:0\n\t" \
  "v_mfma_f32_16x16x32_bf16 v[152:155], a[180:183], v[100:103], v[152:155]\n\t" \
  "ds_read_b64_tr_b16 a[202:203], v13 offset:1024\n\t" \
  "v_mfma_f32_16x16x32_bf16 v[156:159], a[188:191], v[100:103], v[156:159]\n\t" \
  "ds_read_b64_tr_b16 a[204:205], v13 offset:8192\n\t" \
  "v_add_u32 v9, 4294959104, v9\n\t" \
  "v_xor_b32 v9, 0x10000, v9\n\t" \
  "global_load_dwordx4 a[0:3], v9, s[50:51] offset:-4096 sc0 sc1\n\t" \
  "v_mfma_f32_16x16x32_bf16 v[156:159], a[184:187], v[96:99], v[156:159]\n\t" \
  "ds_read_b128 v[96:99], v10 offset:0\n\t" \
  "ds_read_b128 v[100:103], v11 offset:0\n\t" \
  "v_mfma_f32_16x16x32_bf16 v[188:191], a[184:187], v[104:107], v[188:191]\n\t" \
  "ds_read_b64_tr_b16 a[206:207], v13 offset:9216\n\t" \
  "v_mfma_f32_16x16x32_bf16 v[188:191], a[188:191], v[108:111], v[188:191]\n\t" \
  "ds_read_b64_tr_b16 a[208:209], v14 offset:0\n\t" \
  "global_load_dwordx4 a[4:7], v9, s[50:51] offset:-3072 sc0 sc1\n\t" \
  "v_mfma_f32_16x16x32_bf16 v[184:187], a[180:183], v[108:111], v[184:187]\n\t" \
  "ds_read_b64_tr_b16 a[210:211], v14 offset:1024\n\t" \
  "v_mfma_f32_16x16x32_bf16 v[184:187], a[176:179], v[104:107], v[184:187]\n\t" \
  "ds_read_b64_tr_b16 a[212:213], v14 offset:8192\n\t" \
  "v_mfma_f32_16x16x32_bf16 v[180:183], a[168:171], v[104:107], v[180:183]\n\t" \
  "ds_read_b64_tr_b16 a[214:215], v14 offset:9216\n\t" \
  "global_load_dwordx4 a[8:11], v9, s[50:51] offset:-2048 sc0 sc1\n\t" \
  "v_mfma_f32_16x16x32_bf16 v[180:183], a[172:175], v[108:111], v[180:183]\n\t" \
  "ds_read_b64_tr_b16 a[216:217], v15 offset:0\n\t" \
  "v_mfma_f32_16x16x32_bf16 v[176:179], a[164:167], v[108:111], v[176:179]\n\t" \
  "ds_read_b64_tr_b16 a[218:219], v15 offset:1024\n\t" \
  "v_mfma_f32_16x16x32_bf16 v[176:179], a[160:163], v[104:107], v[176:179]\n\t" \
  "ds_read_b64_tr_b16 a[220:221], v15 offset:8192\n\t" \
  "global_load_dwordx4 a[12:15], v9, s[50:51] offset:-1024 sc0 sc1\n\t" \
  "v_mfma_f32_16x16x32_bf16 v[172:175], a[152:155], v[104:107], v[172:175]\n\t" \
  "ds_read_b64_tr_b16 a[222:223], v15 offset:9216\n\t" \
  "v_mfma_f32_16x16x32_bf16 v[172:175], a[156:159], v[108:111], v[172:175]\n\t" \
  "ds_read_b64_tr_b16 a[224:225], v12 offset:32768\n\t" \
  "v_mfma_f32_16x16x32_bf16 v[168:171], a[148:151], v[108:111], v[168:171]\n\t" \
  "ds_read_b64_tr_b16 a[226:227], v12 offset:33792\n\t" \
  "global_load_dwordx4 a[16:19], v9, s[50:51] offset:0 sc0 sc1\n\t" \
  "v_mfma_f32_16x16x32_bf16 v[168:171], a[144:147], v[104:107], v[168:171]\n\t" \
  "ds_read_b64_tr_b16 a[228:229], v12 offset:40960\n\t" \
  "v_mfma_f32_16x16x32_bf16 v[164:167], a[136:139], v[104:107], v[164:167]\n\t" \
  "ds_read_b64_tr_b16 a[230:231], v12 offset:41984\n\t" \
  "v_mfma_f32_16x16x32_bf16 v[164:167], a[140:143], v[108:111], v[164:167]\n\t" \
  "ds_read_b64_tr_b16 a[232:233], v13 offset:32768\n\t" \
  "global_load_dwordx4 a[20:23], v9, s[50:51] offset:1024 sc0 sc1\n\t" \
  "v_mfma_f32_16x16x32_bf16 v[160:163], a[132:135], v[108:111], v[160:163]\n\t" \
  "ds_read_b64_tr_b16 a[234:235], v13 offset:33792\n\t" \
  "v_mfma_f32_16x16x32_bf16 v[160:163], a[128:131], v[104:107], v[160:163]\n\t" \
  "ds_read_b128 v[104:107], v10 offset:2048\n\t" \
  "ds_read_b128 v[108:111], v11 offset:2048\n\t" \
  "v_mfma_f32_16x16x32_bf16 v[192:195], a[128:131], v[112:115], v[192:195]\n\t" \
  "ds_read_b64_tr_b16 a[236:237], v13 offset:40960\n\t" \
  "global_load_dwordx4 a[24:27], v9, s[50:51] offset:2048 sc0 sc1\n\t" \
  "v_mfma_f32_16x16x32_bf16 v[192:195], a[132:135], v[116:119], v[192:195]\n\t" \
  "ds_read_b64_tr_b16 a[238:239], v13 offset:41984\n\t" \
  "v_mfma_f32_16x16x32_bf16 v[196:199], a[140:143], v[116:119], v[196:199]\n\t" \
  "ds_read_b64_tr_b16 a[240:241], v14 offset:32768\n\t" \
  "v_mfma_f32_16x16x32_bf16 v[196:199], a[136:139], v[112:115], v[196:199]\n\t" \
  "ds_read_b64_tr_b16 a[242:243], v14 offset:33792\n\t" \
  "global_load_dwordx4 a[28:31], v9, s[50:51] offset:3072 sc0 sc1\n\t" \
  "v_mfma_f32_16x16x32_bf16 v[200:203], a[144:147], v[112:115], v[200:203]\n\t" \
  "ds_read_b64_tr_b16 a[244:245], v14 offset:40960\n\t" \
  "v_mfma_f32_16x16x32_bf16 v[200:203], a[148:151], v[116:119], v[200:203]\n\t" \
  "ds_read_b64_tr_b16 a[246:247], v14 offset:41984\n\t" \
  "v_mfma_f32_16x16x32_bf16 v[204:207], a[156:159], v[116:119], v[204:207]\n\t" \
  "ds_read_b64_tr_b16 a[248:249], v15 offset:32768\n\t" \
  "v_add_u32 v9, 8192, v9\n\t" \
  "global_load_dwordx4 a[32:35], v9, s[50:51] offset:-4096 sc0 sc1\n\t" \
  "v_mfma_f32_16x16x32_bf16 v[204:207], a[152:155], v[112:115], v[204:207]\n\t" \
  "ds_read_b64_tr_b16 a[250:251], v15 offset:33792\n\t" \
  "v_mfma_f32_16x16x32_bf16 v[208:211], a[160:163], v[112:115], v[208:211]\n\t" \
  "ds_read_b64_tr_b16 a[252:253], v15 offset:40960\n\t" \
  "v_mfma_f32_16x16x32_bf16 v[208:211], a[164:167], v[116:119], v[208:211]\n\t" \
  "ds_read_b64_tr_b16 a[254:255], v15 offset:41984\n\t" \
  "global_load_dwordx4 a[36:39], v9, s[50:51] offset:-3072 sc0 sc1\n\t" \
  "v_mfma_f32_16x16x32_bf16 v[212:215], a[172:175], v[116:119], v[212:215]\n\t" \
  "v_mfma_f32_16x16x32_bf16 v[212:215], a[168:171], v[112:115], v[212:215]\n\t" \
  "v_mfma_f32_16x16x32_bf16 v[216:219], a[176:179], v[112:115], v[216:219]\n\t" \
  "global_load_dwordx4 a[40:43], v9, s[50:51] offset:-2048 sc0 sc1\n\t" \
  "v_mfma_f32_16x16x32_bf16 v[216:219], a[180:183], v[116:119], v[216:219]\n\t" \
  "v_mfma_f32_16x16x32_bf16 v[220:223], a[188:191], v[116:119], v[220:223]\n\t" \
  "v_mfma_f32_16x16x32_bf16 v[220:223], a[184:187], v[112:115], v[220:223]\n\t" \
  "ds_read_b128 v[112:115], v10 offset:4096\n\t" \
  "ds_read_b128 v[116:119], v11 offset:4096\n\t" \
  "global_load_dwordx4 a[44:47], v9, s[50:51] offset:-1024 sc0 sc1\n\t" \
  "v_mfma_f32_16x16x32_bf16 v[252:255], a[184:187], v[120:123], v[252:255]\n\t" \
  "v_mfma_f32_16x16x32_bf16 v[252:255], a[188:191], v[124:127], v[252:255]\n\t" \
  "v_mfma_f32_16x16x32_bf16 v[248:251], a[180:183], v[124:127], v[248:251]\n\t" \
  "global_load_dwordx4 a[48:51], v9, s[50:51] offset:0 sc0 sc1\n\t" \
  "v_mfma_f32_16x16x32_bf16 v[248:251], a[176:179], v[120:123], v[248:251]\n\t" \
  "v_mfma_f32_16x16x32_bf16 v[244:247], a[168:171], v[120:123], v[244:247]\n\t" \
  "v_mfma_f32_16x16x32_bf16 v[244:247], a[172:175], v[124:127], v[244:247]\n\t" \
  "global_load_dwordx4 a[52:55], v9, s[50:51] offset:1024 sc0 sc1\n\t" \
  "v_mfma_f32_16x16x32_bf16 v[240:243], a[164:167], v[124:127], v[240:243]\n\t" \
  "v_mfma_f32_16x16x32_bf16 v[240:243], a[160:163], v[120:123], v[240:243]\n\t" \
  "v_mfma_f32_16x16x32_bf16 v[236:239], a[152:155], v[120:123], v[236:239]\n\t" \
  "global_load_dwordx4 a[56:59], v9, s[50:51] offset:2048 sc0 sc1\n\t" \
  "v_mfma_f32_16x16x32_bf16 v[236:239], a[156:159], v[124:127], v[236:239]\n\t" \
  "v_mfma_f32_16x16x32_bf16 v[232:235], a[148:151], v[124:127], v[232:235]\n\t" \
  "v_mfma_f32_16x16x32_bf16 v[232:235], a[144:147], v[120:123], v[232:235]\n\t" \
  "global_load_dwordx4 a[60:63], v9, s[50:51] offset:3072 sc0 sc1\n\t" \
  "v_mfma_f32_16x16x32_bf16 v[228:231], a[136:139], v[120:123], v[228:231]\n\t" \
  "v_mfma_f32_16x16x32_bf16 v[228:231], a[140:143], v[124:127], v[228:231]\n\t" \
  "v_mfma_f32_16x16x32_bf16 v[224:227], a[132:135], v[124:127], v[224:227]\n\t" \
  "v_mfma_f32_16x16x32_bf16 v[224:227], a[128:131], v[120:123], v[224:227]\n\t" \
  "ds_read_b128 v[120:123], v10 offset:6144\n\t" \
  "ds_read_b128 v[124:127], v11 offset:6144\n\t" \
  "s_waitcnt lgkmcnt(4)\n\t" \
  "v_mfma_f32_16x16x32_bf16 v[128:131], a[192:195], v[96:99], v[128:131]\n\t" \
  "v_mfma_f32_16x16x32_bf16 v[128:131], a[196:199], v[100:103], v[128:131]\n\t" \
  "v_add_u32 v10, 4294950912, v10\n\t" \
  "v_mfma_f32_16x16x32_bf16 v[132:135], a[204:207], v[100:103], v[132:135]\n\t" \
  "v_add_u32 v11, 4294950912, v11\n\t" \
  "v_mfma_f32_16x16x32_bf16 v[132:135], a[200:203], v[96:99], v[132:135]\n\t" \
  "v_add_u32 v12, 16384, v12\n\t" \
  "v_mfma_f32_16x16x32_bf16 v[136:139], a[208:211], v[96:99], v[136:139]\n\t" \
  "v_add_u32 v13, 16384, v13\n\t" \
  "v_mfma_f32_16x16x32_bf16 v[136:139], a[212:215], v[100:103], v[136:139]\n\t" \
  "v_add_u32 v14, 16384, v14\n\t" \
  "v_mfma_f32_16x16x32_bf16 v[140:143], a[220:223], v[100:103], v[140:143]\n\t" \
  "v_add_u32 v15, 16384, v15\n\t" \
  "v_mfma_f32_16x16x32_bf16 v[140:143], a[216:219], v[96:99], v[140:143]\n\t" \
  "s_waitcnt vmcnt(17) lgkmcnt(0)\n\t" \
  "s_barrier\n\t" \
  "v_mfma_f32_16x16x32_bf16 v[144:147], a[224:227], v[96:99], v[144:147]\n\t" \
  "ds_read_b128 v[64:67], v10 offset:0\n\t" \
  "ds_read_b128 v[68:71], v11 offset:0\n\t" \
  "v_mfma_f32_16x16x32_bf16 v[144:147], a[228:231], v[100:103], v[144:147]\n\t" \
  "ds_read_b64_tr_b16 a[128:129], v12 offset:0\n\t" \
  "ds_read_b64_tr_b16 a[130:131], v12 offset:1024\n\t" \
  "v_mfma_f32_16x16x32_bf16 v[148:151], a[236:239], v[100:103], v[148:151]\n\t" \
  "ds_read_b64_tr_b16 a[132:133], v12 offset:8192\n\t" \
  "ds_read_b64_tr_b16 a[134:135], v12 offset:9216\n\t" \
  "v_mfma_f32_16x16x32_bf16 v[148:151], a[232:235], v[96:99], v[148:151]\n\t" \
  "ds_read_b64_tr_b16 a[136:137], v13 offset:0\n\t" \
  "ds_read_b64_tr_b16 a[138:139], v13 offset:1024\n\t" \
  "v_mfma_f32_16x16x32_bf16 v[152:155], a[240:243], v[96:99], v[152:155]\n\t" \
  "ds_read_b64_tr_b16 a[140:141], v13 offset:8192\n\t" \
  "ds_read_b64_tr_b16 a[142:143], v13 offset:9216\n\t" \
  "v_mfma_f32_16x16x32_bf16 v[152:155], a[244:247], v[100:103], v[152:155]\n\t" \
  "ds_read_b64_tr_b16 a[144:145], v14 offset:0\n\t" \
  "ds_read_b64_tr_b16 a[146:147], v14 offset:1024\n\t" \
  "v_mfma_f32_16x16x32_bf16 v[156:159], a[252:255], v[100:103], v[156:159]\n\t" \
  "ds_read_b64_tr_b16 a[148:149], v14 offset:8192\n\t" \
  "ds_read_b64_tr_b16 a[150:151], v14 offset:9216\n\t" \
  "v_mfma_f32_16x16x32_bf16 v[156:159], a[248:251], v[96:99], v[156:159]\n\t" \
  "ds_read_b64_tr_b16 a[152:153], v15 offset:0\n\t" \
  "ds_read_b64_tr_b16 a[154:155], v15 offset:1024\n\t" \
  "v_mfma_f32_16x16x32_bf16 v[188:191], a[248:251], v[104:107], v[188:191]\n\t" \
  "ds_read_b64_tr_b16 a[156:157], v15 offset:8192\n\t" \
  "ds_read_b64_tr_b16 a[158:159], v15 offset:9216\n\t" \
  "v_mfma_f32_16x16x32_bf16 v[188:191], a[252:255], v[108:111], v[188:191]\n\t" \
  "ds_read_b64_tr_b16 a[160:161], v12 offset:32768\n\t" \
  "ds_read_b64_tr_b16 a[162:163], v12 offset:33792\n\t" \
  "v_mfma_f32_16x16x32_bf16 v[184:187], a[244:247], v[108:111], v[184:187]\n\t" \
  "ds_read_b64_tr_b16 a[164:165], v12 offset:40960\n\t" \
  "ds_read_b64_tr_b16 a[166:167], v12 offset:41984\n\t" \
  "v_mfma_f32_16x16x32_bf16 v[184:187], a[240:243], v[104:107], v[184:187]\n\t" \
  "ds_read_b64_tr_b16 a[168:169], v13 offset:32768\n\t" \
  "ds_read_b64_tr_b16 a[170:171], v13 offset:33792\n\t" \
  "v_mfma_f32_16x16x32_bf16 v[180:183], a[232:235], v[104:107], v[180:183]\n\t" \
  "ds_read_b64_tr_b16 a[172:173], v13 offset:40960\n\t" \
  "ds_read_b64_tr_b16 a[174:175], v13 offset:41984\n\t" \
  "v_mfma_f32_16x16x32_bf16 v[180:183], a[236:239], v[108:111], v[180:183]\n\t" \
  "ds_read_b64_tr_b16 a[176:177], v14 offset:32768\n\t" \
  "ds_read_b64_tr_b16 a[178:179], v14 offset:33792\n\t" \
  "v_mfma_f32_16x16x32_bf16 v[176:179], a[228:231], v[108:111], v[176:179]\n\t" \
  "ds_read_b64_tr_b16 a[180:181], v14 offset:40960\n\t" \
  "ds_read_b64_tr_b16 a[182:183], v14 offset:41984\n\t" \
  "v_mfma_f32_16x16x32_bf16 v[176:179], a[224:227], v[104:107], v[176:179]\n\t" \
  "ds_read_b64_tr_b16 a[184:185], v15 offset:32768\n\t" \
  "ds_read_b64_tr_b16 a[186:187], v15 offset:33792\n\t" \
  "v_mfma_f32_16x16x32_bf16 v[172:175], a[216:219], v[104:107], v[172:175]\n\t" \
  "ds_read_b64_tr_b16 a[188:189], v15 offset:40960\n\t" \
  "ds_read_b64_tr_b16 a[190:191], v15 offset:41984\n\t" \
  "v_mfma_f32_16x16x32_bf16 v[172:175], a[220:223], v[108:111], v[172:175]\n\t" \
  "ds_read_b128 v[72:75], v10 offset:2048\n\t" \
  "ds_read_b128 v[76:79], v11 offset:2048\n\t" \
  "v_mfma_f32_16x16x32_bf16 v[168:171], a[212:215], v[108:111], v[168:171]\n\t" \
  "ds_read_b128 v[80:83], v10 offset:4096\n\t" \
  "ds_read_b128 v[84:87], v11 offset:4096\n\t" \
  "v_mfma_f32_16x16x32_bf16 v[168:171], a[208:211], v[104:107], v[168:171]\n\t" \
  "ds_read_b128 v[88:91], v10 offset:6144\n\t" \
  "ds_read_b128 v[92:95], v11 offset:6144\n\t" \
  "v_mfma_f32_16x16x32_bf16 v[164:167], a[200:203], v[104:107], v[164:167]\n\t" \
  "v_mfma_f32_16x16x32_bf16 v[164:167], a[204:207], v[108:111], v[164:167]\n\t" \
  "v_mfma_f32_16x16x32_bf16 v[160:163], a[196:199], v[108:111], v[160:163]\n\t" \
  "v_mfma_f32_16x16x32_bf16 v[160:163], a[192:195], v[104:107], v[160:163]\n\t" \
  "s_waitcnt lgkmcnt(6)\n\t" \
  "v_mfma_f32_16x16x32_bf16 v[128:131], a[128:131], v[64:67], v[128:131]\n\t" \
  "s_waitcnt vmcnt(0)\n\t" \
  "v_mfma_f32_16x16x32_bf16 v[128:131], a[132:135], v[68:71], v[128:131]\n\t" \
  "v_add_u32 v27, 16384, v27\n\t" \
  "v_mfma_f32_16x16x32_bf16 v[128:131], v[42:45], a[0:3], v[128:131]\n\t" \
  "v_add_u32 v28, 16384, v28\n\t" \
  "v_mfma_f32_16x16x32_bf16 v[132:135], a[140:143], v[68:71], v[132:135]\n\t" \
  "v_xor_b32 v0, 0, v27\n\t" \
  "v_mfma_f32_16x16x32_bf16 v[132:135], a[136:139], v[64:67], v[132:135]\n\t" \
  "v_xor_b32 v1, 32, v27\n\t" \
  "v_mfma_f32_16x16x32_bf16 v[132:135], v[40:43], a[0:3], v[132:135]\n\t" \
  "v_xor_b32 v2, 64, v27\n\t" \
  "v_mfma_f32_16x16x32_bf16 v[136:139], a[144:147], v[64:67], v[136:139]\n\t" \
  "v_xor_b32 v3, 96, v27\n\t" \
  "v_mfma_f32_16x16x32_bf16 v[136:139], a[148:151], v[68:71], v[136:139]\n\t" \
  "v_xor_b32 v4, 128, v27\n\t" \
  "v_mfma_f32_16x16x32_bf16 v[136:139], v[42:45], a[4:7], v[136:139]\n\t" \
  "v_xor_b32 v5, 160, v27\n\t" \
  "v_mfma_f32_16x16x32_bf16 v[140:143], a[156:159], v[68:71], v[140:143]\n\t" \
  "v_xor_b32 v6, 192, v27\n\t" \
  "v_mfma_f32_16x16x32_bf16 v[140:143], a[152:155], v[64:67], v[140:143]\n\t" \
  "v_xor_b32 v7, 224, v27\n\t" \
  "v_mfma_f32_16x16x32_bf16 v[140:143], v[40:43], a[4:7], v[140:143]\n\t" \
  "v_xor_b32 v20, 0, v28\n\t" \
  "v_mfma_f32_16x16x32_bf16 v[144:147], a[160:163], v[64:67], v[144:147]\n\t" \
  "v_xor_b32 v21, 64, v28\n\t" \
  "v_mfma_f32_16x16x32_bf16 v[144:147], a[164:167], v[68:71], v[144:147]\n\t" \
  "v_xor_b32 v22, 128, v28\n\t" \
  "v_mfma_f32_16x16x32_bf16 v[144:147], v[42:45], a[8:11], v[144:147]\n\t" \
  "v_xor_b32 v23, 192, v28\n\t" \
  "v_mfma_f32_16x16x32_bf16 v[148:151], a[172:175], v[68:71], v[148:151]\n\t" \
  "v_mfma_f32_16x16x32_bf16 v[148:151], a[168:171], v[64:67], v[148:151]\n\t" \
  "v_readfirstlane_b32 s99, v48\n\t" \
  "v_mfma_f32_16x16x32_bf16 v[148:151], v[40:43], a[8:11], v[148:151]\n\t" \
  "v_mfma_f32_16x16x32_bf16 v[152:155], a[176:179], v[64:67], v[152:155]\n\t" \
  "s_waitcnt lgkmcnt(0)\n\t" \
  "v_mfma_f32_16x16x32_bf16 v[152:155], a[180:183], v[68:71], v[152:155]\n\t" \
  "v_mfma_f32_16x16x32_bf16 v[152:155], v[42:45], a[12:15], v[152:155]\n\t" \
  "s_cmp_eq_u32 s99, s85\n\t" \
  "v_mfma_f32_16x16x32_bf16 v[156:159], a[188:191], v[68:71], v[156:159]\n\t" \
  "v_mfma_f32_16x16x32_bf16 v[156:159], a[184:187], v[64:67], v[156:159]\n\t" \
  "s_cbranch_scc1 NANP_%=\n\t" \
  "v_mfma_f32_16x16x32_bf16 v[156:159], v[40:43], a[12:15], v[156:159]\n\t" \
  "v_mfma_f32_16x16x32_bf16 v[188:191], a[184:187], v[72:75], v[188:191]\n\t" \
  "v_mfma_f32_16x16x32_bf16 v[188:191], a[188:191], v[76:79], v[188:191]\n\t" \
  "v_mfma_f32_16x16x32_bf16 v[188:191], v[40:43], a[28:31], v[188:191]\n\t" \
  "v_mfma_f32_16x16x32_bf16 v[184:187], a[180:183], v[76:79], v[184:187]\n\t" \
  "v_cvt_pk_bf16_f32 v60, v128, v129\n\t" \
  "v_cvt_pk_bf16_f32 v61, v130, v131\n\t" \
  "v_mfma_f32_16x16x32_bf16 v[184:187], a[176:179], v[72:75], v[184:187]\n\t" \
  "ds_write_b64 v0, v[60:61]\n\t" \
  "v_cvt_pk_bf16_f32 v62, v132, v133\n\t" \
  "v_mfma_f32_16x16x32_bf16 v[184:187], v[42:45], a[28:31], v[184:187]\n\t" \
  "v_cvt_pk_bf16_f32 v63, v134, v135\n\t" \
  "ds_write_b64 v1, v[62:63]\n\t" \
  "v_mfma_f32_16x16x32_bf16 v[180:183], a[168:171], v[72:75], v[180:183]\n\t" \
  "v_cvt_pk_bf16_f32 v60, v136, v137\n\t" \
  "v_cvt_pk_bf16_f32 v61, v138, v139\n\t" \
  "v_mfma_f32_16x16x32_bf16 v[180:183], a[172:175], v[76:79], v[180:183]\n\t" \
  "ds_write_b64 v2, v[60:61]\n\t" \
  "v_cvt_pk_bf16_f32 v62, v140, v141\n\t" \
  "v_mfma_f32_16x16x32_bf16 v[180:183], v[40:43], a[24:27], v[180:183]\n\t" \
  "v_cvt_pk_bf16_f32 v63, v142, v143\n\t" \
  "ds_write_b64 v3, v[62:63]\n\t" \
  "v_mfma_f32_16x16x32_bf16 v[176:179], a[164:167], v[76:79], v[176:179]\n\t" \
  "v_cvt_pk_bf16_f32 v60, v144, v145\n\t" \
  "v_cvt_pk_bf16_f32 v61, v146, v147\n\t" \
  "v_mfma_f32_16x16x32_bf16 v[176:179], a[160:163], v[72:75], v[176:179]\n\t" \
  "ds_write_b64 v4, v[60:61]\n\t" \
  "v_cvt_pk_bf16_f32 v62, v148, v149\n\t" \
  "v_mfma_f32_16x16x32_bf16 v[176:179], v[42:45], a[24:27], v[176:179]\n\t" \
  "v_cvt_pk_bf16_f32 v63, v150, v151\n\t" \
  "ds_write_b64 v5, v[62:63]\n\t" \
  "v_mfma_f32_16x16x32_bf16 v[172:175], a[152:155], v[72:75], v[172:175]\n\t" \
  "v_cvt_pk_bf16_f32 v60, v152, v153\n\t" \
  "v_cvt_pk_bf16_f32 v61, v154, v155\n\t" \
  "v_mfma_f32_16x16x32_bf16 v[172:175], a[156:159], v[76:79], v[172:175]\n\t" \
  "ds_write_b64 v6, v[60:61]\n\t" \
  "v_cvt_pk_bf16_f32 v62, v156, v157\n\t" \
  "v_mfma_f32_16x16x32_bf16 v[172:175], v[40:43], a[20:23], v[172:175]\n\t" \
  "v_cvt_pk_bf16_f32 v63, v158, v159\n\t" \
  "ds_write_b64 v7, v[62:63]\n\t" \
  "v_mfma_f32_16x16x32_bf16 v[168:171], a[148:151], v[76:79], v[168:171]\n\t" \
  "ds_read_b128 v[128:131], v20 offset:0\n\t" \
  "ds_read_b128 v[132:135], v21 offset:1024\n\t" \
  "v_mfma_f32_16x16x32_bf16 v[168:171], a[144:147], v[72:75], v[168:171]\n\t" \
  "ds_read_b128 v[136:139], v22 offset:2048\n\t" \
  "ds_read_b128 v[140:143], v23 offset:3072\n\t" \
  "v_mfma_f32_16x16x32_bf16 v[168:171], v[42:45], a[20:23], v[168:171]\n\t" \
  "v_mfma_f32_16x16x32_bf16 v[164:167], a[136:139], v[72:75], v[164:167]\n\t" \
  "v_mfma_f32_16x16x32_bf16 v[164:167], a[140:143], v[76:79], v[164:167]\n\t" \
  "v_mfma_f32_16x16x32_bf16 v[164:167], v[40:43], a[16:19], v[164:167]\n\t" \
  "s_waitcnt lgkmcnt(0)\n\t" \
  "s_mov_b32 s98, 0\n\t" \
  "buffer_store_dwordx4 v[128:131], v29, s[44:47], s98 offen sc0 sc1\n\t" \
  "v_mfma_f32_16x16x32_bf16 v[160:163], a[132:135], v[76:79], v[160:163]\n\t" \
  "v_mfma_f32_16x16x32_bf16 v[160:163], a[128:131], v[72:75], v[160:163]\n\t" \
  "v_mfma_f32_16x16x32_bf16 v[160:163], v[42:45], a[16:19], v[160:163]\n\t" \
  "v_mfma_f32_16x16x32_bf16 v[192:195], a[192:195], v[112:115], v[192:195]\n\t" \
  "v_mfma_f32_16x16x32_bf16 v[192:195], a[196:199], v[116:119], v[192:195]\n\t" \
  "v_mfma_f32_16x16x32_bf16 v[192:195], a[132:135], v[84:87], v[192:195]\n\t" \
  "s_mov_b32 s98, 65536\n\t" \
  "buffer_store_dwordx4 v[132:135], v29, s[44:47], s98 offen sc0 sc1\n\t" \
  "v_mfma_f32_16x16x32_bf16 v[192:195], a[128:131], v[80:83], v[192:195]\n\t" \
  "v_cvt_pk_bf16_f32 v62, v188, v189\n\t" \
  "v_cvt_pk_bf16_f32 v63, v190, v191\n\t" \
  "v_mfma_f32_16x16x32_bf16 v[192:195], v[42:45], a[32:35], v[192:195]\n\t" \
  "ds_write_b64 v7, v[62:63]\n\t" \
  "v_cvt_pk_bf16_f32 v60, v184, v185\n\t" \
  "v_mfma_f32_16x16x32_bf16 v[196:199], a[136:139], v[80:83], v[196:199]\n\t" \
  "v_cvt_pk_bf16_f32 v61, v186, v187\n\t" \
  "ds_write_b64 v6, v[60:61]\n\t" \
  "v_mfma_f32_16x16x32_bf16 v[196:199], a[140:143], v[84:87], v[196:199]\n\t" \
  "v_cvt_pk_bf16_f32 v62, v180, v181\n\t" \
  "v_cvt_pk_bf16_f32 v63, v182, v183\n\t" \
  "v_mfma_f32_16x16x32_bf16 v[196:199], a[204:207], v[116:119], v[196:199]\n\t" \
  "ds_write_b64 v5, v[62:63]\n\t" \
  "v_cvt_pk_bf16_f32 v60, v176, v177\n\t" \
  "v_mfma_f32_16x16x32_bf16 v[196:199], a[200:203], v[112:115], v[196:199]\n\t" \
  "v_cvt_pk_bf16_f32 v61, v178, v179\n\t" \
  "ds_write_b64 v4, v[60:61]\n\t" \
  "s_mov_b32 s98, 131072\n\t" \
  "buffer_store_dwordx4 v[136:139], v29, s[44:47], s98 offen sc0 sc1\n\t" \
  "v_mfma_f32_16x16x32_bf16 v[196:199], v[40:43], a[32:35], v[196:199]\n\t" \
  "v_cvt_pk_bf16_f32 v62, v172, v173\n\t" \
  "v_cvt_pk_bf16_f32 v63, v174, v175\n\t" \
  "v_mfma_f32_16x16x32_bf16 v[200:203], a[208:211], v[112:115], v[200:203]\n\t" \
  "ds_write_b64 v3, v[62:63]\n\t" \
  "v_cvt_pk_bf16_f32 v60, v168, v169\n\t" \
  "v_mfma_f32_16x16x32_bf16 v[200:203], a[212:215], v[116:119], v[200:203]\n\t" \
  "v_cvt_pk_bf16_f32 v61, v170, v171\n\t" \
  "ds_write_b64 v2, v[60:61]\n\t" \
  "v_mfma_f32_16x16x32_bf16 v[200:203], a[148:151], v[84:87], v[200:203]\n\t" \
  "v_cvt_pk_bf16_f32 v62, v164, v165\n\t" \
  "v_cvt_pk_bf16_f32 v63, v166, v167\n\t" \
  "v_mfma_f32_16x16x32_bf16 v[200:203], a[144:147], v[80:83], v[200:203]\n\t" \
  "ds_write_b64 v1, v[62:63]\n\t" \
  "v_cvt_pk_bf16_f32 v60, v160, v161\n\t" \
  "v_mfma_f32_16x16x32_bf16 v[200:203], v[42:45], a[36:39], v[200:203]\n\t" \
  "v_cvt_pk_bf16_f32 v61, v162, v163\n\t" \
  "ds_write_b64 v0, v[60:61]\n\t" \
  "s_mov_b32 s98, 196608\n\t" \
  "buffer_store_dwordx4 v[140:143], v29, s[44:47], s98 offen sc0 sc1\n\t" \
  "v_mfma_f32_16x16x32_bf16 v[204:207], a[152:155], v[80:83], v[204:207]\n\t" \
  "ds_read_b128 v[160:163], v20 offset:0\n\t" \
  "ds_read_b128 v[164:167], v21 offset:1024\n\t" \
  "v_mfma_f32_16x16x32_bf16 v[204:207], a[156:159], v[84:87], v[204:207]\n\t" \
  "ds_read_b128 v[168:171], v22 offset:2048\n\t" \
  "ds_read_b128 v[172:175], v23 offset:3072\n\t" \
  "v_mfma_f32_16x16x32_bf16 v[204:207], a[220:223], v[116:119], v[204:207]\n\t" \
  "v_mfma_f32_16x16x32_bf16 v[204:207], a[216:219], v[112:115], v[204:207]\n\t" \
  "v_mfma_f32_16x16x32_bf16 v[204:207], v[40:43], a[36:39], v[204:207]\n\t" \
  "v_mfma_f32_16x16x32_bf16 v[208:211], a[224:227], v[112:115], v[208:211]\n\t" \
  "s_waitcnt lgkmcnt(0)\n\t" \
  "s_mov_b32 s98, 262144\n\t" \
  "buffer_store_dwordx4 v[160:163], v29, s[44:47], s98 offen sc0 sc1\n\t" \
  "v_mfma_f32_16x16x32_bf16 v[208:211], a[228:231], v[116:119], v[208:211]\n\t" \
  "v_mfma_f32_16x16x32_bf16 v[208:211], a[164:167], v[84:87], v[208:211]\n\t" \
  "v_mfma_f32_16x16x32_bf16 v[208:211], a[160:163], v[80:83], v[208:211]\n\t" \
  "v_mfma_f32_16x16x32_bf16 v[208:211], v[42:45], a[40:43], v[208:211]\n\t" \
  "v_mfma_f32_16x16x32_bf16 v[212:215], a[168:171], v[80:83], v[212:215]\n\t" \
  "v_mfma_f32_16x16x32_bf16 v[212:215], a[172:175], v[84:87], v[212:215]\n\t" \
  "s_mov_b32 s98, 327680\n\t" \
  "buffer_store_dwordx4 v[164:167], v29, s[44:47], s98 offen sc0 sc1\n\t" \
  "v_mfma_f32_16x16x32_bf16 v[212:215], a[236:239], v[116:119], v[212:215]\n\t" \
  "v_mfma_f32_16x16x32_bf16 v[212:215], a[232:235], v[112:115], v[212:215]\n\t" \
  "v_mfma_f32_16x16x32_bf16 v[212:215], v[40:43], a[40:43], v[212:215]\n\t" \
  "v_mfma_f32_16x16x32_bf16 v[216:219], a[240:243], v[112:115], v[216:219]\n\t" \
  "v_mfma_f32_16x16x32_bf16 v[216:219], a[244:247], v[116:119], v[216:219]\n\t" \
  "v_mfma_f32_16x16x32_bf16 v[216:219], a[180:183], v[84:87], v[216:219]\n\t" \
  "s_mov_b32 s98, 393216\n\t" \
  "buffer_store_dwordx4 v[168:171], v29, s[44:47], s98 offen sc0 sc1\n\t" \
  "v_mfma_f32_16x16x32_bf16 v[216:219], a[176:179], v[80:83], v[216:219]\n\t" \
  "v_mfma_f32_16x16x32_bf16 v[216:219], v[42:45], a[44:47], v[216:219]\n\t" \
  "v_mfma_f32_16x16x32_bf16 v[220:223], a[184:187], v[80:83], v[220:223]\n\t" \
  "v_mfma_f32_16x16x32_bf16 v[220:223], a[188:191], v[84:87], v[220:223]\n\t" \
  "v_mfma_f32_16x16x32_bf16 v[220:223], a[252:255], v[116:119], v[220:223]\n\t" \
  "v_mfma_f32_16x16x32_bf16 v[220:223], a[248:251], v[112:115], v[220:223]\n\t" \
  "s_mov_b32 s98, 458752\n\t" \
  "buffer_store_dwordx4 v[172:175], v29, s[44:47], s98 offen sc0 sc1\n\t" \
  "v_mfma_f32_16x16x32_bf16 v[220:223], v[40:43], a[44:47], v[220:223]\n\t" \
  "v_mfma_f32_16x16x32_bf16 v[252:255], a[248:251], v[120:123], v[252:255]\n\t" \
  "v_mfma_f32_16x16x32_bf16 v[252:255], a[252:255], v[124:127], v[252:255]\n\t" \
  "v_mfma_f32_16x16x32_bf16 v[252:255], a[188:191], v[92:95], v[252:255]\n\t" \
  "v_mfma_f32_16x16x32_bf16 v[252:255], a[184:187], v[88:91], v[252:255]\n\t" \
  "v_cvt_pk_bf16_f32 v60, v192, v193\n\t" \
  "v_cvt_pk_bf16_f32 v61, v194, v195\n\t" \
  "v_mfma_f32_16x16x32_bf16 v[252:255], v[40:43], a[60:63], v[252:255]\n\t" \
  "ds_write_b64 v0, v[60:61]\n\t" \
  "v_cvt_pk_bf16_f32 v62, v196, v197\n\t" \
  "v_mfma_f32_16x16x32_bf16 v[248:251], a[176:179], v[88:91], v[248:251]\n\t" \
  "v_cvt_pk_bf16_f32 v63, v198, v199\n\t" \
  "ds_write_b64 v1, v[62:63]\n\t" \
  "v_mfma_f32_16x16x32_bf16 v[248:251], a[180:183], v[92:95], v[248:251]\n\t" \
  "v_cvt_pk_bf16_f32 v60, v200, v201\n\t" \
  "v_cvt_pk_bf16_f32 v61, v202, v203\n\t" \
  "v_mfma_f32_16x16x32_bf16 v[248:251], a[244:247], v[124:127], v[248:251]\n\t" \
  "ds_write_b64 v2, v[60:61]\n\t" \
  "v_cvt_pk_bf16_f32 v62, v204, v205\n\t" \
  "v_mfma_f32_16x16x32_bf16 v[248:251], a[240:243], v[120:123], v[248:251]\n\t" \
  "v_cvt_pk_bf16_f32 v63, v206, v207\n\t" \
  "ds_write_b64 v3, v[62:63]\n\t" \
  "v_mfma_f32_16x16x32_bf16 v[248:251], v[42:45], a[60:63], v[248:251]\n\t" \
  "v_cvt_pk_bf16_f32 v60, v208, v209\n\t" \
  "v_cvt_pk_bf16_f32 v61, v210, v211\n\t" \
  "v_mfma_f32_16x16x32_bf16 v[244:247], a[232:235], v[120:123], v[244:247]\n\t" \
  "ds_write_b64 v4, v[60:61]\n\t" \
  "v_cvt_pk_bf16_f32 v62, v212, v213\n\t" \
  "v_mfma_f32_16x16x32_bf16 v[244:247], a[236:239], v[124:127], v[244:247]\n\t" \
  "v_cvt_pk_bf16_f32 v63, v214, v215\n\t" \
  "ds_write_b64 v5, v[62:63]\n\t" \
  "v_mfma_f32_16x16x32_bf16 v[244:247], a[172:175], v[92:95], v[244:247]\n\t" \
  "v_cvt_pk_bf16_f32 v60, v216, v217\n\t" \
  "v_cvt_pk_bf16_f32 v61, v218, v219\n\t" \
  "v_mfma_f32_16x16x32_bf16 v[244:247], a[168:171], v[88:91], v[244:247]\n\t" \
  "ds_write_b64 v6, v[60:61]\n\t" \
  "v_cvt_pk_bf16_f32 v62, v220, v221\n\t" \
  "v_mfma_f32_16x16x32_bf16 v[244:247], v[40:43], a[56:59], v[244:247]\n\t" \
  "v_cvt_pk_bf16_f32 v63, v222, v223\n\t" \
  "ds_write_b64 v7, v[62:63]\n\t" \
  "v_mfma_f32_16x16x32_bf16 v[240:243], a[160:163], v[88:91], v[240:243]\n\t" \
  "ds_read_b128 v[192:195], v20 offset:0\n\t" \
  "ds_read_b128 v[196:199], v21 offset:1024\n\t" \
  "v_mfma_f32_16x16x32_bf16 v[240:243], a[164:167], v[92:95], v[240:243]\n\t" \
  "ds_read_b128 v[200:203], v22 offset:2048\n\t" \
  "ds_read_b128 v[204:207], v23 offset:3072\n\t" \
  "v_mfma_f32_16x16x32_bf16 v[240:243], a[228:231], v[124:127], v[240:243]\n\t" \
  "v_mfma_f32_16x16x32_bf16 v[240:243], a[224:227], v[120:123], v[240:243]\n\t" \
  "v_mfma_f32_16x16x32_bf16 v[240:243], v[42:45], a[56:59], v[240:243]\n\t" \
  "v_mfma_f32_16x16x32_bf16 v[236:239], a[216:219], v[120:123], v[236:239]\n\t" \
  "s_waitcnt lgkmcnt(0)\n\t" \
  "s_mov_b32 s98, 524288\n\t" \
  "buffer_store_dwordx4 v[192:195], v29, s[44:47], s98 offen sc0 sc1\n\t" \
  "v_mfma_f32_16x16x32_bf16 v[236:239], a[220:223], v[124:127], v[236:239]\n\t" \
  "v_mfma_f32_16x16x32_bf16 v[236:239], a[156:159], v[92:95], v[236:239]\n\t" \
  "v_mfma_f32_16x16x32_bf16 v[236:239], a[152:155], v[88:91], v[236:239]\n\t" \
  "v_mfma_f32_16x16x32_bf16 v[236:239], v[40:43], a[52:55], v[236:239]\n\t" \
  "v_mfma_f32_16x16x32_bf16 v[232:235], a[144:147], v[88:91], v[232:235]\n\t" \
  "v_mfma_f32_16x16x32_bf16 v[232:235], a[148:151], v[92:95], v[232:235]\n\t" \
  "s_mov_b32 s98, 589824\n\t" \
  "buffer_store_dwordx4 v[196:199], v29, s[44:47], s98 offen sc0 sc1\n\t" \
  "v_mfma_f32_16x16x32_bf16 v[232:235], a[212:215], v[124:127], v[232:235]\n\t" \
  "v_mfma_f32_16x16x32_bf16 v[232:235], a[208:211], v[120:123], v[232:235]\n\t" \
  "v_mfma_f32_16x16x32_bf16 v[232:235], v[42:45], a[52:55], v[232:235]\n\t" \
  "v_mfma_f32_16x16x32_bf16 v[228:231], a[200:203], v[120:123], v[228:231]\n\t" \
  "v_mfma_f32_16x16x32_bf16 v[228:231], a[204:207], v[124:127], v[228:231]\n\t" \
  "v_mfma_f32_16x16x32_bf16 v[228:231], a[140:143], v[92:95], v[228:231]\n\t" \
  "s_mov_b32 s98, 655360\n\t" \
  "buffer_store_dwordx4 v[200:203], v29, s[44:47], s98 offen sc0 sc1\n\t" \
  "v_mfma_f32_16x16x32_bf16 v[228:231], a[136:139], v[88:91], v[228:231]\n\t" \
  "v_mfma_f32_16x16x32_bf16 v[228:231], v[40:43], a[48:51], v[228:231]\n\t" \
  "v_mfma_f32_16x16x32_bf16 v[224:227], a[128:131], v[88:91], v[224:227]\n\t" \
  "v_mfma_f32_16x16x32_bf16 v[224:227], a[132:135], v[92:95], v[224:227]\n\t" \
  "v_mfma_f32_16x16x32_bf16 v[224:227], a[196:199], v[124:127], v[224:227]\n\t" \
  "v_mfma_f32_16x16x32_bf16 v[224:227], a[192:195], v[120:123], v[224:227]\n\t" \
  "s_mov_b32 s98, 720896\n\t" \
  "buffer_store_dwordx4 v[204:207], v29, s[44:47], s98 offen sc0 sc1\n\t" \
  "v_mfma_f32_16x16x32_bf16 v[224:227], v[42:45], a[48:51], v[224:227]\n\t" \
  "s_nop 7\n\t" \
  "v_cvt_pk_bf16_f32 v62, v252, v253\n\t" \
  "v_cvt_pk_bf16_f32 v63, v254, v255\n\t" \
  "ds_write_b64 v7, v[62:63]\n\t" \
  "v_cvt_pk_bf16_f32 v60, v248, v249\n\t" \
  "v_cvt_pk_bf16_f32 v61, v250, v251\n\t" \
  "ds_write_b64 v6, v[60:61]\n\t" \
  "v_cvt_pk_bf16_f32 v62, v244, v245\n\t" \
  "v_cvt_pk_bf16_f32 v63, v246, v247\n\t" \
  "ds_write_b64 v5, v[62:63]\n\t" \
  "v_cvt_pk_bf16_f32 v60, v240, v241\n\t" \
  "v_cvt_pk_bf16_f32 v61, v242, v243\n\t" \
  "ds_write_b64 v4, v[60:61]\n\t" \
  "v_cvt_pk_bf16_f32 v62, v236, v237\n\t" \
  "v_cvt_pk_bf16_f32 v63, v238, v239\n\t" \
  "ds_write_b64 v3, v[62:63]\n\t" \
  "v_cvt_pk_bf16_f32 v60, v232, v233\n\t" \
  "v_cvt_pk_bf16_f32 v61, v234, v235\n\t" \
  "ds_write_b64 v2, v[60:61]\n\t" \
  "v_cvt_pk_bf16_f32 v62, v228, v229\n\t" \
  "v_cvt_pk_bf16_f32 v63, v230, v231\n\t" \
  "ds_write_b64 v1, v[62:63]\n\t" \
  "v_cvt_pk_bf16_f32 v60, v224, v225\n\t" \
  "v_cvt_pk_bf16_f32 v61, v226, v227\n\t" \
  "ds_write_b64 v0, v[60:61]\n\t" \
  "ds_read_b128 v[224:227], v20 offset:0\n\t" \
  "ds_read_b128 v[228:231], v21 offset:1024\n\t" \
  "ds_read_b128 v[232:235], v22 offset:2048\n\t" \
  "ds_read_b128 v[236:239], v23 offset:3072\n\t" \
  "s_mov_b32 s98, 786432\n\t" \
  "s_waitcnt lgkmcnt(3)\n\t" \
  "buffer_store_dwordx4 v[224:227], v29, s[44:47], s98 offen sc0 sc1\n\t" \
  "s_mov_b32 s98, 851968\n\t" \
  "s_waitcnt lgkmcnt(2)\n\t" \
  "buffer_store_dwordx4 v[228:231], v29, s[44:47], s98 offen sc0 sc1\n\t" \
  "s_mov_b32 s98, 917504\n\t" \
  "s_waitcnt lgkmcnt(1)\n\t" \
  "buffer_store_dwordx4 v[232:235], v29, s[44:47], s98 offen sc0 sc1\n\t" \
  "s_mov_b32 s98, 983040\n\t" \
  "s_waitcnt lgkmcnt(0)\n\t" \
  "buffer_store_dwordx4 v[236:239], v29, s[44:47], s98 offen sc0 sc1\n\t" \
  "s_branch END_%=\n\t" \
  "NANP_%=:\n\t" \
  "v_mov_b32 v80, 0x7fc07fc0\n\t" \
  "v_mov_b32 v81, 0x7fc07fc0\n\t" \
  "v_mov_b32 v82, 0x7fc07fc0\n\t" \
  "v_mov_b32 v83, 0x7fc07fc0\n\t" \
  "s_mov_b32 s98, 0\n\t" \
  "buffer_store_dwordx4 v[80:83], v29, s[44:47], s98 offen\n\t" \
  "s_mov_b32 s98, 65536\n\t" \
  "buffer_store_dwordx4 v[80:83], v29, s[44:47], s98 offen\n\t" \
  "s_mov_b32 s98, 131072\n\t" \
  "buffer_store_dwordx4 v[80:83], v29, s[44:47], s98 offen\n\t" \
  "s_mov_b32 s98, 196608\n\t" \
  "buffer_store_dwordx4 v[80:83], v29, s[44:47], s98 offen\n\t" \
  "s_mov_b32 s98, 262144\n\t" \
  "buffer_store_dwordx4 v[80:83], v29, s[44:47], s98 offen\n\t" \
  "s_mov_b32 s98, 327680\n\t" \
  "buffer_store_dwordx4 v[80:83], v29, s[44:47], s98 offen\n\t" \
  "s_mov_b32 s98, 393216\n\t" \
  "buffer_store_dwordx4 v[80:83], v29, s[44:47], s98 offen\n\t" \
  "s_mov_b32 s98, 458752\n\t" \
  "buffer_store_dwordx4 v[80:83], v29, s[44:47], s98 offen\n\t" \
  "s_mov_b32 s98, 524288\n\t" \
  "buffer_store_dwordx4 v[80:83], v29, s[44:47], s98 offen\n\t" \
  "s_mov_b32 s98, 589824\n\t" \
  "buffer_store_dwordx4 v[80:83], v29, s[44:47], s98 offen\n\t" \
  "s_mov_b32 s98, 655360\n\t" \
  "buffer_store_dwordx4 v[80:83], v29, s[44:47], s98 offen\n\t" \
  "s_mov_b32 s98, 720896\n\t" \
  "buffer_store_dwordx4 v[80:83], v29, s[44:47], s98 offen\n\t" \
  "s_mov_b32 s98, 786432\n\t" \
  "buffer_store_dwordx4 v[80:83], v29, s[44:47], s98 offen\n\t" \
  "s_mov_b32 s98, 851968\n\t" \
  "buffer_store_dwordx4 v[80:83], v29, s[44:47], s98 offen\n\t" \
  "s_mov_b32 s98, 917504\n\t" \
  "buffer_store_dwordx4 v[80:83], v29, s[44:47], s98 offen\n\t" \
  "s_mov_b32 s98, 983040\n\t" \
  "buffer_store_dwordx4 v[80:83], v29, s[44:47], s98 offen\n\t" \
  "s_branch END_%=\n\t" \
  ".p2align 6\n\t" \
  "EVT1_%=:\n\t" \
  "s_cmp_lg_u32 s30, 0\n\t" \
  "s_cbranch_scc1 PUB3_%=\n\t" \
  "global_store_dwordx4 v39, v[32:35], s[52:53] sc0 sc1\n\t" \
  "v_mfma_f32_16x16x32_bf16 a[72:75], a[144:147], v[80:83], 0\n\t" \
  "ds_read_b64_tr_b16 a[220:221], v15 offset:40960\n\t" \
  "s_and_b32 s28, s26, 15\n\t" \
  "v_mfma_f32_16x16x32_bf16 a[72:75], a[148:151], v[84:87], a[72:75]\n\t" \
  "s_sub_u32 s28, s28, 2\n\t" \
  "s_mov_b32 s87, 1\n\t" \
  "v_mfma_f32_16x16x32_bf16 a[76:79], a[156:159], v[84:87], 0\n\t" \
  "ds_read_b64_tr_b16 a[222:223], v15 offset:41984\n\t" \
  "s_mov_b32 s30, 1\n\t" \
  "v_mfma_f32_16x16x32_bf16 a[76:79], a[152:155], v[80:83], a[76:79]\n\t" \
  "s_lshr_b64 s[26:27], s[26:27], 4\n\t" \
  "v_mfma_f32_16x16x32_bf16 a[80:83], a[160:163], v[80:83], 0\n\t" \
  "s_cmp_eq_u32 s88, 15\n\t" \
  "s_cselect_b32 s74, 0, s74\n\t" \
  "v_mfma_f32_16x16x32_bf16 a[80:83], a[164:167], v[84:87], a[80:83]\n\t" \
  "ds_read_b128 v[116:119], v11 offset:20480\n\t" \
  "v_add_u32 v8, s74, v8\n\t" \
  "v_mfma_f32_16x16x32_bf16 a[84:87], a[172:175], v[84:87], 0\n\t" \
  "ds_read_b128 v[120:123], v10 offset:22528\n\t" \
  "global_load_dwordx4 v[32:35], v8, s[48:49] nt\n\t" \
  "v_mfma_f32_16x16x32_bf16 a[84:87], a[168:171], v[80:83], a[84:87]\n\t" \
  "s_lshl_b32 s98, s74, 10\n\t" \
  "v_add_u32 v39, s98, v39\n\t" \
  "v_mfma_f32_16x16x32_bf16 a[88:91], a[176:179], v[80:83], 0\n\t" \
  "ds_read_b128 v[124:127], v11 offset:22528\n\t" \
  "s_xor_b32 s74, s74, s80\n\t" \
  "v_mfma_f32_16x16x32_bf16 a[88:91], a[180:183], v[84:87], a[88:91]\n\t" \
  "s_add_u32 s88, s88, 1\n\t" \
  "v_mfma_f32_16x16x32_bf16 a[92:95], a[188:191], v[84:87], 0\n\t" \
  "s_branch RET2_%=\n\t" \
  ".p2align 6\n\t" \
  "PUB3_%=:\n\t" \
  "s_cmp_lg_u32 s71, 0\n\t" \
  "s_cbranch_scc1 PSK4_%=\n\t" \
  "s_mov_b64 exec, 1\n\t" \
  "global_atomic_add v38, v31, s[54:55] sc1\n\t" \
  "s_mov_b64 exec, -1\n\t" \
  "PSK4_%=:\n\t" \
  "v_mfma_f32_16x16x32_bf16 a[72:75], a[144:147], v[80:83], 0\n\t" \
  "ds_read_b64_tr_b16 a[220:221], v15 offset:40960\n\t" \
  "v_add_u32 v38, s31, v38\n\t" \
  "v_mfma_f32_16x16x32_bf16 a[72:75], a[148:151], v[84:87], a[72:75]\n\t" \
  "s_xor_b32 s31, s31, s80\n\t" \
  "s_mov_b32 s30, 0\n\t" \
  "v_mfma_f32_16x16x32_bf16 a[76:79], a[156:159], v[84:87], 0\n\t" \
  "ds_read_b64_tr_b16 a[222:223], v15 offset:41984\n\t" \
  "s_add_u32 s28, s28, s72\n\t" \
  "v_mfma_f32_16x16x32_bf16 a[76:79], a[152:155], v[80:83], a[76:79]\n\t" \
  "s_cmp_eq_u32 s88, 16\n\t" \
  "s_cselect_b32 s87, 0x7fffffff, s28\n\t" \
  "v_mfma_f32_16x16x32_bf16 a[80:83], a[160:163], v[80:83], 0\n\t" \
  "s_mov_b32 s72, s73\n\t" \
  "s_mov_b32 s73, 0\n\t" \
  "v_mfma_f32_16x16x32_bf16 a[80:83], a[164:167], v[84:87], a[80:83]\n\t" \
  "ds_read_b128 v[116:119], v11 offset:20480\n\t" \
  "v_mfma_f32_16x16x32_bf16 a[84:87], a[172:175], v[84:87], 0\n\t" \
  "ds_read_b128 v[120:123], v10 offset:22528\n\t" \
  "v_mfma_f32_16x16x32_bf16 a[84:87], a[168:171], v[80:83], a[84:87]\n\t" \
  "v_mfma_f32_16x16x32_bf16 a[88:91], a[176:179], v[80:83], 0\n\t" \
  "ds_read_b128 v[124:127], v11 offset:22528\n\t" \
  "v_mfma_f32_16x16x32_bf16 a[88:91], a[180:183], v[84:87], a[88:91]\n\t" \
  "v_mfma_f32_16x16x32_bf16 a[92:95], a[188:191], v[84:87], 0\n\t" \
  "s_branch RET2_%=\n\t" \
  ".p2align 6\n\t" \
  "EVT7_%=:\n\t" \
  "s_cmp_lg_u32 s30, 0\n\t" \
  "s_cbranch_scc1 PUB9_%=\n\t" \
  "global_store_dwordx4 v39, v[32:35], s[52:53] sc0 sc1\n\t" \
  "v_mfma_f32_16x16x32_bf16 v[168:171], a[148:151], v[108:111], 0\n\t" \
  "ds_read_b64_tr_b16 a[242:243], v14 offset:50176\n\t" \
  "s_and_b32 s28, s26, 15\n\t" \
  "v_mfma_f32_16x16x32_bf16 v[168:171], a[144:147], v[104:107], v[168:171]\n\t" \
  "s_sub_u32 s28, s28, 2\n\t" \
  "s_mov_b32 s87, 1\n\t" \
  "v_mfma_f32_16x16x32_bf16 v[164:167], a[136:139], v[104:107], 0\n\t" \
  "ds_read_b64_tr_b16 a[244:245], v14 offset:57344\n\t" \
  "s_mov_b32 s30, 1\n\t" \
  "v_mfma_f32_16x16x32_bf16 v[164:167], a[140:143], v[108:111], v[164:167]\n\t" \
  "s_lshr_b64 s[26:27], s[26:27], 4\n\t" \
  "v_mfma_f32_16x16x32_bf16 v[160:163], a[132:135], v[108:111], 0\n\t" \
  "ds_read_b64_tr_b16 a[246:247], v14 offset:58368\n\t" \
  "v_mfma_f32_16x16x32_bf16 v[160:163], a[128:131], v[104:107], v[160:163]\n\t" \
  "ds_read_b128 v[104:107], v10 offset:51200\n\t" \
  "ds_read_b128 v[108:111], v11 offset:51200\n\t" \
  "v_mfma_f32_16x16x32_bf16 v[192:195], a[128:131], v[112:115], 0\n\t" \
  "ds_read_b64_tr_b16 a[248:249], v15 offset:49152\n\t" \
  "v_mfma_f32_16x16x32_bf16 v[192:195], a[132:135], v[116:119], v[192:195]\n\t" \
  "s_cmp_eq_u32 s88, 15\n\t" \
  "s_cselect_b32 s74, 0, s74\n\t" \
  "v_mfma_f32_16x16x32_bf16 v[196:199], a[140:143], v[116:119], 0\n\t" \
  "ds_read_b64_tr_b16 a[250:251], v15 offset:50176\n\t" \
  "v_add_u32 v8, s74, v8\n\t" \
  "v_mfma_f32_16x16x32_bf16 v[196:199], a[136:139], v[112:115], v[196:199]\n\t" \
  "global_load_dwordx4 v[32:35], v8, s[48:49] nt\n\t" \
  "s_lshl_b32 s98, s74, 10\n\t" \
  "v_mfma_f32_16x16x32_bf16 v[200:203], a[144:147], v[112:115], 0\n\t" \
  "ds_read_b64_tr_b16 a[252:253], v15 offset:57344\n\t" \
  "v_add_u32 v39, s98, v39\n\t" \
  "v_mfma_f32_16x16x32_bf16 v[200:203], a[148:151], v[116:119], v[200:203]\n\t" \
  "s_xor_b32 s74, s74, s80\n\t" \
  "s_add_u32 s88, s88, 1\n\t" \
  "v_mfma_f32_16x16x32_bf16 v[204:207], a[156:159], v[116:119], 0\n\t" \
  "ds_read_b64_tr_b16 a[254:255], v15 offset:58368\n\t" \
  "s_branch RET8_%=\n\t" \
  ".p2align 6\n\t" \
  "PUB9_%=:\n\t" \
  "s_cmp_lg_u32 s71, 0\n\t" \
  "s_cbranch_scc1 PSK10_%=\n\t" \
  "s_mov_b64 exec, 1\n\t" \
  "global_atomic_add v38, v31, s[54:55] sc1\n\t" \
  "s_mov_b64 exec, -1\n\t" \
  "PSK10_%=:\n\t" \
  "v_mfma_f32_16x16x32_bf16 v[168:171], a[148:151], v[108:111], 0\n\t" \
  "ds_read_b64_tr_b16 a[242:243], v14 offset:50176\n\t" \
  "v_add_u32 v38, s31, v38\n\t" \
  "v_mfma_f32_16x16x32_bf16 v[168:171], a[144:147], v[104:107], v[168:171]\n\t" \
  "s_xor_b32 s31, s31, s80\n\t" \
  "s_mov_b32 s30, 0\n\t" \
  "v_mfma_f32_16x16x32_bf16 v[164:167], a[136:139], v[104:107], 0\n\t" \
  "ds_read_b64_tr_b16 a[244:245], v14 offset:57344\n\t" \
  "s_add_u32 s28, s28, s72\n\t" \
  "v_mfma_f32_16x16x32_bf16 v[164:167], a[140:143], v[108:111], v[164:167]\n\t" \
  "s_cmp_eq_u32 s88, 16\n\t" \
  "s_cselect_b32 s87, 0x7fffffff, s28\n\t" \
  "v_mfma_f32_16x16x32_bf16 v[160:163], a[132:135], v[108:111], 0\n\t" \
  "ds_read_b64_tr_b16 a[246:247], v14 offset:58368\n\t" \
  "s_mov_b32 s72, s73\n\t" \
  "v_mfma_f32_16x16x32_bf16 v[160:163], a[128:131], v[104:107], v[160:163]\n\t" \
  "ds_read_b128 v[104:107], v10 offset:51200\n\t" \
  "ds_read_b128 v[108:111], v11 offset:51200\n\t" \
  "v_mfma_f32_16x16x32_bf16 v[192:195], a[128:131], v[112:115], 0\n\t" \
  "ds_read_b64_tr_b16 a[248:249], v15 offset:49152\n\t" \
  "s_mov_b32 s73, 0\n\t" \
  "v_mfma_f32_16x16x32_bf16 v[192:195], a[132:135], v[116:119], v[192:195]\n\t" \
  "v_mfma_f32_16x16x32_bf16 v[196:199], a[140:143], v[116:119], 0\n\t" \
  "ds_read_b64_tr_b16 a[250:251], v15 offset:50176\n\t" \
  "v_mfma_f32_16x16x32_bf16 v[196:199], a[136:139], v[112:115], v[196:199]\n\t" \
  "v_mfma_f32_16x16x32_bf16 v[200:203], a[144:147], v[112:115], 0\n\t" \
  "ds_read_b64_tr_b16 a[252:253], v15 offset:57344\n\t" \
  "v_mfma_f32_16x16x32_bf16 v[200:203], a[148:151], v[116:119], v[200:203]\n\t" \
  "v_mfma_f32_16x16x32_bf16 v[204:207], a[156:159], v[116:119], 0\n\t" \
  "ds_read_b64_tr_b16 a[254:255], v15 offset:58368\n\t" \
  "s_branch RET8_%=\n\t" \
  ".p2align 6\n\t" \
  "EVT11_%=:\n\t" \
  "s_cmp_lg_u32 s30, 0\n\t" \
  "s_cbranch_scc1 PUB13_%=\n\t" \
  "global_store_dwordx4 v39, v[32:35], s[52:53] sc0 sc1\n\t" \
  "v_mfma_f32_16x16x32_bf16 a[40:43], a[212:215], v[76:79], a[40:43]\n\t" \
  "ds_read_b64_tr_b16 a[146:147], v14 offset:1024\n\t" \
  "s_and_b32 s28, s26, 15\n\t" \
  "v_mfma_f32_16x16x32_bf16 a[40:43], a[208:211], v[72:75], a[40:43]\n\t" \
  "s_sub_u32 s28, s28, 2\n\t" \
  "s_mov_b32 s87, 1\n\t" \
  "v_mfma_f32_16x16x32_bf16 a[36:39], a[200:203], v[72:75], a[36:39]\n\t" \
  "ds_read_b64_tr_b16 a[148:149], v14 offset:8192\n\t" \
  "s_mov_b32 s30, 1\n\t" \
  "v_mfma_f32_16x16x32_bf16 a[36:39], a[204:207], v[76:79], a[36:39]\n\t" \
  "s_lshr_b64 s[26:27], s[26:27], 4\n\t" \
  "v_mfma_f32_16x16x32_bf16 a[32:35], a[196:199], v[76:79], a[32:35]\n\t" \
  "ds_read_b64_tr_b16 a[150:151], v14 offset:9216\n\t" \
  "v_mfma_f32_16x16x32_bf16 a[32:35], a[192:195], v[72:75], a[32:35]\n\t" \
  "ds_read_b128 v[72:75], v10 offset:2048\n\t" \
  "ds_read_b128 v[76:79], v11 offset:2048\n\t" \
  "v_mfma_f32_16x16x32_bf16 a[64:67], a[192:195], v[80:83], a[64:67]\n\t" \
  "ds_read_b64_tr_b16 a[152:153], v15 offset:0\n\t" \
  "v_mfma_f32_16x16x32_bf16 a[64:67], a[196:199], v[84:87], a[64:67]\n\t" \
  "s_cmp_eq_u32 s88, 15\n\t" \
  "s_cselect_b32 s74, 0, s74\n\t" \
  "v_mfma_f32_16x16x32_bf16 a[68:71], a[204:207], v[84:87], a[68:71]\n\t" \
  "ds_read_b64_tr_b16 a[154:155], v15 offset:1024\n\t" \
  "v_add_u32 v8, s74, v8\n\t" \
  "v_mfma_f32_16x16x32_bf16 a[68:71], a[200:203], v[80:83], a[68:71]\n\t" \
  "global_load_dwordx4 v[32:35], v8, s[48:49] nt\n\t" \
  "s_lshl_b32 s98, s74, 10\n\t" \
  "v_mfma_f32_16x16x32_bf16 a[72:75], a[208:211], v[80:83], a[72:75]\n\t" \
  "ds_read_b64_tr_b16 a[156:157], v15 offset:8192\n\t" \
  "v_add_u32 v39, s98, v39\n\t" \
  "v_mfma_f32_16x16x32_bf16 a[72:75], a[212:215], v[84:87], a[72:75]\n\t" \
  "s_xor_b32 s74, s74, s80\n\t" \
  "s_add_u32 s88, s88, 1\n\t" \
  "v_mfma_f32_16x16x32_bf16 a[76:79], a[220:223], v[84:87], a[76:79]\n\t" \
  "ds_read_b64_tr_b16 a[158:159], v15 offset:9216\n\t" \
  "s_branch RET12_%=\n\t" \
  ".p2align 6\n\t" \
  "PUB13_%=:\n\t" \
  "s_cmp_lg_u32 s71, 0\n\t" \
  "s_cbranch_scc1 PSK14_%=\n\t" \
  "s_mov_b64 exec, 1\n\t" \
  "global_atomic_add v38, v31, s[54:55] sc1\n\t" \
  "s_mov_b64 exec, -1\n\t" \
  "PSK14_%=:\n\t" \
  "v_mfma_f32_16x16x32_bf16 a[40:43], a[212:215], v[76:79], a[40:43]\n\t" \
  "ds_read_b64_tr_b16 a[146:147], v14 offset:1024\n\t" \
  "v_add_u32 v38, s31, v38\n\t" \
  "v_mfma_f32_16x16x32_bf16 a[40:43], a[208:211], v[72:75], a[40:43]\n\t" \
  "s_xor_b32 s31, s31, s80\n\t" \
  "s_mov_b32 s30, 0\n\t" \
  "v_mfma_f32_16x16x32_bf16 a[36:39], a[200:203], v[72:75], a[36:39]\n\t" \
  "ds_read_b64_tr_b16 a[148:149], v14 offset:8192\n\t" \
  "s_add_u32 s28, s28, s72\n\t" \
  "v_mfma_f32_16x16x32_bf16 a[36:39], a[204:207], v[76:79], a[36:39]\n\t" \
  "s_cmp_eq_u32 s88, 16\n\t" \
  "s_cselect_b32 s87, 0x7fffffff, s28\n\t" \
  "v_mfma_f32_16x16x32_bf16 a[32:35], a[196:199], v[76:79], a[32:35]\n\t" \
  "ds_read_b64_tr_b16 a[150:151], v14 offset:9216\n\t" \
  "s_mov_b32 s72, s73\n\t" \
  "v_mfma_f32_16x16x32_bf16 a[32:35], a[192:195], v[72:75], a[32:35]\n\t" \
  "ds_read_b128 v[72:75], v10 offset:2048\n\t" \
  "ds_read_b128 v[76:79], v11 offset:2048\n\t" \
  "v_mfma_f32_16x16x32_bf16 a[64:67], a[192:195], v[80:83], a[64:67]\n\t" \
  "ds_read_b64_tr_b16 a[152:153], v15 offset:0\n\t" \
  "s_mov_b32 s73, 0\n\t" \
  "v_mfma_f32_16x16x32_bf16 a[64:67], a[196:199], v[84:87], a[64:67]\n\t" \
  "v_mfma_f32_16x16x32_bf16 a[68:71], a[204:207], v[84:87], a[68:71]\n\t" \
  "ds_read_b64_tr_b16 a[154:155], v15 offset:1024\n\t" \
  "v_mfma_f32_16x16x32_bf16 a[68:71], a[200:203], v[80:83], a[68:71]\n\t" \
  "v_mfma_f32_16x16x32_bf16 a[72:75], a[208:211], v[80:83], a[72:75]\n\t" \
  "ds_read_b64_tr_b16 a[156:157], v15 offset:8192\n\t" \
  "v_mfma_f32_16x16x32_bf16 a[72:75], a[212:215], v[84:87], a[72:75]\n\t" \
  "v_mfma_f32_16x16x32_bf16 a[76:79], a[220:223], v[84:87], a[76:79]\n\t" \
  "ds_read_b64_tr_b16 a[158:159], v15 offset:9216\n\t" \
  "s_branch RET12_%=\n\t" \
  ".p2align 6\n\t" \
  "EVT17_%=:\n\t" \
  "s_cmp_lg_u32 s30, 0\n\t" \
  "s_cbranch_scc1 PUB19_%=\n\t" \
  "global_store_dwordx4 v39, v[32:35], s[52:53] sc0 sc1\n\t" \
  "v_mfma_f32_16x16x32_bf16 v[168:171], a[212:215], v[108:111], v[168:171]\n\t" \
  "ds_read_b64_tr_b16 a[178:179], v14 offset:17408\n\t" \
  "s_and_b32 s28, s26, 15\n\t" \
  "v_mfma_f32_16x16x32_bf16 v[168:171], a[208:211], v[104:107], v[168:171]\n\t" \
  "s_sub_u32 s28, s28, 2\n\t" \
  "s_mov_b32 s87, 1\n\t" \
  "v_mfma_f32_16x16x32_bf16 v[164:167], a[200:203], v[104:107], v[164:167]\n\t" \
  "ds_read_b64_tr_b16 a[180:181], v14 offset:24576\n\t" \
  "s_mov_b32 s30, 1\n\t" \
  "v_mfma_f32_16x16x32_bf16 v[164:167], a[204:207], v[108:111], v[164:167]\n\t" \
  "s_lshr_b64 s[26:27], s[26:27], 4\n\t" \
  "v_mfma_f32_16x16x32_bf16 v[160:163], a[196:199], v[108:111], v[160:163]\n\t" \
  "ds_read_b64_tr_b16 a[182:183], v14 offset:25600\n\t" \
  "v_mfma_f32_16x16x32_bf16 v[160:163], a[192:195], v[104:107], v[160:163]\n\t" \
  "ds_read_b128 v[104:107], v10 offset:18432\n\t" \
  "ds_read_b128 v[108:111], v11 offset:18432\n\t" \
  "v_mfma_f32_16x16x32_bf16 v[192:195], a[192:195], v[112:115], v[192:195]\n\t" \
  "ds_read_b64_tr_b16 a[184:185], v15 offset:16384\n\t" \
  "v_mfma_f32_16x16x32_bf16 v[192:195], a[196:199], v[116:119], v[192:195]\n\t" \
  "s_cmp_eq_u32 s88, 15\n\t" \
  "s_cselect_b32 s74, 0, s74\n\t" \
  "v_mfma_f32_16x16x32_bf16 v[196:199], a[204:207], v[116:119], v[196:199]\n\t" \
  "ds_read_b64_tr_b16 a[186:187], v15 offset:17408\n\t" \
  "v_add_u32 v8, s74, v8\n\t" \
  "v_mfma_f32_16x16x32_bf16 v[196:199], a[200:203], v[112:115], v[196:199]\n\t" \
  "global_load_dwordx4 v[32:35], v8, s[48:49] nt\n\t" \
  "s_lshl_b32 s98, s74, 10\n\t" \
  "v_mfma_f32_16x16x32_bf16 v[200:203], a[208:211], v[112:115], v[200:203]\n\t" \
  "ds_read_b64_tr_b16 a[188:189], v15 offset:24576\n\t" \
  "v_add_u32 v39, s98, v39\n\t" \
  "v_mfma_f32_16x16x32_bf16 v[200:203], a[212:215], v[116:119], v[200:203]\n\t" \
  "s_xor_b32 s74, s74, s80\n\t" \
  "s_add_u32 s88, s88, 1\n\t" \
  "v_mfma_f32_16x16x32_bf16 v[204:207], a[220:223], v[116:119], v[204:207]\n\t" \
  "ds_read_b64_tr_b16 a[190:191], v15 offset:25600\n\t" \
  "s_branch RET18_%=\n\t" \
  ".p2align 6\n\t" \
  "PUB19_%=:\n\t" \
  "s_cmp_lg_u32 s71, 0\n\t" \
  "s_cbranch_scc1 PSK20_%=\n\t" \
  "s_mov_b64 exec, 1\n\t" \
  "global_atomic_add v38, v31, s[54:55] sc1\n\t" \
  "s_mov_b64 exec, -1\n\t" \
  "PSK20_%=:\n\t" \
  "v_mfma_f32_16x16x32_bf16 v[168:171], a[212:215], v[108:111], v[168:171]\n\t" \
  "ds_read_b64_tr_b16 a[178:179], v14 offset:17408\n\t" \
  "v_add_u32 v38, s31, v38\n\t" \
  "v_mfma_f32_16x16x32_bf16 v[168:171], a[208:211], v[104:107], v[168:171]\n\t" \
  "s_xor_b32 s31, s31, s80\n\t" \
  "s_mov_b32 s30, 0\n\t" \
  "v_mfma_f32_16x16x32_bf16 v[164:167], a[200:203], v[104:107], v[164:167]\n\t" \
  "ds_read_b64_tr_b16 a[180:181], v14 offset:24576\n\t" \
  "s_add_u32 s28, s28, s72\n\t" \
  "v_mfma_f32_16x16x32_bf16 v[164:167], a[204:207], v[108:111], v[164:167]\n\t" \
  "s_cmp_eq_u32 s88, 16\n\t" \
  "s_cselect_b32 s87, 0x7fffffff, s28\n\t" \
  "v_mfma_f32_16x16x32_bf16 v[160:163], a[196:199], v[108:111], v[160:163]\n\t" \
  "ds_read_b64_tr_b16 a[182:183], v14 offset:25600\n\t" \
  "s_mov_b32 s72, s73\n\t" \
  "v_mfma_f32_16x16x32_bf16 v[160:163], a[192:195], v[104:107], v[160:163]\n\t" \
  "ds_read_b128 v[104:107], v10 offset:18432\n\t" \
  "ds_read_b128 v[108:111], v11 offset:18432\n\t" \
  "v_mfma_f32_16x16x32_bf16 v[192:195], a[192:195], v[112:115], v[192:195]\n\t" \
  "ds_read_b64_tr_b16 a[184:185], v15 offset:16384\n\t" \
  "s_mov_b32 s73, 0\n\t" \
  "v_mfma_f32_16x16x32_bf16 v[192:195], a[196:199], v[116:119], v[192:195]\n\t" \
  "v_mfma_f32_16x16x32_bf16 v[196:199], a[204:207], v[116:119], v[196:199]\n\t" \
  "ds_read_b64_tr_b16 a[186:187], v15 offset:17408\n\t" \
  "v_mfma_f32_16x16x32_bf16 v[196:199], a[200:203], v[112:115], v[196:199]\n\t" \
  "v_mfma_f32_16x16x32_bf16 v[200:203], a[208:211], v[112:115], v[200:203]\n\t" \
  "ds_read_b64_tr_b16 a[188:189], v15 offset:24576\n\t" \
  "v_mfma_f32_16x16x32_bf16 v[200:203], a[212:215], v[116:119], v[200:203]\n\t" \
  "v_mfma_f32_16x16x32_bf16 v[204:207], a[220:223], v[116:119], v[204:207]\n\t" \
  "ds_read_b64_tr_b16 a[190:191], v15 offset:25600\n\t" \
  "s_branch RET18_%=\n\t" \
  "SLOW24_%=:\n\t" \
  "s_memrealtime s[94:95]\n\t" \
  "s_waitcnt lgkmcnt(0)\n\t" \
  "SL25_%=:\n\t" \
  "v_mov_b32 v46, 0\n\t" \
  "s_lshl_b64 exec, 1, s99\n\t" \
  "global_load_dword v46, v16, s[50:51] sc0 sc1\n\t" \
  "s_mov_b64 exec, 1\n\t" \
  "global_load_dword v47, v30, s[50:51] offset:-128 sc1\n\t" \
  "s_mov_b64 exec, -1\n\t" \
  "s_waitcnt vmcnt(0)\n\t" \
  "v_cmp_le_u32 s[92:93], s86, v46\n\t" \
  "v_readfirstlane_b32 s98, v47\n\t" \
  "s_nop 4\n\t" \
  "s_or_b64 s[90:91], s[90:91], s[92:93]\n\t" \
  "s_bitcmp1_b64 s[90:91], s99\n\t" \
  "s_cbranch_scc1 RET23_%=\n\t" \
  "s_cmp_eq_u32 s98, s85\n\t" \
  "s_cbranch_scc1 SB26_%=\n\t" \
  "s_memrealtime s[96:97]\n\t" \
  "s_waitcnt lgkmcnt(0)\n\t" \
  "s_sub_u32 s96, s96, s94\n\t" \
  "s_cmp_gt_u32 s96, 50000000\n\t" \
  "s_cbranch_scc1 SB26_%=\n\t" \
  "s_sleep 2\n\t" \
  "s_branch SL25_%=\n\t" \
  "SB26_%=:\n\t" \
  "v_mov_b32 v47, s85\n\t" \
  "global_store_dword v30, v47, s[50:51] offset:-128 sc1\n\t" \
  "s_mov_b64 s[90:91], -1\n\t" \
  "s_mov_b64 s[58:59], -1\n\t" \
  "s_waitcnt vmcnt(0)\n\t" \
  "s_branch RET23_%=\n\t" \
  "POLL28_%=:\n\t" \
  "s_lshl_b64 exec, 1, s99\n\t" \
  "global_load_dword v36, v16, s[50:51] sc0 sc1\n\t" \
  "s_mov_b64 exec, -1\n\t" \
  "s_branch RET27_%=\n\t" \
  "STG29_%=:\n\t" \
  "s_waitcnt vmcnt(5)\n\t" \
  "v_cmp_le_u32 s[92:93], s86, v36\n\t" \
  "s_nop 3\n\t" \
  "s_or_b64 s[90:91], s[90:91], s[92:93]\n\t" \
  "s_bitcmp1_b64 s[90:91], s99\n\t" \
  "s_cbranch_scc1 RET23_%=\n\t" \
  "s_branch SLOW24_%=\n\t" \
  ".p2align 6\n\t" \
  "EVT30_%=:\n\t" \
  "s_cmp_lg_u32 s30, 0\n\t" \
  "s_cbranch_scc1 PUB32_%=\n\t" \
  "global_store_dwordx4 v39, v[32:35], s[52:53] sc0 sc1\n\t" \
  "v_mfma_f32_16x16x32_bf16 a[40:43], a[148:151], v[76:79], a[40:43]\n\t" \
  "ds_read_b64_tr_b16 a[210:211], v14 offset:33792\n\t" \
  "v_mfma_f32_16x16x32_bf16 a[40:43], a[144:147], v[72:75], a[40:43]\n\t" \
  "s_bitcmp1_b32 65472, s88\n\t" \
  "s_cselect_b64 exec, 1, 0\n\t" \
  "global_atomic_add v38, v31, s[54:55] sc1\n\t" \
  "v_add_u32 v38, s31, v38\n\t" \
  "s_mov_b64 exec, -1\n\t" \
  "s_cselect_b32 s98, s80, 0\n\t" \
  "v_mfma_f32_16x16x32_bf16 a[36:39], a[136:139], v[72:75], a[36:39]\n\t" \
  "ds_read_b64_tr_b16 a[212:213], v14 offset:40960\n\t" \
  "s_xor_b32 s31, s31, s98\n\t" \
  "v_mfma_f32_16x16x32_bf16 a[36:39], a[140:143], v[76:79], a[36:39]\n\t" \
  "s_and_b32 s87, s26, 15\n\t" \
  "s_sub_u32 s28, s87, 2\n\t" \
  "v_mfma_f32_16x16x32_bf16 a[32:35], a[132:135], v[76:79], a[32:35]\n\t" \
  "ds_read_b64_tr_b16 a[214:215], v14 offset:41984\n\t" \
  "v_mfma_f32_16x16x32_bf16 a[32:35], a[128:131], v[72:75], a[32:35]\n\t" \
  "ds_read_b128 v[72:75], v10 offset:34816\n\t" \
  "ds_read_b128 v[76:79], v11 offset:34816\n\t" \
  "v_mfma_f32_16x16x32_bf16 a[64:67], a[128:131], v[80:83], a[64:67]\n\t" \
  "ds_read_b64_tr_b16 a[216:217], v15 offset:32768\n\t" \
  "v_mfma_f32_16x16x32_bf16 a[64:67], a[132:135], v[84:87], a[64:67]\n\t" \
  "s_bitcmp1_b32 32799, s88\n\t" \
  "s_cselect_b32 s87, 1, s87\n\t" \
  "v_mfma_f32_16x16x32_bf16 a[68:71], a[140:143], v[84:87], a[68:71]\n\t" \
  "ds_read_b64_tr_b16 a[218:219], v15 offset:33792\n\t" \
  "s_cselect_b32 s30, 1, 0\n\t" \
  "v_mfma_f32_16x16x32_bf16 a[68:71], a[136:139], v[80:83], a[68:71]\n\t" \
  "s_lshr_b64 s[26:27], s[26:27], 4\n\t" \
  "v_mfma_f32_16x16x32_bf16 a[72:75], a[144:147], v[80:83], a[72:75]\n\t" \
  "ds_read_b64_tr_b16 a[220:221], v15 offset:40960\n\t" \
  "v_mfma_f32_16x16x32_bf16 a[72:75], a[148:151], v[84:87], a[72:75]\n\t" \
  "s_cmp_eq_u32 s88, 15\n\t" \
  "s_cselect_b32 s74, 0, s74\n\t" \
  "v_mfma_f32_16x16x32_bf16 a[76:79], a[156:159], v[84:87], a[76:79]\n\t" \
  "ds_read_b64_tr_b16 a[222:223], v15 offset:41984\n\t" \
  "v_add_u32 v8, s74, v8\n\t" \
  "v_mfma_f32_16x16x32_bf16 a[76:79], a[152:155], v[80:83], a[76:79]\n\t" \
  "s_add_u32 m0, s68, 16384\n\t" \
  "global_load_dwordx4 v[32:35], v8, s[48:49] nt\n\t" \
  "v_mfma_f32_16x16x32_bf16 a[80:83], a[160:163], v[80:83], a[80:83]\n\t" \
  "buffer_load_dwordx4 v0, s[36:39], s65 offen offset:0 sc0 sc1 lds\n\t" \
  "s_lshl_b32 s98, s74, 10\n\t" \
  "v_mfma_f32_16x16x32_bf16 a[80:83], a[164:167], v[84:87], a[80:83]\n\t" \
  "v_add_u32 v39, s98, v39\n\t" \
  "s_xor_b32 s74, s74, s80\n\t" \
  "v_mfma_f32_16x16x32_bf16 a[84:87], a[172:175], v[84:87], a[84:87]\n\t" \
  "s_add_u32 s88, s88, 1\n\t" \
  "s_branch RET31_%=\n\t" \
  ".p2align 6\n\t" \
  "PUB32_%=:\n\t" \
  "s_mov_b64 exec, 1\n\t" \
  "global_atomic_add v38, v31, s[54:55] sc1\n\t" \
  "s_mov_b64 exec, -1\n\t" \
  "v_mfma_f32_16x16x32_bf16 a[40:43], a[148:151], v[76:79], a[40:43]\n\t" \
  "ds_read_b64_tr_b16 a[210:211], v14 offset:33792\n\t" \
  "v_add_u32 v38, s31, v38\n\t" \
  "v_mfma_f32_16x16x32_bf16 a[40:43], a[144:147], v[72:75], a[40:43]\n\t" \
  "s_xor_b32 s31, s31, s80\n\t" \
  "s_mov_b32 s30, 0\n\t" \
  "v_mfma_f32_16x16x32_bf16 a[36:39], a[136:139], v[72:75], a[36:39]\n\t" \
  "ds_read_b64_tr_b16 a[212:213], v14 offset:40960\n\t" \
  "s_add_u32 s28, s28, s72\n\t" \
  "v_mfma_f32_16x16x32_bf16 a[36:39], a[140:143], v[76:79], a[36:39]\n\t" \
  "s_cmp_eq_u32 s88, 16\n\t" \
  "s_cselect_b32 s87, 0x7fffffff, s28\n\t" \
  "v_mfma_f32_16x16x32_bf16 a[32:35], a[132:135], v[76:79], a[32:35]\n\t" \
  "ds_read_b64_tr_b16 a[214:215], v14 offset:41984\n\t" \
  "s_mov_b32 s72, s73\n\t" \
  "v_mfma_f32_16x16x32_bf16 a[32:35], a[128:131], v[72:75], a[32:35]\n\t" \
  "ds_read_b128 v[72:75], v10 offset:34816\n\t" \
  "ds_read_b128 v[76:79], v11 offset:34816\n\t" \
  "v_mfma_f32_16x16x32_bf16 a[64:67], a[128:131], v[80:83], a[64:67]\n\t" \
  "ds_read_b64_tr_b16 a[216:217], v15 offset:32768\n\t" \
  "s_mov_b32 s73, 0\n\t" \
  "v_mfma_f32_16x16x32_bf16 a[64:67], a[132:135], v[84:87], a[64:67]\n\t" \
  "v_mfma_f32_16x16x32_bf16 a[68:71], a[140:143], v[84:87], a[68:71]\n\t" \
  "ds_read_b64_tr_b16 a[218:219], v15 offset:33792\n\t" \
  "v_mfma_f32_16x16x32_bf16 a[68:71], a[136:139], v[80:83], a[68:71]\n\t" \
  "v_mfma_f32_16x16x32_bf16 a[72:75], a[144:147], v[80:83], a[72:75]\n\t" \
  "ds_read_b64_tr_b16 a[220:221], v15 offset:40960\n\t" \
  "v_mfma_f32_16x16x32_bf16 a[72:75], a[148:151], v[84:87], a[72:75]\n\t" \
  "v_mfma_f32_16x16x32_bf16 a[76:79], a[156:159], v[84:87], a[76:79]\n\t" \
  "ds_read_b64_tr_b16 a[222:223], v15 offset:41984\n\t" \
  "v_mfma_f32_16x16x32_bf16 a[76:79], a[152:155], v[80:83], a[76:79]\n\t" \
  "s_add_u32 m0, s68, 16384\n\t" \
  "v_mfma_f32_16x16x32_bf16 a[80:83], a[160:163], v[80:83], a[80:83]\n\t" \
  "buffer_load_dwordx4 v0, s[36:39], s65 offen offset:0 sc0 sc1 lds\n\t" \
  "v_mfma_f32_16x16x32_bf16 a[80:83], a[164:167], v[84:87], a[80:83]\n\t" \
  "v_mfma_f32_16x16x32_bf16 a[84:87], a[172:175], v[84:87], a[84:87]\n\t" \
  "s_branch RET31_%=\n\t" \
  "SW33_%=:\n\t" \
  "s_mov_b32 s36, s56\n\t" \
  "s_mov_b32 s37, s57\n\t" \
  "s_mov_b32 s38, s63\n\t" \
  "s_mov_b32 s67, 1024\n\t" \
  "s_mov_b32 s89, 0\n\t" \
  "s_lshr_b32 s69, s69, 4\n\t" \
  "s_lshr_b32 s70, s70, 4\n\t" \
  "v_lshrrev_b32 v1, 11, v0\n\t" \
  "v_and_b32 v0, 0x7f, v0\n\t" \
  "v_lshl_add_u32 v0, v1, 7, v0\n\t" \
  "v_xor_b32 v1, 64, v0\n\t" \
  "v_mov_b32 v2, v0\n\t" \
  "v_mov_b32 v3, v1\n\t" \
  "s_branch RET34_%=\n\t" \
  "IDL37_%=:\n\t" \
  "s_sleep 4\n\t" \
  "s_branch IDLR38_%=\n\t" \
  ".p2align 6\n\t" \
  "EVT35_%=:\n\t" \
  "s_cmp_lg_u32 s30, 0\n\t" \
  "s_cbranch_scc1 PUB39_%=\n\t" \
  "global_store_dwordx4 v39, v[32:35], s[52:53] sc0 sc1\n\t" \
  "v_mfma_f32_16x16x32_bf16 v[168:171], a[148:151], v[108:111], v[168:171]\n\t" \
  "ds_read_b64_tr_b16 a[242:243], v14 offset:50176\n\t" \
  "v_mfma_f32_16x16x32_bf16 v[168:171], a[144:147], v[104:107], v[168:171]\n\t" \
  "s_bitcmp1_b32 65472, s88\n\t" \
  "s_cselect_b64 exec, 1, 0\n\t" \
  "global_atomic_add v38, v31, s[54:55] sc1\n\t" \
  "v_add_u32 v38, s31, v38\n\t" \
  "s_mov_b64 exec, -1\n\t" \
  "s_cselect_b32 s98, s80, 0\n\t" \
  "v_mfma_f32_16x16x32_bf16 v[164:167], a[136:139], v[104:107], v[164:167]\n\t" \
  "ds_read_b64_tr_b16 a[244:245], v14 offset:57344\n\t" \
  "s_xor_b32 s31, s31, s98\n\t" \
  "v_mfma_f32_16x16x32_bf16 v[164:167], a[140:143], v[108:111], v[164:167]\n\t" \
  "s_and_b32 s87, s26, 15\n\t" \
  "s_sub_u32 s28, s87, 2\n\t" \
  "v_mfma_f32_16x16x32_bf16 v[160:163], a[132:135], v[108:111], v[160:163]\n\t" \
  "ds_read_b64_tr_b16 a[246:247], v14 offset:58368\n\t" \
  "v_mfma_f32_16x16x32_bf16 v[160:163], a[128:131], v[104:107], v[160:163]\n\t" \
  "ds_read_b128 v[104:107], v10 offset:51200\n\t" \
  "ds_read_b128 v[108:111], v11 offset:51200\n\t" \
  "v_mfma_f32_16x16x32_bf16 v[192:195], a[128:131], v[112:115], v[192:195]\n\t" \
  "ds_read_b64_tr_b16 a[248:249], v15 offset:49152\n\t" \
  "v_mfma_f32_16x16x32_bf16 v[192:195], a[132:135], v[116:119], v[192:195]\n\t" \
  "s_bitcmp1_b32 32799, s88\n\t" \
  "s_cselect_b32 s87, 1, s87\n\t" \
  "v_mfma_f32_16x16x32_bf16 v[196:199], a[140:143], v[116:119], v[196:199]\n\t" \
  "ds_read_b64_tr_b16 a[250:251], v15 offset:50176\n\t" \
  "s_cselect_b32 s30, 1, 0\n\t" \
  "v_mfma_f32_16x16x32_bf16 v[196:199], a[136:139], v[112:115], v[196:199]\n\t" \
  "s_lshr_b64 s[26:27], s[26:27], 4\n\t" \
  "v_mfma_f32_16x16x32_bf16 v[200:203], a[144:147], v[112:115], v[200:203]\n\t" \
  "ds_read_b64_tr_b16 a[252:253], v15 offset:57344\n\t" \
  "v_mfma_f32_16x16x32_bf16 v[200:203], a[148:151], v[116:119], v[200:203]\n\t" \
  "s_cmp_eq_u32 s88, 15\n\t" \
  "s_cselect_b32 s74, 0, s74\n\t" \
  "v_mfma_f32_16x16x32_bf16 v[204:207], a[156:159], v[116:119], v[204:207]\n\t" \
  "ds_read_b64_tr_b16 a[254:255], v15 offset:58368\n\t" \
  "v_add_u32 v8, s74, v8\n\t" \
  "v_mfma_f32_16x16x32_bf16 v[204:207], a[152:155], v[112:115], v[204:207]\n\t" \
  "s_add_u32 m0, s68, 32768\n\t" \
  "global_load_dwordx4 v[32:35], v8, s[48:49] nt\n\t" \
  "v_mfma_f32_16x16x32_bf16 v[208:211], a[160:163], v[112:115], v[208:211]\n\t" \
  "buffer_load_dwordx4 v0, s[36:39], s64 offen offset:0 sc0 sc1 lds\n\t" \
  "s_lshl_b32 s98, s74, 10\n\t" \
  "v_mfma_f32_16x16x32_bf16 v[208:211], a[164:167], v[116:119], v[208:211]\n\t" \
  "v_add_u32 v39, s98, v39\n\t" \
  "s_xor_b32 s74, s74, s80\n\t" \
  "v_mfma_f32_16x16x32_bf16 v[212:215], a[172:175], v[116:119], v[212:215]\n\t" \
  "s_add_u32 s88, s88, 1\n\t" \
  "s_branch RET36_%=\n\t" \
  ".p2align 6\n\t" \
  "PUB39_%=:\n\t" \
  "s_mov_b64 exec, 1\n\t" \
  "global_atomic_add v38, v31, s[54:55] sc1\n\t" \
  "s_mov_b64 exec, -1\n\t" \
  "v_mfma_f32_16x16x32_bf16 v[168:171], a[148:151], v[108:111], v[168:171]\n\t" \
  "ds_read_b64_tr_b16 a[242:243], v14 offset:50176\n\t" \
  "v_add_u32 v38, s31, v38\n\t" \
  "v_mfma_f32_16x16x32_bf16 v[168:171], a[144:147], v[104:107], v[168:171]\n\t" \
  "s_xor_b32 s31, s31, s80\n\t" \
  "s_mov_b32 s30, 0\n\t" \
  "v_mfma_f32_16x16x32_bf16 v[164:167], a[136:139], v[104:107], v[164:167]\n\t" \
  "ds_read_b64_tr_b16 a[244:245], v14 offset:57344\n\t" \
  "s_add_u32 s28, s28, s72\n\t" \
  "v_mfma_f32_16x16x32_bf16 v[164:167], a[140:143], v[108:111], v[164:167]\n\t" \
  "s_cmp_eq_u32 s88, 16\n\t" \
  "s_cselect_b32 s87, 0x7fffffff, s28\n\t" \
  "v_mfma_f32_16x16x32_bf16 v[160:163], a[132:135], v[108:111], v[160:163]\n\t" \
  "ds_read_b64_tr_b16 a[246:247], v14 offset:58368\n\t" \
  "s_mov_b32 s72, s73\n\t" \
  "v_mfma_f32_16x16x32_bf16 v[160:163], a[128:131], v[104:107], v[160:163]\n\t" \
  "ds_read_b128 v[104:107], v10 offset:51200\n\t" \
  "ds_read_b128 v[108:111], v11 offset:51200\n\t" \
  "v_mfma_f32_16x16x32_bf16 v[192:195], a[128:131], v[112:115], v[192:195]\n\t" \
  "ds_read_b64_tr_b16 a[248:249], v15 offset:49152\n\t" \
  "s_mov_b32 s73, 0\n\t" \
  "v_mfma_f32_16x16x32_bf16 v[192:195], a[132:135], v[116:119], v[192:195]\n\t" \
  "v_mfma_f32_16x16x32_bf16 v[196:199], a[140:143], v[116:119], v[196:199]\n\t" \
  "ds_read_b64_tr_b16 a[250:251], v15 offset:50176\n\t" \
  "v_mfma_f32_16x16x32_bf16 v[196:199], a[136:139], v[112:115], v[196:199]\n\t" \
  "v_mfma_f32_16x16x32_bf16 v[200:203], a[144:147], v[112:115], v[200:203]\n\t" \
  "ds_read_b64_tr_b16 a[252:253], v15 offset:57344\n\t" \
  "v_mfma_f32_16x16x32_bf16 v[200:203], a[148:151], v[116:119], v[200:203]\n\t" \
  "v_mfma_f32_16x16x32_bf16 v[204:207], a[156:159], v[116:119], v[204:207]\n\t" \
  "ds_read_b64_tr_b16 a[254:255], v15 offset:58368\n\t" \
  "v_mfma_f32_16x16x32_bf16 v[204:207], a[152:155], v[112:115], v[204:207]\n\t" \
  "s_add_u32 m0, s68, 32768\n\t" \
  "v_mfma_f32_16x16x32_bf16 v[208:211], a[160:163], v[112:115], v[208:211]\n\t" \
  "buffer_load_dwordx4 v0, s[36:39], s64 offen offset:0 sc0 sc1 lds\n\t" \
  "v_mfma_f32_16x16x32_bf16 v[208:211], a[164:167], v[116:119], v[208:211]\n\t" \
  "v_mfma_f32_16x16x32_bf16 v[212:215], a[172:175], v[116:119], v[212:215]\n\t" \
  "s_branch RET36_%=\n\t" \
  "SLOW43_%=:\n\t" \
  "s_memrealtime s[94:95]\n\t" \
  "s_waitcnt lgkmcnt(0)\n\t" \
  "SL44_%=:\n\t" \
  "v_mov_b32 v46, 0\n\t" \
  "s_lshl_b64 exec, 1, s99\n\t" \
  "global_load_dword v46, v16, s[50:51] sc0 sc1\n\t" \
  "s_mov_b64 exec, 1\n\t" \
  "global_load_dword v47, v30, s[50:51] offset:-128 sc1\n\t" \
  "s_mov_b64 exec, -1\n\t" \
  "s_waitcnt vmcnt(0)\n\t" \
  "v_cmp_le_u32 s[92:93], s86, v46\n\t" \
  "v_readfirstlane_b32 s98, v47\n\t" \
  "s_nop 4\n\t" \
  "s_or_b64 s[90:91], s[90:91], s[92:93]\n\t" \
  "s_bitcmp1_b64 s[90:91], s99\n\t" \
  "s_cbranch_scc1 RET42_%=\n\t" \
  "s_cmp_eq_u32 s98, s85\n\t" \
  "s_cbranch_scc1 SB45_%=\n\t" \
  "s_memrealtime s[96:97]\n\t" \
  "s_waitcnt lgkmcnt(0)\n\t" \
  "s_sub_u32 s96, s96, s94\n\t" \
  "s_cmp_gt_u32 s96, 50000000\n\t" \
  "s_cbranch_scc1 SB45_%=\n\t" \
  "s_sleep 2\n\t" \
  "s_branch SL44_%=\n\t" \
  "SB45_%=:\n\t" \
  "v_mov_b32 v47, s85\n\t" \
  "global_store_dword v30, v47, s[50:51] offset:-128 sc1\n\t" \
  "s_mov_b64 s[90:91], -1\n\t" \
  "s_mov_b64 s[58:59], -1\n\t" \
  "s_waitcnt vmcnt(0)\n\t" \
  "s_branch RET42_%=\n\t" \
  "POLL47_%=:\n\t" \
  "s_lshl_b64 exec, 1, s99\n\t" \
  "global_load_dword v36, v16, s[50:51] sc0 sc1\n\t" \
  "s_mov_b64 exec, -1\n\t" \
  "s_branch RET46_%=\n\t" \
  "STG48_%=:\n\t" \
  "s_waitcnt vmcnt(5)\n\t" \
  "v_cmp_le_u32 s[92:93], s86, v36\n\t" \
  "s_nop 3\n\t" \
  "s_or_b64 s[90:91], s[90:91], s[92:93]\n\t" \
  "s_bitcmp1_b64 s[90:91], s99\n\t" \
  "s_cbranch_scc1 RET42_%=\n\t" \
  "s_branch SLOW43_%=\n\t" \
  ".p2align 6\n\t" \
  "EVT49_%=:\n\t" \
  "s_cmp_lg_u32 s30, 0\n\t" \
  "s_cbranch_scc1 PUB51_%=\n\t" \
  "global_store_dwordx4 v39, v[32:35], s[52:53] sc0 sc1\n\t" \
  "v_mfma_f32_16x16x32_bf16 a[40:43], a[212:215], v[76:79], a[40:43]\n\t" \
  "ds_read_b64_tr_b16 a[146:147], v14 offset:1024\n\t" \
  "v_mfma_f32_16x16x32_bf16 a[40:43], a[208:211], v[72:75], a[40:43]\n\t" \
  "s_bitcmp1_b32 65472, s88\n\t" \
  "s_cselect_b64 exec, 1, 0\n\t" \
  "global_atomic_add v38, v31, s[54:55] sc1\n\t" \
  "v_add_u32 v38, s31, v38\n\t" \
  "s_mov_b64 exec, -1\n\t" \
  "s_cselect_b32 s98, s80, 0\n\t" \
  "v_mfma_f32_16x16x32_bf16 a[36:39], a[200:203], v[72:75], a[36:39]\n\t" \
  "ds_read_b64_tr_b16 a[148:149], v14 offset:8192\n\t" \
  "s_xor_b32 s31, s31, s98\n\t" \
  "v_mfma_f32_16x16x32_bf16 a[36:39], a[204:207], v[76:79], a[36:39]\n\t" \
  "s_and_b32 s87, s26, 15\n\t" \
  "s_sub_u32 s28, s87, 2\n\t" \
  "v_mfma_f32_16x16x32_bf16 a[32:35], a[196:199], v[76:79], a[32:35]\n\t" \
  "ds_read_b64_tr_b16 a[150:151], v14 offset:9216\n\t" \
  "v_mfma_f32_16x16x32_bf16 a[32:35], a[192:195], v[72:75], a[32:35]\n\t" \
  "ds_read_b128 v[72:75], v10 offset:2048\n\t" \
  "ds_read_b128 v[76:79], v11 offset:2048\n\t" \
  "v_mfma_f32_16x16x32_bf16 a[64:67], a[192:195], v[80:83], a[64:67]\n\t" \
  "ds_read_b64_tr_b16 a[152:153], v15 offset:0\n\t" \
  "v_mfma_f32_16x16x32_bf16 a[64:67], a[196:199], v[84:87], a[64:67]\n\t" \
  "s_bitcmp1_b32 32799, s88\n\t" \
  "s_cselect_b32 s87, 1, s87\n\t" \
  "v_mfma_f32_16x16x32_bf16 a[68:71], a[204:207], v[84:87], a[68:71]\n\t" \
  "ds_read_b64_tr_b16 a[154:155], v15 offset:1024\n\t" \
  "s_cselect_b32 s30, 1, 0\n\t" \
  "v_mfma_f32_16x16x32_bf16 a[68:71], a[200:203], v[80:83], a[68:71]\n\t" \
  "s_lshr_b64 s[26:27], s[26:27], 4\n\t" \
  "v_mfma_f32_16x16x32_bf16 a[72:75], a[208:211], v[80:83], a[72:75]\n\t" \
  "ds_read_b64_tr_b16 a[156:157], v15 offset:8192\n\t" \
  "v_mfma_f32_16x16x32_bf16 a[72:75], a[212:215], v[84:87], a[72:75]\n\t" \
  "s_cmp_eq_u32 s88, 15\n\t" \
  "s_cselect_b32 s74, 0, s74\n\t" \
  "v_mfma_f32_16x16x32_bf16 a[76:79], a[220:223], v[84:87], a[76:79]\n\t" \
  "ds_read_b64_tr_b16 a[158:159], v15 offset:9216\n\t" \
  "v_add_u32 v8, s74, v8\n\t" \
  "v_mfma_f32_16x16x32_bf16 a[76:79], a[216:219], v[80:83], a[76:79]\n\t" \
  "s_add_u32 m0, s68, 49152\n\t" \
  "global_load_dwordx4 v[32:35], v8, s[48:49] nt\n\t" \
  "v_mfma_f32_16x16x32_bf16 a[80:83], a[224:227], v[80:83], a[80:83]\n\t" \
  "buffer_load_dwordx4 v0, s[36:39], s65 offen offset:0 sc0 sc1 lds\n\t" \
  "s_lshl_b32 s98, s74, 10\n\t" \
  "v_mfma_f32_16x16x32_bf16 a[80:83], a[228:231], v[84:87], a[80:83]\n\t" \
  "v_add_u32 v39, s98, v39\n\t" \
  "s_xor_b32 s74, s74, s80\n\t" \
  "v_mfma_f32_16x16x32_bf16 a[84:87], a[236:239], v[84:87], a[84:87]\n\t" \
  "s_add_u32 s88, s88, 1\n\t" \
  "s_branch RET50_%=\n\t" \
  ".p2align 6\n\t" \
  "PUB51_%=:\n\t" \
  "s_mov_b64 exec, 1\n\t" \
  "global_atomic_add v38, v31, s[54:55] sc1\n\t" \
  "s_mov_b64 exec, -1\n\t" \
  "v_mfma_f32_16x16x32_bf16 a[40:43], a[212:215], v[76:79], a[40:43]\n\t" \
  "ds_read_b64_tr_b16 a[146:147], v14 offset:1024\n\t" \
  "v_add_u32 v38, s31, v38\n\t" \
  "v_mfma_f32_16x16x32_bf16 a[40:43], a[208:211], v[72:75], a[40:43]\n\t" \
  "s_xor_b32 s31, s31, s80\n\t" \
  "s_mov_b32 s30, 0\n\t" \
  "v_mfma_f32_16x16x32_bf16 a[36:39], a[200:203], v[72:75], a[36:39]\n\t" \
  "ds_read_b64_tr_b16 a[148:149], v14 offset:8192\n\t" \
  "s_add_u32 s28, s28, s72\n\t" \
  "v_mfma_f32_16x16x32_bf16 a[36:39], a[204:207], v[76:79], a[36:39]\n\t" \
  "s_cmp_eq_u32 s88, 16\n\t" \
  "s_cselect_b32 s87, 0x7fffffff, s28\n\t" \
  "v_mfma_f32_16x16x32_bf16 a[32:35], a[196:199], v[76:79], a[32:35]\n\t" \
  "ds_read_b64_tr_b16 a[150:151], v14 offset:9216\n\t" \
  "s_mov_b32 s72, s73\n\t" \
  "v_mfma_f32_16x16x32_bf16 a[32:35], a[192:195], v[72:75], a[32:35]\n\t" \
  "ds_read_b128 v[72:75], v10 offset:2048\n\t" \
  "ds_read_b128 v[76:79], v11 offset:2048\n\t" \
  "v_mfma_f32_16x16x32_bf16 a[64:67], a[192:195], v[80:83], a[64:67]\n\t" \
  "ds_read_b64_tr_b16 a[152:153], v15 offset:0\n\t" \
  "s_mov_b32 s73, 0\n\t" \
  "v_mfma_f32_16x16x32_bf16 a[64:67], a[196:199], v[84:87], a[64:67]\n\t" \
  "v_mfma_f32_16x16x32_bf16 a[68:71], a[204:207], v[84:87], a[68:71]\n\t" \
  "ds_read_b64_tr_b16 a[154:155], v15 offset:1024\n\t" \
  "v_mfma_f32_16x16x32_bf16 a[68:71], a[200:203], v[80:83], a[68:71]\n\t" \
  "v_mfma_f32_16x16x32_bf16 a[72:75], a[208:211], v[80:83], a[72:75]\n\t" \
  "ds_read_b64_tr_b16 a[156:157], v15 offset:8192\n\t" \
  "v_mfma_f32_16x16x32_bf16 a[72:75], a[212:215], v[84:87], a[72:75]\n\t" \
  "v_mfma_f32_16x16x32_bf16 a[76:79], a[220:223], v[84:87], a[76:79]\n\t" \
  "ds_read_b64_tr_b16 a[158:159], v15 offset:9216\n\t" \
  "v_mfma_f32_16x16x32_bf16 a[76:79], a[216:219], v[80:83], a[76:79]\n\t" \
  "s_add_u32 m0, s68, 49152\n\t" \
  "v_mfma_f32_16x16x32_bf16 a[80:83], a[224:227], v[80:83], a[80:83]\n\t" \
  "buffer_load_dwordx4 v0, s[36:39], s65 offen offset:0 sc0 sc1 lds\n\t" \
  "v_mfma_f32_16x16x32_bf16 a[80:83], a[228:231], v[84:87], a[80:83]\n\t" \
  "v_mfma_f32_16x16x32_bf16 a[84:87], a[236:239], v[84:87], a[84:87]\n\t" \
  "s_branch RET50_%=\n\t" \
  "SW52_%=:\n\t" \
  "s_mov_b32 s36, s56\n\t" \
  "s_mov_b32 s37, s57\n\t" \
  "s_mov_b32 s38, s63\n\t" \
  "s_mov_b32 s67, 1024\n\t" \
  "s_mov_b32 s89, 0\n\t" \
  "s_lshr_b32 s69, s69, 4\n\t" \
  "s_lshr_b32 s70, s70, 4\n\t" \
  "v_lshrrev_b32 v1, 11, v0\n\t" \
  "v_and_b32 v0, 0x7f, v0\n\t" \
  "v_lshl_add_u32 v0, v1, 7, v0\n\t" \
  "v_xor_b32 v1, 64, v0\n\t" \
  "v_mov_b32 v2, v0\n\t" \
  "v_mov_b32 v3, v1\n\t" \
  "s_branch RET53_%=\n\t" \
  "AD56_%=:\n\t" \
  "s_cmp_lt_u32 s76, 22\n\t" \
  "s_cbranch_scc1 ADS58_%=\n\t" \
  "s_cmp_eq_u32 s76, 26\n\t" \
  "s_cbranch_scc1 ADR57_%=\n\t" \
  "s_mov_b32 s16, s24\n\t" \
  "s_mov_b32 s17, s25\n\t" \
  "s_branch ADR57_%=\n\t" \
  "ADS58_%=:\n\t" \
  "s_cmp_lt_u32 s18, s13\n\t" \
  "s_cselect_b32 s19, 2, s19\n\t" \
  "s_sub_u32 s19, s19, 1\n\t" \
  "s_cmp_lt_u32 s76, 4\n\t" \
  "s_cbranch_scc1 ADR57_%=\n\t" \
  "s_sleep 12\n\t" \
  "s_branch ADR57_%=\n\t" \
  "TRG59_%=:\n\t" \
  "s_cmp_eq_u32 s76, 4\n\t" \
  "s_cbranch_scc1 TRGL61_%=\n\t" \
  "s_memrealtime s[96:97]\n\t" \
  "s_waitcnt lgkmcnt(0)\n\t" \
  "v_mov_b32 v46, s96\n\t" \
  "v_mov_b32 v47, s85\n\t" \
  "s_mov_b64 exec, 1\n\t" \
  "global_store_dwordx2 v22, v[46:47], s[50:51] sc0 sc1\n\t" \
  "s_mov_b64 exec, -1\n\t" \
  "s_branch TRGR60_%=\n\t" \
  "TRGL61_%=:\n\t" \
  "s_mov_b64 exec, 3\n\t" \
  "global_load_dwordx2 v[20:21], v22, s[50:51] sc0 sc1\n\t" \
  "s_mov_b64 exec, -1\n\t" \
  "s_branch TRGR60_%=\n\t" \
  ".p2align 6\n\t" \
  "EVT54_%=:\n\t" \
  "s_cmp_lg_u32 s30, 0\n\t" \
  "s_cbranch_scc1 PUB62_%=\n\t" \
  "global_store_dwordx4 v39, v[32:35], s[52:53] sc0 sc1\n\t" \
  "v_mfma_f32_16x16x32_bf16 v[168:171], a[212:215], v[108:111], v[168:171]\n\t" \
  "ds_read_b64_tr_b16 a[178:179], v14 offset:17408\n\t" \
  "v_mfma_f32_16x16x32_bf16 v[168:171], a[208:211], v[104:107], v[168:171]\n\t" \
  "s_bitcmp1_b32 65472, s88\n\t" \
  "s_cselect_b64 exec, 1, 0\n\t" \
  "global_atomic_add v38, v31, s[54:55] sc1\n\t" \
  "v_add_u32 v38, s31, v38\n\t" \
  "s_mov_b64 exec, -1\n\t" \
  "s_cselect_b32 s98, s80, 0\n\t" \
  "v_mfma_f32_16x16x32_bf16 v[164:167], a[200:203], v[104:107], v[164:167]\n\t" \
  "ds_read_b64_tr_b16 a[180:181], v14 offset:24576\n\t" \
  "s_xor_b32 s31, s31, s98\n\t" \
  "v_mfma_f32_16x16x32_bf16 v[164:167], a[204:207], v[108:111], v[164:167]\n\t" \
  "s_and_b32 s87, s26, 15\n\t" \
  "s_sub_u32 s28, s87, 2\n\t" \
  "v_mfma_f32_16x16x32_bf16 v[160:163], a[196:199], v[108:111], v[160:163]\n\t" \
  "ds_read_b64_tr_b16 a[182:183], v14 offset:25600\n\t" \
  "v_mfma_f32_16x16x32_bf16 v[160:163], a[192:195], v[104:107], v[160:163]\n\t" \
  "ds_read_b128 v[104:107], v10 offset:18432\n\t" \
  "ds_read_b128 v[108:111], v11 offset:18432\n\t" \
  "v_mfma_f32_16x16x32_bf16 v[192:195], a[192:195], v[112:115], v[192:195]\n\t" \
  "ds_read_b64_tr_b16 a[184:185], v15 offset:16384\n\t" \
  "v_mfma_f32_16x16x32_bf16 v[192:195], a[196:199], v[116:119], v[192:195]\n\t" \
  "s_bitcmp1_b32 32799, s88\n\t" \
  "s_cselect_b32 s87, 1, s87\n\t" \
  "v_mfma_f32_16x16x32_bf16 v[196:199], a[204:207], v[116:119], v[196:199]\n\t" \
  "ds_read_b64_tr_b16 a[186:187], v15 offset:17408\n\t" \
  "s_cselect_b32 s30, 1, 0\n\t" \
  "v_mfma_f32_16x16x32_bf16 v[196:199], a[200:203], v[112:115], v[196:199]\n\t" \
  "s_lshr_b64 s[26:27], s[26:27], 4\n\t" \
  "v_mfma_f32_16x16x32_bf16 v[200:203], a[208:211], v[112:115], v[200:203]\n\t" \
  "ds_read_b64_tr_b16 a[188:189], v15 offset:24576\n\t" \
  "v_mfma_f32_16x16x32_bf16 v[200:203], a[212:215], v[116:119], v[200:203]\n\t" \
  "s_cmp_eq_u32 s88, 15\n\t" \
  "s_cselect_b32 s74, 0, s74\n\t" \
  "v_mfma_f32_16x16x32_bf16 v[204:207], a[220:223], v[116:119], v[204:207]\n\t" \
  "ds_read_b64_tr_b16 a[190:191], v15 offset:25600\n\t" \
  "v_add_u32 v8, s74, v8\n\t" \
  "v_mfma_f32_16x16x32_bf16 v[204:207], a[216:219], v[112:115], v[204:207]\n\t" \
  "s_add_u32 m0, s68, 0\n\t" \
  "global_load_dwordx4 v[32:35], v8, s[48:49] nt\n\t" \
  "v_mfma_f32_16x16x32_bf16 v[208:211], a[224:227], v[112:115], v[208:211]\n\t" \
  "buffer_load_dwordx4 v0, s[36:39], s64 offen offset:0 sc0 sc1 lds\n\t" \
  "s_lshl_b32 s98, s74, 10\n\t" \
  "v_mfma_f32_16x16x32_bf16 v[208:211], a[228:231], v[116:119], v[208:211]\n\t" \
  "v_add_u32 v39, s98, v39\n\t" \
  "s_xor_b32 s74, s74, s80\n\t" \
  "v_mfma_f32_16x16x32_bf16 v[212:215], a[236:239], v[116:119], v[212:215]\n\t" \
  "s_add_u32 s88, s88, 1\n\t" \
  "s_branch RET55_%=\n\t" \
  ".p2align 6\n\t" \
  "PUB62_%=:\n\t" \
  "s_mov_b64 exec, 1\n\t" \
  "global_atomic_add v38, v31, s[54:55] sc1\n\t" \
  "s_mov_b64 exec, -1\n\t" \
  "v_mfma_f32_16x16x32_bf16 v[168:171], a[212:215], v[108:111], v[168:171]\n\t" \
  "ds_read_b64_tr_b16 a[178:179], v14 offset:17408\n\t" \
  "v_add_u32 v38, s31, v38\n\t" \
  "v_mfma_f32_16x16x32_bf16 v[168:171], a[208:211], v[104:107], v[168:171]\n\t" \
  "s_xor_b32 s31, s31, s80\n\t" \
  "s_mov_b32 s30, 0\n\t" \
  "v_mfma_f32_16x16x32_bf16 v[164:167], a[200:203], v[104:107], v[164:167]\n\t" \
  "ds_read_b64_tr_b16 a[180:181], v14 offset:24576\n\t" \
  "s_add_u32 s28, s28, s72\n\t" \
  "v_mfma_f32_16x16x32_bf16 v[164:167], a[204:207], v[108:111], v[164:167]\n\t" \
  "s_cmp_eq_u32 s88, 16\n\t" \
  "s_cselect_b32 s87, 0x7fffffff, s28\n\t" \
  "v_mfma_f32_16x16x32_bf16 v[160:163], a[196:199], v[108:111], v[160:163]\n\t" \
  "ds_read_b64_tr_b16 a[182:183], v14 offset:25600\n\t" \
  "s_mov_b32 s72, s73\n\t" \
  "v_mfma_f32_16x16x32_bf16 v[160:163], a[192:195], v[104:107], v[160:163]\n\t" \
  "ds_read_b128 v[104:107], v10 offset:18432\n\t" \
  "ds_read_b128 v[108:111], v11 offset:18432\n\t" \
  "v_mfma_f32_16x16x32_bf16 v[192:195], a[192:195], v[112:115], v[192:195]\n\t" \
  "ds_read_b64_tr_b16 a[184:185], v15 offset:16384\n\t" \
  "s_mov_b32 s73, 0\n\t" \
  "v_mfma_f32_16x16x32_bf16 v[192:195], a[196:199], v[116:119], v[192:195]\n\t" \
  "v_mfma_f32_16x16x32_bf16 v[196:199], a[204:207], v[116:119], v[196:199]\n\t" \
  "ds_read_b64_tr_b16 a[186:187], v15 offset:17408\n\t" \
  "v_mfma_f32_16x16x32_bf16 v[196:199], a[200:203], v[112:115], v[196:199]\n\t" \
  "v_mfma_f32_16x16x32_bf16 v[200:203], a[208:211], v[112:115], v[200:203]\n\t" \
  "ds_read_b64_tr_b16 a[188:189], v15 offset:24576\n\t" \
  "v_mfma_f32_16x16x32_bf16 v[200:203], a[212:215], v[116:119], v[200:203]\n\t" \
  "v_mfma_f32_16x16x32_bf16 v[204:207], a[220:223], v[116:119], v[204:207]\n\t" \
  "ds_read_b64_tr_b16 a[190:191], v15 offset:25600\n\t" \
  "v_mfma_f32_16x16x32_bf16 v[204:207], a[216:219], v[112:115], v[204:207]\n\t" \
  "s_add_u32 m0, s68, 0\n\t" \
  "v_mfma_f32_16x16x32_bf16 v[208:211], a[224:227], v[112:115], v[208:211]\n\t" \
  "buffer_load_dwordx4 v0, s[36:39], s64 offen offset:0 sc0 sc1 lds\n\t" \
  "v_mfma_f32_16x16x32_bf16 v[208:211], a[228:231], v[116:119], v[208:211]\n\t" \
  "v_mfma_f32_16x16x32_bf16 v[212:215], a[236:239], v[116:119], v[212:215]\n\t" \
  "s_branch RET55_%=\n\t" \
  ".p2align 6\n\t" \
  "EVT63_%=:\n\t" \
  "s_cmp_lg_u32 s30, 0\n\t" \
  "s_cbranch_scc1 PUB65_%=\n\t" \
  "global_store_dwordx4 v39, v[32:35], s[52:53] sc0 sc1\n\t" \
  "v_mfma_f32_16x16x32_bf16 a[40:43], a[148:151], v[76:79], a[40:43]\n\t" \
  "ds_read_b64_tr_b16 a[210:211], v14 offset:33792\n\t" \
  "s_and_b32 s87, s26, 15\n\t" \
  "v_mfma_f32_16x16x32_bf16 a[40:43], a[144:147], v[72:75], a[40:43]\n\t" \
  "s_sub_u32 s28, s87, 2\n\t" \
  "v_mfma_f32_16x16x32_bf16 a[36:39], a[136:139], v[72:75], a[36:39]\n\t" \
  "ds_read_b64_tr_b16 a[212:213], v14 offset:40960\n\t" \
  "v_mfma_f32_16x16x32_bf16 a[36:39], a[140:143], v[76:79], a[36:39]\n\t" \
  "s_bitcmp1_b32 32799, s88\n\t" \
  "s_cselect_b32 s87, 1, s87\n\t" \
  "v_mfma_f32_16x16x32_bf16 a[32:35], a[132:135], v[76:79], a[32:35]\n\t" \
  "ds_read_b64_tr_b16 a[214:215], v14 offset:41984\n\t" \
  "s_cselect_b32 s30, 1, 0\n\t" \
  "v_mfma_f32_16x16x32_bf16 a[32:35], a[128:131], v[72:75], a[32:35]\n\t" \
  "ds_read_b128 v[72:75], v10 offset:34816\n\t" \
  "ds_read_b128 v[76:79], v11 offset:34816\n\t" \
  "v_mfma_f32_16x16x32_bf16 a[64:67], a[128:131], v[80:83], a[64:67]\n\t" \
  "ds_read_b64_tr_b16 a[216:217], v15 offset:32768\n\t" \
  "s_lshr_b64 s[26:27], s[26:27], 4\n\t" \
  "v_mfma_f32_16x16x32_bf16 a[64:67], a[132:135], v[84:87], a[64:67]\n\t" \
  "s_cmp_eq_u32 s88, 15\n\t" \
  "s_cselect_b32 s74, 0, s74\n\t" \
  "v_mfma_f32_16x16x32_bf16 a[68:71], a[140:143], v[84:87], a[68:71]\n\t" \
  "ds_read_b64_tr_b16 a[218:219], v15 offset:33792\n\t" \
  "v_add_u32 v8, s74, v8\n\t" \
  "v_mfma_f32_16x16x32_bf16 a[68:71], a[136:139], v[80:83], a[68:71]\n\t" \
  "global_load_dwordx4 v[32:35], v8, s[48:49] nt\n\t" \
  "s_lshl_b32 s98, s74, 10\n\t" \
  "v_mfma_f32_16x16x32_bf16 a[72:75], a[144:147], v[80:83], a[72:75]\n\t" \
  "ds_read_b64_tr_b16 a[220:221], v15 offset:40960\n\t" \
  "v_add_u32 v39, s98, v39\n\t" \
  "v_mfma_f32_16x16x32_bf16 a[72:75], a[148:151], v[84:87], a[72:75]\n\t" \
  "s_xor_b32 s74, s74, s80\n\t" \
  "s_add_u32 s88, s88, 1\n\t" \
  "v_mfma_f32_16x16x32_bf16 a[76:79], a[156:159], v[84:87], a[76:79]\n\t" \
  "ds_read_b64_tr_b16 a[222:223], v15 offset:41984\n\t" \
  "v_mfma_f32_16x16x32_bf16 a[76:79], a[152:155], v[80:83], a[76:79]\n\t" \
  "s_add_u32 m0, s68, 16384\n\t" \
  "v_mfma_f32_16x16x32_bf16 a[80:83], a[160:163], v[80:83], a[80:83]\n\t" \
  "buffer_load_dwordx4 v0, s[36:39], s65 offen offset:0 sc0 sc1 lds\n\t" \
  "v_mfma_f32_16x16x32_bf16 a[80:83], a[164:167], v[84:87], a[80:83]\n\t" \
  "v_mfma_f32_16x16x32_bf16 a[84:87], a[172:175], v[84:87], a[84:87]\n\t" \
  "s_branch RET64_%=\n\t" \
  ".p2align 6\n\t" \
  "PUB65_%=:\n\t" \
  "v_mfma_f32_16x16x32_bf16 a[40:43], a[148:151], v[76:79], a[40:43]\n\t" \
  "ds_read_b64_tr_b16 a[210:211], v14 offset:33792\n\t" \
  "s_mov_b32 s30, 0\n\t" \
  "v_mfma_f32_16x16x32_bf16 a[40:43], a[144:147], v[72:75], a[40:43]\n\t" \
  "s_add_u32 s28, s28, s72\n\t" \
  "v_mfma_f32_16x16x32_bf16 a[36:39], a[136:139], v[72:75], a[36:39]\n\t" \
  "ds_read_b64_tr_b16 a[212:213], v14 offset:40960\n\t" \
  "v_mfma_f32_16x16x32_bf16 a[36:39], a[140:143], v[76:79], a[36:39]\n\t" \
  "s_cmp_eq_u32 s88, 16\n\t" \
  "s_cselect_b32 s87, 0x7fffffff, s28\n\t" \
  "v_mfma_f32_16x16x32_bf16 a[32:35], a[132:135], v[76:79], a[32:35]\n\t" \
  "ds_read_b64_tr_b16 a[214:215], v14 offset:41984\n\t" \
  "s_mov_b32 s72, s73\n\t" \
  "v_mfma_f32_16x16x32_bf16 a[32:35], a[128:131], v[72:75], a[32:35]\n\t" \
  "ds_read_b128 v[72:75], v10 offset:34816\n\t" \
  "ds_read_b128 v[76:79], v11 offset:34816\n\t" \
  "v_mfma_f32_16x16x32_bf16 a[64:67], a[128:131], v[80:83], a[64:67]\n\t" \
  "ds_read_b64_tr_b16 a[216:217], v15 offset:32768\n\t" \
  "s_mov_b32 s73, 0\n\t" \
  "v_mfma_f32_16x16x32_bf16 a[64:67], a[132:135], v[84:87], a[64:67]\n\t" \
  "v_mfma_f32_16x16x32_bf16 a[68:71], a[140:143], v[84:87], a[68:71]\n\t" \
  "ds_read_b64_tr_b16 a[218:219], v15 offset:33792\n\t" \
  "v_mfma_f32_16x16x32_bf16 a[68:71], a[136:139], v[80:83], a[68:71]\n\t" \
  "v_mfma_f32_16x16x32_bf16 a[72:75], a[144:147], v[80:83], a[72:75]\n\t" \
  "ds_read_b64_tr_b16 a[220:221], v15 offset:40960\n\t" \
  "v_mfma_f32_16x16x32_bf16 a[72:75], a[148:151], v[84:87], a[72:75]\n\t" \
  "v_mfma_f32_16x16x32_bf16 a[76:79], a[156:159], v[84:87], a[76:79]\n\t" \
  "ds_read_b64_tr_b16 a[222:223], v15 offset:41984\n\t" \
  "v_mfma_f32_16x16x32_bf16 a[76:79], a[152:155], v[80:83], a[76:79]\n\t" \
  "s_add_u32 m0, s68, 16384\n\t" \
  "v_mfma_f32_16x16x32_bf16 a[80:83], a[160:163], v[80:83], a[80:83]\n\t" \
  "buffer_load_dwordx4 v0, s[36:39], s65 offen offset:0 sc0 sc1 lds\n\t" \
  "v_mfma_f32_16x16x32_bf16 a[80:83], a[164:167], v[84:87], a[80:83]\n\t" \
  "v_mfma_f32_16x16x32_bf16 a[84:87], a[172:175], v[84:87], a[84:87]\n\t" \
  "s_branch RET64_%=\n\t" \
  "SW66_%=:\n\t" \
  "s_mov_b32 s36, s56\n\t" \
  "s_mov_b32 s37, s57\n\t" \
  "s_mov_b32 s38, s63\n\t" \
  "s_mov_b32 s67, 1024\n\t" \
  "s_mov_b32 s89, 0\n\t" \
  "s_lshr_b32 s69, s69, 4\n\t" \
  "s_lshr_b32 s70, s70, 4\n\t" \
  "v_lshrrev_b32 v1, 11, v0\n\t" \
  "v_and_b32 v0, 0x7f, v0\n\t" \
  "v_lshl_add_u32 v0, v1, 7, v0\n\t" \
  "v_xor_b32 v1, 64, v0\n\t" \
  "v_mov_b32 v2, v0\n\t" \
  "v_mov_b32 v3, v1\n\t" \
  "s_branch RET67_%=\n\t" \
  ".p2align 6\n\t" \
  "EVT68_%=:\n\t" \
  "s_cmp_lg_u32 s30, 0\n\t" \
  "s_cbranch_scc1 PUB70_%=\n\t" \
  "global_store_dwordx4 v39, v[32:35], s[52:53] sc0 sc1\n\t" \
  "v_mfma_f32_16x16x32_bf16 v[168:171], a[148:151], v[108:111], v[168:171]\n\t" \
  "ds_read_b64_tr_b16 a[242:243], v14 offset:50176\n\t" \
  "s_and_b32 s87, s26, 15\n\t" \
  "v_mfma_f32_16x16x32_bf16 v[168:171], a[144:147], v[104:107], v[168:171]\n\t" \
  "s_sub_u32 s28, s87, 2\n\t" \
  "v_mfma_f32_16x16x32_bf16 v[164:167], a[136:139], v[104:107], v[164:167]\n\t" \
  "ds_read_b64_tr_b16 a[244:245], v14 offset:57344\n\t" \
  "v_mfma_f32_16x16x32_bf16 v[164:167], a[140:143], v[108:111], v[164:167]\n\t" \
  "s_bitcmp1_b32 32799, s88\n\t" \
  "s_cselect_b32 s87, 1, s87\n\t" \
  "v_mfma_f32_16x16x32_bf16 v[160:163], a[132:135], v[108:111], v[160:163]\n\t" \
  "ds_read_b64_tr_b16 a[246:247], v14 offset:58368\n\t" \
  "s_cselect_b32 s30, 1, 0\n\t" \
  "v_mfma_f32_16x16x32_bf16 v[160:163], a[128:131], v[104:107], v[160:163]\n\t" \
  "ds_read_b128 v[104:107], v10 offset:51200\n\t" \
  "ds_read_b128 v[108:111], v11 offset:51200\n\t" \
  "v_mfma_f32_16x16x32_bf16 v[192:195], a[128:131], v[112:115], v[192:195]\n\t" \
  "ds_read_b64_tr_b16 a[248:249], v15 offset:49152\n\t" \
  "s_lshr_b64 s[26:27], s[26:27], 4\n\t" \
  "v_mfma_f32_16x16x32_bf16 v[192:195], a[132:135], v[116:119], v[192:195]\n\t" \
  "s_cmp_eq_u32 s88, 15\n\t" \
  "s_cselect_b32 s74, 0, s74\n\t" \
  "v_mfma_f32_16x16x32_bf16 v[196:199], a[140:143], v[116:119], v[196:199]\n\t" \
  "ds_read_b64_tr_b16 a[250:251], v15 offset:50176\n\t" \
  "v_add_u32 v8, s74, v8\n\t" \
  "v_mfma_f32_16x16x32_bf16 v[196:199], a[136:139], v[112:115], v[196:199]\n\t" \
  "global_load_dwordx4 v[32:35], v8, s[48:49] nt\n\t" \
  "s_lshl_b32 s98, s74, 10\n\t" \
  "v_mfma_f32_16x16x32_bf16 v[200:203], a[144:147], v[112:115], v[200:203]\n\t" \
  "ds_read_b64_tr_b16 a[252:253], v15 offset:57344\n\t" \
  "v_add_u32 v39, s98, v39\n\t" \
  "v_mfma_f32_16x16x32_bf16 v[200:203], a[148:151], v[116:119], v[200:203]\n\t" \
  "s_xor_b32 s74, s74, s80\n\t" \
  "s_add_u32 s88, s88, 1\n\t" \
  "v_mfma_f32_16x16x32_bf16 v[204:207], a[156:159], v[116:119], v[204:207]\n\t" \
  "ds_read_b64_tr_b16 a[254:255], v15 offset:58368\n\t" \
  "v_mfma_f32_16x16x32_bf16 v[204:207], a[152:155], v[112:115], v[204:207]\n\t" \
  "s_add_u32 m0, s68, 32768\n\t" \
  "v_mfma_f32_16x16x32_bf16 v[208:211], a[160:163], v[112:115], v[208:211]\n\t" \
  "buffer_load_dwordx4 v0, s[36:39], s64 offen offset:0 sc0 sc1 lds\n\t" \
  "v_mfma_f32_16x16x32_bf16 v[208:211], a[164:167], v[116:119], v[208:211]\n\t" \
  "v_mfma_f32_16x16x32_bf16 v[212:215], a[172:175], v[116:119], v[212:215]\n\t" \
  "s_branch RET69_%=\n\t" \
  ".p2align 6\n\t" \
  "PUB70_%=:\n\t" \
  "v_mfma_f32_16x16x32_bf16 v[168:171], a[148:151], v[108:111], v[168:171]\n\t" \
  "ds_read_b64_tr_b16 a[242:243], v14 offset:50176\n\t" \
  "s_mov_b32 s30, 0\n\t" \
  "v_mfma_f32_16x16x32_bf16 v[168:171], a[144:147], v[104:107], v[168:171]\n\t" \
  "s_add_u32 s28, s28, s72\n\t" \
  "v_mfma_f32_16x16x32_bf16 v[164:167], a[136:139], v[104:107], v[164:167]\n\t" \
  "ds_read_b64_tr_b16 a[244:245], v14 offset:57344\n\t" \
  "v_mfma_f32_16x16x32_bf16 v[164:167], a[140:143], v[108:111], v[164:167]\n\t" \
  "s_cmp_eq_u32 s88, 16\n\t" \
  "s_cselect_b32 s87, 0x7fffffff, s28\n\t" \
  "v_mfma_f32_16x16x32_bf16 v[160:163], a[132:135], v[108:111], v[160:163]\n\t" \
  "ds_read_b64_tr_b16 a[246:247], v14 offset:58368\n\t" \
  "s_mov_b32 s72, s73\n\t" \
  "v_mfma_f32_16x16x32_bf16 v[160:163], a[128:131], v[104:107], v[160:163]\n\t" \
  "ds_read_b128 v[104:107], v10 offset:51200\n\t" \
  "ds_read_b128 v[108:111], v11 offset:51200\n\t" \
  "v_mfma_f32_16x16x32_bf16 v[192:195], a[128:131], v[112:115], v[192:195]\n\t" \
  "ds_read_b64_tr_b16 a[248:249], v15 offset:49152\n\t" \
  "s_mov_b32 s73, 0\n\t" \
  "v_mfma_f32_16x16x32_bf16 v[192:195], a[132:135], v[116:119], v[192:195]\n\t" \
  "v_mfma_f32_16x16x32_bf16 v[196:199], a[140:143], v[116:119], v[196:199]\n\t" \
  "ds_read_b64_tr_b16 a[250:251], v15 offset:50176\n\t" \
  "v_mfma_f32_16x16x32_bf16 v[196:199], a[136:139], v[112:115], v[196:199]\n\t" \
  "v_mfma_f32_16x16x32_bf16 v[200:203], a[144:147], v[112:115], v[200:203]\n\t" \
  "ds_read_b64_tr_b16 a[252:253], v15 offset:57344\n\t" \
  "v_mfma_f32_16x16x32_bf16 v[200:203], a[148:151], v[116:119], v[200:203]\n\t" \
  "v_mfma_f32_16x16x32_bf16 v[204:207], a[156:159], v[116:119], v[204:207]\n\t" \
  "ds_read_b64_tr_b16 a[254:255], v15 offset:58368\n\t" \
  "v_mfma_f32_16x16x32_bf16 v[204:207], a[152:155], v[112:115], v[204:207]\n\t" \
  "s_add_u32 m0, s68, 32768\n\t" \
  "v_mfma_f32_16x16x32_bf16 v[208:211], a[160:163], v[112:115], v[208:211]\n\t" \
  "buffer_load_dwordx4 v0, s[36:39], s64 offen offset:0 sc0 sc1 lds\n\t" \
  "v_mfma_f32_16x16x32_bf16 v[208:211], a[164:167], v[116:119], v[208:211]\n\t" \
  "v_mfma_f32_16x16x32_bf16 v[212:215], a[172:175], v[116:119], v[212:215]\n\t" \
  "s_branch RET69_%=\n\t" \
  ".p2align 6\n\t" \
  "EVT71_%=:\n\t" \
  "s_cmp_lg_u32 s30, 0\n\t" \
  "s_cbranch_scc1 PUB73_%=\n\t" \
  "global_store_dwordx4 v39, v[32:35], s[52:53] sc0 sc1\n\t" \
  "v_mfma_f32_16x16x32_bf16 a[40:43], a[212:215], v[76:79], a[40:43]\n\t" \
  "ds_read_b64_tr_b16 a[146:147], v14 offset:1024\n\t" \
  "s_and_b32 s87, s26, 15\n\t" \
  "v_mfma_f32_16x16x32_bf16 a[40:43], a[208:211], v[72:75], a[40:43]\n\t" \
  "s_sub_u32 s28, s87, 2\n\t" \
  "v_mfma_f32_16x16x32_bf16 a[36:39], a[200:203], v[72:75], a[36:39]\n\t" \
  "ds_read_b64_tr_b16 a[148:149], v14 offset:8192\n\t" \
  "v_mfma_f32_16x16x32_bf16 a[36:39], a[204:207], v[76:79], a[36:39]\n\t" \
  "s_bitcmp1_b32 32799, s88\n\t" \
  "s_cselect_b32 s87, 1, s87\n\t" \
  "v_mfma_f32_16x16x32_bf16 a[32:35], a[196:199], v[76:79], a[32:35]\n\t" \
  "ds_read_b64_tr_b16 a[150:151], v14 offset:9216\n\t" \
  "s_cselect_b32 s30, 1, 0\n\t" \
  "v_mfma_f32_16x16x32_bf16 a[32:35], a[192:195], v[72:75], a[32:35]\n\t" \
  "ds_read_b128 v[72:75], v10 offset:2048\n\t" \
  "ds_read_b128 v[76:79], v11 offset:2048\n\t" \
  "v_mfma_f32_16x16x32_bf16 a[64:67], a[192:195], v[80:83], a[64:67]\n\t" \
  "ds_read_b64_tr_b16 a[152:153], v15 offset:0\n\t" \
  "s_lshr_b64 s[26:27], s[26:27], 4\n\t" \
  "v_mfma_f32_16x16x32_bf16 a[64:67], a[196:199], v[84:87], a[64:67]\n\t" \
  "s_cmp_eq_u32 s88, 15\n\t" \
  "s_cselect_b32 s74, 0, s74\n\t" \
  "v_mfma_f32_16x16x32_bf16 a[68:71], a[204:207], v[84:87], a[68:71]\n\t" \
  "ds_read_b64_tr_b16 a[154:155], v15 offset:1024\n\t" \
  "v_add_u32 v8, s74, v8\n\t" \
  "v_mfma_f32_16x16x32_bf16 a[68:71], a[200:203], v[80:83], a[68:71]\n\t" \
  "global_load_dwordx4 v[32:35], v8, s[48:49] nt\n\t" \
  "s_lshl_b32 s98, s74, 10\n\t" \
  "v_mfma_f32_16x16x32_bf16 a[72:75], a[208:211], v[80:83], a[72:75]\n\t" \
  "ds_read_b64_tr_b16 a[156:157], v15 offset:8192\n\t" \
  "v_add_u32 v39, s98, v39\n\t" \
  "v_mfma_f32_16x16x32_bf16 a[72:75], a[212:215], v[84:87], a[72:75]\n\t" \
  "s_xor_b32 s74, s74, s80\n\t" \
  "s_add_u32 s88, s88, 1\n\t" \
  "v_mfma_f32_16x16x32_bf16 a[76:79], a[220:223], v[84:87], a[76:79]\n\t" \
  "ds_read_b64_tr_b16 a[158:159], v15 offset:9216\n\t" \
  "v_mfma_f32_16x16x32_bf16 a[76:79], a[216:219], v[80:83], a[76:79]\n\t" \
  "s_add_u32 m0, s68, 49152\n\t" \
  "v_mfma_f32_16x16x32_bf16 a[80:83], a[224:227], v[80:83], a[80:83]\n\t" \
  "buffer_load_dwordx4 v0, s[36:39], s65 offen offset:0 sc0 sc1 lds\n\t" \
  "v_mfma_f32_16x16x32_bf16 a[80:83], a[228:231], v[84:87], a[80:83]\n\t" \
  "v_mfma_f32_16x16x32_bf16 a[84:87], a[236:239], v[84:87], a[84:87]\n\t" \
  "s_branch RET72_%=\n\t" \
  ".p2align 6\n\t" \
  "PUB73_%=:\n\t" \
  "v_mfma_f32_16x16x32_bf16 a[40:43], a[212:215], v[76:79], a[40:43]\n\t" \
  "ds_read_b64_tr_b16 a[146:147], v14 offset:1024\n\t" \
  "s_mov_b32 s30, 0\n\t" \
  "v_mfma_f32_16x16x32_bf16 a[40:43], a[208:211], v[72:75], a[40:43]\n\t" \
  "s_add_u32 s28, s28, s72\n\t" \
  "v_mfma_f32_16x16x32_bf16 a[36:39], a[200:203], v[72:75], a[36:39]\n\t" \
  "ds_read_b64_tr_b16 a[148:149], v14 offset:8192\n\t" \
  "v_mfma_f32_16x16x32_bf16 a[36:39], a[204:207], v[76:79], a[36:39]\n\t" \
  "s_cmp_eq_u32 s88, 16\n\t" \
  "s_cselect_b32 s87, 0x7fffffff, s28\n\t" \
  "v_mfma_f32_16x16x32_bf16 a[32:35], a[196:199], v[76:79], a[32:35]\n\t" \
  "ds_read_b64_tr_b16 a[150:151], v14 offset:9216\n\t" \
  "s_mov_b32 s72, s73\n\t" \
  "v_mfma_f32_16x16x32_bf16 a[32:35], a[192:195], v[72:75], a[32:35]\n\t" \
  "ds_read_b128 v[72:75], v10 offset:2048\n\t" \
  "ds_read_b128 v[76:79], v11 offset:2048\n\t" \
  "v_mfma_f32_16x16x32_bf16 a[64:67], a[192:195], v[80:83], a[64:67]\n\t" \
  "ds_read_b64_tr_b16 a[152:153], v15 offset:0\n\t" \
  "s_mov_b32 s73, 0\n\t" \
  "v_mfma_f32_16x16x32_bf16 a[64:67], a[196:199], v[84:87], a[64:67]\n\t" \
  "v_mfma_f32_16x16x32_bf16 a[68:71], a[204:207], v[84:87], a[68:71]\n\t" \
  "ds_read_b64_tr_b16 a[154:155], v15 offset:1024\n\t" \
  "v_mfma_f32_16x16x32_bf16 a[68:71], a[200:203], v[80:83], a[68:71]\n\t" \
  "v_mfma_f32_16x16x32_bf16 a[72:75], a[208:211], v[80:83], a[72:75]\n\t" \
  "ds_read_b64_tr_b16 a[156:157], v15 offset:8192\n\t" \
  "v_mfma_f32_16x16x32_bf16 a[72:75], a[212:215], v[84:87], a[72:75]\n\t" \
  "v_mfma_f32_16x16x32_bf16 a[76:79], a[220:223], v[84:87], a[76:79]\n\t" \
  "ds_read_b64_tr_b16 a[158:159], v15 offset:9216\n\t" \
  "v_mfma_f32_16x16x32_bf16 a[76:79], a[216:219], v[80:83], a[76:79]\n\t" \
  "s_add_u32 m0, s68, 49152\n\t" \
  "v_mfma_f32_16x16x32_bf16 a[80:83], a[224:227], v[80:83], a[80:83]\n\t" \
  "buffer_load_dwordx4 v0, s[36:39], s65 offen offset:0 sc0 sc1 lds\n\t" \
  "v_mfma_f32_16x16x32_bf16 a[80:83], a[228:231], v[84:87], a[80:83]\n\t" \
  "v_mfma_f32_16x16x32_bf16 a[84:87], a[236:239], v[84:87], a[84:87]\n\t" \
  "s_branch RET72_%=\n\t" \
  "SW74_%=:\n\t" \
  "s_mov_b32 s36, s56\n\t" \
  "s_mov_b32 s37, s57\n\t" \
  "s_mov_b32 s38, s63\n\t" \
  "s_mov_b32 s67, 1024\n\t" \
  "s_mov_b32 s89, 0\n\t" \
  "s_lshr_b32 s69, s69, 4\n\t" \
  "s_lshr_b32 s70, s70, 4\n\t" \
  "v_lshrrev_b32 v1, 11, v0\n\t" \
  "v_and_b32 v0, 0x7f, v0\n\t" \
  "v_lshl_add_u32 v0, v1, 7, v0\n\t" \
  "v_xor_b32 v1, 64, v0\n\t" \
  "v_mov_b32 v2, v0\n\t" \
  "v_mov_b32 v3, v1\n\t" \
  "s_branch RET75_%=\n\t" \
  "TRG78_%=:\n\t" \
  "s_cmp_eq_u32 s76, 4\n\t" \
  "s_cbranch_scc1 TRGL80_%=\n\t" \
  "s_branch TRGR79_%=\n\t" \
  "s_memrealtime s[96:97]\n\t" \
  "s_waitcnt lgkmcnt(0)\n\t" \
  "v_mov_b32 v46, s96\n\t" \
  "v_mov_b32 v47, s85\n\t" \
  "s_mov_b64 exec, 1\n\t" \
  "global_store_dwordx2 v22, v[46:47], s[50:51] sc0 sc1\n\t" \
  "s_mov_b64 exec, -1\n\t" \
  "s_branch TRGR79_%=\n\t" \
  "TRGL80_%=:\n\t" \
  "s_mov_b64 exec, 3\n\t" \
  "global_load_dwordx2 v[20:21], v22, s[50:51] sc0 sc1\n\t" \
  "s_mov_b64 exec, -1\n\t" \
  "s_branch TRGR79_%=\n\t" \
  ".p2align 6\n\t" \
  "EVT76_%=:\n\t" \
  "s_cmp_lg_u32 s30, 0\n\t" \
  "s_cbranch_scc1 PUB81_%=\n\t" \
  "global_store_dwordx4 v39, v[32:35], s[52:53] sc0 sc1\n\t" \
  "v_mfma_f32_16x16x32_bf16 v[168:171], a[212:215], v[108:111], v[168:171]\n\t" \
  "ds_read_b64_tr_b16 a[178:179], v14 offset:17408\n\t" \
  "s_and_b32 s87, s26, 15\n\t" \
  "v_mfma_f32_16x16x32_bf16 v[168:171], a[208:211], v[104:107], v[168:171]\n\t" \
  "s_sub_u32 s28, s87, 2\n\t" \
  "v_mfma_f32_16x16x32_bf16 v[164:167], a[200:203], v[104:107], v[164:167]\n\t" \
  "ds_read_b64_tr_b16 a[180:181], v14 offset:24576\n\t" \
  "v_mfma_f32_16x16x32_bf16 v[164:167], a[204:207], v[108:111], v[164:167]\n\t" \
  "s_bitcmp1_b32 32799, s88\n\t" \
  "s_cselect_b32 s87, 1, s87\n\t" \
  "v_mfma_f32_16x16x32_bf16 v[160:163], a[196:199], v[108:111], v[160:163]\n\t" \
  "ds_read_b64_tr_b16 a[182:183], v14 offset:25600\n\t" \
  "s_cselect_b32 s30, 1, 0\n\t" \
  "v_mfma_f32_16x16x32_bf16 v[160:163], a[192:195], v[104:107], v[160:163]\n\t" \
  "ds_read_b128 v[104:107], v10 offset:18432\n\t" \
  "ds_read_b128 v[108:111], v11 offset:18432\n\t" \
  "v_mfma_f32_16x16x32_bf16 v[192:195], a[192:195], v[112:115], v[192:195]\n\t" \
  "ds_read_b64_tr_b16 a[184:185], v15 offset:16384\n\t" \
  "s_lshr_b64 s[26:27], s[26:27], 4\n\t" \
  "v_mfma_f32_16x16x32_bf16 v[192:195], a[196:199], v[116:119], v[192:195]\n\t" \
  "s_cmp_eq_u32 s88, 15\n\t" \
  "s_cselect_b32 s74, 0, s74\n\t" \
  "v_mfma_f32_16x16x32_bf16 v[196:199], a[204:207], v[116:119], v[196:199]\n\t" \
  "ds_read_b64_tr_b16 a[186:187], v15 offset:17408\n\t" \
  "v_add_u32 v8, s74, v8\n\t" \
  "v_mfma_f32_16x16x32_bf16 v[196:199], a[200:203], v[112:115], v[196:199]\n\t" \
  "global_load_dwordx4 v[32:35], v8, s[48:49] nt\n\t" \
  "s_lshl_b32 s98, s74, 10\n\t" \
  "v_mfma_f32_16x16x32_bf16 v[200:203], a[208:211], v[112:115], v[200:203]\n\t" \
  "ds_read_b64_tr_b16 a[188:189], v15 offset:24576\n\t" \
  "v_add_u32 v39, s98, v39\n\t" \
  "v_mfma_f32_16x16x32_bf16 v[200:203], a[212:215], v[116:119], v[200:203]\n\t" \
  "s_xor_b32 s74, s74, s80\n\t" \
  "s_add_u32 s88, s88, 1\n\t" \
  "v_mfma_f32_16x16x32_bf16 v[204:207], a[220:223], v[116:119], v[204:207]\n\t" \
  "ds_read_b64_tr_b16 a[190:191], v15 offset:25600\n\t" \
  "v_mfma_f32_16x16x32_bf16 v[204:207], a[216:219], v[112:115], v[204:207]\n\t" \
  "s_add_u32 m0, s68, 0\n\t" \
  "v_mfma_f32_16x16x32_bf16 v[208:211], a[224:227], v[112:115], v[208:211]\n\t" \
  "buffer_load_dwordx4 v0, s[36:39], s64 offen offset:0 sc0 sc1 lds\n\t" \
  "v_mfma_f32_16x16x32_bf16 v[208:211], a[228:231], v[116:119], v[208:211]\n\t" \
  "v_mfma_f32_16x16x32_bf16 v[212:215], a[236:239], v[116:119], v[212:215]\n\t" \
  "s_branch RET77_%=\n\t" \
  ".p2align 6\n\t" \
  "PUB81_%=:\n\t" \
  "v_mfma_f32_16x16x32_bf16 v[168:171], a[212:215], v[108:111], v[168:171]\n\t" \
  "ds_read_b64_tr_b16 a[178:179], v14 offset:17408\n\t" \
  "s_mov_b32 s30, 0\n\t" \
  "v_mfma_f32_16x16x32_bf16 v[168:171], a[208:211], v[104:107], v[168:171]\n\t" \
  "s_add_u32 s28, s28, s72\n\t" \
  "v_mfma_f32_16x16x32_bf16 v[164:167], a[200:203], v[104:107], v[164:167]\n\t" \
  "ds_read_b64_tr_b16 a[180:181], v14 offset:24576\n\t" \
  "v_mfma_f32_16x16x32_bf16 v[164:167], a[204:207], v[108:111], v[164:167]\n\t" \
  "s_cmp_eq_u32 s88, 16\n\t" \
  "s_cselect_b32 s87, 0x7fffffff, s28\n\t" \
  "v_mfma_f32_16x16x32_bf16 v[160:163], a[196:199], v[108:111], v[160:163]\n\t" \
  "ds_read_b64_tr_b16 a[182:183], v14 offset:25600\n\t" \
  "s_mov_b32 s72, s73\n\t" \
  "v_mfma_f32_16x16x32_bf16 v[160:163], a[192:195], v[104:107], v[160:163]\n\t" \
  "ds_read_b128 v[104:107], v10 offset:18432\n\t" \
  "ds_read_b128 v[108:111], v11 offset:18432\n\t" \
  "v_mfma_f32_16x16x32_bf16 v[192:195], a[192:195], v[112:115], v[192:195]\n\t" \
  "ds_read_b64_tr_b16 a[184:185], v15 offset:16384\n\t" \
  "s_mov_b32 s73, 0\n\t" \
  "v_mfma_f32_16x16x32_bf16 v[192:195], a[196:199], v[116:119], v[192:195]\n\t" \
  "v_mfma_f32_16x16x32_bf16 v[196:199], a[204:207], v[116:119], v[196:199]\n\t" \
  "ds_read_b64_tr_b16 a[186:187], v15 offset:17408\n\t" \
  "v_mfma_f32_16x16x32_bf16 v[196:199], a[200:203], v[112:115], v[196:199]\n\t" \
  "v_mfma_f32_16x16x32_bf16 v[200:203], a[208:211], v[112:115], v[200:203]\n\t" \
  "ds_read_b64_tr_b16 a[188:189], v15 offset:24576\n\t" \
  "v_mfma_f32_16x16x32_bf16 v[200:203], a[212:215], v[116:119], v[200:203]\n\t" \
  "v_mfma_f32_16x16x32_bf16 v[204:207], a[220:223], v[116:119], v[204:207]\n\t" \
  "ds_read_b64_tr_b16 a[190:191], v15 offset:25600\n\t" \
  "v_mfma_f32_16x16x32_bf16 v[204:207], a[216:219], v[112:115], v[204:207]\n\t" \
  "s_add_u32 m0, s68, 0\n\t" \
  "v_mfma_f32_16x16x32_bf16 v[208:211], a[224:227], v[112:115], v[208:211]\n\t" \
  "buffer_load_dwordx4 v0, s[36:39], s64 offen offset:0 sc0 sc1 lds\n\t" \
  "v_mfma_f32_16x16x32_bf16 v[208:211], a[228:231], v[116:119], v[208:211]\n\t" \
  "v_mfma_f32_16x16x32_bf16 v[212:215], a[236:239], v[116:119], v[212:215]\n\t" \
  "s_branch RET77_%=\n\t" \
  ".p2align 6\n\t" \
  "EVT82_%=:\n\t" \
  "s_cmp_lg_u32 s30, 0\n\t" \
  "s_cbranch_scc1 PUB84_%=\n\t" \
  "global_store_dwordx4 v39, v[32:35], s[52:53] sc0 sc1\n\t" \
  "v_mfma_f32_16x16x32_bf16 a[40:43], a[148:151], v[76:79], a[40:43]\n\t" \
  "ds_read_b64_tr_b16 a[210:211], v14 offset:33792\n\t" \
  "s_and_b32 s87, s26, 15\n\t" \
  "v_mfma_f32_16x16x32_bf16 a[40:43], a[144:147], v[72:75], a[40:43]\n\t" \
  "s_sub_u32 s28, s87, 2\n\t" \
  "v_mfma_f32_16x16x32_bf16 a[36:39], a[136:139], v[72:75], a[36:39]\n\t" \
  "ds_read_b64_tr_b16 a[212:213], v14 offset:40960\n\t" \
  "v_mfma_f32_16x16x32_bf16 a[36:39], a[140:143], v[76:79], a[36:39]\n\t" \
  "s_bitcmp1_b32 32799, s88\n\t" \
  "s_cselect_b32 s87, 1, s87\n\t" \
  "v_mfma_f32_16x16x32_bf16 a[32:35], a[132:135], v[76:79], a[32:35]\n\t" \
  "ds_read_b64_tr_b16 a[214:215], v14 offset:41984\n\t" \
  "s_cselect_b32 s30, 1, 0\n\t" \
  "v_mfma_f32_16x16x32_bf16 a[32:35], a[128:131], v[72:75], a[32:35]\n\t" \
  "ds_read_b128 v[72:75], v10 offset:34816\n\t" \
  "ds_read_b128 v[76:79], v11 offset:34816\n\t" \
  "v_mfma_f32_16x16x32_bf16 a[64:67], a[128:131], v[80:83], a[64:67]\n\t" \
  "ds_read_b64_tr_b16 a[216:217], v15 offset:32768\n\t" \
  "s_lshr_b64 s[26:27], s[26:27], 4\n\t" \
  "v_mfma_f32_16x16x32_bf16 a[64:67], a[132:135], v[84:87], a[64:67]\n\t" \
  "s_cmp_eq_u32 s88, 15\n\t" \
  "s_cselect_b32 s74, 0, s74\n\t" \
  "v_mfma_f32_16x16x32_bf16 a[68:71], a[140:143], v[84:87], a[68:71]\n\t" \
  "ds_read_b64_tr_b16 a[218:219], v15 offset:33792\n\t" \
  "v_add_u32 v8, s74, v8\n\t" \
  "v_mfma_f32_16x16x32_bf16 a[68:71], a[136:139], v[80:83], a[68:71]\n\t" \
  "global_load_dwordx4 v[32:35], v8, s[48:49] nt\n\t" \
  "s_lshl_b32 s98, s74, 10\n\t" \
  "v_mfma_f32_16x16x32_bf16 a[72:75], a[144:147], v[80:83], a[72:75]\n\t" \
  "ds_read_b64_tr_b16 a[220:221], v15 offset:40960\n\t" \
  "v_add_u32 v39, s98, v39\n\t" \
  "v_mfma_f32_16x16x32_bf16 a[72:75], a[148:151], v[84:87], a[72:75]\n\t" \
  "s_xor_b32 s74, s74, s80\n\t" \
  "s_add_u32 s88, s88, 1\n\t" \
  "v_mfma_f32_16x16x32_bf16 a[76:79], a[156:159], v[84:87], a[76:79]\n\t" \
  "ds_read_b64_tr_b16 a[222:223], v15 offset:41984\n\t" \
  "v_mfma_f32_16x16x32_bf16 a[76:79], a[152:155], v[80:83], a[76:79]\n\t" \
  "s_add_u32 m0, s68, 16384\n\t" \
  "v_mfma_f32_16x16x32_bf16 a[80:83], a[160:163], v[80:83], a[80:83]\n\t" \
  "buffer_load_dwordx4 v0, s[36:39], s65 offen offset:0 sc0 sc1 lds\n\t" \
  "v_mfma_f32_16x16x32_bf16 a[80:83], a[164:167], v[84:87], a[80:83]\n\t" \
  "v_mfma_f32_16x16x32_bf16 a[84:87], a[172:175], v[84:87], a[84:87]\n\t" \
  "s_branch RET83_%=\n\t" \
  ".p2align 6\n\t" \
  "PUB84_%=:\n\t" \
  "v_mfma_f32_16x16x32_bf16 a[40:43], a[148:151], v[76:79], a[40:43]\n\t" \
  "ds_read_b64_tr_b16 a[210:211], v14 offset:33792\n\t" \
  "s_mov_b32 s30, 0\n\t" \
  "v_mfma_f32_16x16x32_bf16 a[40:43], a[144:147], v[72:75], a[40:43]\n\t" \
  "s_add_u32 s28, s28, s72\n\t" \
  "v_mfma_f32_16x16x32_bf16 a[36:39], a[136:139], v[72:75], a[36:39]\n\t" \
  "ds_read_b64_tr_b16 a[212:213], v14 offset:40960\n\t" \
  "v_mfma_f32_16x16x32_bf16 a[36:39], a[140:143], v[76:79], a[36:39]\n\t" \
  "s_cmp_eq_u32 s88, 16\n\t" \
  "s_cselect_b32 s87, 0x7fffffff, s28\n\t" \
  "v_mfma_f32_16x16x32_bf16 a[32:35], a[132:135], v[76:79], a[32:35]\n\t" \
  "ds_read_b64_tr_b16 a[214:215], v14 offset:41984\n\t" \
  "s_mov_b32 s72, s73\n\t" \
  "v_mfma_f32_16x16x32_bf16 a[32:35], a[128:131], v[72:75], a[32:35]\n\t" \
  "ds_read_b128 v[72:75], v10 offset:34816\n\t" \
  "ds_read_b128 v[76:79], v11 offset:34816\n\t" \
  "v_mfma_f32_16x16x32_bf16 a[64:67], a[128:131], v[80:83], a[64:67]\n\t" \
  "ds_read_b64_tr_b16 a[216:217], v15 offset:32768\n\t" \
  "s_mov_b32 s73, 0\n\t" \
  "v_mfma_f32_16x16x32_bf16 a[64:67], a[132:135], v[84:87], a[64:67]\n\t" \
  "v_mfma_f32_16x16x32_bf16 a[68:71], a[140:143], v[84:87], a[68:71]\n\t" \
  "ds_read_b64_tr_b16 a[218:219], v15 offset:33792\n\t" \
  "v_mfma_f32_16x16x32_bf16 a[68:71], a[136:139], v[80:83], a[68:71]\n\t" \
  "v_mfma_f32_16x16x32_bf16 a[72:75], a[144:147], v[80:83], a[72:75]\n\t" \
  "ds_read_b64_tr_b16 a[220:221], v15 offset:40960\n\t" \
  "v_mfma_f32_16x16x32_bf16 a[72:75], a[148:151], v[84:87], a[72:75]\n\t" \
  "v_mfma_f32_16x16x32_bf16 a[76:79], a[156:159], v[84:87], a[76:79]\n\t" \
  "ds_read_b64_tr_b16 a[222:223], v15 offset:41984\n\t" \
  "v_mfma_f32_16x16x32_bf16 a[76:79], a[152:155], v[80:83], a[76:79]\n\t" \
  "s_add_u32 m0, s68, 16384\n\t" \
  "v_mfma_f32_16x16x32_bf16 a[80:83], a[160:163], v[80:83], a[80:83]\n\t" \
  "buffer_load_dwordx4 v0, s[36:39], s65 offen offset:0 sc0 sc1 lds\n\t" \
  "v_mfma_f32_16x16x32_bf16 a[80:83], a[164:167], v[84:87], a[80:83]\n\t" \
  "v_mfma_f32_16x16x32_bf16 a[84:87], a[172:175], v[84:87], a[84:87]\n\t" \
  "s_branch RET83_%=\n\t" \
  "SW85_%=:\n\t" \
  "s_mov_b32 s36, s56\n\t" \
  "s_mov_b32 s37, s57\n\t" \
  "s_mov_b32 s38, s63\n\t" \
  "s_mov_b32 s67, 1024\n\t" \
  "s_mov_b32 s89, 0\n\t" \
  "s_lshr_b32 s69, s69, 4\n\t" \
  "s_lshr_b32 s70, s70, 4\n\t" \
  "v_lshrrev_b32 v1, 11, v0\n\t" \
  "v_and_b32 v0, 0x7f, v0\n\t" \
  "v_lshl_add_u32 v0, v1, 7, v0\n\t" \
  "v_xor_b32 v1, 64, v0\n\t" \
  "v_mov_b32 v2, v0\n\t" \
  "v_mov_b32 v3, v1\n\t" \
  "s_branch RET86_%=\n\t" \
  ".p2align 6\n\t" \
  "EVT87_%=:\n\t" \
  "s_cmp_lg_u32 s30, 0\n\t" \
  "s_cbranch_scc1 PUB89_%=\n\t" \
  "global_store_dwordx4 v39, v[32:35], s[52:53] sc0 sc1\n\t" \
  "v_mfma_f32_16x16x32_bf16 v[168:171], a[148:151], v[108:111], v[168:171]\n\t" \
  "ds_read_b64_tr_b16 a[242:243], v14 offset:50176\n\t" \
  "s_and_b32 s87, s26, 15\n\t" \
  "v_mfma_f32_16x16x32_bf16 v[168:171], a[144:147], v[104:107], v[168:171]\n\t" \
  "s_sub_u32 s28, s87, 2\n\t" \
  "v_mfma_f32_16x16x32_bf16 v[164:167], a[136:139], v[104:107], v[164:167]\n\t" \
  "ds_read_b64_tr_b16 a[244:245], v14 offset:57344\n\t" \
  "v_mfma_f32_16x16x32_bf16 v[164:167], a[140:143], v[108:111], v[164:167]\n\t" \
  "s_bitcmp1_b32 32799, s88\n\t" \
  "s_cselect_b32 s87, 1, s87\n\t" \
  "v_mfma_f32_16x16x32_bf16 v[160:163], a[132:135], v[108:111], v[160:163]\n\t" \
  "ds_read_b64_tr_b16 a[246:247], v14 offset:58368\n\t" \
  "s_cselect_b32 s30, 1, 0\n\t" \
  "v_mfma_f32_16x16x32_bf16 v[160:163], a[128:131], v[104:107], v[160:163]\n\t" \
  "ds_read_b128 v[104:107], v10 offset:51200\n\t" \
  "ds_read_b128 v[108:111], v11 offset:51200\n\t" \
  "v_mfma_f32_16x16x32_bf16 v[192:195], a[128:131], v[112:115], v[192:195]\n\t" \
  "ds_read_b64_tr_b16 a[248:249], v15 offset:49152\n\t" \
  "s_lshr_b64 s[26:27], s[26:27], 4\n\t" \
  "v_mfma_f32_16x16x32_bf16 v[192:195], a[132:135], v[116:119], v[192:195]\n\t" \
  "s_cmp_eq_u32 s88, 15\n\t" \
  "s_cselect_b32 s74, 0, s74\n\t" \
  "v_mfma_f32_16x16x32_bf16 v[196:199], a[140:143], v[116:119], v[196:199]\n\t" \
  "ds_read_b64_tr_b16 a[250:251], v15 offset:50176\n\t" \
  "v_add_u32 v8, s74, v8\n\t" \
  "v_mfma_f32_16x16x32_bf16 v[196:199], a[136:139], v[112:115], v[196:199]\n\t" \
  "global_load_dwordx4 v[32:35], v8, s[48:49] nt\n\t" \
  "s_lshl_b32 s98, s74, 10\n\t" \
  "v_mfma_f32_16x16x32_bf16 v[200:203], a[144:147], v[112:115], v[200:203]\n\t" \
  "ds_read_b64_tr_b16 a[252:253], v15 offset:57344\n\t" \
  "v_add_u32 v39, s98, v39\n\t" \
  "v_mfma_f32_16x16x32_bf16 v[200:203], a[148:151], v[116:119], v[200:203]\n\t" \
  "s_xor_b32 s74, s74, s80\n\t" \
  "s_add_u32 s88, s88, 1\n\t" \
  "v_mfma_f32_16x16x32_bf16 v[204:207], a[156:159], v[116:119], v[204:207]\n\t" \
  "ds_read_b64_tr_b16 a[254:255], v15 offset:58368\n\t" \
  "v_mfma_f32_16x16x32_bf16 v[204:207], a[152:155], v[112:115], v[204:207]\n\t" \
  "s_add_u32 m0, s68, 32768\n\t" \
  "v_mfma_f32_16x16x32_bf16 v[208:211], a[160:163], v[112:115], v[208:211]\n\t" \
  "buffer_load_dwordx4 v0, s[36:39], s64 offen offset:0 sc0 sc1 lds\n\t" \
  "v_mfma_f32_16x16x32_bf16 v[208:211], a[164:167], v[116:119], v[208:211]\n\t" \
  "v_mfma_f32_16x16x32_bf16 v[212:215], a[172:175], v[116:119], v[212:215]\n\t" \
  "s_branch RET88_%=\n\t" \
  ".p2align 6\n\t" \
  "PUB89_%=:\n\t" \
  "v_mfma_f32_16x16x32_bf16 v[168:171], a[148:151], v[108:111], v[168:171]\n\t" \
  "ds_read_b64_tr_b16 a[242:243], v14 offset:50176\n\t" \
  "s_mov_b32 s30, 0\n\t" \
  "v_mfma_f32_16x16x32_bf16 v[168:171], a[144:147], v[104:107], v[168:171]\n\t" \
  "s_add_u32 s28, s28, s72\n\t" \
  "v_mfma_f32_16x16x32_bf16 v[164:167], a[136:139], v[104:107], v[164:167]\n\t" \
  "ds_read_b64_tr_b16 a[244:245], v14 offset:57344\n\t" \
  "v_mfma_f32_16x16x32_bf16 v[164:167], a[140:143], v[108:111], v[164:167]\n\t" \
  "s_cmp_eq_u32 s88, 16\n\t" \
  "s_cselect_b32 s87, 0x7fffffff, s28\n\t" \
  "v_mfma_f32_16x16x32_bf16 v[160:163], a[132:135], v[108:111], v[160:163]\n\t" \
  "ds_read_b64_tr_b16 a[246:247], v14 offset:58368\n\t" \
  "s_mov_b32 s72, s73\n\t" \
  "v_mfma_f32_16x16x32_bf16 v[160:163], a[128:131], v[104:107], v[160:163]\n\t" \
  "ds_read_b128 v[104:107], v10 offset:51200\n\t" \
  "ds_read_b128 v[108:111], v11 offset:51200\n\t" \
  "v_mfma_f32_16x16x32_bf16 v[192:195], a[128:131], v[112:115], v[192:195]\n\t" \
  "ds_read_b64_tr_b16 a[248:249], v15 offset:49152\n\t" \
  "s_mov_b32 s73, 0\n\t" \
  "v_mfma_f32_16x16x32_bf16 v[192:195], a[132:135], v[116:119], v[192:195]\n\t" \
  "v_mfma_f32_16x16x32_bf16 v[196:199], a[140:143], v[116:119], v[196:199]\n\t" \
  "ds_read_b64_tr_b16 a[250:251], v15 offset:50176\n\t" \
  "v_mfma_f32_16x16x32_bf16 v[196:199], a[136:139], v[112:115], v[196:199]\n\t" \
  "v_mfma_f32_16x16x32_bf16 v[200:203], a[144:147], v[112:115], v[200:203]\n\t" \
  "ds_read_b64_tr_b16 a[252:253], v15 offset:57344\n\t" \
  "v_mfma_f32_16x16x32_bf16 v[200:203], a[148:151], v[116:119], v[200:203]\n\t" \
  "v_mfma_f32_16x16x32_bf16 v[204:207], a[156:159], v[116:119], v[204:207]\n\t" \
  "ds_read_b64_tr_b16 a[254:255], v15 offset:58368\n\t" \
  "v_mfma_f32_16x16x32_bf16 v[204:207], a[152:155], v[112:115], v[204:207]\n\t" \
  "s_add_u32 m0, s68, 32768\n\t" \
  "v_mfma_f32_16x16x32_bf16 v[208:211], a[160:163], v[112:115], v[208:211]\n\t" \
  "buffer_load_dwordx4 v0, s[36:39], s64 offen offset:0 sc0 sc1 lds\n\t" \
  "v_mfma_f32_16x16x32_bf16 v[208:211], a[164:167], v[116:119], v[208:211]\n\t" \
  "v_mfma_f32_16x16x32_bf16 v[212:215], a[172:175], v[116:119], v[212:215]\n\t" \
  "s_branch RET88_%=\n\t" \
  ".p2align 6\n\t" \
  "EVT90_%=:\n\t" \
  "s_cmp_lg_u32 s30, 0\n\t" \
  "s_cbranch_scc1 PUB92_%=\n\t" \
  "global_store_dwordx4 v39, v[32:35], s[52:53] sc0 sc1\n\t" \
  "v_mfma_f32_16x16x32_bf16 a[40:43], a[212:215], v[76:79], a[40:43]\n\t" \
  "ds_read_b64_tr_b16 a[146:147], v14 offset:1024\n\t" \
  "s_and_b32 s87, s26, 15\n\t" \
  "v_mfma_f32_16x16x32_bf16 a[40:43], a[208:211], v[72:75], a[40:43]\n\t" \
  "s_sub_u32 s28, s87, 2\n\t" \
  "v_mfma_f32_16x16x32_bf16 a[36:39], a[200:203], v[72:75], a[36:39]\n\t" \
  "ds_read_b64_tr_b16 a[148:149], v14 offset:8192\n\t" \
  "v_mfma_f32_16x16x32_bf16 a[36:39], a[204:207], v[76:79], a[36:39]\n\t" \
  "s_bitcmp1_b32 32799, s88\n\t" \
  "s_cselect_b32 s87, 1, s87\n\t" \
  "v_mfma_f32_16x16x32_bf16 a[32:35], a[196:199], v[76:79], a[32:35]\n\t" \
  "ds_read_b64_tr_b16 a[150:151], v14 offset:9216\n\t" \
  "s_cselect_b32 s30, 1, 0\n\t" \
  "v_mfma_f32_16x16x32_bf16 a[32:35], a[192:195], v[72:75], a[32:35]\n\t" \
  "ds_read_b128 v[72:75], v10 offset:2048\n\t" \
  "ds_read_b128 v[76:79], v11 offset:2048\n\t" \
  "v_mfma_f32_16x16x32_bf16 a[64:67], a[192:195], v[80:83], a[64:67]\n\t" \
  "ds_read_b64_tr_b16 a[152:153], v15 offset:0\n\t" \
  "s_lshr_b64 s[26:27], s[26:27], 4\n\t" \
  "v_mfma_f32_16x16x32_bf16 a[64:67], a[196:199], v[84:87], a[64:67]\n\t" \
  "s_cmp_eq_u32 s88, 15\n\t" \
  "s_cselect_b32 s74, 0, s74\n\t" \
  "v_mfma_f32_16x16x32_bf16 a[68:71], a[204:207], v[84:87], a[68:71]\n\t" \
  "ds_read_b64_tr_b16 a[154:155], v15 offset:1024\n\t" \
  "v_add_u32 v8, s74, v8\n\t" \
  "v_mfma_f32_16x16x32_bf16 a[68:71], a[200:203], v[80:83], a[68:71]\n\t" \
  "global_load_dwordx4 v[32:35], v8, s[48:49] nt\n\t" \
  "s_lshl_b32 s98, s74, 10\n\t" \
  "v_mfma_f32_16x16x32_bf16 a[72:75], a[208:211], v[80:83], a[72:75]\n\t" \
  "ds_read_b64_tr_b16 a[156:157], v15 offset:8192\n\t" \
  "v_add_u32 v39, s98, v39\n\t" \
  "v_mfma_f32_16x16x32_bf16 a[72:75], a[212:215], v[84:87], a[72:75]\n\t" \
  "s_xor_b32 s74, s74, s80\n\t" \
  "s_add_u32 s88, s88, 1\n\t" \
  "v_mfma_f32_16x16x32_bf16 a[76:79], a[220:223], v[84:87], a[76:79]\n\t" \
  "ds_read_b64_tr_b16 a[158:159], v15 offset:9216\n\t" \
  "v_mfma_f32_16x16x32_bf16 a[76:79], a[216:219], v[80:83], a[76:79]\n\t" \
  "s_add_u32 m0, s68, 49152\n\t" \
  "v_mfma_f32_16x16x32_bf16 a[80:83], a[224:227], v[80:83], a[80:83]\n\t" \
  "buffer_load_dwordx4 v0, s[36:39], s65 offen offset:0 sc0 sc1 lds\n\t" \
  "v_mfma_f32_16x16x32_bf16 a[80:83], a[228:231], v[84:87], a[80:83]\n\t" \
  "v_mfma_f32_16x16x32_bf16 a[84:87], a[236:239], v[84:87], a[84:87]\n\t" \
  "s_branch RET91_%=\n\t" \
  ".p2align 6\n\t" \
  "PUB92_%=:\n\t" \
  "v_mfma_f32_16x16x32_bf16 a[40:43], a[212:215], v[76:79], a[40:43]\n\t" \
  "ds_read_b64_tr_b16 a[146:147], v14 offset:1024\n\t" \
  "s_mov_b32 s30, 0\n\t" \
  "v_mfma_f32_16x16x32_bf16 a[40:43], a[208:211], v[72:75], a[40:43]\n\t" \
  "s_add_u32 s28, s28, s72\n\t" \
  "v_mfma_f32_16x16x32_bf16 a[36:39], a[200:203], v[72:75], a[36:39]\n\t" \
  "ds_read_b64_tr_b16 a[148:149], v14 offset:8192\n\t" \
  "v_mfma_f32_16x16x32_bf16 a[36:39], a[204:207], v[76:79], a[36:39]\n\t" \
  "s_cmp_eq_u32 s88, 16\n\t" \
  "s_cselect_b32 s87, 0x7fffffff, s28\n\t" \
  "v_mfma_f32_16x16x32_bf16 a[32:35], a[196:199], v[76:79], a[32:35]\n\t" \
  "ds_read_b64_tr_b16 a[150:151], v14 offset:9216\n\t" \
  "s_mov_b32 s72, s73\n\t" \
  "v_mfma_f32_16x16x32_bf16 a[32:35], a[192:195], v[72:75], a[32:35]\n\t" \
  "ds_read_b128 v[72:75], v10 offset:2048\n\t" \
  "ds_read_b128 v[76:79], v11 offset:2048\n\t" \
  "v_mfma_f32_16x16x32_bf16 a[64:67], a[192:195], v[80:83], a[64:67]\n\t" \
  "ds_read_b64_tr_b16 a[152:153], v15 offset:0\n\t" \
  "s_mov_b32 s73, 0\n\t" \
  "v_mfma_f32_16x16x32_bf16 a[64:67], a[196:199], v[84:87], a[64:67]\n\t" \
  "v_mfma_f32_16x16x32_bf16 a[68:71], a[204:207], v[84:87], a[68:71]\n\t" \
  "ds_read_b64_tr_b16 a[154:155], v15 offset:1024\n\t" \
  "v_mfma_f32_16x16x32_bf16 a[68:71], a[200:203], v[80:83], a[68:71]\n\t" \
  "v_mfma_f32_16x16x32_bf16 a[72:75], a[208:211], v[80:83], a[72:75]\n\t" \
  "ds_read_b64_tr_b16 a[156:157], v15 offset:8192\n\t" \
  "v_mfma_f32_16x16x32_bf16 a[72:75], a[212:215], v[84:87], a[72:75]\n\t" \
  "v_mfma_f32_16x16x32_bf16 a[76:79], a[220:223], v[84:87], a[76:79]\n\t" \
  "ds_read_b64_tr_b16 a[158:159], v15 offset:9216\n\t" \
  "v_mfma_f32_16x16x32_bf16 a[76:79], a[216:219], v[80:83], a[76:79]\n\t" \
  "s_add_u32 m0, s68, 49152\n\t" \
  "v_mfma_f32_16x16x32_bf16 a[80:83], a[224:227], v[80:83], a[80:83]\n\t" \
  "buffer_load_dwordx4 v0, s[36:39], s65 offen offset:0 sc0 sc1 lds\n\t" \
  "v_mfma_f32_16x16x32_bf16 a[80:83], a[228:231], v[84:87], a[80:83]\n\t" \
  "v_mfma_f32_16x16x32_bf16 a[84:87], a[236:239], v[84:87], a[84:87]\n\t" \
  "s_branch RET91_%=\n\t" \
  "SW93_%=:\n\t" \
  "s_mov_b32 s36, s56\n\t" \
  "s_mov_b32 s37, s57\n\t" \
  "s_mov_b32 s38, s63\n\t" \
  "s_mov_b32 s67, 1024\n\t" \
  "s_mov_b32 s89, 0\n\t" \
  "s_lshr_b32 s69, s69, 4\n\t" \
  "s_lshr_b32 s70, s70, 4\n\t" \
  "v_lshrrev_b32 v1, 11, v0\n\t" \
  "v_and_b32 v0, 0x7f, v0\n\t" \
  "v_lshl_add_u32 v0, v1, 7, v0\n\t" \
  "v_xor_b32 v1, 64, v0\n\t" \
  "v_mov_b32 v2, v0\n\t" \
  "v_mov_b32 v3, v1\n\t" \
  "s_branch RET94_%=\n\t" \
  "TRG97_%=:\n\t" \
  "s_cmp_eq_u32 s76, 4\n\t" \
  "s_cbranch_scc1 TRGL99_%=\n\t" \
  "s_branch TRGR98_%=\n\t" \
  "s_memrealtime s[96:97]\n\t" \
  "s_waitcnt lgkmcnt(0)\n\t" \
  "v_mov_b32 v46, s96\n\t" \
  "v_mov_b32 v47, s85\n\t" \
  "s_mov_b64 exec, 1\n\t" \
  "global_store_dwordx2 v22, v[46:47], s[50:51] sc0 sc1\n\t" \
  "s_mov_b64 exec, -1\n\t" \
  "s_branch TRGR98_%=\n\t" \
  "TRGL99_%=:\n\t" \
  "s_mov_b64 exec, 3\n\t" \
  "global_load_dwordx2 v[20:21], v22, s[50:51] sc0 sc1\n\t" \
  "s_mov_b64 exec, -1\n\t" \
  "s_branch TRGR98_%=\n\t" \
  ".p2align 6\n\t" \
  "EVT95_%=:\n\t" \
  "s_cmp_lg_u32 s30, 0\n\t" \
  "s_cbranch_scc1 PUB100_%=\n\t" \
  "global_store_dwordx4 v39, v[32:35], s[52:53] sc0 sc1\n\t" \
  "v_mfma_f32_16x16x32_bf16 v[168:171], a[212:215], v[108:111], v[168:171]\n\t" \
  "ds_read_b64_tr_b16 a[178:179], v14 offset:17408\n\t" \
  "s_and_b32 s87, s26, 15\n\t" \
  "v_mfma_f32_16x16x32_bf16 v[168:171], a[208:211], v[104:107], v[168:171]\n\t" \
  "s_sub_u32 s28, s87, 2\n\t" \
  "v_mfma_f32_16x16x32_bf16 v[164:167], a[200:203], v[104:107], v[164:167]\n\t" \
  "ds_read_b64_tr_b16 a[180:181], v14 offset:24576\n\t" \
  "v_mfma_f32_16x16x32_bf16 v[164:167], a[204:207], v[108:111], v[164:167]\n\t" \
  "s_bitcmp1_b32 32799, s88\n\t" \
  "s_cselect_b32 s87, 1, s87\n\t" \
  "v_mfma_f32_16x16x32_bf16 v[160:163], a[196:199], v[108:111], v[160:163]\n\t" \
  "ds_read_b64_tr_b16 a[182:183], v14 offset:25600\n\t" \
  "s_cselect_b32 s30, 1, 0\n\t" \
  "v_mfma_f32_16x16x32_bf16 v[160:163], a[192:195], v[104:107], v[160:163]\n\t" \
  "ds_read_b128 v[104:107], v10 offset:18432\n\t" \
  "ds_read_b128 v[108:111], v11 offset:18432\n\t" \
  "v_mfma_f32_16x16x32_bf16 v[192:195], a[192:195], v[112:115], v[192:195]\n\t" \
  "ds_read_b64_tr_b16 a[184:185], v15 offset:16384\n\t" \
  "s_lshr_b64 s[26:27], s[26:27], 4\n\t" \
  "v_mfma_f32_16x16x32_bf16 v[192:195], a[196:199], v[116:119], v[192:195]\n\t" \
  "s_cmp_eq_u32 s88, 15\n\t" \
  "s_cselect_b32 s74, 0, s74\n\t" \
  "v_mfma_f32_16x16x32_bf16 v[196:199], a[204:207], v[116:119], v[196:199]\n\t" \
  "ds_read_b64_tr_b16 a[186:187], v15 offset:17408\n\t" \
  "v_add_u32 v8, s74, v8\n\t" \
  "v_mfma_f32_16x16x32_bf16 v[196:199], a[200:203], v[112:115], v[196:199]\n\t" \
  "global_load_dwordx4 v[32:35], v8, s[48:49] nt\n\t" \
  "s_lshl_b32 s98, s74, 10\n\t" \
  "v_mfma_f32_16x16x32_bf16 v[200:203], a[208:211], v[112:115], v[200:203]\n\t" \
  "ds_read_b64_tr_b16 a[188:189], v15 offset:24576\n\t" \
  "v_add_u32 v39, s98, v39\n\t" \
  "v_mfma_f32_16x16x32_bf16 v[200:203], a[212:215], v[116:119], v[200:203]\n\t" \
  "s_xor_b32 s74, s74, s80\n\t" \
  "s_add_u32 s88, s88, 1\n\t" \
  "v_mfma_f32_16x16x32_bf16 v[204:207], a[220:223], v[116:119], v[204:207]\n\t" \
  "ds_read_b64_tr_b16 a[190:191], v15 offset:25600\n\t" \
  "v_mfma_f32_16x16x32_bf16 v[204:207], a[216:219], v[112:115], v[204:207]\n\t" \
  "s_add_u32 m0, s68, 0\n\t" \
  "v_mfma_f32_16x16x32_bf16 v[208:211], a[224:227], v[112:115], v[208:211]\n\t" \
  "buffer_load_dwordx4 v0, s[36:39], s64 offen offset:0 sc0 sc1 lds\n\t" \
  "v_mfma_f32_16x16x32_bf16 v[208:211], a[228:231], v[116:119], v[208:211]\n\t" \
  "v_mfma_f32_16x16x32_bf16 v[212:215], a[236:239], v[116:119], v[212:215]\n\t" \
  "s_branch RET96_%=\n\t" \
  ".p2align 6\n\t" \
  "PUB100_%=:\n\t" \
  "v_mfma_f32_16x16x32_bf16 v[168:171], a[212:215], v[108:111], v[168:171]\n\t" \
  "ds_read_b64_tr_b16 a[178:179], v14 offset:17408\n\t" \
  "s_mov_b32 s30, 0\n\t" \
  "v_mfma_f32_16x16x32_bf16 v[168:171], a[208:211], v[104:107], v[168:171]\n\t" \
  "s_add_u32 s28, s28, s72\n\t" \
  "v_mfma_f32_16x16x32_bf16 v[164:167], a[200:203], v[104:107], v[164:167]\n\t" \
  "ds_read_b64_tr_b16 a[180:181], v14 offset:24576\n\t" \
  "v_mfma_f32_16x16x32_bf16 v[164:167], a[204:207], v[108:111], v[164:167]\n\t" \
  "s_cmp_eq_u32 s88, 16\n\t" \
  "s_cselect_b32 s87, 0x7fffffff, s28\n\t" \
  "v_mfma_f32_16x16x32_bf16 v[160:163], a[196:199], v[108:111], v[160:163]\n\t" \
  "ds_read_b64_tr_b16 a[182:183], v14 offset:25600\n\t" \
  "s_mov_b32 s72, s73\n\t" \
  "v_mfma_f32_16x16x32_bf16 v[160:163], a[192:195], v[104:107], v[160:163]\n\t" \
  "ds_read_b128 v[104:107], v10 offset:18432\n\t" \
  "ds_read_b128 v[108:111], v11 offset:18432\n\t" \
  "v_mfma_f32_16x16x32_bf16 v[192:195], a[192:195], v[112:115], v[192:195]\n\t" \
  "ds_read_b64_tr_b16 a[184:185], v15 offset:16384\n\t" \
  "s_mov_b32 s73, 0\n\t" \
  "v_mfma_f32_16x16x32_bf16 v[192:195], a[196:199], v[116:119], v[192:195]\n\t" \
  "v_mfma_f32_16x16x32_bf16 v[196:199], a[204:207], v[116:119], v[196:199]\n\t" \
  "ds_read_b64_tr_b16 a[186:187], v15 offset:17408\n\t" \
  "v_mfma_f32_16x16x32_bf16 v[196:199], a[200:203], v[112:115], v[196:199]\n\t" \
  "v_mfma_f32_16x16x32_bf16 v[200:203], a[208:211], v[112:115], v[200:203]\n\t" \
  "ds_read_b64_tr_b16 a[188:189], v15 offset:24576\n\t" \
  "v_mfma_f32_16x16x32_bf16 v[200:203], a[212:215], v[116:119], v[200:203]\n\t" \
  "v_mfma_f32_16x16x32_bf16 v[204:207], a[220:223], v[116:119], v[204:207]\n\t" \
  "ds_read_b64_tr_b16 a[190:191], v15 offset:25600\n\t" \
  "v_mfma_f32_16x16x32_bf16 v[204:207], a[216:219], v[112:115], v[204:207]\n\t" \
  "s_add_u32 m0, s68, 0\n\t" \
  "v_mfma_f32_16x16x32_bf16 v[208:211], a[224:227], v[112:115], v[208:211]\n\t" \
  "buffer_load_dwordx4 v0, s[36:39], s64 offen offset:0 sc0 sc1 lds\n\t" \
  "v_mfma_f32_16x16x32_bf16 v[208:211], a[228:231], v[116:119], v[208:211]\n\t" \
  "v_mfma_f32_16x16x32_bf16 v[212:215], a[236:239], v[116:119], v[212:215]\n\t" \
  "s_branch RET96_%=\n\t" \
  ".p2align 6\n\t" \
  "EVT101_%=:\n\t" \
  "s_cmp_lg_u32 s30, 0\n\t" \
  "s_cbranch_scc1 PUB103_%=\n\t" \
  "global_store_dwordx4 v39, v[32:35], s[52:53] sc0 sc1\n\t" \
  "v_mfma_f32_16x16x32_bf16 a[40:43], a[148:151], v[76:79], a[40:43]\n\t" \
  "ds_read_b64_tr_b16 a[210:211], v14 offset:33792\n\t" \
  "s_and_b32 s87, s26, 15\n\t" \
  "v_mfma_f32_16x16x32_bf16 a[40:43], a[144:147], v[72:75], a[40:43]\n\t" \
  "s_sub_u32 s28, s87, 2\n\t" \
  "v_mfma_f32_16x16x32_bf16 a[36:39], a[136:139], v[72:75], a[36:39]\n\t" \
  "ds_read_b64_tr_b16 a[212:213], v14 offset:40960\n\t" \
  "v_mfma_f32_16x16x32_bf16 a[36:39], a[140:143], v[76:79], a[36:39]\n\t" \
  "s_bitcmp1_b32 32799, s88\n\t" \
  "s_cselect_b32 s87, 1, s87\n\t" \
  "v_mfma_f32_16x16x32_bf16 a[32:35], a[132:135], v[76:79], a[32:35]\n\t" \
  "ds_read_b64_tr_b16 a[214:215], v14 offset:41984\n\t" \
  "s_cselect_b32 s30, 1, 0\n\t" \
  "v_mfma_f32_16x16x32_bf16 a[32:35], a[128:131], v[72:75], a[32:35]\n\t" \
  "ds_read_b128 v[72:75], v10 offset:34816\n\t" \
  "ds_read_b128 v[76:79], v11 offset:34816\n\t" \
  "v_mfma_f32_16x16x32_bf16 a[64:67], a[128:131], v[80:83], a[64:67]\n\t" \
  "ds_read_b64_tr_b16 a[216:217], v15 offset:32768\n\t" \
  "s_lshr_b64 s[26:27], s[26:27], 4\n\t" \
  "v_mfma_f32_16x16x32_bf16 a[64:67], a[132:135], v[84:87], a[64:67]\n\t" \
  "s_cmp_eq_u32 s88, 15\n\t" \
  "s_cselect_b32 s74, 0, s74\n\t" \
  "v_mfma_f32_16x16x32_bf16 a[68:71], a[140:143], v[84:87], a[68:71]\n\t" \
  "ds_read_b64_tr_b16 a[218:219], v15 offset:33792\n\t" \
  "v_add_u32 v8, s74, v8\n\t" \
  "v_mfma_f32_16x16x32_bf16 a[68:71], a[136:139], v[80:83], a[68:71]\n\t" \
  "global_load_dwordx4 v[32:35], v8, s[48:49] nt\n\t" \
  "s_lshl_b32 s98, s74, 10\n\t" \
  "v_mfma_f32_16x16x32_bf16 a[72:75], a[144:147], v[80:83], a[72:75]\n\t" \
  "ds_read_b64_tr_b16 a[220:221], v15 offset:40960\n\t" \
  "v_add_u32 v39, s98, v39\n\t" \
  "v_mfma_f32_16x16x32_bf16 a[72:75], a[148:151], v[84:87], a[72:75]\n\t" \
  "s_xor_b32 s74, s74, s80\n\t" \
  "s_add_u32 s88, s88, 1\n\t" \
  "v_mfma_f32_16x16x32_bf16 a[76:79], a[156:159], v[84:87], a[76:79]\n\t" \
  "ds_read_b64_tr_b16 a[222:223], v15 offset:41984\n\t" \
  "v_mfma_f32_16x16x32_bf16 a[76:79], a[152:155], v[80:83], a[76:79]\n\t" \
  "s_add_u32 m0, s68, 16384\n\t" \
  "v_mfma_f32_16x16x32_bf16 a[80:83], a[160:163], v[80:83], a[80:83]\n\t" \
  "buffer_load_dwordx4 v0, s[36:39], s65 offen offset:0 sc0 sc1 lds\n\t" \
  "v_mfma_f32_16x16x32_bf16 a[80:83], a[164:167], v[84:87], a[80:83]\n\t" \
  "v_mfma_f32_16x16x32_bf16 a[84:87], a[172:175], v[84:87], a[84:87]\n\t" \
  "s_branch RET102_%=\n\t" \
  ".p2align 6\n\t" \
  "PUB103_%=:\n\t" \
  "v_mfma_f32_16x16x32_bf16 a[40:43], a[148:151], v[76:79], a[40:43]\n\t" \
  "ds_read_b64_tr_b16 a[210:211], v14 offset:33792\n\t" \
  "s_mov_b32 s30, 0\n\t" \
  "v_mfma_f32_16x16x32_bf16 a[40:43], a[144:147], v[72:75], a[40:43]\n\t" \
  "s_add_u32 s28, s28, s72\n\t" \
  "v_mfma_f32_16x16x32_bf16 a[36:39], a[136:139], v[72:75], a[36:39]\n\t" \
  "ds_read_b64_tr_b16 a[212:213], v14 offset:40960\n\t" \
  "v_mfma_f32_16x16x32_bf16 a[36:39], a[140:143], v[76:79], a[36:39]\n\t" \
  "s_cmp_eq_u32 s88, 16\n\t" \
  "s_cselect_b32 s87, 0x7fffffff, s28\n\t" \
  "v_mfma_f32_16x16x32_bf16 a[32:35], a[132:135], v[76:79], a[32:35]\n\t" \
  "ds_read_b64_tr_b16 a[214:215], v14 offset:41984\n\t" \
  "s_mov_b32 s72, s73\n\t" \
  "v_mfma_f32_16x16x32_bf16 a[32:35], a[128:131], v[72:75], a[32:35]\n\t" \
  "ds_read_b128 v[72:75], v10 offset:34816\n\t" \
  "ds_read_b128 v[76:79], v11 offset:34816\n\t" \
  "v_mfma_f32_16x16x32_bf16 a[64:67], a[128:131], v[80:83], a[64:67]\n\t" \
  "ds_read_b64_tr_b16 a[216:217], v15 offset:32768\n\t" \
  "s_mov_b32 s73, 0\n\t" \
  "v_mfma_f32_16x16x32_bf16 a[64:67], a[132:135], v[84:87], a[64:67]\n\t" \
  "v_mfma_f32_16x16x32_bf16 a[68:71], a[140:143], v[84:87], a[68:71]\n\t" \
  "ds_read_b64_tr_b16 a[218:219], v15 offset:33792\n\t" \
  "v_mfma_f32_16x16x32_bf16 a[68:71], a[136:139], v[80:83], a[68:71]\n\t" \
  "v_mfma_f32_16x16x32_bf16 a[72:75], a[144:147], v[80:83], a[72:75]\n\t" \
  "ds_read_b64_tr_b16 a[220:221], v15 offset:40960\n\t" \
  "v_mfma_f32_16x16x32_bf16 a[72:75], a[148:151], v[84:87], a[72:75]\n\t" \
  "v_mfma_f32_16x16x32_bf16 a[76:79], a[156:159], v[84:87], a[76:79]\n\t" \
  "ds_read_b64_tr_b16 a[222:223], v15 offset:41984\n\t" \
  "v_mfma_f32_16x16x32_bf16 a[76:79], a[152:155], v[80:83], a[76:79]\n\t" \
  "s_add_u32 m0, s68, 16384\n\t" \
  "v_mfma_f32_16x16x32_bf16 a[80:83], a[160:163], v[80:83], a[80:83]\n\t" \
  "buffer_load_dwordx4 v0, s[36:39], s65 offen offset:0 sc0 sc1 lds\n\t" \
  "v_mfma_f32_16x16x32_bf16 a[80:83], a[164:167], v[84:87], a[80:83]\n\t" \
  "v_mfma_f32_16x16x32_bf16 a[84:87], a[172:175], v[84:87], a[84:87]\n\t" \
  "s_branch RET102_%=\n\t" \
  "SW104_%=:\n\t" \
  "s_mov_b32 s36, s56\n\t" \
  "s_mov_b32 s37, s57\n\t" \
  "s_mov_b32 s38, s63\n\t" \
  "s_mov_b32 s67, 1024\n\t" \
  "s_mov_b32 s89, 0\n\t" \
  "s_lshr_b32 s69, s69, 4\n\t" \
  "s_lshr_b32 s70, s70, 4\n\t" \
  "v_lshrrev_b32 v1, 11, v0\n\t" \
  "v_and_b32 v0, 0x7f, v0\n\t" \
  "v_lshl_add_u32 v0, v1, 7, v0\n\t" \
  "v_xor_b32 v1, 64, v0\n\t" \
  "v_mov_b32 v2, v0\n\t" \
  "v_mov_b32 v3, v1\n\t" \
  "s_branch RET105_%=\n\t" \
  ".p2align 6\n\t" \
  "EVT106_%=:\n\t" \
  "s_cmp_lg_u32 s30, 0\n\t" \
  "s_cbranch_scc1 PUB108_%=\n\t" \
  "global_store_dwordx4 v39, v[32:35], s[52:53] sc0 sc1\n\t" \
  "v_mfma_f32_16x16x32_bf16 v[168:171], a[148:151], v[108:111], v[168:171]\n\t" \
  "ds_read_b64_tr_b16 a[242:243], v14 offset:50176\n\t" \
  "s_and_b32 s87, s26, 15\n\t" \
  "v_mfma_f32_16x16x32_bf16 v[168:171], a[144:147], v[104:107], v[168:171]\n\t" \
  "s_sub_u32 s28, s87, 2\n\t" \
  "v_mfma_f32_16x16x32_bf16 v[164:167], a[136:139], v[104:107], v[164:167]\n\t" \
  "ds_read_b64_tr_b16 a[244:245], v14 offset:57344\n\t" \
  "v_mfma_f32_16x16x32_bf16 v[164:167], a[140:143], v[108:111], v[164:167]\n\t" \
  "s_bitcmp1_b32 32799, s88\n\t" \
  "s_cselect_b32 s87, 1, s87\n\t" \
  "v_mfma_f32_16x16x32_bf16 v[160:163], a[132:135], v[108:111], v[160:163]\n\t" \
  "ds_read_b64_tr_b16 a[246:247], v14 offset:58368\n\t" \
  "s_cselect_b32 s30, 1, 0\n\t" \
  "v_mfma_f32_16x16x32_bf16 v[160:163], a[128:131], v[104:107], v[160:163]\n\t" \
  "ds_read_b128 v[104:107], v10 offset:51200\n\t" \
  "ds_read_b128 v[108:111], v11 offset:51200\n\t" \
  "v_mfma_f32_16x16x32_bf16 v[192:195], a[128:131], v[112:115], v[192:195]\n\t" \
  "ds_read_b64_tr_b16 a[248:249], v15 offset:49152\n\t" \
  "s_lshr_b64 s[26:27], s[26:27], 4\n\t" \
  "v_mfma_f32_16x16x32_bf16 v[192:195], a[132:135], v[116:119], v[192:195]\n\t" \
  "s_cmp_eq_u32 s88, 15\n\t" \
  "s_cselect_b32 s74, 0, s74\n\t" \
  "v_mfma_f32_16x16x32_bf16 v[196:199], a[140:143], v[116:119], v[196:199]\n\t" \
  "ds_read_b64_tr_b16 a[250:251], v15 offset:50176\n\t" \
  "v_add_u32 v8, s74, v8\n\t" \
  "v_mfma_f32_16x16x32_bf16 v[196:199], a[136:139], v[112:115], v[196:199]\n\t" \
  "global_load_dwordx4 v[32:35], v8, s[48:49] nt\n\t" \
  "s_lshl_b32 s98, s74, 10\n\t" \
  "v_mfma_f32_16x16x32_bf16 v[200:203], a[144:147], v[112:115], v[200:203]\n\t" \
  "ds_read_b64_tr_b16 a[252:253], v15 offset:57344\n\t" \
  "v_add_u32 v39, s98, v39\n\t" \
  "v_mfma_f32_16x16x32_bf16 v[200:203], a[148:151], v[116:119], v[200:203]\n\t" \
  "s_xor_b32 s74, s74, s80\n\t" \
  "s_add_u32 s88, s88, 1\n\t" \
  "v_mfma_f32_16x16x32_bf16 v[204:207], a[156:159], v[116:119], v[204:207]\n\t" \
  "ds_read_b64_tr_b16 a[254:255], v15 offset:58368\n\t" \
  "v_mfma_f32_16x16x32_bf16 v[204:207], a[152:155], v[112:115], v[204:207]\n\t" \
  "s_add_u32 m0, s68, 32768\n\t" \
  "v_mfma_f32_16x16x32_bf16 v[208:211], a[160:163], v[112:115], v[208:211]\n\t" \
  "buffer_load_dwordx4 v0, s[36:39], s64 offen offset:0 sc0 sc1 lds\n\t" \
  "v_mfma_f32_16x16x32_bf16 v[208:211], a[164:167], v[116:119], v[208:211]\n\t" \
  "v_mfma_f32_16x16x32_bf16 v[212:215], a[172:175], v[116:119], v[212:215]\n\t" \
  "s_branch RET107_%=\n\t" \
  ".p2align 6\n\t" \
  "PUB108_%=:\n\t" \
  "v_mfma_f32_16x16x32_bf16 v[168:171], a[148:151], v[108:111], v[168:171]\n\t" \
  "ds_read_b64_tr_b16 a[242:243], v14 offset:50176\n\t" \
  "s_mov_b32 s30, 0\n\t" \
  "v_mfma_f32_16x16x32_bf16 v[168:171], a[144:147], v[104:107], v[168:171]\n\t" \
  "s_add_u32 s28, s28, s72\n\t" \
  "v_mfma_f32_16x16x32_bf16 v[164:167], a[136:139], v[104:107], v[164:167]\n\t" \
  "ds_read_b64_tr_b16 a[244:245], v14 offset:57344\n\t" \
  "v_mfma_f32_16x16x32_bf16 v[164:167], a[140:143], v[108:111], v[164:167]\n\t" \
  "s_cmp_eq_u32 s88, 16\n\t" \
  "s_cselect_b32 s87, 0x7fffffff, s28\n\t" \
  "v_mfma_f32_16x16x32_bf16 v[160:163], a[132:135], v[108:111], v[160:163]\n\t" \
  "ds_read_b64_tr_b16 a[246:247], v14 offset:58368\n\t" \
  "s_mov_b32 s72, s73\n\t" \
  "v_mfma_f32_16x16x32_bf16 v[160:163], a[128:131], v[104:107], v[160:163]\n\t" \
  "ds_read_b128 v[104:107], v10 offset:51200\n\t" \
  "ds_read_b128 v[108:111], v11 offset:51200\n\t" \
  "v_mfma_f32_16x16x32_bf16 v[192:195], a[128:131], v[112:115], v[192:195]\n\t" \
  "ds_read_b64_tr_b16 a[248:249], v15 offset:49152\n\t" \
  "s_mov_b32 s73, 0\n\t" \
  "v_mfma_f32_16x16x32_bf16 v[192:195], a[132:135], v[116:119], v[192:195]\n\t" \
  "v_mfma_f32_16x16x32_bf16 v[196:199], a[140:143], v[116:119], v[196:199]\n\t" \
  "ds_read_b64_tr_b16 a[250:251], v15 offset:50176\n\t" \
  "v_mfma_f32_16x16x32_bf16 v[196:199], a[136:139], v[112:115], v[196:199]\n\t" \
  "v_mfma_f32_16x16x32_bf16 v[200:203], a[144:147], v[112:115], v[200:203]\n\t" \
  "ds_read_b64_tr_b16 a[252:253], v15 offset:57344\n\t" \
  "v_mfma_f32_16x16x32_bf16 v[200:203], a[148:151], v[116:119], v[200:203]\n\t" \
  "v_mfma_f32_16x16x32_bf16 v[204:207], a[156:159], v[116:119], v[204:207]\n\t" \
  "ds_read_b64_tr_b16 a[254:255], v15 offset:58368\n\t" \
  "v_mfma_f32_16x16x32_bf16 v[204:207], a[152:155], v[112:115], v[204:207]\n\t" \
  "s_add_u32 m0, s68, 32768\n\t" \
  "v_mfma_f32_16x16x32_bf16 v[208:211], a[160:163], v[112:115], v[208:211]\n\t" \
  "buffer_load_dwordx4 v0, s[36:39], s64 offen offset:0 sc0 sc1 lds\n\t" \
  "v_mfma_f32_16x16x32_bf16 v[208:211], a[164:167], v[116:119], v[208:211]\n\t" \
  "v_mfma_f32_16x16x32_bf16 v[212:215], a[172:175], v[116:119], v[212:215]\n\t" \
  "s_branch RET107_%=\n\t" \
  ".p2align 6\n\t" \
  "EVT109_%=:\n\t" \
  "s_cmp_lg_u32 s30, 0\n\t" \
  "s_cbranch_scc1 PUB111_%=\n\t" \
  "global_store_dwordx4 v39, v[32:35], s[52:53] sc0 sc1\n\t" \
  "v_mfma_f32_16x16x32_bf16 a[40:43], a[212:215], v[76:79], a[40:43]\n\t" \
  "ds_read_b64_tr_b16 a[146:147], v14 offset:1024\n\t" \
  "s_and_b32 s87, s26, 15\n\t" \
  "v_mfma_f32_16x16x32_bf16 a[40:43], a[208:211], v[72:75], a[40:43]\n\t" \
  "s_sub_u32 s28, s87, 2\n\t" \
  "v_mfma_f32_16x16x32_bf16 a[36:39], a[200:203], v[72:75], a[36:39]\n\t" \
  "ds_read_b64_tr_b16 a[148:149], v14 offset:8192\n\t" \
  "v_mfma_f32_16x16x32_bf16 a[36:39], a[204:207], v[76:79], a[36:39]\n\t" \
  "s_bitcmp1_b32 32799, s88\n\t" \
  "s_cselect_b32 s87, 1, s87\n\t" \
  "v_mfma_f32_16x16x32_bf16 a[32:35], a[196:199], v[76:79], a[32:35]\n\t" \
  "ds_read_b64_tr_b16 a[150:151], v14 offset:9216\n\t" \
  "s_cselect_b32 s30, 1, 0\n\t" \
  "v_mfma_f32_16x16x32_bf16 a[32:35], a[192:195], v[72:75], a[32:35]\n\t" \
  "ds_read_b128 v[72:75], v10 offset:2048\n\t" \
  "ds_read_b128 v[76:79], v11 offset:2048\n\t" \
  "v_mfma_f32_16x16x32_bf16 a[64:67], a[192:195], v[80:83], a[64:67]\n\t" \
  "ds_read_b64_tr_b16 a[152:153], v15 offset:0\n\t" \
  "s_lshr_b64 s[26:27], s[26:27], 4\n\t" \
  "v_mfma_f32_16x16x32_bf16 a[64:67], a[196:199], v[84:87], a[64:67]\n\t" \
  "s_cmp_eq_u32 s88, 15\n\t" \
  "s_cselect_b32 s74, 0, s74\n\t" \
  "v_mfma_f32_16x16x32_bf16 a[68:71], a[204:207], v[84:87], a[68:71]\n\t" \
  "ds_read_b64_tr_b16 a[154:155], v15 offset:1024\n\t" \
  "v_add_u32 v8, s74, v8\n\t" \
  "v_mfma_f32_16x16x32_bf16 a[68:71], a[200:203], v[80:83], a[68:71]\n\t" \
  "global_load_dwordx4 v[32:35], v8, s[48:49] nt\n\t" \
  "s_lshl_b32 s98, s74, 10\n\t" \
  "v_mfma_f32_16x16x32_bf16 a[72:75], a[208:211], v[80:83], a[72:75]\n\t" \
  "ds_read_b64_tr_b16 a[156:157], v15 offset:8192\n\t" \
  "v_add_u32 v39, s98, v39\n\t" \
  "v_mfma_f32_16x16x32_bf16 a[72:75], a[212:215], v[84:87], a[72:75]\n\t" \
  "s_xor_b32 s74, s74, s80\n\t" \
  "s_add_u32 s88, s88, 1\n\t" \
  "v_mfma_f32_16x16x32_bf16 a[76:79], a[220:223], v[84:87], a[76:79]\n\t" \
  "ds_read_b64_tr_b16 a[158:159], v15 offset:9216\n\t" \
  "v_mfma_f32_16x16x32_bf16 a[76:79], a[216:219], v[80:83], a[76:79]\n\t" \
  "s_add_u32 m0, s68, 49152\n\t" \
  "v_mfma_f32_16x16x32_bf16 a[80:83], a[224:227], v[80:83], a[80:83]\n\t" \
  "buffer_load_dwordx4 v0, s[36:39], s65 offen offset:0 sc0 sc1 lds\n\t" \
  "v_mfma_f32_16x16x32_bf16 a[80:83], a[228:231], v[84:87], a[80:83]\n\t" \
  "v_mfma_f32_16x16x32_bf16 a[84:87], a[236:239], v[84:87], a[84:87]\n\t" \
  "s_branch RET110_%=\n\t" \
  ".p2align 6\n\t" \
  "PUB111_%=:\n\t" \
  "v_mfma_f32_16x16x32_bf16 a[40:43], a[212:215], v[76:79], a[40:43]\n\t" \
  "ds_read_b64_tr_b16 a[146:147], v14 offset:1024\n\t" \
  "s_mov_b32 s30, 0\n\t" \
  "v_mfma_f32_16x16x32_bf16 a[40:43], a[208:211], v[72:75], a[40:43]\n\t" \
  "s_add_u32 s28, s28, s72\n\t" \
  "v_mfma_f32_16x16x32_bf16 a[36:39], a[200:203], v[72:75], a[36:39]\n\t" \
  "ds_read_b64_tr_b16 a[148:149], v14 offset:8192\n\t" \
  "v_mfma_f32_16x16x32_bf16 a[36:39], a[204:207], v[76:79], a[36:39]\n\t" \
  "s_cmp_eq_u32 s88, 16\n\t" \
  "s_cselect_b32 s87, 0x7fffffff, s28\n\t" \
  "v_mfma_f32_16x16x32_bf16 a[32:35], a[196:199], v[76:79], a[32:35]\n\t" \
  "ds_read_b64_tr_b16 a[150:151], v14 offset:9216\n\t" \
  "s_mov_b32 s72, s73\n\t" \
  "v_mfma_f32_16x16x32_bf16 a[32:35], a[192:195], v[72:75], a[32:35]\n\t" \
  "ds_read_b128 v[72:75], v10 offset:2048\n\t" \
  "ds_read_b128 v[76:79], v11 offset:2048\n\t" \
  "v_mfma_f32_16x16x32_bf16 a[64:67], a[192:195], v[80:83], a[64:67]\n\t" \
  "ds_read_b64_tr_b16 a[152:153], v15 offset:0\n\t" \
  "s_mov_b32 s73, 0\n\t" \
  "v_mfma_f32_16x16x32_bf16 a[64:67], a[196:199], v[84:87], a[64:67]\n\t" \
  "v_mfma_f32_16x16x32_bf16 a[68:71], a[204:207], v[84:87], a[68:71]\n\t" \
  "ds_read_b64_tr_b16 a[154:155], v15 offset:1024\n\t" \
  "v_mfma_f32_16x16x32_bf16 a[68:71], a[200:203], v[80:83], a[68:71]\n\t" \
  "v_mfma_f32_16x16x32_bf16 a[72:75], a[208:211], v[80:83], a[72:75]\n\t" \
  "ds_read_b64_tr_b16 a[156:157], v15 offset:8192\n\t" \
  "v_mfma_f32_16x16x32_bf16 a[72:75], a[212:215], v[84:87], a[72:75]\n\t" \
  "v_mfma_f32_16x16x32_bf16 a[76:79], a[220:223], v[84:87], a[76:79]\n\t" \
  "ds_read_b64_tr_b16 a[158:159], v15 offset:9216\n\t" \
  "v_mfma_f32_16x16x32_bf16 a[76:79], a[216:219], v[80:83], a[76:79]\n\t" \
  "s_add_u32 m0, s68, 49152\n\t" \
  "v_mfma_f32_16x16x32_bf16 a[80:83], a[224:227], v[80:83], a[80:83]\n\t" \
  "buffer_load_dwordx4 v0, s[36:39], s65 offen offset:0 sc0 sc1 lds\n\t" \
  "v_mfma_f32_16x16x32_bf16 a[80:83], a[228:231], v[84:87], a[80:83]\n\t" \
  "v_mfma_f32_16x16x32_bf16 a[84:87], a[236:239], v[84:87], a[84:87]\n\t" \
  "s_branch RET110_%=\n\t" \
  "SW112_%=:\n\t" \
  "s_mov_b32 s36, s56\n\t" \
  "s_mov_b32 s37, s57\n\t" \
  "s_mov_b32 s38, s63\n\t" \
  "s_mov_b32 s67, 1024\n\t" \
  "s_mov_b32 s89, 0\n\t" \
  "s_lshr_b32 s69, s69, 4\n\t" \
  "s_lshr_b32 s70, s70, 4\n\t" \
  "v_lshrrev_b32 v1, 11, v0\n\t" \
  "v_and_b32 v0, 0x7f, v0\n\t" \
  "v_lshl_add_u32 v0, v1, 7, v0\n\t" \
  "v_xor_b32 v1, 64, v0\n\t" \
  "v_mov_b32 v2, v0\n\t" \
  "v_mov_b32 v3, v1\n\t" \
  "s_branch RET113_%=\n\t" \
  "TRG116_%=:\n\t" \
  "s_cmp_eq_u32 s76, 4\n\t" \
  "s_cbranch_scc1 TRGL118_%=\n\t" \
  "s_branch TRGR117_%=\n\t" \
  "s_memrealtime s[96:97]\n\t" \
  "s_waitcnt lgkmcnt(0)\n\t" \
  "v_mov_b32 v46, s96\n\t" \
  "v_mov_b32 v47, s85\n\t" \
  "s_mov_b64 exec, 1\n\t" \
  "global_store_dwordx2 v22, v[46:47], s[50:51] sc0 sc1\n\t" \
  "s_mov_b64 exec, -1\n\t" \
  "s_branch TRGR117_%=\n\t" \
  "TRGL118_%=:\n\t" \
  "s_mov_b64 exec, 3\n\t" \
  "global_load_dwordx2 v[20:21], v22, s[50:51] sc0 sc1\n\t" \
  "s_mov_b64 exec, -1\n\t" \
  "s_branch TRGR117_%=\n\t" \
  ".p2align 6\n\t" \
  "EVT114_%=:\n\t" \
  "s_cmp_lg_u32 s30, 0\n\t" \
  "s_cbranch_scc1 PUB119_%=\n\t" \
  "global_store_dwordx4 v39, v[32:35], s[52:53] sc0 sc1\n\t" \
  "v_mfma_f32_16x16x32_bf16 v[168:171], a[212:215], v[108:111], v[168:171]\n\t" \
  "ds_read_b64_tr_b16 a[178:179], v14 offset:17408\n\t" \
  "s_and_b32 s87, s26, 15\n\t" \
  "v_mfma_f32_16x16x32_bf16 v[168:171], a[208:211], v[104:107], v[168:171]\n\t" \
  "s_sub_u32 s28, s87, 2\n\t" \
  "v_mfma_f32_16x16x32_bf16 v[164:167], a[200:203], v[104:107], v[164:167]\n\t" \
  "ds_read_b64_tr_b16 a[180:181], v14 offset:24576\n\t" \
  "v_mfma_f32_16x16x32_bf16 v[164:167], a[204:207], v[108:111], v[164:167]\n\t" \
  "s_bitcmp1_b32 32799, s88\n\t" \
  "s_cselect_b32 s87, 1, s87\n\t" \
  "v_mfma_f32_16x16x32_bf16 v[160:163], a[196:199], v[108:111], v[160:163]\n\t" \
  "ds_read_b64_tr_b16 a[182:183], v14 offset:25600\n\t" \
  "s_cselect_b32 s30, 1, 0\n\t" \
  "v_mfma_f32_16x16x32_bf16 v[160:163], a[192:195], v[104:107], v[160:163]\n\t" \
  "ds_read_b128 v[104:107], v10 offset:18432\n\t" \
  "ds_read_b128 v[108:111], v11 offset:18432\n\t" \
  "v_mfma_f32_16x16x32_bf16 v[192:195], a[192:195], v[112:115], v[192:195]\n\t" \
  "ds_read_b64_tr_b16 a[184:185], v15 offset:16384\n\t" \
  "s_lshr_b64 s[26:27], s[26:27], 4\n\t" \
  "v_mfma_f32_16x16x32_bf16 v[192:195], a[196:199], v[116:119], v[192:195]\n\t" \
  "s_cmp_eq_u32 s88, 15\n\t" \
  "s_cselect_b32 s74, 0, s74\n\t" \
  "v_mfma_f32_16x16x32_bf16 v[196:199], a[204:207], v[116:119], v[196:199]\n\t" \
  "ds_read_b64_tr_b16 a[186:187], v15 offset:17408\n\t" \
  "v_add_u32 v8, s74, v8\n\t" \
  "v_mfma_f32_16x16x32_bf16 v[196:199], a[200:203], v[112:115], v[196:199]\n\t" \
  "global_load_dwordx4 v[32:35], v8, s[48:49] nt\n\t" \
  "s_lshl_b32 s98, s74, 10\n\t" \
  "v_mfma_f32_16x16x32_bf16 v[200:203], a[208:211], v[112:115], v[200:203]\n\t" \
  "ds_read_b64_tr_b16 a[188:189], v15 offset:24576\n\t" \
  "v_add_u32 v39, s98, v39\n\t" \
  "v_mfma_f32_16x16x32_bf16 v[200:203], a[212:215], v[116:119], v[200:203]\n\t" \
  "s_xor_b32 s74, s74, s80\n\t" \
  "s_add_u32 s88, s88, 1\n\t" \
  "v_mfma_f32_16x16x32_bf16 v[204:207], a[220:223], v[116:119], v[204:207]\n\t" \
  "ds_read_b64_tr_b16 a[190:191], v15 offset:25600\n\t" \
  "v_mfma_f32_16x16x32_bf16 v[204:207], a[216:219], v[112:115], v[204:207]\n\t" \
  "s_add_u32 m0, s68, 0\n\t" \
  "v_mfma_f32_16x16x32_bf16 v[208:211], a[224:227], v[112:115], v[208:211]\n\t" \
  "buffer_load_dwordx4 v0, s[36:39], s64 offen offset:0 sc0 sc1 lds\n\t" \
  "v_mfma_f32_16x16x32_bf16 v[208:211], a[228:231], v[116:119], v[208:211]\n\t" \
  "v_mfma_f32_16x16x32_bf16 v[212:215], a[236:239], v[116:119], v[212:215]\n\t" \
  "s_branch RET115_%=\n\t" \
  ".p2align 6\n\t" \
  "PUB119_%=:\n\t" \
  "v_mfma_f32_16x16x32_bf16 v[168:171], a[212:215], v[108:111], v[168:171]\n\t" \
  "ds_read_b64_tr_b16 a[178:179], v14 offset:17408\n\t" \
  "s_mov_b32 s30, 0\n\t" \
  "v_mfma_f32_16x16x32_bf16 v[168:171], a[208:211], v[104:107], v[168:171]\n\t" \
  "s_add_u32 s28, s28, s72\n\t" \
  "v_mfma_f32_16x16x32_bf16 v[164:167], a[200:203], v[104:107], v[164:167]\n\t" \
  "ds_read_b64_tr_b16 a[180:181], v14 offset:24576\n\t" \
  "v_mfma_f32_16x16x32_bf16 v[164:167], a[204:207], v[108:111], v[164:167]\n\t" \
  "s_cmp_eq_u32 s88, 16\n\t" \
  "s_cselect_b32 s87, 0x7fffffff, s28\n\t" \
  "v_mfma_f32_16x16x32_bf16 v[160:163], a[196:199], v[108:111], v[160:163]\n\t" \
  "ds_read_b64_tr_b16 a[182:183], v14 offset:25600\n\t" \
  "s_mov_b32 s72, s73\n\t" \
  "v_mfma_f32_16x16x32_bf16 v[160:163], a[192:195], v[104:107], v[160:163]\n\t" \
  "ds_read_b128 v[104:107], v10 offset:18432\n\t" \
  "ds_read_b128 v[108:111], v11 offset:18432\n\t" \
  "v_mfma_f32_16x16x32_bf16 v[192:195], a[192:195], v[112:115], v[192:195]\n\t" \
  "ds_read_b64_tr_b16 a[184:185], v15 offset:16384\n\t" \
  "s_mov_b32 s73, 0\n\t" \
  "v_mfma_f32_16x16x32_bf16 v[192:195], a[196:199], v[116:119], v[192:195]\n\t" \
  "v_mfma_f32_16x16x32_bf16 v[196:199], a[204:207], v[116:119], v[196:199]\n\t" \
  "ds_read_b64_tr_b16 a[186:187], v15 offset:17408\n\t" \
  "v_mfma_f32_16x16x32_bf16 v[196:199], a[200:203], v[112:115], v[196:199]\n\t" \
  "v_mfma_f32_16x16x32_bf16 v[200:203], a[208:211], v[112:115], v[200:203]\n\t" \
  "ds_read_b64_tr_b16 a[188:189], v15 offset:24576\n\t" \
  "v_mfma_f32_16x16x32_bf16 v[200:203], a[212:215], v[116:119], v[200:203]\n\t" \
  "v_mfma_f32_16x16x32_bf16 v[204:207], a[220:223], v[116:119], v[204:207]\n\t" \
  "ds_read_b64_tr_b16 a[190:191], v15 offset:25600\n\t" \
  "v_mfma_f32_16x16x32_bf16 v[204:207], a[216:219], v[112:115], v[204:207]\n\t" \
  "s_add_u32 m0, s68, 0\n\t" \
  "v_mfma_f32_16x16x32_bf16 v[208:211], a[224:227], v[112:115], v[208:211]\n\t" \
  "buffer_load_dwordx4 v0, s[36:39], s64 offen offset:0 sc0 sc1 lds\n\t" \
  "v_mfma_f32_16x16x32_bf16 v[208:211], a[228:231], v[116:119], v[208:211]\n\t" \
  "v_mfma_f32_16x16x32_bf16 v[212:215], a[236:239], v[116:119], v[212:215]\n\t" \
  "s_branch RET115_%=\n\t" \
  "DSLOW_%=:\n\t" \
  "s_memrealtime s[94:95]\n\t" \
  "s_waitcnt lgkmcnt(0)\n\t" \
  "DSL_%=:\n\t" \
  "s_mov_b64 exec, 3\n\t" \
  "global_load_dwordx2 v[20:21], v22, s[50:51] sc0 sc1\n\t" \
  "s_mov_b64 exec, -1\n\t" \
  "global_load_dword v47, v30, s[50:51] offset:-128 sc1\n\t" \
  "s_waitcnt vmcnt(0)\n\t" \
  "v_readlane_b32 s98, v21, 0\n\t" \
  "v_readlane_b32 s99, v21, 1\n\t" \
  "s_nop 1\n\t" \
  "s_min_u32 s99, s98, s99\n\t" \
  "v_readfirstlane_b32 s98, v47\n\t" \
  "s_nop 1\n\t" \
  "s_cmp_ge_u32 s99, s85\n\t" \
  "s_cbranch_scc1 DEC_%=\n\t" \
  "s_cmp_eq_u32 s98, s85\n\t" \
  "s_cbranch_scc1 DSB_%=\n\t" \
  "s_memrealtime s[96:97]\n\t" \
  "s_waitcnt lgkmcnt(0)\n\t" \
  "s_sub_u32 s96, s96, s94\n\t" \
  "s_cmp_gt_u32 s96, 50000000\n\t" \
  "s_cbranch_scc1 DSB_%=\n\t" \
  "s_sleep 2\n\t" \
  "s_branch DSL_%=\n\t" \
  "DSB_%=:\n\t" \
  "v_mov_b32 v47, s85\n\t" \
  "global_store_dword v30, v47, s[50:51] offset:-128 sc1\n\t" \
  "s_mov_b32 s98, 0\n\t" \
  "s_waitcnt vmcnt(0)\n\t" \
  "s_branch DECB_%=\n\t" \
  "SLOW121_%=:\n\t" \
  "s_memrealtime s[94:95]\n\t" \
  "s_waitcnt lgkmcnt(0)\n\t" \
  "SL122_%=:\n\t" \
  "v_mov_b32 v46, 0\n\t" \
  "s_lshl_b64 exec, 1, s99\n\t" \
  "global_load_dword v46, v16, s[50:51] sc0 sc1\n\t" \
  "s_mov_b64 exec, 1\n\t" \
  "global_load_dword v47, v30, s[50:51] offset:-128 sc1\n\t" \
  "s_mov_b64 exec, -1\n\t" \
  "s_waitcnt vmcnt(0)\n\t" \
  "v_cmp_le_u32 s[92:93], s86, v46\n\t" \
  "v_readfirstlane_b32 s98, v47\n\t" \
  "s_nop 4\n\t" \
  "s_or_b64 s[90:91], s[90:91], s[92:93]\n\t" \
  "s_bitcmp1_b64 s[90:91], s99\n\t" \
  "s_cbranch_scc1 RET120_%=\n\t" \
  "s_cmp_eq_u32 s98, s85\n\t" \
  "s_cbranch_scc1 SB123_%=\n\t" \
  "s_memrealtime s[96:97]\n\t" \
  "s_waitcnt lgkmcnt(0)\n\t" \
  "s_sub_u32 s96, s96, s94\n\t" \
  "s_cmp_gt_u32 s96, 50000000\n\t" \
  "s_cbranch_scc1 SB123_%=\n\t" \
  "s_sleep 2\n\t" \
  "s_branch SL122_%=\n\t" \
  "SB123_%=:\n\t" \
  "v_mov_b32 v47, s85\n\t" \
  "global_store_dword v30, v47, s[50:51] offset:-128 sc1\n\t" \
  "s_mov_b64 s[90:91], -1\n\t" \
  "s_mov_b64 s[58:59], -1\n\t" \
  "s_waitcnt vmcnt(0)\n\t" \
  "s_branch RET120_%=\n\t" \
  "POLL125_%=:\n\t" \
  "s_lshl_b64 exec, 1, s99\n\t" \
  "global_load_dword v36, v16, s[50:51] sc0 sc1\n\t" \
  "s_mov_b64 exec, -1\n\t" \
  "s_branch RET124_%=\n\t" \
  "STG126_%=:\n\t" \
  "s_waitcnt vmcnt(5)\n\t" \
  "v_cmp_le_u32 s[92:93], s86, v36\n\t" \
  "s_nop 3\n\t" \
  "s_or_b64 s[90:91], s[90:91], s[92:93]\n\t" \
  "s_bitcmp1_b64 s[90:91], s99\n\t" \
  "s_cbranch_scc1 RET120_%=\n\t" \
  "s_branch SLOW121_%=\n\t" \
  "POLL128_%=:\n\t" \
  "s_mov_b64 exec, s[92:93]\n\t" \
  "global_load_dword v37, v17, s[50:51] sc0 sc1\n\t" \
  "s_mov_b64 exec, -1\n\t" \
  "s_branch RET127_%=\n\t" \
  "POLL132_%=:\n\t" \
  "s_mov_b64 exec, s[92:93]\n\t" \
  "global_load_dword v37, v17, s[50:51] sc0 sc1\n\t" \
  "s_mov_b64 exec, -1\n\t" \
  "s_branch RET131_%=\n\t" \
  "SLOW134_%=:\n\t" \
  "s_memrealtime s[94:95]\n\t" \
  "s_waitcnt lgkmcnt(0)\n\t" \
  "SL135_%=:\n\t" \
  "v_mov_b32 v46, 0\n\t" \
  "s_lshl_b64 exec, 1, 0\n\t" \
  "global_load_dword v46, v17, s[50:51] sc0 sc1\n\t" \
  "s_mov_b64 exec, 1\n\t" \
  "global_load_dword v47, v30, s[50:51] offset:-128 sc1\n\t" \
  "s_mov_b64 exec, -1\n\t" \
  "s_waitcnt vmcnt(0)\n\t" \
  "v_cmp_le_u32 s[92:93], s86, v46\n\t" \
  "v_readfirstlane_b32 s98, v47\n\t" \
  "s_nop 4\n\t" \
  "s_or_b64 s[58:59], s[58:59], s[92:93]\n\t" \
  "s_bitcmp1_b64 s[58:59], 0\n\t" \
  "s_cbranch_scc1 RET133_%=\n\t" \
  "s_cmp_eq_u32 s98, s85\n\t" \
  "s_cbranch_scc1 SB136_%=\n\t" \
  "s_memrealtime s[96:97]\n\t" \
  "s_waitcnt lgkmcnt(0)\n\t" \
  "s_sub_u32 s96, s96, s94\n\t" \
  "s_cmp_gt_u32 s96, 50000000\n\t" \
  "s_cbranch_scc1 SB136_%=\n\t" \
  "s_sleep 2\n\t" \
  "s_branch SL135_%=\n\t" \
  "SB136_%=:\n\t" \
  "v_mov_b32 v47, s85\n\t" \
  "global_store_dword v30, v47, s[50:51] offset:-128 sc1\n\t" \
  "s_mov_b64 s[90:91], -1\n\t" \
  "s_mov_b64 s[58:59], -1\n\t" \
  "s_waitcnt vmcnt(0)\n\t" \
  "s_branch RET133_%=\n\t" \
  "POLL138_%=:\n\t" \
  "s_mov_b64 exec, s[92:93]\n\t" \
  "global_load_dword v37, v17, s[50:51] sc0 sc1\n\t" \
  "s_mov_b64 exec, -1\n\t" \
  "s_branch RET137_%=\n\t" \
  "SLOW140_%=:\n\t" \
  "s_memrealtime s[94:95]\n\t" \
  "s_waitcnt lgkmcnt(0)\n\t" \
  "SL141_%=:\n\t" \
  "v_mov_b32 v46, 0\n\t" \
  "s_lshl_b64 exec, 1, 1\n\t" \
  "global_load_dword v46, v17, s[50:51] sc0 sc1\n\t" \
  "s_mov_b64 exec, 1\n\t" \
  "global_load_dword v47, v30, s[50:51] offset:-128 sc1\n\t" \
  "s_mov_b64 exec, -1\n\t" \
  "s_waitcnt vmcnt(0)\n\t" \
  "v_cmp_le_u32 s[92:93], s86, v46\n\t" \
  "v_readfirstlane_b32 s98, v47\n\t" \
  "s_nop 4\n\t" \
  "s_or_b64 s[58:59], s[58:59], s[92:93]\n\t" \
  "s_bitcmp1_b64 s[58:59], 1\n\t" \
  "s_cbranch_scc1 RET139_%=\n\t" \
  "s_cmp_eq_u32 s98, s85\n\t" \
  "s_cbranch_scc1 SB142_%=\n\t" \
  "s_memrealtime s[96:97]\n\t" \
  "s_waitcnt lgkmcnt(0)\n\t" \
  "s_sub_u32 s96, s96, s94\n\t" \
  "s_cmp_gt_u32 s96, 50000000\n\t" \
  "s_cbranch_scc1 SB142_%=\n\t" \
  "s_sleep 2\n\t" \
  "s_branch SL141_%=\n\t" \
  "SB142_%=:\n\t" \
  "v_mov_b32 v47, s85\n\t" \
  "global_store_dword v30, v47, s[50:51] offset:-128 sc1\n\t" \
  "s_mov_b64 s[90:91], -1\n\t" \
  "s_mov_b64 s[58:59], -1\n\t" \
  "s_waitcnt vmcnt(0)\n\t" \
  "s_branch RET139_%=\n\t" \
  "SLOW144_%=:\n\t" \
  "s_memrealtime s[94:95]\n\t" \
  "s_waitcnt lgkmcnt(0)\n\t" \
  "SL145_%=:\n\t" \
  "v_mov_b32 v46, 0\n\t" \
  "s_lshl_b64 exec, 1, 2\n\t" \
  "global_load_dword v46, v17, s[50:51] sc0 sc1\n\t" \
  "s_mov_b64 exec, 1\n\t" \
  "global_load_dword v47, v30, s[50:51] offset:-128 sc1\n\t" \
  "s_mov_b64 exec, -1\n\t" \
  "s_waitcnt vmcnt(0)\n\t" \
  "v_cmp_le_u32 s[92:93], s86, v46\n\t" \
  "v_readfirstlane_b32 s98, v47\n\t" \
  "s_nop 4\n\t" \
  "s_or_b64 s[58:59], s[58:59], s[92:93]\n\t" \
  "s_bitcmp1_b64 s[58:59], 2\n\t" \
  "s_cbranch_scc1 RET143_%=\n\t" \
  "s_cmp_eq_u32 s98, s85\n\t" \
  "s_cbranch_scc1 SB146_%=\n\t" \
  "s_memrealtime s[96:97]\n\t" \
  "s_waitcnt lgkmcnt(0)\n\t" \
  "s_sub_u32 s96, s96, s94\n\t" \
  "s_cmp_gt_u32 s96, 50000000\n\t" \
  "s_cbranch_scc1 SB146_%=\n\t" \
  "s_sleep 2\n\t" \
  "s_branch SL145_%=\n\t" \
  "SB146_%=:\n\t" \
  "v_mov_b32 v47, s85\n\t" \
  "global_store_dword v30, v47, s[50:51] offset:-128 sc1\n\t" \
  "s_mov_b64 s[90:91], -1\n\t" \
  "s_mov_b64 s[58:59], -1\n\t" \
  "s_waitcnt vmcnt(0)\n\t" \
  "s_branch RET143_%=\n\t" \
  "POLL148_%=:\n\t" \
  "s_mov_b64 exec, s[92:93]\n\t" \
  "global_load_dword v37, v17, s[50:51] sc0 sc1\n\t" \
  "s_mov_b64 exec, -1\n\t" \
  "s_branch RET147_%=\n\t" \
  "SLOW150_%=:\n\t" \
  "s_memrealtime s[94:95]\n\t" \
  "s_waitcnt lgkmcnt(0)\n\t" \
  "SL151_%=:\n\t" \
  "v_mov_b32 v46, 0\n\t" \
  "s_lshl_b64 exec, 1, 3\n\t" \
  "global_load_dword v46, v17, s[50:51] sc0 sc1\n\t" \
  "s_mov_b64 exec, 1\n\t" \
  "global_load_dword v47, v30, s[50:51] offset:-128 sc1\n\t" \
  "s_mov_b64 exec, -1\n\t" \
  "s_waitcnt vmcnt(0)\n\t" \
  "v_cmp_le_u32 s[92:93], s86, v46\n\t" \
  "v_readfirstlane_b32 s98, v47\n\t" \
  "s_nop 4\n\t" \
  "s_or_b64 s[58:59], s[58:59], s[92:93]\n\t" \
  "s_bitcmp1_b64 s[58:59], 3\n\t" \
  "s_cbranch_scc1 RET149_%=\n\t" \
  "s_cmp_eq_u32 s98, s85\n\t" \
  "s_cbranch_scc1 SB152_%=\n\t" \
  "s_memrealtime s[96:97]\n\t" \
  "s_waitcnt lgkmcnt(0)\n\t" \
  "s_sub_u32 s96, s96, s94\n\t" \
  "s_cmp_gt_u32 s96, 50000000\n\t" \
  "s_cbranch_scc1 SB152_%=\n\t" \
  "s_sleep 2\n\t" \
  "s_branch SL151_%=\n\t" \
  "SB152_%=:\n\t" \
  "v_mov_b32 v47, s85\n\t" \
  "global_store_dword v30, v47, s[50:51] offset:-128 sc1\n\t" \
  "s_mov_b64 s[90:91], -1\n\t" \
  "s_mov_b64 s[58:59], -1\n\t" \
  "s_waitcnt vmcnt(0)\n\t" \
  "s_branch RET149_%=\n\t" \
  "POLL154_%=:\n\t" \
  "s_mov_b64 exec, s[92:93]\n\t" \
  "global_load_dword v37, v17, s[50:51] sc0 sc1\n\t" \
  "s_mov_b64 exec, -1\n\t" \
  "s_branch RET153_%=\n\t" \
  "SLOW156_%=:\n\t" \
  "s_memrealtime s[94:95]\n\t" \
  "s_waitcnt lgkmcnt(0)\n\t" \
  "SL157_%=:\n\t" \
  "v_mov_b32 v46, 0\n\t" \
  "s_lshl_b64 exec, 1, 4\n\t" \
  "global_load_dword v46, v17, s[50:51] sc0 sc1\n\t" \
  "s_mov_b64 exec, 1\n\t" \
  "global_load_dword v47, v30, s[50:51] offset:-128 sc1\n\t" \
  "s_mov_b64 exec, -1\n\t" \
  "s_waitcnt vmcnt(0)\n\t" \
  "v_cmp_le_u32 s[92:93], s86, v46\n\t" \
  "v_readfirstlane_b32 s98, v47\n\t" \
  "s_nop 4\n\t" \
  "s_or_b64 s[58:59], s[58:59], s[92:93]\n\t" \
  "s_bitcmp1_b64 s[58:59], 4\n\t" \
  "s_cbranch_scc1 RET155_%=\n\t" \
  "s_cmp_eq_u32 s98, s85\n\t" \
  "s_cbranch_scc1 SB158_%=\n\t" \
  "s_memrealtime s[96:97]\n\t" \
  "s_waitcnt lgkmcnt(0)\n\t" \
  "s_sub_u32 s96, s96, s94\n\t" \
  "s_cmp_gt_u32 s96, 50000000\n\t" \
  "s_cbranch_scc1 SB158_%=\n\t" \
  "s_sleep 2\n\t" \
  "s_branch SL157_%=\n\t" \
  "SB158_%=:\n\t" \
  "v_mov_b32 v47, s85\n\t" \
  "global_store_dword v30, v47, s[50:51] offset:-128 sc1\n\t" \
  "s_mov_b64 s[90:91], -1\n\t" \
  "s_mov_b64 s[58:59], -1\n\t" \
  "s_waitcnt vmcnt(0)\n\t" \
  "s_branch RET155_%=\n\t" \
  "SLOW160_%=:\n\t" \
  "s_memrealtime s[94:95]\n\t" \
  "s_waitcnt lgkmcnt(0)\n\t" \
  "SL161_%=:\n\t" \
  "v_mov_b32 v46, 0\n\t" \
  "s_lshl_b64 exec, 1, 5\n\t" \
  "global_load_dword v46, v17, s[50:51] sc0 sc1\n\t" \
  "s_mov_b64 exec, 1\n\t" \
  "global_load_dword v47, v30, s[50:51] offset:-128 sc1\n\t" \
  "s_mov_b64 exec, -1\n\t" \
  "s_waitcnt vmcnt(0)\n\t" \
  "v_cmp_le_u32 s[92:93], s86, v46\n\t" \
  "v_readfirstlane_b32 s98, v47\n\t" \
  "s_nop 4\n\t" \
  "s_or_b64 s[58:59], s[58:59], s[92:93]\n\t" \
  "s_bitcmp1_b64 s[58:59], 5\n\t" \
  "s_cbranch_scc1 RET159_%=\n\t" \
  "s_cmp_eq_u32 s98, s85\n\t" \
  "s_cbranch_scc1 SB162_%=\n\t" \
  "s_memrealtime s[96:97]\n\t" \
  "s_waitcnt lgkmcnt(0)\n\t" \
  "s_sub_u32 s96, s96, s94\n\t" \
  "s_cmp_gt_u32 s96, 50000000\n\t" \
  "s_cbranch_scc1 SB162_%=\n\t" \
  "s_sleep 2\n\t" \
  "s_branch SL161_%=\n\t" \
  "SB162_%=:\n\t" \
  "v_mov_b32 v47, s85\n\t" \
  "global_store_dword v30, v47, s[50:51] offset:-128 sc1\n\t" \
  "s_mov_b64 s[90:91], -1\n\t" \
  "s_mov_b64 s[58:59], -1\n\t" \
  "s_waitcnt vmcnt(0)\n\t" \
  "s_branch RET159_%=\n\t" \
  "SLOW164_%=:\n\t" \
  "s_memrealtime s[94:95]\n\t" \
  "s_waitcnt lgkmcnt(0)\n\t" \
  "SL165_%=:\n\t" \
  "v_mov_b32 v46, 0\n\t" \
  "s_lshl_b64 exec, 1, 6\n\t" \
  "global_load_dword v46, v17, s[50:51] sc0 sc1\n\t" \
  "s_mov_b64 exec, 1\n\t" \
  "global_load_dword v47, v30, s[50:51] offset:-128 sc1\n\t" \
  "s_mov_b64 exec, -1\n\t" \
  "s_waitcnt vmcnt(0)\n\t" \
  "v_cmp_le_u32 s[92:93], s86, v46\n\t" \
  "v_readfirstlane_b32 s98, v47\n\t" \
  "s_nop 4\n\t" \
  "s_or_b64 s[58:59], s[58:59], s[92:93]\n\t" \
  "s_bitcmp1_b64 s[58:59], 6\n\t" \
  "s_cbranch_scc1 RET163_%=\n\t" \
  "s_cmp_eq_u32 s98, s85\n\t" \
  "s_cbranch_scc1 SB166_%=\n\t" \
  "s_memrealtime s[96:97]\n\t" \
  "s_waitcnt lgkmcnt(0)\n\t" \
  "s_sub_u32 s96, s96, s94\n\t" \
  "s_cmp_gt_u32 s96, 50000000\n\t" \
  "s_cbranch_scc1 SB166_%=\n\t" \
  "s_sleep 2\n\t" \
  "s_branch SL165_%=\n\t" \
  "SB166_%=:\n\t" \
  "v_mov_b32 v47, s85\n\t" \
  "global_store_dword v30, v47, s[50:51] offset:-128 sc1\n\t" \
  "s_mov_b64 s[90:91], -1\n\t" \
  "s_mov_b64 s[58:59], -1\n\t" \
  "s_waitcnt vmcnt(0)\n\t" \
  "s_branch RET163_%=\n\t" \
  "SLOW169_%=:\n\t" \
  "s_memrealtime s[94:95]\n\t" \
  "s_waitcnt lgkmcnt(0)\n\t" \
  "SL170_%=:\n\t" \
  "v_mov_b32 v46, 0\n\t" \
  "s_lshl_b64 exec, 1, 7\n\t" \
  "global_load_dword v46, v17, s[50:51] sc0 sc1\n\t" \
  "s_mov_b64 exec, 1\n\t" \
  "global_load_dword v47, v30, s[50:51] offset:-128 sc1\n\t" \
  "s_mov_b64 exec, -1\n\t" \
  "s_waitcnt vmcnt(0)\n\t" \
  "v_cmp_le_u32 s[92:93], s86, v46\n\t" \
  "v_readfirstlane_b32 s98, v47\n\t" \
  "s_nop 4\n\t" \
  "s_or_b64 s[58:59], s[58:59], s[92:93]\n\t" \
  "s_bitcmp1_b64 s[58:59], 7\n\t" \
  "s_cbranch_scc1 RET168_%=\n\t" \
  "s_cmp_eq_u32 s98, s85\n\t" \
  "s_cbranch_scc1 SB171_%=\n\t" \
  "s_memrealtime s[96:97]\n\t" \
  "s_waitcnt lgkmcnt(0)\n\t" \
  "s_sub_u32 s96, s96, s94\n\t" \
  "s_cmp_gt_u32 s96, 50000000\n\t" \
  "s_cbranch_scc1 SB171_%=\n\t" \
  "s_sleep 2\n\t" \
  "s_branch SL170_%=\n\t" \
  "SB171_%=:\n\t" \
  "v_mov_b32 v47, s85\n\t" \
  "global_store_dword v30, v47, s[50:51] offset:-128 sc1\n\t" \
  "s_mov_b64 s[90:91], -1\n\t" \
  "s_mov_b64 s[58:59], -1\n\t" \
  "s_waitcnt vmcnt(0)\n\t" \
  "s_branch RET168_%=\n\t" \
  "SLOW173_%=:\n\t" \
  "s_memrealtime s[94:95]\n\t" \
  "s_waitcnt lgkmcnt(0)\n\t" \
  "SL174_%=:\n\t" \
  "global_load_dword v46, v19, s[50:51] sc0 sc1\n\t" \
  "global_load_dword v47, v30, s[50:51] offset:-128 sc1\n\t" \
  "s_waitcnt vmcnt(0)\n\t" \
  "v_readfirstlane_b32 s99, v46\n\t" \
  "v_readfirstlane_b32 s98, v47\n\t" \
  "s_nop 4\n\t" \
  "s_cmp_ge_u32 s99, s85\n\t" \
  "s_cbranch_scc1 RET172_%=\n\t" \
  "s_cmp_eq_u32 s98, s85\n\t" \
  "s_cbranch_scc1 SB175_%=\n\t" \
  "s_memrealtime s[96:97]\n\t" \
  "s_waitcnt lgkmcnt(0)\n\t" \
  "s_sub_u32 s96, s96, s94\n\t" \
  "s_cmp_gt_u32 s96, 50000000\n\t" \
  "s_cbranch_scc1 SB175_%=\n\t" \
  "s_sleep 2\n\t" \
  "s_branch SL174_%=\n\t" \
  "SB175_%=:\n\t" \
  "v_mov_b32 v47, s85\n\t" \
  "global_store_dword v30, v47, s[50:51] offset:-128 sc1\n\t" \
  "s_waitcnt vmcnt(0)\n\t" \
  "s_branch RET172_%=\n\t" \
  "POLL177_%=:\n\t" \
  "s_lshl_b64 exec, 1, s66\n\t" \
  "global_load_dword v36, v16, s[50:51] sc0 sc1\n\t" \
  "s_mov_b64 exec, -1\n\t" \
  "s_branch RET176_%=\n\t" \
  "SLOW179_%=:\n\t" \
  "s_memrealtime s[94:95]\n\t" \
  "s_waitcnt lgkmcnt(0)\n\t" \
  "SL180_%=:\n\t" \
  "v_mov_b32 v46, 0\n\t" \
  "s_lshl_b64 exec, 1, s99\n\t" \
  "global_load_dword v46, v16, s[50:51] sc0 sc1\n\t" \
  "s_mov_b64 exec, 1\n\t" \
  "global_load_dword v47, v30, s[50:51] offset:-128 sc1\n\t" \
  "s_mov_b64 exec, -1\n\t" \
  "s_waitcnt vmcnt(0)\n\t" \
  "v_cmp_le_u32 s[92:93], s86, v46\n\t" \
  "v_readfirstlane_b32 s98, v47\n\t" \
  "s_nop 4\n\t" \
  "s_or_b64 s[90:91], s[90:91], s[92:93]\n\t" \
  "s_bitcmp1_b64 s[90:91], s99\n\t" \
  "s_cbranch_scc1 RET178_%=\n\t" \
  "s_cmp_eq_u32 s98, s85\n\t" \
  "s_cbranch_scc1 SB181_%=\n\t" \
  "s_memrealtime s[96:97]\n\t" \
  "s_waitcnt lgkmcnt(0)\n\t" \
  "s_sub_u32 s96, s96, s94\n\t" \
  "s_cmp_gt_u32 s96, 50000000\n\t" \
  "s_cbranch_scc1 SB181_%=\n\t" \
  "s_sleep 2\n\t" \
  "s_branch SL180_%=\n\t" \
  "SB181_%=:\n\t" \
  "v_mov_b32 v47, s85\n\t" \
  "global_store_dword v30, v47, s[50:51] offset:-128 sc1\n\t" \
  "s_mov_b64 s[90:91], -1\n\t" \
  "s_mov_b64 s[58:59], -1\n\t" \
  "s_waitcnt vmcnt(0)\n\t" \
  "s_branch RET178_%=\n\t" \
  "POLL183_%=:\n\t" \
  "s_lshl_b64 exec, 1, s99\n\t" \
  "global_load_dword v36, v16, s[50:51] sc0 sc1\n\t" \
  "s_mov_b64 exec, -1\n\t" \
  "s_branch RET182_%=\n\t" \
  "STG184_%=:\n\t" \
  "s_waitcnt vmcnt(5)\n\t" \
  "v_cmp_le_u32 s[92:93], s86, v36\n\t" \
  "s_nop 3\n\t" \
  "s_or_b64 s[90:91], s[90:91], s[92:93]\n\t" \
  "s_bitcmp1_b64 s[90:91], s99\n\t" \
  "s_cbranch_scc1 RET178_%=\n\t" \
  "s_branch SLOW179_%=\n\t" \
  "POLL186_%=:\n\t" \
  "s_mov_b64 exec, s[92:93]\n\t" \
  "global_load_dword v37, v17, s[50:51] sc0 sc1\n\t" \
  "s_mov_b64 exec, -1\n\t" \
  "s_branch RET185_%=\n\t" \
  "SLOW190_%=:\n\t" \
  "s_memrealtime s[94:95]\n\t" \
  "s_waitcnt lgkmcnt(0)\n\t" \
  "SL191_%=:\n\t" \
  "v_mov_b32 v46, 0\n\t" \
  "s_lshl_b64 exec, 1, s99\n\t" \
  "global_load_dword v46, v16, s[50:51] sc0 sc1\n\t" \
  "s_mov_b64 exec, 1\n\t" \
  "global_load_dword v47, v30, s[50:51] offset:-128 sc1\n\t" \
  "s_mov_b64 exec, -1\n\t" \
  "s_waitcnt vmcnt(0)\n\t" \
  "v_cmp_le_u32 s[92:93], s86, v46\n\t" \
  "v_readfirstlane_b32 s98, v47\n\t" \
  "s_nop 4\n\t" \
  "s_or_b64 s[90:91], s[90:91], s[92:93]\n\t" \
  "s_bitcmp1_b64 s[90:91], s99\n\t" \
  "s_cbranch_scc1 RET189_%=\n\t" \
  "s_cmp_eq_u32 s98, s85\n\t" \
  "s_cbranch_scc1 SB192_%=\n\t" \
  "s_memrealtime s[96:97]\n\t" \
  "s_waitcnt lgkmcnt(0)\n\t" \
  "s_sub_u32 s96, s96, s94\n\t" \
  "s_cmp_gt_u32 s96, 50000000\n\t" \
  "s_cbranch_scc1 SB192_%=\n\t" \
  "s_sleep 2\n\t" \
  "s_branch SL191_%=\n\t" \
  "SB192_%=:\n\t" \
  "v_mov_b32 v47, s85\n\t" \
  "global_store_dword v30, v47, s[50:51] offset:-128 sc1\n\t" \
  "s_mov_b64 s[90:91], -1\n\t" \
  "s_mov_b64 s[58:59], -1\n\t" \
  "s_waitcnt vmcnt(0)\n\t" \
  "s_branch RET189_%=\n\t" \
  "POLL194_%=:\n\t" \
  "s_lshl_b64 exec, 1, s99\n\t" \
  "global_load_dword v36, v16, s[50:51] sc0 sc1\n\t" \
  "s_mov_b64 exec, -1\n\t" \
  "s_branch RET193_%=\n\t" \
  "STG195_%=:\n\t" \
  "s_waitcnt vmcnt(5)\n\t" \
  "v_cmp_le_u32 s[92:93], s86, v36\n\t" \
  "s_nop 3\n\t" \
  "s_or_b64 s[90:91], s[90:91], s[92:93]\n\t" \
  "s_bitcmp1_b64 s[90:91], s99\n\t" \
  "s_cbranch_scc1 RET189_%=\n\t" \
  "s_branch SLOW190_%=\n\t" \
  "POLL197_%=:\n\t" \
  "s_mov_b64 exec, s[92:93]\n\t" \
  "global_load_dword v37, v17, s[50:51] sc0 sc1\n\t" \
  "s_mov_b64 exec, -1\n\t" \
  "s_branch RET196_%=\n\t" \
  "POLL201_%=:\n\t" \
  "s_mov_b64 exec, s[92:93]\n\t" \
  "global_load_dword v37, v17, s[50:51] sc0 sc1\n\t" \
  "s_mov_b64 exec, -1\n\t" \
  "s_branch RET200_%=\n\t" \
  "SLOW203_%=:\n\t" \
  "s_memrealtime s[94:95]\n\t" \
  "s_waitcnt lgkmcnt(0)\n\t" \
  "SL204_%=:\n\t" \
  "v_mov_b32 v46, 0\n\t" \
  "s_lshl_b64 exec, 1, 0\n\t" \
  "global_load_dword v46, v17, s[50:51] sc0 sc1\n\t" \
  "s_mov_b64 exec, 1\n\t" \
  "global_load_dword v47, v30, s[50:51] offset:-128 sc1\n\t" \
  "s_mov_b64 exec, -1\n\t" \
  "s_waitcnt vmcnt(0)\n\t" \
  "v_cmp_le_u32 s[92:93], s86, v46\n\t" \
  "v_readfirstlane_b32 s98, v47\n\t" \
  "s_nop 4\n\t" \
  "s_or_b64 s[58:59], s[58:59], s[92:93]\n\t" \
  "s_bitcmp1_b64 s[58:59], 0\n\t" \
  "s_cbranch_scc1 RET202_%=\n\t" \
  "s_cmp_eq_u32 s98, s85\n\t" \
  "s_cbranch_scc1 SB205_%=\n\t" \
  "s_memrealtime s[96:97]\n\t" \
  "s_waitcnt lgkmcnt(0)\n\t" \
  "s_sub_u32 s96, s96, s94\n\t" \
  "s_cmp_gt_u32 s96, 50000000\n\t" \
  "s_cbranch_scc1 SB205_%=\n\t" \
  "s_sleep 2\n\t" \
  "s_branch SL204_%=\n\t" \
  "SB205_%=:\n\t" \
  "v_mov_b32 v47, s85\n\t" \
  "global_store_dword v30, v47, s[50:51] offset:-128 sc1\n\t" \
  "s_mov_b64 s[90:91], -1\n\t" \
  "s_mov_b64 s[58:59], -1\n\t" \
  "s_waitcnt vmcnt(0)\n\t" \
  "s_branch RET202_%=\n\t" \
  "POLL207_%=:\n\t" \
  "s_mov_b64 exec, s[92:93]\n\t" \
  "global_load_dword v37, v17, s[50:51] sc0 sc1\n\t" \
  "s_mov_b64 exec, -1\n\t" \
  "s_branch RET206_%=\n\t" \
  "SLOW209_%=:\n\t" \
  "s_memrealtime s[94:95]\n\t" \
  "s_waitcnt lgkmcnt(0)\n\t" \
  "SL210_%=:\n\t" \
  "v_mov_b32 v46, 0\n\t" \
  "s_lshl_b64 exec, 1, 1\n\t" \
  "global_load_dword v46, v17, s[50:51] sc0 sc1\n\t" \
  "s_mov_b64 exec, 1\n\t" \
  "global_load_dword v47, v30, s[50:51] offset:-128 sc1\n\t" \
  "s_mov_b64 exec, -1\n\t" \
  "s_waitcnt vmcnt(0)\n\t" \
  "v_cmp_le_u32 s[92:93], s86, v46\n\t" \
  "v_readfirstlane_b32 s98, v47\n\t" \
  "s_nop 4\n\t" \
  "s_or_b64 s[58:59], s[58:59], s[92:93]\n\t" \
  "s_bitcmp1_b64 s[58:59], 1\n\t" \
  "s_cbranch_scc1 RET208_%=\n\t" \
  "s_cmp_eq_u32 s98, s85\n\t" \
  "s_cbranch_scc1 SB211_%=\n\t" \
  "s_memrealtime s[96:97]\n\t" \
  "s_waitcnt lgkmcnt(0)\n\t" \
  "s_sub_u32 s96, s96, s94\n\t" \
  "s_cmp_gt_u32 s96, 50000000\n\t" \
  "s_cbranch_scc1 SB211_%=\n\t" \
  "s_sleep 2\n\t" \
  "s_branch SL210_%=\n\t" \
  "SB211_%=:\n\t" \
  "v_mov_b32 v47, s85\n\t" \
  "global_store_dword v30, v47, s[50:51] offset:-128 sc1\n\t" \
  "s_mov_b64 s[90:91], -1\n\t" \
  "s_mov_b64 s[58:59], -1\n\t" \
  "s_waitcnt vmcnt(0)\n\t" \
  "s_branch RET208_%=\n\t" \
  "SLOW213_%=:\n\t" \
  "s_memrealtime s[94:95]\n\t" \
  "s_waitcnt lgkmcnt(0)\n\t" \
  "SL214_%=:\n\t" \
  "v_mov_b32 v46, 0\n\t" \
  "s_lshl_b64 exec, 1, 2\n\t" \
  "global_load_dword v46, v17, s[50:51] sc0 sc1\n\t" \
  "s_mov_b64 exec, 1\n\t" \
  "global_load_dword v47, v30, s[50:51] offset:-128 sc1\n\t" \
  "s_mov_b64 exec, -1\n\t" \
  "s_waitcnt vmcnt(0)\n\t" \
  "v_cmp_le_u32 s[92:93], s86, v46\n\t" \
  "v_readfirstlane_b32 s98, v47\n\t" \
  "s_nop 4\n\t" \
  "s_or_b64 s[58:59], s[58:59], s[92:93]\n\t" \
  "s_bitcmp1_b64 s[58:59], 2\n\t" \
  "s_cbranch_scc1 RET212_%=\n\t" \
  "s_cmp_eq_u32 s98, s85\n\t" \
  "s_cbranch_scc1 SB215_%=\n\t" \
  "s_memrealtime s[96:97]\n\t" \
  "s_waitcnt lgkmcnt(0)\n\t" \
  "s_sub_u32 s96, s96, s94\n\t" \
  "s_cmp_gt_u32 s96, 50000000\n\t" \
  "s_cbranch_scc1 SB215_%=\n\t" \
  "s_sleep 2\n\t" \
  "s_branch SL214_%=\n\t" \
  "SB215_%=:\n\t" \
  "v_mov_b32 v47, s85\n\t" \
  "global_store_dword v30, v47, s[50:51] offset:-128 sc1\n\t" \
  "s_mov_b64 s[90:91], -1\n\t" \
  "s_mov_b64 s[58:59], -1\n\t" \
  "s_waitcnt vmcnt(0)\n\t" \
  "s_branch RET212_%=\n\t" \
  "POLL217_%=:\n\t" \
  "s_mov_b64 exec, s[92:93]\n\t" \
  "global_load_dword v37, v17, s[50:51] sc0 sc1\n\t" \
  "s_mov_b64 exec, -1\n\t" \
  "s_branch RET216_%=\n\t" \
  "SLOW219_%=:\n\t" \
  "s_memrealtime s[94:95]\n\t" \
  "s_waitcnt lgkmcnt(0)\n\t" \
  "SL220_%=:\n\t" \
  "v_mov_b32 v46, 0\n\t" \
  "s_lshl_b64 exec, 1, 3\n\t" \
  "global_load_dword v46, v17, s[50:51] sc0 sc1\n\t" \
  "s_mov_b64 exec, 1\n\t" \
  "global_load_dword v47, v30, s[50:51] offset:-128 sc1\n\t" \
  "s_mov_b64 exec, -1\n\t" \
  "s_waitcnt vmcnt(0)\n\t" \
  "v_cmp_le_u32 s[92:93], s86, v46\n\t" \
  "v_readfirstlane_b32 s98, v47\n\t" \
  "s_nop 4\n\t" \
  "s_or_b64 s[58:59], s[58:59], s[92:93]\n\t" \
  "s_bitcmp1_b64 s[58:59], 3\n\t" \
  "s_cbranch_scc1 RET218_%=\n\t" \
  "s_cmp_eq_u32 s98, s85\n\t" \
  "s_cbranch_scc1 SB221_%=\n\t" \
  "s_memrealtime s[96:97]\n\t" \
  "s_waitcnt lgkmcnt(0)\n\t" \
  "s_sub_u32 s96, s96, s94\n\t" \
  "s_cmp_gt_u32 s96, 50000000\n\t" \
  "s_cbranch_scc1 SB221_%=\n\t" \
  "s_sleep 2\n\t" \
  "s_branch SL220_%=\n\t" \
  "SB221_%=:\n\t" \
  "v_mov_b32 v47, s85\n\t" \
  "global_store_dword v30, v47, s[50:51] offset:-128 sc1\n\t" \
  "s_mov_b64 s[90:91], -1\n\t" \
  "s_mov_b64 s[58:59], -1\n\t" \
  "s_waitcnt vmcnt(0)\n\t" \
  "s_branch RET218_%=\n\t" \
  "POLL223_%=:\n\t" \
  "s_mov_b64 exec, s[92:93]\n\t" \
  "global_load_dword v37, v17, s[50:51] sc0 sc1\n\t" \
  "s_mov_b64 exec, -1\n\t" \
  "s_branch RET222_%=\n\t" \
  "SLOW225_%=:\n\t" \
  "s_memrealtime s[94:95]\n\t" \
  "s_waitcnt lgkmcnt(0)\n\t" \
  "SL226_%=:\n\t" \
  "v_mov_b32 v46, 0\n\t" \
  "s_lshl_b64 exec, 1, 4\n\t" \
  "global_load_dword v46, v17, s[50:51] sc0 sc1\n\t" \
  "s_mov_b64 exec, 1\n\t" \
  "global_load_dword v47, v30, s[50:51] offset:-128 sc1\n\t" \
  "s_mov_b64 exec, -1\n\t" \
  "s_waitcnt vmcnt(0)\n\t" \
  "v_cmp_le_u32 s[92:93], s86, v46\n\t" \
  "v_readfirstlane_b32 s98, v47\n\t" \
  "s_nop 4\n\t" \
  "s_or_b64 s[58:59], s[58:59], s[92:93]\n\t" \
  "s_bitcmp1_b64 s[58:59], 4\n\t" \
  "s_cbranch_scc1 RET224_%=\n\t" \
  "s_cmp_eq_u32 s98, s85\n\t" \
  "s_cbranch_scc1 SB227_%=\n\t" \
  "s_memrealtime s[96:97]\n\t" \
  "s_waitcnt lgkmcnt(0)\n\t" \
  "s_sub_u32 s96, s96, s94\n\t" \
  "s_cmp_gt_u32 s96, 50000000\n\t" \
  "s_cbranch_scc1 SB227_%=\n\t" \
  "s_sleep 2\n\t" \
  "s_branch SL226_%=\n\t" \
  "SB227_%=:\n\t" \
  "v_mov_b32 v47, s85\n\t" \
  "global_store_dword v30, v47, s[50:51] offset:-128 sc1\n\t" \
  "s_mov_b64 s[90:91], -1\n\t" \
  "s_mov_b64 s[58:59], -1\n\t" \
  "s_waitcnt vmcnt(0)\n\t" \
  "s_branch RET224_%=\n\t" \
  "SLOW229_%=:\n\t" \
  "s_memrealtime s[94:95]\n\t" \
  "s_waitcnt lgkmcnt(0)\n\t" \
  "SL230_%=:\n\t" \
  "v_mov_b32 v46, 0\n\t" \
  "s_lshl_b64 exec, 1, 5\n\t" \
  "global_load_dword v46, v17, s[50:51] sc0 sc1\n\t" \
  "s_mov_b64 exec, 1\n\t" \
  "global_load_dword v47, v30, s[50:51] offset:-128 sc1\n\t" \
  "s_mov_b64 exec, -1\n\t" \
  "s_waitcnt vmcnt(0)\n\t" \
  "v_cmp_le_u32 s[92:93], s86, v46\n\t" \
  "v_readfirstlane_b32 s98, v47\n\t" \
  "s_nop 4\n\t" \
  "s_or_b64 s[58:59], s[58:59], s[92:93]\n\t" \
  "s_bitcmp1_b64 s[58:59], 5\n\t" \
  "s_cbranch_scc1 RET228_%=\n\t" \
  "s_cmp_eq_u32 s98, s85\n\t" \
  "s_cbranch_scc1 SB231_%=\n\t" \
  "s_memrealtime s[96:97]\n\t" \
  "s_waitcnt lgkmcnt(0)\n\t" \
  "s_sub_u32 s96, s96, s94\n\t" \
  "s_cmp_gt_u32 s96, 50000000\n\t" \
  "s_cbranch_scc1 SB231_%=\n\t" \
  "s_sleep 2\n\t" \
  "s_branch SL230_%=\n\t" \
  "SB231_%=:\n\t" \
  "v_mov_b32 v47, s85\n\t" \
  "global_store_dword v30, v47, s[50:51] offset:-128 sc1\n\t" \
  "s_mov_b64 s[90:91], -1\n\t" \
  "s_mov_b64 s[58:59], -1\n\t" \
  "s_waitcnt vmcnt(0)\n\t" \
  "s_branch RET228_%=\n\t" \
  "SLOW233_%=:\n\t" \
  "s_memrealtime s[94:95]\n\t" \
  "s_waitcnt lgkmcnt(0)\n\t" \
  "SL234_%=:\n\t" \
  "v_mov_b32 v46, 0\n\t" \
  "s_lshl_b64 exec, 1, 6\n\t" \
  "global_load_dword v46, v17, s[50:51] sc0 sc1\n\t" \
  "s_mov_b64 exec, 1\n\t" \
  "global_load_dword v47, v30, s[50:51] offset:-128 sc1\n\t" \
  "s_mov_b64 exec, -1\n\t" \
  "s_waitcnt vmcnt(0)\n\t" \
  "v_cmp_le_u32 s[92:93], s86, v46\n\t" \
  "v_readfirstlane_b32 s98, v47\n\t" \
  "s_nop 4\n\t" \
  "s_or_b64 s[58:59], s[58:59], s[92:93]\n\t" \
  "s_bitcmp1_b64 s[58:59], 6\n\t" \
  "s_cbranch_scc1 RET232_%=\n\t" \
  "s_cmp_eq_u32 s98, s85\n\t" \
  "s_cbranch_scc1 SB235_%=\n\t" \
  "s_memrealtime s[96:97]\n\t" \
  "s_waitcnt lgkmcnt(0)\n\t" \
  "s_sub_u32 s96, s96, s94\n\t" \
  "s_cmp_gt_u32 s96, 50000000\n\t" \
  "s_cbranch_scc1 SB235_%=\n\t" \
  "s_sleep 2\n\t" \
  "s_branch SL234_%=\n\t" \
  "SB235_%=:\n\t" \
  "v_mov_b32 v47, s85\n\t" \
  "global_store_dword v30, v47, s[50:51] offset:-128 sc1\n\t" \
  "s_mov_b64 s[90:91], -1\n\t" \
  "s_mov_b64 s[58:59], -1\n\t" \
  "s_waitcnt vmcnt(0)\n\t" \
  "s_branch RET232_%=\n\t" \
  "SLOW238_%=:\n\t" \
  "s_memrealtime s[94:95]\n\t" \
  "s_waitcnt lgkmcnt(0)\n\t" \
  "SL239_%=:\n\t" \
  "v_mov_b32 v46, 0\n\t" \
  "s_lshl_b64 exec, 1, 7\n\t" \
  "global_load_dword v46, v17, s[50:51] sc0 sc1\n\t" \
  "s_mov_b64 exec, 1\n\t" \
  "global_load_dword v47, v30, s[50:51] offset:-128 sc1\n\t" \
  "s_mov_b64 exec, -1\n\t" \
  "s_waitcnt vmcnt(0)\n\t" \
  "v_cmp_le_u32 s[92:93], s86, v46\n\t" \
  "v_readfirstlane_b32 s98, v47\n\t" \
  "s_nop 4\n\t" \
  "s_or_b64 s[58:59], s[58:59], s[92:93]\n\t" \
  "s_bitcmp1_b64 s[58:59], 7\n\t" \
  "s_cbranch_scc1 RET237_%=\n\t" \
  "s_cmp_eq_u32 s98, s85\n\t" \
  "s_cbranch_scc1 SB240_%=\n\t" \
  "s_memrealtime s[96:97]\n\t" \
  "s_waitcnt lgkmcnt(0)\n\t" \
  "s_sub_u32 s96, s96, s94\n\t" \
  "s_cmp_gt_u32 s96, 50000000\n\t" \
  "s_cbranch_scc1 SB240_%=\n\t" \
  "s_sleep 2\n\t" \
  "s_branch SL239_%=\n\t" \
  "SB240_%=:\n\t" \
  "v_mov_b32 v47, s85\n\t" \
  "global_store_dword v30, v47, s[50:51] offset:-128 sc1\n\t" \
  "s_mov_b64 s[90:91], -1\n\t" \
  "s_mov_b64 s[58:59], -1\n\t" \
  "s_waitcnt vmcnt(0)\n\t" \
  "s_branch RET237_%=\n\t" \
  "SLOW242_%=:\n\t" \
  "s_memrealtime s[94:95]\n\t" \
  "s_waitcnt lgkmcnt(0)\n\t" \
  "SL243_%=:\n\t" \
  "global_load_dword v46, v19, s[50:51] sc0 sc1\n\t" \
  "global_load_dword v47, v30, s[50:51] offset:-128 sc1\n\t" \
  "s_waitcnt vmcnt(0)\n\t" \
  "v_readfirstlane_b32 s99, v46\n\t" \
  "v_readfirstlane_b32 s98, v47\n\t" \
  "s_nop 4\n\t" \
  "s_cmp_ge_u32 s99, s85\n\t" \
  "s_cbranch_scc1 RET241_%=\n\t" \
  "s_cmp_eq_u32 s98, s85\n\t" \
  "s_cbranch_scc1 SB244_%=\n\t" \
  "s_memrealtime s[96:97]\n\t" \
  "s_waitcnt lgkmcnt(0)\n\t" \
  "s_sub_u32 s96, s96, s94\n\t" \
  "s_cmp_gt_u32 s96, 50000000\n\t" \
  "s_cbranch_scc1 SB244_%=\n\t" \
  "s_sleep 2\n\t" \
  "s_branch SL243_%=\n\t" \
  "SB244_%=:\n\t" \
  "v_mov_b32 v47, s85\n\t" \
  "global_store_dword v30, v47, s[50:51] offset:-128 sc1\n\t" \
  "s_waitcnt vmcnt(0)\n\t" \
  "s_branch RET241_%=\n\t" \
  "END_%=:\n\t" \
  ""

#define K_CLOB2 "v17","v18","v19","v20","v21","v22","v23","v25","v26","v27","v28","v29","v30","v31","v36","v37","v38","v39","v40","v41","v42","v43","v44","v45","v47","v48","v49","v50","v51","v52","v53","v54","v55","v56","v57","v58","v59","v60","v61","v62","v63","v64","v65","v66","v67","v68","v69","v70","v71","v72","v73","v74","v75","v76","v77","v78","v79","v80","v81","v82","v83","v84","v85","v86","v87","v88","v89","v90","v91","v92","v93","v94","v95","v96","v97","v98","v99","v100","v101","v102","v103","v104","v105","v106","v107","v108","v109","v110","v111","v112","v113","v114","v115","v116","v117","v118","v119","v120","v121","v122","v123","v124","v125","v126","v127","v128","v129","v130","v131","v132","v133","v134","v135","v136","v137","v138","v139","v140","v141","v142","v143","v144","v145","v146","v147","v148","v149","v150","v151","v152","v153","v154","v155","v156","v157","v158","v159","v160","v161","v162","v163","v164","v165","v166","v167","v168","v169","v170","v171","v172","v173","v174","v175","v176","v177","v178","v179","v180","v181","v182","v183","v184","v185","v186","v187","v188","v189","v190","v191","v192","v193","v194","v195","v196","v197","v198","v199","v200","v201","v202","v203","v204","v205","v206","v207","v208","v209","v210","v211","v212","v213","v214","v215","v216","v217","v218","v219","v220","v221","v222","v223","v224","v225","v226","v227","v228","v229","v230","v231","v232","v233","v234","v235","v236","v237","v238","v239","v240","v241","v242","v243","v244","v245","v246","v247","v248","v249","v250","v251","v252","v253","v254","v255","v9","a0","a1","a2","a3","a4","a5","a6","a7","a8","a9","a10","a11","a12","a13","a14","a15","a16","a17","a18","a19","a20","a21","a22","a23","a24","a25","a26","a27","a28","a29","a30","a31","a32","a33","a34","a35","a36","a37","a38","a39","a40","a41","a42","a43","a44","a45","a46","a47","a48","a49","a50","a51","a52","a53","a54","a55","a56","a57","a58","a59","a60","a61","a62","a63","a64","a65","a66","a67","a68","a69","a70","a71","a72","a73","a74","a75","a76","a77","a78","a79","a80","a81","a82","a83","a84","a85","a86","a87","a88","a89","a90","a91","a92","a93","a94","a95","a96","a97","a98","a99","a100","a101","a102","a103","a104","a105","a106","a107","a108","a109","a110","a111","a112","a113","a114","a115","a116","a117","a118","a119","a120","a121","a122","a123","a124","a125","a126","a127","a128","a129","a130","a131","a132","a133","a134","a135","a136","a137","a138","a139","a140","a141","a142","a143","a144","a145","a146","a147","a148","a149","a150","a151","a152","a153","a154","a155","a156","a157","a158","a159","a160","a161","a162","a163","a164","a165","a166","a167","a168","a169","a170","a171","a172","a173","a174","a175","a176","a177","a178","a179","a180","a181","a182","a183","a184","a185","a186","a187","a188","a189","a190","a191","a192","a193","a194","a195","a196","a197","a198","a199","a200","a201","a202","a203","a204","a205","a206","a207","a208","a209","a210","a211","a212","a213","a214","a215","a216","a217","a218","a219","a220","a221","a222","a223","a224","a225","a226","a227","a228","a229","a230","a231","a232","a233","a234","a235","a236","a237","a238","a239","a240","a241","a242","a243","a244","a245","a246","a247","a248","a249","a250","a251","a252","a253","a254","a255","s76","s77","s79","s80","s81","s84","s85","s86","s87","s88","s89","s90","s91","s92","s93","s94","s95","s96","s97","s98","s99","s12","s13","s14","s15","s16","s17","s18","s19","s20","s21","s22","s23","s24","s25","s26","s27","s28","s29","s30","s31","s33","s34","s35","s53","s54","s55","s64","s65","s66","s67","vcc","scc","memory"

typedef unsigned long long u64;
typedef int i32x8 __attribute__((ext_vector_type(8)));
typedef int i32x4 __attribute__((ext_vector_type(4)));
typedef int i32x2 __attribute__((ext_vector_type(2)));
constexpr int LDS_BYTES = 163840;
constexpr size_t CHUNK = 2u << 20;
constexpr size_t OFF_FLAG = 64u << 20;
constexpr unsigned REL_XF = 16384;
constexpr unsigned REL_CNT = REL_XF + 256 * 128;
constexpr unsigned REL_DBG = REL_CNT + 1024 * 128;
constexpr unsigned REL_PART = 1u << 20;
constexpr size_t OFF_DBG = OFF_FLAG + REL_DBG;
constexpr size_t WS_NEED = OFF_FLAG + REL_PART + (16u << 20);
struct Link { char* pdst; char* pflag; };
struct P { const char* x; const char* W; char* wsf; int rank; int pad; char* ws; char* out; Link link[7]; unsigned short tab[2][64]; };
#define LO(x) (int)(unsigned)(u64)(x)
#define HI(x) (int)(unsigned)((u64)(x) >> 32)

__global__ void __launch_bounds__(256, 1) a2a_gemm(P p) {
  extern __shared__ __attribute__((aligned(16))) char smem[];
  int lane = threadIdx.x & 63; int w = __builtin_amdgcn_readfirstlane(threadIdx.x >> 6);
  const unsigned lds0 = (unsigned)(size_t)smem;
  int bid = blockIdx.x; const int rank = p.rank;
  i32x8 vo; int vpush, vtab; i32x8 sa; i32x4 sb; i32x2 ska; i32x4 sc; int sw0, stk; int vtick; i32x4 vdata; i32x2 st0;
  {
    const int h = bid & 1, tm = (bid >> 3) & 3, tn = ((bid & 7) >> 1) * 8 + (bid >> 5), r = lane & 15, g = lane >> 4;
#pragma unroll
    for (int i = 0; i < 4; ++i) {
      vo[i] = ((w >> 1) * 128 + (w & 1) * 32 + 8 * i + (lane >> 3)) * 2048 + (((lane & 7) ^ (((i & 1) << 2) | g)) << 4) - 1024 * i;
      const int c = (r >> 1) ^ ((((i >> 1) & 1) << 2) | g);
      vo[4 + i] = (16 * w + 4 * i + g) * 16384 + (c >> 2) * (K_WC ? 128 : 256) + (c & 3) * 32 + (r & 1) * 16 - 1024 * i;
    }
    const int ps = (bid * 9363) >> 16, dl = bid - ps * 7;
    vpush = ((ps & 31) * 32 + 8 * w + (lane >> 3)) * 2048 + (lane & 7) * 16 + (dl & 1) * 128;
    vtab = (int)offsetof(P, tab) + h * 128 + lane * 2;
    const char* a0 = p.x + tm * 256 * 2048; const char* Wb = p.W + tn * 512;
    sa[0] = LO(a0); sa[1] = HI(a0) & 0xffff; sa[2] = (16 << 20) - tm * 256 * 2048; sa[3] = 0x20000;
    sa[4] = LO(Wb); sa[5] = HI(Wb) & 0xffff; sa[6] = (128 << 20) - tn * 512; sa[7] = 0x20000;
    const char* psrc = p.x + (size_t)((rank + 1 + dl) & 7) * CHUNK;
    sb[0] = LO(psrc); sb[1] = HI(psrc); sb[2] = LO(p.wsf); sb[3] = HI(p.wsf);
    const u64 ka = (u64)__builtin_amdgcn_kernarg_segment_ptr(); ska[0] = LO(ka); ska[1] = HI(ka);
    stk = REL_CNT + (bid * 4 + w) * 128;
    const int a00 = rank * (int)CHUNK + h * 128;
    sc[0] = lds0 + w * 4096; sc[1] = (1 - h) * 131072 + a00; sc[2] = h * 131072 + a00; sc[3] = w; sw0 = (rank * 16 + h) << 20;
  }
  asm volatile(K_ASM1 : "+{v[0:7]}"(vo), "+{v8}"(vpush), "+{v16}"(vtab), "+{s[36:43]}"(sa), "+{s[48:51]}"(sb), "+{s[58:59]}"(ska), "+{s62}"(stk), "+{s[68:71]}"(sc),
               "+{s78}"(sw0), "={v46}"(vtick), "={v[32:35]}"(vdata), "={s[82:83]}"(st0), "+{v24}"(lane), "+s"(bid) : : "v31", "v47", "s79", "scc", "memory");
  w = __builtin_amdgcn_readfirstlane(sc[3]); bid = __builtin_amdgcn_readfirstlane(bid);
  const int h = bid & 1, tm = (bid >> 3) & 3, tn = ((bid & 7) >> 1) * 8 + (bid >> 5);
  const int wm = w >> 1, wn = w & 1, r = lane & 15, g = lane >> 4, q4 = r >> 2, pp = lane & 3;
  const int ps = (bid * 9363) >> 16, dl = bid - ps * 7;
  i32x2 vaa; i32x4 vab; i32x4 so; i32x2 swb, sx; i32x4 sd;
  vaa[0] = lds0 + wm * 8192 + r * 128 + ((g ^ ((r >> 1) & 7)) << 4); vaa[1] = vaa[0] ^ 64;
  const int sw = ((g & 1) << 2) | q4;
#pragma unroll
  for (int n = 0; n < 4; ++n) vab[n] = K_KH ? lds0 + 65536 + (8 * g + q4) * 512 + wn * 256 + ((n ^ sw) << 5) + 8 * pp : lds0 + 65536 + (8 * g + q4) * 256 + (((wn * 4 + n) ^ sw) << 5) + 8 * pp;
  const char* Ob = p.out + (size_t)(tm * 256 + wm * 128 + h * 64) * 16384 + (tn * 256 + wn * (K_WC ? 64 : 128)) * 2;
  so[0] = LO(Ob); so[1] = HI(Ob) & 0xffff; so[2] = 63 * 16384 + (K_WC ? 384 : 256); so[3] = 0x20000;
  const char* wb = p.ws + tm * 256 * 128; swb[0] = LO(wb); swb[1] = HI(wb);
  sx[0] = REL_PART + bid * 65536 + w * 16384 + 4096; sx[1] = REL_XF + bid * 128;
  const int sI = ps % K_I, s0 = sI * K_I0 / K_I;
  sd[0] = bid < 224 ? s0 : 0x7fffffff; sd[1] = sI - s0; sd[2] = (dl & 1) ? -128 : 128; sd[3] = REL_DBG + bid * 128;
  int slo = (int)offsetof(P, link) + dl * 16;
  int sa00 = rank * (int)CHUNK + h * 128, stch = (tm * 2 + h) * 8 * 16384;
  asm volatile(K_ASM2 : "+{v[0:7]}"(vo), "+{v8}"(vpush), "+{v[10:11]}"(vaa), "+{v[12:15]}"(vab), "+{v16}"(vtab), "+{v24}"(lane),
               "+{v46}"(vtick), "+{v[32:35]}"(vdata), "+{s[36:43]}"(sa), "+{s[44:47]}"(so), "+{s[48:51]}"(sb), "+{s52}"(slo), "+{s[58:59]}"(ska), "+{s[56:57]}"(swb),
               "+{s[60:61]}"(sx), "+{s62}"(stch), "+{s63}"(sa00), "+{s[68:71]}"(sc), "+{s[72:75]}"(sd), "+{s78}"(sw0), "+{s[82:83]}"(st0) : : K_CLOB2);
}

extern "C" void kernel_launch(int rank, int nranks, ncclComm_t comm, hipStream_t stream,
                              void* const* d_in, const int* in_sizes, int n_in,
                              void* d_out, int out_size, const int* in_dtypes, int out_dtype,
                              void* const* d_out_all, void* const* d_ws_all, size_t ws_bytes) {
  static bool init[16] = {};
  if (nranks != 8 || (unsigned)rank >= 8u || ws_bytes < WS_NEED) return;
  if (!init[rank]) { (void)hipFuncSetAttribute((const void*)a2a_gemm, hipFuncAttributeMaxDynamicSharedMemorySize, LDS_BYTES); init[rank] = true; }
  P p;
  p.x = (const char*)d_in[0]; p.W = (const char*)d_in[1]; p.out = (char*)d_out; p.ws = (char*)d_ws_all[rank]; p.wsf = p.ws + OFF_FLAG;
  for (int d = 0; d < 7; ++d) { char* q = (char*)d_ws_all[(rank + 1 + d) & 7]; p.link[d].pdst = q + (size_t)rank * CHUNK; p.link[d].pflag = q + OFF_FLAG + rank * 2048; }
  int seq[2][64];
  for (int h = 0; h < 2; ++h) {
    int n = 0;
    for (int t = 0; t < 8; ++t) seq[h][n++] = rank * 16 + 2 * t + h;
    for (int rd = 0; rd < 16; ++rd)
      for (int d = 0; d < 7; ++d) if (((rd ^ d) & 1) == h && !(rd >= 14 && d >= 7 - K_NR / 2)) seq[h][n++] = ((rank - 1 - d) & 7) * 16 + (rd ^ (d & 1));
    while (n < 64) seq[h][n++] = rank * 16;
  }
  for (int h = 0; h < 2; ++h) {
    const int ta = 64 - K_NR / 2; int* a = seq[h];
    a[ta] = seq[h ^ 1][ta - 1]; a[ta + 1] = seq[h ^ 1][ta - 2];
    for (int i = 0; i < 64; ++i) { const int nl = K_NR / 2, b = i < K_NR ? i : 0, d = 7 - nl + (b < nl ? b : b - nl);
      p.tab[h][i] = (unsigned short)(a[i] | ((((rank - 1 - d) & 7) * 16 + ((b < nl ? 14 : 15) ^ (d & 1))) << 8)); }
  }
  p.rank = rank; p.pad = 0;
  a2a_gemm<<<256, 256, LDS_BYTES, stream>>>(p);
}
```
